```python
import numpy as np

kernel_name = "gpu_a2a_gemm_m4096_k4096_n16384_e4m3e5m2_dequant"


ABSMAX_RTOL = 0.05

COMPUTE_DTYPE = "float32"
EPILOGUE = "dequant"
IN_QUANT = None
OUT_QUANT = None
IN_FP8 = "e4m3e5m2"
IN_DTYPES = {"x": "e4m3", "w_mat": "e5m2",
             "scale_x": "float32", "scale_w": "float32"}
WORLD_SIZE = 8

def _snap_fp8(x, mant_bits, max_abs, min_normal):
    a = np.abs(x.astype(np.float64))
    _m, _e = np.frexp(a)
    q = np.where(a >= min_normal, np.ldexp(1.0, _e - 1 - mant_bits),
                 min_normal * 2.0 ** -mant_bits)
    snapped = np.minimum(np.round(a / q) * q, max_abs)
    return (np.sign(x) * snapped).astype(np.float32)

def setup_inputs():

    x = _snap_fp8(np.random.randn(4096, 4096).astype(np.float32), 3, 448.0, 2.0 ** -6)
    w_mat = _snap_fp8(np.random.randn(4096, 16384).astype(np.float32), 2, 57344.0, 2.0 ** -14)
    scale_x = (0.02 * (1.0 + np.random.rand(1))).astype(np.float32)
    scale_w = (0.02 * (1.0 + np.random.rand(1))).astype(np.float32)
    return {"x": x, "w_mat": w_mat, "scale_x": scale_x, "scale_w": scale_w}

def reference(x, w_mat, scale_x, scale_w):
    acc = (x @ w_mat).astype(np.float32)
    y = (acc * (scale_x * scale_w)).astype(np.float32)
    return y
```

```cpp
#include <hip/hip_runtime.h>
#include <rccl/rccl.h>
#include <cstdio>

#define KERN_THREADS 256
#define KERN_WPE 1
#define KERN_LDS 163840
#define KERN_NBAR 33
#define KERN_ASM \
  "s_mov_b64 s[8:9], %1\n\t" \
  "s_mov_b32 s10, %2\n\t" \
  "s_load_dwordx8 s[16:23], s[8:9], 0x0\n\t" \
  "s_load_dwordx4 s[24:27], s[8:9], 0x20\n\t" \
  "s_load_dword s28, s[8:9], 0x30\n\t" \
  "s_mul_i32 s84, s10, 9363\n\t" \
  "s_lshr_b32 s84, s84, 16\n\t" \
  "s_mul_i32 s85, s84, 7\n\t" \
  "s_sub_u32 s85, s10, s85\n\t" \
  "s_lshl_b32 s86, s85, 3\n\t" \
  "s_add_u32 s86, s86, 0x38\n\t" \
  "s_load_dwordx2 s[12:13], s[8:9], s86\n\t" \
  "s_memrealtime s[14:15]\n\t" \
  "s_mov_b32 s11, %3\n\t" \
  "v_mov_b32 v212, %0\n\t" \
  "v_lshrrev_b32 v213, 6, v212\n\t" \
  "v_and_b32 v212, 63, v212\n\t" \
  "v_readfirstlane_b32 s29, v213\n\t" \
  "s_lshr_b32 s98, s10, 3\n\t" \
  "s_lshr_b32 s99, s98, 3\n\t" \
  "s_and_b32 s87, s98, 1\n\t" \
  "s_lshl1_add_u32 s99, s99, s87\n\t" \
  "s_lshl_b32 s99, s99, 10\n\t" \
  "v_and_b32 v234, 7, v212\n\t" \
  "v_lshl_add_u32 v234, v234, 7, s99\n\t" \
  "s_getpc_b64 s[88:89]\n\t" \
  "H_pc:\n\t" \
  "s_bfe_u32 s87, s10, 65540\n\t" \
  "s_mov_b32 s99, H_PH0-H_pc\n\t" \
  "s_cmp_eq_u32 s87, 1\n\t" \
  "s_cselect_b32 s99, H_PH1-H_pc, s99\n\t" \
  "s_cmp_eq_u32 s29, 1\n\t" \
  "s_cselect_b32 s87, 8192, 0\n\t" \
  "s_add_u32 s99, s99, s87\n\t" \
  "s_cmp_eq_u32 s29, 3\n\t" \
  "s_cselect_b32 s99, 0, s99\n\t" \
  "s_add_u32 s88, s88, s99\n\t" \
  "s_addc_u32 s89, s89, 0\n\t" \
  "s_cmp_eq_u32 s29, 0\n\t" \
  "s_cbranch_scc1 H_ct\n\t" \
  "s_mov_b64 exec, 0xff\n\t" \
  "global_load_dword v210, v234, s[88:89]\n\t" \
  "s_mov_b64 exec, -1\n\t" \
  "H_ct:\n\t" \
  "s_and_b32 s87, s10, 7\n\t" \
  "s_lshr_b32 s98, s10, 3\n\t" \
  "s_and_b32 s31, s98, 1\n\t" \
  "s_lshr_b32 s99, s98, 1\n\t" \
  "s_lshl4_add_u32 s30, s87, s99\n\t" \
  "s_lshl_b32 s83, s10, 6\n\t" \
  "s_add_u32 s83, s83, 6815744\n\t" \
  "v_and_b32 v213, 15, v212\n\t" \
  "v_lshrrev_b32 v214, 4, v212\n\t" \
  "v_lshlrev_b32 v199, 9, v213\n\t" \
  "v_lshl_add_u32 v199, v214, 4, v199\n\t" \
  "s_lshl_b32 s90, s29, 7\n\t" \
  "v_add_u32 v199, s90, v199\n\t" \
  "s_cmp_lt_u32 s10, 224\n\t" \
  "s_cselect_b32 s50, 8192, 0\n\t" \
  "s_cselect_b32 s54, 2048, 0\n\t" \
  "s_cselect_b32 s58, 4, 0\n\t" \
  "s_mov_b32 s51, 0x20000\n\t" \
  "s_lshl_b32 s86, s98, 13\n\t" \
  "v_lshl_add_u32 v215, v212, 7, s86\n\t" \
  "s_lshl_b32 s96, s30, 7\n\t" \
  "s_sub_u32 s42, 0x4000000, s96\n\t" \
  "s_mov_b32 s43, 0x20000\n\t" \
  "v_lshrrev_b32 v216, 3, v212\n\t" \
  "v_and_b32 v217, 7, v212\n\t" \
  "v_xor_b32 v217, v217, v214\n\t" \
  "v_lshlrev_b32 v190, 9, v216\n\t" \
  "v_lshl_add_u32 v190, v217, 4, v190\n\t" \
  "v_xor_b32 v191, 64, v190\n\t" \
  "s_and_b32 s86, s29, 1\n\t" \
  "s_lshl_b32 s86, s86, 2\n\t" \
  "v_xor_b32 v217, s86, v217\n\t" \
  "s_lshl_b32 s86, s29, 4\n\t" \
  "s_lshl_b32 s97, s31, 6\n\t" \
  "s_add_u32 s97, s97, s86\n\t" \
  "v_add_u32 v218, s97, v216\n\t" \
  "v_lshlrev_b32 v218, 14, v218\n\t" \
  "v_lshl_add_u32 v184, v217, 4, v218\n\t" \
  "v_add_u32 v185, 130048, v184\n\t" \
  "v_xor_b32 v186, 1048576, v184\n\t" \
  "v_add_u32 v187, 130048, v186\n\t" \
  "s_lshl_b32 s97, s97, 7\n\t" \
  "s_add_u32 s76, s97, s11\n\t" \
  "s_xor_b32 s97, s97, 8192\n\t" \
  "s_add_u32 s77, s97, s11\n\t" \
  "v_bfe_u32 v222, v212, 4, 1\n\t" \
  "v_lshlrev_b32 v222, 1, v222\n\t" \
  "v_xor_b32 v222, s98, v222\n\t" \
  "v_and_b32 v218, 3, v212\n\t" \
  "v_lshl_add_u32 v222, v222, 2, v218\n\t" \
  "v_lshlrev_b32 v222, 14, v222\n\t" \
  "v_bfe_u32 v219, v212, 2, 2\n\t" \
  "v_lshlrev_b32 v219, 3, v219\n\t" \
  "v_min_u32 v219, 12, v219\n\t" \
  "s_lshl_b32 s87, s87, 11\n\t" \
  "s_sub_u32 s62, 0x4000000, s87\n\t" \
  "s_waitcnt lgkmcnt(0)\n\t" \
  "s_load_dword s79, s[26:27], s83\n\t" \
  "s_load_dword s88, s[20:21], 0x0\n\t" \
  "s_load_dword s89, s[22:23], 0x0\n\t" \
  "s_sub_u32 s86, s28, s85\n\t" \
  "s_sub_u32 s86, s86, 1\n\t" \
  "s_and_b32 s86, s86, 7\n\t" \
  "s_lshl_b32 s86, s86, 5\n\t" \
  "s_add_u32 s86, s86, s84\n\t" \
  "s_lshl_b32 s86, s86, 13\n\t" \
  "s_add_u32 s48, s16, s86\n\t" \
  "s_addc_u32 s49, s17, 0\n\t" \
  "s_and_b32 s49, s49, 0xffff\n\t" \
  "s_cmp_eq_u32 s29, 0\n\t" \
  "s_cbranch_scc0 H_pl0\n\t" \
  "buffer_load_dwordx4 v[200:203], v199, s[48:51], 0 offen offset:0\n\t" \
  "buffer_load_dwordx4 v[204:207], v199, s[48:51], 0 offen offset:64\n\t" \
  "H_pl0:\n\t" \
  "s_lshl_b32 s75, s28, 23\n\t" \
  "s_cmp_eq_u32 s29, 2\n\t" \
  "s_cbranch_scc0 H_rt2\n\t" \
  "s_mov_b32 s64, s16\n\t" \
  "s_and_b32 s65, s17, 0xffff\n\t" \
  "s_mov_b32 s66, 2097152\n\t" \
  "s_mov_b32 s67, 0x20000\n\t" \
  "s_lshl_b32 s86, s28, 18\n\t" \
  "buffer_load_dword v210, v215, s[64:67], s86 offen\n\t" \
  "H_rt2:\n\t" \
  "s_add_u32 s40, s18, s96\n\t" \
  "s_addc_u32 s41, s19, 0\n\t" \
  "s_and_b32 s41, s41, 0xffff\n\t" \
  "s_add_u32 m0, s76, 0\n\t" \
  "s_add_u32 s90, s75, 0\n\t" \
  "buffer_load_dwordx4 v184, s[40:43], s90 offen offset:0 lds\n\t" \
  "buffer_load_dwordx4 v185, s[40:43], s90 offen offset:1024 lds\n\t" \
  "s_add_u32 m0, s76, 16384\n\t" \
  "s_add_u32 s90, s75, 2097152\n\t" \
  "buffer_load_dwordx4 v184, s[40:43], s90 offen offset:0 lds\n\t" \
  "buffer_load_dwordx4 v185, s[40:43], s90 offen offset:1024 lds\n\t" \
  "s_cmp_ge_u32 s29, 1\n\t" \
  "s_cbranch_scc0 H_pl1\n\t" \
  "buffer_load_dwordx4 v[200:203], v199, s[48:51], 0 offen offset:0\n\t" \
  "buffer_load_dwordx4 v[204:207], v199, s[48:51], 0 offen offset:64\n\t" \
  "H_pl1:\n\t" \
  "s_add_u32 s60, s18, s87\n\t" \
  "s_addc_u32 s61, s19, 0\n\t" \
  "s_and_b32 s61, s61, 0xffff\n\t" \
  "s_mov_b32 s63, 0x20000\n\t" \
  "s_lshr_b32 s86, s18, 7\n\t" \
  "s_sub_u32 s101, 3, s86\n\t" \
  "s_and_b32 s101, s101, 7\n\t" \
  "v_add_u32 v219, s101, v219\n\t" \
  "v_and_b32 v219, 15, v219\n\t" \
  "v_lshl_add_u32 v222, v219, 7, v222\n\t" \
  "s_mov_b32 s15, 0xfff\n\t" \
  "s_xor_b32 s101, s30, 1\n\t" \
  "s_add_u32 s101, s101, s86\n\t" \
  "s_and_b32 s100, s101, 127\n\t" \
  "s_cmp_eq_u32 s100, 63\n\t" \
  "s_cselect_b32 s101, 3, s101\n\t" \
  "s_and_b32 s101, s101, 7\n\t" \
  "s_cmp_eq_u32 s101, 3\n\t" \
  "s_cselect_b32 s15, 0x0fff0fff, s15\n\t" \
  "s_add_u32 s101, s30, s86\n\t" \
  "s_and_b32 s100, s101, 127\n\t" \
  "s_cmp_eq_u32 s100, 63\n\t" \
  "s_cselect_b32 s101, 3, s101\n\t" \
  "s_and_b32 s101, s101, 7\n\t" \
  "s_cmp_eq_u32 s101, 3\n\t" \
  "s_cselect_b32 s15, 0, s15\n\t" \
  "s_add_u32 s82, s29, 8\n\t" \
  "s_mul_i32 s86, s29, 7\n\t" \
  "s_add_u32 s86, s86, s85\n\t" \
  "s_lshl_b32 s86, s86, 16\n\t" \
  "s_lshl_b32 s87, s84, 11\n\t" \
  "s_add_u32 s91, s86, s87\n\t" \
  "s_lshr_b32 s86, s84, 2\n\t" \
  "s_lshl_b32 s86, s86, 7\n\t" \
  "s_lshl4_add_u32 s86, s85, s86\n\t" \
  "s_and_b32 s87, s84, 3\n\t" \
  "s_lshl2_add_u32 s86, s87, s86\n\t" \
  "s_lshl_b32 s87, s29, 10\n\t" \
  "s_add_u32 s86, s86, s87\n\t" \
  "s_add_u32 s86, s86, 6291456\n\t" \
  "s_add_u32 s56, s12, s86\n\t" \
  "s_addc_u32 s57, s13, 0\n\t" \
  "s_and_b32 s57, s57, 0xffff\n\t" \
  "s_mov_b32 s59, 0x20000\n\t" \
  "s_lshl2_add_u32 s80, s31, s29\n\t" \
  "s_add_u32 s68, s28, 1\n\t" \
  "s_and_b32 s68, s68, 7\n\t" \
  "s_lshl_b32 s68, s68, 23\n\t" \
  "s_add_u32 s69, s28, 2\n\t" \
  "s_and_b32 s69, s69, 7\n\t" \
  "s_lshl_b32 s69, s69, 23\n\t" \
  "s_add_u32 s70, s28, 3\n\t" \
  "s_and_b32 s70, s70, 7\n\t" \
  "s_lshl_b32 s70, s70, 23\n\t" \
  "s_add_u32 s71, s28, 4\n\t" \
  "s_and_b32 s71, s71, 7\n\t" \
  "s_lshl_b32 s71, s71, 23\n\t" \
  "s_add_u32 s72, s28, 5\n\t" \
  "s_and_b32 s72, s72, 7\n\t" \
  "s_lshl_b32 s72, s72, 23\n\t" \
  "s_add_u32 s73, s28, 6\n\t" \
  "s_and_b32 s73, s73, 7\n\t" \
  "s_lshl_b32 s73, s73, 23\n\t" \
  "s_add_u32 s74, s28, 7\n\t" \
  "s_and_b32 s74, s74, 7\n\t" \
  "s_lshl_b32 s74, s74, 23\n\t" \
  "s_lshl_b32 s78, s29, 13\n\t" \
  "s_add_u32 s78, s78, s11\n\t" \
  "s_lshl3_add_u32 s86, s28, s80\n\t" \
  "s_lshl_b32 s86, s86, 15\n\t" \
  "s_add_u32 s32, s16, s86\n\t" \
  "s_addc_u32 s33, s17, 0\n\t" \
  "s_and_b32 s33, s33, 0xffff\n\t" \
  "s_mov_b32 s34, 32768\n\t" \
  "s_mov_b32 s35, 0x20000\n\t" \
  "v_lshlrev_b32 v188, 4, v212\n\t" \
  "v_add_u32 v189, 4096, v188\n\t" \
  "v_mov_b32 v209, 0\n\t" \
  "v_bfe_u32 v216, v212, 1, 3\n\t" \
  "v_lshl_add_u32 v215, v214, 4, v216\n\t" \
  "v_lshlrev_b32 v215, 7, v215\n\t" \
  "v_lshrrev_b32 v217, 1, v216\n\t" \
  "v_and_b32 v218, 1, v214\n\t" \
  "v_lshl_add_u32 v217, v218, 2, v217\n\t" \
  "v_lshl_add_u32 v215, v217, 4, v215\n\t" \
  "v_and_b32 v218, 1, v212\n\t" \
  "v_lshl_add_u32 v215, v218, 3, v215\n\t" \
  "v_add_u32 v215, s11, v215\n\t" \
  "v_xor_b32 v160, 0, v215\n\t" \
  "v_add_u32 v168, 65536, v160\n\t" \
  "v_add_u32 v176, 131072, v160\n\t" \
  "v_xor_b32 v161, 16, v215\n\t" \
  "v_add_u32 v169, 65536, v161\n\t" \
  "v_add_u32 v177, 131072, v161\n\t" \
  "v_xor_b32 v162, 32, v215\n\t" \
  "v_add_u32 v170, 65536, v162\n\t" \
  "v_add_u32 v178, 131072, v162\n\t" \
  "v_xor_b32 v163, 48, v215\n\t" \
  "v_add_u32 v171, 65536, v163\n\t" \
  "v_add_u32 v179, 131072, v163\n\t" \
  "v_xor_b32 v164, 64, v215\n\t" \
  "v_add_u32 v172, 65536, v164\n\t" \
  "v_add_u32 v180, 131072, v164\n\t" \
  "v_xor_b32 v165, 80, v215\n\t" \
  "v_add_u32 v173, 65536, v165\n\t" \
  "v_add_u32 v181, 131072, v165\n\t" \
  "v_xor_b32 v166, 96, v215\n\t" \
  "v_add_u32 v174, 65536, v166\n\t" \
  "v_add_u32 v182, 131072, v166\n\t" \
  "v_xor_b32 v167, 112, v215\n\t" \
  "v_add_u32 v175, 65536, v167\n\t" \
  "v_add_u32 v183, 131072, v167\n\t" \
  "v_xor_b32 v217, v216, v214\n\t" \
  "v_lshlrev_b32 v218, 7, v213\n\t" \
  "v_lshl_add_u32 v217, v217, 4, v218\n\t" \
  "v_add_u32 v192, s78, v217\n\t" \
  "v_add_u32 v192, 65536, v192\n\t" \
  "v_xor_b32 v193, 64, v192\n\t" \
  "v_add_u32 v224, 65536, v192\n\t" \
  "v_add_u32 v225, 65536, v193\n\t" \
  "s_waitcnt lgkmcnt(0)\n\t" \
  "s_add_u32 s79, s79, 1\n\t" \
  "s_mul_hi_u32 s86, s79, 0xaaaaaaab\n\t" \
  "s_lshr_b32 s86, s86, 1\n\t" \
  "s_mul_i32 s86, s86, 3\n\t" \
  "s_sub_u32 s86, s79, s86\n\t" \
  "s_sub_u32 s100, s86, 1\n\t" \
  "s_min_u32 s100, s100, 2\n\t" \
  "s_lshl_b32 s100, s100, 21\n\t" \
  "s_lshl_b32 s101, s10, 11\n\t" \
  "s_add_u32 s100, s100, s101\n\t" \
  "s_mul_i32 s86, s86, 2097152\n\t" \
  "s_add_u32 s87, s91, s86\n\t" \
  "s_add_u32 s52, s12, s87\n\t" \
  "s_addc_u32 s53, s13, 0\n\t" \
  "s_and_b32 s53, s53, 0xffff\n\t" \
  "s_mov_b32 s55, 0x20000\n\t" \
  "v_mov_b32 v208, s79\n\t" \
  "s_lshl_b32 s87, s80, 13\n\t" \
  "s_add_u32 s86, s86, s87\n\t" \
  "s_add_u32 s36, s26, s86\n\t" \
  "s_addc_u32 s37, s27, 0\n\t" \
  "s_and_b32 s37, s37, 0xffff\n\t" \
  "s_mov_b32 s39, 0x20000\n\t" \
  "s_sub_u32 s38, 1835008, s87\n\t" \
  "v_mov_b32 v219, s88\n\t" \
  "v_mul_f32 v219, s89, v219\n\t" \
  "s_mov_b32 s97, 0\n\t" \
  "s_mov_b32 s96, 0\n\t" \
  "v_readfirstlane_b32 s81, v219\n\t" \
  "s_cmp_eq_u32 s29, 0\n\t" \
  "s_cbranch_scc0 H_ps0\n\t" \
  "s_waitcnt vmcnt(4)\n\t" \
  "buffer_store_dwordx4 v[200:203], v188, s[52:55], 0 offen offset:0 sc0 sc1\n\t" \
  "buffer_store_dwordx4 v[204:207], v188, s[52:55], 0 offen offset:1024 sc0 sc1\n\t" \
  "H_ps0:\n\t" \
  "s_barrier\n\t" \
  "s_lshr_b32 s84, s10, 4\n\t" \
  "s_and_b32 s84, s84, 1\n\t" \
  "s_cmp_eq_u32 s84, 1\n\t" \
  "s_cbranch_scc1 H_PH1\n\t" \
  "H_PH0:\n\t" \
  "s_add_u32 m0, s77, 0\n\t" \
  "s_add_u32 s90, s75, 0\n\t" \
  "buffer_load_dwordx4 v186, s[40:43], s90 offen offset:0  lds\n\t" \
  "s_add_u32 m0, s77, 0\n\t" \
  "s_add_u32 s90, s75, 0\n\t" \
  "buffer_load_dwordx4 v187, s[40:43], s90 offen offset:1024  lds\n\t" \
  "s_add_u32 m0, s78, 65536\n\t" \
  "s_movk_i32 s84, 0\n\t" \
  "buffer_load_dwordx4 v190, s[32:35], s84 offen offset:0 lds\n\t" \
  "s_add_u32 m0, s78, 65536\n\t" \
  "s_movk_i32 s84, 3072\n\t" \
  "buffer_load_dwordx4 v191, s[32:35], s84 offen offset:1024 lds\n\t" \
  "s_add_u32 m0, s78, 65536\n\t" \
  "s_movk_i32 s84, 6144\n\t" \
  "buffer_load_dwordx4 v190, s[32:35], s84 offen offset:2048 lds\n\t" \
  "s_add_u32 m0, s78, 65536\n\t" \
  "s_movk_i32 s84, 9216\n\t" \
  "buffer_load_dwordx4 v191, s[32:35], s84 offen offset:3072 lds\n\t" \
  "s_add_u32 m0, s78, 69632\n\t" \
  "s_movk_i32 s84, 16384\n\t" \
  "buffer_load_dwordx4 v190, s[32:35], s84 offen offset:0 lds\n\t" \
  "s_add_u32 m0, s78, 69632\n\t" \
  "s_movk_i32 s84, 19456\n\t" \
  "buffer_load_dwordx4 v191, s[32:35], s84 offen offset:1024 lds\n\t" \
  "s_add_u32 m0, s78, 69632\n\t" \
  "s_movk_i32 s84, 22528\n\t" \
  "buffer_load_dwordx4 v190, s[32:35], s84 offen offset:2048 lds\n\t" \
  "s_add_u32 m0, s78, 69632\n\t" \
  "s_movk_i32 s84, 25600\n\t" \
  "buffer_load_dwordx4 v191, s[32:35], s84 offen offset:3072 lds\n\t" \
  "s_add_u32 m0, s77, 16384\n\t" \
  "s_add_u32 s90, s75, 2097152\n\t" \
  "buffer_load_dwordx4 v186, s[40:43], s90 offen offset:0  lds\n\t" \
  "s_add_u32 m0, s77, 16384\n\t" \
  "s_add_u32 s90, s75, 2097152\n\t" \
  "buffer_load_dwordx4 v187, s[40:43], s90 offen offset:1024  lds\n\t" \
  "s_add_u32 m0, s78, 98304\n\t" \
  "s_movk_i32 s84, 128\n\t" \
  "buffer_load_dwordx4 v190, s[32:35], s84 offen offset:0 lds\n\t" \
  "s_add_u32 m0, s78, 98304\n\t" \
  "s_movk_i32 s84, 3200\n\t" \
  "buffer_load_dwordx4 v191, s[32:35], s84 offen offset:1024 lds\n\t" \
  "s_add_u32 m0, s78, 98304\n\t" \
  "s_movk_i32 s84, 6272\n\t" \
  "buffer_load_dwordx4 v190, s[32:35], s84 offen offset:2048 lds\n\t" \
  "s_add_u32 m0, s78, 98304\n\t" \
  "s_movk_i32 s84, 9344\n\t" \
  "buffer_load_dwordx4 v191, s[32:35], s84 offen offset:3072 lds\n\t" \
  "s_add_u32 m0, s78, 102400\n\t" \
  "s_movk_i32 s84, 16512\n\t" \
  "buffer_load_dwordx4 v190, s[32:35], s84 offen offset:0 lds\n\t" \
  "s_add_u32 m0, s78, 102400\n\t" \
  "s_movk_i32 s84, 19584\n\t" \
  "buffer_load_dwordx4 v191, s[32:35], s84 offen offset:1024 lds\n\t" \
  "s_add_u32 m0, s78, 102400\n\t" \
  "s_movk_i32 s84, 22656\n\t" \
  "buffer_load_dwordx4 v190, s[32:35], s84 offen offset:2048 lds\n\t" \
  "s_add_u32 m0, s78, 102400\n\t" \
  "s_movk_i32 s84, 25728\n\t" \
  "buffer_load_dwordx4 v191, s[32:35], s84 offen offset:3072 lds\n\t" \
  "v_lshlrev_b32 v194, 2, v213\n\t" \
  "v_lshl_add_u32 v194, v214, 18, v194\n\t" \
  "s_lshl_b32 s86, s80, 22\n\t" \
  "s_lshl_b32 s85, s30, 9\n\t" \
  "s_add_u32 s86, s86, s85\n\t" \
  "s_add_u32 s44, s24, s86\n\t" \
  "s_addc_u32 s45, s25, 0\n\t" \
  "s_and_b32 s45, s45, 0xffff\n\t" \
  "s_mov_b32 s46, 4129280\n\t" \
  "s_mov_b32 s47, 0x20000\n\t" \
  "s_lshl_b32 s86, s98, 2\n\t" \
  "v_add_u32 v197, s86, v214\n\t" \
  "v_lshlrev_b32 v197, 14, v197\n\t" \
  "v_lshl_add_u32 v197, v213, 7, v197\n\t" \
  "s_lshl2_add_u32 s86, s31, s29\n\t" \
  "s_lshl_b32 s86, s86, 4\n\t" \
  "v_add_u32 v241, s86, v213\n\t" \
  "v_lshlrev_b32 v241, 14, v241\n\t" \
  "v_min_u32 v217, 27, v212\n\t" \
  "s_lshl_b32 s86, s80, 7\n\t" \
  "s_add_u32 s86, s86, 6291456\n\t" \
  "v_lshl_add_u32 v195, v217, 2, s86\n\t" \
  "v_lshrrev_b32 v218, 2, v217\n\t" \
  "v_lshlrev_b32 v218, 16, v218\n\t" \
  "v_and_b32 v217, 3, v217\n\t" \
  "v_lshl_add_u32 v218, v217, 11, v218\n\t" \
  "s_lshl_b32 s86, s99, 7\n\t" \
  "v_add_u32 v198, s86, v218\n\t" \
  "v_subrev_u32 v217, 4, v212\n\t" \
  "v_and_b32 v217, 63, v217\n\t" \
  "v_mul_u32_u24 v218, 37, v217\n\t" \
  "v_lshrrev_b32 v218, 8, v218\n\t" \
  "v_mul_u32_u24 v219, 7, v218\n\t" \
  "v_sub_u32 v217, v217, v219\n\t" \
  "v_add3_u32 v217, v217, s28, 1\n\t" \
  "v_and_b32 v217, 7, v217\n\t" \
  "v_lshl_add_u32 v217, v217, 2, v218\n\t" \
  "s_lshl_b32 s84, s28, 2\n\t" \
  "v_add_u32 v218, s84, v212\n\t" \
  "v_cmp_gt_u32 vcc, 4, v212\n\t" \
  "s_nop 1\n\t" \
  "v_cndmask_b32 v217, v217, v218, vcc\n\t" \
  "v_lshlrev_b32 v211, 21, v217\n\t" \
  "s_waitcnt vmcnt(10)\n\t" \
  "s_cmp_eq_u32 s29, 0\n\t" \
  "s_cbranch_scc0 L_fl0_0\n\t" \
  "s_mov_b64 exec, 1\n\t" \
  "buffer_store_dword v208, v209, s[56:59], 0 offen sc0 sc1\n\t" \
  "s_mov_b64 exec, -1\n\t" \
  "L_fl0_0:\n\t" \
  "ds_read_b128 a[0:3], v192 offset:0\n\t" \
  "ds_read_b128 a[4:7], v193 offset:0\n\t" \
  "ds_read_b128 a[8:11], v192 offset:2048\n\t" \
  "ds_read_b128 a[12:15], v193 offset:2048\n\t" \
  "ds_read_b128 a[16:19], v192 offset:4096\n\t" \
  "ds_read_b128 a[20:23], v193 offset:4096\n\t" \
  "ds_read_b128 a[24:27], v192 offset:6144\n\t" \
  "ds_read_b128 a[28:31], v193 offset:6144\n\t" \
  "s_waitcnt lgkmcnt(0)\n\t" \
  "s_add_u32 m0, s76, 32768\n\t" \
  "s_add_u32 s90, s75, 4194304\n\t" \
  "buffer_load_dwordx4 v184, s[40:43], s90 offen offset:0  lds\n\t" \
  "s_add_u32 m0, s76, 32768\n\t" \
  "s_add_u32 s90, s75, 4194304\n\t" \
  "buffer_load_dwordx4 v185, s[40:43], s90 offen offset:1024  lds\n\t" \
  "s_add_u32 m0, s76, 49152\n\t" \
  "s_add_u32 s90, s75, 6291456\n\t" \
  "buffer_load_dwordx4 v184, s[40:43], s90 offen offset:0  lds\n\t" \
  "s_add_u32 m0, s76, 49152\n\t" \
  "s_add_u32 s90, s75, 6291456\n\t" \
  "buffer_load_dwordx4 v185, s[40:43], s90 offen offset:1024  lds\n\t" \
  "s_cmp_eq_u32 s29, 1\n\t" \
  "s_cbranch_scc0 L_ps1_0\n\t" \
  "buffer_store_dwordx4 v[200:203], v188, s[52:55], 0 offen offset:0 sc0 sc1\n\t" \
  "buffer_store_dwordx4 v[204:207], v188, s[52:55], 0 offen offset:1024 sc0 sc1\n\t" \
  "L_ps1_0:\n\t" \
  "s_barrier\n\t" \
  "ds_read_b64_tr_b8 v[128:129], v160 offset:0\n\t" \
  "ds_read_b64_tr_b8 v[130:131], v160 offset:1024\n\t" \
  "ds_read_b64_tr_b8 v[132:133], v160 offset:8192\n\t" \
  "ds_read_b64_tr_b8 v[134:135], v160 offset:9216\n\t" \
  "ds_read_b64_tr_b8 v[136:137], v161 offset:0\n\t" \
  "ds_read_b64_tr_b8 v[138:139], v161 offset:1024\n\t" \
  "ds_read_b64_tr_b8 v[140:141], v161 offset:8192\n\t" \
  "ds_read_b64_tr_b8 v[142:143], v161 offset:9216\n\t" \
  "s_waitcnt lgkmcnt(4)\n\t" \
  "v_mfma_f32_16x16x128_f8f6f4 v[0:3], a[0:7], v[128:135], 0 cbsz:0 blgp:1\n\t" \
  "ds_read_b64_tr_b8 v[144:145], v162 offset:0\n\t" \
  "s_add_u32 m0, s78, 131072\n\t" \
  "s_movk_i32 s84, 256\n\t" \
  "buffer_load_dwordx4 v190, s[32:35], s84 offen offset:0 lds\n\t" \
  "v_mfma_f32_16x16x128_f8f6f4 v[32:35], a[8:15], v[128:135], 0 cbsz:0 blgp:1\n\t" \
  "ds_read_b64_tr_b8 v[146:147], v162 offset:1024\n\t" \
  "s_add_u32 s99, s29, 4\n\t" \
  "s_min_u32 s99, s99, 31\n\t" \
  "v_readlane_b32 s99, v211, s99\n\t" \
  "s_mov_b32 exec_lo, s15\n\t" \
  "s_mov_b32 exec_hi, 0\n\t" \
  "s_cbranch_execz L_c4_0\n\t" \
  "s_nop 1\n\t" \
  "buffer_load_dword v210, v222, s[60:63], s99 offen\n\t" \
  "L_c4_0:\n\t" \
  "s_mov_b64 exec, -1\n\t" \
  "v_mfma_f32_16x16x128_f8f6f4 v[64:67], a[16:23], v[128:135], 0 cbsz:0 blgp:1\n\t" \
  "ds_read_b64_tr_b8 v[148:149], v162 offset:8192\n\t" \
  "s_add_u32 m0, s78, 131072\n\t" \
  "s_movk_i32 s84, 3328\n\t" \
  "buffer_load_dwordx4 v191, s[32:35], s84 offen offset:1024 lds\n\t" \
  "v_mfma_f32_16x16x128_f8f6f4 v[96:99], a[24:31], v[128:135], 0 cbsz:0 blgp:1\n\t" \
  "ds_read_b64_tr_b8 v[150:151], v162 offset:9216\n\t" \
  "s_add_u32 s99, s29, 8\n\t" \
  "s_min_u32 s99, s99, 31\n\t" \
  "v_readlane_b32 s99, v211, s99\n\t" \
  "s_mov_b32 exec_lo, s15\n\t" \
  "s_mov_b32 exec_hi, 0\n\t" \
  "s_cbranch_execz L_c8_0\n\t" \
  "s_nop 1\n\t" \
  "buffer_load_dword v210, v222, s[60:63], s99 offen\n\t" \
  "L_c8_0:\n\t" \
  "s_mov_b64 exec, -1\n\t" \
  "s_waitcnt lgkmcnt(4)\n\t" \
  "v_mfma_f32_16x16x128_f8f6f4 v[4:7], a[0:7], v[136:143], 0 cbsz:0 blgp:1\n\t" \
  "ds_read_b64_tr_b8 v[152:153], v163 offset:0\n\t" \
  "s_add_u32 m0, s78, 131072\n\t" \
  "s_movk_i32 s84, 6400\n\t" \
  "buffer_load_dwordx4 v190, s[32:35], s84 offen offset:2048 lds\n\t" \
  "v_mfma_f32_16x16x128_f8f6f4 v[36:39], a[8:15], v[136:143], 0 cbsz:0 blgp:1\n\t" \
  "ds_read_b64_tr_b8 v[154:155], v163 offset:1024\n\t" \
  "v_mfma_f32_16x16x128_f8f6f4 v[68:71], a[16:23], v[136:143], 0 cbsz:0 blgp:1\n\t" \
  "ds_read_b64_tr_b8 v[156:157], v163 offset:8192\n\t" \
  "s_add_u32 m0, s78, 131072\n\t" \
  "s_movk_i32 s84, 9472\n\t" \
  "buffer_load_dwordx4 v191, s[32:35], s84 offen offset:3072 lds\n\t" \
  "v_mfma_f32_16x16x128_f8f6f4 v[100:103], a[24:31], v[136:143], 0 cbsz:0 blgp:1\n\t" \
  "ds_read_b64_tr_b8 v[158:159], v163 offset:9216\n\t" \
  "s_waitcnt lgkmcnt(4)\n\t" \
  "v_mfma_f32_16x16x128_f8f6f4 v[8:11], a[0:7], v[144:151], 0 cbsz:0 blgp:1\n\t" \
  "ds_read_b64_tr_b8 v[128:129], v164 offset:0\n\t" \
  "s_waitcnt vmcnt(14)\n\t" \
  "ds_read_b128 a[32:35], v192 offset:32768\n\t" \
  "s_add_u32 m0, s78, 135168\n\t" \
  "s_movk_i32 s84, 16640\n\t" \
  "buffer_load_dwordx4 v190, s[32:35], s84 offen offset:0 lds\n\t" \
  "v_mfma_f32_16x16x128_f8f6f4 v[40:43], a[8:15], v[144:151], 0 cbsz:0 blgp:1\n\t" \
  "ds_read_b64_tr_b8 v[130:131], v164 offset:1024\n\t" \
  "ds_read_b128 a[36:39], v193 offset:32768\n\t" \
  "v_mfma_f32_16x16x128_f8f6f4 v[72:75], a[16:23], v[144:151], 0 cbsz:0 blgp:1\n\t" \
  "ds_read_b64_tr_b8 v[132:133], v164 offset:8192\n\t" \
  "s_add_u32 m0, s78, 135168\n\t" \
  "s_movk_i32 s84, 19712\n\t" \
  "buffer_load_dwordx4 v191, s[32:35], s84 offen offset:1024 lds\n\t" \
  "v_mfma_f32_16x16x128_f8f6f4 v[104:107], a[24:31], v[144:151], 0 cbsz:0 blgp:1\n\t" \
  "ds_read_b64_tr_b8 v[134:135], v164 offset:9216\n\t" \
  "s_waitcnt lgkmcnt(6)\n\t" \
  "v_mfma_f32_16x16x128_f8f6f4 v[12:15], a[0:7], v[152:159], 0 cbsz:0 blgp:1\n\t" \
  "ds_read_b64_tr_b8 v[136:137], v165 offset:0\n\t" \
  "s_waitcnt vmcnt(14)\n\t" \
  "ds_read_b128 a[40:43], v192 offset:34816\n\t" \
  "s_add_u32 m0, s78, 135168\n\t" \
  "s_movk_i32 s84, 22784\n\t" \
  "buffer_load_dwordx4 v190, s[32:35], s84 offen offset:2048 lds\n\t" \
  "v_mfma_f32_16x16x128_f8f6f4 v[44:47], a[8:15], v[152:159], 0 cbsz:0 blgp:1\n\t" \
  "ds_read_b64_tr_b8 v[138:139], v165 offset:1024\n\t" \
  "ds_read_b128 a[44:47], v193 offset:34816\n\t" \
  "v_mfma_f32_16x16x128_f8f6f4 v[76:79], a[16:23], v[152:159], 0 cbsz:0 blgp:1\n\t" \
  "ds_read_b64_tr_b8 v[140:141], v165 offset:8192\n\t" \
  "s_add_u32 m0, s78, 135168\n\t" \
  "s_movk_i32 s84, 25856\n\t" \
  "buffer_load_dwordx4 v191, s[32:35], s84 offen offset:3072 lds\n\t" \
  "v_mfma_f32_16x16x128_f8f6f4 v[108:111], a[24:31], v[152:159], 0 cbsz:0 blgp:1\n\t" \
  "ds_read_b64_tr_b8 v[142:143], v165 offset:9216\n\t" \
  "s_waitcnt lgkmcnt(6)\n\t" \
  "v_mfma_f32_16x16x128_f8f6f4 v[16:19], a[0:7], v[128:135], 0 cbsz:0 blgp:1\n\t" \
  "ds_read_b64_tr_b8 v[144:145], v166 offset:0\n\t" \
  "s_waitcnt vmcnt(14)\n\t" \
  "ds_read_b128 a[48:51], v192 offset:36864\n\t" \
  "s_add_u32 m0, s78, 65536\n\t" \
  "s_movk_i32 s84, 384\n\t" \
  "buffer_load_dwordx4 v190, s[32:35], s84 offen offset:0 lds\n\t" \
  "v_mfma_f32_16x16x128_f8f6f4 v[48:51], a[8:15], v[128:135], 0 cbsz:0 blgp:1\n\t" \
  "ds_read_b64_tr_b8 v[146:147], v166 offset:1024\n\t" \
  "ds_read_b128 a[52:55], v193 offset:36864\n\t" \
  "v_mfma_f32_16x16x128_f8f6f4 v[80:83], a[16:23], v[128:135], 0 cbsz:0 blgp:1\n\t" \
  "ds_read_b64_tr_b8 v[148:149], v166 offset:8192\n\t" \
  "s_add_u32 m0, s78, 65536\n\t" \
  "s_movk_i32 s84, 3456\n\t" \
  "buffer_load_dwordx4 v191, s[32:35], s84 offen offset:1024 lds\n\t" \
  "v_mfma_f32_16x16x128_f8f6f4 v[112:115], a[24:31], v[128:135], 0 cbsz:0 blgp:1\n\t" \
  "ds_read_b64_tr_b8 v[150:151], v166 offset:9216\n\t" \
  "s_waitcnt lgkmcnt(6)\n\t" \
  "v_mfma_f32_16x16x128_f8f6f4 v[20:23], a[0:7], v[136:143], 0 cbsz:0 blgp:1\n\t" \
  "ds_read_b64_tr_b8 v[152:153], v167 offset:0\n\t" \
  "s_waitcnt vmcnt(14)\n\t" \
  "ds_read_b128 a[56:59], v192 offset:38912\n\t" \
  "s_add_u32 m0, s78, 65536\n\t" \
  "s_movk_i32 s84, 6528\n\t" \
  "buffer_load_dwordx4 v190, s[32:35], s84 offen offset:2048 lds\n\t" \
  "v_mfma_f32_16x16x128_f8f6f4 v[52:55], a[8:15], v[136:143], 0 cbsz:0 blgp:1\n\t" \
  "ds_read_b64_tr_b8 v[154:155], v167 offset:1024\n\t" \
  "ds_read_b128 a[60:63], v193 offset:38912\n\t" \
  "v_mfma_f32_16x16x128_f8f6f4 v[84:87], a[16:23], v[136:143], 0 cbsz:0 blgp:1\n\t" \
  "ds_read_b64_tr_b8 v[156:157], v167 offset:8192\n\t" \
  "s_add_u32 m0, s78, 65536\n\t" \
  "s_movk_i32 s84, 9600\n\t" \
  "buffer_load_dwordx4 v191, s[32:35], s84 offen offset:3072 lds\n\t" \
  "v_mfma_f32_16x16x128_f8f6f4 v[116:119], a[24:31], v[136:143], 0 cbsz:0 blgp:1\n\t" \
  "ds_read_b64_tr_b8 v[158:159], v167 offset:9216\n\t" \
  "s_waitcnt lgkmcnt(6)\n\t" \
  "v_mfma_f32_16x16x128_f8f6f4 v[24:27], a[0:7], v[144:151], 0 cbsz:0 blgp:1\n\t" \
  "s_waitcnt lgkmcnt(2)\n\t" \
  "s_barrier\n\t" \
  "ds_read_b64_tr_b8 v[128:129], v160 offset:16384\n\t" \
  "v_mfma_f32_16x16x128_f8f6f4 v[56:59], a[8:15], v[144:151], 0 cbsz:0 blgp:1\n\t" \
  "ds_read_b64_tr_b8 v[130:131], v160 offset:17408\n\t" \
  "s_add_u32 m0, s78, 69632\n\t" \
  "s_movk_i32 s84, 16768\n\t" \
  "buffer_load_dwordx4 v190, s[32:35], s84 offen offset:0 lds\n\t" \
  "v_mfma_f32_16x16x128_f8f6f4 v[88:91], a[16:23], v[144:151], 0 cbsz:0 blgp:1\n\t" \
  "ds_read_b64_tr_b8 v[132:133], v160 offset:24576\n\t" \
  "v_mfma_f32_16x16x128_f8f6f4 v[120:123], a[24:31], v[144:151], 0 cbsz:0 blgp:1\n\t" \
  "ds_read_b64_tr_b8 v[134:135], v160 offset:25600\n\t" \
  "s_add_u32 m0, s78, 69632\n\t" \
  "s_movk_i32 s84, 19840\n\t" \
  "buffer_load_dwordx4 v191, s[32:35], s84 offen offset:1024 lds\n\t" \
  "s_waitcnt lgkmcnt(4)\n\t" \
  "v_mfma_f32_16x16x128_f8f6f4 v[28:31], a[0:7], v[152:159], 0 cbsz:0 blgp:1\n\t" \
  "ds_read_b64_tr_b8 v[136:137], v161 offset:16384\n\t" \
  "v_mfma_f32_16x16x128_f8f6f4 v[60:63], a[8:15], v[152:159], 0 cbsz:0 blgp:1\n\t" \
  "ds_read_b64_tr_b8 v[138:139], v161 offset:17408\n\t" \
  "s_add_u32 m0, s78, 69632\n\t" \
  "s_movk_i32 s84, 22912\n\t" \
  "buffer_load_dwordx4 v190, s[32:35], s84 offen offset:2048 lds\n\t" \
  "v_mfma_f32_16x16x128_f8f6f4 v[92:95], a[16:23], v[152:159], 0 cbsz:0 blgp:1\n\t" \
  "ds_read_b64_tr_b8 v[140:141], v161 offset:24576\n\t" \
  "s_cmp_eq_u32 s29, 3\n\t" \
  "s_cbranch_scc0 L_ct_0\n\t" \
  "v_mov_b32 v212, s83\n\t" \
  "s_mov_b64 exec, 1\n\t" \
  "global_store_dword v212, v208, s[26:27] sc1\n\t" \
  "s_mov_b64 exec, -1\n\t" \
  "L_ct_0:\n\t" \
  "v_mfma_f32_16x16x128_f8f6f4 v[124:127], a[24:31], v[152:159], 0 cbsz:0 blgp:1\n\t" \
  "ds_read_b64_tr_b8 v[142:143], v161 offset:25600\n\t" \
  "s_add_u32 m0, s78, 69632\n\t" \
  "s_movk_i32 s84, 25984\n\t" \
  "buffer_load_dwordx4 v191, s[32:35], s84 offen offset:3072 lds\n\t" \
  "s_waitcnt lgkmcnt(4)\n\t" \
  "v_mfma_f32_16x16x128_f8f6f4 v[0:3], a[32:39], v[128:135], v[0:3] cbsz:0 blgp:1\n\t" \
  "ds_read_b64_tr_b8 v[144:145], v162 offset:16384\n\t" \
  "v_mfma_f32_16x16x128_f8f6f4 v[32:35], a[40:47], v[128:135], v[32:35] cbsz:0 blgp:1\n\t" \
  "ds_read_b64_tr_b8 v[146:147], v162 offset:17408\n\t" \
  "v_mfma_f32_16x16x128_f8f6f4 v[64:67], a[48:55], v[128:135], v[64:67] cbsz:0 blgp:1\n\t" \
  "ds_read_b64_tr_b8 v[148:149], v162 offset:24576\n\t" \
  "s_add_u32 m0, s77, 32768\n\t" \
  "s_add_u32 s90, s75, 4194304\n\t" \
  "buffer_load_dwordx4 v186, s[40:43], s90 offen offset:0  lds\n\t" \
  "v_mfma_f32_16x16x128_f8f6f4 v[96:99], a[56:63], v[128:135], v[96:99] cbsz:0 blgp:1\n\t" \
  "ds_read_b64_tr_b8 v[150:151], v162 offset:25600\n\t" \
  "s_add_u32 m0, s77, 32768\n\t" \
  "s_add_u32 s90, s75, 4194304\n\t" \
  "buffer_load_dwordx4 v187, s[40:43], s90 offen offset:1024  lds\n\t" \
  "s_waitcnt lgkmcnt(4)\n\t" \
  "v_mfma_f32_16x16x128_f8f6f4 v[4:7], a[32:39], v[136:143], v[4:7] cbsz:0 blgp:1\n\t" \
  "ds_read_b64_tr_b8 v[152:153], v163 offset:16384\n\t" \
  "s_add_u32 m0, s77, 49152\n\t" \
  "s_add_u32 s90, s75, 6291456\n\t" \
  "buffer_load_dwordx4 v186, s[40:43], s90 offen offset:0  lds\n\t" \
  "v_mfma_f32_16x16x128_f8f6f4 v[36:39], a[40:47], v[136:143], v[36:39] cbsz:0 blgp:1\n\t" \
  "ds_read_b64_tr_b8 v[154:155], v163 offset:17408\n\t" \
  "s_add_u32 m0, s77, 49152\n\t" \
  "s_add_u32 s90, s75, 6291456\n\t" \
  "buffer_load_dwordx4 v187, s[40:43], s90 offen offset:1024  lds\n\t" \
  "v_mfma_f32_16x16x128_f8f6f4 v[68:71], a[48:55], v[136:143], v[68:71] cbsz:0 blgp:1\n\t" \
  "ds_read_b64_tr_b8 v[156:157], v163 offset:24576\n\t" \
  "v_mfma_f32_16x16x128_f8f6f4 v[100:103], a[56:63], v[136:143], v[100:103] cbsz:0 blgp:1\n\t" \
  "ds_read_b64_tr_b8 v[158:159], v163 offset:25600\n\t" \
  "s_waitcnt lgkmcnt(4)\n\t" \
  "v_mfma_f32_16x16x128_f8f6f4 v[8:11], a[32:39], v[144:151], v[8:11] cbsz:0 blgp:1\n\t" \
  "ds_read_b64_tr_b8 v[128:129], v164 offset:16384\n\t" \
  "s_waitcnt vmcnt(18)\n\t" \
  "ds_read_b128 a[64:67], v224 offset:0\n\t" \
  "v_mfma_f32_16x16x128_f8f6f4 v[40:43], a[40:47], v[144:151], v[40:43] cbsz:0 blgp:1\n\t" \
  "ds_read_b64_tr_b8 v[130:131], v164 offset:17408\n\t" \
  "ds_read_b128 a[68:71], v225 offset:0\n\t" \
  "v_mfma_f32_16x16x128_f8f6f4 v[72:75], a[48:55], v[144:151], v[72:75] cbsz:0 blgp:1\n\t" \
  "ds_read_b64_tr_b8 v[132:133], v164 offset:24576\n\t" \
  "s_waitcnt vmcnt(16)\n\t" \
  "ds_read_b128 a[72:75], v224 offset:2048\n\t" \
  "v_mfma_f32_16x16x128_f8f6f4 v[104:107], a[56:63], v[144:151], v[104:107] cbsz:0 blgp:1\n\t" \
  "ds_read_b64_tr_b8 v[134:135], v164 offset:25600\n\t" \
  "ds_read_b128 a[76:79], v225 offset:2048\n\t" \
  "s_waitcnt lgkmcnt(8)\n\t" \
  "v_mfma_f32_16x16x128_f8f6f4 v[12:15], a[32:39], v[152:159], v[12:15] cbsz:0 blgp:1\n\t" \
  "ds_read_b64_tr_b8 v[136:137], v165 offset:16384\n\t" \
  "s_waitcnt vmcnt(14)\n\t" \
  "ds_read_b128 a[80:83], v224 offset:4096\n\t" \
  "v_mfma_f32_16x16x128_f8f6f4 v[44:47], a[40:47], v[152:159], v[44:47] cbsz:0 blgp:1\n\t" \
  "ds_read_b64_tr_b8 v[138:139], v165 offset:17408\n\t" \
  "ds_read_b128 a[84:87], v225 offset:4096\n\t" \
  "v_mfma_f32_16x16x128_f8f6f4 v[76:79], a[48:55], v[152:159], v[76:79] cbsz:0 blgp:1\n\t" \
  "ds_read_b64_tr_b8 v[140:141], v165 offset:24576\n\t" \
  "s_waitcnt vmcnt(12)\n\t" \
  "ds_read_b128 a[88:91], v224 offset:6144\n\t" \
  "v_mfma_f32_16x16x128_f8f6f4 v[108:111], a[56:63], v[152:159], v[108:111] cbsz:0 blgp:1\n\t" \
  "ds_read_b64_tr_b8 v[142:143], v165 offset:25600\n\t" \
  "ds_read_b128 a[92:95], v225 offset:6144\n\t" \
  "s_waitcnt vmcnt(10)\n\t" \
  "ds_read_b128 a[96:99], v192 offset:0\n\t" \
  "s_waitcnt lgkmcnt(10)\n\t" \
  "v_mfma_f32_16x16x128_f8f6f4 v[16:19], a[32:39], v[128:135], v[16:19] cbsz:0 blgp:1\n\t" \
  "ds_read_b64_tr_b8 v[144:145], v166 offset:16384\n\t" \
  "ds_read_b128 a[100:103], v193 offset:0\n\t" \
  "v_mfma_f32_16x16x128_f8f6f4 v[48:51], a[40:47], v[128:135], v[48:51] cbsz:0 blgp:1\n\t" \
  "ds_read_b64_tr_b8 v[146:147], v166 offset:17408\n\t" \
  "s_waitcnt vmcnt(8)\n\t" \
  "ds_read_b128 a[104:107], v192 offset:2048\n\t" \
  "v_mfma_f32_16x16x128_f8f6f4 v[80:83], a[48:55], v[128:135], v[80:83] cbsz:0 blgp:1\n\t" \
  "ds_read_b64_tr_b8 v[148:149], v166 offset:24576\n\t" \
  "ds_read_b128 a[108:111], v193 offset:2048\n\t" \
  "s_mov_b64 exec, 0xfffffff\n\t" \
  "s_mov_b32 s84, 0\n\t" \
  "buffer_load_dword v210, v198, s[36:39], s84 offen sc1\n\t" \
  "s_mov_b64 exec, -1\n\t" \
  "v_mfma_f32_16x16x128_f8f6f4 v[112:115], a[56:63], v[128:135], v[112:115] cbsz:0 blgp:1\n\t" \
  "ds_read_b64_tr_b8 v[150:151], v166 offset:25600\n\t" \
  "s_waitcnt vmcnt(7)\n\t" \
  "ds_read_b128 a[112:115], v192 offset:4096\n\t" \
  "s_mov_b32 s87, 0\n\t" \
  "buffer_load_dwordx4 a[128:131], v188, s[36:39], s87 offen offset:0 sc1\n\t" \
  "s_waitcnt lgkmcnt(10)\n\t" \
  "v_mfma_f32_16x16x128_f8f6f4 v[20:23], a[32:39], v[136:143], v[20:23] cbsz:0 blgp:1\n\t" \
  "ds_read_b64_tr_b8 v[152:153], v167 offset:16384\n\t" \
  "ds_read_b128 a[116:119], v193 offset:4096\n\t" \
  "buffer_load_dwordx4 a[132:135], v188, s[36:39], s87 offen offset:1024 sc1\n\t" \
  "v_mfma_f32_16x16x128_f8f6f4 v[52:55], a[40:47], v[136:143], v[52:55] cbsz:0 blgp:1\n\t" \
  "ds_read_b64_tr_b8 v[154:155], v167 offset:17408\n\t" \
  "s_waitcnt vmcnt(7)\n\t" \
  "ds_read_b128 a[120:123], v192 offset:6144\n\t" \
  "buffer_load_dwordx4 a[136:139], v188, s[36:39], s87 offen offset:2048 sc1\n\t" \
  "v_mfma_f32_16x16x128_f8f6f4 v[84:87], a[48:55], v[136:143], v[84:87] cbsz:0 blgp:1\n\t" \
  "ds_read_b64_tr_b8 v[156:157], v167 offset:24576\n\t" \
  "ds_read_b128 a[124:127], v193 offset:6144\n\t" \
  "buffer_load_dwordx4 a[140:143], v188, s[36:39], s87 offen offset:3072 sc1\n\t" \
  "v_mfma_f32_16x16x128_f8f6f4 v[116:119], a[56:63], v[136:143], v[116:119] cbsz:0 blgp:1\n\t" \
  "ds_read_b64_tr_b8 v[158:159], v167 offset:25600\n\t" \
  "buffer_load_dwordx4 a[144:147], v189, s[36:39], s87 offen offset:0 sc1\n\t" \
  "s_waitcnt lgkmcnt(8)\n\t" \
  "v_mfma_f32_16x16x128_f8f6f4 v[24:27], a[32:39], v[144:151], v[24:27] cbsz:0 blgp:1\n\t" \
  "s_waitcnt vmcnt(8) lgkmcnt(1)\n\t" \
  "s_barrier\n\t" \
  "ds_read_b64_tr_b8 v[128:129], v160 offset:32768\n\t" \
  "v_mfma_f32_16x16x128_f8f6f4 v[56:59], a[40:47], v[144:151], v[56:59] cbsz:0 blgp:1\n\t" \
  "ds_read_b64_tr_b8 v[130:131], v160 offset:33792\n\t" \
  "s_add_u32 m0, s76, 65536\n\t" \
  "s_add_u32 s90, s68, 0\n\t" \
  "buffer_load_dwordx4 v184, s[40:43], s90 offen offset:0  lds\n\t" \
  "s_add_u32 m0, s76, 65536\n\t" \
  "s_add_u32 s90, s68, 0\n\t" \
  "buffer_load_dwordx4 v185, s[40:43], s90 offen offset:1024  lds\n\t" \
  "s_add_u32 m0, s76, 81920\n\t" \
  "s_add_u32 s90, s69, 0\n\t" \
  "buffer_load_dwordx4 v184, s[40:43], s90 offen offset:0  lds\n\t" \
  "s_add_u32 m0, s76, 81920\n\t" \
  "s_add_u32 s90, s69, 0\n\t" \
  "buffer_load_dwordx4 v185, s[40:43], s90 offen offset:1024  lds\n\t" \
  "s_add_u32 s99, s29, 12\n\t" \
  "s_min_u32 s99, s99, 31\n\t" \
  "v_readlane_b32 s99, v211, s99\n\t" \
  "s_mov_b32 exec_lo, s15\n\t" \
  "s_mov_b32 exec_hi, 0\n\t" \
  "s_cbranch_execz L_c12_0\n\t" \
  "s_nop 1\n\t" \
  "buffer_load_dword v210, v222, s[60:63], s99 offen\n\t" \
  "L_c12_0:\n\t" \
  "s_mov_b64 exec, -1\n\t" \
  "v_mfma_f32_16x16x128_f8f6f4 v[88:91], a[48:55], v[144:151], v[88:91] cbsz:0 blgp:1\n\t" \
  "ds_read_b64_tr_b8 v[132:133], v160 offset:40960\n\t" \
  "s_cmp_eq_u32 s29, 2\n\t" \
  "s_cbranch_scc0 L_ps2_0\n\t" \
  "buffer_store_dwordx4 v[200:203], v188, s[52:55], 0 offen offset:0 sc0 sc1\n\t" \
  "buffer_store_dwordx4 v[204:207], v188, s[52:55], 0 offen offset:1024 sc0 sc1\n\t" \
  "L_ps2_0:\n\t" \
  "s_cmp_lt_u32 s29, 2\n\t" \
  "s_cbranch_scc1 L_sk_0\n\t" \
  "s_getpc_b64 s[88:89]\n\t" \
  "L_skp_0:\n\t" \
  "s_lshl_b32 s84, s29, 13\n\t" \
  "s_add_u32 s84, s84, L_slow_0-L_skp_0-16384\n\t" \
  "s_add_u32 s88, s88, s84\n\t" \
  "s_addc_u32 s89, s89, 0\n\t" \
  "s_mov_b64 exec, 0xff\n\t" \
  "global_load_dword v210, v234, s[88:89]\n\t" \
  "s_mov_b64 exec, -1\n\t" \
  "L_sk_0:\n\t" \
  "buffer_load_dwordx4 a[148:151], v189, s[36:39], s87 offen offset:1024 sc1\n\t" \
  "v_mfma_f32_16x16x128_f8f6f4 v[120:123], a[56:63], v[144:151], v[120:123] cbsz:0 blgp:1\n\t" \
  "ds_read_b64_tr_b8 v[134:135], v160 offset:41984\n\t" \
  "buffer_load_dwordx4 a[152:155], v189, s[36:39], s87 offen offset:2048 sc1\n\t" \
  "s_waitcnt lgkmcnt(4)\n\t" \
  "v_mfma_f32_16x16x128_f8f6f4 v[28:31], a[32:39], v[152:159], v[28:31] cbsz:0 blgp:1\n\t" \
  "ds_read_b64_tr_b8 v[136:137], v161 offset:32768\n\t" \
  "s_cmp_eq_u32 s29, 1\n\t" \
  "s_cbranch_scc0 L_fl1_0\n\t" \
  "s_waitcnt vmcnt(32)\n\t" \
  "s_mov_b64 exec, 1\n\t" \
  "buffer_store_dword v208, v209, s[56:59], 0 offen sc0 sc1\n\t" \
  "s_mov_b64 exec, -1\n\t" \
  "L_fl1_0:\n\t" \
  "buffer_load_dwordx4 a[156:159], v189, s[36:39], s87 offen offset:3072 sc1\n\t" \
  "v_mfma_f32_16x16x128_f8f6f4 v[60:63], a[40:47], v[152:159], v[60:63] cbsz:0 blgp:1\n\t" \
  "ds_read_b64_tr_b8 v[138:139], v161 offset:33792\n\t" \
  "v_mfma_f32_16x16x128_f8f6f4 v[92:95], a[48:55], v[152:159], v[92:95] cbsz:0 blgp:1\n\t" \
  "ds_read_b64_tr_b8 v[140:141], v161 offset:40960\n\t" \
  "v_mfma_f32_16x16x128_f8f6f4 v[124:127], a[56:63], v[152:159], v[124:127] cbsz:0 blgp:1\n\t" \
  "ds_read_b64_tr_b8 v[142:143], v161 offset:41984\n\t" \
  "s_waitcnt lgkmcnt(4)\n\t" \
  "v_mfma_f32_16x16x128_f8f6f4 v[0:3], a[64:71], v[128:135], v[0:3] cbsz:0 blgp:1\n\t" \
  "ds_read_b64_tr_b8 v[144:145], v162 offset:32768\n\t" \
  "v_mfma_f32_16x16x128_f8f6f4 v[32:35], a[72:79], v[128:135], v[32:35] cbsz:0 blgp:1\n\t" \
  "ds_read_b64_tr_b8 v[146:147], v162 offset:33792\n\t" \
  "v_mfma_f32_16x16x128_f8f6f4 v[64:67], a[80:87], v[128:135], v[64:67] cbsz:0 blgp:1\n\t" \
  "ds_read_b64_tr_b8 v[148:149], v162 offset:40960\n\t" \
  "v_mfma_f32_16x16x128_f8f6f4 v[96:99], a[88:95], v[128:135], v[96:99] cbsz:0 blgp:1\n\t" \
  "ds_read_b64_tr_b8 v[150:151], v162 offset:41984\n\t" \
  "s_waitcnt lgkmcnt(4)\n\t" \
  "v_mfma_f32_16x16x128_f8f6f4 v[4:7], a[64:71], v[136:143], v[4:7] cbsz:0 blgp:1\n\t" \
  "ds_read_b64_tr_b8 v[152:153], v163 offset:32768\n\t" \
  "v_mfma_f32_16x16x128_f8f6f4 v[36:39], a[72:79], v[136:143], v[36:39] cbsz:0 blgp:1\n\t" \
  "ds_read_b64_tr_b8 v[154:155], v163 offset:33792\n\t" \
  "v_mfma_f32_16x16x128_f8f6f4 v[68:71], a[80:87], v[136:143], v[68:71] cbsz:0 blgp:1\n\t" \
  "ds_read_b64_tr_b8 v[156:157], v163 offset:40960\n\t" \
  "v_mfma_f32_16x16x128_f8f6f4 v[100:103], a[88:95], v[136:143], v[100:103] cbsz:0 blgp:1\n\t" \
  "ds_read_b64_tr_b8 v[158:159], v163 offset:41984\n\t" \
  "s_waitcnt lgkmcnt(4)\n\t" \
  "v_mfma_f32_16x16x128_f8f6f4 v[8:11], a[64:71], v[144:151], v[8:11] cbsz:0 blgp:1\n\t" \
  "ds_read_b64_tr_b8 v[128:129], v164 offset:32768\n\t" \
  "s_waitcnt vmcnt(11)\n\t" \
  "v_accvgpr_read_b32 v229, a128\n\t" \
  "v_accvgpr_read_b32 v230, a129\n\t" \
  "v_mfma_f32_16x16x128_f8f6f4 v[40:43], a[72:79], v[144:151], v[40:43] cbsz:0 blgp:1\n\t" \
  "ds_read_b64_tr_b8 v[130:131], v164 offset:33792\n\t" \
  "v_accvgpr_read_b32 v231, a130\n\t" \
  "v_accvgpr_read_b32 v232, a131\n\t" \
  "v_mfma_f32_16x16x128_f8f6f4 v[72:75], a[80:87], v[144:151], v[72:75] cbsz:0 blgp:1\n\t" \
  "ds_read_b64_tr_b8 v[132:133], v164 offset:40960\n\t" \
  "v_min3_u32 v228, v229, v230, v229\n\t" \
  "v_min3_u32 v228, v231, v232, v228\n\t" \
  "v_mfma_f32_16x16x128_f8f6f4 v[104:107], a[88:95], v[144:151], v[104:107] cbsz:0 blgp:1\n\t" \
  "ds_read_b64_tr_b8 v[134:135], v164 offset:41984\n\t" \
  "s_waitcnt vmcnt(10)\n\t" \
  "v_accvgpr_read_b32 v229, a132\n\t" \
  "v_accvgpr_read_b32 v230, a133\n\t" \
  "s_waitcnt lgkmcnt(4)\n\t" \
  "v_mfma_f32_16x16x128_f8f6f4 v[12:15], a[64:71], v[152:159], v[12:15] cbsz:0 blgp:1\n\t" \
  "ds_read_b64_tr_b8 v[136:137], v165 offset:32768\n\t" \
  "v_accvgpr_read_b32 v231, a134\n\t" \
  "v_accvgpr_read_b32 v232, a135\n\t" \
  "v_mfma_f32_16x16x128_f8f6f4 v[44:47], a[72:79], v[152:159], v[44:47] cbsz:0 blgp:1\n\t" \
  "ds_read_b64_tr_b8 v[138:139], v165 offset:33792\n\t" \
  "v_min3_u32 v228, v229, v230, v228\n\t" \
  "v_min3_u32 v228, v231, v232, v228\n\t" \
  "v_mfma_f32_16x16x128_f8f6f4 v[76:79], a[80:87], v[152:159], v[76:79] cbsz:0 blgp:1\n\t" \
  "ds_read_b64_tr_b8 v[140:141], v165 offset:40960\n\t" \
  "s_waitcnt vmcnt(9)\n\t" \
  "v_accvgpr_read_b32 v229, a136\n\t" \
  "v_accvgpr_read_b32 v230, a137\n\t" \
  "v_mfma_f32_16x16x128_f8f6f4 v[108:111], a[88:95], v[152:159], v[108:111] cbsz:0 blgp:1\n\t" \
  "ds_read_b64_tr_b8 v[142:143], v165 offset:41984\n\t" \
  "v_accvgpr_read_b32 v231, a138\n\t" \
  "v_accvgpr_read_b32 v232, a139\n\t" \
  "s_waitcnt lgkmcnt(4)\n\t" \
  "v_mfma_f32_16x16x128_f8f6f4 v[16:19], a[64:71], v[128:135], v[16:19] cbsz:0 blgp:1\n\t" \
  "ds_read_b64_tr_b8 v[144:145], v166 offset:32768\n\t" \
  "v_min3_u32 v228, v229, v230, v228\n\t" \
  "v_min3_u32 v228, v231, v232, v228\n\t" \
  "v_mfma_f32_16x16x128_f8f6f4 v[48:51], a[72:79], v[128:135], v[48:51] cbsz:0 blgp:1\n\t" \
  "ds_read_b64_tr_b8 v[146:147], v166 offset:33792\n\t" \
  "s_waitcnt vmcnt(8)\n\t" \
  "v_accvgpr_read_b32 v229, a140\n\t" \
  "v_accvgpr_read_b32 v230, a141\n\t" \
  "s_add_u32 m0, s77, 65536\n\t" \
  "s_add_u32 s90, s68, 0\n\t" \
  "buffer_load_dwordx4 v186, s[40:43], s90 offen offset:0  lds\n\t" \
  "v_mfma_f32_16x16x128_f8f6f4 v[80:83], a[80:87], v[128:135], v[80:83] cbsz:0 blgp:1\n\t" \
  "ds_read_b64_tr_b8 v[148:149], v166 offset:40960\n\t" \
  "v_accvgpr_read_b32 v231, a142\n\t" \
  "v_accvgpr_read_b32 v232, a143\n\t" \
  "v_mfma_f32_16x16x128_f8f6f4 v[112:115], a[88:95], v[128:135], v[112:115] cbsz:0 blgp:1\n\t" \
  "ds_read_b64_tr_b8 v[150:151], v166 offset:41984\n\t" \
  "v_min3_u32 v228, v229, v230, v228\n\t" \
  "v_min3_u32 v228, v231, v232, v228\n\t" \
  "s_add_u32 m0, s77, 65536\n\t" \
  "s_add_u32 s90, s68, 0\n\t" \
  "buffer_load_dwordx4 v187, s[40:43], s90 offen offset:1024  lds\n\t" \
  "s_waitcnt lgkmcnt(4)\n\t" \
  "v_mfma_f32_16x16x128_f8f6f4 v[20:23], a[64:71], v[136:143], v[20:23] cbsz:0 blgp:1\n\t" \
  "ds_read_b64_tr_b8 v[152:153], v167 offset:32768\n\t" \
  "s_waitcnt vmcnt(9)\n\t" \
  "v_accvgpr_read_b32 v229, a144\n\t" \
  "v_accvgpr_read_b32 v230, a145\n\t" \
  "v_mfma_f32_16x16x128_f8f6f4 v[52:55], a[72:79], v[136:143], v[52:55] cbsz:0 blgp:1\n\t" \
  "ds_read_b64_tr_b8 v[154:155], v167 offset:33792\n\t" \
  "v_accvgpr_read_b32 v231, a146\n\t" \
  "v_accvgpr_read_b32 v232, a147\n\t" \
  "s_add_u32 m0, s77, 81920\n\t" \
  "s_add_u32 s90, s69, 0\n\t" \
  "buffer_load_dwordx4 v186, s[40:43], s90 offen offset:0  lds\n\t" \
  "v_mfma_f32_16x16x128_f8f6f4 v[84:87], a[80:87], v[136:143], v[84:87] cbsz:0 blgp:1\n\t" \
  "ds_read_b64_tr_b8 v[156:157], v167 offset:40960\n\t" \
  "v_min3_u32 v228, v229, v230, v228\n\t" \
  "v_min3_u32 v228, v231, v232, v228\n\t" \
  "v_mfma_f32_16x16x128_f8f6f4 v[116:119], a[88:95], v[136:143], v[116:119] cbsz:0 blgp:1\n\t" \
  "ds_read_b64_tr_b8 v[158:159], v167 offset:41984\n\t" \
  "s_waitcnt vmcnt(5)\n\t" \
  "v_accvgpr_read_b32 v229, a148\n\t" \
  "v_accvgpr_read_b32 v230, a149\n\t" \
  "s_add_u32 m0, s77, 81920\n\t" \
  "s_add_u32 s90, s69, 0\n\t" \
  "buffer_load_dwordx4 v187, s[40:43], s90 offen offset:1024  lds\n\t" \
  "s_waitcnt lgkmcnt(4)\n\t" \
  "v_mfma_f32_16x16x128_f8f6f4 v[24:27], a[64:71], v[144:151], v[24:27] cbsz:0 blgp:1\n\t" \
  "s_barrier\n\t" \
  "ds_read_b64_tr_b8 v[128:129], v160 offset:49152\n\t" \
  "v_accvgpr_read_b32 v231, a150\n\t" \
  "v_accvgpr_read_b32 v232, a151\n\t" \
  "v_mfma_f32_16x16x128_f8f6f4 v[56:59], a[72:79], v[144:151], v[56:59] cbsz:0 blgp:1\n\t" \
  "ds_read_b64_tr_b8 v[130:131], v160 offset:50176\n\t" \
  "v_min3_u32 v228, v229, v230, v228\n\t" \
  "v_min3_u32 v228, v231, v232, v228\n\t" \
  "v_mfma_f32_16x16x128_f8f6f4 v[88:91], a[80:87], v[144:151], v[88:91] cbsz:0 blgp:1\n\t" \
  "ds_read_b64_tr_b8 v[132:133], v160 offset:57344\n\t" \
  "s_waitcnt vmcnt(5)\n\t" \
  "v_accvgpr_read_b32 v229, a152\n\t" \
  "v_accvgpr_read_b32 v230, a153\n\t" \
  "v_mfma_f32_16x16x128_f8f6f4 v[120:123], a[88:95], v[144:151], v[120:123] cbsz:0 blgp:1\n\t" \
  "ds_read_b64_tr_b8 v[134:135], v160 offset:58368\n\t" \
  "v_accvgpr_read_b32 v231, a154\n\t" \
  "v_accvgpr_read_b32 v232, a155\n\t" \
  "s_waitcnt lgkmcnt(4)\n\t" \
  "v_mfma_f32_16x16x128_f8f6f4 v[28:31], a[64:71], v[152:159], v[28:31] cbsz:0 blgp:1\n\t" \
  "ds_read_b64_tr_b8 v[136:137], v161 offset:49152\n\t" \
  "v_min3_u32 v228, v229, v230, v228\n\t" \
  "v_min3_u32 v228, v231, v232, v228\n\t" \
  "s_add_u32 m0, s76, 98304\n\t" \
  "s_add_u32 s90, s70, 0\n\t" \
  "buffer_load_dwordx4 v184, s[40:43], s90 offen offset:0  lds\n\t" \
  "v_mfma_f32_16x16x128_f8f6f4 v[60:63], a[72:79], v[152:159], v[60:63] cbsz:0 blgp:1\n\t" \
  "ds_read_b64_tr_b8 v[138:139], v161 offset:50176\n\t" \
  "s_waitcnt vmcnt(5)\n\t" \
  "v_accvgpr_read_b32 v229, a156\n\t" \
  "v_accvgpr_read_b32 v230, a157\n\t" \
  "s_add_u32 m0, s76, 98304\n\t" \
  "s_add_u32 s90, s70, 0\n\t" \
  "buffer_load_dwordx4 v185, s[40:43], s90 offen offset:1024  lds\n\t" \
  "v_mfma_f32_16x16x128_f8f6f4 v[92:95], a[80:87], v[152:159], v[92:95] cbsz:0 blgp:1\n\t" \
  "ds_read_b64_tr_b8 v[140:141], v161 offset:57344\n\t" \
  "v_accvgpr_read_b32 v231, a158\n\t" \
  "v_accvgpr_read_b32 v232, a159\n\t" \
  "s_add_u32 m0, s76, 114688\n\t" \
  "s_add_u32 s90, s71, 0\n\t" \
  "buffer_load_dwordx4 v184, s[40:43], s90 offen offset:0  lds\n\t" \
  "v_mfma_f32_16x16x128_f8f6f4 v[124:127], a[88:95], v[152:159], v[124:127] cbsz:0 blgp:1\n\t" \
  "ds_read_b64_tr_b8 v[142:143], v161 offset:58368\n\t" \
  "v_min3_u32 v228, v229, v230, v228\n\t" \
  "v_min3_u32 v228, v231, v232, v228\n\t" \
  "s_add_u32 m0, s76, 114688\n\t" \
  "s_add_u32 s90, s71, 0\n\t" \
  "buffer_load_dwordx4 v185, s[40:43], s90 offen offset:1024  lds\n\t" \
  "s_waitcnt lgkmcnt(4)\n\t" \
  "v_mfma_f32_16x16x128_f8f6f4 v[0:3], a[96:103], v[128:135], v[0:3] cbsz:0 blgp:1\n\t" \
  "ds_read_b64_tr_b8 v[144:145], v162 offset:49152\n\t" \
  "v_mfma_f32_16x16x128_f8f6f4 v[32:35], a[104:111], v[128:135], v[32:35] cbsz:0 blgp:1\n\t" \
  "ds_read_b64_tr_b8 v[146:147], v162 offset:50176\n\t" \
  "v_cmp_eq_u32 vcc, 0, v228\n\t" \
  "s_nop 4\n\t" \
  "s_cmp_lg_u64 vcc, 0\n\t" \
  "s_cbranch_scc1 L_v4_0\n\t" \
  "L_vr4_0:\n\t" \
  "v_mfma_f32_16x16x128_f8f6f4 v[64:67], a[112:119], v[128:135], v[64:67] cbsz:0 blgp:1\n\t" \
  "ds_read_b64_tr_b8 v[148:149], v162 offset:57344\n\t" \
  "v_mfma_f32_16x16x128_f8f6f4 v[96:99], a[120:127], v[128:135], v[96:99] cbsz:0 blgp:1\n\t" \
  "ds_read_b64_tr_b8 v[150:151], v162 offset:58368\n\t" \
  "s_mov_b32 s87, 65536\n\t" \
  "buffer_load_dwordx4 a[160:163], v188, s[36:39], s87 offen offset:0 sc1\n\t" \
  "s_waitcnt lgkmcnt(4)\n\t" \
  "v_mfma_f32_16x16x128_f8f6f4 v[4:7], a[96:103], v[136:143], v[4:7] cbsz:0 blgp:1\n\t" \
  "ds_read_b64_tr_b8 v[152:153], v163 offset:49152\n\t" \
  "v_mfma_f32_16x16x128_f8f6f4 v[36:39], a[104:111], v[136:143], v[36:39] cbsz:0 blgp:1\n\t" \
  "ds_read_b64_tr_b8 v[154:155], v163 offset:50176\n\t" \
  "buffer_load_dwordx4 a[164:167], v188, s[36:39], s87 offen offset:1024 sc1\n\t" \
  "v_mfma_f32_16x16x128_f8f6f4 v[68:71], a[112:119], v[136:143], v[68:71] cbsz:0 blgp:1\n\t" \
  "ds_read_b64_tr_b8 v[156:157], v163 offset:57344\n\t" \
  "v_mfma_f32_16x16x128_f8f6f4 v[100:103], a[120:127], v[136:143], v[100:103] cbsz:0 blgp:1\n\t" \
  "ds_read_b64_tr_b8 v[158:159], v163 offset:58368\n\t" \
  "buffer_load_dwordx4 a[168:171], v188, s[36:39], s87 offen offset:2048 sc1\n\t" \
  "s_waitcnt lgkmcnt(4)\n\t" \
  "v_mfma_f32_16x16x128_f8f6f4 v[8:11], a[96:103], v[144:151], v[8:11] cbsz:0 blgp:1\n\t" \
  "ds_read_b64_tr_b8 v[128:129], v164 offset:49152\n\t" \
  "v_mfma_f32_16x16x128_f8f6f4 v[40:43], a[104:111], v[144:151], v[40:43] cbsz:0 blgp:1\n\t" \
  "ds_read_b64_tr_b8 v[130:131], v164 offset:50176\n\t" \
  "buffer_load_dwordx4 a[172:175], v188, s[36:39], s87 offen offset:3072 sc1\n\t" \
  "v_mfma_f32_16x16x128_f8f6f4 v[72:75], a[112:119], v[144:151], v[72:75] cbsz:0 blgp:1\n\t" \
  "ds_read_b64_tr_b8 v[132:133], v164 offset:57344\n\t" \
  "v_mfma_f32_16x16x128_f8f6f4 v[104:107], a[120:127], v[144:151], v[104:107] cbsz:0 blgp:1\n\t" \
  "ds_read_b64_tr_b8 v[134:135], v164 offset:58368\n\t" \
  "buffer_load_dwordx4 a[176:179], v189, s[36:39], s87 offen offset:0 sc1\n\t" \
  "s_waitcnt lgkmcnt(4)\n\t" \
  "v_mfma_f32_16x16x128_f8f6f4 v[12:15], a[96:103], v[152:159], v[12:15] cbsz:0 blgp:1\n\t" \
  "ds_read_b64_tr_b8 v[136:137], v165 offset:49152\n\t" \
  "s_cmp_eq_u32 s29, 3\n\t" \
  "s_cbranch_scc0 L_ps3_0\n\t" \
  "buffer_store_dwordx4 v[200:203], v188, s[52:55], 0 offen offset:0 sc0 sc1\n\t" \
  "buffer_store_dwordx4 v[204:207], v188, s[52:55], 0 offen offset:1024 sc0 sc1\n\t" \
  "L_ps3_0:\n\t" \
  "v_mfma_f32_16x16x128_f8f6f4 v[44:47], a[104:111], v[152:159], v[44:47] cbsz:0 blgp:1\n\t" \
  "ds_read_b64_tr_b8 v[138:139], v165 offset:50176\n\t" \
  "buffer_load_dwordx4 a[180:183], v189, s[36:39], s87 offen offset:1024 sc1\n\t" \
  "v_mfma_f32_16x16x128_f8f6f4 v[76:79], a[112:119], v[152:159], v[76:79] cbsz:0 blgp:1\n\t" \
  "ds_read_b64_tr_b8 v[140:141], v165 offset:57344\n\t" \
  "v_mfma_f32_16x16x128_f8f6f4 v[108:111], a[120:127], v[152:159], v[108:111] cbsz:0 blgp:1\n\t" \
  "ds_read_b64_tr_b8 v[142:143], v165 offset:58368\n\t" \
  "buffer_load_dwordx4 a[184:187], v189, s[36:39], s87 offen offset:2048 sc1\n\t" \
  "s_waitcnt lgkmcnt(4)\n\t" \
  "v_mfma_f32_16x16x128_f8f6f4 v[16:19], a[96:103], v[128:135], v[16:19] cbsz:0 blgp:1\n\t" \
  "ds_read_b64_tr_b8 v[144:145], v166 offset:49152\n\t" \
  "v_mfma_f32_16x16x128_f8f6f4 v[48:51], a[104:111], v[128:135], v[48:51] cbsz:0 blgp:1\n\t" \
  "ds_read_b64_tr_b8 v[146:147], v166 offset:50176\n\t" \
  "buffer_load_dwordx4 a[188:191], v189, s[36:39], s87 offen offset:3072 sc1\n\t" \
  "v_mfma_f32_16x16x128_f8f6f4 v[80:83], a[112:119], v[128:135], v[80:83] cbsz:0 blgp:1\n\t" \
  "ds_read_b64_tr_b8 v[148:149], v166 offset:57344\n\t" \
  "v_mfma_f32_16x16x128_f8f6f4 v[112:115], a[120:127], v[128:135], v[112:115] cbsz:0 blgp:1\n\t" \
  "ds_read_b64_tr_b8 v[150:151], v166 offset:58368\n\t" \
  "s_mov_b32 s87, 131072\n\t" \
  "buffer_load_dwordx4 a[192:195], v188, s[36:39], s87 offen offset:0 sc1\n\t" \
  "s_waitcnt lgkmcnt(4)\n\t" \
  "v_mfma_f32_16x16x128_f8f6f4 v[20:23], a[96:103], v[136:143], v[20:23] cbsz:0 blgp:1\n\t" \
  "ds_read_b64_tr_b8 v[152:153], v167 offset:49152\n\t" \
  "v_mfma_f32_16x16x128_f8f6f4 v[52:55], a[104:111], v[136:143], v[52:55] cbsz:0 blgp:1\n\t" \
  "ds_read_b64_tr_b8 v[154:155], v167 offset:50176\n\t" \
  "buffer_load_dwordx4 a[196:199], v188, s[36:39], s87 offen offset:1024 sc1\n\t" \
  "v_mfma_f32_16x16x128_f8f6f4 v[84:87], a[112:119], v[136:143], v[84:87] cbsz:0 blgp:1\n\t" \
  "ds_read_b64_tr_b8 v[156:157], v167 offset:57344\n\t" \
  "v_mfma_f32_16x16x128_f8f6f4 v[116:119], a[120:127], v[136:143], v[116:119] cbsz:0 blgp:1\n\t" \
  "ds_read_b64_tr_b8 v[158:159], v167 offset:58368\n\t" \
  "buffer_load_dwordx4 a[200:203], v188, s[36:39], s87 offen offset:2048 sc1\n\t" \
  "s_waitcnt lgkmcnt(4)\n\t" \
  "v_mfma_f32_16x16x128_f8f6f4 v[24:27], a[96:103], v[144:151], v[24:27] cbsz:0 blgp:1\n\t" \
  "s_waitcnt vmcnt(17)\n\t" \
  "s_barrier\n\t" \
  "ds_read_b64_tr_b8 v[128:129], v168 offset:0\n\t" \
  "v_mfma_f32_16x16x128_f8f6f4 v[56:59], a[104:111], v[144:151], v[56:59] cbsz:0 blgp:1\n\t" \
  "ds_read_b64_tr_b8 v[130:131], v168 offset:1024\n\t" \
  "buffer_load_dwordx4 a[204:207], v188, s[36:39], s87 offen offset:3072 sc1\n\t" \
  "v_mfma_f32_16x16x128_f8f6f4 v[88:91], a[112:119], v[144:151], v[88:91] cbsz:0 blgp:1\n\t" \
  "ds_read_b64_tr_b8 v[132:133], v168 offset:8192\n\t" \
  "s_cmp_eq_u32 s29, 2\n\t" \
  "s_cbranch_scc0 L_fl2_0\n\t" \
  "s_waitcnt vmcnt(23)\n\t" \
  "s_mov_b64 exec, 1\n\t" \
  "buffer_store_dword v208, v209, s[56:59], 0 offen sc0 sc1\n\t" \
  "s_mov_b64 exec, -1\n\t" \
  "L_fl2_0:\n\t" \
  "v_mfma_f32_16x16x128_f8f6f4 v[120:123], a[120:127], v[144:151], v[120:123] cbsz:0 blgp:1\n\t" \
  "ds_read_b64_tr_b8 v[134:135], v168 offset:9216\n\t" \
  "buffer_load_dwordx4 a[208:211], v189, s[36:39], s87 offen offset:0 sc1\n\t" \
  "s_waitcnt lgkmcnt(4)\n\t" \
  "v_mfma_f32_16x16x128_f8f6f4 v[28:31], a[96:103], v[152:159], v[28:31] cbsz:0 blgp:1\n\t" \
  "ds_read_b64_tr_b8 v[136:137], v169 offset:0\n\t" \
  "v_mfma_f32_16x16x128_f8f6f4 v[60:63], a[104:111], v[152:159], v[60:63] cbsz:0 blgp:1\n\t" \
  "ds_read_b64_tr_b8 v[138:139], v169 offset:1024\n\t" \
  "s_waitcnt vmcnt(12)\n\t" \
  "v_accvgpr_read_b32 v229, a160\n\t" \
  "v_accvgpr_read_b32 v230, a161\n\t" \
  "buffer_load_dwordx4 a[212:215], v189, s[36:39], s87 offen offset:1024 sc1\n\t" \
  "v_mfma_f32_16x16x128_f8f6f4 v[92:95], a[112:119], v[152:159], v[92:95] cbsz:0 blgp:1\n\t" \
  "ds_read_b64_tr_b8 v[140:141], v169 offset:8192\n\t" \
  "v_accvgpr_read_b32 v231, a162\n\t" \
  "v_accvgpr_read_b32 v232, a163\n\t" \
  "v_mfma_f32_16x16x128_f8f6f4 v[124:127], a[120:127], v[152:159], v[124:127] cbsz:0 blgp:1\n\t" \
  "ds_read_b64_tr_b8 v[142:143], v169 offset:9216\n\t" \
  "v_min3_u32 v228, v229, v230, v229\n\t" \
  "v_min3_u32 v228, v231, v232, v228\n\t" \
  "buffer_load_dwordx4 a[216:219], v189, s[36:39], s87 offen offset:2048 sc1\n\t" \
  "s_waitcnt lgkmcnt(4)\n\t" \
  "v_mfma_f32_16x16x128_f8f6f4 v[0:3], a[128:135], v[128:135], v[0:3] cbsz:0 blgp:1\n\t" \
  "ds_read_b64_tr_b8 v[144:145], v170 offset:0\n\t" \
  "s_waitcnt vmcnt(13)\n\t" \
  "v_accvgpr_read_b32 v229, a164\n\t" \
  "v_accvgpr_read_b32 v230, a165\n\t" \
  "v_mfma_f32_16x16x128_f8f6f4 v[32:35], a[136:143], v[128:135], v[32:35] cbsz:0 blgp:1\n\t" \
  "ds_read_b64_tr_b8 v[146:147], v170 offset:1024\n\t" \
  "v_accvgpr_read_b32 v231, a166\n\t" \
  "v_accvgpr_read_b32 v232, a167\n\t" \
  "buffer_load_dwordx4 a[220:223], v189, s[36:39], s87 offen offset:3072 sc1\n\t" \
  "v_mfma_f32_16x16x128_f8f6f4 v[64:67], a[144:151], v[128:135], v[64:67] cbsz:0 blgp:1\n\t" \
  "ds_read_b64_tr_b8 v[148:149], v170 offset:8192\n\t" \
  "v_min3_u32 v228, v229, v230, v228\n\t" \
  "v_min3_u32 v228, v231, v232, v228\n\t" \
  "v_mfma_f32_16x16x128_f8f6f4 v[96:99], a[152:159], v[128:135], v[96:99] cbsz:0 blgp:1\n\t" \
  "ds_read_b64_tr_b8 v[150:151], v170 offset:9216\n\t" \
  "s_waitcnt vmcnt(13)\n\t" \
  "v_accvgpr_read_b32 v229, a168\n\t" \
  "v_accvgpr_read_b32 v230, a169\n\t" \
  "s_mov_b32 s87, 196608\n\t" \
  "buffer_load_dwordx4 a[224:227], v188, s[36:39], s87 offen offset:0 sc1\n\t" \
  "s_waitcnt lgkmcnt(4)\n\t" \
  "v_mfma_f32_16x16x128_f8f6f4 v[4:7], a[128:135], v[136:143], v[4:7] cbsz:0 blgp:1\n\t" \
  "ds_read_b64_tr_b8 v[152:153], v171 offset:0\n\t" \
  "v_accvgpr_read_b32 v231, a170\n\t" \
  "v_accvgpr_read_b32 v232, a171\n\t" \
  "v_mfma_f32_16x16x128_f8f6f4 v[36:39], a[136:143], v[136:143], v[36:39] cbsz:0 blgp:1\n\t" \
  "ds_read_b64_tr_b8 v[154:155], v171 offset:1024\n\t" \
  "v_min3_u32 v228, v229, v230, v228\n\t" \
  "v_min3_u32 v228, v231, v232, v228\n\t" \
  "v_mfma_f32_16x16x128_f8f6f4 v[68:71], a[144:151], v[136:143], v[68:71] cbsz:0 blgp:1\n\t" \
  "ds_read_b64_tr_b8 v[156:157], v171 offset:8192\n\t" \
  "s_waitcnt vmcnt(13)\n\t" \
  "v_accvgpr_read_b32 v229, a172\n\t" \
  "v_accvgpr_read_b32 v230, a173\n\t" \
  "buffer_load_dwordx4 a[228:231], v188, s[36:39], s87 offen offset:1024 sc1\n\t" \
  "v_mfma_f32_16x16x128_f8f6f4 v[100:103], a[152:159], v[136:143], v[100:103] cbsz:0 blgp:1\n\t" \
  "ds_read_b64_tr_b8 v[158:159], v171 offset:9216\n\t" \
  "v_accvgpr_read_b32 v231, a174\n\t" \
  "v_accvgpr_read_b32 v232, a175\n\t" \
  "s_waitcnt lgkmcnt(4)\n\t" \
  "v_mfma_f32_16x16x128_f8f6f4 v[8:11], a[128:135], v[144:151], v[8:11] cbsz:0 blgp:1\n\t" \
  "ds_read_b64_tr_b8 v[128:129], v172 offset:0\n\t" \
  "v_min3_u32 v228, v229, v230, v228\n\t" \
  "v_min3_u32 v228, v231, v232, v228\n\t" \
  "v_mfma_f32_16x16x128_f8f6f4 v[40:43], a[136:143], v[144:151], v[40:43] cbsz:0 blgp:1\n\t" \
  "ds_read_b64_tr_b8 v[130:131], v172 offset:1024\n\t" \
  "s_waitcnt vmcnt(13)\n\t" \
  "v_accvgpr_read_b32 v229, a176\n\t" \
  "v_accvgpr_read_b32 v230, a177\n\t" \
  "buffer_load_dwordx4 a[232:235], v188, s[36:39], s87 offen offset:2048 sc1\n\t" \
  "v_mfma_f32_16x16x128_f8f6f4 v[72:75], a[144:151], v[144:151], v[72:75] cbsz:0 blgp:1\n\t" \
  "ds_read_b64_tr_b8 v[132:133], v172 offset:8192\n\t" \
  "v_accvgpr_read_b32 v231, a178\n\t" \
  "v_accvgpr_read_b32 v232, a179\n\t" \
  "v_mfma_f32_16x16x128_f8f6f4 v[104:107], a[152:159], v[144:151], v[104:107] cbsz:0 blgp:1\n\t" \
  "ds_read_b64_tr_b8 v[134:135], v172 offset:9216\n\t" \
  "v_min3_u32 v228, v229, v230, v228\n\t" \
  "v_min3_u32 v228, v231, v232, v228\n\t" \
  "s_waitcnt lgkmcnt(4)\n\t" \
  "v_mfma_f32_16x16x128_f8f6f4 v[12:15], a[128:135], v[152:159], v[12:15] cbsz:0 blgp:1\n\t" \
  "ds_read_b64_tr_b8 v[136:137], v173 offset:0\n\t" \
  "s_waitcnt vmcnt(13)\n\t" \
  "v_accvgpr_read_b32 v229, a180\n\t" \
  "v_accvgpr_read_b32 v230, a181\n\t" \
  "buffer_load_dwordx4 a[236:239], v188, s[36:39], s87 offen offset:3072 sc1\n\t" \
  "v_mfma_f32_16x16x128_f8f6f4 v[44:47], a[136:143], v[152:159], v[44:47] cbsz:0 blgp:1\n\t" \
  "ds_read_b64_tr_b8 v[138:139], v173 offset:1024\n\t" \
  "v_accvgpr_read_b32 v231, a182\n\t" \
  "v_accvgpr_read_b32 v232, a183\n\t" \
  "v_mfma_f32_16x16x128_f8f6f4 v[76:79], a[144:151], v[152:159], v[76:79] cbsz:0 blgp:1\n\t" \
  "ds_read_b64_tr_b8 v[140:141], v173 offset:8192\n\t" \
  "v_min3_u32 v228, v229, v230, v228\n\t" \
  "v_min3_u32 v228, v231, v232, v228\n\t" \
  "v_mfma_f32_16x16x128_f8f6f4 v[108:111], a[152:159], v[152:159], v[108:111] cbsz:0 blgp:1\n\t" \
  "ds_read_b64_tr_b8 v[142:143], v173 offset:9216\n\t" \
  "s_waitcnt vmcnt(13)\n\t" \
  "v_accvgpr_read_b32 v229, a184\n\t" \
  "v_accvgpr_read_b32 v230, a185\n\t" \
  "s_add_u32 m0, s77, 98304\n\t" \
  "s_add_u32 s90, s70, 0\n\t" \
  "buffer_load_dwordx4 v186, s[40:43], s90 offen offset:0  lds\n\t" \
  "s_waitcnt lgkmcnt(4)\n\t" \
  "v_mfma_f32_16x16x128_f8f6f4 v[16:19], a[128:135], v[128:135], v[16:19] cbsz:0 blgp:1\n\t" \
  "ds_read_b64_tr_b8 v[144:145], v174 offset:0\n\t" \
  "v_accvgpr_read_b32 v231, a186\n\t" \
  "v_accvgpr_read_b32 v232, a187\n\t" \
  "s_add_u32 m0, s77, 114688\n\t" \
  "s_add_u32 s90, s71, 0\n\t" \
  "buffer_load_dwordx4 v186, s[40:43], s90 offen offset:0  lds\n\t" \
  "v_mfma_f32_16x16x128_f8f6f4 v[48:51], a[136:143], v[128:135], v[48:51] cbsz:0 blgp:1\n\t" \
  "ds_read_b64_tr_b8 v[146:147], v174 offset:1024\n\t" \
  "v_min3_u32 v228, v229, v230, v228\n\t" \
  "v_min3_u32 v228, v231, v232, v228\n\t" \
  "s_add_u32 m0, s77, 98304\n\t" \
  "s_add_u32 s90, s70, 0\n\t" \
  "buffer_load_dwordx4 v187, s[40:43], s90 offen offset:1024  lds\n\t" \
  "v_mfma_f32_16x16x128_f8f6f4 v[80:83], a[144:151], v[128:135], v[80:83] cbsz:0 blgp:1\n\t" \
  "ds_read_b64_tr_b8 v[148:149], v174 offset:8192\n\t" \
  "s_waitcnt vmcnt(15)\n\t" \
  "v_accvgpr_read_b32 v229, a188\n\t" \
  "v_accvgpr_read_b32 v230, a189\n\t" \
  "s_add_u32 m0, s76, 131072\n\t" \
  "s_add_u32 s90, s72, 0\n\t" \
  "buffer_load_dwordx4 v184, s[40:43], s90 offen offset:0  lds\n\t" \
  "v_mfma_f32_16x16x128_f8f6f4 v[112:115], a[152:159], v[128:135], v[112:115] cbsz:0 blgp:1\n\t" \
  "ds_read_b64_tr_b8 v[150:151], v174 offset:9216\n\t" \
  "v_accvgpr_read_b32 v231, a190\n\t" \
  "v_accvgpr_read_b32 v232, a191\n\t" \
  "s_add_u32 m0, s76, 131072\n\t" \
  "s_add_u32 s90, s72, 0\n\t" \
  "buffer_load_dwordx4 v185, s[40:43], s90 offen offset:1024  lds\n\t" \
  "s_waitcnt lgkmcnt(4)\n\t" \
  "v_mfma_f32_16x16x128_f8f6f4 v[20:23], a[128:135], v[136:143], v[20:23] cbsz:0 blgp:1\n\t" \
  "ds_read_b64_tr_b8 v[152:153], v175 offset:0\n\t" \
  "v_min3_u32 v228, v229, v230, v228\n\t" \
  "v_min3_u32 v228, v231, v232, v228\n\t" \
  "s_cmp_gt_u32 s29, 1\n\t" \
  "s_cbranch_scc1 L_k148_0\n\t" \
  "s_getpc_b64 s[88:89]\n\t" \
  "s_lshl_b32 s84, s29, 13\n\t" \
  "s_add_u32 s84, s84, 2048\n\t" \
  "s_add_u32 s88, s88, s84\n\t" \
  "s_addc_u32 s89, s89, 0\n\t" \
  "s_mov_b64 exec, 0xff\n\t" \
  "global_load_dword v210, v234, s[88:89]\n\t" \
  "s_mov_b64 exec, -1\n\t" \
  "L_k148_0:\n\t" \
  "s_add_u32 m0, s76, 147456\n\t" \
  "s_add_u32 s90, s73, 0\n\t" \
  "buffer_load_dwordx4 v184, s[40:43], s90 offen offset:0  lds\n\t" \
  "v_mfma_f32_16x16x128_f8f6f4 v[52:55], a[136:143], v[136:143], v[52:55] cbsz:0 blgp:1\n\t" \
  "ds_read_b64_tr_b8 v[154:155], v175 offset:1024\n\t" \
  "s_add_u32 m0, s76, 147456\n\t" \
  "s_add_u32 s90, s73, 0\n\t" \
  "buffer_load_dwordx4 v185, s[40:43], s90 offen offset:1024  lds\n\t" \
  "v_mfma_f32_16x16x128_f8f6f4 v[84:87], a[144:151], v[136:143], v[84:87] cbsz:0 blgp:1\n\t" \
  "ds_read_b64_tr_b8 v[156:157], v175 offset:8192\n\t" \
  "v_cmp_eq_u32 vcc, 0, v228\n\t" \
  "s_nop 4\n\t" \
  "s_cmp_lg_u64 vcc, 0\n\t" \
  "s_cbranch_scc1 L_v5_0\n\t" \
  "L_vr5_0:\n\t" \
  "s_add_u32 m0, s76, 0\n\t" \
  "s_add_u32 s90, s74, 0\n\t" \
  "buffer_load_dwordx4 v184, s[40:43], s90 offen offset:0  lds\n\t" \
  "v_mfma_f32_16x16x128_f8f6f4 v[116:119], a[152:159], v[136:143], v[116:119] cbsz:0 blgp:1\n\t" \
  "ds_read_b64_tr_b8 v[158:159], v175 offset:9216\n\t" \
  "s_add_u32 m0, s76, 0\n\t" \
  "s_add_u32 s90, s74, 0\n\t" \
  "buffer_load_dwordx4 v185, s[40:43], s90 offen offset:1024  lds\n\t" \
  "s_waitcnt lgkmcnt(4)\n\t" \
  "v_mfma_f32_16x16x128_f8f6f4 v[24:27], a[128:135], v[144:151], v[24:27] cbsz:0 blgp:1\n\t" \
  "s_barrier\n\t" \
  "ds_read_b64_tr_b8 v[128:129], v168 offset:16384\n\t" \
  "v_mfma_f32_16x16x128_f8f6f4 v[56:59], a[136:143], v[144:151], v[56:59] cbsz:0 blgp:1\n\t" \
  "ds_read_b64_tr_b8 v[130:131], v168 offset:17408\n\t" \
  "s_add_u32 m0, s76, 16384\n\t" \
  "s_add_u32 s90, s68, 2097152\n\t" \
  "buffer_load_dwordx4 v184, s[40:43], s90 offen offset:0  lds\n\t" \
  "v_mfma_f32_16x16x128_f8f6f4 v[88:91], a[144:151], v[144:151], v[88:91] cbsz:0 blgp:1\n\t" \
  "ds_read_b64_tr_b8 v[132:133], v168 offset:24576\n\t" \
  "s_add_u32 m0, s76, 16384\n\t" \
  "s_add_u32 s90, s68, 2097152\n\t" \
  "buffer_load_dwordx4 v185, s[40:43], s90 offen offset:1024  lds\n\t" \
  "v_mfma_f32_16x16x128_f8f6f4 v[120:123], a[152:159], v[144:151], v[120:123] cbsz:0 blgp:1\n\t" \
  "ds_read_b64_tr_b8 v[134:135], v168 offset:25600\n\t" \
  "s_add_u32 m0, s77, 114688\n\t" \
  "s_add_u32 s90, s71, 0\n\t" \
  "buffer_load_dwordx4 v187, s[40:43], s90 offen offset:1024  lds\n\t" \
  "s_waitcnt lgkmcnt(4)\n\t" \
  "v_mfma_f32_16x16x128_f8f6f4 v[28:31], a[128:135], v[152:159], v[28:31] cbsz:0 blgp:1\n\t" \
  "ds_read_b64_tr_b8 v[136:137], v169 offset:16384\n\t" \
  "s_add_u32 s99, s29, 16\n\t" \
  "s_min_u32 s99, s99, 31\n\t" \
  "v_readlane_b32 s99, v211, s99\n\t" \
  "s_mov_b32 exec_lo, s15\n\t" \
  "s_mov_b32 exec_hi, 0\n\t" \
  "s_cbranch_execz L_c16_0\n\t" \
  "s_nop 1\n\t" \
  "buffer_load_dword v210, v222, s[60:63], s99 offen\n\t" \
  "L_c16_0:\n\t" \
  "s_mov_b64 exec, -1\n\t" \
  "v_mfma_f32_16x16x128_f8f6f4 v[60:63], a[136:143], v[152:159], v[60:63] cbsz:0 blgp:1\n\t" \
  "ds_read_b64_tr_b8 v[138:139], v169 offset:17408\n\t" \
  "s_waitcnt vmcnt(23)\n\t" \
  "v_accvgpr_read_b32 v229, a192\n\t" \
  "v_accvgpr_read_b32 v230, a193\n\t" \
  "v_mfma_f32_16x16x128_f8f6f4 v[92:95], a[144:151], v[152:159], v[92:95] cbsz:0 blgp:1\n\t" \
  "ds_read_b64_tr_b8 v[140:141], v169 offset:24576\n\t" \
  "v_accvgpr_read_b32 v231, a194\n\t" \
  "v_accvgpr_read_b32 v232, a195\n\t" \
  "v_mfma_f32_16x16x128_f8f6f4 v[124:127], a[152:159], v[152:159], v[124:127] cbsz:0 blgp:1\n\t" \
  "ds_read_b64_tr_b8 v[142:143], v169 offset:25600\n\t" \
  "v_min3_u32 v228, v229, v230, v229\n\t" \
  "v_min3_u32 v228, v231, v232, v228\n\t" \
  "s_waitcnt lgkmcnt(4)\n\t" \
  "v_mfma_f32_16x16x128_f8f6f4 v[0:3], a[160:167], v[128:135], v[0:3] cbsz:0 blgp:1\n\t" \
  "ds_read_b64_tr_b8 v[144:145], v170 offset:16384\n\t" \
  "s_waitcnt vmcnt(22)\n\t" \
  "v_accvgpr_read_b32 v229, a196\n\t" \
  "v_accvgpr_read_b32 v230, a197\n\t" \
  "v_mfma_f32_16x16x128_f8f6f4 v[32:35], a[168:175], v[128:135], v[32:35] cbsz:0 blgp:1\n\t" \
  "ds_read_b64_tr_b8 v[146:147], v170 offset:17408\n\t" \
  "v_accvgpr_read_b32 v231, a198\n\t" \
  "v_accvgpr_read_b32 v232, a199\n\t" \
  "v_mfma_f32_16x16x128_f8f6f4 v[64:67], a[176:183], v[128:135], v[64:67] cbsz:0 blgp:1\n\t" \
  "ds_read_b64_tr_b8 v[148:149], v170 offset:24576\n\t" \
  "v_min3_u32 v228, v229, v230, v228\n\t" \
  "v_min3_u32 v228, v231, v232, v228\n\t" \
  "v_mfma_f32_16x16x128_f8f6f4 v[96:99], a[184:191], v[128:135], v[96:99] cbsz:0 blgp:1\n\t" \
  "ds_read_b64_tr_b8 v[150:151], v170 offset:25600\n\t" \
  "s_waitcnt vmcnt(21)\n\t" \
  "v_accvgpr_read_b32 v229, a200\n\t" \
  "v_accvgpr_read_b32 v230, a201\n\t" \
  "s_waitcnt lgkmcnt(4)\n\t" \
  "v_mfma_f32_16x16x128_f8f6f4 v[4:7], a[160:167], v[136:143], v[4:7] cbsz:0 blgp:1\n\t" \
  "ds_read_b64_tr_b8 v[152:153], v171 offset:16384\n\t" \
  "v_accvgpr_read_b32 v231, a202\n\t" \
  "v_accvgpr_read_b32 v232, a203\n\t" \
  "v_mfma_f32_16x16x128_f8f6f4 v[36:39], a[168:175], v[136:143], v[36:39] cbsz:0 blgp:1\n\t" \
  "ds_read_b64_tr_b8 v[154:155], v171 offset:17408\n\t" \
  "v_min3_u32 v228, v229, v230, v228\n\t" \
  "v_min3_u32 v228, v231, v232, v228\n\t" \
  "v_mfma_f32_16x16x128_f8f6f4 v[68:71], a[176:183], v[136:143], v[68:71] cbsz:0 blgp:1\n\t" \
  "ds_read_b64_tr_b8 v[156:157], v171 offset:24576\n\t" \
  "s_waitcnt vmcnt(20)\n\t" \
  "v_accvgpr_read_b32 v229, a204\n\t" \
  "v_accvgpr_read_b32 v230, a205\n\t" \
  "v_mfma_f32_16x16x128_f8f6f4 v[100:103], a[184:191], v[136:143], v[100:103] cbsz:0 blgp:1\n\t" \
  "ds_read_b64_tr_b8 v[158:159], v171 offset:25600\n\t" \
  "v_accvgpr_read_b32 v231, a206\n\t" \
  "v_accvgpr_read_b32 v232, a207\n\t" \
  "s_waitcnt lgkmcnt(4)\n\t" \
  "v_mfma_f32_16x16x128_f8f6f4 v[8:11], a[160:167], v[144:151], v[8:11] cbsz:0 blgp:1\n\t" \
  "ds_read_b64_tr_b8 v[128:129], v172 offset:16384\n\t" \
  "v_min3_u32 v228, v229, v230, v228\n\t" \
  "v_min3_u32 v228, v231, v232, v228\n\t" \
  "v_mfma_f32_16x16x128_f8f6f4 v[40:43], a[168:175], v[144:151], v[40:43] cbsz:0 blgp:1\n\t" \
  "ds_read_b64_tr_b8 v[130:131], v172 offset:17408\n\t" \
  "s_waitcnt vmcnt(19)\n\t" \
  "v_accvgpr_read_b32 v229, a208\n\t" \
  "v_accvgpr_read_b32 v230, a209\n\t" \
  "v_mfma_f32_16x16x128_f8f6f4 v[72:75], a[176:183], v[144:151], v[72:75] cbsz:0 blgp:1\n\t" \
  "ds_read_b64_tr_b8 v[132:133], v172 offset:24576\n\t" \
  "v_accvgpr_read_b32 v231, a210\n\t" \
  "v_accvgpr_read_b32 v232, a211\n\t" \
  "v_mfma_f32_16x16x128_f8f6f4 v[104:107], a[184:191], v[144:151], v[104:107] cbsz:0 blgp:1\n\t" \
  "ds_read_b64_tr_b8 v[134:135], v172 offset:25600\n\t" \
  "v_min3_u32 v228, v229, v230, v228\n\t" \
  "v_min3_u32 v228, v231, v232, v228\n\t" \
  "s_waitcnt lgkmcnt(4)\n\t" \
  "v_mfma_f32_16x16x128_f8f6f4 v[12:15], a[160:167], v[152:159], v[12:15] cbsz:0 blgp:1\n\t" \
  "ds_read_b64_tr_b8 v[136:137], v173 offset:16384\n\t" \
  "s_waitcnt vmcnt(18)\n\t" \
  "v_accvgpr_read_b32 v229, a212\n\t" \
  "v_accvgpr_read_b32 v230, a213\n\t" \
  "s_cmp_eq_u32 s29, 3\n\t" \
  "s_cbranch_scc0 L_fl3_0\n\t" \
  "s_waitcnt vmcnt(27)\n\t" \
  "s_mov_b64 exec, 1\n\t" \
  "buffer_store_dword v208, v209, s[56:59], 0 offen sc0 sc1\n\t" \
  "s_mov_b64 exec, -1\n\t" \
  "L_fl3_0:\n\t" \
  "v_mfma_f32_16x16x128_f8f6f4 v[44:47], a[168:175], v[152:159], v[44:47] cbsz:0 blgp:1\n\t" \
  "ds_read_b64_tr_b8 v[138:139], v173 offset:17408\n\t" \
  "v_accvgpr_read_b32 v231, a214\n\t" \
  "v_accvgpr_read_b32 v232, a215\n\t" \
  "v_mfma_f32_16x16x128_f8f6f4 v[76:79], a[176:183], v[152:159], v[76:79] cbsz:0 blgp:1\n\t" \
  "ds_read_b64_tr_b8 v[140:141], v173 offset:24576\n\t" \
  "v_min3_u32 v228, v229, v230, v228\n\t" \
  "v_min3_u32 v228, v231, v232, v228\n\t" \
  "v_mfma_f32_16x16x128_f8f6f4 v[108:111], a[184:191], v[152:159], v[108:111] cbsz:0 blgp:1\n\t" \
  "ds_read_b64_tr_b8 v[142:143], v173 offset:25600\n\t" \
  "s_waitcnt vmcnt(17)\n\t" \
  "v_accvgpr_read_b32 v229, a216\n\t" \
  "v_accvgpr_read_b32 v230, a217\n\t" \
  "s_waitcnt lgkmcnt(4)\n\t" \
  "v_mfma_f32_16x16x128_f8f6f4 v[16:19], a[160:167], v[128:135], v[16:19] cbsz:0 blgp:1\n\t" \
  "ds_read_b64_tr_b8 v[144:145], v174 offset:16384\n\t" \
  "v_accvgpr_read_b32 v231, a218\n\t" \
  "v_accvgpr_read_b32 v232, a219\n\t" \
  "v_mfma_f32_16x16x128_f8f6f4 v[48:51], a[168:175], v[128:135], v[48:51] cbsz:0 blgp:1\n\t" \
  "ds_read_b64_tr_b8 v[146:147], v174 offset:17408\n\t" \
  "v_min3_u32 v228, v229, v230, v228\n\t" \
  "v_min3_u32 v228, v231, v232, v228\n\t" \
  "v_mfma_f32_16x16x128_f8f6f4 v[80:83], a[176:183], v[128:135], v[80:83] cbsz:0 blgp:1\n\t" \
  "ds_read_b64_tr_b8 v[148:149], v174 offset:24576\n\t" \
  "s_waitcnt vmcnt(16)\n\t" \
  "v_accvgpr_read_b32 v229, a220\n\t" \
  "v_accvgpr_read_b32 v230, a221\n\t" \
  "v_mfma_f32_16x16x128_f8f6f4 v[112:115], a[184:191], v[128:135], v[112:115] cbsz:0 blgp:1\n\t" \
  "ds_read_b64_tr_b8 v[150:151], v174 offset:25600\n\t" \
  "v_accvgpr_read_b32 v231, a222\n\t" \
  "v_accvgpr_read_b32 v232, a223\n\t" \
  "s_waitcnt lgkmcnt(4)\n\t" \
  "v_mfma_f32_16x16x128_f8f6f4 v[20:23], a[160:167], v[136:143], v[20:23] cbsz:0 blgp:1\n\t" \
  "ds_read_b64_tr_b8 v[152:153], v175 offset:16384\n\t" \
  "v_min3_u32 v228, v229, v230, v228\n\t" \
  "v_min3_u32 v228, v231, v232, v228\n\t" \
  "v_mfma_f32_16x16x128_f8f6f4 v[52:55], a[168:175], v[136:143], v[52:55] cbsz:0 blgp:1\n\t" \
  "ds_read_b64_tr_b8 v[154:155], v175 offset:17408\n\t" \
  "v_mfma_f32_16x16x128_f8f6f4 v[84:87], a[176:183], v[136:143], v[84:87] cbsz:0 blgp:1\n\t" \
  "ds_read_b64_tr_b8 v[156:157], v175 offset:24576\n\t" \
  "v_cmp_eq_u32 vcc, 0, v228\n\t" \
  "s_nop 4\n\t" \
  "s_cmp_lg_u64 vcc, 0\n\t" \
  "s_cbranch_scc1 L_v6_0\n\t" \
  "L_vr6_0:\n\t" \
  "v_mfma_f32_16x16x128_f8f6f4 v[116:119], a[184:191], v[136:143], v[116:119] cbsz:0 blgp:1\n\t" \
  "ds_read_b64_tr_b8 v[158:159], v175 offset:25600\n\t" \
  "s_waitcnt lgkmcnt(4)\n\t" \
  "v_mfma_f32_16x16x128_f8f6f4 v[24:27], a[160:167], v[144:151], v[24:27] cbsz:0 blgp:1\n\t" \
  "s_waitcnt vmcnt(9)\n\t" \
  "s_barrier\n\t" \
  "ds_read_b64_tr_b8 v[128:129], v168 offset:32768\n\t" \
  "v_mfma_f32_16x16x128_f8f6f4 v[56:59], a[168:175], v[144:151], v[56:59] cbsz:0 blgp:1\n\t" \
  "ds_read_b64_tr_b8 v[130:131], v168 offset:33792\n\t" \
  "v_mfma_f32_16x16x128_f8f6f4 v[88:91], a[176:183], v[144:151], v[88:91] cbsz:0 blgp:1\n\t" \
  "ds_read_b64_tr_b8 v[132:133], v168 offset:40960\n\t" \
  "v_mfma_f32_16x16x128_f8f6f4 v[120:123], a[184:191], v[144:151], v[120:123] cbsz:0 blgp:1\n\t" \
  "ds_read_b64_tr_b8 v[134:135], v168 offset:41984\n\t" \
  "buffer_load_dwordx4 a[240:243], v189, s[36:39], s87 offen offset:0 sc1\n\t" \
  "s_waitcnt lgkmcnt(4)\n\t" \
  "v_mfma_f32_16x16x128_f8f6f4 v[28:31], a[160:167], v[152:159], v[28:31] cbsz:0 blgp:1\n\t" \
  "ds_read_b64_tr_b8 v[136:137], v169 offset:32768\n\t" \
  "v_mfma_f32_16x16x128_f8f6f4 v[60:63], a[168:175], v[152:159], v[60:63] cbsz:0 blgp:1\n\t" \
  "ds_read_b64_tr_b8 v[138:139], v169 offset:33792\n\t" \
  "buffer_load_dwordx4 a[244:247], v189, s[36:39], s87 offen offset:1024 sc1\n\t" \
  "v_mfma_f32_16x16x128_f8f6f4 v[92:95], a[176:183], v[152:159], v[92:95] cbsz:0 blgp:1\n\t" \
  "ds_read_b64_tr_b8 v[140:141], v169 offset:40960\n\t" \
  "v_mfma_f32_16x16x128_f8f6f4 v[124:127], a[184:191], v[152:159], v[124:127] cbsz:0 blgp:1\n\t" \
  "ds_read_b64_tr_b8 v[142:143], v169 offset:41984\n\t" \
  "v_accvgpr_read_b32 v229, a224\n\t" \
  "v_accvgpr_read_b32 v230, a225\n\t" \
  "buffer_load_dwordx4 a[248:251], v189, s[36:39], s87 offen offset:2048 sc1\n\t" \
  "s_waitcnt lgkmcnt(4)\n\t" \
  "v_mfma_f32_16x16x128_f8f6f4 v[0:3], a[192:199], v[128:135], v[0:3] cbsz:0 blgp:1\n\t" \
  "ds_read_b64_tr_b8 v[144:145], v170 offset:32768\n\t" \
  "v_accvgpr_read_b32 v231, a226\n\t" \
  "v_accvgpr_read_b32 v232, a227\n\t" \
  "v_mfma_f32_16x16x128_f8f6f4 v[32:35], a[200:207], v[128:135], v[32:35] cbsz:0 blgp:1\n\t" \
  "ds_read_b64_tr_b8 v[146:147], v170 offset:33792\n\t" \
  "v_min3_u32 v228, v229, v230, v229\n\t" \
  "v_min3_u32 v228, v231, v232, v228\n\t" \
  "buffer_load_dwordx4 a[252:255], v189, s[36:39], s87 offen offset:3072 sc1\n\t" \
  "v_mfma_f32_16x16x128_f8f6f4 v[64:67], a[208:215], v[128:135], v[64:67] cbsz:0 blgp:1\n\t" \
  "ds_read_b64_tr_b8 v[148:149], v170 offset:40960\n\t" \
  "v_accvgpr_read_b32 v229, a228\n\t" \
  "v_accvgpr_read_b32 v230, a229\n\t" \
  "s_mov_b32 s87, 262144\n\t" \
  "buffer_load_dwordx4 a[0:3], v188, s[36:39], s87 offen offset:0 sc1\n\t" \
  "v_mfma_f32_16x16x128_f8f6f4 v[96:99], a[216:223], v[128:135], v[96:99] cbsz:0 blgp:1\n\t" \
  "ds_read_b64_tr_b8 v[150:151], v170 offset:41984\n\t" \
  "v_accvgpr_read_b32 v231, a230\n\t" \
  "v_accvgpr_read_b32 v232, a231\n\t" \
  "s_waitcnt lgkmcnt(4)\n\t" \
  "v_mfma_f32_16x16x128_f8f6f4 v[4:7], a[192:199], v[136:143], v[4:7] cbsz:0 blgp:1\n\t" \
  "ds_read_b64_tr_b8 v[152:153], v171 offset:32768\n\t" \
  "v_min3_u32 v228, v229, v230, v228\n\t" \
  "v_min3_u32 v228, v231, v232, v228\n\t" \
  "buffer_load_dwordx4 a[4:7], v188, s[36:39], s87 offen offset:1024 sc1\n\t" \
  "v_mfma_f32_16x16x128_f8f6f4 v[36:39], a[200:207], v[136:143], v[36:39] cbsz:0 blgp:1\n\t" \
  "ds_read_b64_tr_b8 v[154:155], v171 offset:33792\n\t" \
  "v_accvgpr_read_b32 v229, a232\n\t" \
  "v_accvgpr_read_b32 v230, a233\n\t" \
  "v_mfma_f32_16x16x128_f8f6f4 v[68:71], a[208:215], v[136:143], v[68:71] cbsz:0 blgp:1\n\t" \
  "ds_read_b64_tr_b8 v[156:157], v171 offset:40960\n\t" \
  "v_accvgpr_read_b32 v231, a234\n\t" \
  "v_accvgpr_read_b32 v232, a235\n\t" \
  "buffer_load_dwordx4 a[8:11], v188, s[36:39], s87 offen offset:2048 sc1\n\t" \
  "v_mfma_f32_16x16x128_f8f6f4 v[100:103], a[216:223], v[136:143], v[100:103] cbsz:0 blgp:1\n\t" \
  "ds_read_b64_tr_b8 v[158:159], v171 offset:41984\n\t" \
  "v_min3_u32 v228, v229, v230, v228\n\t" \
  "v_min3_u32 v228, v231, v232, v228\n\t" \
  "s_waitcnt lgkmcnt(4)\n\t" \
  "v_mfma_f32_16x16x128_f8f6f4 v[8:11], a[192:199], v[144:151], v[8:11] cbsz:0 blgp:1\n\t" \
  "ds_read_b64_tr_b8 v[128:129], v172 offset:32768\n\t" \
  "v_accvgpr_read_b32 v229, a236\n\t" \
  "v_accvgpr_read_b32 v230, a237\n\t" \
  "buffer_load_dwordx4 a[12:15], v188, s[36:39], s87 offen offset:3072 sc1\n\t" \
  "v_mfma_f32_16x16x128_f8f6f4 v[40:43], a[200:207], v[144:151], v[40:43] cbsz:0 blgp:1\n\t" \
  "ds_read_b64_tr_b8 v[130:131], v172 offset:33792\n\t" \
  "v_accvgpr_read_b32 v231, a238\n\t" \
  "v_accvgpr_read_b32 v232, a239\n\t" \
  "v_mfma_f32_16x16x128_f8f6f4 v[72:75], a[208:215], v[144:151], v[72:75] cbsz:0 blgp:1\n\t" \
  "ds_read_b64_tr_b8 v[132:133], v172 offset:40960\n\t" \
  "v_min3_u32 v228, v229, v230, v228\n\t" \
  "v_min3_u32 v228, v231, v232, v228\n\t" \
  "s_add_u32 m0, s77, 131072\n\t" \
  "s_add_u32 s90, s72, 0\n\t" \
  "buffer_load_dwordx4 v186, s[40:43], s90 offen offset:0  lds\n\t" \
  "v_mfma_f32_16x16x128_f8f6f4 v[104:107], a[216:223], v[144:151], v[104:107] cbsz:0 blgp:1\n\t" \
  "ds_read_b64_tr_b8 v[134:135], v172 offset:41984\n\t" \
  "s_waitcnt vmcnt(8)\n\t" \
  "v_accvgpr_read_b32 v229, a240\n\t" \
  "v_accvgpr_read_b32 v230, a241\n\t" \
  "s_add_u32 m0, s77, 147456\n\t" \
  "s_add_u32 s90, s73, 0\n\t" \
  "buffer_load_dwordx4 v186, s[40:43], s90 offen offset:0  lds\n\t" \
  "s_waitcnt lgkmcnt(4)\n\t" \
  "v_mfma_f32_16x16x128_f8f6f4 v[12:15], a[192:199], v[152:159], v[12:15] cbsz:0 blgp:1\n\t" \
  "ds_read_b64_tr_b8 v[136:137], v173 offset:32768\n\t" \
  "v_accvgpr_read_b32 v231, a242\n\t" \
  "v_accvgpr_read_b32 v232, a243\n\t" \
  "s_add_u32 m0, s77, 131072\n\t" \
  "s_add_u32 s90, s72, 0\n\t" \
  "buffer_load_dwordx4 v187, s[40:43], s90 offen offset:1024  lds\n\t" \
  "v_mfma_f32_16x16x128_f8f6f4 v[44:47], a[200:207], v[152:159], v[44:47] cbsz:0 blgp:1\n\t" \
  "ds_read_b64_tr_b8 v[138:139], v173 offset:33792\n\t" \
  "v_min3_u32 v228, v229, v230, v228\n\t" \
  "v_min3_u32 v228, v231, v232, v228\n\t" \
  "s_add_u32 m0, s77, 147456\n\t" \
  "s_add_u32 s90, s73, 0\n\t" \
  "buffer_load_dwordx4 v187, s[40:43], s90 offen offset:1024  lds\n\t" \
  "v_mfma_f32_16x16x128_f8f6f4 v[76:79], a[208:215], v[152:159], v[76:79] cbsz:0 blgp:1\n\t" \
  "ds_read_b64_tr_b8 v[140:141], v173 offset:40960\n\t" \
  "s_waitcnt vmcnt(10)\n\t" \
  "v_accvgpr_read_b32 v229, a244\n\t" \
  "v_accvgpr_read_b32 v230, a245\n\t" \
  "s_add_u32 m0, s77, 0\n\t" \
  "s_add_u32 s90, s74, 0\n\t" \
  "buffer_load_dwordx4 v186, s[40:43], s90 offen offset:0  lds\n\t" \
  "v_mfma_f32_16x16x128_f8f6f4 v[108:111], a[216:223], v[152:159], v[108:111] cbsz:0 blgp:1\n\t" \
  "ds_read_b64_tr_b8 v[142:143], v173 offset:41984\n\t" \
  "v_accvgpr_read_b32 v231, a246\n\t" \
  "v_accvgpr_read_b32 v232, a247\n\t" \
  "s_add_u32 m0, s77, 16384\n\t" \
  "s_add_u32 s90, s68, 2097152\n\t" \
  "buffer_load_dwordx4 v186, s[40:43], s90 offen offset:0  lds\n\t" \
  "s_waitcnt lgkmcnt(4)\n\t" \
  "v_mfma_f32_16x16x128_f8f6f4 v[16:19], a[192:199], v[128:135], v[16:19] cbsz:0 blgp:1\n\t" \
  "ds_read_b64_tr_b8 v[144:145], v174 offset:32768\n\t" \
  "v_min3_u32 v228, v229, v230, v228\n\t" \
  "v_min3_u32 v228, v231, v232, v228\n\t" \
  "s_mov_b32 s64, 0\n\t" \
  "s_add_u32 m0, s77, 0\n\t" \
  "s_add_u32 s90, s74, 0\n\t" \
  "buffer_load_dwordx4 v187, s[40:43], s90 offen offset:1024  lds\n\t" \
  "v_mfma_f32_16x16x128_f8f6f4 v[48:51], a[200:207], v[128:135], v[48:51] cbsz:0 blgp:1\n\t" \
  "ds_read_b64_tr_b8 v[146:147], v174 offset:33792\n\t" \
  "s_waitcnt vmcnt(12)\n\t" \
  "v_accvgpr_read_b32 v229, a248\n\t" \
  "v_accvgpr_read_b32 v230, a249\n\t" \
  "global_load_dword v196, v195, s[26:27] offset:0 sc1\n\t" \
  "s_add_u32 m0, s77, 16384\n\t" \
  "s_add_u32 s90, s68, 2097152\n\t" \
  "buffer_load_dwordx4 v187, s[40:43], s90 offen offset:1024  lds\n\t" \
  "v_mfma_f32_16x16x128_f8f6f4 v[80:83], a[208:215], v[128:135], v[80:83] cbsz:0 blgp:1\n\t" \
  "ds_read_b64_tr_b8 v[148:149], v174 offset:40960\n\t" \
  "v_accvgpr_read_b32 v231, a250\n\t" \
  "v_accvgpr_read_b32 v232, a251\n\t" \
  "buffer_load_dwordx4 a[16:19], v189, s[36:39], s87 offen offset:0 sc1\n\t" \
  "v_mfma_f32_16x16x128_f8f6f4 v[112:115], a[216:223], v[128:135], v[112:115] cbsz:0 blgp:1\n\t" \
  "ds_read_b64_tr_b8 v[150:151], v174 offset:41984\n\t" \
  "v_min3_u32 v228, v229, v230, v228\n\t" \
  "v_min3_u32 v228, v231, v232, v228\n\t" \
  "s_waitcnt lgkmcnt(4)\n\t" \
  "v_mfma_f32_16x16x128_f8f6f4 v[20:23], a[192:199], v[136:143], v[20:23] cbsz:0 blgp:1\n\t" \
  "ds_read_b64_tr_b8 v[152:153], v175 offset:32768\n\t" \
  "s_waitcnt vmcnt(14)\n\t" \
  "v_accvgpr_read_b32 v229, a252\n\t" \
  "v_accvgpr_read_b32 v230, a253\n\t" \
  "buffer_load_dwordx4 a[20:23], v189, s[36:39], s87 offen offset:1024 sc1\n\t" \
  "v_mfma_f32_16x16x128_f8f6f4 v[52:55], a[200:207], v[136:143], v[52:55] cbsz:0 blgp:1\n\t" \
  "ds_read_b64_tr_b8 v[154:155], v175 offset:33792\n\t" \
  "v_accvgpr_read_b32 v231, a254\n\t" \
  "v_accvgpr_read_b32 v232, a255\n\t" \
  "v_mfma_f32_16x16x128_f8f6f4 v[84:87], a[208:215], v[136:143], v[84:87] cbsz:0 blgp:1\n\t" \
  "ds_read_b64_tr_b8 v[156:157], v175 offset:40960\n\t" \
  "v_min3_u32 v228, v229, v230, v228\n\t" \
  "v_min3_u32 v228, v231, v232, v228\n\t" \
  "buffer_load_dwordx4 a[24:27], v189, s[36:39], s87 offen offset:2048 sc1\n\t" \
  "v_mfma_f32_16x16x128_f8f6f4 v[116:119], a[216:223], v[136:143], v[116:119] cbsz:0 blgp:1\n\t" \
  "ds_read_b64_tr_b8 v[158:159], v175 offset:41984\n\t" \
  "s_waitcnt lgkmcnt(4)\n\t" \
  "v_mfma_f32_16x16x128_f8f6f4 v[24:27], a[192:199], v[144:151], v[24:27] cbsz:0 blgp:1\n\t" \
  "s_barrier\n\t" \
  "ds_read_b64_tr_b8 v[128:129], v168 offset:49152\n\t" \
  "v_cmp_eq_u32 vcc, 0, v228\n\t" \
  "s_nop 4\n\t" \
  "s_cmp_lg_u64 vcc, 0\n\t" \
  "s_cbranch_scc1 L_v7_0\n\t" \
  "L_vr7_0:\n\t" \
  "v_mfma_f32_16x16x128_f8f6f4 v[56:59], a[200:207], v[144:151], v[56:59] cbsz:0 blgp:1\n\t" \
  "ds_read_b64_tr_b8 v[130:131], v168 offset:50176\n\t" \
  "buffer_load_dwordx4 a[28:31], v189, s[36:39], s87 offen offset:3072 sc1\n\t" \
  "v_mfma_f32_16x16x128_f8f6f4 v[88:91], a[208:215], v[144:151], v[88:91] cbsz:0 blgp:1\n\t" \
  "ds_read_b64_tr_b8 v[132:133], v168 offset:57344\n\t" \
  "v_mfma_f32_16x16x128_f8f6f4 v[120:123], a[216:223], v[144:151], v[120:123] cbsz:0 blgp:1\n\t" \
  "ds_read_b64_tr_b8 v[134:135], v168 offset:58368\n\t" \
  "s_mov_b32 s87, 327680\n\t" \
  "buffer_load_dwordx4 a[32:35], v188, s[36:39], s87 offen offset:0 sc1\n\t" \
  "s_waitcnt lgkmcnt(4)\n\t" \
  "v_mfma_f32_16x16x128_f8f6f4 v[28:31], a[192:199], v[152:159], v[28:31] cbsz:0 blgp:1\n\t" \
  "ds_read_b64_tr_b8 v[136:137], v169 offset:49152\n\t" \
  "v_mfma_f32_16x16x128_f8f6f4 v[60:63], a[200:207], v[152:159], v[60:63] cbsz:0 blgp:1\n\t" \
  "ds_read_b64_tr_b8 v[138:139], v169 offset:50176\n\t" \
  "s_waitcnt vmcnt(17)\n\t" \
  "v_accvgpr_read_b32 v229, a0\n\t" \
  "v_accvgpr_read_b32 v230, a1\n\t" \
  "buffer_load_dwordx4 a[36:39], v188, s[36:39], s87 offen offset:1024 sc1\n\t" \
  "v_mfma_f32_16x16x128_f8f6f4 v[92:95], a[208:215], v[152:159], v[92:95] cbsz:0 blgp:1\n\t" \
  "ds_read_b64_tr_b8 v[140:141], v169 offset:57344\n\t" \
  "v_accvgpr_read_b32 v231, a2\n\t" \
  "v_accvgpr_read_b32 v232, a3\n\t" \
  "v_mfma_f32_16x16x128_f8f6f4 v[124:127], a[216:223], v[152:159], v[124:127] cbsz:0 blgp:1\n\t" \
  "ds_read_b64_tr_b8 v[142:143], v169 offset:58368\n\t" \
  "v_min3_u32 v228, v229, v230, v229\n\t" \
  "v_min3_u32 v228, v231, v232, v228\n\t" \
  "buffer_load_dwordx4 a[40:43], v188, s[36:39], s87 offen offset:2048 sc1\n\t" \
  "s_waitcnt lgkmcnt(4)\n\t" \
  "v_mfma_f32_16x16x128_f8f6f4 v[0:3], a[224:231], v[128:135], v[0:3] cbsz:0 blgp:1\n\t" \
  "ds_read_b64_tr_b8 v[144:145], v170 offset:49152\n\t" \
  "s_waitcnt vmcnt(18)\n\t" \
  "v_accvgpr_read_b32 v229, a4\n\t" \
  "v_accvgpr_read_b32 v230, a5\n\t" \
  "v_mfma_f32_16x16x128_f8f6f4 v[32:35], a[232:239], v[128:135], v[32:35] cbsz:0 blgp:1\n\t" \
  "ds_read_b64_tr_b8 v[146:147], v170 offset:50176\n\t" \
  "v_accvgpr_read_b32 v231, a6\n\t" \
  "v_accvgpr_read_b32 v232, a7\n\t" \
  "buffer_load_dwordx4 a[44:47], v188, s[36:39], s87 offen offset:3072 sc1\n\t" \
  "v_mfma_f32_16x16x128_f8f6f4 v[64:67], a[240:247], v[128:135], v[64:67] cbsz:0 blgp:1\n\t" \
  "ds_read_b64_tr_b8 v[148:149], v170 offset:57344\n\t" \
  "v_min3_u32 v228, v229, v230, v228\n\t" \
  "v_min3_u32 v228, v231, v232, v228\n\t" \
  "v_mfma_f32_16x16x128_f8f6f4 v[96:99], a[248:255], v[128:135], v[96:99] cbsz:0 blgp:1\n\t" \
  "ds_read_b64_tr_b8 v[150:151], v170 offset:58368\n\t" \
  "s_waitcnt vmcnt(18)\n\t" \
  "v_accvgpr_read_b32 v229, a8\n\t" \
  "v_accvgpr_read_b32 v230, a9\n\t" \
  "buffer_load_dwordx4 a[48:51], v189, s[36:39], s87 offen offset:0 sc1\n\t" \
  "s_waitcnt lgkmcnt(4)\n\t" \
  "v_mfma_f32_16x16x128_f8f6f4 v[4:7], a[224:231], v[136:143], v[4:7] cbsz:0 blgp:1\n\t" \
  "ds_read_b64_tr_b8 v[152:153], v171 offset:49152\n\t" \
  "v_accvgpr_read_b32 v231, a10\n\t" \
  "v_accvgpr_read_b32 v232, a11\n\t" \
  "v_mfma_f32_16x16x128_f8f6f4 v[36:39], a[232:239], v[136:143], v[36:39] cbsz:0 blgp:1\n\t" \
  "ds_read_b64_tr_b8 v[154:155], v171 offset:50176\n\t" \
  "v_min3_u32 v228, v229, v230, v228\n\t" \
  "v_min3_u32 v228, v231, v232, v228\n\t" \
  "s_waitcnt vmcnt(10)\n\t" \
  "v_cmp_le_u32 vcc, s79, v196\n\t" \
  "s_nop 4\n\t" \
  "s_andn2_b32 s85, vcc_lo, s64\n\t" \
  "s_or_b32 s64, s64, vcc_lo\n\t" \
  "buffer_load_dwordx4 a[52:55], v189, s[36:39], s87 offen offset:1024 sc1\n\t" \
  "v_mfma_f32_16x16x128_f8f6f4 v[68:71], a[240:247], v[136:143], v[68:71] cbsz:0 blgp:1\n\t" \
  "ds_read_b64_tr_b8 v[156:157], v171 offset:57344\n\t" \
  "v_accvgpr_read_b32 v229, a12\n\t" \
  "v_accvgpr_read_b32 v230, a13\n\t" \
  "s_mov_b32 s96, 0\n\t" \
  "v_mfma_f32_16x16x128_f8f6f4 v[100:103], a[248:255], v[136:143], v[100:103] cbsz:0 blgp:1\n\t" \
  "ds_read_b64_tr_b8 v[158:159], v171 offset:58368\n\t" \
  "v_accvgpr_read_b32 v231, a14\n\t" \
  "v_accvgpr_read_b32 v232, a15\n\t" \
  "global_load_dword v196, v195, s[26:27] offset:1024 sc1\n\t" \
  "buffer_load_dwordx4 a[56:59], v189, s[36:39], s87 offen offset:2048 sc1\n\t" \
  "s_waitcnt lgkmcnt(4)\n\t" \
  "v_mfma_f32_16x16x128_f8f6f4 v[8:11], a[224:231], v[144:151], v[8:11] cbsz:0 blgp:1\n\t" \
  "ds_read_b64_tr_b8 v[128:129], v172 offset:49152\n\t" \
  "v_min3_u32 v228, v229, v230, v228\n\t" \
  "v_min3_u32 v228, v231, v232, v228\n\t" \
  "v_mfma_f32_16x16x128_f8f6f4 v[40:43], a[232:239], v[144:151], v[40:43] cbsz:0 blgp:1\n\t" \
  "ds_read_b64_tr_b8 v[130:131], v172 offset:50176\n\t" \
  "s_waitcnt vmcnt(11)\n\t" \
  "v_accvgpr_read_b32 v229, a16\n\t" \
  "v_accvgpr_read_b32 v230, a17\n\t" \
  "buffer_load_dwordx4 a[60:63], v189, s[36:39], s87 offen offset:3072 sc1\n\t" \
  "v_mfma_f32_16x16x128_f8f6f4 v[72:75], a[240:247], v[144:151], v[72:75] cbsz:0 blgp:1\n\t" \
  "ds_read_b64_tr_b8 v[132:133], v172 offset:57344\n\t" \
  "v_accvgpr_read_b32 v231, a18\n\t" \
  "v_accvgpr_read_b32 v232, a19\n\t" \
  "v_mfma_f32_16x16x128_f8f6f4 v[104:107], a[248:255], v[144:151], v[104:107] cbsz:0 blgp:1\n\t" \
  "ds_read_b64_tr_b8 v[134:135], v172 offset:58368\n\t" \
  "v_min3_u32 v228, v229, v230, v228\n\t" \
  "v_min3_u32 v228, v231, v232, v228\n\t" \
  "s_bfe_u32 s84, s64, 262168\n\t" \
  "s_cmp_eq_u32 s84, 15\n\t" \
  "s_cbranch_scc0 L_v10_0\n\t" \
  "L_vr10_0:\n\t" \
  "s_mov_b32 s87, 393216\n\t" \
  "buffer_load_dwordx4 a[64:67], v188, s[36:39], s87 offen offset:0 sc1\n\t" \
  "s_waitcnt lgkmcnt(4)\n\t" \
  "v_mfma_f32_16x16x128_f8f6f4 v[12:15], a[224:231], v[152:159], v[12:15] cbsz:0 blgp:1\n\t" \
  "ds_read_b64_tr_b8 v[136:137], v173 offset:49152\n\t" \
  "s_waitcnt vmcnt(12)\n\t" \
  "v_accvgpr_read_b32 v229, a20\n\t" \
  "v_accvgpr_read_b32 v230, a21\n\t" \
  "v_mfma_f32_16x16x128_f8f6f4 v[44:47], a[232:239], v[152:159], v[44:47] cbsz:0 blgp:1\n\t" \
  "ds_read_b64_tr_b8 v[138:139], v173 offset:50176\n\t" \
  "v_accvgpr_read_b32 v231, a22\n\t" \
  "v_accvgpr_read_b32 v232, a23\n\t" \
  "buffer_load_dwordx4 a[68:71], v188, s[36:39], s87 offen offset:1024 sc1\n\t" \
  "v_mfma_f32_16x16x128_f8f6f4 v[76:79], a[240:247], v[152:159], v[76:79] cbsz:0 blgp:1\n\t" \
  "ds_read_b64_tr_b8 v[140:141], v173 offset:57344\n\t" \
  "v_min3_u32 v228, v229, v230, v228\n\t" \
  "v_min3_u32 v228, v231, v232, v228\n\t" \
  "v_mfma_f32_16x16x128_f8f6f4 v[108:111], a[248:255], v[152:159], v[108:111] cbsz:0 blgp:1\n\t" \
  "ds_read_b64_tr_b8 v[142:143], v173 offset:58368\n\t" \
  "s_waitcnt vmcnt(12)\n\t" \
  "v_accvgpr_read_b32 v229, a24\n\t" \
  "v_accvgpr_read_b32 v230, a25\n\t" \
  "buffer_load_dwordx4 a[72:75], v188, s[36:39], s87 offen offset:2048 sc1\n\t" \
  "s_waitcnt lgkmcnt(4)\n\t" \
  "v_mfma_f32_16x16x128_f8f6f4 v[16:19], a[224:231], v[128:135], v[16:19] cbsz:0 blgp:1\n\t" \
  "ds_read_b64_tr_b8 v[144:145], v174 offset:49152\n\t" \
  "v_accvgpr_read_b32 v231, a26\n\t" \
  "v_accvgpr_read_b32 v232, a27\n\t" \
  "v_mfma_f32_16x16x128_f8f6f4 v[48:51], a[232:239], v[128:135], v[48:51] cbsz:0 blgp:1\n\t" \
  "ds_read_b64_tr_b8 v[146:147], v174 offset:50176\n\t" \
  "v_min3_u32 v228, v229, v230, v228\n\t" \
  "v_min3_u32 v228, v231, v232, v228\n\t" \
  "buffer_load_dwordx4 a[76:79], v188, s[36:39], s87 offen offset:3072 sc1\n\t" \
  "v_mfma_f32_16x16x128_f8f6f4 v[80:83], a[240:247], v[128:135], v[80:83] cbsz:0 blgp:1\n\t" \
  "ds_read_b64_tr_b8 v[148:149], v174 offset:57344\n\t" \
  "s_waitcnt vmcnt(13)\n\t" \
  "v_accvgpr_read_b32 v229, a28\n\t" \
  "v_accvgpr_read_b32 v230, a29\n\t" \
  "v_mfma_f32_16x16x128_f8f6f4 v[112:115], a[248:255], v[128:135], v[112:115] cbsz:0 blgp:1\n\t" \
  "ds_read_b64_tr_b8 v[150:151], v174 offset:58368\n\t" \
  "v_accvgpr_read_b32 v231, a30\n\t" \
  "v_accvgpr_read_b32 v232, a31\n\t" \
  "buffer_load_dwordx4 a[80:83], v189, s[36:39], s87 offen offset:0 sc1\n\t" \
  "s_waitcnt lgkmcnt(4)\n\t" \
  "v_mfma_f32_16x16x128_f8f6f4 v[20:23], a[224:231], v[136:143], v[20:23] cbsz:0 blgp:1\n\t" \
  "ds_read_b64_tr_b8 v[152:153], v175 offset:49152\n\t" \
  "v_min3_u32 v228, v229, v230, v228\n\t" \
  "v_min3_u32 v228, v231, v232, v228\n\t" \
  "v_mfma_f32_16x16x128_f8f6f4 v[52:55], a[232:239], v[136:143], v[52:55] cbsz:0 blgp:1\n\t" \
  "ds_read_b64_tr_b8 v[154:155], v175 offset:50176\n\t" \
  "buffer_load_dwordx4 a[84:87], v189, s[36:39], s87 offen offset:1024 sc1\n\t" \
  "v_mfma_f32_16x16x128_f8f6f4 v[84:87], a[240:247], v[136:143], v[84:87] cbsz:0 blgp:1\n\t" \
  "ds_read_b64_tr_b8 v[156:157], v175 offset:57344\n\t" \
  "v_cmp_eq_u32 vcc, 0, v228\n\t" \
  "s_nop 4\n\t" \
  "s_cmp_lg_u64 vcc, 0\n\t" \
  "s_cbranch_scc1 L_v8_0\n\t" \
  "L_vr8_0:\n\t" \
  "v_mfma_f32_16x16x128_f8f6f4 v[116:119], a[248:255], v[136:143], v[116:119] cbsz:0 blgp:1\n\t" \
  "ds_read_b64_tr_b8 v[158:159], v175 offset:58368\n\t" \
  "buffer_load_dwordx4 a[88:91], v189, s[36:39], s87 offen offset:2048 sc1\n\t" \
  "s_waitcnt lgkmcnt(4)\n\t" \
  "v_mfma_f32_16x16x128_f8f6f4 v[24:27], a[224:231], v[144:151], v[24:27] cbsz:0 blgp:1\n\t" \
  "s_barrier\n\t" \
  "ds_read_b64_tr_b8 v[128:129], v176 offset:0\n\t" \
  "v_mfma_f32_16x16x128_f8f6f4 v[56:59], a[232:239], v[144:151], v[56:59] cbsz:0 blgp:1\n\t" \
  "ds_read_b64_tr_b8 v[130:131], v176 offset:1024\n\t" \
  "buffer_load_dwordx4 a[92:95], v189, s[36:39], s87 offen offset:3072 sc1\n\t" \
  "v_mfma_f32_16x16x128_f8f6f4 v[88:91], a[240:247], v[144:151], v[88:91] cbsz:0 blgp:1\n\t" \
  "ds_read_b64_tr_b8 v[132:133], v176 offset:8192\n\t" \
  "v_mfma_f32_16x16x128_f8f6f4 v[120:123], a[248:255], v[144:151], v[120:123] cbsz:0 blgp:1\n\t" \
  "ds_read_b64_tr_b8 v[134:135], v176 offset:9216\n\t" \
  "s_waitcnt vmcnt(10)\n\t" \
  "v_cmp_le_u32 vcc, s79, v196\n\t" \
  "s_nop 4\n\t" \
  "s_andn2_b32 s85, vcc_lo, s96\n\t" \
  "s_or_b32 s96, s96, vcc_lo\n\t" \
  "s_and_b32 exec_lo, s85, 0xfffffff\n\t" \
  "s_mov_b32 exec_hi, 0\n\t" \
  "s_cbranch_execz L_u2_0\n\t" \
  "s_mov_b32 s84, 458752\n\t" \
  "buffer_load_dword v210, v198, s[36:39], s84 offen \n\t" \
  "L_u2_0:\n\t" \
  "s_mov_b64 exec, -1\n\t" \
  "s_bfe_u32 s84, s96, 262144\n\t" \
  "s_cmp_eq_u32 s84, 15\n\t" \
  "s_cbranch_scc0 L_q11_0\n\t" \
  "L_qr11_0:\n\t" \
  "s_mov_b32 s87, 458752\n\t" \
  "buffer_load_dwordx4 a[96:99], v188, s[36:39], s87 offen offset:0 \n\t" \
  "s_waitcnt lgkmcnt(4)\n\t" \
  "v_mfma_f32_16x16x128_f8f6f4 v[28:31], a[224:231], v[152:159], v[28:31] cbsz:0 blgp:1\n\t" \
  "ds_read_b64_tr_b8 v[136:137], v177 offset:0\n\t" \
  "v_mfma_f32_16x16x128_f8f6f4 v[60:63], a[232:239], v[152:159], v[60:63] cbsz:0 blgp:1\n\t" \
  "ds_read_b64_tr_b8 v[138:139], v177 offset:1024\n\t" \
  "v_accvgpr_read_b32 v229, a32\n\t" \
  "v_accvgpr_read_b32 v230, a33\n\t" \
  "buffer_load_dwordx4 a[100:103], v188, s[36:39], s87 offen offset:1024 \n\t" \
  "v_mfma_f32_16x16x128_f8f6f4 v[92:95], a[240:247], v[152:159], v[92:95] cbsz:0 blgp:1\n\t" \
  "ds_read_b64_tr_b8 v[140:141], v177 offset:8192\n\t" \
  "v_accvgpr_read_b32 v231, a34\n\t" \
  "v_accvgpr_read_b32 v232, a35\n\t" \
  "s_add_u32 m0, s76, 32768\n\t" \
  "s_add_u32 s90, s69, 2097152\n\t" \
  "buffer_load_dwordx4 v184, s[40:43], s90 offen offset:0  lds\n\t" \
  "v_mfma_f32_16x16x128_f8f6f4 v[124:127], a[248:255], v[152:159], v[124:127] cbsz:0 blgp:1\n\t" \
  "ds_read_b64_tr_b8 v[142:143], v177 offset:9216\n\t" \
  "v_min3_u32 v228, v229, v230, v229\n\t" \
  "v_min3_u32 v228, v231, v232, v228\n\t" \
  "s_add_u32 m0, s76, 32768\n\t" \
  "s_add_u32 s90, s69, 2097152\n\t" \
  "buffer_load_dwordx4 v185, s[40:43], s90 offen offset:1024  lds\n\t" \
  "s_waitcnt lgkmcnt(4)\n\t" \
  "v_mfma_f32_16x16x128_f8f6f4 v[0:3], a[0:7], v[128:135], v[0:3] cbsz:0 blgp:1\n\t" \
  "ds_read_b64_tr_b8 v[144:145], v178 offset:0\n\t" \
  "v_accvgpr_read_b32 v229, a36\n\t" \
  "v_accvgpr_read_b32 v230, a37\n\t" \
  "s_add_u32 m0, s76, 49152\n\t" \
  "s_add_u32 s90, s70, 2097152\n\t" \
  "buffer_load_dwordx4 v184, s[40:43], s90 offen offset:0  lds\n\t" \
  "v_mfma_f32_16x16x128_f8f6f4 v[32:35], a[8:15], v[128:135], v[32:35] cbsz:0 blgp:1\n\t" \
  "ds_read_b64_tr_b8 v[146:147], v178 offset:1024\n\t" \
  "v_accvgpr_read_b32 v231, a38\n\t" \
  "v_accvgpr_read_b32 v232, a39\n\t" \
  "s_add_u32 m0, s76, 49152\n\t" \
  "s_add_u32 s90, s70, 2097152\n\t" \
  "buffer_load_dwordx4 v185, s[40:43], s90 offen offset:1024  lds\n\t" \
  "v_mfma_f32_16x16x128_f8f6f4 v[64:67], a[16:23], v[128:135], v[64:67] cbsz:0 blgp:1\n\t" \
  "ds_read_b64_tr_b8 v[148:149], v178 offset:8192\n\t" \
  "v_min3_u32 v228, v229, v230, v228\n\t" \
  "v_min3_u32 v228, v231, v232, v228\n\t" \
  "s_add_u32 m0, s76, 65536\n\t" \
  "s_add_u32 s90, s71, 2097152\n\t" \
  "buffer_load_dwordx4 v184, s[40:43], s90 offen offset:0  lds\n\t" \
  "v_mfma_f32_16x16x128_f8f6f4 v[96:99], a[24:31], v[128:135], v[96:99] cbsz:0 blgp:1\n\t" \
  "ds_read_b64_tr_b8 v[150:151], v178 offset:9216\n\t" \
  "v_accvgpr_read_b32 v229, a40\n\t" \
  "v_accvgpr_read_b32 v230, a41\n\t" \
  "s_add_u32 m0, s76, 65536\n\t" \
  "s_add_u32 s90, s71, 2097152\n\t" \
  "buffer_load_dwordx4 v185, s[40:43], s90 offen offset:1024  lds\n\t" \
  "s_waitcnt lgkmcnt(4)\n\t" \
  "v_mfma_f32_16x16x128_f8f6f4 v[4:7], a[0:7], v[136:143], v[4:7] cbsz:0 blgp:1\n\t" \
  "ds_read_b64_tr_b8 v[152:153], v179 offset:0\n\t" \
  "v_accvgpr_read_b32 v231, a42\n\t" \
  "v_accvgpr_read_b32 v232, a43\n\t" \
  "s_add_u32 m0, s76, 81920\n\t" \
  "s_add_u32 s90, s72, 2097152\n\t" \
  "buffer_load_dwordx4 v184, s[40:43], s90 offen offset:0  lds\n\t" \
  "v_mfma_f32_16x16x128_f8f6f4 v[36:39], a[8:15], v[136:143], v[36:39] cbsz:0 blgp:1\n\t" \
  "ds_read_b64_tr_b8 v[154:155], v179 offset:1024\n\t" \
  "v_min3_u32 v228, v229, v230, v228\n\t" \
  "v_min3_u32 v228, v231, v232, v228\n\t" \
  "s_add_u32 m0, s76, 81920\n\t" \
  "s_add_u32 s90, s72, 2097152\n\t" \
  "buffer_load_dwordx4 v185, s[40:43], s90 offen offset:1024  lds\n\t" \
  "v_mfma_f32_16x16x128_f8f6f4 v[68:71], a[16:23], v[136:143], v[68:71] cbsz:0 blgp:1\n\t" \
  "ds_read_b64_tr_b8 v[156:157], v179 offset:8192\n\t" \
  "v_accvgpr_read_b32 v229, a44\n\t" \
  "v_accvgpr_read_b32 v230, a45\n\t" \
  "s_add_u32 s99, s29, 20\n\t" \
  "s_min_u32 s99, s99, 31\n\t" \
  "v_readlane_b32 s99, v211, s99\n\t" \
  "s_mov_b32 exec_lo, s15\n\t" \
  "s_mov_b32 exec_hi, 0\n\t" \
  "s_cbranch_execz L_c20_0\n\t" \
  "s_nop 1\n\t" \
  "buffer_load_dword v210, v222, s[60:63], s99 offen\n\t" \
  "L_c20_0:\n\t" \
  "s_mov_b64 exec, -1\n\t" \
  "v_mfma_f32_16x16x128_f8f6f4 v[100:103], a[24:31], v[136:143], v[100:103] cbsz:0 blgp:1\n\t" \
  "ds_read_b64_tr_b8 v[158:159], v179 offset:9216\n\t" \
  "v_accvgpr_read_b32 v231, a46\n\t" \
  "v_accvgpr_read_b32 v232, a47\n\t" \
  "s_cmp_eq_u32 s96, -1\n\t" \
  "s_cbranch_scc0 L_ap2_0\n\t" \
  "L_apr2_0:\n\t" \
  "buffer_load_dwordx4 a[104:107], v188, s[36:39], s87 offen offset:2048 \n\t" \
  "s_waitcnt lgkmcnt(4)\n\t" \
  "v_mfma_f32_16x16x128_f8f6f4 v[8:11], a[0:7], v[144:151], v[8:11] cbsz:0 blgp:1\n\t" \
  "ds_read_b64_tr_b8 v[128:129], v180 offset:0\n\t" \
  "v_min3_u32 v228, v229, v230, v228\n\t" \
  "v_min3_u32 v228, v231, v232, v228\n\t" \
  "v_mfma_f32_16x16x128_f8f6f4 v[40:43], a[8:15], v[144:151], v[40:43] cbsz:0 blgp:1\n\t" \
  "ds_read_b64_tr_b8 v[130:131], v180 offset:1024\n\t" \
  "v_accvgpr_read_b32 v229, a48\n\t" \
  "v_accvgpr_read_b32 v230, a49\n\t" \
  "v_mfma_f32_16x16x128_f8f6f4 v[72:75], a[16:23], v[144:151], v[72:75] cbsz:0 blgp:1\n\t" \
  "ds_read_b64_tr_b8 v[132:133], v180 offset:8192\n\t" \
  "v_accvgpr_read_b32 v231, a50\n\t" \
  "v_accvgpr_read_b32 v232, a51\n\t" \
  "v_mfma_f32_16x16x128_f8f6f4 v[104:107], a[24:31], v[144:151], v[104:107] cbsz:0 blgp:1\n\t" \
  "ds_read_b64_tr_b8 v[134:135], v180 offset:9216\n\t" \
  "v_min3_u32 v228, v229, v230, v228\n\t" \
  "v_min3_u32 v228, v231, v232, v228\n\t" \
  "s_waitcnt lgkmcnt(4)\n\t" \
  "v_mfma_f32_16x16x128_f8f6f4 v[12:15], a[0:7], v[152:159], v[12:15] cbsz:0 blgp:1\n\t" \
  "ds_read_b64_tr_b8 v[136:137], v181 offset:0\n\t" \
  "v_accvgpr_read_b32 v229, a52\n\t" \
  "v_accvgpr_read_b32 v230, a53\n\t" \
  "v_mfma_f32_16x16x128_f8f6f4 v[44:47], a[8:15], v[152:159], v[44:47] cbsz:0 blgp:1\n\t" \
  "ds_read_b64_tr_b8 v[138:139], v181 offset:1024\n\t" \
  "v_accvgpr_read_b32 v231, a54\n\t" \
  "v_accvgpr_read_b32 v232, a55\n\t" \
  "v_mfma_f32_16x16x128_f8f6f4 v[76:79], a[16:23], v[152:159], v[76:79] cbsz:0 blgp:1\n\t" \
  "ds_read_b64_tr_b8 v[140:141], v181 offset:8192\n\t" \
  "v_min3_u32 v228, v229, v230, v228\n\t" \
  "v_min3_u32 v228, v231, v232, v228\n\t" \
  "v_mfma_f32_16x16x128_f8f6f4 v[108:111], a[24:31], v[152:159], v[108:111] cbsz:0 blgp:1\n\t" \
  "ds_read_b64_tr_b8 v[142:143], v181 offset:9216\n\t" \
  "s_waitcnt vmcnt(20)\n\t" \
  "v_accvgpr_read_b32 v229, a56\n\t" \
  "v_accvgpr_read_b32 v230, a57\n\t" \
  "s_waitcnt lgkmcnt(4)\n\t" \
  "v_mfma_f32_16x16x128_f8f6f4 v[16:19], a[0:7], v[128:135], v[16:19] cbsz:0 blgp:1\n\t" \
  "ds_read_b64_tr_b8 v[144:145], v182 offset:0\n\t" \
  "v_accvgpr_read_b32 v231, a58\n\t" \
  "v_accvgpr_read_b32 v232, a59\n\t" \
  "v_mfma_f32_16x16x128_f8f6f4 v[48:51], a[8:15], v[128:135], v[48:51] cbsz:0 blgp:1\n\t" \
  "ds_read_b64_tr_b8 v[146:147], v182 offset:1024\n\t" \
  "v_min3_u32 v228, v229, v230, v228\n\t" \
  "v_min3_u32 v228, v231, v232, v228\n\t" \
  "v_mfma_f32_16x16x128_f8f6f4 v[80:83], a[16:23], v[128:135], v[80:83] cbsz:0 blgp:1\n\t" \
  "ds_read_b64_tr_b8 v[148:149], v182 offset:8192\n\t" \
  "s_waitcnt vmcnt(19)\n\t" \
  "v_accvgpr_read_b32 v229, a60\n\t" \
  "v_accvgpr_read_b32 v230, a61\n\t" \
  "v_mfma_f32_16x16x128_f8f6f4 v[112:115], a[24:31], v[128:135], v[112:115] cbsz:0 blgp:1\n\t" \
  "ds_read_b64_tr_b8 v[150:151], v182 offset:9216\n\t" \
  "v_accvgpr_read_b32 v231, a62\n\t" \
  "v_accvgpr_read_b32 v232, a63\n\t" \
  "s_waitcnt lgkmcnt(4)\n\t" \
  "v_mfma_f32_16x16x128_f8f6f4 v[20:23], a[0:7], v[136:143], v[20:23] cbsz:0 blgp:1\n\t" \
  "ds_read_b64_tr_b8 v[152:153], v183 offset:0\n\t" \
  "v_min3_u32 v228, v229, v230, v228\n\t" \
  "v_min3_u32 v228, v231, v232, v228\n\t" \
  "v_mfma_f32_16x16x128_f8f6f4 v[52:55], a[8:15], v[136:143], v[52:55] cbsz:0 blgp:1\n\t" \
  "ds_read_b64_tr_b8 v[154:155], v183 offset:1024\n\t" \
  "v_mfma_f32_16x16x128_f8f6f4 v[84:87], a[16:23], v[136:143], v[84:87] cbsz:0 blgp:1\n\t" \
  "ds_read_b64_tr_b8 v[156:157], v183 offset:8192\n\t" \
  "v_cmp_eq_u32 vcc, 0, v228\n\t" \
  "s_nop 4\n\t" \
  "s_cmp_lg_u64 vcc, 0\n\t" \
  "s_cbranch_scc1 L_v9_0\n\t" \
  "L_vr9_0:\n\t" \
  "v_mfma_f32_16x16x128_f8f6f4 v[116:119], a[24:31], v[136:143], v[116:119] cbsz:0 blgp:1\n\t" \
  "ds_read_b64_tr_b8 v[158:159], v183 offset:9216\n\t" \
  "s_waitcnt lgkmcnt(4)\n\t" \
  "v_mfma_f32_16x16x128_f8f6f4 v[24:27], a[0:7], v[144:151], v[24:27] cbsz:0 blgp:1\n\t" \
  "s_barrier\n\t" \
  "ds_read_b64_tr_b8 v[128:129], v176 offset:16384\n\t" \
  "v_mfma_f32_16x16x128_f8f6f4 v[56:59], a[8:15], v[144:151], v[56:59] cbsz:0 blgp:1\n\t" \
  "ds_read_b64_tr_b8 v[130:131], v176 offset:17408\n\t" \
  "v_mfma_f32_16x16x128_f8f6f4 v[88:91], a[16:23], v[144:151], v[88:91] cbsz:0 blgp:1\n\t" \
  "ds_read_b64_tr_b8 v[132:133], v176 offset:24576\n\t" \
  "v_mfma_f32_16x16x128_f8f6f4 v[120:123], a[24:31], v[144:151], v[120:123] cbsz:0 blgp:1\n\t" \
  "ds_read_b64_tr_b8 v[134:135], v176 offset:25600\n\t" \
  "s_cmp_eq_u32 s96, -1\n\t" \
  "s_cbranch_scc0 L_au2_0\n\t" \
  "L_aur2_0:\n\t" \
  "s_waitcnt lgkmcnt(4)\n\t" \
  "v_mfma_f32_16x16x128_f8f6f4 v[28:31], a[0:7], v[152:159], v[28:31] cbsz:0 blgp:1\n\t" \
  "ds_read_b64_tr_b8 v[136:137], v177 offset:16384\n\t" \
  "v_mfma_f32_16x16x128_f8f6f4 v[60:63], a[8:15], v[152:159], v[60:63] cbsz:0 blgp:1\n\t" \
  "ds_read_b64_tr_b8 v[138:139], v177 offset:17408\n\t" \
  "v_mfma_f32_16x16x128_f8f6f4 v[92:95], a[16:23], v[152:159], v[92:95] cbsz:0 blgp:1\n\t" \
  "ds_read_b64_tr_b8 v[140:141], v177 offset:24576\n\t" \
  "v_mfma_f32_16x16x128_f8f6f4 v[124:127], a[24:31], v[152:159], v[124:127] cbsz:0 blgp:1\n\t" \
  "ds_read_b64_tr_b8 v[142:143], v177 offset:25600\n\t" \
  "s_waitcnt lgkmcnt(4)\n\t" \
  "v_mfma_f32_16x16x128_f8f6f4 v[0:3], a[32:39], v[128:135], v[0:3] cbsz:0 blgp:1\n\t" \
  "ds_read_b64_tr_b8 v[144:145], v178 offset:16384\n\t" \
  "v_mfma_f32_16x16x128_f8f6f4 v[32:35], a[40:47], v[128:135], v[32:35] cbsz:0 blgp:1\n\t" \
  "ds_read_b64_tr_b8 v[146:147], v178 offset:17408\n\t" \
  "v_mfma_f32_16x16x128_f8f6f4 v[64:67], a[48:55], v[128:135], v[64:67] cbsz:0 blgp:1\n\t" \
  "ds_read_b64_tr_b8 v[148:149], v178 offset:24576\n\t" \
  "v_mfma_f32_16x16x128_f8f6f4 v[96:99], a[56:63], v[128:135], v[96:99] cbsz:0 blgp:1\n\t" \
  "ds_read_b64_tr_b8 v[150:151], v178 offset:25600\n\t" \
  "s_waitcnt lgkmcnt(4)\n\t" \
  "v_mfma_f32_16x16x128_f8f6f4 v[4:7], a[32:39], v[136:143], v[4:7] cbsz:0 blgp:1\n\t" \
  "ds_read_b64_tr_b8 v[152:153], v179 offset:16384\n\t" \
  "v_mfma_f32_16x16x128_f8f6f4 v[36:39], a[40:47], v[136:143], v[36:39] cbsz:0 blgp:1\n\t" \
  "ds_read_b64_tr_b8 v[154:155], v179 offset:17408\n\t" \
  "v_mfma_f32_16x16x128_f8f6f4 v[68:71], a[48:55], v[136:143], v[68:71] cbsz:0 blgp:1\n\t" \
  "ds_read_b64_tr_b8 v[156:157], v179 offset:24576\n\t" \
  "buffer_load_dwordx4 a[108:111], v188, s[36:39], s87 offen offset:3072 \n\t" \
  "v_mfma_f32_16x16x128_f8f6f4 v[100:103], a[56:63], v[136:143], v[100:103] cbsz:0 blgp:1\n\t" \
  "ds_read_b64_tr_b8 v[158:159], v179 offset:25600\n\t" \
  "s_cmp_eq_u32 s96, -1\n\t" \
  "s_cbranch_scc0 L_ap3_0\n\t" \
  "L_apr3_0:\n\t" \
  "s_waitcnt lgkmcnt(4)\n\t" \
  "v_mfma_f32_16x16x128_f8f6f4 v[8:11], a[32:39], v[144:151], v[8:11] cbsz:0 blgp:1\n\t" \
  "ds_read_b64_tr_b8 v[128:129], v180 offset:16384\n\t" \
  "buffer_load_dwordx4 a[112:115], v189, s[36:39], s87 offen offset:0 \n\t" \
  "v_mfma_f32_16x16x128_f8f6f4 v[40:43], a[40:47], v[144:151], v[40:43] cbsz:0 blgp:1\n\t" \
  "ds_read_b64_tr_b8 v[130:131], v180 offset:17408\n\t" \
  "v_mfma_f32_16x16x128_f8f6f4 v[72:75], a[48:55], v[144:151], v[72:75] cbsz:0 blgp:1\n\t" \
  "ds_read_b64_tr_b8 v[132:133], v180 offset:24576\n\t" \
  "buffer_load_dwordx4 a[116:119], v189, s[36:39], s87 offen offset:1024 \n\t" \
  "v_mfma_f32_16x16x128_f8f6f4 v[104:107], a[56:63], v[144:151], v[104:107] cbsz:0 blgp:1\n\t" \
  "ds_read_b64_tr_b8 v[134:135], v180 offset:25600\n\t" \
  "s_waitcnt lgkmcnt(4)\n\t" \
  "v_mfma_f32_16x16x128_f8f6f4 v[12:15], a[32:39], v[152:159], v[12:15] cbsz:0 blgp:1\n\t" \
  "ds_read_b64_tr_b8 v[136:137], v181 offset:16384\n\t" \
  "s_cmp_eq_u32 s29, 3\n\t" \
  "s_cbranch_scc0 L_z_0\n\t" \
  "s_cmp_lt_u32 s10, 224\n\t" \
  "s_cbranch_scc0 L_z_0\n\t" \
  "v_mov_b32 v200, 0\n\t" \
  "v_mov_b32 v201, 0\n\t" \
  "v_mov_b32 v202, 0\n\t" \
  "v_mov_b32 v203, 0\n\t" \
  "v_add_u32 v212, s100, v188\n\t" \
  "global_store_dwordx4 v212, v[200:203], s[26:27] sc1\n\t" \
  "global_store_dwordx4 v212, v[200:203], s[26:27] offset:1024 sc1\n\t" \
  "L_z_0:\n\t" \
  "buffer_load_dwordx4 a[120:123], v189, s[36:39], s87 offen offset:2048 \n\t" \
  "v_mfma_f32_16x16x128_f8f6f4 v[44:47], a[40:47], v[152:159], v[44:47] cbsz:0 blgp:1\n\t" \
  "ds_read_b64_tr_b8 v[138:139], v181 offset:17408\n\t" \
  "v_mfma_f32_16x16x128_f8f6f4 v[76:79], a[48:55], v[152:159], v[76:79] cbsz:0 blgp:1\n\t" \
  "ds_read_b64_tr_b8 v[140:141], v181 offset:24576\n\t" \
  "buffer_load_dwordx4 a[124:127], v189, s[36:39], s87 offen offset:3072 \n\t" \
  "v_mfma_f32_16x16x128_f8f6f4 v[108:111], a[56:63], v[152:159], v[108:111] cbsz:0 blgp:1\n\t" \
  "ds_read_b64_tr_b8 v[142:143], v181 offset:25600\n\t" \
  "s_waitcnt lgkmcnt(4)\n\t" \
  "v_mfma_f32_16x16x128_f8f6f4 v[16:19], a[32:39], v[128:135], v[16:19] cbsz:0 blgp:1\n\t" \
  "ds_read_b64_tr_b8 v[144:145], v182 offset:16384\n\t" \
  "s_bfe_u32 s84, s96, 262148\n\t" \
  "s_cmp_eq_u32 s84, 15\n\t" \
  "s_cbranch_scc0 L_q12_0\n\t" \
  "L_qr12_0:\n\t" \
  "s_mov_b32 s87, 524288\n\t" \
  "buffer_load_dwordx4 a[128:131], v188, s[36:39], s87 offen offset:0 \n\t" \
  "v_mfma_f32_16x16x128_f8f6f4 v[48:51], a[40:47], v[128:135], v[48:51] cbsz:0 blgp:1\n\t" \
  "ds_read_b64_tr_b8 v[146:147], v182 offset:17408\n\t" \
  "v_mfma_f32_16x16x128_f8f6f4 v[80:83], a[48:55], v[128:135], v[80:83] cbsz:0 blgp:1\n\t" \
  "ds_read_b64_tr_b8 v[148:149], v182 offset:24576\n\t" \
  "buffer_load_dwordx4 a[132:135], v188, s[36:39], s87 offen offset:1024 \n\t" \
  "v_mfma_f32_16x16x128_f8f6f4 v[112:115], a[56:63], v[128:135], v[112:115] cbsz:0 blgp:1\n\t" \
  "ds_read_b64_tr_b8 v[150:151], v182 offset:25600\n\t" \
  "s_waitcnt lgkmcnt(4)\n\t" \
  "v_mfma_f32_16x16x128_f8f6f4 v[20:23], a[32:39], v[136:143], v[20:23] cbsz:0 blgp:1\n\t" \
  "ds_read_b64_tr_b8 v[152:153], v183 offset:16384\n\t" \
  "buffer_load_dwordx4 a[136:139], v188, s[36:39], s87 offen offset:2048 \n\t" \
  "v_mfma_f32_16x16x128_f8f6f4 v[52:55], a[40:47], v[136:143], v[52:55] cbsz:0 blgp:1\n\t" \
  "ds_read_b64_tr_b8 v[154:155], v183 offset:17408\n\t" \
  "s_add_u32 m0, s77, 32768\n\t" \
  "s_add_u32 s90, s69, 2097152\n\t" \
  "buffer_load_dwordx4 v186, s[40:43], s90 offen offset:0  lds\n\t" \
  "v_mfma_f32_16x16x128_f8f6f4 v[84:87], a[48:55], v[136:143], v[84:87] cbsz:0 blgp:1\n\t" \
  "ds_read_b64_tr_b8 v[156:157], v183 offset:24576\n\t" \
  "s_add_u32 m0, s77, 49152\n\t" \
  "s_add_u32 s90, s70, 2097152\n\t" \
  "buffer_load_dwordx4 v186, s[40:43], s90 offen offset:0  lds\n\t" \
  "v_mfma_f32_16x16x128_f8f6f4 v[116:119], a[56:63], v[136:143], v[116:119] cbsz:0 blgp:1\n\t" \
  "ds_read_b64_tr_b8 v[158:159], v183 offset:25600\n\t" \
  "s_add_u32 m0, s77, 32768\n\t" \
  "s_add_u32 s90, s69, 2097152\n\t" \
  "buffer_load_dwordx4 v187, s[40:43], s90 offen offset:1024  lds\n\t" \
  "s_waitcnt lgkmcnt(4)\n\t" \
  "v_mfma_f32_16x16x128_f8f6f4 v[24:27], a[32:39], v[144:151], v[24:27] cbsz:0 blgp:1\n\t" \
  "s_barrier\n\t" \
  "ds_read_b64_tr_b8 v[128:129], v160 offset:0\n\t" \
  "v_mfma_f32_16x16x128_f8f6f4 v[56:59], a[40:47], v[144:151], v[56:59] cbsz:0 blgp:1\n\t" \
  "ds_read_b64_tr_b8 v[130:131], v160 offset:1024\n\t" \
  "s_add_u32 m0, s77, 49152\n\t" \
  "s_add_u32 s90, s70, 2097152\n\t" \
  "buffer_load_dwordx4 v187, s[40:43], s90 offen offset:1024  lds\n\t" \
  "v_mfma_f32_16x16x128_f8f6f4 v[88:91], a[48:55], v[144:151], v[88:91] cbsz:0 blgp:1\n\t" \
  "ds_read_b64_tr_b8 v[132:133], v160 offset:8192\n\t" \
  "s_add_u32 m0, s77, 65536\n\t" \
  "s_add_u32 s90, s71, 2097152\n\t" \
  "buffer_load_dwordx4 v186, s[40:43], s90 offen offset:0  lds\n\t" \
  "v_mfma_f32_16x16x128_f8f6f4 v[120:123], a[56:63], v[144:151], v[120:123] cbsz:0 blgp:1\n\t" \
  "ds_read_b64_tr_b8 v[134:135], v160 offset:9216\n\t" \
  "s_cmp_eq_u32 s96, -1\n\t" \
  "s_cbranch_scc0 L_au3_0\n\t" \
  "L_aur3_0:\n\t" \
  "s_add_u32 m0, s77, 81920\n\t" \
  "s_add_u32 s90, s72, 2097152\n\t" \
  "buffer_load_dwordx4 v186, s[40:43], s90 offen offset:0  lds\n\t" \
  "s_waitcnt lgkmcnt(4)\n\t" \
  "v_mfma_f32_16x16x128_f8f6f4 v[28:31], a[32:39], v[152:159], v[28:31] cbsz:0 blgp:1\n\t" \
  "ds_read_b64_tr_b8 v[136:137], v161 offset:0\n\t" \
  "s_add_u32 m0, s77, 65536\n\t" \
  "s_add_u32 s90, s71, 2097152\n\t" \
  "buffer_load_dwordx4 v187, s[40:43], s90 offen offset:1024  lds\n\t" \
  "v_mfma_f32_16x16x128_f8f6f4 v[60:63], a[40:47], v[152:159], v[60:63] cbsz:0 blgp:1\n\t" \
  "ds_read_b64_tr_b8 v[138:139], v161 offset:1024\n\t" \
  "s_add_u32 m0, s77, 81920\n\t" \
  "s_add_u32 s90, s72, 2097152\n\t" \
  "buffer_load_dwordx4 v187, s[40:43], s90 offen offset:1024  lds\n\t" \
  "v_mfma_f32_16x16x128_f8f6f4 v[92:95], a[48:55], v[152:159], v[92:95] cbsz:0 blgp:1\n\t" \
  "ds_read_b64_tr_b8 v[140:141], v161 offset:8192\n\t" \
  "buffer_load_dwordx4 a[140:143], v188, s[36:39], s87 offen offset:3072 \n\t" \
  "v_mfma_f32_16x16x128_f8f6f4 v[124:127], a[56:63], v[152:159], v[124:127] cbsz:0 blgp:1\n\t" \
  "ds_read_b64_tr_b8 v[142:143], v161 offset:9216\n\t" \
  "s_waitcnt vmcnt(34) lgkmcnt(4)\n\t" \
  "v_mfma_f32_16x16x128_f8f6f4 v[0:3], a[64:71], v[128:135], v[0:3] cbsz:0 blgp:1\n\t" \
  "ds_read_b64_tr_b8 v[144:145], v162 offset:0\n\t" \
  "buffer_load_dwordx4 a[144:147], v189, s[36:39], s87 offen offset:0 \n\t" \
  "s_waitcnt vmcnt(33)\n\t" \
  "v_mfma_f32_16x16x128_f8f6f4 v[32:35], a[72:79], v[128:135], v[32:35] cbsz:0 blgp:1\n\t" \
  "ds_read_b64_tr_b8 v[146:147], v162 offset:1024\n\t" \
  "s_waitcnt vmcnt(31)\n\t" \
  "v_mfma_f32_16x16x128_f8f6f4 v[64:67], a[80:87], v[128:135], v[64:67] cbsz:0 blgp:1\n\t" \
  "ds_read_b64_tr_b8 v[148:149], v162 offset:8192\n\t" \
  "buffer_load_dwordx4 a[148:151], v189, s[36:39], s87 offen offset:1024 \n\t" \
  "s_waitcnt vmcnt(30)\n\t" \
  "v_mfma_f32_16x16x128_f8f6f4 v[96:99], a[88:95], v[128:135], v[96:99] cbsz:0 blgp:1\n\t" \
  "ds_read_b64_tr_b8 v[150:151], v162 offset:9216\n\t" \
  "s_waitcnt lgkmcnt(4)\n\t" \
  "v_mfma_f32_16x16x128_f8f6f4 v[4:7], a[64:71], v[136:143], v[4:7] cbsz:0 blgp:1\n\t" \
  "ds_read_b64_tr_b8 v[152:153], v163 offset:0\n\t" \
  "buffer_load_dwordx4 a[152:155], v189, s[36:39], s87 offen offset:2048 \n\t" \
  "v_mfma_f32_16x16x128_f8f6f4 v[36:39], a[72:79], v[136:143], v[36:39] cbsz:0 blgp:1\n\t" \
  "ds_read_b64_tr_b8 v[154:155], v163 offset:1024\n\t" \
  "v_mfma_f32_16x16x128_f8f6f4 v[68:71], a[80:87], v[136:143], v[68:71] cbsz:0 blgp:1\n\t" \
  "ds_read_b64_tr_b8 v[156:157], v163 offset:8192\n\t" \
  "buffer_load_dwordx4 a[156:159], v189, s[36:39], s87 offen offset:3072 \n\t" \
  "v_mfma_f32_16x16x128_f8f6f4 v[100:103], a[88:95], v[136:143], v[100:103] cbsz:0 blgp:1\n\t" \
  "ds_read_b64_tr_b8 v[158:159], v163 offset:9216\n\t" \
  "s_waitcnt lgkmcnt(4)\n\t" \
  "v_mfma_f32_16x16x128_f8f6f4 v[8:11], a[64:71], v[144:151], v[8:11] cbsz:0 blgp:1\n\t" \
  "ds_read_b64_tr_b8 v[128:129], v164 offset:0\n\t" \
  "s_bfe_u32 s84, s96, 262152\n\t" \
  "s_cmp_eq_u32 s84, 15\n\t" \
  "s_cbranch_scc0 L_q13_0\n\t" \
  "L_qr13_0:\n\t" \
  "s_mov_b32 s87, 589824\n\t" \
  "buffer_load_dwordx4 a[160:163], v188, s[36:39], s87 offen offset:0 \n\t" \
  "v_mfma_f32_16x16x128_f8f6f4 v[40:43], a[72:79], v[144:151], v[40:43] cbsz:0 blgp:1\n\t" \
  "ds_read_b64_tr_b8 v[130:131], v164 offset:1024\n\t" \
  "v_mfma_f32_16x16x128_f8f6f4 v[72:75], a[80:87], v[144:151], v[72:75] cbsz:0 blgp:1\n\t" \
  "ds_read_b64_tr_b8 v[132:133], v164 offset:8192\n\t" \
  "buffer_load_dwordx4 a[164:167], v188, s[36:39], s87 offen offset:1024 \n\t" \
  "v_mfma_f32_16x16x128_f8f6f4 v[104:107], a[88:95], v[144:151], v[104:107] cbsz:0 blgp:1\n\t" \
  "ds_read_b64_tr_b8 v[134:135], v164 offset:9216\n\t" \
  "s_waitcnt lgkmcnt(4)\n\t" \
  "v_mfma_f32_16x16x128_f8f6f4 v[12:15], a[64:71], v[152:159], v[12:15] cbsz:0 blgp:1\n\t" \
  "ds_read_b64_tr_b8 v[136:137], v165 offset:0\n\t" \
  "buffer_load_dwordx4 a[168:171], v188, s[36:39], s87 offen offset:2048 \n\t" \
  "v_mfma_f32_16x16x128_f8f6f4 v[44:47], a[72:79], v[152:159], v[44:47] cbsz:0 blgp:1\n\t" \
  "ds_read_b64_tr_b8 v[138:139], v165 offset:1024\n\t" \
  "v_mfma_f32_16x16x128_f8f6f4 v[76:79], a[80:87], v[152:159], v[76:79] cbsz:0 blgp:1\n\t" \
  "ds_read_b64_tr_b8 v[140:141], v165 offset:8192\n\t" \
  "s_mov_b32 s8, 0\n\t" \
  "buffer_load_dwordx4 a[172:175], v188, s[36:39], s87 offen offset:3072 \n\t" \
  "v_mfma_f32_16x16x128_f8f6f4 v[108:111], a[88:95], v[152:159], v[108:111] cbsz:0 blgp:1\n\t" \
  "ds_read_b64_tr_b8 v[142:143], v165 offset:9216\n\t" \
  "global_load_dword v196, v195, s[26:27] offset:2048 sc1\n\t" \
  "s_waitcnt lgkmcnt(4)\n\t" \
  "v_mfma_f32_16x16x128_f8f6f4 v[16:19], a[64:71], v[128:135], v[16:19] cbsz:0 blgp:1\n\t" \
  "ds_read_b64_tr_b8 v[144:145], v166 offset:0\n\t" \
  "buffer_load_dwordx4 a[176:179], v189, s[36:39], s87 offen offset:0 \n\t" \
  "v_mfma_f32_16x16x128_f8f6f4 v[48:51], a[72:79], v[128:135], v[48:51] cbsz:0 blgp:1\n\t" \
  "ds_read_b64_tr_b8 v[146:147], v166 offset:1024\n\t" \
  "v_mfma_f32_16x16x128_f8f6f4 v[80:83], a[80:87], v[128:135], v[80:83] cbsz:0 blgp:1\n\t" \
  "ds_read_b64_tr_b8 v[148:149], v166 offset:8192\n\t" \
  "buffer_load_dwordx4 a[180:183], v189, s[36:39], s87 offen offset:1024 \n\t" \
  "v_mfma_f32_16x16x128_f8f6f4 v[112:115], a[88:95], v[128:135], v[112:115] cbsz:0 blgp:1\n\t" \
  "ds_read_b64_tr_b8 v[150:151], v166 offset:9216\n\t" \
  "s_waitcnt lgkmcnt(4)\n\t" \
  "v_mfma_f32_16x16x128_f8f6f4 v[20:23], a[64:71], v[136:143], v[20:23] cbsz:0 blgp:1\n\t" \
  "ds_read_b64_tr_b8 v[152:153], v167 offset:0\n\t" \
  "buffer_load_dwordx4 a[184:187], v189, s[36:39], s87 offen offset:2048 \n\t" \
  "v_mfma_f32_16x16x128_f8f6f4 v[52:55], a[72:79], v[136:143], v[52:55] cbsz:0 blgp:1\n\t" \
  "ds_read_b64_tr_b8 v[154:155], v167 offset:1024\n\t" \
  "v_mfma_f32_16x16x128_f8f6f4 v[84:87], a[80:87], v[136:143], v[84:87] cbsz:0 blgp:1\n\t" \
  "ds_read_b64_tr_b8 v[156:157], v167 offset:8192\n\t" \
  "buffer_load_dwordx4 a[188:191], v189, s[36:39], s87 offen offset:3072 \n\t" \
  "v_mfma_f32_16x16x128_f8f6f4 v[116:119], a[88:95], v[136:143], v[116:119] cbsz:0 blgp:1\n\t" \
  "ds_read_b64_tr_b8 v[158:159], v167 offset:9216\n\t" \
  "s_waitcnt lgkmcnt(4)\n\t" \
  "v_mfma_f32_16x16x128_f8f6f4 v[24:27], a[64:71], v[144:151], v[24:27] cbsz:0 blgp:1\n\t" \
  "s_barrier\n\t" \
  "ds_read_b64_tr_b8 v[128:129], v160 offset:16384\n\t" \
  "v_mfma_f32_16x16x128_f8f6f4 v[56:59], a[72:79], v[144:151], v[56:59] cbsz:0 blgp:1\n\t" \
  "ds_read_b64_tr_b8 v[130:131], v160 offset:17408\n\t" \
  "s_bfe_u32 s84, s96, 262156\n\t" \
  "s_cmp_eq_u32 s84, 15\n\t" \
  "s_cbranch_scc0 L_q14_0\n\t" \
  "L_qr14_0:\n\t" \
  "s_mov_b32 s87, 655360\n\t" \
  "buffer_load_dwordx4 a[192:195], v188, s[36:39], s87 offen offset:0 \n\t" \
  "v_mfma_f32_16x16x128_f8f6f4 v[88:91], a[80:87], v[144:151], v[88:91] cbsz:0 blgp:1\n\t" \
  "ds_read_b64_tr_b8 v[132:133], v160 offset:24576\n\t" \
  "v_mfma_f32_16x16x128_f8f6f4 v[120:123], a[88:95], v[144:151], v[120:123] cbsz:0 blgp:1\n\t" \
  "ds_read_b64_tr_b8 v[134:135], v160 offset:25600\n\t" \
  "buffer_load_dwordx4 a[196:199], v188, s[36:39], s87 offen offset:1024 \n\t" \
  "s_waitcnt lgkmcnt(4)\n\t" \
  "v_mfma_f32_16x16x128_f8f6f4 v[28:31], a[64:71], v[152:159], v[28:31] cbsz:0 blgp:1\n\t" \
  "ds_read_b64_tr_b8 v[136:137], v161 offset:16384\n\t" \
  "v_mfma_f32_16x16x128_f8f6f4 v[60:63], a[72:79], v[152:159], v[60:63] cbsz:0 blgp:1\n\t" \
  "ds_read_b64_tr_b8 v[138:139], v161 offset:17408\n\t" \
  "buffer_load_dwordx4 a[200:203], v188, s[36:39], s87 offen offset:2048 \n\t" \
  "v_mfma_f32_16x16x128_f8f6f4 v[92:95], a[80:87], v[152:159], v[92:95] cbsz:0 blgp:1\n\t" \
  "ds_read_b64_tr_b8 v[140:141], v161 offset:24576\n\t" \
  "v_mfma_f32_16x16x128_f8f6f4 v[124:127], a[88:95], v[152:159], v[124:127] cbsz:0 blgp:1\n\t" \
  "ds_read_b64_tr_b8 v[142:143], v161 offset:25600\n\t" \
  "buffer_load_dwordx4 a[204:207], v188, s[36:39], s87 offen offset:3072 \n\t" \
  "s_waitcnt vmcnt(43) lgkmcnt(4)\n\t" \
  "v_mfma_f32_16x16x128_f8f6f4 v[0:3], a[96:103], v[128:135], v[0:3] cbsz:0 blgp:1\n\t" \
  "ds_read_b64_tr_b8 v[144:145], v162 offset:16384\n\t" \
  "s_waitcnt vmcnt(33)\n\t" \
  "v_mfma_f32_16x16x128_f8f6f4 v[32:35], a[104:111], v[128:135], v[32:35] cbsz:0 blgp:1\n\t" \
  "ds_read_b64_tr_b8 v[146:147], v162 offset:17408\n\t" \
  "buffer_load_dwordx4 a[208:211], v189, s[36:39], s87 offen offset:0 \n\t" \
  "s_waitcnt vmcnt(32)\n\t" \
  "v_mfma_f32_16x16x128_f8f6f4 v[64:67], a[112:119], v[128:135], v[64:67] cbsz:0 blgp:1\n\t" \
  "ds_read_b64_tr_b8 v[148:149], v162 offset:24576\n\t" \
  "s_waitcnt vmcnt(30)\n\t" \
  "v_mfma_f32_16x16x128_f8f6f4 v[96:99], a[120:127], v[128:135], v[96:99] cbsz:0 blgp:1\n\t" \
  "ds_read_b64_tr_b8 v[150:151], v162 offset:25600\n\t" \
  "s_waitcnt vmcnt(9)\n\t" \
  "v_cmp_le_u32 vcc, s79, v196\n\t" \
  "s_nop 4\n\t" \
  "s_andn2_b32 s85, vcc_lo, s8\n\t" \
  "s_or_b32 s8, s8, vcc_lo\n\t" \
  "s_and_b32 exec_lo, s85, 0xfffffff\n\t" \
  "s_mov_b32 exec_hi, 0\n\t" \
  "s_cbranch_execz L_u3_0\n\t" \
  "s_mov_b32 s84, 917504\n\t" \
  "buffer_load_dword v210, v198, s[36:39], s84 offen \n\t" \
  "L_u3_0:\n\t" \
  "s_mov_b64 exec, -1\n\t" \
  "buffer_load_dwordx4 a[212:215], v189, s[36:39], s87 offen offset:1024 \n\t" \
  "s_waitcnt lgkmcnt(4)\n\t" \
  "v_mfma_f32_16x16x128_f8f6f4 v[4:7], a[96:103], v[136:143], v[4:7] cbsz:0 blgp:1\n\t" \
  "ds_read_b64_tr_b8 v[152:153], v163 offset:16384\n\t" \
  "v_mfma_f32_16x16x128_f8f6f4 v[36:39], a[104:111], v[136:143], v[36:39] cbsz:0 blgp:1\n\t" \
  "ds_read_b64_tr_b8 v[154:155], v163 offset:17408\n\t" \
  "buffer_load_dwordx4 a[216:219], v189, s[36:39], s87 offen offset:2048 \n\t" \
  "v_mfma_f32_16x16x128_f8f6f4 v[68:71], a[112:119], v[136:143], v[68:71] cbsz:0 blgp:1\n\t" \
  "ds_read_b64_tr_b8 v[156:157], v163 offset:24576\n\t" \
  "v_mfma_f32_16x16x128_f8f6f4 v[100:103], a[120:127], v[136:143], v[100:103] cbsz:0 blgp:1\n\t" \
  "ds_read_b64_tr_b8 v[158:159], v163 offset:25600\n\t" \
  "buffer_load_dwordx4 a[220:223], v189, s[36:39], s87 offen offset:3072 \n\t" \
  "s_waitcnt lgkmcnt(4)\n\t" \
  "v_mfma_f32_16x16x128_f8f6f4 v[8:11], a[96:103], v[144:151], v[8:11] cbsz:0 blgp:1\n\t" \
  "ds_read_b64_tr_b8 v[128:129], v164 offset:16384\n\t" \
  "v_mfma_f32_16x16x128_f8f6f4 v[40:43], a[104:111], v[144:151], v[40:43] cbsz:0 blgp:1\n\t" \
  "ds_read_b64_tr_b8 v[130:131], v164 offset:17408\n\t" \
  "s_bfe_u32 s84, s96, 262160\n\t" \
  "s_cmp_eq_u32 s84, 15\n\t" \
  "s_cbranch_scc0 L_q15_0\n\t" \
  "L_qr15_0:\n\t" \
  "s_mov_b32 s87, 720896\n\t" \
  "buffer_load_dwordx4 a[224:227], v188, s[36:39], s87 offen offset:0 \n\t" \
  "v_mfma_f32_16x16x128_f8f6f4 v[72:75], a[112:119], v[144:151], v[72:75] cbsz:0 blgp:1\n\t" \
  "ds_read_b64_tr_b8 v[132:133], v164 offset:24576\n\t" \
  "s_add_u32 m0, s76, 98304\n\t" \
  "s_add_u32 s90, s73, 2097152\n\t" \
  "buffer_load_dwordx4 v184, s[40:43], s90 offen offset:0  lds\n\t" \
  "v_mfma_f32_16x16x128_f8f6f4 v[104:107], a[120:127], v[144:151], v[104:107] cbsz:0 blgp:1\n\t" \
  "ds_read_b64_tr_b8 v[134:135], v164 offset:25600\n\t" \
  "s_add_u32 m0, s76, 98304\n\t" \
  "s_add_u32 s90, s73, 2097152\n\t" \
  "buffer_load_dwordx4 v185, s[40:43], s90 offen offset:1024  lds\n\t" \
  "s_waitcnt lgkmcnt(4)\n\t" \
  "v_mfma_f32_16x16x128_f8f6f4 v[12:15], a[96:103], v[152:159], v[12:15] cbsz:0 blgp:1\n\t" \
  "ds_read_b64_tr_b8 v[136:137], v165 offset:16384\n\t" \
  "s_add_u32 m0, s76, 114688\n\t" \
  "s_add_u32 s90, s74, 2097152\n\t" \
  "buffer_load_dwordx4 v184, s[40:43], s90 offen offset:0  lds\n\t" \
  "v_mfma_f32_16x16x128_f8f6f4 v[44:47], a[104:111], v[152:159], v[44:47] cbsz:0 blgp:1\n\t" \
  "ds_read_b64_tr_b8 v[138:139], v165 offset:17408\n\t" \
  "s_add_u32 m0, s76, 114688\n\t" \
  "s_add_u32 s90, s74, 2097152\n\t" \
  "buffer_load_dwordx4 v185, s[40:43], s90 offen offset:1024  lds\n\t" \
  "v_mfma_f32_16x16x128_f8f6f4 v[76:79], a[112:119], v[152:159], v[76:79] cbsz:0 blgp:1\n\t" \
  "ds_read_b64_tr_b8 v[140:141], v165 offset:24576\n\t" \
  "s_add_u32 m0, s76, 131072\n\t" \
  "s_add_u32 s90, s68, 4194304\n\t" \
  "buffer_load_dwordx4 v184, s[40:43], s90 offen offset:0  lds\n\t" \
  "v_mfma_f32_16x16x128_f8f6f4 v[108:111], a[120:127], v[152:159], v[108:111] cbsz:0 blgp:1\n\t" \
  "ds_read_b64_tr_b8 v[142:143], v165 offset:25600\n\t" \
  "s_cmp_eq_u32 s8, -1\n\t" \
  "s_cbranch_scc0 L_ap5_0\n\t" \
  "L_apr5_0:\n\t" \
  "s_add_u32 m0, s76, 131072\n\t" \
  "s_add_u32 s90, s68, 4194304\n\t" \
  "buffer_load_dwordx4 v185, s[40:43], s90 offen offset:1024  lds\n\t" \
  "s_waitcnt lgkmcnt(4)\n\t" \
  "v_mfma_f32_16x16x128_f8f6f4 v[16:19], a[96:103], v[128:135], v[16:19] cbsz:0 blgp:1\n\t" \
  "ds_read_b64_tr_b8 v[144:145], v166 offset:16384\n\t" \
  "s_add_u32 m0, s76, 147456\n\t" \
  "s_add_u32 s90, s69, 4194304\n\t" \
  "buffer_load_dwordx4 v184, s[40:43], s90 offen offset:0  lds\n\t" \
  "v_mfma_f32_16x16x128_f8f6f4 v[48:51], a[104:111], v[128:135], v[48:51] cbsz:0 blgp:1\n\t" \
  "ds_read_b64_tr_b8 v[146:147], v166 offset:17408\n\t" \
  "s_add_u32 m0, s76, 147456\n\t" \
  "s_add_u32 s90, s69, 4194304\n\t" \
  "buffer_load_dwordx4 v185, s[40:43], s90 offen offset:1024  lds\n\t" \
  "v_mfma_f32_16x16x128_f8f6f4 v[80:83], a[112:119], v[128:135], v[80:83] cbsz:0 blgp:1\n\t" \
  "ds_read_b64_tr_b8 v[148:149], v166 offset:24576\n\t" \
  "s_add_u32 s99, s29, 24\n\t" \
  "s_min_u32 s99, s99, 31\n\t" \
  "v_readlane_b32 s99, v211, s99\n\t" \
  "s_mov_b32 exec_lo, s15\n\t" \
  "s_mov_b32 exec_hi, 0\n\t" \
  "s_cbranch_execz L_c24_0\n\t" \
  "s_nop 1\n\t" \
  "buffer_load_dword v210, v222, s[60:63], s99 offen\n\t" \
  "L_c24_0:\n\t" \
  "s_mov_b64 exec, -1\n\t" \
  "v_mfma_f32_16x16x128_f8f6f4 v[112:115], a[120:127], v[128:135], v[112:115] cbsz:0 blgp:1\n\t" \
  "ds_read_b64_tr_b8 v[150:151], v166 offset:25600\n\t" \
  "buffer_load_dwordx4 a[228:231], v188, s[36:39], s87 offen offset:1024 \n\t" \
  "s_waitcnt lgkmcnt(4)\n\t" \
  "v_mfma_f32_16x16x128_f8f6f4 v[20:23], a[96:103], v[136:143], v[20:23] cbsz:0 blgp:1\n\t" \
  "ds_read_b64_tr_b8 v[152:153], v167 offset:16384\n\t" \
  "v_mfma_f32_16x16x128_f8f6f4 v[52:55], a[104:111], v[136:143], v[52:55] cbsz:0 blgp:1\n\t" \
  "ds_read_b64_tr_b8 v[154:155], v167 offset:17408\n\t" \
  "v_mfma_f32_16x16x128_f8f6f4 v[84:87], a[112:119], v[136:143], v[84:87] cbsz:0 blgp:1\n\t" \
  "ds_read_b64_tr_b8 v[156:157], v167 offset:24576\n\t" \
  "v_mfma_f32_16x16x128_f8f6f4 v[116:119], a[120:127], v[136:143], v[116:119] cbsz:0 blgp:1\n\t" \
  "ds_read_b64_tr_b8 v[158:159], v167 offset:25600\n\t" \
  "s_waitcnt lgkmcnt(4)\n\t" \
  "v_mfma_f32_16x16x128_f8f6f4 v[24:27], a[96:103], v[144:151], v[24:27] cbsz:0 blgp:1\n\t" \
  "s_barrier\n\t" \
  "ds_read_b64_tr_b8 v[128:129], v160 offset:32768\n\t" \
  "v_mfma_f32_16x16x128_f8f6f4 v[56:59], a[104:111], v[144:151], v[56:59] cbsz:0 blgp:1\n\t" \
  "ds_read_b64_tr_b8 v[130:131], v160 offset:33792\n\t" \
  "v_mfma_f32_16x16x128_f8f6f4 v[88:91], a[112:119], v[144:151], v[88:91] cbsz:0 blgp:1\n\t" \
  "ds_read_b64_tr_b8 v[132:133], v160 offset:40960\n\t" \
  "v_mfma_f32_16x16x128_f8f6f4 v[120:123], a[120:127], v[144:151], v[120:123] cbsz:0 blgp:1\n\t" \
  "ds_read_b64_tr_b8 v[134:135], v160 offset:41984\n\t" \
  "s_waitcnt lgkmcnt(4)\n\t" \
  "v_mfma_f32_16x16x128_f8f6f4 v[28:31], a[96:103], v[152:159], v[28:31] cbsz:0 blgp:1\n\t" \
  "ds_read_b64_tr_b8 v[136:137], v161 offset:32768\n\t" \
  "v_mfma_f32_16x16x128_f8f6f4 v[60:63], a[104:111], v[152:159], v[60:63] cbsz:0 blgp:1\n\t" \
  "ds_read_b64_tr_b8 v[138:139], v161 offset:33792\n\t" \
  "v_mfma_f32_16x16x128_f8f6f4 v[92:95], a[112:119], v[152:159], v[92:95] cbsz:0 blgp:1\n\t" \
  "ds_read_b64_tr_b8 v[140:141], v161 offset:40960\n\t" \
  "v_mfma_f32_16x16x128_f8f6f4 v[124:127], a[120:127], v[152:159], v[124:127] cbsz:0 blgp:1\n\t" \
  "ds_read_b64_tr_b8 v[142:143], v161 offset:41984\n\t" \
  "s_waitcnt lgkmcnt(4)\n\t" \
  "v_mfma_f32_16x16x128_f8f6f4 v[0:3], a[128:135], v[128:135], v[0:3] cbsz:0 blgp:1\n\t" \
  "ds_read_b64_tr_b8 v[144:145], v162 offset:32768\n\t" \
  "v_mfma_f32_16x16x128_f8f6f4 v[32:35], a[136:143], v[128:135], v[32:35] cbsz:0 blgp:1\n\t" \
  "ds_read_b64_tr_b8 v[146:147], v162 offset:33792\n\t" \
  "v_mfma_f32_16x16x128_f8f6f4 v[64:67], a[144:151], v[128:135], v[64:67] cbsz:0 blgp:1\n\t" \
  "ds_read_b64_tr_b8 v[148:149], v162 offset:40960\n\t" \
  "v_mfma_f32_16x16x128_f8f6f4 v[96:99], a[152:159], v[128:135], v[96:99] cbsz:0 blgp:1\n\t" \
  "ds_read_b64_tr_b8 v[150:151], v162 offset:41984\n\t" \
  "s_cmp_eq_u32 s8, -1\n\t" \
  "s_cbranch_scc0 L_au5_0\n\t" \
  "L_aur5_0:\n\t" \
  "s_waitcnt lgkmcnt(4)\n\t" \
  "v_mfma_f32_16x16x128_f8f6f4 v[4:7], a[128:135], v[136:143], v[4:7] cbsz:0 blgp:1\n\t" \
  "ds_read_b64_tr_b8 v[152:153], v163 offset:32768\n\t" \
  "v_mfma_f32_16x16x128_f8f6f4 v[36:39], a[136:143], v[136:143], v[36:39] cbsz:0 blgp:1\n\t" \
  "ds_read_b64_tr_b8 v[154:155], v163 offset:33792\n\t" \
  "v_mfma_f32_16x16x128_f8f6f4 v[68:71], a[144:151], v[136:143], v[68:71] cbsz:0 blgp:1\n\t" \
  "ds_read_b64_tr_b8 v[156:157], v163 offset:40960\n\t" \
  "v_mfma_f32_16x16x128_f8f6f4 v[100:103], a[152:159], v[136:143], v[100:103] cbsz:0 blgp:1\n\t" \
  "ds_read_b64_tr_b8 v[158:159], v163 offset:41984\n\t" \
  "s_waitcnt lgkmcnt(4)\n\t" \
  "v_mfma_f32_16x16x128_f8f6f4 v[8:11], a[128:135], v[144:151], v[8:11] cbsz:0 blgp:1\n\t" \
  "ds_read_b64_tr_b8 v[128:129], v164 offset:32768\n\t" \
  "v_mfma_f32_16x16x128_f8f6f4 v[40:43], a[136:143], v[144:151], v[40:43] cbsz:0 blgp:1\n\t" \
  "ds_read_b64_tr_b8 v[130:131], v164 offset:33792\n\t" \
  "v_mfma_f32_16x16x128_f8f6f4 v[72:75], a[144:151], v[144:151], v[72:75] cbsz:0 blgp:1\n\t" \
  "ds_read_b64_tr_b8 v[132:133], v164 offset:40960\n\t" \
  "v_mfma_f32_16x16x128_f8f6f4 v[104:107], a[152:159], v[144:151], v[104:107] cbsz:0 blgp:1\n\t" \
  "ds_read_b64_tr_b8 v[134:135], v164 offset:41984\n\t" \
  "s_waitcnt lgkmcnt(4)\n\t" \
  "v_mfma_f32_16x16x128_f8f6f4 v[12:15], a[128:135], v[152:159], v[12:15] cbsz:0 blgp:1\n\t" \
  "ds_read_b64_tr_b8 v[136:137], v165 offset:32768\n\t" \
  "v_mfma_f32_16x16x128_f8f6f4 v[44:47], a[136:143], v[152:159], v[44:47] cbsz:0 blgp:1\n\t" \
  "ds_read_b64_tr_b8 v[138:139], v165 offset:33792\n\t" \
  "v_mfma_f32_16x16x128_f8f6f4 v[76:79], a[144:151], v[152:159], v[76:79] cbsz:0 blgp:1\n\t" \
  "ds_read_b64_tr_b8 v[140:141], v165 offset:40960\n\t" \
  "v_mfma_f32_16x16x128_f8f6f4 v[108:111], a[152:159], v[152:159], v[108:111] cbsz:0 blgp:1\n\t" \
  "ds_read_b64_tr_b8 v[142:143], v165 offset:41984\n\t" \
  "s_cmp_eq_u32 s8, -1\n\t" \
  "s_cbranch_scc0 L_ap6_0\n\t" \
  "L_apr6_0:\n\t" \
  "s_waitcnt lgkmcnt(4)\n\t" \
  "v_mfma_f32_16x16x128_f8f6f4 v[16:19], a[128:135], v[128:135], v[16:19] cbsz:0 blgp:1\n\t" \
  "ds_read_b64_tr_b8 v[144:145], v166 offset:32768\n\t" \
  "v_mfma_f32_16x16x128_f8f6f4 v[48:51], a[136:143], v[128:135], v[48:51] cbsz:0 blgp:1\n\t" \
  "ds_read_b64_tr_b8 v[146:147], v166 offset:33792\n\t" \
  "v_mfma_f32_16x16x128_f8f6f4 v[80:83], a[144:151], v[128:135], v[80:83] cbsz:0 blgp:1\n\t" \
  "ds_read_b64_tr_b8 v[148:149], v166 offset:40960\n\t" \
  "buffer_load_dwordx4 a[232:235], v188, s[36:39], s87 offen offset:2048 \n\t" \
  "v_mfma_f32_16x16x128_f8f6f4 v[112:115], a[152:159], v[128:135], v[112:115] cbsz:0 blgp:1\n\t" \
  "ds_read_b64_tr_b8 v[150:151], v166 offset:41984\n\t" \
  "s_waitcnt lgkmcnt(4)\n\t" \
  "v_mfma_f32_16x16x128_f8f6f4 v[20:23], a[128:135], v[136:143], v[20:23] cbsz:0 blgp:1\n\t" \
  "ds_read_b64_tr_b8 v[152:153], v167 offset:32768\n\t" \
  "buffer_load_dwordx4 a[236:239], v188, s[36:39], s87 offen offset:3072 \n\t" \
  "v_mfma_f32_16x16x128_f8f6f4 v[52:55], a[136:143], v[136:143], v[52:55] cbsz:0 blgp:1\n\t" \
  "ds_read_b64_tr_b8 v[154:155], v167 offset:33792\n\t" \
  "v_mfma_f32_16x16x128_f8f6f4 v[84:87], a[144:151], v[136:143], v[84:87] cbsz:0 blgp:1\n\t" \
  "ds_read_b64_tr_b8 v[156:157], v167 offset:40960\n\t" \
  "buffer_load_dwordx4 a[240:243], v189, s[36:39], s87 offen offset:0 \n\t" \
  "v_mfma_f32_16x16x128_f8f6f4 v[116:119], a[152:159], v[136:143], v[116:119] cbsz:0 blgp:1\n\t" \
  "ds_read_b64_tr_b8 v[158:159], v167 offset:41984\n\t" \
  "s_waitcnt lgkmcnt(4)\n\t" \
  "v_mfma_f32_16x16x128_f8f6f4 v[24:27], a[128:135], v[144:151], v[24:27] cbsz:0 blgp:1\n\t" \
  "s_barrier\n\t" \
  "ds_read_b64_tr_b8 v[128:129], v160 offset:49152\n\t" \
  "v_mfma_f32_16x16x128_f8f6f4 v[56:59], a[136:143], v[144:151], v[56:59] cbsz:0 blgp:1\n\t" \
  "ds_read_b64_tr_b8 v[130:131], v160 offset:50176\n\t" \
  "buffer_load_dwordx4 a[244:247], v189, s[36:39], s87 offen offset:1024 \n\t" \
  "v_mfma_f32_16x16x128_f8f6f4 v[88:91], a[144:151], v[144:151], v[88:91] cbsz:0 blgp:1\n\t" \
  "ds_read_b64_tr_b8 v[132:133], v160 offset:57344\n\t" \
  "v_mfma_f32_16x16x128_f8f6f4 v[120:123], a[152:159], v[144:151], v[120:123] cbsz:0 blgp:1\n\t" \
  "ds_read_b64_tr_b8 v[134:135], v160 offset:58368\n\t" \
  "buffer_load_dwordx4 a[248:251], v189, s[36:39], s87 offen offset:2048 \n\t" \
  "s_waitcnt lgkmcnt(4)\n\t" \
  "v_mfma_f32_16x16x128_f8f6f4 v[28:31], a[128:135], v[152:159], v[28:31] cbsz:0 blgp:1\n\t" \
  "ds_read_b64_tr_b8 v[136:137], v161 offset:49152\n\t" \
  "v_mfma_f32_16x16x128_f8f6f4 v[60:63], a[136:143], v[152:159], v[60:63] cbsz:0 blgp:1\n\t" \
  "ds_read_b64_tr_b8 v[138:139], v161 offset:50176\n\t" \
  "buffer_load_dwordx4 a[252:255], v189, s[36:39], s87 offen offset:3072 \n\t" \
  "v_mfma_f32_16x16x128_f8f6f4 v[92:95], a[144:151], v[152:159], v[92:95] cbsz:0 blgp:1\n\t" \
  "ds_read_b64_tr_b8 v[140:141], v161 offset:57344\n\t" \
  "v_mfma_f32_16x16x128_f8f6f4 v[124:127], a[152:159], v[152:159], v[124:127] cbsz:0 blgp:1\n\t" \
  "ds_read_b64_tr_b8 v[142:143], v161 offset:58368\n\t" \
  "s_bfe_u32 s84, s96, 262164\n\t" \
  "s_cmp_eq_u32 s84, 15\n\t" \
  "s_cbranch_scc0 L_q16_0\n\t" \
  "L_qr16_0:\n\t" \
  "s_mov_b32 s87, 786432\n\t" \
  "buffer_load_dwordx4 a[0:3], v188, s[36:39], s87 offen offset:0 \n\t" \
  "s_waitcnt lgkmcnt(4)\n\t" \
  "v_mfma_f32_16x16x128_f8f6f4 v[0:3], a[160:167], v[128:135], v[0:3] cbsz:0 blgp:1\n\t" \
  "ds_read_b64_tr_b8 v[144:145], v162 offset:49152\n\t" \
  "v_mfma_f32_16x16x128_f8f6f4 v[32:35], a[168:175], v[128:135], v[32:35] cbsz:0 blgp:1\n\t" \
  "ds_read_b64_tr_b8 v[146:147], v162 offset:50176\n\t" \
  "s_add_u32 m0, s77, 98304\n\t" \
  "s_add_u32 s90, s73, 2097152\n\t" \
  "buffer_load_dwordx4 v186, s[40:43], s90 offen offset:0  lds\n\t" \
  "s_waitcnt vmcnt(28)\n\t" \
  "v_mfma_f32_16x16x128_f8f6f4 v[64:67], a[176:183], v[128:135], v[64:67] cbsz:0 blgp:1\n\t" \
  "ds_read_b64_tr_b8 v[148:149], v162 offset:57344\n\t" \
  "s_add_u32 m0, s77, 114688\n\t" \
  "s_add_u32 s90, s74, 2097152\n\t" \
  "buffer_load_dwordx4 v186, s[40:43], s90 offen offset:0  lds\n\t" \
  "s_waitcnt vmcnt(27)\n\t" \
  "v_mfma_f32_16x16x128_f8f6f4 v[96:99], a[184:191], v[128:135], v[96:99] cbsz:0 blgp:1\n\t" \
  "ds_read_b64_tr_b8 v[150:151], v162 offset:58368\n\t" \
  "s_cmp_eq_u32 s8, -1\n\t" \
  "s_cbranch_scc0 L_au6_0\n\t" \
  "L_aur6_0:\n\t" \
  "s_add_u32 m0, s77, 98304\n\t" \
  "s_add_u32 s90, s73, 2097152\n\t" \
  "buffer_load_dwordx4 v187, s[40:43], s90 offen offset:1024  lds\n\t" \
  "s_waitcnt lgkmcnt(4)\n\t" \
  "v_mfma_f32_16x16x128_f8f6f4 v[4:7], a[160:167], v[136:143], v[4:7] cbsz:0 blgp:1\n\t" \
  "ds_read_b64_tr_b8 v[152:153], v163 offset:49152\n\t" \
  "s_add_u32 m0, s77, 114688\n\t" \
  "s_add_u32 s90, s74, 2097152\n\t" \
  "buffer_load_dwordx4 v187, s[40:43], s90 offen offset:1024  lds\n\t" \
  "v_mfma_f32_16x16x128_f8f6f4 v[36:39], a[168:175], v[136:143], v[36:39] cbsz:0 blgp:1\n\t" \
  "ds_read_b64_tr_b8 v[154:155], v163 offset:50176\n\t" \
  "s_add_u32 m0, s77, 131072\n\t" \
  "s_add_u32 s90, s68, 4194304\n\t" \
  "buffer_load_dwordx4 v186, s[40:43], s90 offen offset:0  lds\n\t" \
  "v_mfma_f32_16x16x128_f8f6f4 v[68:71], a[176:183], v[136:143], v[68:71] cbsz:0 blgp:1\n\t" \
  "ds_read_b64_tr_b8 v[156:157], v163 offset:57344\n\t" \
  "s_add_u32 m0, s77, 147456\n\t" \
  "s_add_u32 s90, s69, 4194304\n\t" \
  "buffer_load_dwordx4 v186, s[40:43], s90 offen offset:0  lds\n\t" \
  "v_mfma_f32_16x16x128_f8f6f4 v[100:103], a[184:191], v[136:143], v[100:103] cbsz:0 blgp:1\n\t" \
  "ds_read_b64_tr_b8 v[158:159], v163 offset:58368\n\t" \
  "s_add_u32 m0, s77, 131072\n\t" \
  "s_add_u32 s90, s68, 4194304\n\t" \
  "buffer_load_dwordx4 v187, s[40:43], s90 offen offset:1024  lds\n\t" \
  "s_waitcnt lgkmcnt(4)\n\t" \
  "v_mfma_f32_16x16x128_f8f6f4 v[8:11], a[160:167], v[144:151], v[8:11] cbsz:0 blgp:1\n\t" \
  "ds_read_b64_tr_b8 v[128:129], v164 offset:49152\n\t" \
  "s_add_u32 m0, s77, 147456\n\t" \
  "s_add_u32 s90, s69, 4194304\n\t" \
  "buffer_load_dwordx4 v187, s[40:43], s90 offen offset:1024  lds\n\t" \
  "v_mfma_f32_16x16x128_f8f6f4 v[40:43], a[168:175], v[144:151], v[40:43] cbsz:0 blgp:1\n\t" \
  "ds_read_b64_tr_b8 v[130:131], v164 offset:50176\n\t" \
  "buffer_load_dwordx4 a[4:7], v188, s[36:39], s87 offen offset:1024 \n\t" \
  "v_mfma_f32_16x16x128_f8f6f4 v[72:75], a[176:183], v[144:151], v[72:75] cbsz:0 blgp:1\n\t" \
  "ds_read_b64_tr_b8 v[132:133], v164 offset:57344\n\t" \
  "v_mfma_f32_16x16x128_f8f6f4 v[104:107], a[184:191], v[144:151], v[104:107] cbsz:0 blgp:1\n\t" \
  "ds_read_b64_tr_b8 v[134:135], v164 offset:58368\n\t" \
  "buffer_load_dwordx4 a[8:11], v188, s[36:39], s87 offen offset:2048 \n\t" \
  "s_waitcnt lgkmcnt(4)\n\t" \
  "v_mfma_f32_16x16x128_f8f6f4 v[12:15], a[160:167], v[152:159], v[12:15] cbsz:0 blgp:1\n\t" \
  "ds_read_b64_tr_b8 v[136:137], v165 offset:49152\n\t" \
  "v_mfma_f32_16x16x128_f8f6f4 v[44:47], a[168:175], v[152:159], v[44:47] cbsz:0 blgp:1\n\t" \
  "ds_read_b64_tr_b8 v[138:139], v165 offset:50176\n\t" \
  "buffer_load_dwordx4 a[12:15], v188, s[36:39], s87 offen offset:3072 \n\t" \
  "v_mfma_f32_16x16x128_f8f6f4 v[76:79], a[176:183], v[152:159], v[76:79] cbsz:0 blgp:1\n\t" \
  "ds_read_b64_tr_b8 v[140:141], v165 offset:57344\n\t" \
  "v_mfma_f32_16x16x128_f8f6f4 v[108:111], a[184:191], v[152:159], v[108:111] cbsz:0 blgp:1\n\t" \
  "ds_read_b64_tr_b8 v[142:143], v165 offset:58368\n\t" \
  "s_cmp_eq_u32 s8, -1\n\t" \
  "s_cbranch_scc0 L_ap7_0\n\t" \
  "L_apr7_0:\n\t" \
  "buffer_load_dwordx4 a[16:19], v189, s[36:39], s87 offen offset:0 \n\t" \
  "s_waitcnt lgkmcnt(4)\n\t" \
  "v_mfma_f32_16x16x128_f8f6f4 v[16:19], a[160:167], v[128:135], v[16:19] cbsz:0 blgp:1\n\t" \
  "ds_read_b64_tr_b8 v[144:145], v166 offset:49152\n\t" \
  "v_mfma_f32_16x16x128_f8f6f4 v[48:51], a[168:175], v[128:135], v[48:51] cbsz:0 blgp:1\n\t" \
  "ds_read_b64_tr_b8 v[146:147], v166 offset:50176\n\t" \
  "buffer_load_dwordx4 a[20:23], v189, s[36:39], s87 offen offset:1024 \n\t" \
  "v_mfma_f32_16x16x128_f8f6f4 v[80:83], a[176:183], v[128:135], v[80:83] cbsz:0 blgp:1\n\t" \
  "ds_read_b64_tr_b8 v[148:149], v166 offset:57344\n\t" \
  "v_mfma_f32_16x16x128_f8f6f4 v[112:115], a[184:191], v[128:135], v[112:115] cbsz:0 blgp:1\n\t" \
  "ds_read_b64_tr_b8 v[150:151], v166 offset:58368\n\t" \
  "buffer_load_dwordx4 a[24:27], v189, s[36:39], s87 offen offset:2048 \n\t" \
  "s_waitcnt lgkmcnt(4)\n\t" \
  "v_mfma_f32_16x16x128_f8f6f4 v[20:23], a[160:167], v[136:143], v[20:23] cbsz:0 blgp:1\n\t" \
  "ds_read_b64_tr_b8 v[152:153], v167 offset:49152\n\t" \
  "v_mfma_f32_16x16x128_f8f6f4 v[52:55], a[168:175], v[136:143], v[52:55] cbsz:0 blgp:1\n\t" \
  "ds_read_b64_tr_b8 v[154:155], v167 offset:50176\n\t" \
  "buffer_load_dwordx4 a[28:31], v189, s[36:39], s87 offen offset:3072 \n\t" \
  "v_mfma_f32_16x16x128_f8f6f4 v[84:87], a[176:183], v[136:143], v[84:87] cbsz:0 blgp:1\n\t" \
  "ds_read_b64_tr_b8 v[156:157], v167 offset:57344\n\t" \
  "v_mfma_f32_16x16x128_f8f6f4 v[116:119], a[184:191], v[136:143], v[116:119] cbsz:0 blgp:1\n\t" \
  "ds_read_b64_tr_b8 v[158:159], v167 offset:58368\n\t" \
  "s_bfe_u32 s84, s96, 262168\n\t" \
  "s_cmp_eq_u32 s84, 15\n\t" \
  "s_cbranch_scc0 L_q17_0\n\t" \
  "L_qr17_0:\n\t" \
  "s_mov_b32 s87, 851968\n\t" \
  "buffer_load_dwordx4 a[32:35], v188, s[36:39], s87 offen offset:0 \n\t" \
  "s_waitcnt lgkmcnt(4)\n\t" \
  "v_mfma_f32_16x16x128_f8f6f4 v[24:27], a[160:167], v[144:151], v[24:27] cbsz:0 blgp:1\n\t" \
  "s_barrier\n\t" \
  "ds_read_b64_tr_b8 v[128:129], v168 offset:0\n\t" \
  "v_mfma_f32_16x16x128_f8f6f4 v[56:59], a[168:175], v[144:151], v[56:59] cbsz:0 blgp:1\n\t" \
  "ds_read_b64_tr_b8 v[130:131], v168 offset:1024\n\t" \
  "buffer_load_dwordx4 a[36:39], v188, s[36:39], s87 offen offset:1024 \n\t" \
  "v_mfma_f32_16x16x128_f8f6f4 v[88:91], a[176:183], v[144:151], v[88:91] cbsz:0 blgp:1\n\t" \
  "ds_read_b64_tr_b8 v[132:133], v168 offset:8192\n\t" \
  "v_mfma_f32_16x16x128_f8f6f4 v[120:123], a[184:191], v[144:151], v[120:123] cbsz:0 blgp:1\n\t" \
  "ds_read_b64_tr_b8 v[134:135], v168 offset:9216\n\t" \
  "buffer_load_dwordx4 a[40:43], v188, s[36:39], s87 offen offset:2048 \n\t" \
  "s_waitcnt lgkmcnt(4)\n\t" \
  "v_mfma_f32_16x16x128_f8f6f4 v[28:31], a[160:167], v[152:159], v[28:31] cbsz:0 blgp:1\n\t" \
  "ds_read_b64_tr_b8 v[136:137], v169 offset:0\n\t" \
  "v_mfma_f32_16x16x128_f8f6f4 v[60:63], a[168:175], v[152:159], v[60:63] cbsz:0 blgp:1\n\t" \
  "ds_read_b64_tr_b8 v[138:139], v169 offset:1024\n\t" \
  "buffer_load_dwordx4 a[44:47], v188, s[36:39], s87 offen offset:3072 \n\t" \
  "v_mfma_f32_16x16x128_f8f6f4 v[92:95], a[176:183], v[152:159], v[92:95] cbsz:0 blgp:1\n\t" \
  "ds_read_b64_tr_b8 v[140:141], v169 offset:8192\n\t" \
  "v_mfma_f32_16x16x128_f8f6f4 v[124:127], a[184:191], v[152:159], v[124:127] cbsz:0 blgp:1\n\t" \
  "ds_read_b64_tr_b8 v[142:143], v169 offset:9216\n\t" \
  "buffer_load_dwordx4 a[48:51], v189, s[36:39], s87 offen offset:0 \n\t" \
  "s_waitcnt vmcnt(43) lgkmcnt(4)\n\t" \
  "v_mfma_f32_16x16x128_f8f6f4 v[0:3], a[192:199], v[128:135], v[0:3] cbsz:0 blgp:1\n\t" \
  "ds_read_b64_tr_b8 v[144:145], v170 offset:0\n\t" \
  "s_waitcnt vmcnt(41)\n\t" \
  "v_mfma_f32_16x16x128_f8f6f4 v[32:35], a[200:207], v[128:135], v[32:35] cbsz:0 blgp:1\n\t" \
  "ds_read_b64_tr_b8 v[146:147], v170 offset:1024\n\t" \
  "buffer_load_dwordx4 a[52:55], v189, s[36:39], s87 offen offset:1024 \n\t" \
  "s_waitcnt vmcnt(40)\n\t" \
  "v_mfma_f32_16x16x128_f8f6f4 v[64:67], a[208:215], v[128:135], v[64:67] cbsz:0 blgp:1\n\t" \
  "ds_read_b64_tr_b8 v[148:149], v170 offset:8192\n\t" \
  "s_waitcnt vmcnt(38)\n\t" \
  "v_mfma_f32_16x16x128_f8f6f4 v[96:99], a[216:223], v[128:135], v[96:99] cbsz:0 blgp:1\n\t" \
  "ds_read_b64_tr_b8 v[150:151], v170 offset:9216\n\t" \
  "s_cmp_eq_u32 s8, -1\n\t" \
  "s_cbranch_scc0 L_au7_0\n\t" \
  "L_aur7_0:\n\t" \
  "buffer_load_dwordx4 a[56:59], v189, s[36:39], s87 offen offset:2048 \n\t" \
  "s_waitcnt lgkmcnt(4)\n\t" \
  "v_mfma_f32_16x16x128_f8f6f4 v[4:7], a[192:199], v[136:143], v[4:7] cbsz:0 blgp:1\n\t" \
  "ds_read_b64_tr_b8 v[152:153], v171 offset:0\n\t" \
  "v_mfma_f32_16x16x128_f8f6f4 v[36:39], a[200:207], v[136:143], v[36:39] cbsz:0 blgp:1\n\t" \
  "ds_read_b64_tr_b8 v[154:155], v171 offset:1024\n\t" \
  "buffer_load_dwordx4 a[60:63], v189, s[36:39], s87 offen offset:3072 \n\t" \
  "v_mfma_f32_16x16x128_f8f6f4 v[68:71], a[208:215], v[136:143], v[68:71] cbsz:0 blgp:1\n\t" \
  "ds_read_b64_tr_b8 v[156:157], v171 offset:8192\n\t" \
  "v_mfma_f32_16x16x128_f8f6f4 v[100:103], a[216:223], v[136:143], v[100:103] cbsz:0 blgp:1\n\t" \
  "ds_read_b64_tr_b8 v[158:159], v171 offset:9216\n\t" \
  "s_bfe_u32 s84, s8, 262144\n\t" \
  "s_cmp_eq_u32 s84, 15\n\t" \
  "s_cbranch_scc0 L_q18_0\n\t" \
  "L_qr18_0:\n\t" \
  "s_mov_b32 s87, 917504\n\t" \
  "buffer_load_dwordx4 a[64:67], v188, s[36:39], s87 offen offset:0 \n\t" \
  "s_waitcnt lgkmcnt(4)\n\t" \
  "v_mfma_f32_16x16x128_f8f6f4 v[8:11], a[192:199], v[144:151], v[8:11] cbsz:0 blgp:1\n\t" \
  "ds_read_b64_tr_b8 v[128:129], v172 offset:0\n\t" \
  "v_mfma_f32_16x16x128_f8f6f4 v[40:43], a[200:207], v[144:151], v[40:43] cbsz:0 blgp:1\n\t" \
  "ds_read_b64_tr_b8 v[130:131], v172 offset:1024\n\t" \
  "buffer_load_dwordx4 a[68:71], v188, s[36:39], s87 offen offset:1024 \n\t" \
  "v_mfma_f32_16x16x128_f8f6f4 v[72:75], a[208:215], v[144:151], v[72:75] cbsz:0 blgp:1\n\t" \
  "ds_read_b64_tr_b8 v[132:133], v172 offset:8192\n\t" \
  "v_mfma_f32_16x16x128_f8f6f4 v[104:107], a[216:223], v[144:151], v[104:107] cbsz:0 blgp:1\n\t" \
  "ds_read_b64_tr_b8 v[134:135], v172 offset:9216\n\t" \
  "buffer_load_dwordx4 a[72:75], v188, s[36:39], s87 offen offset:2048 \n\t" \
  "s_waitcnt lgkmcnt(4)\n\t" \
  "v_mfma_f32_16x16x128_f8f6f4 v[12:15], a[192:199], v[152:159], v[12:15] cbsz:0 blgp:1\n\t" \
  "ds_read_b64_tr_b8 v[136:137], v173 offset:0\n\t" \
  "v_mfma_f32_16x16x128_f8f6f4 v[44:47], a[200:207], v[152:159], v[44:47] cbsz:0 blgp:1\n\t" \
  "ds_read_b64_tr_b8 v[138:139], v173 offset:1024\n\t" \
  "buffer_load_dwordx4 a[76:79], v188, s[36:39], s87 offen offset:3072 \n\t" \
  "v_mfma_f32_16x16x128_f8f6f4 v[76:79], a[208:215], v[152:159], v[76:79] cbsz:0 blgp:1\n\t" \
  "ds_read_b64_tr_b8 v[140:141], v173 offset:8192\n\t" \
  "v_mfma_f32_16x16x128_f8f6f4 v[108:111], a[216:223], v[152:159], v[108:111] cbsz:0 blgp:1\n\t" \
  "ds_read_b64_tr_b8 v[142:143], v173 offset:9216\n\t" \
  "s_cmp_eq_u32 s8, -1\n\t" \
  "s_cbranch_scc0 L_ap8_0\n\t" \
  "L_apr8_0:\n\t" \
  "buffer_load_dwordx4 a[80:83], v189, s[36:39], s87 offen offset:0 \n\t" \
  "s_waitcnt lgkmcnt(4)\n\t" \
  "v_mfma_f32_16x16x128_f8f6f4 v[16:19], a[192:199], v[128:135], v[16:19] cbsz:0 blgp:1\n\t" \
  "ds_read_b64_tr_b8 v[144:145], v174 offset:0\n\t" \
  "v_mfma_f32_16x16x128_f8f6f4 v[48:51], a[200:207], v[128:135], v[48:51] cbsz:0 blgp:1\n\t" \
  "ds_read_b64_tr_b8 v[146:147], v174 offset:1024\n\t" \
  "buffer_load_dwordx4 a[84:87], v189, s[36:39], s87 offen offset:1024 \n\t" \
  "v_mfma_f32_16x16x128_f8f6f4 v[80:83], a[208:215], v[128:135], v[80:83] cbsz:0 blgp:1\n\t" \
  "ds_read_b64_tr_b8 v[148:149], v174 offset:8192\n\t" \
  "v_mfma_f32_16x16x128_f8f6f4 v[112:115], a[216:223], v[128:135], v[112:115] cbsz:0 blgp:1\n\t" \
  "ds_read_b64_tr_b8 v[150:151], v174 offset:9216\n\t" \
  "buffer_load_dwordx4 a[88:91], v189, s[36:39], s87 offen offset:2048 \n\t" \
  "s_waitcnt lgkmcnt(4)\n\t" \
  "v_mfma_f32_16x16x128_f8f6f4 v[20:23], a[192:199], v[136:143], v[20:23] cbsz:0 blgp:1\n\t" \
  "ds_read_b64_tr_b8 v[152:153], v175 offset:0\n\t" \
  "v_mfma_f32_16x16x128_f8f6f4 v[52:55], a[200:207], v[136:143], v[52:55] cbsz:0 blgp:1\n\t" \
  "ds_read_b64_tr_b8 v[154:155], v175 offset:1024\n\t" \
  "buffer_load_dwordx4 a[92:95], v189, s[36:39], s87 offen offset:3072 \n\t" \
  "v_mfma_f32_16x16x128_f8f6f4 v[84:87], a[208:215], v[136:143], v[84:87] cbsz:0 blgp:1\n\t" \
  "ds_read_b64_tr_b8 v[156:157], v175 offset:8192\n\t" \
  "s_add_u32 m0, s76, 0\n\t" \
  "s_add_u32 s90, s70, 4194304\n\t" \
  "buffer_load_dwordx4 v184, s[40:43], s90 offen offset:0  lds\n\t" \
  "v_mfma_f32_16x16x128_f8f6f4 v[116:119], a[216:223], v[136:143], v[116:119] cbsz:0 blgp:1\n\t" \
  "ds_read_b64_tr_b8 v[158:159], v175 offset:9216\n\t" \
  "s_add_u32 m0, s76, 0\n\t" \
  "s_add_u32 s90, s70, 4194304\n\t" \
  "buffer_load_dwordx4 v185, s[40:43], s90 offen offset:1024  lds\n\t" \
  "s_waitcnt lgkmcnt(4)\n\t" \
  "v_mfma_f32_16x16x128_f8f6f4 v[24:27], a[192:199], v[144:151], v[24:27] cbsz:0 blgp:1\n\t" \
  "s_barrier\n\t" \
  "ds_read_b64_tr_b8 v[128:129], v168 offset:16384\n\t" \
  "s_cmp_gt_u32 s29, 1\n\t" \
  "s_cbranch_scc1 L_k472_0\n\t" \
  "s_getpc_b64 s[88:89]\n\t" \
  "s_lshl_b32 s84, s29, 13\n\t" \
  "s_add_u32 s84, s84, 2048\n\t" \
  "s_add_u32 s88, s88, s84\n\t" \
  "s_addc_u32 s89, s89, 0\n\t" \
  "s_mov_b64 exec, 0xff\n\t" \
  "global_load_dword v210, v234, s[88:89]\n\t" \
  "s_mov_b64 exec, -1\n\t" \
  "L_k472_0:\n\t" \
  "v_mfma_f32_16x16x128_f8f6f4 v[56:59], a[200:207], v[144:151], v[56:59] cbsz:0 blgp:1\n\t" \
  "ds_read_b64_tr_b8 v[130:131], v168 offset:17408\n\t" \
  "s_add_u32 m0, s76, 16384\n\t" \
  "s_add_u32 s90, s71, 4194304\n\t" \
  "buffer_load_dwordx4 v184, s[40:43], s90 offen offset:0  lds\n\t" \
  "v_mfma_f32_16x16x128_f8f6f4 v[88:91], a[208:215], v[144:151], v[88:91] cbsz:0 blgp:1\n\t" \
  "ds_read_b64_tr_b8 v[132:133], v168 offset:24576\n\t" \
  "s_add_u32 m0, s76, 16384\n\t" \
  "s_add_u32 s90, s71, 4194304\n\t" \
  "buffer_load_dwordx4 v185, s[40:43], s90 offen offset:1024  lds\n\t" \
  "v_mfma_f32_16x16x128_f8f6f4 v[120:123], a[216:223], v[144:151], v[120:123] cbsz:0 blgp:1\n\t" \
  "ds_read_b64_tr_b8 v[134:135], v168 offset:25600\n\t" \
  "s_add_u32 m0, s76, 32768\n\t" \
  "s_add_u32 s90, s72, 4194304\n\t" \
  "buffer_load_dwordx4 v184, s[40:43], s90 offen offset:0  lds\n\t" \
  "s_waitcnt lgkmcnt(4)\n\t" \
  "v_mfma_f32_16x16x128_f8f6f4 v[28:31], a[192:199], v[152:159], v[28:31] cbsz:0 blgp:1\n\t" \
  "ds_read_b64_tr_b8 v[136:137], v169 offset:16384\n\t" \
  "s_add_u32 m0, s76, 32768\n\t" \
  "s_add_u32 s90, s72, 4194304\n\t" \
  "buffer_load_dwordx4 v185, s[40:43], s90 offen offset:1024  lds\n\t" \
  "v_mfma_f32_16x16x128_f8f6f4 v[60:63], a[200:207], v[152:159], v[60:63] cbsz:0 blgp:1\n\t" \
  "ds_read_b64_tr_b8 v[138:139], v169 offset:17408\n\t" \
  "s_add_u32 m0, s76, 49152\n\t" \
  "s_add_u32 s90, s73, 4194304\n\t" \
  "buffer_load_dwordx4 v184, s[40:43], s90 offen offset:0  lds\n\t" \
  "v_mfma_f32_16x16x128_f8f6f4 v[92:95], a[208:215], v[152:159], v[92:95] cbsz:0 blgp:1\n\t" \
  "ds_read_b64_tr_b8 v[140:141], v169 offset:24576\n\t" \
  "s_add_u32 m0, s76, 49152\n\t" \
  "s_add_u32 s90, s73, 4194304\n\t" \
  "buffer_load_dwordx4 v185, s[40:43], s90 offen offset:1024  lds\n\t" \
  "v_mfma_f32_16x16x128_f8f6f4 v[124:127], a[216:223], v[152:159], v[124:127] cbsz:0 blgp:1\n\t" \
  "ds_read_b64_tr_b8 v[142:143], v169 offset:25600\n\t" \
  "s_add_u32 s99, s29, 28\n\t" \
  "s_min_u32 s99, s99, 31\n\t" \
  "v_readlane_b32 s99, v211, s99\n\t" \
  "s_mov_b32 exec_lo, s15\n\t" \
  "s_mov_b32 exec_hi, 0\n\t" \
  "s_cbranch_execz L_c28_0\n\t" \
  "s_nop 1\n\t" \
  "buffer_load_dword v210, v222, s[60:63], s99 offen\n\t" \
  "L_c28_0:\n\t" \
  "s_mov_b64 exec, -1\n\t" \
  "s_waitcnt vmcnt(46) lgkmcnt(4)\n\t" \
  "v_mfma_f32_16x16x128_f8f6f4 v[0:3], a[224:231], v[128:135], v[0:3] cbsz:0 blgp:1\n\t" \
  "ds_read_b64_tr_b8 v[144:145], v170 offset:16384\n\t" \
  "s_bfe_u32 s84, s8, 262148\n\t" \
  "s_cmp_eq_u32 s84, 15\n\t" \
  "s_cbranch_scc0 L_q19_0\n\t" \
  "L_qr19_0:\n\t" \
  "s_mov_b32 s87, 983040\n\t" \
  "buffer_load_dwordx4 a[96:99], v188, s[36:39], s87 offen offset:0 \n\t" \
  "s_waitcnt vmcnt(45)\n\t" \
  "v_mfma_f32_16x16x128_f8f6f4 v[32:35], a[232:239], v[128:135], v[32:35] cbsz:0 blgp:1\n\t" \
  "ds_read_b64_tr_b8 v[146:147], v170 offset:17408\n\t" \
  "s_waitcnt vmcnt(43)\n\t" \
  "v_mfma_f32_16x16x128_f8f6f4 v[64:67], a[240:247], v[128:135], v[64:67] cbsz:0 blgp:1\n\t" \
  "ds_read_b64_tr_b8 v[148:149], v170 offset:24576\n\t" \
  "s_waitcnt vmcnt(41)\n\t" \
  "v_mfma_f32_16x16x128_f8f6f4 v[96:99], a[248:255], v[128:135], v[96:99] cbsz:0 blgp:1\n\t" \
  "ds_read_b64_tr_b8 v[150:151], v170 offset:25600\n\t" \
  "s_cmp_eq_u32 s8, -1\n\t" \
  "s_cbranch_scc0 L_au8_0\n\t" \
  "L_aur8_0:\n\t" \
  "s_waitcnt lgkmcnt(4)\n\t" \
  "v_mfma_f32_16x16x128_f8f6f4 v[4:7], a[224:231], v[136:143], v[4:7] cbsz:0 blgp:1\n\t" \
  "ds_read_b64_tr_b8 v[152:153], v171 offset:16384\n\t" \
  "v_mfma_f32_16x16x128_f8f6f4 v[36:39], a[232:239], v[136:143], v[36:39] cbsz:0 blgp:1\n\t" \
  "ds_read_b64_tr_b8 v[154:155], v171 offset:17408\n\t" \
  "v_mfma_f32_16x16x128_f8f6f4 v[68:71], a[240:247], v[136:143], v[68:71] cbsz:0 blgp:1\n\t" \
  "ds_read_b64_tr_b8 v[156:157], v171 offset:24576\n\t" \
  "v_mfma_f32_16x16x128_f8f6f4 v[100:103], a[248:255], v[136:143], v[100:103] cbsz:0 blgp:1\n\t" \
  "ds_read_b64_tr_b8 v[158:159], v171 offset:25600\n\t" \
  "s_waitcnt lgkmcnt(4)\n\t" \
  "v_mfma_f32_16x16x128_f8f6f4 v[8:11], a[224:231], v[144:151], v[8:11] cbsz:0 blgp:1\n\t" \
  "ds_read_b64_tr_b8 v[128:129], v172 offset:16384\n\t" \
  "v_mfma_f32_16x16x128_f8f6f4 v[40:43], a[232:239], v[144:151], v[40:43] cbsz:0 blgp:1\n\t" \
  "ds_read_b64_tr_b8 v[130:131], v172 offset:17408\n\t" \
  "v_mfma_f32_16x16x128_f8f6f4 v[72:75], a[240:247], v[144:151], v[72:75] cbsz:0 blgp:1\n\t" \
  "ds_read_b64_tr_b8 v[132:133], v172 offset:24576\n\t" \
  "v_mfma_f32_16x16x128_f8f6f4 v[104:107], a[248:255], v[144:151], v[104:107] cbsz:0 blgp:1\n\t" \
  "ds_read_b64_tr_b8 v[134:135], v172 offset:25600\n\t" \
  "s_waitcnt lgkmcnt(4)\n\t" \
  "v_mfma_f32_16x16x128_f8f6f4 v[12:15], a[224:231], v[152:159], v[12:15] cbsz:0 blgp:1\n\t" \
  "ds_read_b64_tr_b8 v[136:137], v173 offset:16384\n\t" \
  "v_mfma_f32_16x16x128_f8f6f4 v[44:47], a[232:239], v[152:159], v[44:47] cbsz:0 blgp:1\n\t" \
  "ds_read_b64_tr_b8 v[138:139], v173 offset:17408\n\t" \
  "v_mfma_f32_16x16x128_f8f6f4 v[76:79], a[240:247], v[152:159], v[76:79] cbsz:0 blgp:1\n\t" \
  "ds_read_b64_tr_b8 v[140:141], v173 offset:24576\n\t" \
  "v_mfma_f32_16x16x128_f8f6f4 v[108:111], a[248:255], v[152:159], v[108:111] cbsz:0 blgp:1\n\t" \
  "ds_read_b64_tr_b8 v[142:143], v173 offset:25600\n\t" \
  "s_cmp_eq_u32 s8, -1\n\t" \
  "s_cbranch_scc0 L_ap9_0\n\t" \
  "L_apr9_0:\n\t" \
  "s_waitcnt lgkmcnt(4)\n\t" \
  "v_mfma_f32_16x16x128_f8f6f4 v[16:19], a[224:231], v[128:135], v[16:19] cbsz:0 blgp:1\n\t" \
  "ds_read_b64_tr_b8 v[144:145], v174 offset:16384\n\t" \
  "v_mfma_f32_16x16x128_f8f6f4 v[48:51], a[232:239], v[128:135], v[48:51] cbsz:0 blgp:1\n\t" \
  "ds_read_b64_tr_b8 v[146:147], v174 offset:17408\n\t" \
  "v_mfma_f32_16x16x128_f8f6f4 v[80:83], a[240:247], v[128:135], v[80:83] cbsz:0 blgp:1\n\t" \
  "ds_read_b64_tr_b8 v[148:149], v174 offset:24576\n\t" \
  "v_mfma_f32_16x16x128_f8f6f4 v[112:115], a[248:255], v[128:135], v[112:115] cbsz:0 blgp:1\n\t" \
  "ds_read_b64_tr_b8 v[150:151], v174 offset:25600\n\t" \
  "s_waitcnt lgkmcnt(4)\n\t" \
  "v_mfma_f32_16x16x128_f8f6f4 v[20:23], a[224:231], v[136:143], v[20:23] cbsz:0 blgp:1\n\t" \
  "ds_read_b64_tr_b8 v[152:153], v175 offset:16384\n\t" \
  "v_mfma_f32_16x16x128_f8f6f4 v[52:55], a[232:239], v[136:143], v[52:55] cbsz:0 blgp:1\n\t" \
  "ds_read_b64_tr_b8 v[154:155], v175 offset:17408\n\t" \
  "v_mfma_f32_16x16x128_f8f6f4 v[84:87], a[240:247], v[136:143], v[84:87] cbsz:0 blgp:1\n\t" \
  "ds_read_b64_tr_b8 v[156:157], v175 offset:24576\n\t" \
  "v_mfma_f32_16x16x128_f8f6f4 v[116:119], a[248:255], v[136:143], v[116:119] cbsz:0 blgp:1\n\t" \
  "ds_read_b64_tr_b8 v[158:159], v175 offset:25600\n\t" \
  "s_waitcnt lgkmcnt(4)\n\t" \
  "v_mfma_f32_16x16x128_f8f6f4 v[24:27], a[224:231], v[144:151], v[24:27] cbsz:0 blgp:1\n\t" \
  "s_waitcnt vmcnt(37)\n\t" \
  "s_barrier\n\t" \
  "ds_read_b64_tr_b8 v[128:129], v168 offset:32768\n\t" \
  "v_mfma_f32_16x16x128_f8f6f4 v[56:59], a[232:239], v[144:151], v[56:59] cbsz:0 blgp:1\n\t" \
  "ds_read_b64_tr_b8 v[130:131], v168 offset:33792\n\t" \
  "v_mfma_f32_16x16x128_f8f6f4 v[88:91], a[240:247], v[144:151], v[88:91] cbsz:0 blgp:1\n\t" \
  "ds_read_b64_tr_b8 v[132:133], v168 offset:40960\n\t" \
  "v_mfma_f32_16x16x128_f8f6f4 v[120:123], a[248:255], v[144:151], v[120:123] cbsz:0 blgp:1\n\t" \
  "ds_read_b64_tr_b8 v[134:135], v168 offset:41984\n\t" \
  "s_waitcnt lgkmcnt(4)\n\t" \
  "v_mfma_f32_16x16x128_f8f6f4 v[28:31], a[224:231], v[152:159], v[28:31] cbsz:0 blgp:1\n\t" \
  "ds_read_b64_tr_b8 v[136:137], v169 offset:32768\n\t" \
  "v_mfma_f32_16x16x128_f8f6f4 v[60:63], a[232:239], v[152:159], v[60:63] cbsz:0 blgp:1\n\t" \
  "ds_read_b64_tr_b8 v[138:139], v169 offset:33792\n\t" \
  "v_mfma_f32_16x16x128_f8f6f4 v[92:95], a[240:247], v[152:159], v[92:95] cbsz:0 blgp:1\n\t" \
  "ds_read_b64_tr_b8 v[140:141], v169 offset:40960\n\t" \
  "v_mfma_f32_16x16x128_f8f6f4 v[124:127], a[248:255], v[152:159], v[124:127] cbsz:0 blgp:1\n\t" \
  "ds_read_b64_tr_b8 v[142:143], v169 offset:41984\n\t" \
  "buffer_load_dwordx4 a[100:103], v188, s[36:39], s87 offen offset:1024 \n\t" \
  "s_waitcnt vmcnt(32) lgkmcnt(4)\n\t" \
  "v_mfma_f32_16x16x128_f8f6f4 v[0:3], a[0:7], v[128:135], v[0:3] cbsz:0 blgp:1\n\t" \
  "ds_read_b64_tr_b8 v[144:145], v170 offset:32768\n\t" \
  "s_waitcnt vmcnt(30)\n\t" \
  "v_mfma_f32_16x16x128_f8f6f4 v[32:35], a[8:15], v[128:135], v[32:35] cbsz:0 blgp:1\n\t" \
  "ds_read_b64_tr_b8 v[146:147], v170 offset:33792\n\t" \
  "buffer_load_dwordx4 a[104:107], v188, s[36:39], s87 offen offset:2048 \n\t" \
  "s_waitcnt vmcnt(29)\n\t" \
  "v_mfma_f32_16x16x128_f8f6f4 v[64:67], a[16:23], v[128:135], v[64:67] cbsz:0 blgp:1\n\t" \
  "ds_read_b64_tr_b8 v[148:149], v170 offset:40960\n\t" \
  "s_waitcnt vmcnt(27)\n\t" \
  "v_mfma_f32_16x16x128_f8f6f4 v[96:99], a[24:31], v[128:135], v[96:99] cbsz:0 blgp:1\n\t" \
  "ds_read_b64_tr_b8 v[150:151], v170 offset:41984\n\t" \
  "s_cmp_eq_u32 s8, -1\n\t" \
  "s_cbranch_scc0 L_au9_0\n\t" \
  "L_aur9_0:\n\t" \
  "buffer_load_dwordx4 a[108:111], v188, s[36:39], s87 offen offset:3072 \n\t" \
  "s_waitcnt lgkmcnt(4)\n\t" \
  "v_mfma_f32_16x16x128_f8f6f4 v[4:7], a[0:7], v[136:143], v[4:7] cbsz:0 blgp:1\n\t" \
  "ds_read_b64_tr_b8 v[152:153], v171 offset:32768\n\t" \
  "v_mfma_f32_16x16x128_f8f6f4 v[36:39], a[8:15], v[136:143], v[36:39] cbsz:0 blgp:1\n\t" \
  "ds_read_b64_tr_b8 v[154:155], v171 offset:33792\n\t" \
  "buffer_load_dwordx4 a[112:115], v189, s[36:39], s87 offen offset:0 \n\t" \
  "v_mfma_f32_16x16x128_f8f6f4 v[68:71], a[16:23], v[136:143], v[68:71] cbsz:0 blgp:1\n\t" \
  "ds_read_b64_tr_b8 v[156:157], v171 offset:40960\n\t" \
  "v_mfma_f32_16x16x128_f8f6f4 v[100:103], a[24:31], v[136:143], v[100:103] cbsz:0 blgp:1\n\t" \
  "ds_read_b64_tr_b8 v[158:159], v171 offset:41984\n\t" \
  "buffer_load_dwordx4 a[116:119], v189, s[36:39], s87 offen offset:1024 \n\t" \
  "s_waitcnt lgkmcnt(4)\n\t" \
  "v_mfma_f32_16x16x128_f8f6f4 v[8:11], a[0:7], v[144:151], v[8:11] cbsz:0 blgp:1\n\t" \
  "ds_read_b64_tr_b8 v[128:129], v172 offset:32768\n\t" \
  "v_mfma_f32_16x16x128_f8f6f4 v[40:43], a[8:15], v[144:151], v[40:43] cbsz:0 blgp:1\n\t" \
  "ds_read_b64_tr_b8 v[130:131], v172 offset:33792\n\t" \
  "buffer_load_dwordx4 a[120:123], v189, s[36:39], s87 offen offset:2048 \n\t" \
  "v_mfma_f32_16x16x128_f8f6f4 v[72:75], a[16:23], v[144:151], v[72:75] cbsz:0 blgp:1\n\t" \
  "ds_read_b64_tr_b8 v[132:133], v172 offset:40960\n\t" \
  "v_mfma_f32_16x16x128_f8f6f4 v[104:107], a[24:31], v[144:151], v[104:107] cbsz:0 blgp:1\n\t" \
  "ds_read_b64_tr_b8 v[134:135], v172 offset:41984\n\t" \
  "buffer_load_dwordx4 a[124:127], v189, s[36:39], s87 offen offset:3072 \n\t" \
  "s_waitcnt lgkmcnt(4)\n\t" \
  "v_mfma_f32_16x16x128_f8f6f4 v[12:15], a[0:7], v[152:159], v[12:15] cbsz:0 blgp:1\n\t" \
  "ds_read_b64_tr_b8 v[136:137], v173 offset:32768\n\t" \
  "v_mfma_f32_16x16x128_f8f6f4 v[44:47], a[8:15], v[152:159], v[44:47] cbsz:0 blgp:1\n\t" \
  "ds_read_b64_tr_b8 v[138:139], v173 offset:33792\n\t" \
  "s_bfe_u32 s84, s8, 262152\n\t" \
  "s_cmp_eq_u32 s84, 15\n\t" \
  "s_cbranch_scc0 L_q20_0\n\t" \
  "L_qr20_0:\n\t" \
  "s_mov_b32 s87, 1048576\n\t" \
  "buffer_load_dwordx4 a[128:131], v188, s[36:39], s87 offen offset:0 \n\t" \
  "v_mfma_f32_16x16x128_f8f6f4 v[76:79], a[16:23], v[152:159], v[76:79] cbsz:0 blgp:1\n\t" \
  "ds_read_b64_tr_b8 v[140:141], v173 offset:40960\n\t" \
  "s_add_u32 m0, s77, 0\n\t" \
  "s_add_u32 s90, s70, 4194304\n\t" \
  "buffer_load_dwordx4 v186, s[40:43], s90 offen offset:0  lds\n\t" \
  "v_mfma_f32_16x16x128_f8f6f4 v[108:111], a[24:31], v[152:159], v[108:111] cbsz:0 blgp:1\n\t" \
  "ds_read_b64_tr_b8 v[142:143], v173 offset:41984\n\t" \
  "s_cmp_eq_u32 s8, -1\n\t" \
  "s_cbranch_scc0 L_ap10_0\n\t" \
  "L_apr10_0:\n\t" \
  "s_add_u32 m0, s77, 16384\n\t" \
  "s_add_u32 s90, s71, 4194304\n\t" \
  "buffer_load_dwordx4 v186, s[40:43], s90 offen offset:0  lds\n\t" \
  "s_waitcnt lgkmcnt(4)\n\t" \
  "v_mfma_f32_16x16x128_f8f6f4 v[16:19], a[0:7], v[128:135], v[16:19] cbsz:0 blgp:1\n\t" \
  "ds_read_b64_tr_b8 v[144:145], v174 offset:32768\n\t" \
  "s_add_u32 m0, s77, 0\n\t" \
  "s_add_u32 s90, s70, 4194304\n\t" \
  "buffer_load_dwordx4 v187, s[40:43], s90 offen offset:1024  lds\n\t" \
  "v_mfma_f32_16x16x128_f8f6f4 v[48:51], a[8:15], v[128:135], v[48:51] cbsz:0 blgp:1\n\t" \
  "ds_read_b64_tr_b8 v[146:147], v174 offset:33792\n\t" \
  "s_add_u32 m0, s77, 16384\n\t" \
  "s_add_u32 s90, s71, 4194304\n\t" \
  "buffer_load_dwordx4 v187, s[40:43], s90 offen offset:1024  lds\n\t" \
  "v_mfma_f32_16x16x128_f8f6f4 v[80:83], a[16:23], v[128:135], v[80:83] cbsz:0 blgp:1\n\t" \
  "ds_read_b64_tr_b8 v[148:149], v174 offset:40960\n\t" \
  "s_add_u32 m0, s77, 32768\n\t" \
  "s_add_u32 s90, s72, 4194304\n\t" \
  "buffer_load_dwordx4 v186, s[40:43], s90 offen offset:0  lds\n\t" \
  "v_mfma_f32_16x16x128_f8f6f4 v[112:115], a[24:31], v[128:135], v[112:115] cbsz:0 blgp:1\n\t" \
  "ds_read_b64_tr_b8 v[150:151], v174 offset:41984\n\t" \
  "s_add_u32 m0, s77, 49152\n\t" \
  "s_add_u32 s90, s73, 4194304\n\t" \
  "buffer_load_dwordx4 v186, s[40:43], s90 offen offset:0  lds\n\t" \
  "s_waitcnt lgkmcnt(4)\n\t" \
  "v_mfma_f32_16x16x128_f8f6f4 v[20:23], a[0:7], v[136:143], v[20:23] cbsz:0 blgp:1\n\t" \
  "ds_read_b64_tr_b8 v[152:153], v175 offset:32768\n\t" \
  "s_add_u32 m0, s77, 32768\n\t" \
  "s_add_u32 s90, s72, 4194304\n\t" \
  "buffer_load_dwordx4 v187, s[40:43], s90 offen offset:1024  lds\n\t" \
  "v_mfma_f32_16x16x128_f8f6f4 v[52:55], a[8:15], v[136:143], v[52:55] cbsz:0 blgp:1\n\t" \
  "ds_read_b64_tr_b8 v[154:155], v175 offset:33792\n\t" \
  "s_add_u32 m0, s77, 49152\n\t" \
  "s_add_u32 s90, s73, 4194304\n\t" \
  "buffer_load_dwordx4 v187, s[40:43], s90 offen offset:1024  lds\n\t" \
  "v_mfma_f32_16x16x128_f8f6f4 v[84:87], a[16:23], v[136:143], v[84:87] cbsz:0 blgp:1\n\t" \
  "ds_read_b64_tr_b8 v[156:157], v175 offset:40960\n\t" \
  "buffer_load_dwordx4 a[132:135], v188, s[36:39], s87 offen offset:1024 \n\t" \
  "v_mfma_f32_16x16x128_f8f6f4 v[116:119], a[24:31], v[136:143], v[116:119] cbsz:0 blgp:1\n\t" \
  "ds_read_b64_tr_b8 v[158:159], v175 offset:41984\n\t" \
  "s_waitcnt lgkmcnt(4)\n\t" \
  "v_mfma_f32_16x16x128_f8f6f4 v[24:27], a[0:7], v[144:151], v[24:27] cbsz:0 blgp:1\n\t" \
  "s_barrier\n\t" \
  "ds_read_b64_tr_b8 v[128:129], v168 offset:49152\n\t" \
  "v_mfma_f32_16x16x128_f8f6f4 v[56:59], a[8:15], v[144:151], v[56:59] cbsz:0 blgp:1\n\t" \
  "ds_read_b64_tr_b8 v[130:131], v168 offset:50176\n\t" \
  "buffer_load_dwordx4 a[136:139], v188, s[36:39], s87 offen offset:2048 \n\t" \
  "v_mfma_f32_16x16x128_f8f6f4 v[88:91], a[16:23], v[144:151], v[88:91] cbsz:0 blgp:1\n\t" \
  "ds_read_b64_tr_b8 v[132:133], v168 offset:57344\n\t" \
  "v_mfma_f32_16x16x128_f8f6f4 v[120:123], a[24:31], v[144:151], v[120:123] cbsz:0 blgp:1\n\t" \
  "ds_read_b64_tr_b8 v[134:135], v168 offset:58368\n\t" \
  "buffer_load_dwordx4 a[140:143], v188, s[36:39], s87 offen offset:3072 \n\t" \
  "s_waitcnt lgkmcnt(4)\n\t" \
  "v_mfma_f32_16x16x128_f8f6f4 v[28:31], a[0:7], v[152:159], v[28:31] cbsz:0 blgp:1\n\t" \
  "ds_read_b64_tr_b8 v[136:137], v169 offset:49152\n\t" \
  "v_mfma_f32_16x16x128_f8f6f4 v[60:63], a[8:15], v[152:159], v[60:63] cbsz:0 blgp:1\n\t" \
  "ds_read_b64_tr_b8 v[138:139], v169 offset:50176\n\t" \
  "buffer_load_dwordx4 a[144:147], v189, s[36:39], s87 offen offset:0 \n\t" \
  "v_mfma_f32_16x16x128_f8f6f4 v[92:95], a[16:23], v[152:159], v[92:95] cbsz:0 blgp:1\n\t" \
  "ds_read_b64_tr_b8 v[140:141], v169 offset:57344\n\t" \
  "v_mfma_f32_16x16x128_f8f6f4 v[124:127], a[24:31], v[152:159], v[124:127] cbsz:0 blgp:1\n\t" \
  "ds_read_b64_tr_b8 v[142:143], v169 offset:58368\n\t" \
  "buffer_load_dwordx4 a[148:151], v189, s[36:39], s87 offen offset:1024 \n\t" \
  "s_waitcnt vmcnt(44) lgkmcnt(4)\n\t" \
  "v_mfma_f32_16x16x128_f8f6f4 v[0:3], a[32:39], v[128:135], v[0:3] cbsz:0 blgp:1\n\t" \
  "ds_read_b64_tr_b8 v[144:145], v170 offset:49152\n\t" \
  "s_waitcnt vmcnt(42)\n\t" \
  "v_mfma_f32_16x16x128_f8f6f4 v[32:35], a[40:47], v[128:135], v[32:35] cbsz:0 blgp:1\n\t" \
  "ds_read_b64_tr_b8 v[146:147], v170 offset:50176\n\t" \
  "buffer_load_dwordx4 a[152:155], v189, s[36:39], s87 offen offset:2048 \n\t" \
  "s_waitcnt vmcnt(41)\n\t" \
  "v_mfma_f32_16x16x128_f8f6f4 v[64:67], a[48:55], v[128:135], v[64:67] cbsz:0 blgp:1\n\t" \
  "ds_read_b64_tr_b8 v[148:149], v170 offset:57344\n\t" \
  "s_waitcnt vmcnt(39)\n\t" \
  "v_mfma_f32_16x16x128_f8f6f4 v[96:99], a[56:63], v[128:135], v[96:99] cbsz:0 blgp:1\n\t" \
  "ds_read_b64_tr_b8 v[150:151], v170 offset:58368\n\t" \
  "s_cmp_eq_u32 s8, -1\n\t" \
  "s_cbranch_scc0 L_au10_0\n\t" \
  "L_aur10_0:\n\t" \
  "buffer_load_dwordx4 a[156:159], v189, s[36:39], s87 offen offset:3072 \n\t" \
  "s_waitcnt lgkmcnt(4)\n\t" \
  "v_mfma_f32_16x16x128_f8f6f4 v[4:7], a[32:39], v[136:143], v[4:7] cbsz:0 blgp:1\n\t" \
  "ds_read_b64_tr_b8 v[152:153], v171 offset:49152\n\t" \
  "v_mfma_f32_16x16x128_f8f6f4 v[36:39], a[40:47], v[136:143], v[36:39] cbsz:0 blgp:1\n\t" \
  "ds_read_b64_tr_b8 v[154:155], v171 offset:50176\n\t" \
  "s_bfe_u32 s84, s8, 262156\n\t" \
  "s_cmp_eq_u32 s84, 15\n\t" \
  "s_cbranch_scc0 L_q21_0\n\t" \
  "L_qr21_0:\n\t" \
  "s_mov_b32 s87, 1114112\n\t" \
  "buffer_load_dwordx4 a[160:163], v188, s[36:39], s87 offen offset:0 \n\t" \
  "v_mfma_f32_16x16x128_f8f6f4 v[68:71], a[48:55], v[136:143], v[68:71] cbsz:0 blgp:1\n\t" \
  "ds_read_b64_tr_b8 v[156:157], v171 offset:57344\n\t" \
  "v_mfma_f32_16x16x128_f8f6f4 v[100:103], a[56:63], v[136:143], v[100:103] cbsz:0 blgp:1\n\t" \
  "ds_read_b64_tr_b8 v[158:159], v171 offset:58368\n\t" \
  "buffer_load_dwordx4 a[164:167], v188, s[36:39], s87 offen offset:1024 \n\t" \
  "s_waitcnt lgkmcnt(4)\n\t" \
  "v_mfma_f32_16x16x128_f8f6f4 v[8:11], a[32:39], v[144:151], v[8:11] cbsz:0 blgp:1\n\t" \
  "ds_read_b64_tr_b8 v[128:129], v172 offset:49152\n\t" \
  "v_mfma_f32_16x16x128_f8f6f4 v[40:43], a[40:47], v[144:151], v[40:43] cbsz:0 blgp:1\n\t" \
  "ds_read_b64_tr_b8 v[130:131], v172 offset:50176\n\t" \
  "buffer_load_dwordx4 a[168:171], v188, s[36:39], s87 offen offset:2048 \n\t" \
  "v_mfma_f32_16x16x128_f8f6f4 v[72:75], a[48:55], v[144:151], v[72:75] cbsz:0 blgp:1\n\t" \
  "ds_read_b64_tr_b8 v[132:133], v172 offset:57344\n\t" \
  "v_mfma_f32_16x16x128_f8f6f4 v[104:107], a[56:63], v[144:151], v[104:107] cbsz:0 blgp:1\n\t" \
  "ds_read_b64_tr_b8 v[134:135], v172 offset:58368\n\t" \
  "buffer_load_dwordx4 a[172:175], v188, s[36:39], s87 offen offset:3072 \n\t" \
  "s_waitcnt lgkmcnt(4)\n\t" \
  "v_mfma_f32_16x16x128_f8f6f4 v[12:15], a[32:39], v[152:159], v[12:15] cbsz:0 blgp:1\n\t" \
  "ds_read_b64_tr_b8 v[136:137], v173 offset:49152\n\t" \
  "v_mfma_f32_16x16x128_f8f6f4 v[44:47], a[40:47], v[152:159], v[44:47] cbsz:0 blgp:1\n\t" \
  "ds_read_b64_tr_b8 v[138:139], v173 offset:50176\n\t" \
  "buffer_load_dwordx4 a[176:179], v189, s[36:39], s87 offen offset:0 \n\t" \
  "v_mfma_f32_16x16x128_f8f6f4 v[76:79], a[48:55], v[152:159], v[76:79] cbsz:0 blgp:1\n\t" \
  "ds_read_b64_tr_b8 v[140:141], v173 offset:57344\n\t" \
  "v_mfma_f32_16x16x128_f8f6f4 v[108:111], a[56:63], v[152:159], v[108:111] cbsz:0 blgp:1\n\t" \
  "ds_read_b64_tr_b8 v[142:143], v173 offset:58368\n\t" \
  "buffer_load_dwordx4 a[180:183], v189, s[36:39], s87 offen offset:1024 \n\t" \
  "s_waitcnt lgkmcnt(4)\n\t" \
  "v_mfma_f32_16x16x128_f8f6f4 v[16:19], a[32:39], v[128:135], v[16:19] cbsz:0 blgp:1\n\t" \
  "ds_read_b64_tr_b8 v[144:145], v174 offset:49152\n\t" \
  "v_mfma_f32_16x16x128_f8f6f4 v[48:51], a[40:47], v[128:135], v[48:51] cbsz:0 blgp:1\n\t" \
  "ds_read_b64_tr_b8 v[146:147], v174 offset:50176\n\t" \
  "buffer_load_dwordx4 a[184:187], v189, s[36:39], s87 offen offset:2048 \n\t" \
  "v_mfma_f32_16x16x128_f8f6f4 v[80:83], a[48:55], v[128:135], v[80:83] cbsz:0 blgp:1\n\t" \
  "ds_read_b64_tr_b8 v[148:149], v174 offset:57344\n\t" \
  "v_mfma_f32_16x16x128_f8f6f4 v[112:115], a[56:63], v[128:135], v[112:115] cbsz:0 blgp:1\n\t" \
  "ds_read_b64_tr_b8 v[150:151], v174 offset:58368\n\t" \
  "buffer_load_dwordx4 a[188:191], v189, s[36:39], s87 offen offset:3072 \n\t" \
  "s_waitcnt lgkmcnt(4)\n\t" \
  "v_mfma_f32_16x16x128_f8f6f4 v[20:23], a[32:39], v[136:143], v[20:23] cbsz:0 blgp:1\n\t" \
  "ds_read_b64_tr_b8 v[152:153], v175 offset:49152\n\t" \
  "v_mfma_f32_16x16x128_f8f6f4 v[52:55], a[40:47], v[136:143], v[52:55] cbsz:0 blgp:1\n\t" \
  "ds_read_b64_tr_b8 v[154:155], v175 offset:50176\n\t" \
  "s_bfe_u32 s84, s8, 262160\n\t" \
  "s_cmp_eq_u32 s84, 15\n\t" \
  "s_cbranch_scc0 L_q22_0\n\t" \
  "L_qr22_0:\n\t" \
  "s_mov_b32 s87, 1179648\n\t" \
  "buffer_load_dwordx4 a[192:195], v188, s[36:39], s87 offen offset:0 \n\t" \
  "v_mfma_f32_16x16x128_f8f6f4 v[84:87], a[48:55], v[136:143], v[84:87] cbsz:0 blgp:1\n\t" \
  "ds_read_b64_tr_b8 v[156:157], v175 offset:57344\n\t" \
  "v_mfma_f32_16x16x128_f8f6f4 v[116:119], a[56:63], v[136:143], v[116:119] cbsz:0 blgp:1\n\t" \
  "ds_read_b64_tr_b8 v[158:159], v175 offset:58368\n\t" \
  "buffer_load_dwordx4 a[196:199], v188, s[36:39], s87 offen offset:1024 \n\t" \
  "s_waitcnt lgkmcnt(4)\n\t" \
  "v_mfma_f32_16x16x128_f8f6f4 v[24:27], a[32:39], v[144:151], v[24:27] cbsz:0 blgp:1\n\t" \
  "s_barrier\n\t" \
  "ds_read_b64_tr_b8 v[128:129], v176 offset:0\n\t" \
  "v_mfma_f32_16x16x128_f8f6f4 v[56:59], a[40:47], v[144:151], v[56:59] cbsz:0 blgp:1\n\t" \
  "ds_read_b64_tr_b8 v[130:131], v176 offset:1024\n\t" \
  "buffer_load_dwordx4 a[200:203], v188, s[36:39], s87 offen offset:2048 \n\t" \
  "v_mfma_f32_16x16x128_f8f6f4 v[88:91], a[48:55], v[144:151], v[88:91] cbsz:0 blgp:1\n\t" \
  "ds_read_b64_tr_b8 v[132:133], v176 offset:8192\n\t" \
  "v_mfma_f32_16x16x128_f8f6f4 v[120:123], a[56:63], v[144:151], v[120:123] cbsz:0 blgp:1\n\t" \
  "ds_read_b64_tr_b8 v[134:135], v176 offset:9216\n\t" \
  "buffer_load_dwordx4 a[204:207], v188, s[36:39], s87 offen offset:3072 \n\t" \
  "s_waitcnt lgkmcnt(4)\n\t" \
  "v_mfma_f32_16x16x128_f8f6f4 v[28:31], a[32:39], v[152:159], v[28:31] cbsz:0 blgp:1\n\t" \
  "ds_read_b64_tr_b8 v[136:137], v177 offset:0\n\t" \
  "v_mfma_f32_16x16x128_f8f6f4 v[60:63], a[40:47], v[152:159], v[60:63] cbsz:0 blgp:1\n\t" \
  "ds_read_b64_tr_b8 v[138:139], v177 offset:1024\n\t" \
  "buffer_load_dwordx4 a[208:211], v189, s[36:39], s87 offen offset:0 \n\t" \
  "v_mfma_f32_16x16x128_f8f6f4 v[92:95], a[48:55], v[152:159], v[92:95] cbsz:0 blgp:1\n\t" \
  "ds_read_b64_tr_b8 v[140:141], v177 offset:8192\n\t" \
  "v_mfma_f32_16x16x128_f8f6f4 v[124:127], a[56:63], v[152:159], v[124:127] cbsz:0 blgp:1\n\t" \
  "ds_read_b64_tr_b8 v[142:143], v177 offset:9216\n\t" \
  "buffer_load_dwordx4 a[212:215], v189, s[36:39], s87 offen offset:1024 \n\t" \
  "s_waitcnt vmcnt(52) lgkmcnt(4)\n\t" \
  "v_mfma_f32_16x16x128_f8f6f4 v[0:3], a[64:71], v[128:135], v[0:3] cbsz:0 blgp:1\n\t" \
  "ds_read_b64_tr_b8 v[144:145], v178 offset:0\n\t" \
  "s_waitcnt vmcnt(50)\n\t" \
  "v_mfma_f32_16x16x128_f8f6f4 v[32:35], a[72:79], v[128:135], v[32:35] cbsz:0 blgp:1\n\t" \
  "ds_read_b64_tr_b8 v[146:147], v178 offset:1024\n\t" \
  "buffer_load_dwordx4 a[216:219], v189, s[36:39], s87 offen offset:2048 \n\t" \
  "s_waitcnt vmcnt(49)\n\t" \
  "v_mfma_f32_16x16x128_f8f6f4 v[64:67], a[80:87], v[128:135], v[64:67] cbsz:0 blgp:1\n\t" \
  "ds_read_b64_tr_b8 v[148:149], v178 offset:8192\n\t" \
  "s_add_u32 m0, s76, 65536\n\t" \
  "s_add_u32 s90, s74, 4194304\n\t" \
  "buffer_load_dwordx4 v184, s[40:43], s90 offen offset:0  lds\n\t" \
  "s_waitcnt vmcnt(48)\n\t" \
  "v_mfma_f32_16x16x128_f8f6f4 v[96:99], a[88:95], v[128:135], v[96:99] cbsz:0 blgp:1\n\t" \
  "ds_read_b64_tr_b8 v[150:151], v178 offset:9216\n\t" \
  "s_add_u32 m0, s76, 65536\n\t" \
  "s_add_u32 s90, s74, 4194304\n\t" \
  "buffer_load_dwordx4 v185, s[40:43], s90 offen offset:1024  lds\n\t" \
  "s_waitcnt lgkmcnt(4)\n\t" \
  "v_mfma_f32_16x16x128_f8f6f4 v[4:7], a[64:71], v[136:143], v[4:7] cbsz:0 blgp:1\n\t" \
  "ds_read_b64_tr_b8 v[152:153], v179 offset:0\n\t" \
  "s_mov_b32 s9, 0\n\t" \
  "s_add_u32 m0, s76, 81920\n\t" \
  "s_add_u32 s90, s68, 6291456\n\t" \
  "buffer_load_dwordx4 v184, s[40:43], s90 offen offset:0  lds\n\t" \
  "v_mfma_f32_16x16x128_f8f6f4 v[36:39], a[72:79], v[136:143], v[36:39] cbsz:0 blgp:1\n\t" \
  "ds_read_b64_tr_b8 v[154:155], v179 offset:1024\n\t" \
  "global_load_dword v196, v195, s[26:27] offset:3072 sc1\n\t" \
  "s_add_u32 m0, s76, 81920\n\t" \
  "s_add_u32 s90, s68, 6291456\n\t" \
  "buffer_load_dwordx4 v185, s[40:43], s90 offen offset:1024  lds\n\t" \
  "v_mfma_f32_16x16x128_f8f6f4 v[68:71], a[80:87], v[136:143], v[68:71] cbsz:0 blgp:1\n\t" \
  "ds_read_b64_tr_b8 v[156:157], v179 offset:8192\n\t" \
  "s_add_u32 m0, s76, 98304\n\t" \
  "s_add_u32 s90, s69, 6291456\n\t" \
  "buffer_load_dwordx4 v184, s[40:43], s90 offen offset:0  lds\n\t" \
  "v_mfma_f32_16x16x128_f8f6f4 v[100:103], a[88:95], v[136:143], v[100:103] cbsz:0 blgp:1\n\t" \
  "ds_read_b64_tr_b8 v[158:159], v179 offset:9216\n\t" \
  "s_add_u32 m0, s76, 98304\n\t" \
  "s_add_u32 s90, s69, 6291456\n\t" \
  "buffer_load_dwordx4 v185, s[40:43], s90 offen offset:1024  lds\n\t" \
  "s_waitcnt lgkmcnt(4)\n\t" \
  "v_mfma_f32_16x16x128_f8f6f4 v[8:11], a[64:71], v[144:151], v[8:11] cbsz:0 blgp:1\n\t" \
  "ds_read_b64_tr_b8 v[128:129], v180 offset:0\n\t" \
  "s_add_u32 s90, s70, 6291456\n\t" \
  "s_mov_b64 exec, 0xffff\n\t" \
  "buffer_load_dword v210, v241, s[40:43], s90 offen\n\t" \
  "s_mov_b64 exec, -1\n\t" \
  "v_mfma_f32_16x16x128_f8f6f4 v[40:43], a[72:79], v[144:151], v[40:43] cbsz:0 blgp:1\n\t" \
  "ds_read_b64_tr_b8 v[130:131], v180 offset:1024\n\t" \
  "buffer_load_dwordx4 a[220:223], v189, s[36:39], s87 offen offset:3072 \n\t" \
  "v_mfma_f32_16x16x128_f8f6f4 v[72:75], a[80:87], v[144:151], v[72:75] cbsz:0 blgp:1\n\t" \
  "ds_read_b64_tr_b8 v[132:133], v180 offset:8192\n\t" \
  "v_mfma_f32_16x16x128_f8f6f4 v[104:107], a[88:95], v[144:151], v[104:107] cbsz:0 blgp:1\n\t" \
  "ds_read_b64_tr_b8 v[134:135], v180 offset:9216\n\t" \
  "s_waitcnt lgkmcnt(4)\n\t" \
  "v_mfma_f32_16x16x128_f8f6f4 v[12:15], a[64:71], v[152:159], v[12:15] cbsz:0 blgp:1\n\t" \
  "ds_read_b64_tr_b8 v[136:137], v181 offset:0\n\t" \
  "v_mfma_f32_16x16x128_f8f6f4 v[44:47], a[72:79], v[152:159], v[44:47] cbsz:0 blgp:1\n\t" \
  "ds_read_b64_tr_b8 v[138:139], v181 offset:1024\n\t" \
  "v_mfma_f32_16x16x128_f8f6f4 v[76:79], a[80:87], v[152:159], v[76:79] cbsz:0 blgp:1\n\t" \
  "ds_read_b64_tr_b8 v[140:141], v181 offset:8192\n\t" \
  "v_mfma_f32_16x16x128_f8f6f4 v[108:111], a[88:95], v[152:159], v[108:111] cbsz:0 blgp:1\n\t" \
  "ds_read_b64_tr_b8 v[142:143], v181 offset:9216\n\t" \
  "s_waitcnt lgkmcnt(4)\n\t" \
  "v_mfma_f32_16x16x128_f8f6f4 v[16:19], a[64:71], v[128:135], v[16:19] cbsz:0 blgp:1\n\t" \
  "ds_read_b64_tr_b8 v[144:145], v182 offset:0\n\t" \
  "v_mfma_f32_16x16x128_f8f6f4 v[48:51], a[72:79], v[128:135], v[48:51] cbsz:0 blgp:1\n\t" \
  "ds_read_b64_tr_b8 v[146:147], v182 offset:1024\n\t" \
  "v_mfma_f32_16x16x128_f8f6f4 v[80:83], a[80:87], v[128:135], v[80:83] cbsz:0 blgp:1\n\t" \
  "ds_read_b64_tr_b8 v[148:149], v182 offset:8192\n\t" \
  "v_mfma_f32_16x16x128_f8f6f4 v[112:115], a[88:95], v[128:135], v[112:115] cbsz:0 blgp:1\n\t" \
  "ds_read_b64_tr_b8 v[150:151], v182 offset:9216\n\t" \
  "s_waitcnt lgkmcnt(4)\n\t" \
  "v_mfma_f32_16x16x128_f8f6f4 v[20:23], a[64:71], v[136:143], v[20:23] cbsz:0 blgp:1\n\t" \
  "ds_read_b64_tr_b8 v[152:153], v183 offset:0\n\t" \
  "v_mfma_f32_16x16x128_f8f6f4 v[52:55], a[72:79], v[136:143], v[52:55] cbsz:0 blgp:1\n\t" \
  "ds_read_b64_tr_b8 v[154:155], v183 offset:1024\n\t" \
  "v_mfma_f32_16x16x128_f8f6f4 v[84:87], a[80:87], v[136:143], v[84:87] cbsz:0 blgp:1\n\t" \
  "ds_read_b64_tr_b8 v[156:157], v183 offset:8192\n\t" \
  "v_mfma_f32_16x16x128_f8f6f4 v[116:119], a[88:95], v[136:143], v[116:119] cbsz:0 blgp:1\n\t" \
  "ds_read_b64_tr_b8 v[158:159], v183 offset:9216\n\t" \
  "s_waitcnt lgkmcnt(4)\n\t" \
  "v_mfma_f32_16x16x128_f8f6f4 v[24:27], a[64:71], v[144:151], v[24:27] cbsz:0 blgp:1\n\t" \
  "s_barrier\n\t" \
  "ds_read_b64_tr_b8 v[128:129], v176 offset:16384\n\t" \
  "v_mfma_f32_16x16x128_f8f6f4 v[56:59], a[72:79], v[144:151], v[56:59] cbsz:0 blgp:1\n\t" \
  "ds_read_b64_tr_b8 v[130:131], v176 offset:17408\n\t" \
  "s_waitcnt vmcnt(5)\n\t" \
  "v_cmp_le_u32 vcc, s79, v196\n\t" \
  "s_nop 4\n\t" \
  "s_andn2_b32 s85, vcc_lo, s9\n\t" \
  "s_or_b32 s9, s9, vcc_lo\n\t" \
  "s_and_b32 exec_lo, s85, 0xfffffff\n\t" \
  "s_mov_b32 exec_hi, 0\n\t" \
  "s_cbranch_execz L_u4_0\n\t" \
  "s_mov_b32 s84, 1376256\n\t" \
  "buffer_load_dword v210, v198, s[36:39], s84 offen \n\t" \
  "L_u4_0:\n\t" \
  "s_mov_b64 exec, -1\n\t" \
  "v_mfma_f32_16x16x128_f8f6f4 v[88:91], a[80:87], v[144:151], v[88:91] cbsz:0 blgp:1\n\t" \
  "ds_read_b64_tr_b8 v[132:133], v176 offset:24576\n\t" \
  "v_mfma_f32_16x16x128_f8f6f4 v[120:123], a[88:95], v[144:151], v[120:123] cbsz:0 blgp:1\n\t" \
  "ds_read_b64_tr_b8 v[134:135], v176 offset:25600\n\t" \
  "s_waitcnt lgkmcnt(4)\n\t" \
  "v_mfma_f32_16x16x128_f8f6f4 v[28:31], a[64:71], v[152:159], v[28:31] cbsz:0 blgp:1\n\t" \
  "ds_read_b64_tr_b8 v[136:137], v177 offset:16384\n\t" \
  "v_mfma_f32_16x16x128_f8f6f4 v[60:63], a[72:79], v[152:159], v[60:63] cbsz:0 blgp:1\n\t" \
  "ds_read_b64_tr_b8 v[138:139], v177 offset:17408\n\t" \
  "v_mfma_f32_16x16x128_f8f6f4 v[92:95], a[80:87], v[152:159], v[92:95] cbsz:0 blgp:1\n\t" \
  "ds_read_b64_tr_b8 v[140:141], v177 offset:24576\n\t" \
  "v_mfma_f32_16x16x128_f8f6f4 v[124:127], a[88:95], v[152:159], v[124:127] cbsz:0 blgp:1\n\t" \
  "ds_read_b64_tr_b8 v[142:143], v177 offset:25600\n\t" \
  "s_waitcnt lgkmcnt(4)\n\t" \
  "v_mfma_f32_16x16x128_f8f6f4 v[0:3], a[96:103], v[128:135], v[0:3] cbsz:0 blgp:1\n\t" \
  "ds_read_b64_tr_b8 v[144:145], v178 offset:16384\n\t" \
  "v_mfma_f32_16x16x128_f8f6f4 v[32:35], a[104:111], v[128:135], v[32:35] cbsz:0 blgp:1\n\t" \
  "ds_read_b64_tr_b8 v[146:147], v178 offset:17408\n\t" \
  "v_mfma_f32_16x16x128_f8f6f4 v[64:67], a[112:119], v[128:135], v[64:67] cbsz:0 blgp:1\n\t" \
  "ds_read_b64_tr_b8 v[148:149], v178 offset:24576\n\t" \
  "v_mfma_f32_16x16x128_f8f6f4 v[96:99], a[120:127], v[128:135], v[96:99] cbsz:0 blgp:1\n\t" \
  "ds_read_b64_tr_b8 v[150:151], v178 offset:25600\n\t" \
  "s_waitcnt lgkmcnt(4)\n\t" \
  "v_mfma_f32_16x16x128_f8f6f4 v[4:7], a[96:103], v[136:143], v[4:7] cbsz:0 blgp:1\n\t" \
  "ds_read_b64_tr_b8 v[152:153], v179 offset:16384\n\t" \
  "v_mfma_f32_16x16x128_f8f6f4 v[36:39], a[104:111], v[136:143], v[36:39] cbsz:0 blgp:1\n\t" \
  "ds_read_b64_tr_b8 v[154:155], v179 offset:17408\n\t" \
  "s_cmp_eq_u32 s9, -1\n\t" \
  "s_cbranch_scc0 L_ap12_0\n\t" \
  "L_apr12_0:\n\t" \
  "v_mfma_f32_16x16x128_f8f6f4 v[68:71], a[112:119], v[136:143], v[68:71] cbsz:0 blgp:1\n\t" \
  "ds_read_b64_tr_b8 v[156:157], v179 offset:24576\n\t" \
  "v_mfma_f32_16x16x128_f8f6f4 v[100:103], a[120:127], v[136:143], v[100:103] cbsz:0 blgp:1\n\t" \
  "ds_read_b64_tr_b8 v[158:159], v179 offset:25600\n\t" \
  "s_waitcnt lgkmcnt(4)\n\t" \
  "v_mfma_f32_16x16x128_f8f6f4 v[8:11], a[96:103], v[144:151], v[8:11] cbsz:0 blgp:1\n\t" \
  "ds_read_b64_tr_b8 v[128:129], v180 offset:16384\n\t" \
  "v_mfma_f32_16x16x128_f8f6f4 v[40:43], a[104:111], v[144:151], v[40:43] cbsz:0 blgp:1\n\t" \
  "ds_read_b64_tr_b8 v[130:131], v180 offset:17408\n\t" \
  "s_bfe_u32 s84, s8, 262164\n\t" \
  "s_cmp_eq_u32 s84, 15\n\t" \
  "s_cbranch_scc0 L_q23_0\n\t" \
  "L_qr23_0:\n\t" \
  "s_mov_b32 s87, 1245184\n\t" \
  "buffer_load_dwordx4 a[224:227], v188, s[36:39], s87 offen offset:0 \n\t" \
  "v_mfma_f32_16x16x128_f8f6f4 v[72:75], a[112:119], v[144:151], v[72:75] cbsz:0 blgp:1\n\t" \
  "ds_read_b64_tr_b8 v[132:133], v180 offset:24576\n\t" \
  "v_mfma_f32_16x16x128_f8f6f4 v[104:107], a[120:127], v[144:151], v[104:107] cbsz:0 blgp:1\n\t" \
  "ds_read_b64_tr_b8 v[134:135], v180 offset:25600\n\t" \
  "buffer_load_dwordx4 a[228:231], v188, s[36:39], s87 offen offset:1024 \n\t" \
  "s_waitcnt lgkmcnt(4)\n\t" \
  "v_mfma_f32_16x16x128_f8f6f4 v[12:15], a[96:103], v[152:159], v[12:15] cbsz:0 blgp:1\n\t" \
  "ds_read_b64_tr_b8 v[136:137], v181 offset:16384\n\t" \
  "v_mfma_f32_16x16x128_f8f6f4 v[44:47], a[104:111], v[152:159], v[44:47] cbsz:0 blgp:1\n\t" \
  "ds_read_b64_tr_b8 v[138:139], v181 offset:17408\n\t" \
  "buffer_load_dwordx4 a[232:235], v188, s[36:39], s87 offen offset:2048 \n\t" \
  "v_mfma_f32_16x16x128_f8f6f4 v[76:79], a[112:119], v[152:159], v[76:79] cbsz:0 blgp:1\n\t" \
  "ds_read_b64_tr_b8 v[140:141], v181 offset:24576\n\t" \
  "v_mfma_f32_16x16x128_f8f6f4 v[108:111], a[120:127], v[152:159], v[108:111] cbsz:0 blgp:1\n\t" \
  "ds_read_b64_tr_b8 v[142:143], v181 offset:25600\n\t" \
  "buffer_load_dwordx4 a[236:239], v188, s[36:39], s87 offen offset:3072 \n\t" \
  "s_waitcnt lgkmcnt(4)\n\t" \
  "v_mfma_f32_16x16x128_f8f6f4 v[16:19], a[96:103], v[128:135], v[16:19] cbsz:0 blgp:1\n\t" \
  "ds_read_b64_tr_b8 v[144:145], v182 offset:16384\n\t" \
  "v_mfma_f32_16x16x128_f8f6f4 v[48:51], a[104:111], v[128:135], v[48:51] cbsz:0 blgp:1\n\t" \
  "ds_read_b64_tr_b8 v[146:147], v182 offset:17408\n\t" \
  "buffer_load_dwordx4 a[240:243], v189, s[36:39], s87 offen offset:0 \n\t" \
  "v_mfma_f32_16x16x128_f8f6f4 v[80:83], a[112:119], v[128:135], v[80:83] cbsz:0 blgp:1\n\t" \
  "ds_read_b64_tr_b8 v[148:149], v182 offset:24576\n\t" \
  "v_mfma_f32_16x16x128_f8f6f4 v[112:115], a[120:127], v[128:135], v[112:115] cbsz:0 blgp:1\n\t" \
  "ds_read_b64_tr_b8 v[150:151], v182 offset:25600\n\t" \
  "buffer_load_dwordx4 a[244:247], v189, s[36:39], s87 offen offset:1024 \n\t" \
  "s_waitcnt lgkmcnt(4)\n\t" \
  "v_mfma_f32_16x16x128_f8f6f4 v[20:23], a[96:103], v[136:143], v[20:23] cbsz:0 blgp:1\n\t" \
  "ds_read_b64_tr_b8 v[152:153], v183 offset:16384\n\t" \
  "v_mfma_f32_16x16x128_f8f6f4 v[52:55], a[104:111], v[136:143], v[52:55] cbsz:0 blgp:1\n\t" \
  "ds_read_b64_tr_b8 v[154:155], v183 offset:17408\n\t" \
  "buffer_load_dwordx4 a[248:251], v189, s[36:39], s87 offen offset:2048 \n\t" \
  "v_mfma_f32_16x16x128_f8f6f4 v[84:87], a[112:119], v[136:143], v[84:87] cbsz:0 blgp:1\n\t" \
  "ds_read_b64_tr_b8 v[156:157], v183 offset:24576\n\t" \
  "v_mfma_f32_16x16x128_f8f6f4 v[116:119], a[120:127], v[136:143], v[116:119] cbsz:0 blgp:1\n\t" \
  "ds_read_b64_tr_b8 v[158:159], v183 offset:25600\n\t" \
  "buffer_load_dwordx4 a[252:255], v189, s[36:39], s87 offen offset:3072 \n\t" \
  "s_waitcnt lgkmcnt(4)\n\t" \
  "v_mfma_f32_16x16x128_f8f6f4 v[24:27], a[96:103], v[144:151], v[24:27] cbsz:0 blgp:1\n\t" \
  "s_barrier\n\t" \
  "ds_read_b64_tr_b8 v[128:129], v160 offset:0\n\t" \
  "v_mfma_f32_16x16x128_f8f6f4 v[56:59], a[104:111], v[144:151], v[56:59] cbsz:0 blgp:1\n\t" \
  "ds_read_b64_tr_b8 v[130:131], v160 offset:1024\n\t" \
  "s_cmp_eq_u32 s9, -1\n\t" \
  "s_cbranch_scc0 L_au12_0\n\t" \
  "L_aur12_0:\n\t" \
  "s_add_u32 m0, s77, 65536\n\t" \
  "s_add_u32 s90, s74, 4194304\n\t" \
  "buffer_load_dwordx4 v186, s[40:43], s90 offen offset:0  lds\n\t" \
  "v_mfma_f32_16x16x128_f8f6f4 v[88:91], a[112:119], v[144:151], v[88:91] cbsz:0 blgp:1\n\t" \
  "ds_read_b64_tr_b8 v[132:133], v160 offset:8192\n\t" \
  "s_add_u32 m0, s77, 81920\n\t" \
  "s_add_u32 s90, s68, 6291456\n\t" \
  "buffer_load_dwordx4 v186, s[40:43], s90 offen offset:0  lds\n\t" \
  "v_mfma_f32_16x16x128_f8f6f4 v[120:123], a[120:127], v[144:151], v[120:123] cbsz:0 blgp:1\n\t" \
  "ds_read_b64_tr_b8 v[134:135], v160 offset:9216\n\t" \
  "s_add_u32 m0, s77, 65536\n\t" \
  "s_add_u32 s90, s74, 4194304\n\t" \
  "buffer_load_dwordx4 v187, s[40:43], s90 offen offset:1024  lds\n\t" \
  "s_waitcnt lgkmcnt(4)\n\t" \
  "v_mfma_f32_16x16x128_f8f6f4 v[28:31], a[96:103], v[152:159], v[28:31] cbsz:0 blgp:1\n\t" \
  "ds_read_b64_tr_b8 v[136:137], v161 offset:0\n\t" \
  "s_add_u32 m0, s77, 81920\n\t" \
  "s_add_u32 s90, s68, 6291456\n\t" \
  "buffer_load_dwordx4 v187, s[40:43], s90 offen offset:1024  lds\n\t" \
  "v_mfma_f32_16x16x128_f8f6f4 v[60:63], a[104:111], v[152:159], v[60:63] cbsz:0 blgp:1\n\t" \
  "ds_read_b64_tr_b8 v[138:139], v161 offset:1024\n\t" \
  "s_add_u32 m0, s77, 98304\n\t" \
  "s_add_u32 s90, s69, 6291456\n\t" \
  "buffer_load_dwordx4 v186, s[40:43], s90 offen offset:0  lds\n\t" \
  "v_mfma_f32_16x16x128_f8f6f4 v[92:95], a[112:119], v[152:159], v[92:95] cbsz:0 blgp:1\n\t" \
  "ds_read_b64_tr_b8 v[140:141], v161 offset:8192\n\t" \
  "s_add_u32 m0, s76, 114688\n\t" \
  "s_add_u32 s90, s70, 6291456\n\t" \
  "buffer_load_dwordx4 v184, s[40:43], s90 offen offset:0  lds\n\t" \
  "v_mfma_f32_16x16x128_f8f6f4 v[124:127], a[120:127], v[152:159], v[124:127] cbsz:0 blgp:1\n\t" \
  "ds_read_b64_tr_b8 v[142:143], v161 offset:9216\n\t" \
  "s_add_u32 m0, s77, 98304\n\t" \
  "s_add_u32 s90, s69, 6291456\n\t" \
  "buffer_load_dwordx4 v187, s[40:43], s90 offen offset:1024  lds\n\t" \
  "s_waitcnt lgkmcnt(4)\n\t" \
  "v_mfma_f32_16x16x128_f8f6f4 v[0:3], a[128:135], v[128:135], v[0:3] cbsz:0 blgp:1\n\t" \
  "ds_read_b64_tr_b8 v[144:145], v162 offset:0\n\t" \
  "s_add_u32 m0, s76, 131072\n\t" \
  "s_add_u32 s90, s71, 6291456\n\t" \
  "buffer_load_dwordx4 v184, s[40:43], s90 offen offset:0  lds\n\t" \
  "v_mfma_f32_16x16x128_f8f6f4 v[32:35], a[136:143], v[128:135], v[32:35] cbsz:0 blgp:1\n\t" \
  "ds_read_b64_tr_b8 v[146:147], v162 offset:1024\n\t" \
  "s_add_u32 m0, s76, 131072\n\t" \
  "s_add_u32 s90, s71, 6291456\n\t" \
  "buffer_load_dwordx4 v185, s[40:43], s90 offen offset:1024  lds\n\t" \
  "v_mfma_f32_16x16x128_f8f6f4 v[64:67], a[144:151], v[128:135], v[64:67] cbsz:0 blgp:1\n\t" \
  "ds_read_b64_tr_b8 v[148:149], v162 offset:8192\n\t" \
  "s_cmp_gt_u32 s29, 1\n\t" \
  "s_cbranch_scc1 L_k642_0\n\t" \
  "s_getpc_b64 s[88:89]\n\t" \
  "s_lshl_b32 s84, s29, 13\n\t" \
  "s_add_u32 s84, s84, 2048\n\t" \
  "s_add_u32 s88, s88, s84\n\t" \
  "s_addc_u32 s89, s89, 0\n\t" \
  "s_mov_b64 exec, 0xff\n\t" \
  "global_load_dword v210, v234, s[88:89]\n\t" \
  "s_mov_b64 exec, -1\n\t" \
  "L_k642_0:\n\t" \
  "s_add_u32 s90, s72, 6291456\n\t" \
  "s_mov_b64 exec, 0xffff\n\t" \
  "buffer_load_dword v210, v241, s[40:43], s90 offen\n\t" \
  "s_mov_b64 exec, -1\n\t" \
  "v_mfma_f32_16x16x128_f8f6f4 v[96:99], a[152:159], v[128:135], v[96:99] cbsz:0 blgp:1\n\t" \
  "ds_read_b64_tr_b8 v[150:151], v162 offset:9216\n\t" \
  "s_add_u32 s90, s73, 6291456\n\t" \
  "s_mov_b64 exec, 0xffff\n\t" \
  "buffer_load_dword v210, v241, s[40:43], s90 offen\n\t" \
  "s_mov_b64 exec, -1\n\t" \
  "s_waitcnt lgkmcnt(4)\n\t" \
  "v_mfma_f32_16x16x128_f8f6f4 v[4:7], a[128:135], v[136:143], v[4:7] cbsz:0 blgp:1\n\t" \
  "ds_read_b64_tr_b8 v[152:153], v163 offset:0\n\t" \
  "s_add_u32 s90, s74, 6291456\n\t" \
  "s_mov_b64 exec, 0xffff\n\t" \
  "buffer_load_dword v210, v241, s[40:43], s90 offen\n\t" \
  "s_mov_b64 exec, -1\n\t" \
  "v_mfma_f32_16x16x128_f8f6f4 v[36:39], a[136:143], v[136:143], v[36:39] cbsz:0 blgp:1\n\t" \
  "ds_read_b64_tr_b8 v[154:155], v163 offset:1024\n\t" \
  "s_cmp_eq_u32 s9, -1\n\t" \
  "s_cbranch_scc0 L_ap13_0\n\t" \
  "L_apr13_0:\n\t" \
  "s_add_u32 m0, s76, 114688\n\t" \
  "s_add_u32 s90, s70, 6291456\n\t" \
  "buffer_load_dwordx4 v185, s[40:43], s90 offen offset:1024  lds\n\t" \
  "v_mfma_f32_16x16x128_f8f6f4 v[68:71], a[144:151], v[136:143], v[68:71] cbsz:0 blgp:1\n\t" \
  "ds_read_b64_tr_b8 v[156:157], v163 offset:8192\n\t" \
  "s_bfe_u32 s84, s8, 262168\n\t" \
  "s_cmp_eq_u32 s84, 15\n\t" \
  "s_cbranch_scc0 L_q24_0\n\t" \
  "L_qr24_0:\n\t" \
  "s_mov_b32 s87, 1310720\n\t" \
  "buffer_load_dwordx4 a[0:3], v188, s[36:39], s87 offen offset:0 \n\t" \
  "v_mfma_f32_16x16x128_f8f6f4 v[100:103], a[152:159], v[136:143], v[100:103] cbsz:0 blgp:1\n\t" \
  "ds_read_b64_tr_b8 v[158:159], v163 offset:9216\n\t" \
  "s_waitcnt lgkmcnt(4)\n\t" \
  "v_mfma_f32_16x16x128_f8f6f4 v[8:11], a[128:135], v[144:151], v[8:11] cbsz:0 blgp:1\n\t" \
  "ds_read_b64_tr_b8 v[128:129], v164 offset:0\n\t" \
  "v_mfma_f32_16x16x128_f8f6f4 v[40:43], a[136:143], v[144:151], v[40:43] cbsz:0 blgp:1\n\t" \
  "ds_read_b64_tr_b8 v[130:131], v164 offset:1024\n\t" \
  "v_mfma_f32_16x16x128_f8f6f4 v[72:75], a[144:151], v[144:151], v[72:75] cbsz:0 blgp:1\n\t" \
  "ds_read_b64_tr_b8 v[132:133], v164 offset:8192\n\t" \
  "v_mfma_f32_16x16x128_f8f6f4 v[104:107], a[152:159], v[144:151], v[104:107] cbsz:0 blgp:1\n\t" \
  "ds_read_b64_tr_b8 v[134:135], v164 offset:9216\n\t" \
  "s_waitcnt lgkmcnt(4)\n\t" \
  "v_mfma_f32_16x16x128_f8f6f4 v[12:15], a[128:135], v[152:159], v[12:15] cbsz:0 blgp:1\n\t" \
  "ds_read_b64_tr_b8 v[136:137], v165 offset:0\n\t" \
  "v_mfma_f32_16x16x128_f8f6f4 v[44:47], a[136:143], v[152:159], v[44:47] cbsz:0 blgp:1\n\t" \
  "ds_read_b64_tr_b8 v[138:139], v165 offset:1024\n\t" \
  "v_mfma_f32_16x16x128_f8f6f4 v[76:79], a[144:151], v[152:159], v[76:79] cbsz:0 blgp:1\n\t" \
  "ds_read_b64_tr_b8 v[140:141], v165 offset:8192\n\t" \
  "v_mfma_f32_16x16x128_f8f6f4 v[108:111], a[152:159], v[152:159], v[108:111] cbsz:0 blgp:1\n\t" \
  "ds_read_b64_tr_b8 v[142:143], v165 offset:9216\n\t" \
  "s_waitcnt lgkmcnt(4)\n\t" \
  "v_mfma_f32_16x16x128_f8f6f4 v[16:19], a[128:135], v[128:135], v[16:19] cbsz:0 blgp:1\n\t" \
  "ds_read_b64_tr_b8 v[144:145], v166 offset:0\n\t" \
  "v_mfma_f32_16x16x128_f8f6f4 v[48:51], a[136:143], v[128:135], v[48:51] cbsz:0 blgp:1\n\t" \
  "ds_read_b64_tr_b8 v[146:147], v166 offset:1024\n\t" \
  "v_mfma_f32_16x16x128_f8f6f4 v[80:83], a[144:151], v[128:135], v[80:83] cbsz:0 blgp:1\n\t" \
  "ds_read_b64_tr_b8 v[148:149], v166 offset:8192\n\t" \
  "v_mfma_f32_16x16x128_f8f6f4 v[112:115], a[152:159], v[128:135], v[112:115] cbsz:0 blgp:1\n\t" \
  "ds_read_b64_tr_b8 v[150:151], v166 offset:9216\n\t" \
  "s_waitcnt lgkmcnt(4)\n\t" \
  "v_mfma_f32_16x16x128_f8f6f4 v[20:23], a[128:135], v[136:143], v[20:23] cbsz:0 blgp:1\n\t" \
  "ds_read_b64_tr_b8 v[152:153], v167 offset:0\n\t" \
  "v_mfma_f32_16x16x128_f8f6f4 v[52:55], a[136:143], v[136:143], v[52:55] cbsz:0 blgp:1\n\t" \
  "ds_read_b64_tr_b8 v[154:155], v167 offset:1024\n\t" \
  "v_mfma_f32_16x16x128_f8f6f4 v[84:87], a[144:151], v[136:143], v[84:87] cbsz:0 blgp:1\n\t" \
  "ds_read_b64_tr_b8 v[156:157], v167 offset:8192\n\t" \
  "v_mfma_f32_16x16x128_f8f6f4 v[116:119], a[152:159], v[136:143], v[116:119] cbsz:0 blgp:1\n\t" \
  "ds_read_b64_tr_b8 v[158:159], v167 offset:9216\n\t" \
  "s_waitcnt lgkmcnt(4)\n\t" \
  "v_mfma_f32_16x16x128_f8f6f4 v[24:27], a[128:135], v[144:151], v[24:27] cbsz:0 blgp:1\n\t" \
  "s_barrier\n\t" \
  "ds_read_b64_tr_b8 v[128:129], v160 offset:16384\n\t" \
  "v_mfma_f32_16x16x128_f8f6f4 v[56:59], a[136:143], v[144:151], v[56:59] cbsz:0 blgp:1\n\t" \
  "ds_read_b64_tr_b8 v[130:131], v160 offset:17408\n\t" \
  "s_cmp_eq_u32 s9, -1\n\t" \
  "s_cbranch_scc0 L_au13_0\n\t" \
  "L_aur13_0:\n\t" \
  "v_mfma_f32_16x16x128_f8f6f4 v[88:91], a[144:151], v[144:151], v[88:91] cbsz:0 blgp:1\n\t" \
  "ds_read_b64_tr_b8 v[132:133], v160 offset:24576\n\t" \
  "v_mfma_f32_16x16x128_f8f6f4 v[120:123], a[152:159], v[144:151], v[120:123] cbsz:0 blgp:1\n\t" \
  "ds_read_b64_tr_b8 v[134:135], v160 offset:25600\n\t" \
  "s_waitcnt lgkmcnt(4)\n\t" \
  "v_mfma_f32_16x16x128_f8f6f4 v[28:31], a[128:135], v[152:159], v[28:31] cbsz:0 blgp:1\n\t" \
  "ds_read_b64_tr_b8 v[136:137], v161 offset:16384\n\t" \
  "v_mfma_f32_16x16x128_f8f6f4 v[60:63], a[136:143], v[152:159], v[60:63] cbsz:0 blgp:1\n\t" \
  "ds_read_b64_tr_b8 v[138:139], v161 offset:17408\n\t" \
  "v_mfma_f32_16x16x128_f8f6f4 v[92:95], a[144:151], v[152:159], v[92:95] cbsz:0 blgp:1\n\t" \
  "ds_read_b64_tr_b8 v[140:141], v161 offset:24576\n\t" \
  "v_mfma_f32_16x16x128_f8f6f4 v[124:127], a[152:159], v[152:159], v[124:127] cbsz:0 blgp:1\n\t" \
  "ds_read_b64_tr_b8 v[142:143], v161 offset:25600\n\t" \
  "s_waitcnt lgkmcnt(4)\n\t" \
  "v_mfma_f32_16x16x128_f8f6f4 v[0:3], a[160:167], v[128:135], v[0:3] cbsz:0 blgp:1\n\t" \
  "ds_read_b64_tr_b8 v[144:145], v162 offset:16384\n\t" \
  "v_mfma_f32_16x16x128_f8f6f4 v[32:35], a[168:175], v[128:135], v[32:35] cbsz:0 blgp:1\n\t" \
  "ds_read_b64_tr_b8 v[146:147], v162 offset:17408\n\t" \
  "v_mfma_f32_16x16x128_f8f6f4 v[64:67], a[176:183], v[128:135], v[64:67] cbsz:0 blgp:1\n\t" \
  "ds_read_b64_tr_b8 v[148:149], v162 offset:24576\n\t" \
  "v_mfma_f32_16x16x128_f8f6f4 v[96:99], a[184:191], v[128:135], v[96:99] cbsz:0 blgp:1\n\t" \
  "ds_read_b64_tr_b8 v[150:151], v162 offset:25600\n\t" \
  "s_waitcnt lgkmcnt(4)\n\t" \
  "v_mfma_f32_16x16x128_f8f6f4 v[4:7], a[160:167], v[136:143], v[4:7] cbsz:0 blgp:1\n\t" \
  "ds_read_b64_tr_b8 v[152:153], v163 offset:16384\n\t" \
  "v_mfma_f32_16x16x128_f8f6f4 v[36:39], a[168:175], v[136:143], v[36:39] cbsz:0 blgp:1\n\t" \
  "ds_read_b64_tr_b8 v[154:155], v163 offset:17408\n\t" \
  "s_cmp_eq_u32 s9, -1\n\t" \
  "s_cbranch_scc0 L_ap14_0\n\t" \
  "L_apr14_0:\n\t" \
  "s_add_u32 m0, s77, 114688\n\t" \
  "s_add_u32 s90, s70, 6291456\n\t" \
  "buffer_load_dwordx4 v186, s[40:43], s90 offen offset:0  lds\n\t" \
  "v_mfma_f32_16x16x128_f8f6f4 v[68:71], a[176:183], v[136:143], v[68:71] cbsz:0 blgp:1\n\t" \
  "ds_read_b64_tr_b8 v[156:157], v163 offset:24576\n\t" \
  "buffer_load_dwordx4 a[4:7], v188, s[36:39], s87 offen offset:1024 \n\t" \
  "v_mfma_f32_16x16x128_f8f6f4 v[100:103], a[184:191], v[136:143], v[100:103] cbsz:0 blgp:1\n\t" \
  "ds_read_b64_tr_b8 v[158:159], v163 offset:25600\n\t" \
  "s_add_u32 m0, s77, 114688\n\t" \
  "s_add_u32 s90, s70, 6291456\n\t" \
  "buffer_load_dwordx4 v187, s[40:43], s90 offen offset:1024  lds\n\t" \
  "s_waitcnt lgkmcnt(4)\n\t" \
  "v_mfma_f32_16x16x128_f8f6f4 v[8:11], a[160:167], v[144:151], v[8:11] cbsz:0 blgp:1\n\t" \
  "ds_read_b64_tr_b8 v[128:129], v164 offset:16384\n\t" \
  "buffer_load_dwordx4 a[8:11], v188, s[36:39], s87 offen offset:2048 \n\t" \
  "v_mfma_f32_16x16x128_f8f6f4 v[40:43], a[168:175], v[144:151], v[40:43] cbsz:0 blgp:1\n\t" \
  "ds_read_b64_tr_b8 v[130:131], v164 offset:17408\n\t" \
  "v_mfma_f32_16x16x128_f8f6f4 v[72:75], a[176:183], v[144:151], v[72:75] cbsz:0 blgp:1\n\t" \
  "ds_read_b64_tr_b8 v[132:133], v164 offset:24576\n\t" \
  "buffer_load_dwordx4 a[12:15], v188, s[36:39], s87 offen offset:3072 \n\t" \
  "v_mfma_f32_16x16x128_f8f6f4 v[104:107], a[184:191], v[144:151], v[104:107] cbsz:0 blgp:1\n\t" \
  "ds_read_b64_tr_b8 v[134:135], v164 offset:25600\n\t" \
  "s_waitcnt lgkmcnt(4)\n\t" \
  "v_mfma_f32_16x16x128_f8f6f4 v[12:15], a[160:167], v[152:159], v[12:15] cbsz:0 blgp:1\n\t" \
  "ds_read_b64_tr_b8 v[136:137], v165 offset:16384\n\t" \
  "buffer_load_dwordx4 a[16:19], v189, s[36:39], s87 offen offset:0 \n\t" \
  "v_mfma_f32_16x16x128_f8f6f4 v[44:47], a[168:175], v[152:159], v[44:47] cbsz:0 blgp:1\n\t" \
  "ds_read_b64_tr_b8 v[138:139], v165 offset:17408\n\t" \
  "v_mfma_f32_16x16x128_f8f6f4 v[76:79], a[176:183], v[152:159], v[76:79] cbsz:0 blgp:1\n\t" \
  "ds_read_b64_tr_b8 v[140:141], v165 offset:24576\n\t" \
  "buffer_load_dwordx4 a[20:23], v189, s[36:39], s87 offen offset:1024 \n\t" \
  "v_mfma_f32_16x16x128_f8f6f4 v[108:111], a[184:191], v[152:159], v[108:111] cbsz:0 blgp:1\n\t" \
  "ds_read_b64_tr_b8 v[142:143], v165 offset:25600\n\t" \
  "s_waitcnt lgkmcnt(4)\n\t" \
  "v_mfma_f32_16x16x128_f8f6f4 v[16:19], a[160:167], v[128:135], v[16:19] cbsz:0 blgp:1\n\t" \
  "ds_read_b64_tr_b8 v[144:145], v166 offset:16384\n\t" \
  "buffer_load_dwordx4 a[24:27], v189, s[36:39], s87 offen offset:2048 \n\t" \
  "v_mfma_f32_16x16x128_f8f6f4 v[48:51], a[168:175], v[128:135], v[48:51] cbsz:0 blgp:1\n\t" \
  "ds_read_b64_tr_b8 v[146:147], v166 offset:17408\n\t" \
  "v_mfma_f32_16x16x128_f8f6f4 v[80:83], a[176:183], v[128:135], v[80:83] cbsz:0 blgp:1\n\t" \
  "ds_read_b64_tr_b8 v[148:149], v166 offset:24576\n\t" \
  "buffer_load_dwordx4 a[28:31], v189, s[36:39], s87 offen offset:3072 \n\t" \
  "v_mfma_f32_16x16x128_f8f6f4 v[112:115], a[184:191], v[128:135], v[112:115] cbsz:0 blgp:1\n\t" \
  "ds_read_b64_tr_b8 v[150:151], v166 offset:25600\n\t" \
  "s_waitcnt lgkmcnt(4)\n\t" \
  "v_mfma_f32_16x16x128_f8f6f4 v[20:23], a[160:167], v[136:143], v[20:23] cbsz:0 blgp:1\n\t" \
  "ds_read_b64_tr_b8 v[152:153], v167 offset:16384\n\t" \
  "s_add_u32 m0, s77, 131072\n\t" \
  "s_add_u32 s90, s71, 6291456\n\t" \
  "buffer_load_dwordx4 v186, s[40:43], s90 offen offset:0  lds\n\t" \
  "v_mfma_f32_16x16x128_f8f6f4 v[52:55], a[168:175], v[136:143], v[52:55] cbsz:0 blgp:1\n\t" \
  "ds_read_b64_tr_b8 v[154:155], v167 offset:17408\n\t" \
  "s_add_u32 m0, s76, 147456\n\t" \
  "s_add_u32 s90, s72, 6291456\n\t" \
  "buffer_load_dwordx4 v184, s[40:43], s90 offen offset:0  lds\n\t" \
  "v_mfma_f32_16x16x128_f8f6f4 v[84:87], a[176:183], v[136:143], v[84:87] cbsz:0 blgp:1\n\t" \
  "ds_read_b64_tr_b8 v[156:157], v167 offset:24576\n\t" \
  "s_add_u32 m0, s77, 131072\n\t" \
  "s_add_u32 s90, s71, 6291456\n\t" \
  "buffer_load_dwordx4 v187, s[40:43], s90 offen offset:1024  lds\n\t" \
  "v_mfma_f32_16x16x128_f8f6f4 v[116:119], a[184:191], v[136:143], v[116:119] cbsz:0 blgp:1\n\t" \
  "ds_read_b64_tr_b8 v[158:159], v167 offset:25600\n\t" \
  "s_add_u32 m0, s76, 147456\n\t" \
  "s_add_u32 s90, s72, 6291456\n\t" \
  "buffer_load_dwordx4 v185, s[40:43], s90 offen offset:1024  lds\n\t" \
  "s_waitcnt lgkmcnt(4)\n\t" \
  "v_mfma_f32_16x16x128_f8f6f4 v[24:27], a[160:167], v[144:151], v[24:27] cbsz:0 blgp:1\n\t" \
  "s_barrier\n\t" \
  "ds_read_b64_tr_b8 v[128:129], v160 offset:32768\n\t" \
  "v_mfma_f32_16x16x128_f8f6f4 v[56:59], a[168:175], v[144:151], v[56:59] cbsz:0 blgp:1\n\t" \
  "ds_read_b64_tr_b8 v[130:131], v160 offset:33792\n\t" \
  "s_cmp_eq_u32 s9, -1\n\t" \
  "s_cbranch_scc0 L_au14_0\n\t" \
  "L_aur14_0:\n\t" \
  "s_add_u32 m0, s77, 147456\n\t" \
  "s_add_u32 s90, s72, 6291456\n\t" \
  "buffer_load_dwordx4 v186, s[40:43], s90 offen offset:0  lds\n\t" \
  "v_mfma_f32_16x16x128_f8f6f4 v[88:91], a[176:183], v[144:151], v[88:91] cbsz:0 blgp:1\n\t" \
  "ds_read_b64_tr_b8 v[132:133], v160 offset:40960\n\t" \
  "s_add_u32 m0, s76, 0\n\t" \
  "s_add_u32 s90, s73, 6291456\n\t" \
  "buffer_load_dwordx4 v184, s[40:43], s90 offen offset:0  lds\n\t" \
  "v_mfma_f32_16x16x128_f8f6f4 v[120:123], a[184:191], v[144:151], v[120:123] cbsz:0 blgp:1\n\t" \
  "ds_read_b64_tr_b8 v[134:135], v160 offset:41984\n\t" \
  "s_add_u32 m0, s77, 147456\n\t" \
  "s_add_u32 s90, s72, 6291456\n\t" \
  "buffer_load_dwordx4 v187, s[40:43], s90 offen offset:1024  lds\n\t" \
  "s_waitcnt lgkmcnt(4)\n\t" \
  "v_mfma_f32_16x16x128_f8f6f4 v[28:31], a[160:167], v[152:159], v[28:31] cbsz:0 blgp:1\n\t" \
  "ds_read_b64_tr_b8 v[136:137], v161 offset:32768\n\t" \
  "s_add_u32 m0, s76, 0\n\t" \
  "s_add_u32 s90, s73, 6291456\n\t" \
  "buffer_load_dwordx4 v185, s[40:43], s90 offen offset:1024  lds\n\t" \
  "v_mfma_f32_16x16x128_f8f6f4 v[60:63], a[168:175], v[152:159], v[60:63] cbsz:0 blgp:1\n\t" \
  "ds_read_b64_tr_b8 v[138:139], v161 offset:33792\n\t" \
  "s_bfe_u32 s84, s9, 262144\n\t" \
  "s_cmp_eq_u32 s84, 15\n\t" \
  "s_cbranch_scc0 L_q25_0\n\t" \
  "L_qr25_0:\n\t" \
  "s_mov_b32 s87, 1376256\n\t" \
  "buffer_load_dwordx4 a[32:35], v188, s[36:39], s87 offen offset:0 \n\t" \
  "v_mfma_f32_16x16x128_f8f6f4 v[92:95], a[176:183], v[152:159], v[92:95] cbsz:0 blgp:1\n\t" \
  "ds_read_b64_tr_b8 v[140:141], v161 offset:40960\n\t" \
  "s_add_u32 m0, s77, 0\n\t" \
  "s_add_u32 s90, s73, 6291456\n\t" \
  "buffer_load_dwordx4 v186, s[40:43], s90 offen offset:0  lds\n\t" \
  "v_mfma_f32_16x16x128_f8f6f4 v[124:127], a[184:191], v[152:159], v[124:127] cbsz:0 blgp:1\n\t" \
  "ds_read_b64_tr_b8 v[142:143], v161 offset:41984\n\t" \
  "buffer_load_dwordx4 a[36:39], v188, s[36:39], s87 offen offset:1024 \n\t" \
  "s_waitcnt lgkmcnt(4)\n\t" \
  "v_mfma_f32_16x16x128_f8f6f4 v[0:3], a[192:199], v[128:135], v[0:3] cbsz:0 blgp:1\n\t" \
  "ds_read_b64_tr_b8 v[144:145], v162 offset:32768\n\t" \
  "s_add_u32 m0, s77, 0\n\t" \
  "s_add_u32 s90, s73, 6291456\n\t" \
  "buffer_load_dwordx4 v187, s[40:43], s90 offen offset:1024  lds\n\t" \
  "v_mfma_f32_16x16x128_f8f6f4 v[32:35], a[200:207], v[128:135], v[32:35] cbsz:0 blgp:1\n\t" \
  "ds_read_b64_tr_b8 v[146:147], v162 offset:33792\n\t" \
  "buffer_load_dwordx4 a[40:43], v188, s[36:39], s87 offen offset:2048 \n\t" \
  "v_mfma_f32_16x16x128_f8f6f4 v[64:67], a[208:215], v[128:135], v[64:67] cbsz:0 blgp:1\n\t" \
  "ds_read_b64_tr_b8 v[148:149], v162 offset:40960\n\t" \
  "s_waitcnt vmcnt(44)\n\t" \
  "v_mfma_f32_16x16x128_f8f6f4 v[96:99], a[216:223], v[128:135], v[96:99] cbsz:0 blgp:1\n\t" \
  "ds_read_b64_tr_b8 v[150:151], v162 offset:41984\n\t" \
  "buffer_load_dwordx4 a[44:47], v188, s[36:39], s87 offen offset:3072 \n\t" \
  "s_waitcnt lgkmcnt(4)\n\t" \
  "v_mfma_f32_16x16x128_f8f6f4 v[4:7], a[192:199], v[136:143], v[4:7] cbsz:0 blgp:1\n\t" \
  "ds_read_b64_tr_b8 v[152:153], v163 offset:32768\n\t" \
  "v_mfma_f32_16x16x128_f8f6f4 v[36:39], a[200:207], v[136:143], v[36:39] cbsz:0 blgp:1\n\t" \
  "ds_read_b64_tr_b8 v[154:155], v163 offset:33792\n\t" \
  "s_cmp_eq_u32 s9, -1\n\t" \
  "s_cbranch_scc0 L_ap15_0\n\t" \
  "L_apr15_0:\n\t" \
  "buffer_load_dwordx4 a[48:51], v189, s[36:39], s87 offen offset:0 \n\t" \
  "v_mfma_f32_16x16x128_f8f6f4 v[68:71], a[208:215], v[136:143], v[68:71] cbsz:0 blgp:1\n\t" \
  "ds_read_b64_tr_b8 v[156:157], v163 offset:40960\n\t" \
  "v_mfma_f32_16x16x128_f8f6f4 v[100:103], a[216:223], v[136:143], v[100:103] cbsz:0 blgp:1\n\t" \
  "ds_read_b64_tr_b8 v[158:159], v163 offset:41984\n\t" \
  "buffer_load_dwordx4 a[52:55], v189, s[36:39], s87 offen offset:1024 \n\t" \
  "s_waitcnt lgkmcnt(4)\n\t" \
  "v_mfma_f32_16x16x128_f8f6f4 v[8:11], a[192:199], v[144:151], v[8:11] cbsz:0 blgp:1\n\t" \
  "ds_read_b64_tr_b8 v[128:129], v164 offset:32768\n\t" \
  "v_mfma_f32_16x16x128_f8f6f4 v[40:43], a[200:207], v[144:151], v[40:43] cbsz:0 blgp:1\n\t" \
  "ds_read_b64_tr_b8 v[130:131], v164 offset:33792\n\t" \
  "buffer_load_dwordx4 a[56:59], v189, s[36:39], s87 offen offset:2048 \n\t" \
  "v_mfma_f32_16x16x128_f8f6f4 v[72:75], a[208:215], v[144:151], v[72:75] cbsz:0 blgp:1\n\t" \
  "ds_read_b64_tr_b8 v[132:133], v164 offset:40960\n\t" \
  "v_mfma_f32_16x16x128_f8f6f4 v[104:107], a[216:223], v[144:151], v[104:107] cbsz:0 blgp:1\n\t" \
  "ds_read_b64_tr_b8 v[134:135], v164 offset:41984\n\t" \
  "buffer_load_dwordx4 a[60:63], v189, s[36:39], s87 offen offset:3072 \n\t" \
  "s_waitcnt lgkmcnt(4)\n\t" \
  "v_mfma_f32_16x16x128_f8f6f4 v[12:15], a[192:199], v[152:159], v[12:15] cbsz:0 blgp:1\n\t" \
  "ds_read_b64_tr_b8 v[136:137], v165 offset:32768\n\t" \
  "v_mfma_f32_16x16x128_f8f6f4 v[44:47], a[200:207], v[152:159], v[44:47] cbsz:0 blgp:1\n\t" \
  "ds_read_b64_tr_b8 v[138:139], v165 offset:33792\n\t" \
  "s_bfe_u32 s84, s9, 262148\n\t" \
  "s_cmp_eq_u32 s84, 15\n\t" \
  "s_cbranch_scc0 L_q26_0\n\t" \
  "L_qr26_0:\n\t" \
  "s_mov_b32 s87, 1441792\n\t" \
  "buffer_load_dwordx4 a[64:67], v188, s[36:39], s87 offen offset:0 \n\t" \
  "v_mfma_f32_16x16x128_f8f6f4 v[76:79], a[208:215], v[152:159], v[76:79] cbsz:0 blgp:1\n\t" \
  "ds_read_b64_tr_b8 v[140:141], v165 offset:40960\n\t" \
  "v_mfma_f32_16x16x128_f8f6f4 v[108:111], a[216:223], v[152:159], v[108:111] cbsz:0 blgp:1\n\t" \
  "ds_read_b64_tr_b8 v[142:143], v165 offset:41984\n\t" \
  "buffer_load_dwordx4 a[68:71], v188, s[36:39], s87 offen offset:1024 \n\t" \
  "s_waitcnt lgkmcnt(4)\n\t" \
  "v_mfma_f32_16x16x128_f8f6f4 v[16:19], a[192:199], v[128:135], v[16:19] cbsz:0 blgp:1\n\t" \
  "ds_read_b64_tr_b8 v[144:145], v166 offset:32768\n\t" \
  "v_mfma_f32_16x16x128_f8f6f4 v[48:51], a[200:207], v[128:135], v[48:51] cbsz:0 blgp:1\n\t" \
  "ds_read_b64_tr_b8 v[146:147], v166 offset:33792\n\t" \
  "buffer_load_dwordx4 a[72:75], v188, s[36:39], s87 offen offset:2048 \n\t" \
  "v_mfma_f32_16x16x128_f8f6f4 v[80:83], a[208:215], v[128:135], v[80:83] cbsz:0 blgp:1\n\t" \
  "ds_read_b64_tr_b8 v[148:149], v166 offset:40960\n\t" \
  "v_mfma_f32_16x16x128_f8f6f4 v[112:115], a[216:223], v[128:135], v[112:115] cbsz:0 blgp:1\n\t" \
  "ds_read_b64_tr_b8 v[150:151], v166 offset:41984\n\t" \
  "buffer_load_dwordx4 a[76:79], v188, s[36:39], s87 offen offset:3072 \n\t" \
  "s_waitcnt lgkmcnt(4)\n\t" \
  "v_mfma_f32_16x16x128_f8f6f4 v[20:23], a[192:199], v[136:143], v[20:23] cbsz:0 blgp:1\n\t" \
  "ds_read_b64_tr_b8 v[152:153], v167 offset:32768\n\t" \
  "v_mfma_f32_16x16x128_f8f6f4 v[52:55], a[200:207], v[136:143], v[52:55] cbsz:0 blgp:1\n\t" \
  "ds_read_b64_tr_b8 v[154:155], v167 offset:33792\n\t" \
  "buffer_load_dwordx4 a[80:83], v189, s[36:39], s87 offen offset:0 \n\t" \
  "v_mfma_f32_16x16x128_f8f6f4 v[84:87], a[208:215], v[136:143], v[84:87] cbsz:0 blgp:1\n\t" \
  "ds_read_b64_tr_b8 v[156:157], v167 offset:40960\n\t" \
  "v_mfma_f32_16x16x128_f8f6f4 v[116:119], a[216:223], v[136:143], v[116:119] cbsz:0 blgp:1\n\t" \
  "ds_read_b64_tr_b8 v[158:159], v167 offset:41984\n\t" \
  "buffer_load_dwordx4 a[84:87], v189, s[36:39], s87 offen offset:1024 \n\t" \
  "s_waitcnt lgkmcnt(4)\n\t" \
  "v_mfma_f32_16x16x128_f8f6f4 v[24:27], a[192:199], v[144:151], v[24:27] cbsz:0 blgp:1\n\t" \
  "s_barrier\n\t" \
  "ds_read_b64_tr_b8 v[128:129], v160 offset:49152\n\t" \
  "v_mfma_f32_16x16x128_f8f6f4 v[56:59], a[200:207], v[144:151], v[56:59] cbsz:0 blgp:1\n\t" \
  "ds_read_b64_tr_b8 v[130:131], v160 offset:50176\n\t" \
  "s_cmp_eq_u32 s9, -1\n\t" \
  "s_cbranch_scc0 L_au15_0\n\t" \
  "L_aur15_0:\n\t" \
  "s_add_u32 m0, s76, 16384\n\t" \
  "s_add_u32 s90, s74, 6291456\n\t" \
  "buffer_load_dwordx4 v184, s[40:43], s90 offen offset:0  lds\n\t" \
  "v_mfma_f32_16x16x128_f8f6f4 v[88:91], a[208:215], v[144:151], v[88:91] cbsz:0 blgp:1\n\t" \
  "ds_read_b64_tr_b8 v[132:133], v160 offset:57344\n\t" \
  "buffer_load_dwordx4 a[88:91], v189, s[36:39], s87 offen offset:2048 \n\t" \
  "v_mfma_f32_16x16x128_f8f6f4 v[120:123], a[216:223], v[144:151], v[120:123] cbsz:0 blgp:1\n\t" \
  "ds_read_b64_tr_b8 v[134:135], v160 offset:58368\n\t" \
  "s_add_u32 m0, s76, 16384\n\t" \
  "s_add_u32 s90, s74, 6291456\n\t" \
  "buffer_load_dwordx4 v185, s[40:43], s90 offen offset:1024  lds\n\t" \
  "s_waitcnt lgkmcnt(4)\n\t" \
  "v_mfma_f32_16x16x128_f8f6f4 v[28:31], a[192:199], v[152:159], v[28:31] cbsz:0 blgp:1\n\t" \
  "ds_read_b64_tr_b8 v[136:137], v161 offset:49152\n\t" \
  "buffer_load_dwordx4 a[92:95], v189, s[36:39], s87 offen offset:3072 \n\t" \
  "v_mfma_f32_16x16x128_f8f6f4 v[60:63], a[200:207], v[152:159], v[60:63] cbsz:0 blgp:1\n\t" \
  "ds_read_b64_tr_b8 v[138:139], v161 offset:50176\n\t" \
  "s_add_u32 m0, s77, 16384\n\t" \
  "s_add_u32 s90, s74, 6291456\n\t" \
  "buffer_load_dwordx4 v186, s[40:43], s90 offen offset:0  lds\n\t" \
  "v_mfma_f32_16x16x128_f8f6f4 v[92:95], a[208:215], v[152:159], v[92:95] cbsz:0 blgp:1\n\t" \
  "ds_read_b64_tr_b8 v[140:141], v161 offset:57344\n\t" \
  "s_bfe_u32 s84, s9, 262152\n\t" \
  "s_cmp_eq_u32 s84, 15\n\t" \
  "s_cbranch_scc0 L_q27_0\n\t" \
  "L_qr27_0:\n\t" \
  "s_mov_b32 s87, 1507328\n\t" \
  "buffer_load_dwordx4 a[96:99], v188, s[36:39], s87 offen offset:0 \n\t" \
  "v_mfma_f32_16x16x128_f8f6f4 v[124:127], a[216:223], v[152:159], v[124:127] cbsz:0 blgp:1\n\t" \
  "ds_read_b64_tr_b8 v[142:143], v161 offset:58368\n\t" \
  "s_add_u32 m0, s77, 16384\n\t" \
  "s_add_u32 s90, s74, 6291456\n\t" \
  "buffer_load_dwordx4 v187, s[40:43], s90 offen offset:1024  lds\n\t" \
  "s_waitcnt vmcnt(60) lgkmcnt(4)\n\t" \
  "v_mfma_f32_16x16x128_f8f6f4 v[0:3], a[224:231], v[128:135], v[0:3] cbsz:0 blgp:1\n\t" \
  "ds_read_b64_tr_b8 v[144:145], v162 offset:49152\n\t" \
  "buffer_load_dwordx4 a[100:103], v188, s[36:39], s87 offen offset:1024 \n\t" \
  "s_waitcnt vmcnt(59)\n\t" \
  "v_mfma_f32_16x16x128_f8f6f4 v[32:35], a[232:239], v[128:135], v[32:35] cbsz:0 blgp:1\n\t" \
  "ds_read_b64_tr_b8 v[146:147], v162 offset:50176\n\t" \
  "s_waitcnt vmcnt(57)\n\t" \
  "v_mfma_f32_16x16x128_f8f6f4 v[64:67], a[240:247], v[128:135], v[64:67] cbsz:0 blgp:1\n\t" \
  "ds_read_b64_tr_b8 v[148:149], v162 offset:57344\n\t" \
  "buffer_load_dwordx4 a[104:107], v188, s[36:39], s87 offen offset:2048 \n\t" \
  "s_waitcnt vmcnt(56)\n\t" \
  "v_mfma_f32_16x16x128_f8f6f4 v[96:99], a[248:255], v[128:135], v[96:99] cbsz:0 blgp:1\n\t" \
  "ds_read_b64_tr_b8 v[150:151], v162 offset:58368\n\t" \
  "s_waitcnt lgkmcnt(4)\n\t" \
  "v_mfma_f32_16x16x128_f8f6f4 v[4:7], a[224:231], v[136:143], v[4:7] cbsz:0 blgp:1\n\t" \
  "ds_read_b64_tr_b8 v[152:153], v163 offset:49152\n\t" \
  "buffer_load_dwordx4 a[108:111], v188, s[36:39], s87 offen offset:3072 \n\t" \
  "v_mfma_f32_16x16x128_f8f6f4 v[36:39], a[232:239], v[136:143], v[36:39] cbsz:0 blgp:1\n\t" \
  "ds_read_b64_tr_b8 v[154:155], v163 offset:50176\n\t" \
  "s_cmp_eq_u32 s9, -1\n\t" \
  "s_cbranch_scc0 L_ap16_0\n\t" \
  "L_apr16_0:\n\t" \
  "v_mfma_f32_16x16x128_f8f6f4 v[68:71], a[240:247], v[136:143], v[68:71] cbsz:0 blgp:1\n\t" \
  "ds_read_b64_tr_b8 v[156:157], v163 offset:57344\n\t" \
  "buffer_load_dwordx4 a[112:115], v189, s[36:39], s87 offen offset:0 \n\t" \
  "v_mfma_f32_16x16x128_f8f6f4 v[100:103], a[248:255], v[136:143], v[100:103] cbsz:0 blgp:1\n\t" \
  "ds_read_b64_tr_b8 v[158:159], v163 offset:58368\n\t" \
  "s_waitcnt lgkmcnt(4)\n\t" \
  "v_mfma_f32_16x16x128_f8f6f4 v[8:11], a[224:231], v[144:151], v[8:11] cbsz:0 blgp:1\n\t" \
  "ds_read_b64_tr_b8 v[128:129], v164 offset:49152\n\t" \
  "buffer_load_dwordx4 a[116:119], v189, s[36:39], s87 offen offset:1024 \n\t" \
  "v_mfma_f32_16x16x128_f8f6f4 v[40:43], a[232:239], v[144:151], v[40:43] cbsz:0 blgp:1\n\t" \
  "ds_read_b64_tr_b8 v[130:131], v164 offset:50176\n\t" \
  "v_mfma_f32_16x16x128_f8f6f4 v[72:75], a[240:247], v[144:151], v[72:75] cbsz:0 blgp:1\n\t" \
  "ds_read_b64_tr_b8 v[132:133], v164 offset:57344\n\t" \
  "buffer_load_dwordx4 a[120:123], v189, s[36:39], s87 offen offset:2048 \n\t" \
  "v_mfma_f32_16x16x128_f8f6f4 v[104:107], a[248:255], v[144:151], v[104:107] cbsz:0 blgp:1\n\t" \
  "ds_read_b64_tr_b8 v[134:135], v164 offset:58368\n\t" \
  "s_waitcnt lgkmcnt(4)\n\t" \
  "v_mfma_f32_16x16x128_f8f6f4 v[12:15], a[224:231], v[152:159], v[12:15] cbsz:0 blgp:1\n\t" \
  "ds_read_b64_tr_b8 v[136:137], v165 offset:49152\n\t" \
  "buffer_load_dwordx4 a[124:127], v189, s[36:39], s87 offen offset:3072 \n\t" \
  "v_mfma_f32_16x16x128_f8f6f4 v[44:47], a[232:239], v[152:159], v[44:47] cbsz:0 blgp:1\n\t" \
  "ds_read_b64_tr_b8 v[138:139], v165 offset:50176\n\t" \
  "v_mfma_f32_16x16x128_f8f6f4 v[76:79], a[240:247], v[152:159], v[76:79] cbsz:0 blgp:1\n\t" \
  "ds_read_b64_tr_b8 v[140:141], v165 offset:57344\n\t" \
  "s_bfe_u32 s84, s9, 262156\n\t" \
  "s_cmp_eq_u32 s84, 15\n\t" \
  "s_cbranch_scc0 L_q28_0\n\t" \
  "L_qr28_0:\n\t" \
  "s_mov_b32 s87, 1572864\n\t" \
  "buffer_load_dwordx4 a[128:131], v188, s[36:39], s87 offen offset:0 \n\t" \
  "v_mfma_f32_16x16x128_f8f6f4 v[108:111], a[248:255], v[152:159], v[108:111] cbsz:0 blgp:1\n\t" \
  "ds_read_b64_tr_b8 v[142:143], v165 offset:58368\n\t" \
  "s_waitcnt lgkmcnt(4)\n\t" \
  "v_mfma_f32_16x16x128_f8f6f4 v[16:19], a[224:231], v[128:135], v[16:19] cbsz:0 blgp:1\n\t" \
  "ds_read_b64_tr_b8 v[144:145], v166 offset:49152\n\t" \
  "buffer_load_dwordx4 a[132:135], v188, s[36:39], s87 offen offset:1024 \n\t" \
  "v_mfma_f32_16x16x128_f8f6f4 v[48:51], a[232:239], v[128:135], v[48:51] cbsz:0 blgp:1\n\t" \
  "ds_read_b64_tr_b8 v[146:147], v166 offset:50176\n\t" \
  "v_mfma_f32_16x16x128_f8f6f4 v[80:83], a[240:247], v[128:135], v[80:83] cbsz:0 blgp:1\n\t" \
  "ds_read_b64_tr_b8 v[148:149], v166 offset:57344\n\t" \
  "buffer_load_dwordx4 a[136:139], v188, s[36:39], s87 offen offset:2048 \n\t" \
  "v_mfma_f32_16x16x128_f8f6f4 v[112:115], a[248:255], v[128:135], v[112:115] cbsz:0 blgp:1\n\t" \
  "ds_read_b64_tr_b8 v[150:151], v166 offset:58368\n\t" \
  "s_waitcnt lgkmcnt(4)\n\t" \
  "v_mfma_f32_16x16x128_f8f6f4 v[20:23], a[224:231], v[136:143], v[20:23] cbsz:0 blgp:1\n\t" \
  "ds_read_b64_tr_b8 v[152:153], v167 offset:49152\n\t" \
  "buffer_load_dwordx4 a[140:143], v188, s[36:39], s87 offen offset:3072 \n\t" \
  "v_mfma_f32_16x16x128_f8f6f4 v[52:55], a[232:239], v[136:143], v[52:55] cbsz:0 blgp:1\n\t" \
  "ds_read_b64_tr_b8 v[154:155], v167 offset:50176\n\t" \
  "v_mfma_f32_16x16x128_f8f6f4 v[84:87], a[240:247], v[136:143], v[84:87] cbsz:0 blgp:1\n\t" \
  "ds_read_b64_tr_b8 v[156:157], v167 offset:57344\n\t" \
  "buffer_load_dwordx4 a[144:147], v189, s[36:39], s87 offen offset:0 \n\t" \
  "v_mfma_f32_16x16x128_f8f6f4 v[116:119], a[248:255], v[136:143], v[116:119] cbsz:0 blgp:1\n\t" \
  "ds_read_b64_tr_b8 v[158:159], v167 offset:58368\n\t" \
  "s_waitcnt lgkmcnt(4)\n\t" \
  "v_mfma_f32_16x16x128_f8f6f4 v[24:27], a[224:231], v[144:151], v[24:27] cbsz:0 blgp:1\n\t" \
  "s_waitcnt vmcnt(63)\n\t" \
  "s_barrier\n\t" \
  "ds_read_b64_tr_b8 v[128:129], v168 offset:0\n\t" \
  "v_mfma_f32_16x16x128_f8f6f4 v[56:59], a[232:239], v[144:151], v[56:59] cbsz:0 blgp:1\n\t" \
  "ds_read_b64_tr_b8 v[130:131], v168 offset:1024\n\t" \
  "s_cmp_eq_u32 s9, -1\n\t" \
  "s_cbranch_scc0 L_au16_0\n\t" \
  "L_aur16_0:\n\t" \
  "buffer_load_dwordx4 a[148:151], v189, s[36:39], s87 offen offset:1024 \n\t" \
  "v_mfma_f32_16x16x128_f8f6f4 v[88:91], a[240:247], v[144:151], v[88:91] cbsz:0 blgp:1\n\t" \
  "ds_read_b64_tr_b8 v[132:133], v168 offset:8192\n\t" \
  "v_mfma_f32_16x16x128_f8f6f4 v[120:123], a[248:255], v[144:151], v[120:123] cbsz:0 blgp:1\n\t" \
  "ds_read_b64_tr_b8 v[134:135], v168 offset:9216\n\t" \
  "buffer_load_dwordx4 a[152:155], v189, s[36:39], s87 offen offset:2048 \n\t" \
  "s_waitcnt lgkmcnt(4)\n\t" \
  "v_mfma_f32_16x16x128_f8f6f4 v[28:31], a[224:231], v[152:159], v[28:31] cbsz:0 blgp:1\n\t" \
  "s_waitcnt vmcnt(63)\n\t" \
  "s_barrier\n\t" \
  "ds_read_b64_tr_b8 v[136:137], v168 offset:16384\n\t" \
  "v_mfma_f32_16x16x128_f8f6f4 v[60:63], a[232:239], v[152:159], v[60:63] cbsz:0 blgp:1\n\t" \
  "ds_read_b64_tr_b8 v[138:139], v168 offset:17408\n\t" \
  "buffer_load_dwordx4 a[156:159], v189, s[36:39], s87 offen offset:3072 \n\t" \
  "v_mfma_f32_16x16x128_f8f6f4 v[92:95], a[240:247], v[152:159], v[92:95] cbsz:0 blgp:1\n\t" \
  "ds_read_b64_tr_b8 v[140:141], v168 offset:24576\n\t" \
  "v_mfma_f32_16x16x128_f8f6f4 v[124:127], a[248:255], v[152:159], v[124:127] cbsz:0 blgp:1\n\t" \
  "ds_read_b64_tr_b8 v[142:143], v168 offset:25600\n\t" \
  "s_bfe_u32 s84, s9, 262160\n\t" \
  "s_cmp_eq_u32 s84, 15\n\t" \
  "s_cbranch_scc0 L_q29_0\n\t" \
  "L_qr29_0:\n\t" \
  "s_mov_b32 s87, 1638400\n\t" \
  "buffer_load_dwordx4 a[160:163], v188, s[36:39], s87 offen offset:0 \n\t" \
  "s_waitcnt vmcnt(54) lgkmcnt(4)\n\t" \
  "v_mfma_f32_16x16x128_f8f6f4 v[0:3], a[0:7], v[128:135], v[0:3] cbsz:0 blgp:1\n\t" \
  "s_barrier\n\t" \
  "ds_read_b64_tr_b8 v[144:145], v168 offset:32768\n\t" \
  "s_waitcnt vmcnt(51)\n\t" \
  "v_mfma_f32_16x16x128_f8f6f4 v[32:35], a[8:15], v[128:135], v[32:35] cbsz:0 blgp:1\n\t" \
  "ds_read_b64_tr_b8 v[146:147], v168 offset:33792\n\t" \
  "buffer_load_dwordx4 a[164:167], v188, s[36:39], s87 offen offset:1024 \n\t" \
  "s_waitcnt vmcnt(50)\n\t" \
  "v_mfma_f32_16x16x128_f8f6f4 v[64:67], a[16:23], v[128:135], v[64:67] cbsz:0 blgp:1\n\t" \
  "ds_read_b64_tr_b8 v[148:149], v168 offset:40960\n\t" \
  "s_waitcnt vmcnt(48)\n\t" \
  "v_mfma_f32_16x16x128_f8f6f4 v[96:99], a[24:31], v[128:135], v[96:99] cbsz:0 blgp:1\n\t" \
  "ds_read_b64_tr_b8 v[150:151], v168 offset:41984\n\t" \
  "buffer_load_dwordx4 a[168:171], v188, s[36:39], s87 offen offset:2048 \n\t" \
  "s_waitcnt vmcnt(38) lgkmcnt(4)\n\t" \
  "v_mfma_f32_16x16x128_f8f6f4 v[0:3], a[32:39], v[136:143], v[0:3] cbsz:0 blgp:1\n\t" \
  "ds_read_b64_tr_b8 v[152:153], v169 offset:0\n\t" \
  "s_waitcnt vmcnt(35)\n\t" \
  "v_mfma_f32_16x16x128_f8f6f4 v[32:35], a[40:47], v[136:143], v[32:35] cbsz:0 blgp:1\n\t" \
  "ds_read_b64_tr_b8 v[154:155], v169 offset:1024\n\t" \
  "s_cmp_eq_u32 s9, -1\n\t" \
  "s_cbranch_scc0 L_ap17_0\n\t" \
  "L_apr17_0:\n\t" \
  "buffer_load_dwordx4 a[172:175], v188, s[36:39], s87 offen offset:3072 \n\t" \
  "s_waitcnt vmcnt(34)\n\t" \
  "v_mfma_f32_16x16x128_f8f6f4 v[64:67], a[48:55], v[136:143], v[64:67] cbsz:0 blgp:1\n\t" \
  "ds_read_b64_tr_b8 v[156:157], v169 offset:8192\n\t" \
  "s_waitcnt vmcnt(32)\n\t" \
  "v_mfma_f32_16x16x128_f8f6f4 v[96:99], a[56:63], v[136:143], v[96:99] cbsz:0 blgp:1\n\t" \
  "ds_read_b64_tr_b8 v[158:159], v169 offset:9216\n\t" \
  "buffer_load_dwordx4 a[176:179], v189, s[36:39], s87 offen offset:0 \n\t" \
  "s_waitcnt vmcnt(31) lgkmcnt(4)\n\t" \
  "v_mfma_f32_16x16x128_f8f6f4 v[0:3], a[64:71], v[144:151], v[0:3] cbsz:0 blgp:1\n\t" \
  "s_barrier\n\t" \
  "ds_read_b64_tr_b8 v[128:129], v168 offset:49152\n\t" \
  "s_waitcnt vmcnt(29)\n\t" \
  "v_mfma_f32_16x16x128_f8f6f4 v[32:35], a[72:79], v[144:151], v[32:35] cbsz:0 blgp:1\n\t" \
  "ds_read_b64_tr_b8 v[130:131], v168 offset:50176\n\t" \
  "buffer_load_dwordx4 a[180:183], v189, s[36:39], s87 offen offset:1024 \n\t" \
  "s_waitcnt vmcnt(28)\n\t" \
  "v_mfma_f32_16x16x128_f8f6f4 v[64:67], a[80:87], v[144:151], v[64:67] cbsz:0 blgp:1\n\t" \
  "ds_read_b64_tr_b8 v[132:133], v168 offset:57344\n\t" \
  "s_waitcnt vmcnt(24)\n\t" \
  "v_mfma_f32_16x16x128_f8f6f4 v[96:99], a[88:95], v[144:151], v[96:99] cbsz:0 blgp:1\n\t" \
  "ds_read_b64_tr_b8 v[134:135], v168 offset:58368\n\t" \
  "buffer_load_dwordx4 a[184:187], v189, s[36:39], s87 offen offset:2048 \n\t" \
  "s_waitcnt lgkmcnt(4)\n\t" \
  "v_mfma_f32_16x16x128_f8f6f4 v[4:7], a[0:7], v[152:159], v[4:7] cbsz:0 blgp:1\n\t" \
  "ds_read_b64_tr_b8 v[136:137], v169 offset:16384\n\t" \
  "v_mfma_f32_16x16x128_f8f6f4 v[36:39], a[8:15], v[152:159], v[36:39] cbsz:0 blgp:1\n\t" \
  "ds_read_b64_tr_b8 v[138:139], v169 offset:17408\n\t" \
  "buffer_load_dwordx4 a[188:191], v189, s[36:39], s87 offen offset:3072 \n\t" \
  "v_mfma_f32_16x16x128_f8f6f4 v[68:71], a[16:23], v[152:159], v[68:71] cbsz:0 blgp:1\n\t" \
  "ds_read_b64_tr_b8 v[140:141], v169 offset:24576\n\t" \
  "v_mfma_f32_16x16x128_f8f6f4 v[100:103], a[24:31], v[152:159], v[100:103] cbsz:0 blgp:1\n\t" \
  "ds_read_b64_tr_b8 v[142:143], v169 offset:25600\n\t" \
  "s_bfe_u32 s84, s9, 262164\n\t" \
  "s_cmp_eq_u32 s84, 15\n\t" \
  "s_cbranch_scc0 L_q30_0\n\t" \
  "L_qr30_0:\n\t" \
  "s_mov_b32 s87, 1703936\n\t" \
  "buffer_load_dwordx4 a[192:195], v188, s[36:39], s87 offen offset:0 \n\t" \
  "s_waitcnt vmcnt(23) lgkmcnt(4)\n\t" \
  "v_mfma_f32_16x16x128_f8f6f4 v[0:3], a[96:103], v[128:135], v[0:3] cbsz:0 blgp:1\n\t" \
  "s_barrier\n\t" \
  "ds_read_b64_tr_b8 v[144:145], v176 offset:0\n\t" \
  "s_waitcnt vmcnt(21)\n\t" \
  "v_mfma_f32_16x16x128_f8f6f4 v[32:35], a[104:111], v[128:135], v[32:35] cbsz:0 blgp:1\n\t" \
  "ds_read_b64_tr_b8 v[146:147], v176 offset:1024\n\t" \
  "buffer_load_dwordx4 a[196:199], v188, s[36:39], s87 offen offset:1024 \n\t" \
  "s_waitcnt vmcnt(20)\n\t" \
  "v_mfma_f32_16x16x128_f8f6f4 v[64:67], a[112:119], v[128:135], v[64:67] cbsz:0 blgp:1\n\t" \
  "ds_read_b64_tr_b8 v[148:149], v176 offset:8192\n\t" \
  "s_waitcnt vmcnt(18)\n\t" \
  "v_mfma_f32_16x16x128_f8f6f4 v[96:99], a[120:127], v[128:135], v[96:99] cbsz:0 blgp:1\n\t" \
  "ds_read_b64_tr_b8 v[150:151], v176 offset:9216\n\t" \
  "buffer_load_dwordx4 a[200:203], v188, s[36:39], s87 offen offset:2048 \n\t" \
  "s_waitcnt lgkmcnt(4)\n\t" \
  "v_mfma_f32_16x16x128_f8f6f4 v[4:7], a[32:39], v[136:143], v[4:7] cbsz:0 blgp:1\n\t" \
  "ds_read_b64_tr_b8 v[152:153], v169 offset:32768\n\t" \
  "v_mfma_f32_16x16x128_f8f6f4 v[36:39], a[40:47], v[136:143], v[36:39] cbsz:0 blgp:1\n\t" \
  "ds_read_b64_tr_b8 v[154:155], v169 offset:33792\n\t" \
  "buffer_load_dwordx4 a[204:207], v188, s[36:39], s87 offen offset:3072 \n\t" \
  "v_mfma_f32_16x16x128_f8f6f4 v[68:71], a[48:55], v[136:143], v[68:71] cbsz:0 blgp:1\n\t" \
  "ds_read_b64_tr_b8 v[156:157], v169 offset:40960\n\t" \
  "v_mfma_f32_16x16x128_f8f6f4 v[100:103], a[56:63], v[136:143], v[100:103] cbsz:0 blgp:1\n\t" \
  "ds_read_b64_tr_b8 v[158:159], v169 offset:41984\n\t" \
  "buffer_load_dwordx4 a[208:211], v189, s[36:39], s87 offen offset:0 \n\t" \
  "s_waitcnt vmcnt(19) lgkmcnt(4)\n\t" \
  "v_mfma_f32_16x16x128_f8f6f4 v[0:3], a[128:135], v[144:151], v[0:3] cbsz:0 blgp:1\n\t" \
  "ds_read_b64_tr_b8 v[128:129], v170 offset:0\n\t" \
  "s_waitcnt vmcnt(17)\n\t" \
  "v_mfma_f32_16x16x128_f8f6f4 v[32:35], a[136:143], v[144:151], v[32:35] cbsz:0 blgp:1\n\t" \
  "ds_read_b64_tr_b8 v[130:131], v170 offset:1024\n\t" \
  "s_cmp_eq_u32 s9, -1\n\t" \
  "s_cbranch_scc0 L_au17_0\n\t" \
  "L_aur17_0:\n\t" \
  "buffer_load_dwordx4 a[212:215], v189, s[36:39], s87 offen offset:1024 \n\t" \
  "s_waitcnt vmcnt(16)\n\t" \
  "v_mfma_f32_16x16x128_f8f6f4 v[64:67], a[144:151], v[144:151], v[64:67] cbsz:0 blgp:1\n\t" \
  "ds_read_b64_tr_b8 v[132:133], v170 offset:8192\n\t" \
  "s_waitcnt vmcnt(14)\n\t" \
  "v_mfma_f32_16x16x128_f8f6f4 v[96:99], a[152:159], v[144:151], v[96:99] cbsz:0 blgp:1\n\t" \
  "ds_read_b64_tr_b8 v[134:135], v170 offset:9216\n\t" \
  "buffer_load_dwordx4 a[216:219], v189, s[36:39], s87 offen offset:2048 \n\t" \
  "s_waitcnt lgkmcnt(4)\n\t" \
  "v_mfma_f32_16x16x128_f8f6f4 v[4:7], a[64:71], v[152:159], v[4:7] cbsz:0 blgp:1\n\t" \
  "s_barrier\n\t" \
  "ds_read_b64_tr_b8 v[136:137], v176 offset:16384\n\t" \
  "v_mfma_f32_16x16x128_f8f6f4 v[36:39], a[72:79], v[152:159], v[36:39] cbsz:0 blgp:1\n\t" \
  "ds_read_b64_tr_b8 v[138:139], v176 offset:17408\n\t" \
  "buffer_load_dwordx4 a[220:223], v189, s[36:39], s87 offen offset:3072 \n\t" \
  "v_mfma_f32_16x16x128_f8f6f4 v[68:71], a[80:87], v[152:159], v[68:71] cbsz:0 blgp:1\n\t" \
  "ds_read_b64_tr_b8 v[140:141], v176 offset:24576\n\t" \
  "v_mfma_f32_16x16x128_f8f6f4 v[100:103], a[88:95], v[152:159], v[100:103] cbsz:0 blgp:1\n\t" \
  "ds_read_b64_tr_b8 v[142:143], v176 offset:25600\n\t" \
  "s_bfe_u32 s84, s9, 262168\n\t" \
  "s_cmp_eq_u32 s84, 15\n\t" \
  "s_cbranch_scc0 L_q31_0\n\t" \
  "L_qr31_0:\n\t" \
  "s_mov_b32 s87, 1769472\n\t" \
  "buffer_load_dwordx4 a[224:227], v188, s[36:39], s87 offen offset:0 \n\t" \
  "s_waitcnt lgkmcnt(4)\n\t" \
  "v_mfma_f32_16x16x128_f8f6f4 v[8:11], a[0:7], v[128:135], v[8:11] cbsz:0 blgp:1\n\t" \
  "ds_read_b64_tr_b8 v[144:145], v169 offset:49152\n\t" \
  "v_mfma_f32_16x16x128_f8f6f4 v[40:43], a[8:15], v[128:135], v[40:43] cbsz:0 blgp:1\n\t" \
  "ds_read_b64_tr_b8 v[146:147], v169 offset:50176\n\t" \
  "buffer_load_dwordx4 a[228:231], v188, s[36:39], s87 offen offset:1024 \n\t" \
  "v_mfma_f32_16x16x128_f8f6f4 v[72:75], a[16:23], v[128:135], v[72:75] cbsz:0 blgp:1\n\t" \
  "ds_read_b64_tr_b8 v[148:149], v169 offset:57344\n\t" \
  "v_mfma_f32_16x16x128_f8f6f4 v[104:107], a[24:31], v[128:135], v[104:107] cbsz:0 blgp:1\n\t" \
  "ds_read_b64_tr_b8 v[150:151], v169 offset:58368\n\t" \
  "buffer_load_dwordx4 a[232:235], v188, s[36:39], s87 offen offset:2048 \n\t" \
  "s_waitcnt vmcnt(17) lgkmcnt(4)\n\t" \
  "v_mfma_f32_16x16x128_f8f6f4 v[0:3], a[160:167], v[136:143], v[0:3] cbsz:0 blgp:1\n\t" \
  "ds_read_b64_tr_b8 v[152:153], v170 offset:16384\n\t" \
  "s_waitcnt vmcnt(15)\n\t" \
  "v_mfma_f32_16x16x128_f8f6f4 v[32:35], a[168:175], v[136:143], v[32:35] cbsz:0 blgp:1\n\t" \
  "ds_read_b64_tr_b8 v[154:155], v170 offset:17408\n\t" \
  "buffer_load_dwordx4 a[236:239], v188, s[36:39], s87 offen offset:3072 \n\t" \
  "s_waitcnt vmcnt(14)\n\t" \
  "v_mfma_f32_16x16x128_f8f6f4 v[64:67], a[176:183], v[136:143], v[64:67] cbsz:0 blgp:1\n\t" \
  "ds_read_b64_tr_b8 v[156:157], v170 offset:24576\n\t" \
  "s_waitcnt vmcnt(12)\n\t" \
  "v_mfma_f32_16x16x128_f8f6f4 v[96:99], a[184:191], v[136:143], v[96:99] cbsz:0 blgp:1\n\t" \
  "ds_read_b64_tr_b8 v[158:159], v170 offset:25600\n\t" \
  "buffer_load_dwordx4 a[240:243], v189, s[36:39], s87 offen offset:0 \n\t" \
  "s_waitcnt lgkmcnt(4)\n\t" \
  "v_mfma_f32_16x16x128_f8f6f4 v[4:7], a[96:103], v[144:151], v[4:7] cbsz:0 blgp:1\n\t" \
  "s_barrier\n\t" \
  "ds_read_b64_tr_b8 v[128:129], v160 offset:0\n\t" \
  "v_mfma_f32_16x16x128_f8f6f4 v[36:39], a[104:111], v[144:151], v[36:39] cbsz:0 blgp:1\n\t" \
  "ds_read_b64_tr_b8 v[130:131], v160 offset:1024\n\t" \
  "buffer_load_dwordx4 a[244:247], v189, s[36:39], s87 offen offset:1024 \n\t" \
  "v_mfma_f32_16x16x128_f8f6f4 v[68:71], a[112:119], v[144:151], v[68:71] cbsz:0 blgp:1\n\t" \
  "ds_read_b64_tr_b8 v[132:133], v160 offset:8192\n\t" \
  "v_mfma_f32_16x16x128_f8f6f4 v[100:103], a[120:127], v[144:151], v[100:103] cbsz:0 blgp:1\n\t" \
  "ds_read_b64_tr_b8 v[134:135], v160 offset:9216\n\t" \
  "buffer_load_dwordx4 a[248:251], v189, s[36:39], s87 offen offset:2048 \n\t" \
  "s_waitcnt lgkmcnt(4)\n\t" \
  "v_mfma_f32_16x16x128_f8f6f4 v[8:11], a[32:39], v[152:159], v[8:11] cbsz:0 blgp:1\n\t" \
  "ds_read_b64_tr_b8 v[136:137], v177 offset:0\n\t" \
  "v_mfma_f32_16x16x128_f8f6f4 v[40:43], a[40:47], v[152:159], v[40:43] cbsz:0 blgp:1\n\t" \
  "ds_read_b64_tr_b8 v[138:139], v177 offset:1024\n\t" \
  "buffer_load_dwordx4 a[252:255], v189, s[36:39], s87 offen offset:3072 \n\t" \
  "v_mfma_f32_16x16x128_f8f6f4 v[72:75], a[48:55], v[152:159], v[72:75] cbsz:0 blgp:1\n\t" \
  "ds_read_b64_tr_b8 v[140:141], v177 offset:8192\n\t" \
  "v_mfma_f32_16x16x128_f8f6f4 v[104:107], a[56:63], v[152:159], v[104:107] cbsz:0 blgp:1\n\t" \
  "ds_read_b64_tr_b8 v[142:143], v177 offset:9216\n\t" \
  "s_waitcnt vmcnt(14) lgkmcnt(4)\n\t" \
  "v_mfma_f32_16x16x128_f8f6f4 v[0:3], a[192:199], v[128:135], v[0:3] cbsz:0 blgp:1\n\t" \
  "ds_read_b64_tr_b8 v[144:145], v170 offset:32768\n\t" \
  "s_waitcnt vmcnt(12)\n\t" \
  "v_mfma_f32_16x16x128_f8f6f4 v[32:35], a[200:207], v[128:135], v[32:35] cbsz:0 blgp:1\n\t" \
  "ds_read_b64_tr_b8 v[146:147], v170 offset:33792\n\t" \
  "s_waitcnt vmcnt(10)\n\t" \
  "v_mfma_f32_16x16x128_f8f6f4 v[64:67], a[208:215], v[128:135], v[64:67] cbsz:0 blgp:1\n\t" \
  "ds_read_b64_tr_b8 v[148:149], v170 offset:40960\n\t" \
  "s_waitcnt vmcnt(8)\n\t" \
  "v_mfma_f32_16x16x128_f8f6f4 v[96:99], a[216:223], v[128:135], v[96:99] cbsz:0 blgp:1\n\t" \
  "ds_read_b64_tr_b8 v[150:151], v170 offset:41984\n\t" \
  "s_waitcnt lgkmcnt(4)\n\t" \
  "v_mfma_f32_16x16x128_f8f6f4 v[4:7], a[128:135], v[136:143], v[4:7] cbsz:0 blgp:1\n\t" \
  "ds_read_b64_tr_b8 v[152:153], v171 offset:0\n\t" \
  "v_mfma_f32_16x16x128_f8f6f4 v[36:39], a[136:143], v[136:143], v[36:39] cbsz:0 blgp:1\n\t" \
  "ds_read_b64_tr_b8 v[154:155], v171 offset:1024\n\t" \
  "v_mfma_f32_16x16x128_f8f6f4 v[68:71], a[144:151], v[136:143], v[68:71] cbsz:0 blgp:1\n\t" \
  "ds_read_b64_tr_b8 v[156:157], v171 offset:8192\n\t" \
  "v_mfma_f32_16x16x128_f8f6f4 v[100:103], a[152:159], v[136:143], v[100:103] cbsz:0 blgp:1\n\t" \
  "ds_read_b64_tr_b8 v[158:159], v171 offset:9216\n\t" \
  "s_waitcnt lgkmcnt(4)\n\t" \
  "v_mfma_f32_16x16x128_f8f6f4 v[8:11], a[64:71], v[144:151], v[8:11] cbsz:0 blgp:1\n\t" \
  "s_barrier\n\t" \
  "ds_read_b64_tr_b8 v[128:129], v160 offset:16384\n\t" \
  "v_mfma_f32_16x16x128_f8f6f4 v[40:43], a[72:79], v[144:151], v[40:43] cbsz:0 blgp:1\n\t" \
  "ds_read_b64_tr_b8 v[130:131], v160 offset:17408\n\t" \
  "v_mfma_f32_16x16x128_f8f6f4 v[72:75], a[80:87], v[144:151], v[72:75] cbsz:0 blgp:1\n\t" \
  "ds_read_b64_tr_b8 v[132:133], v160 offset:24576\n\t" \
  "v_mfma_f32_16x16x128_f8f6f4 v[104:107], a[88:95], v[144:151], v[104:107] cbsz:0 blgp:1\n\t" \
  "ds_read_b64_tr_b8 v[134:135], v160 offset:25600\n\t" \
  "s_waitcnt lgkmcnt(4)\n\t" \
  "v_mfma_f32_16x16x128_f8f6f4 v[12:15], a[0:7], v[152:159], v[12:15] cbsz:0 blgp:1\n\t" \
  "ds_read_b64_tr_b8 v[136:137], v177 offset:16384\n\t" \
  "v_mfma_f32_16x16x128_f8f6f4 v[44:47], a[8:15], v[152:159], v[44:47] cbsz:0 blgp:1\n\t" \
  "ds_read_b64_tr_b8 v[138:139], v177 offset:17408\n\t" \
  "v_mfma_f32_16x16x128_f8f6f4 v[76:79], a[16:23], v[152:159], v[76:79] cbsz:0 blgp:1\n\t" \
  "ds_read_b64_tr_b8 v[140:141], v177 offset:24576\n\t" \
  "v_mfma_f32_16x16x128_f8f6f4 v[108:111], a[24:31], v[152:159], v[108:111] cbsz:0 blgp:1\n\t" \
  "ds_read_b64_tr_b8 v[142:143], v177 offset:25600\n\t" \
  "s_waitcnt vmcnt(6) lgkmcnt(4)\n\t" \
  "v_mfma_f32_16x16x128_f8f6f4 v[0:3], a[224:231], v[128:135], v[0:3] cbsz:0 blgp:1\n\t" \
  "ds_read_b64_tr_b8 v[144:145], v170 offset:49152\n\t" \
  "s_waitcnt vmcnt(4)\n\t" \
  "v_mfma_f32_16x16x128_f8f6f4 v[32:35], a[232:239], v[128:135], v[32:35] cbsz:0 blgp:1\n\t" \
  "ds_read_b64_tr_b8 v[146:147], v170 offset:50176\n\t" \
  "s_waitcnt vmcnt(2)\n\t" \
  "v_mfma_f32_16x16x128_f8f6f4 v[64:67], a[240:247], v[128:135], v[64:67] cbsz:0 blgp:1\n\t" \
  "ds_read_b64_tr_b8 v[148:149], v170 offset:57344\n\t" \
  "s_waitcnt vmcnt(0)\n\t" \
  "v_mfma_f32_16x16x128_f8f6f4 v[96:99], a[248:255], v[128:135], v[96:99] cbsz:0 blgp:1\n\t" \
  "ds_read_b64_tr_b8 v[150:151], v170 offset:58368\n\t" \
  "s_waitcnt lgkmcnt(4)\n\t" \
  "v_mfma_f32_16x16x128_f8f6f4 v[4:7], a[160:167], v[136:143], v[4:7] cbsz:0 blgp:1\n\t" \
  "ds_read_b64_tr_b8 v[152:153], v171 offset:16384\n\t" \
  "v_mfma_f32_16x16x128_f8f6f4 v[36:39], a[168:175], v[136:143], v[36:39] cbsz:0 blgp:1\n\t" \
  "ds_read_b64_tr_b8 v[154:155], v171 offset:17408\n\t" \
  "v_mfma_f32_16x16x128_f8f6f4 v[68:71], a[176:183], v[136:143], v[68:71] cbsz:0 blgp:1\n\t" \
  "ds_read_b64_tr_b8 v[156:157], v171 offset:24576\n\t" \
  "v_mul_f32 v0, s81, v0\n\t" \
  "v_mfma_f32_16x16x128_f8f6f4 v[100:103], a[184:191], v[136:143], v[100:103] cbsz:0 blgp:1\n\t" \
  "ds_read_b64_tr_b8 v[158:159], v171 offset:25600\n\t" \
  "s_mov_b32 s84, 0\n\t" \
  "buffer_store_dword v0, v194, s[44:47], s84 offen offset:0 sc1\n\t" \
  "v_mul_f32 v1, s81, v1\n\t" \
  "s_waitcnt lgkmcnt(4)\n\t" \
  "v_mfma_f32_16x16x128_f8f6f4 v[8:11], a[96:103], v[144:151], v[8:11] cbsz:0 blgp:1\n\t" \
  "ds_read_b64_tr_b8 v[128:129], v161 offset:0\n\t" \
  "s_mov_b32 s84, 65536\n\t" \
  "buffer_store_dword v1, v194, s[44:47], s84 offen offset:0 sc1\n\t" \
  "v_mul_f32 v2, s81, v2\n\t" \
  "v_mfma_f32_16x16x128_f8f6f4 v[40:43], a[104:111], v[144:151], v[40:43] cbsz:0 blgp:1\n\t" \
  "ds_read_b64_tr_b8 v[130:131], v161 offset:1024\n\t" \
  "s_mov_b32 s84, 131072\n\t" \
  "buffer_store_dword v2, v194, s[44:47], s84 offen offset:0 sc1\n\t" \
  "v_mul_f32 v3, s81, v3\n\t" \
  "v_mfma_f32_16x16x128_f8f6f4 v[72:75], a[112:119], v[144:151], v[72:75] cbsz:0 blgp:1\n\t" \
  "ds_read_b64_tr_b8 v[132:133], v161 offset:8192\n\t" \
  "s_mov_b32 s84, 196608\n\t" \
  "buffer_store_dword v3, v194, s[44:47], s84 offen offset:0 sc1\n\t" \
  "v_mfma_f32_16x16x128_f8f6f4 v[104:107], a[120:127], v[144:151], v[104:107] cbsz:0 blgp:1\n\t" \
  "ds_read_b64_tr_b8 v[134:135], v161 offset:9216\n\t" \
  "s_waitcnt lgkmcnt(4)\n\t" \
  "v_mfma_f32_16x16x128_f8f6f4 v[12:15], a[32:39], v[152:159], v[12:15] cbsz:0 blgp:1\n\t" \
  "ds_read_b64_tr_b8 v[136:137], v178 offset:0\n\t" \
  "v_mul_f32 v32, s81, v32\n\t" \
  "v_mfma_f32_16x16x128_f8f6f4 v[44:47], a[40:47], v[152:159], v[44:47] cbsz:0 blgp:1\n\t" \
  "ds_read_b64_tr_b8 v[138:139], v178 offset:1024\n\t" \
  "s_mov_b32 s84, 1048576\n\t" \
  "buffer_store_dword v32, v194, s[44:47], s84 offen offset:0 sc1\n\t" \
  "v_mul_f32 v33, s81, v33\n\t" \
  "v_mfma_f32_16x16x128_f8f6f4 v[76:79], a[48:55], v[152:159], v[76:79] cbsz:0 blgp:1\n\t" \
  "ds_read_b64_tr_b8 v[140:141], v178 offset:8192\n\t" \
  "s_mov_b32 s84, 1114112\n\t" \
  "buffer_store_dword v33, v194, s[44:47], s84 offen offset:0 sc1\n\t" \
  "v_mul_f32 v34, s81, v34\n\t" \
  "v_mfma_f32_16x16x128_f8f6f4 v[108:111], a[56:63], v[152:159], v[108:111] cbsz:0 blgp:1\n\t" \
  "ds_read_b64_tr_b8 v[142:143], v178 offset:9216\n\t" \
  "s_mov_b32 s84, 1179648\n\t" \
  "buffer_store_dword v34, v194, s[44:47], s84 offen offset:0 sc1\n\t" \
  "v_mul_f32 v35, s81, v35\n\t" \
  "s_waitcnt lgkmcnt(4)\n\t" \
  "v_mfma_f32_16x16x128_f8f6f4 v[4:7], a[192:199], v[128:135], v[4:7] cbsz:0 blgp:1\n\t" \
  "ds_read_b64_tr_b8 v[144:145], v171 offset:32768\n\t" \
  "s_mov_b32 s84, 1245184\n\t" \
  "buffer_store_dword v35, v194, s[44:47], s84 offen offset:0 sc1\n\t" \
  "v_mfma_f32_16x16x128_f8f6f4 v[36:39], a[200:207], v[128:135], v[36:39] cbsz:0 blgp:1\n\t" \
  "ds_read_b64_tr_b8 v[146:147], v171 offset:33792\n\t" \
  "v_mfma_f32_16x16x128_f8f6f4 v[68:71], a[208:215], v[128:135], v[68:71] cbsz:0 blgp:1\n\t" \
  "ds_read_b64_tr_b8 v[148:149], v171 offset:40960\n\t" \
  "v_mul_f32 v64, s81, v64\n\t" \
  "v_mfma_f32_16x16x128_f8f6f4 v[100:103], a[216:223], v[128:135], v[100:103] cbsz:0 blgp:1\n\t" \
  "ds_read_b64_tr_b8 v[150:151], v171 offset:41984\n\t" \
  "s_mov_b32 s84, 2097152\n\t" \
  "buffer_store_dword v64, v194, s[44:47], s84 offen offset:0 sc1\n\t" \
  "v_mul_f32 v65, s81, v65\n\t" \
  "s_waitcnt lgkmcnt(4)\n\t" \
  "v_mfma_f32_16x16x128_f8f6f4 v[8:11], a[128:135], v[136:143], v[8:11] cbsz:0 blgp:1\n\t" \
  "ds_read_b64_tr_b8 v[152:153], v172 offset:0\n\t" \
  "s_mov_b32 s84, 2162688\n\t" \
  "buffer_store_dword v65, v194, s[44:47], s84 offen offset:0 sc1\n\t" \
  "v_mul_f32 v66, s81, v66\n\t" \
  "v_mfma_f32_16x16x128_f8f6f4 v[40:43], a[136:143], v[136:143], v[40:43] cbsz:0 blgp:1\n\t" \
  "ds_read_b64_tr_b8 v[154:155], v172 offset:1024\n\t" \
  "s_mov_b32 s84, 2228224\n\t" \
  "buffer_store_dword v66, v194, s[44:47], s84 offen offset:0 sc1\n\t" \
  "v_mul_f32 v67, s81, v67\n\t" \
  "v_mfma_f32_16x16x128_f8f6f4 v[72:75], a[144:151], v[136:143], v[72:75] cbsz:0 blgp:1\n\t" \
  "ds_read_b64_tr_b8 v[156:157], v172 offset:8192\n\t" \
  "s_mov_b32 s84, 2293760\n\t" \
  "buffer_store_dword v67, v194, s[44:47], s84 offen offset:0 sc1\n\t" \
  "v_mfma_f32_16x16x128_f8f6f4 v[104:107], a[152:159], v[136:143], v[104:107] cbsz:0 blgp:1\n\t" \
  "ds_read_b64_tr_b8 v[158:159], v172 offset:9216\n\t" \
  "s_waitcnt lgkmcnt(4)\n\t" \
  "v_mfma_f32_16x16x128_f8f6f4 v[12:15], a[64:71], v[144:151], v[12:15] cbsz:0 blgp:1\n\t" \
  "ds_read_b64_tr_b8 v[128:129], v161 offset:16384\n\t" \
  "v_mul_f32 v96, s81, v96\n\t" \
  "v_mfma_f32_16x16x128_f8f6f4 v[44:47], a[72:79], v[144:151], v[44:47] cbsz:0 blgp:1\n\t" \
  "ds_read_b64_tr_b8 v[130:131], v161 offset:17408\n\t" \
  "s_mov_b32 s84, 3145728\n\t" \
  "buffer_store_dword v96, v194, s[44:47], s84 offen offset:0 sc1\n\t" \
  "v_mul_f32 v97, s81, v97\n\t" \
  "v_mfma_f32_16x16x128_f8f6f4 v[76:79], a[80:87], v[144:151], v[76:79] cbsz:0 blgp:1\n\t" \
  "ds_read_b64_tr_b8 v[132:133], v161 offset:24576\n\t" \
  "s_mov_b32 s84, 3211264\n\t" \
  "buffer_store_dword v97, v194, s[44:47], s84 offen offset:0 sc1\n\t" \
  "v_mul_f32 v98, s81, v98\n\t" \
  "v_mfma_f32_16x16x128_f8f6f4 v[108:111], a[88:95], v[144:151], v[108:111] cbsz:0 blgp:1\n\t" \
  "ds_read_b64_tr_b8 v[134:135], v161 offset:25600\n\t" \
  "s_mov_b32 s84, 3276800\n\t" \
  "buffer_store_dword v98, v194, s[44:47], s84 offen offset:0 sc1\n\t" \
  "v_mul_f32 v99, s81, v99\n\t" \
  "s_waitcnt lgkmcnt(4)\n\t" \
  "v_mfma_f32_16x16x128_f8f6f4 v[16:19], a[0:7], v[152:159], v[16:19] cbsz:0 blgp:1\n\t" \
  "ds_read_b64_tr_b8 v[136:137], v178 offset:16384\n\t" \
  "s_mov_b32 s84, 3342336\n\t" \
  "buffer_store_dword v99, v194, s[44:47], s84 offen offset:0 sc1\n\t" \
  "v_mfma_f32_16x16x128_f8f6f4 v[48:51], a[8:15], v[152:159], v[48:51] cbsz:0 blgp:1\n\t" \
  "ds_read_b64_tr_b8 v[138:139], v178 offset:17408\n\t" \
  "v_mfma_f32_16x16x128_f8f6f4 v[80:83], a[16:23], v[152:159], v[80:83] cbsz:0 blgp:1\n\t" \
  "ds_read_b64_tr_b8 v[140:141], v178 offset:24576\n\t" \
  "v_mfma_f32_16x16x128_f8f6f4 v[112:115], a[24:31], v[152:159], v[112:115] cbsz:0 blgp:1\n\t" \
  "ds_read_b64_tr_b8 v[142:143], v178 offset:25600\n\t" \
  "s_waitcnt lgkmcnt(4)\n\t" \
  "v_mfma_f32_16x16x128_f8f6f4 v[4:7], a[224:231], v[128:135], v[4:7] cbsz:0 blgp:1\n\t" \
  "ds_read_b64_tr_b8 v[144:145], v171 offset:49152\n\t" \
  "v_mfma_f32_16x16x128_f8f6f4 v[36:39], a[232:239], v[128:135], v[36:39] cbsz:0 blgp:1\n\t" \
  "ds_read_b64_tr_b8 v[146:147], v171 offset:50176\n\t" \
  "v_mfma_f32_16x16x128_f8f6f4 v[68:71], a[240:247], v[128:135], v[68:71] cbsz:0 blgp:1\n\t" \
  "ds_read_b64_tr_b8 v[148:149], v171 offset:57344\n\t" \
  "v_mfma_f32_16x16x128_f8f6f4 v[100:103], a[248:255], v[128:135], v[100:103] cbsz:0 blgp:1\n\t" \
  "ds_read_b64_tr_b8 v[150:151], v171 offset:58368\n\t" \
  "s_waitcnt lgkmcnt(4)\n\t" \
  "v_mfma_f32_16x16x128_f8f6f4 v[8:11], a[160:167], v[136:143], v[8:11] cbsz:0 blgp:1\n\t" \
  "ds_read_b64_tr_b8 v[152:153], v172 offset:16384\n\t" \
  "v_mfma_f32_16x16x128_f8f6f4 v[40:43], a[168:175], v[136:143], v[40:43] cbsz:0 blgp:1\n\t" \
  "ds_read_b64_tr_b8 v[154:155], v172 offset:17408\n\t" \
  "v_mfma_f32_16x16x128_f8f6f4 v[72:75], a[176:183], v[136:143], v[72:75] cbsz:0 blgp:1\n\t" \
  "ds_read_b64_tr_b8 v[156:157], v172 offset:24576\n\t" \
  "v_mul_f32 v4, s81, v4\n\t" \
  "v_mfma_f32_16x16x128_f8f6f4 v[104:107], a[184:191], v[136:143], v[104:107] cbsz:0 blgp:1\n\t" \
  "ds_read_b64_tr_b8 v[158:159], v172 offset:25600\n\t" \
  "s_mov_b32 s84, 0\n\t" \
  "buffer_store_dword v4, v194, s[44:47], s84 offen offset:64 sc1\n\t" \
  "v_mul_f32 v5, s81, v5\n\t" \
  "s_waitcnt lgkmcnt(4)\n\t" \
  "v_mfma_f32_16x16x128_f8f6f4 v[12:15], a[96:103], v[144:151], v[12:15] cbsz:0 blgp:1\n\t" \
  "ds_read_b64_tr_b8 v[128:129], v162 offset:0\n\t" \
  "s_mov_b32 s84, 65536\n\t" \
  "buffer_store_dword v5, v194, s[44:47], s84 offen offset:64 sc1\n\t" \
  "v_mul_f32 v6, s81, v6\n\t" \
  "v_mfma_f32_16x16x128_f8f6f4 v[44:47], a[104:111], v[144:151], v[44:47] cbsz:0 blgp:1\n\t" \
  "ds_read_b64_tr_b8 v[130:131], v162 offset:1024\n\t" \
  "s_mov_b32 s84, 131072\n\t" \
  "buffer_store_dword v6, v194, s[44:47], s84 offen offset:64 sc1\n\t" \
  "v_mul_f32 v7, s81, v7\n\t" \
  "v_mfma_f32_16x16x128_f8f6f4 v[76:79], a[112:119], v[144:151], v[76:79] cbsz:0 blgp:1\n\t" \
  "ds_read_b64_tr_b8 v[132:133], v162 offset:8192\n\t" \
  "s_mov_b32 s84, 196608\n\t" \
  "buffer_store_dword v7, v194, s[44:47], s84 offen offset:64 sc1\n\t" \
  "v_mfma_f32_16x16x128_f8f6f4 v[108:111], a[120:127], v[144:151], v[108:111] cbsz:0 blgp:1\n\t" \
  "ds_read_b64_tr_b8 v[134:135], v162 offset:9216\n\t" \
  "s_waitcnt lgkmcnt(4)\n\t" \
  "v_mfma_f32_16x16x128_f8f6f4 v[16:19], a[32:39], v[152:159], v[16:19] cbsz:0 blgp:1\n\t" \
  "ds_read_b64_tr_b8 v[136:137], v179 offset:0\n\t" \
  "v_mul_f32 v36, s81, v36\n\t" \
  "v_mfma_f32_16x16x128_f8f6f4 v[48:51], a[40:47], v[152:159], v[48:51] cbsz:0 blgp:1\n\t" \
  "ds_read_b64_tr_b8 v[138:139], v179 offset:1024\n\t" \
  "s_mov_b32 s84, 1048576\n\t" \
  "buffer_store_dword v36, v194, s[44:47], s84 offen offset:64 sc1\n\t" \
  "v_mul_f32 v37, s81, v37\n\t" \
  "v_mfma_f32_16x16x128_f8f6f4 v[80:83], a[48:55], v[152:159], v[80:83] cbsz:0 blgp:1\n\t" \
  "ds_read_b64_tr_b8 v[140:141], v179 offset:8192\n\t" \
  "s_mov_b32 s84, 1114112\n\t" \
  "buffer_store_dword v37, v194, s[44:47], s84 offen offset:64 sc1\n\t" \
  "v_mul_f32 v38, s81, v38\n\t" \
  "v_mfma_f32_16x16x128_f8f6f4 v[112:115], a[56:63], v[152:159], v[112:115] cbsz:0 blgp:1\n\t" \
  "ds_read_b64_tr_b8 v[142:143], v179 offset:9216\n\t" \
  "s_mov_b32 s84, 1179648\n\t" \
  "buffer_store_dword v38, v194, s[44:47], s84 offen offset:64 sc1\n\t" \
  "v_mul_f32 v39, s81, v39\n\t" \
  "s_waitcnt lgkmcnt(4)\n\t" \
  "v_mfma_f32_16x16x128_f8f6f4 v[8:11], a[192:199], v[128:135], v[8:11] cbsz:0 blgp:1\n\t" \
  "ds_read_b64_tr_b8 v[144:145], v172 offset:32768\n\t" \
  "s_mov_b32 s84, 1245184\n\t" \
  "buffer_store_dword v39, v194, s[44:47], s84 offen offset:64 sc1\n\t" \
  "v_mfma_f32_16x16x128_f8f6f4 v[40:43], a[200:207], v[128:135], v[40:43] cbsz:0 blgp:1\n\t" \
  "ds_read_b64_tr_b8 v[146:147], v172 offset:33792\n\t" \
  "v_mfma_f32_16x16x128_f8f6f4 v[72:75], a[208:215], v[128:135], v[72:75] cbsz:0 blgp:1\n\t" \
  "ds_read_b64_tr_b8 v[148:149], v172 offset:40960\n\t" \
  "v_mul_f32 v68, s81, v68\n\t" \
  "v_mfma_f32_16x16x128_f8f6f4 v[104:107], a[216:223], v[128:135], v[104:107] cbsz:0 blgp:1\n\t" \
  "ds_read_b64_tr_b8 v[150:151], v172 offset:41984\n\t" \
  "s_mov_b32 s84, 2097152\n\t" \
  "buffer_store_dword v68, v194, s[44:47], s84 offen offset:64 sc1\n\t" \
  "v_mul_f32 v69, s81, v69\n\t" \
  "s_waitcnt lgkmcnt(4)\n\t" \
  "v_mfma_f32_16x16x128_f8f6f4 v[12:15], a[128:135], v[136:143], v[12:15] cbsz:0 blgp:1\n\t" \
  "ds_read_b64_tr_b8 v[152:153], v173 offset:0\n\t" \
  "s_mov_b32 s84, 2162688\n\t" \
  "buffer_store_dword v69, v194, s[44:47], s84 offen offset:64 sc1\n\t" \
  "v_mul_f32 v70, s81, v70\n\t" \
  "v_mfma_f32_16x16x128_f8f6f4 v[44:47], a[136:143], v[136:143], v[44:47] cbsz:0 blgp:1\n\t" \
  "ds_read_b64_tr_b8 v[154:155], v173 offset:1024\n\t" \
  "s_mov_b32 s84, 2228224\n\t" \
  "buffer_store_dword v70, v194, s[44:47], s84 offen offset:64 sc1\n\t" \
  "v_mul_f32 v71, s81, v71\n\t" \
  "v_mfma_f32_16x16x128_f8f6f4 v[76:79], a[144:151], v[136:143], v[76:79] cbsz:0 blgp:1\n\t" \
  "ds_read_b64_tr_b8 v[156:157], v173 offset:8192\n\t" \
  "s_mov_b32 s84, 2293760\n\t" \
  "buffer_store_dword v71, v194, s[44:47], s84 offen offset:64 sc1\n\t" \
  "v_mfma_f32_16x16x128_f8f6f4 v[108:111], a[152:159], v[136:143], v[108:111] cbsz:0 blgp:1\n\t" \
  "ds_read_b64_tr_b8 v[158:159], v173 offset:9216\n\t" \
  "s_waitcnt lgkmcnt(4)\n\t" \
  "v_mfma_f32_16x16x128_f8f6f4 v[16:19], a[64:71], v[144:151], v[16:19] cbsz:0 blgp:1\n\t" \
  "ds_read_b64_tr_b8 v[128:129], v162 offset:16384\n\t" \
  "v_mul_f32 v100, s81, v100\n\t" \
  "v_mfma_f32_16x16x128_f8f6f4 v[48:51], a[72:79], v[144:151], v[48:51] cbsz:0 blgp:1\n\t" \
  "ds_read_b64_tr_b8 v[130:131], v162 offset:17408\n\t" \
  "s_mov_b32 s84, 3145728\n\t" \
  "buffer_store_dword v100, v194, s[44:47], s84 offen offset:64 sc1\n\t" \
  "v_mul_f32 v101, s81, v101\n\t" \
  "v_mfma_f32_16x16x128_f8f6f4 v[80:83], a[80:87], v[144:151], v[80:83] cbsz:0 blgp:1\n\t" \
  "ds_read_b64_tr_b8 v[132:133], v162 offset:24576\n\t" \
  "s_mov_b32 s84, 3211264\n\t" \
  "buffer_store_dword v101, v194, s[44:47], s84 offen offset:64 sc1\n\t" \
  "v_mul_f32 v102, s81, v102\n\t" \
  "v_mfma_f32_16x16x128_f8f6f4 v[112:115], a[88:95], v[144:151], v[112:115] cbsz:0 blgp:1\n\t" \
  "ds_read_b64_tr_b8 v[134:135], v162 offset:25600\n\t" \
  "s_mov_b32 s84, 3276800\n\t" \
  "buffer_store_dword v102, v194, s[44:47], s84 offen offset:64 sc1\n\t" \
  "v_mul_f32 v103, s81, v103\n\t" \
  "s_waitcnt lgkmcnt(4)\n\t" \
  "v_mfma_f32_16x16x128_f8f6f4 v[20:23], a[0:7], v[152:159], v[20:23] cbsz:0 blgp:1\n\t" \
  "ds_read_b64_tr_b8 v[136:137], v179 offset:16384\n\t" \
  "s_mov_b32 s84, 3342336\n\t" \
  "buffer_store_dword v103, v194, s[44:47], s84 offen offset:64 sc1\n\t" \
  "v_mfma_f32_16x16x128_f8f6f4 v[52:55], a[8:15], v[152:159], v[52:55] cbsz:0 blgp:1\n\t" \
  "ds_read_b64_tr_b8 v[138:139], v179 offset:17408\n\t" \
  "v_mfma_f32_16x16x128_f8f6f4 v[84:87], a[16:23], v[152:159], v[84:87] cbsz:0 blgp:1\n\t" \
  "ds_read_b64_tr_b8 v[140:141], v179 offset:24576\n\t" \
  "v_mfma_f32_16x16x128_f8f6f4 v[116:119], a[24:31], v[152:159], v[116:119] cbsz:0 blgp:1\n\t" \
  "ds_read_b64_tr_b8 v[142:143], v179 offset:25600\n\t" \
  "s_waitcnt lgkmcnt(4)\n\t" \
  "v_mfma_f32_16x16x128_f8f6f4 v[8:11], a[224:231], v[128:135], v[8:11] cbsz:0 blgp:1\n\t" \
  "ds_read_b64_tr_b8 v[144:145], v172 offset:49152\n\t" \
  "v_mfma_f32_16x16x128_f8f6f4 v[40:43], a[232:239], v[128:135], v[40:43] cbsz:0 blgp:1\n\t" \
  "ds_read_b64_tr_b8 v[146:147], v172 offset:50176\n\t" \
  "v_mfma_f32_16x16x128_f8f6f4 v[72:75], a[240:247], v[128:135], v[72:75] cbsz:0 blgp:1\n\t" \
  "ds_read_b64_tr_b8 v[148:149], v172 offset:57344\n\t" \
  "v_mfma_f32_16x16x128_f8f6f4 v[104:107], a[248:255], v[128:135], v[104:107] cbsz:0 blgp:1\n\t" \
  "ds_read_b64_tr_b8 v[150:151], v172 offset:58368\n\t" \
  "s_waitcnt lgkmcnt(4)\n\t" \
  "v_mfma_f32_16x16x128_f8f6f4 v[12:15], a[160:167], v[136:143], v[12:15] cbsz:0 blgp:1\n\t" \
  "ds_read_b64_tr_b8 v[152:153], v173 offset:16384\n\t" \
  "v_mfma_f32_16x16x128_f8f6f4 v[44:47], a[168:175], v[136:143], v[44:47] cbsz:0 blgp:1\n\t" \
  "ds_read_b64_tr_b8 v[154:155], v173 offset:17408\n\t" \
  "v_mfma_f32_16x16x128_f8f6f4 v[76:79], a[176:183], v[136:143], v[76:79] cbsz:0 blgp:1\n\t" \
  "ds_read_b64_tr_b8 v[156:157], v173 offset:24576\n\t" \
  "v_mul_f32 v8, s81, v8\n\t" \
  "v_mfma_f32_16x16x128_f8f6f4 v[108:111], a[184:191], v[136:143], v[108:111] cbsz:0 blgp:1\n\t" \
  "ds_read_b64_tr_b8 v[158:159], v173 offset:25600\n\t" \
  "s_mov_b32 s84, 0\n\t" \
  "buffer_store_dword v8, v194, s[44:47], s84 offen offset:128 sc1\n\t" \
  "v_mul_f32 v9, s81, v9\n\t" \
  "s_waitcnt lgkmcnt(4)\n\t" \
  "v_mfma_f32_16x16x128_f8f6f4 v[16:19], a[96:103], v[144:151], v[16:19] cbsz:0 blgp:1\n\t" \
  "ds_read_b64_tr_b8 v[128:129], v163 offset:0\n\t" \
  "s_mov_b32 s84, 65536\n\t" \
  "buffer_store_dword v9, v194, s[44:47], s84 offen offset:128 sc1\n\t" \
  "v_mul_f32 v10, s81, v10\n\t" \
  "v_mfma_f32_16x16x128_f8f6f4 v[48:51], a[104:111], v[144:151], v[48:51] cbsz:0 blgp:1\n\t" \
  "ds_read_b64_tr_b8 v[130:131], v163 offset:1024\n\t" \
  "s_mov_b32 s84, 131072\n\t" \
  "buffer_store_dword v10, v194, s[44:47], s84 offen offset:128 sc1\n\t" \
  "v_mul_f32 v11, s81, v11\n\t" \
  "v_mfma_f32_16x16x128_f8f6f4 v[80:83], a[112:119], v[144:151], v[80:83] cbsz:0 blgp:1\n\t" \
  "ds_read_b64_tr_b8 v[132:133], v163 offset:8192\n\t" \
  "s_mov_b32 s84, 196608\n\t" \
  "buffer_store_dword v11, v194, s[44:47], s84 offen offset:128 sc1\n\t" \
  "v_mfma_f32_16x16x128_f8f6f4 v[112:115], a[120:127], v[144:151], v[112:115] cbsz:0 blgp:1\n\t" \
  "ds_read_b64_tr_b8 v[134:135], v163 offset:9216\n\t" \
  "s_waitcnt lgkmcnt(4)\n\t" \
  "v_mfma_f32_16x16x128_f8f6f4 v[20:23], a[32:39], v[152:159], v[20:23] cbsz:0 blgp:1\n\t" \
  "ds_read_b64_tr_b8 v[136:137], v180 offset:0\n\t" \
  "v_mul_f32 v40, s81, v40\n\t" \
  "v_mfma_f32_16x16x128_f8f6f4 v[52:55], a[40:47], v[152:159], v[52:55] cbsz:0 blgp:1\n\t" \
  "ds_read_b64_tr_b8 v[138:139], v180 offset:1024\n\t" \
  "s_mov_b32 s84, 1048576\n\t" \
  "buffer_store_dword v40, v194, s[44:47], s84 offen offset:128 sc1\n\t" \
  "v_mul_f32 v41, s81, v41\n\t" \
  "v_mfma_f32_16x16x128_f8f6f4 v[84:87], a[48:55], v[152:159], v[84:87] cbsz:0 blgp:1\n\t" \
  "ds_read_b64_tr_b8 v[140:141], v180 offset:8192\n\t" \
  "s_mov_b32 s84, 1114112\n\t" \
  "buffer_store_dword v41, v194, s[44:47], s84 offen offset:128 sc1\n\t" \
  "v_mul_f32 v42, s81, v42\n\t" \
  "v_mfma_f32_16x16x128_f8f6f4 v[116:119], a[56:63], v[152:159], v[116:119] cbsz:0 blgp:1\n\t" \
  "ds_read_b64_tr_b8 v[142:143], v180 offset:9216\n\t" \
  "s_mov_b32 s84, 1179648\n\t" \
  "buffer_store_dword v42, v194, s[44:47], s84 offen offset:128 sc1\n\t" \
  "v_mul_f32 v43, s81, v43\n\t" \
  "s_waitcnt lgkmcnt(4)\n\t" \
  "v_mfma_f32_16x16x128_f8f6f4 v[12:15], a[192:199], v[128:135], v[12:15] cbsz:0 blgp:1\n\t" \
  "ds_read_b64_tr_b8 v[144:145], v173 offset:32768\n\t" \
  "s_mov_b32 s84, 1245184\n\t" \
  "buffer_store_dword v43, v194, s[44:47], s84 offen offset:128 sc1\n\t" \
  "v_mfma_f32_16x16x128_f8f6f4 v[44:47], a[200:207], v[128:135], v[44:47] cbsz:0 blgp:1\n\t" \
  "ds_read_b64_tr_b8 v[146:147], v173 offset:33792\n\t" \
  "v_mfma_f32_16x16x128_f8f6f4 v[76:79], a[208:215], v[128:135], v[76:79] cbsz:0 blgp:1\n\t" \
  "ds_read_b64_tr_b8 v[148:149], v173 offset:40960\n\t" \
  "v_mul_f32 v72, s81, v72\n\t" \
  "v_mfma_f32_16x16x128_f8f6f4 v[108:111], a[216:223], v[128:135], v[108:111] cbsz:0 blgp:1\n\t" \
  "ds_read_b64_tr_b8 v[150:151], v173 offset:41984\n\t" \
  "s_mov_b32 s84, 2097152\n\t" \
  "buffer_store_dword v72, v194, s[44:47], s84 offen offset:128 sc1\n\t" \
  "v_mul_f32 v73, s81, v73\n\t" \
  "s_waitcnt lgkmcnt(4)\n\t" \
  "v_mfma_f32_16x16x128_f8f6f4 v[16:19], a[128:135], v[136:143], v[16:19] cbsz:0 blgp:1\n\t" \
  "ds_read_b64_tr_b8 v[152:153], v174 offset:0\n\t" \
  "s_mov_b32 s84, 2162688\n\t" \
  "buffer_store_dword v73, v194, s[44:47], s84 offen offset:128 sc1\n\t" \
  "v_mul_f32 v74, s81, v74\n\t" \
  "v_mfma_f32_16x16x128_f8f6f4 v[48:51], a[136:143], v[136:143], v[48:51] cbsz:0 blgp:1\n\t" \
  "ds_read_b64_tr_b8 v[154:155], v174 offset:1024\n\t" \
  "s_mov_b32 s84, 2228224\n\t" \
  "buffer_store_dword v74, v194, s[44:47], s84 offen offset:128 sc1\n\t" \
  "v_mul_f32 v75, s81, v75\n\t" \
  "v_mfma_f32_16x16x128_f8f6f4 v[80:83], a[144:151], v[136:143], v[80:83] cbsz:0 blgp:1\n\t" \
  "ds_read_b64_tr_b8 v[156:157], v174 offset:8192\n\t" \
  "s_mov_b32 s84, 2293760\n\t" \
  "buffer_store_dword v75, v194, s[44:47], s84 offen offset:128 sc1\n\t" \
  "v_mfma_f32_16x16x128_f8f6f4 v[112:115], a[152:159], v[136:143], v[112:115] cbsz:0 blgp:1\n\t" \
  "ds_read_b64_tr_b8 v[158:159], v174 offset:9216\n\t" \
  "s_waitcnt lgkmcnt(4)\n\t" \
  "v_mfma_f32_16x16x128_f8f6f4 v[20:23], a[64:71], v[144:151], v[20:23] cbsz:0 blgp:1\n\t" \
  "ds_read_b64_tr_b8 v[128:129], v163 offset:16384\n\t" \
  "v_mul_f32 v104, s81, v104\n\t" \
  "v_mfma_f32_16x16x128_f8f6f4 v[52:55], a[72:79], v[144:151], v[52:55] cbsz:0 blgp:1\n\t" \
  "ds_read_b64_tr_b8 v[130:131], v163 offset:17408\n\t" \
  "s_mov_b32 s84, 3145728\n\t" \
  "buffer_store_dword v104, v194, s[44:47], s84 offen offset:128 sc1\n\t" \
  "v_mul_f32 v105, s81, v105\n\t" \
  "v_mfma_f32_16x16x128_f8f6f4 v[84:87], a[80:87], v[144:151], v[84:87] cbsz:0 blgp:1\n\t" \
  "ds_read_b64_tr_b8 v[132:133], v163 offset:24576\n\t" \
  "s_mov_b32 s84, 3211264\n\t" \
  "buffer_store_dword v105, v194, s[44:47], s84 offen offset:128 sc1\n\t" \
  "v_mul_f32 v106, s81, v106\n\t" \
  "v_mfma_f32_16x16x128_f8f6f4 v[116:119], a[88:95], v[144:151], v[116:119] cbsz:0 blgp:1\n\t" \
  "ds_read_b64_tr_b8 v[134:135], v163 offset:25600\n\t" \
  "s_mov_b32 s84, 3276800\n\t" \
  "buffer_store_dword v106, v194, s[44:47], s84 offen offset:128 sc1\n\t" \
  "v_mul_f32 v107, s81, v107\n\t" \
  "s_waitcnt lgkmcnt(4)\n\t" \
  "v_mfma_f32_16x16x128_f8f6f4 v[24:27], a[0:7], v[152:159], v[24:27] cbsz:0 blgp:1\n\t" \
  "ds_read_b64_tr_b8 v[136:137], v180 offset:16384\n\t" \
  "s_mov_b32 s84, 3342336\n\t" \
  "buffer_store_dword v107, v194, s[44:47], s84 offen offset:128 sc1\n\t" \
  "v_mfma_f32_16x16x128_f8f6f4 v[56:59], a[8:15], v[152:159], v[56:59] cbsz:0 blgp:1\n\t" \
  "ds_read_b64_tr_b8 v[138:139], v180 offset:17408\n\t" \
  "v_mfma_f32_16x16x128_f8f6f4 v[88:91], a[16:23], v[152:159], v[88:91] cbsz:0 blgp:1\n\t" \
  "ds_read_b64_tr_b8 v[140:141], v180 offset:24576\n\t" \
  "v_mfma_f32_16x16x128_f8f6f4 v[120:123], a[24:31], v[152:159], v[120:123] cbsz:0 blgp:1\n\t" \
  "ds_read_b64_tr_b8 v[142:143], v180 offset:25600\n\t" \
  "s_waitcnt lgkmcnt(4)\n\t" \
  "v_mfma_f32_16x16x128_f8f6f4 v[12:15], a[224:231], v[128:135], v[12:15] cbsz:0 blgp:1\n\t" \
  "ds_read_b64_tr_b8 v[144:145], v173 offset:49152\n\t" \
  "v_mfma_f32_16x16x128_f8f6f4 v[44:47], a[232:239], v[128:135], v[44:47] cbsz:0 blgp:1\n\t" \
  "ds_read_b64_tr_b8 v[146:147], v173 offset:50176\n\t" \
  "v_mfma_f32_16x16x128_f8f6f4 v[76:79], a[240:247], v[128:135], v[76:79] cbsz:0 blgp:1\n\t" \
  "ds_read_b64_tr_b8 v[148:149], v173 offset:57344\n\t" \
  "v_mfma_f32_16x16x128_f8f6f4 v[108:111], a[248:255], v[128:135], v[108:111] cbsz:0 blgp:1\n\t" \
  "ds_read_b64_tr_b8 v[150:151], v173 offset:58368\n\t" \
  "s_waitcnt lgkmcnt(4)\n\t" \
  "v_mfma_f32_16x16x128_f8f6f4 v[16:19], a[160:167], v[136:143], v[16:19] cbsz:0 blgp:1\n\t" \
  "ds_read_b64_tr_b8 v[152:153], v174 offset:16384\n\t" \
  "v_mfma_f32_16x16x128_f8f6f4 v[48:51], a[168:175], v[136:143], v[48:51] cbsz:0 blgp:1\n\t" \
  "ds_read_b64_tr_b8 v[154:155], v174 offset:17408\n\t" \
  "v_mfma_f32_16x16x128_f8f6f4 v[80:83], a[176:183], v[136:143], v[80:83] cbsz:0 blgp:1\n\t" \
  "ds_read_b64_tr_b8 v[156:157], v174 offset:24576\n\t" \
  "v_mul_f32 v12, s81, v12\n\t" \
  "v_mfma_f32_16x16x128_f8f6f4 v[112:115], a[184:191], v[136:143], v[112:115] cbsz:0 blgp:1\n\t" \
  "ds_read_b64_tr_b8 v[158:159], v174 offset:25600\n\t" \
  "s_mov_b32 s84, 0\n\t" \
  "buffer_store_dword v12, v194, s[44:47], s84 offen offset:192 sc1\n\t" \
  "v_mul_f32 v13, s81, v13\n\t" \
  "s_waitcnt lgkmcnt(4)\n\t" \
  "v_mfma_f32_16x16x128_f8f6f4 v[20:23], a[96:103], v[144:151], v[20:23] cbsz:0 blgp:1\n\t" \
  "ds_read_b64_tr_b8 v[128:129], v164 offset:0\n\t" \
  "s_mov_b32 s84, 65536\n\t" \
  "buffer_store_dword v13, v194, s[44:47], s84 offen offset:192 sc1\n\t" \
  "v_mul_f32 v14, s81, v14\n\t" \
  "v_mfma_f32_16x16x128_f8f6f4 v[52:55], a[104:111], v[144:151], v[52:55] cbsz:0 blgp:1\n\t" \
  "ds_read_b64_tr_b8 v[130:131], v164 offset:1024\n\t" \
  "s_mov_b32 s84, 131072\n\t" \
  "buffer_store_dword v14, v194, s[44:47], s84 offen offset:192 sc1\n\t" \
  "v_mul_f32 v15, s81, v15\n\t" \
  "v_mfma_f32_16x16x128_f8f6f4 v[84:87], a[112:119], v[144:151], v[84:87] cbsz:0 blgp:1\n\t" \
  "ds_read_b64_tr_b8 v[132:133], v164 offset:8192\n\t" \
  "s_mov_b32 s84, 196608\n\t" \
  "buffer_store_dword v15, v194, s[44:47], s84 offen offset:192 sc1\n\t" \
  "v_mfma_f32_16x16x128_f8f6f4 v[116:119], a[120:127], v[144:151], v[116:119] cbsz:0 blgp:1\n\t" \
  "ds_read_b64_tr_b8 v[134:135], v164 offset:9216\n\t" \
  "s_waitcnt lgkmcnt(4)\n\t" \
  "v_mfma_f32_16x16x128_f8f6f4 v[24:27], a[32:39], v[152:159], v[24:27] cbsz:0 blgp:1\n\t" \
  "ds_read_b64_tr_b8 v[136:137], v181 offset:0\n\t" \
  "v_mul_f32 v44, s81, v44\n\t" \
  "v_mfma_f32_16x16x128_f8f6f4 v[56:59], a[40:47], v[152:159], v[56:59] cbsz:0 blgp:1\n\t" \
  "ds_read_b64_tr_b8 v[138:139], v181 offset:1024\n\t" \
  "s_mov_b32 s84, 1048576\n\t" \
  "buffer_store_dword v44, v194, s[44:47], s84 offen offset:192 sc1\n\t" \
  "v_mul_f32 v45, s81, v45\n\t" \
  "v_mfma_f32_16x16x128_f8f6f4 v[88:91], a[48:55], v[152:159], v[88:91] cbsz:0 blgp:1\n\t" \
  "ds_read_b64_tr_b8 v[140:141], v181 offset:8192\n\t" \
  "s_mov_b32 s84, 1114112\n\t" \
  "buffer_store_dword v45, v194, s[44:47], s84 offen offset:192 sc1\n\t" \
  "v_mul_f32 v46, s81, v46\n\t" \
  "v_mfma_f32_16x16x128_f8f6f4 v[120:123], a[56:63], v[152:159], v[120:123] cbsz:0 blgp:1\n\t" \
  "ds_read_b64_tr_b8 v[142:143], v181 offset:9216\n\t" \
  "s_mov_b32 s84, 1179648\n\t" \
  "buffer_store_dword v46, v194, s[44:47], s84 offen offset:192 sc1\n\t" \
  "v_mul_f32 v47, s81, v47\n\t" \
  "s_waitcnt lgkmcnt(4)\n\t" \
  "v_mfma_f32_16x16x128_f8f6f4 v[16:19], a[192:199], v[128:135], v[16:19] cbsz:0 blgp:1\n\t" \
  "ds_read_b64_tr_b8 v[144:145], v174 offset:32768\n\t" \
  "s_mov_b32 s84, 1245184\n\t" \
  "buffer_store_dword v47, v194, s[44:47], s84 offen offset:192 sc1\n\t" \
  "v_mfma_f32_16x16x128_f8f6f4 v[48:51], a[200:207], v[128:135], v[48:51] cbsz:0 blgp:1\n\t" \
  "ds_read_b64_tr_b8 v[146:147], v174 offset:33792\n\t" \
  "v_mfma_f32_16x16x128_f8f6f4 v[80:83], a[208:215], v[128:135], v[80:83] cbsz:0 blgp:1\n\t" \
  "ds_read_b64_tr_b8 v[148:149], v174 offset:40960\n\t" \
  "v_mul_f32 v76, s81, v76\n\t" \
  "v_mfma_f32_16x16x128_f8f6f4 v[112:115], a[216:223], v[128:135], v[112:115] cbsz:0 blgp:1\n\t" \
  "ds_read_b64_tr_b8 v[150:151], v174 offset:41984\n\t" \
  "s_mov_b32 s84, 2097152\n\t" \
  "buffer_store_dword v76, v194, s[44:47], s84 offen offset:192 sc1\n\t" \
  "v_mul_f32 v77, s81, v77\n\t" \
  "s_waitcnt lgkmcnt(4)\n\t" \
  "v_mfma_f32_16x16x128_f8f6f4 v[20:23], a[128:135], v[136:143], v[20:23] cbsz:0 blgp:1\n\t" \
  "ds_read_b64_tr_b8 v[152:153], v175 offset:0\n\t" \
  "s_mov_b32 s84, 2162688\n\t" \
  "buffer_store_dword v77, v194, s[44:47], s84 offen offset:192 sc1\n\t" \
  "v_mul_f32 v78, s81, v78\n\t" \
  "v_mfma_f32_16x16x128_f8f6f4 v[52:55], a[136:143], v[136:143], v[52:55] cbsz:0 blgp:1\n\t" \
  "ds_read_b64_tr_b8 v[154:155], v175 offset:1024\n\t" \
  "s_mov_b32 s84, 2228224\n\t" \
  "buffer_store_dword v78, v194, s[44:47], s84 offen offset:192 sc1\n\t" \
  "v_mul_f32 v79, s81, v79\n\t" \
  "v_mfma_f32_16x16x128_f8f6f4 v[84:87], a[144:151], v[136:143], v[84:87] cbsz:0 blgp:1\n\t" \
  "ds_read_b64_tr_b8 v[156:157], v175 offset:8192\n\t" \
  "s_mov_b32 s84, 2293760\n\t" \
  "buffer_store_dword v79, v194, s[44:47], s84 offen offset:192 sc1\n\t" \
  "v_mfma_f32_16x16x128_f8f6f4 v[116:119], a[152:159], v[136:143], v[116:119] cbsz:0 blgp:1\n\t" \
  "ds_read_b64_tr_b8 v[158:159], v175 offset:9216\n\t" \
  "s_waitcnt lgkmcnt(4)\n\t" \
  "v_mfma_f32_16x16x128_f8f6f4 v[24:27], a[64:71], v[144:151], v[24:27] cbsz:0 blgp:1\n\t" \
  "ds_read_b64_tr_b8 v[128:129], v164 offset:16384\n\t" \
  "v_mul_f32 v108, s81, v108\n\t" \
  "v_mfma_f32_16x16x128_f8f6f4 v[56:59], a[72:79], v[144:151], v[56:59] cbsz:0 blgp:1\n\t" \
  "ds_read_b64_tr_b8 v[130:131], v164 offset:17408\n\t" \
  "s_mov_b32 s84, 3145728\n\t" \
  "buffer_store_dword v108, v194, s[44:47], s84 offen offset:192 sc1\n\t" \
  "v_mul_f32 v109, s81, v109\n\t" \
  "v_mfma_f32_16x16x128_f8f6f4 v[88:91], a[80:87], v[144:151], v[88:91] cbsz:0 blgp:1\n\t" \
  "ds_read_b64_tr_b8 v[132:133], v164 offset:24576\n\t" \
  "s_mov_b32 s84, 3211264\n\t" \
  "buffer_store_dword v109, v194, s[44:47], s84 offen offset:192 sc1\n\t" \
  "v_mul_f32 v110, s81, v110\n\t" \
  "v_mfma_f32_16x16x128_f8f6f4 v[120:123], a[88:95], v[144:151], v[120:123] cbsz:0 blgp:1\n\t" \
  "ds_read_b64_tr_b8 v[134:135], v164 offset:25600\n\t" \
  "s_mov_b32 s84, 3276800\n\t" \
  "buffer_store_dword v110, v194, s[44:47], s84 offen offset:192 sc1\n\t" \
  "v_mul_f32 v111, s81, v111\n\t" \
  "s_waitcnt lgkmcnt(4)\n\t" \
  "v_mfma_f32_16x16x128_f8f6f4 v[28:31], a[0:7], v[152:159], v[28:31] cbsz:0 blgp:1\n\t" \
  "ds_read_b64_tr_b8 v[136:137], v181 offset:16384\n\t" \
  "s_mov_b32 s84, 3342336\n\t" \
  "buffer_store_dword v111, v194, s[44:47], s84 offen offset:192 sc1\n\t" \
  "v_mfma_f32_16x16x128_f8f6f4 v[60:63], a[8:15], v[152:159], v[60:63] cbsz:0 blgp:1\n\t" \
  "ds_read_b64_tr_b8 v[138:139], v181 offset:17408\n\t" \
  "v_mfma_f32_16x16x128_f8f6f4 v[92:95], a[16:23], v[152:159], v[92:95] cbsz:0 blgp:1\n\t" \
  "ds_read_b64_tr_b8 v[140:141], v181 offset:24576\n\t" \
  "v_mfma_f32_16x16x128_f8f6f4 v[124:127], a[24:31], v[152:159], v[124:127] cbsz:0 blgp:1\n\t" \
  "ds_read_b64_tr_b8 v[142:143], v181 offset:25600\n\t" \
  "s_waitcnt lgkmcnt(4)\n\t" \
  "v_mfma_f32_16x16x128_f8f6f4 v[16:19], a[224:231], v[128:135], v[16:19] cbsz:0 blgp:1\n\t" \
  "ds_read_b64_tr_b8 v[144:145], v174 offset:49152\n\t" \
  "v_mfma_f32_16x16x128_f8f6f4 v[48:51], a[232:239], v[128:135], v[48:51] cbsz:0 blgp:1\n\t" \
  "ds_read_b64_tr_b8 v[146:147], v174 offset:50176\n\t" \
  "v_mfma_f32_16x16x128_f8f6f4 v[80:83], a[240:247], v[128:135], v[80:83] cbsz:0 blgp:1\n\t" \
  "ds_read_b64_tr_b8 v[148:149], v174 offset:57344\n\t" \
  "v_mfma_f32_16x16x128_f8f6f4 v[112:115], a[248:255], v[128:135], v[112:115] cbsz:0 blgp:1\n\t" \
  "ds_read_b64_tr_b8 v[150:151], v174 offset:58368\n\t" \
  "s_waitcnt lgkmcnt(4)\n\t" \
  "v_mfma_f32_16x16x128_f8f6f4 v[20:23], a[160:167], v[136:143], v[20:23] cbsz:0 blgp:1\n\t" \
  "ds_read_b64_tr_b8 v[152:153], v175 offset:16384\n\t" \
  "v_mfma_f32_16x16x128_f8f6f4 v[52:55], a[168:175], v[136:143], v[52:55] cbsz:0 blgp:1\n\t" \
  "ds_read_b64_tr_b8 v[154:155], v175 offset:17408\n\t" \
  "v_mfma_f32_16x16x128_f8f6f4 v[84:87], a[176:183], v[136:143], v[84:87] cbsz:0 blgp:1\n\t" \
  "ds_read_b64_tr_b8 v[156:157], v175 offset:24576\n\t" \
  "v_mul_f32 v16, s81, v16\n\t" \
  "v_mfma_f32_16x16x128_f8f6f4 v[116:119], a[184:191], v[136:143], v[116:119] cbsz:0 blgp:1\n\t" \
  "ds_read_b64_tr_b8 v[158:159], v175 offset:25600\n\t" \
  "s_mov_b32 s84, 0\n\t" \
  "buffer_store_dword v16, v194, s[44:47], s84 offen offset:256 sc1\n\t" \
  "v_mul_f32 v17, s81, v17\n\t" \
  "s_waitcnt lgkmcnt(4)\n\t" \
  "v_mfma_f32_16x16x128_f8f6f4 v[24:27], a[96:103], v[144:151], v[24:27] cbsz:0 blgp:1\n\t" \
  "ds_read_b64_tr_b8 v[128:129], v165 offset:0\n\t" \
  "s_mov_b32 s84, 65536\n\t" \
  "buffer_store_dword v17, v194, s[44:47], s84 offen offset:256 sc1\n\t" \
  "v_mul_f32 v18, s81, v18\n\t" \
  "v_mfma_f32_16x16x128_f8f6f4 v[56:59], a[104:111], v[144:151], v[56:59] cbsz:0 blgp:1\n\t" \
  "ds_read_b64_tr_b8 v[130:131], v165 offset:1024\n\t" \
  "s_mov_b32 s84, 131072\n\t" \
  "buffer_store_dword v18, v194, s[44:47], s84 offen offset:256 sc1\n\t" \
  "v_mul_f32 v19, s81, v19\n\t" \
  "v_mfma_f32_16x16x128_f8f6f4 v[88:91], a[112:119], v[144:151], v[88:91] cbsz:0 blgp:1\n\t" \
  "ds_read_b64_tr_b8 v[132:133], v165 offset:8192\n\t" \
  "s_mov_b32 s84, 196608\n\t" \
  "buffer_store_dword v19, v194, s[44:47], s84 offen offset:256 sc1\n\t" \
  "v_mfma_f32_16x16x128_f8f6f4 v[120:123], a[120:127], v[144:151], v[120:123] cbsz:0 blgp:1\n\t" \
  "ds_read_b64_tr_b8 v[134:135], v165 offset:9216\n\t" \
  "s_waitcnt lgkmcnt(4)\n\t" \
  "v_mfma_f32_16x16x128_f8f6f4 v[28:31], a[32:39], v[152:159], v[28:31] cbsz:0 blgp:1\n\t" \
  "ds_read_b64_tr_b8 v[136:137], v182 offset:0\n\t" \
  "v_mul_f32 v48, s81, v48\n\t" \
  "v_mfma_f32_16x16x128_f8f6f4 v[60:63], a[40:47], v[152:159], v[60:63] cbsz:0 blgp:1\n\t" \
  "ds_read_b64_tr_b8 v[138:139], v182 offset:1024\n\t" \
  "s_mov_b32 s84, 1048576\n\t" \
  "buffer_store_dword v48, v194, s[44:47], s84 offen offset:256 sc1\n\t" \
  "v_mul_f32 v49, s81, v49\n\t" \
  "v_mfma_f32_16x16x128_f8f6f4 v[92:95], a[48:55], v[152:159], v[92:95] cbsz:0 blgp:1\n\t" \
  "ds_read_b64_tr_b8 v[140:141], v182 offset:8192\n\t" \
  "s_mov_b32 s84, 1114112\n\t" \
  "buffer_store_dword v49, v194, s[44:47], s84 offen offset:256 sc1\n\t" \
  "v_mul_f32 v50, s81, v50\n\t" \
  "v_mfma_f32_16x16x128_f8f6f4 v[124:127], a[56:63], v[152:159], v[124:127] cbsz:0 blgp:1\n\t" \
  "ds_read_b64_tr_b8 v[142:143], v182 offset:9216\n\t" \
  "s_mov_b32 s84, 1179648\n\t" \
  "buffer_store_dword v50, v194, s[44:47], s84 offen offset:256 sc1\n\t" \
  "v_mul_f32 v51, s81, v51\n\t" \
  "s_waitcnt lgkmcnt(4)\n\t" \
  "v_mfma_f32_16x16x128_f8f6f4 v[20:23], a[192:199], v[128:135], v[20:23] cbsz:0 blgp:1\n\t" \
  "ds_read_b64_tr_b8 v[144:145], v175 offset:32768\n\t" \
  "s_mov_b32 s84, 1245184\n\t" \
  "buffer_store_dword v51, v194, s[44:47], s84 offen offset:256 sc1\n\t" \
  "v_mfma_f32_16x16x128_f8f6f4 v[52:55], a[200:207], v[128:135], v[52:55] cbsz:0 blgp:1\n\t" \
  "ds_read_b64_tr_b8 v[146:147], v175 offset:33792\n\t" \
  "v_mfma_f32_16x16x128_f8f6f4 v[84:87], a[208:215], v[128:135], v[84:87] cbsz:0 blgp:1\n\t" \
  "ds_read_b64_tr_b8 v[148:149], v175 offset:40960\n\t" \
  "v_mul_f32 v80, s81, v80\n\t" \
  "v_mfma_f32_16x16x128_f8f6f4 v[116:119], a[216:223], v[128:135], v[116:119] cbsz:0 blgp:1\n\t" \
  "ds_read_b64_tr_b8 v[150:151], v175 offset:41984\n\t" \
  "s_mov_b32 s84, 2097152\n\t" \
  "buffer_store_dword v80, v194, s[44:47], s84 offen offset:256 sc1\n\t" \
  "v_mul_f32 v81, s81, v81\n\t" \
  "s_waitcnt lgkmcnt(4)\n\t" \
  "v_mfma_f32_16x16x128_f8f6f4 v[24:27], a[128:135], v[136:143], v[24:27] cbsz:0 blgp:1\n\t" \
  "ds_read_b64_tr_b8 v[152:153], v165 offset:16384\n\t" \
  "s_mov_b32 s84, 2162688\n\t" \
  "buffer_store_dword v81, v194, s[44:47], s84 offen offset:256 sc1\n\t" \
  "v_mul_f32 v82, s81, v82\n\t" \
  "v_mfma_f32_16x16x128_f8f6f4 v[56:59], a[136:143], v[136:143], v[56:59] cbsz:0 blgp:1\n\t" \
  "ds_read_b64_tr_b8 v[154:155], v165 offset:17408\n\t" \
  "s_mov_b32 s84, 2228224\n\t" \
  "buffer_store_dword v82, v194, s[44:47], s84 offen offset:256 sc1\n\t" \
  "v_mul_f32 v83, s81, v83\n\t" \
  "v_mfma_f32_16x16x128_f8f6f4 v[88:91], a[144:151], v[136:143], v[88:91] cbsz:0 blgp:1\n\t" \
  "ds_read_b64_tr_b8 v[156:157], v165 offset:24576\n\t" \
  "s_mov_b32 s84, 2293760\n\t" \
  "buffer_store_dword v83, v194, s[44:47], s84 offen offset:256 sc1\n\t" \
  "v_mfma_f32_16x16x128_f8f6f4 v[120:123], a[152:159], v[136:143], v[120:123] cbsz:0 blgp:1\n\t" \
  "ds_read_b64_tr_b8 v[158:159], v165 offset:25600\n\t" \
  "s_waitcnt lgkmcnt(4)\n\t" \
  "v_mfma_f32_16x16x128_f8f6f4 v[28:31], a[64:71], v[144:151], v[28:31] cbsz:0 blgp:1\n\t" \
  "ds_read_b64_tr_b8 v[128:129], v182 offset:16384\n\t" \
  "v_mul_f32 v112, s81, v112\n\t" \
  "v_mfma_f32_16x16x128_f8f6f4 v[60:63], a[72:79], v[144:151], v[60:63] cbsz:0 blgp:1\n\t" \
  "ds_read_b64_tr_b8 v[130:131], v182 offset:17408\n\t" \
  "s_mov_b32 s84, 3145728\n\t" \
  "buffer_store_dword v112, v194, s[44:47], s84 offen offset:256 sc1\n\t" \
  "v_mul_f32 v113, s81, v113\n\t" \
  "v_mfma_f32_16x16x128_f8f6f4 v[92:95], a[80:87], v[144:151], v[92:95] cbsz:0 blgp:1\n\t" \
  "ds_read_b64_tr_b8 v[132:133], v182 offset:24576\n\t" \
  "s_mov_b32 s84, 3211264\n\t" \
  "buffer_store_dword v113, v194, s[44:47], s84 offen offset:256 sc1\n\t" \
  "v_mul_f32 v114, s81, v114\n\t" \
  "v_mfma_f32_16x16x128_f8f6f4 v[124:127], a[88:95], v[144:151], v[124:127] cbsz:0 blgp:1\n\t" \
  "ds_read_b64_tr_b8 v[134:135], v182 offset:25600\n\t" \
  "s_mov_b32 s84, 3276800\n\t" \
  "buffer_store_dword v114, v194, s[44:47], s84 offen offset:256 sc1\n\t" \
  "v_mul_f32 v115, s81, v115\n\t" \
  "s_waitcnt lgkmcnt(4)\n\t" \
  "v_mfma_f32_16x16x128_f8f6f4 v[20:23], a[224:231], v[152:159], v[20:23] cbsz:0 blgp:1\n\t" \
  "ds_read_b64_tr_b8 v[136:137], v175 offset:49152\n\t" \
  "s_mov_b32 s84, 3342336\n\t" \
  "buffer_store_dword v115, v194, s[44:47], s84 offen offset:256 sc1\n\t" \
  "v_mfma_f32_16x16x128_f8f6f4 v[52:55], a[232:239], v[152:159], v[52:55] cbsz:0 blgp:1\n\t" \
  "ds_read_b64_tr_b8 v[138:139], v175 offset:50176\n\t" \
  "v_mfma_f32_16x16x128_f8f6f4 v[84:87], a[240:247], v[152:159], v[84:87] cbsz:0 blgp:1\n\t" \
  "ds_read_b64_tr_b8 v[140:141], v175 offset:57344\n\t" \
  "v_mfma_f32_16x16x128_f8f6f4 v[116:119], a[248:255], v[152:159], v[116:119] cbsz:0 blgp:1\n\t" \
  "ds_read_b64_tr_b8 v[142:143], v175 offset:58368\n\t" \
  "s_waitcnt lgkmcnt(4)\n\t" \
  "v_mfma_f32_16x16x128_f8f6f4 v[24:27], a[160:167], v[128:135], v[24:27] cbsz:0 blgp:1\n\t" \
  "ds_read_b64_tr_b8 v[144:145], v166 offset:0\n\t" \
  "v_mfma_f32_16x16x128_f8f6f4 v[56:59], a[168:175], v[128:135], v[56:59] cbsz:0 blgp:1\n\t" \
  "ds_read_b64_tr_b8 v[146:147], v166 offset:1024\n\t" \
  "v_mfma_f32_16x16x128_f8f6f4 v[88:91], a[176:183], v[128:135], v[88:91] cbsz:0 blgp:1\n\t" \
  "ds_read_b64_tr_b8 v[148:149], v166 offset:8192\n\t" \
  "v_mul_f32 v20, s81, v20\n\t" \
  "v_mfma_f32_16x16x128_f8f6f4 v[120:123], a[184:191], v[128:135], v[120:123] cbsz:0 blgp:1\n\t" \
  "ds_read_b64_tr_b8 v[150:151], v166 offset:9216\n\t" \
  "s_mov_b32 s84, 0\n\t" \
  "buffer_store_dword v20, v194, s[44:47], s84 offen offset:320 sc1\n\t" \
  "v_mul_f32 v21, s81, v21\n\t" \
  "s_waitcnt lgkmcnt(4)\n\t" \
  "v_mfma_f32_16x16x128_f8f6f4 v[28:31], a[96:103], v[136:143], v[28:31] cbsz:0 blgp:1\n\t" \
  "ds_read_b64_tr_b8 v[152:153], v183 offset:0\n\t" \
  "s_mov_b32 s84, 65536\n\t" \
  "buffer_store_dword v21, v194, s[44:47], s84 offen offset:320 sc1\n\t" \
  "v_mul_f32 v22, s81, v22\n\t" \
  "v_mfma_f32_16x16x128_f8f6f4 v[60:63], a[104:111], v[136:143], v[60:63] cbsz:0 blgp:1\n\t" \
  "ds_read_b64_tr_b8 v[154:155], v183 offset:1024\n\t" \
  "s_mov_b32 s84, 131072\n\t" \
  "buffer_store_dword v22, v194, s[44:47], s84 offen offset:320 sc1\n\t" \
  "v_mul_f32 v23, s81, v23\n\t" \
  "v_mfma_f32_16x16x128_f8f6f4 v[92:95], a[112:119], v[136:143], v[92:95] cbsz:0 blgp:1\n\t" \
  "ds_read_b64_tr_b8 v[156:157], v183 offset:8192\n\t" \
  "s_mov_b32 s84, 196608\n\t" \
  "buffer_store_dword v23, v194, s[44:47], s84 offen offset:320 sc1\n\t" \
  "v_mfma_f32_16x16x128_f8f6f4 v[124:127], a[120:127], v[136:143], v[124:127] cbsz:0 blgp:1\n\t" \
  "ds_read_b64_tr_b8 v[158:159], v183 offset:9216\n\t" \
  "s_waitcnt lgkmcnt(4)\n\t" \
  "v_mfma_f32_16x16x128_f8f6f4 v[24:27], a[192:199], v[144:151], v[24:27] cbsz:0 blgp:1\n\t" \
  "ds_read_b64_tr_b8 v[128:129], v166 offset:16384\n\t" \
  "v_mul_f32 v52, s81, v52\n\t" \
  "v_mfma_f32_16x16x128_f8f6f4 v[56:59], a[200:207], v[144:151], v[56:59] cbsz:0 blgp:1\n\t" \
  "ds_read_b64_tr_b8 v[130:131], v166 offset:17408\n\t" \
  "s_mov_b32 s84, 1048576\n\t" \
  "buffer_store_dword v52, v194, s[44:47], s84 offen offset:320 sc1\n\t" \
  "v_mul_f32 v53, s81, v53\n\t" \
  "v_mfma_f32_16x16x128_f8f6f4 v[88:91], a[208:215], v[144:151], v[88:91] cbsz:0 blgp:1\n\t" \
  "ds_read_b64_tr_b8 v[132:133], v166 offset:24576\n\t" \
  "s_mov_b32 s84, 1114112\n\t" \
  "buffer_store_dword v53, v194, s[44:47], s84 offen offset:320 sc1\n\t" \
  "v_mul_f32 v54, s81, v54\n\t" \
  "v_mfma_f32_16x16x128_f8f6f4 v[120:123], a[216:223], v[144:151], v[120:123] cbsz:0 blgp:1\n\t" \
  "ds_read_b64_tr_b8 v[134:135], v166 offset:25600\n\t" \
  "s_mov_b32 s84, 1179648\n\t" \
  "buffer_store_dword v54, v194, s[44:47], s84 offen offset:320 sc1\n\t" \
  "v_mul_f32 v55, s81, v55\n\t" \
  "s_waitcnt lgkmcnt(4)\n\t" \
  "v_mfma_f32_16x16x128_f8f6f4 v[28:31], a[128:135], v[152:159], v[28:31] cbsz:0 blgp:1\n\t" \
  "ds_read_b64_tr_b8 v[136:137], v183 offset:16384\n\t" \
  "s_mov_b32 s84, 1245184\n\t" \
  "buffer_store_dword v55, v194, s[44:47], s84 offen offset:320 sc1\n\t" \
  "v_mfma_f32_16x16x128_f8f6f4 v[60:63], a[136:143], v[152:159], v[60:63] cbsz:0 blgp:1\n\t" \
  "ds_read_b64_tr_b8 v[138:139], v183 offset:17408\n\t" \
  "v_mfma_f32_16x16x128_f8f6f4 v[92:95], a[144:151], v[152:159], v[92:95] cbsz:0 blgp:1\n\t" \
  "ds_read_b64_tr_b8 v[140:141], v183 offset:24576\n\t" \
  "v_mul_f32 v84, s81, v84\n\t" \
  "v_mfma_f32_16x16x128_f8f6f4 v[124:127], a[152:159], v[152:159], v[124:127] cbsz:0 blgp:1\n\t" \
  "ds_read_b64_tr_b8 v[142:143], v183 offset:25600\n\t" \
  "s_mov_b32 s84, 2097152\n\t" \
  "buffer_store_dword v84, v194, s[44:47], s84 offen offset:320 sc1\n\t" \
  "v_mul_f32 v85, s81, v85\n\t" \
  "s_waitcnt lgkmcnt(4)\n\t" \
  "v_mfma_f32_16x16x128_f8f6f4 v[24:27], a[224:231], v[128:135], v[24:27] cbsz:0 blgp:1\n\t" \
  "ds_read_b64_tr_b8 v[144:145], v167 offset:0\n\t" \
  "s_mov_b32 s84, 2162688\n\t" \
  "buffer_store_dword v85, v194, s[44:47], s84 offen offset:320 sc1\n\t" \
  "v_mul_f32 v86, s81, v86\n\t" \
  "v_mfma_f32_16x16x128_f8f6f4 v[56:59], a[232:239], v[128:135], v[56:59] cbsz:0 blgp:1\n\t" \
  "ds_read_b64_tr_b8 v[146:147], v167 offset:1024\n\t" \
  "s_mov_b32 s84, 2228224\n\t" \
  "buffer_store_dword v86, v194, s[44:47], s84 offen offset:320 sc1\n\t" \
  "v_mul_f32 v87, s81, v87\n\t" \
  "v_mfma_f32_16x16x128_f8f6f4 v[88:91], a[240:247], v[128:135], v[88:91] cbsz:0 blgp:1\n\t" \
  "ds_read_b64_tr_b8 v[148:149], v167 offset:8192\n\t" \
  "s_mov_b32 s84, 2293760\n\t" \
  "buffer_store_dword v87, v194, s[44:47], s84 offen offset:320 sc1\n\t" \
  "v_mfma_f32_16x16x128_f8f6f4 v[120:123], a[248:255], v[128:135], v[120:123] cbsz:0 blgp:1\n\t" \
  "ds_read_b64_tr_b8 v[150:151], v167 offset:9216\n\t" \
  "s_waitcnt lgkmcnt(4)\n\t" \
  "v_mfma_f32_16x16x128_f8f6f4 v[28:31], a[160:167], v[136:143], v[28:31] cbsz:0 blgp:1\n\t" \
  "ds_read_b64_tr_b8 v[152:153], v167 offset:16384\n\t" \
  "v_mul_f32 v116, s81, v116\n\t" \
  "v_mfma_f32_16x16x128_f8f6f4 v[60:63], a[168:175], v[136:143], v[60:63] cbsz:0 blgp:1\n\t" \
  "ds_read_b64_tr_b8 v[154:155], v167 offset:17408\n\t" \
  "s_mov_b32 s84, 3145728\n\t" \
  "buffer_store_dword v116, v194, s[44:47], s84 offen offset:320 sc1\n\t" \
  "v_mul_f32 v117, s81, v117\n\t" \
  "v_mfma_f32_16x16x128_f8f6f4 v[92:95], a[176:183], v[136:143], v[92:95] cbsz:0 blgp:1\n\t" \
  "ds_read_b64_tr_b8 v[156:157], v167 offset:24576\n\t" \
  "s_mov_b32 s84, 3211264\n\t" \
  "buffer_store_dword v117, v194, s[44:47], s84 offen offset:320 sc1\n\t" \
  "v_mul_f32 v118, s81, v118\n\t" \
  "v_mfma_f32_16x16x128_f8f6f4 v[124:127], a[184:191], v[136:143], v[124:127] cbsz:0 blgp:1\n\t" \
  "ds_read_b64_tr_b8 v[158:159], v167 offset:25600\n\t" \
  "s_mov_b32 s84, 3276800\n\t" \
  "buffer_store_dword v118, v194, s[44:47], s84 offen offset:320 sc1\n\t" \
  "v_mul_f32 v119, s81, v119\n\t" \
  "s_waitcnt lgkmcnt(4)\n\t" \
  "v_mfma_f32_16x16x128_f8f6f4 v[28:31], a[192:199], v[144:151], v[28:31] cbsz:0 blgp:1\n\t" \
  "s_mov_b32 s84, 3342336\n\t" \
  "buffer_store_dword v119, v194, s[44:47], s84 offen offset:320 sc1\n\t" \
  "v_mfma_f32_16x16x128_f8f6f4 v[60:63], a[200:207], v[144:151], v[60:63] cbsz:0 blgp:1\n\t" \
  "v_mfma_f32_16x16x128_f8f6f4 v[92:95], a[208:215], v[144:151], v[92:95] cbsz:0 blgp:1\n\t" \
  "v_mul_f32 v24, s81, v24\n\t" \
  "v_mfma_f32_16x16x128_f8f6f4 v[124:127], a[216:223], v[144:151], v[124:127] cbsz:0 blgp:1\n\t" \
  "s_mov_b32 s84, 0\n\t" \
  "buffer_store_dword v24, v194, s[44:47], s84 offen offset:384 sc1\n\t" \
  "v_mul_f32 v25, s81, v25\n\t" \
  "s_waitcnt lgkmcnt(0)\n\t" \
  "v_mfma_f32_16x16x128_f8f6f4 v[28:31], a[224:231], v[152:159], v[28:31] cbsz:0 blgp:1\n\t" \
  "s_mov_b32 s84, 65536\n\t" \
  "buffer_store_dword v25, v194, s[44:47], s84 offen offset:384 sc1\n\t" \
  "v_mul_f32 v26, s81, v26\n\t" \
  "v_mfma_f32_16x16x128_f8f6f4 v[60:63], a[232:239], v[152:159], v[60:63] cbsz:0 blgp:1\n\t" \
  "s_mov_b32 s84, 131072\n\t" \
  "buffer_store_dword v26, v194, s[44:47], s84 offen offset:384 sc1\n\t" \
  "v_mul_f32 v27, s81, v27\n\t" \
  "v_mfma_f32_16x16x128_f8f6f4 v[92:95], a[240:247], v[152:159], v[92:95] cbsz:0 blgp:1\n\t" \
  "s_mov_b32 s84, 196608\n\t" \
  "buffer_store_dword v27, v194, s[44:47], s84 offen offset:384 sc1\n\t" \
  "v_mfma_f32_16x16x128_f8f6f4 v[124:127], a[248:255], v[152:159], v[124:127] cbsz:0 blgp:1\n\t" \
  "s_nop 7\n\t" \
  "s_nop 3\n\t" \
  "v_mul_f32 v56, s81, v56\n\t" \
  "v_mul_f32 v57, s81, v57\n\t" \
  "v_mul_f32 v58, s81, v58\n\t" \
  "v_mul_f32 v59, s81, v59\n\t" \
  "s_mov_b32 s84, 1048576\n\t" \
  "buffer_store_dword v56, v194, s[44:47], s84 offen offset:384 sc1\n\t" \
  "s_mov_b32 s84, 1114112\n\t" \
  "buffer_store_dword v57, v194, s[44:47], s84 offen offset:384 sc1\n\t" \
  "s_mov_b32 s84, 1179648\n\t" \
  "buffer_store_dword v58, v194, s[44:47], s84 offen offset:384 sc1\n\t" \
  "s_mov_b32 s84, 1245184\n\t" \
  "buffer_store_dword v59, v194, s[44:47], s84 offen offset:384 sc1\n\t" \
  "v_mul_f32 v88, s81, v88\n\t" \
  "v_mul_f32 v89, s81, v89\n\t" \
  "v_mul_f32 v90, s81, v90\n\t" \
  "v_mul_f32 v91, s81, v91\n\t" \
  "s_mov_b32 s84, 2097152\n\t" \
  "buffer_store_dword v88, v194, s[44:47], s84 offen offset:384 sc1\n\t" \
  "s_mov_b32 s84, 2162688\n\t" \
  "buffer_store_dword v89, v194, s[44:47], s84 offen offset:384 sc1\n\t" \
  "s_mov_b32 s84, 2228224\n\t" \
  "buffer_store_dword v90, v194, s[44:47], s84 offen offset:384 sc1\n\t" \
  "s_mov_b32 s84, 2293760\n\t" \
  "buffer_store_dword v91, v194, s[44:47], s84 offen offset:384 sc1\n\t" \
  "v_mul_f32 v120, s81, v120\n\t" \
  "v_mul_f32 v121, s81, v121\n\t" \
  "v_mul_f32 v122, s81, v122\n\t" \
  "v_mul_f32 v123, s81, v123\n\t" \
  "s_mov_b32 s84, 3145728\n\t" \
  "buffer_store_dword v120, v194, s[44:47], s84 offen offset:384 sc1\n\t" \
  "s_mov_b32 s84, 3211264\n\t" \
  "buffer_store_dword v121, v194, s[44:47], s84 offen offset:384 sc1\n\t" \
  "s_mov_b32 s84, 3276800\n\t" \
  "buffer_store_dword v122, v194, s[44:47], s84 offen offset:384 sc1\n\t" \
  "s_mov_b32 s84, 3342336\n\t" \
  "buffer_store_dword v123, v194, s[44:47], s84 offen offset:384 sc1\n\t" \
  "v_mul_f32 v28, s81, v28\n\t" \
  "v_mul_f32 v29, s81, v29\n\t" \
  "v_mul_f32 v30, s81, v30\n\t" \
  "v_mul_f32 v31, s81, v31\n\t" \
  "s_mov_b32 s84, 0\n\t" \
  "buffer_store_dword v28, v194, s[44:47], s84 offen offset:448 sc1\n\t" \
  "s_mov_b32 s84, 65536\n\t" \
  "buffer_store_dword v29, v194, s[44:47], s84 offen offset:448 sc1\n\t" \
  "s_mov_b32 s84, 131072\n\t" \
  "buffer_store_dword v30, v194, s[44:47], s84 offen offset:448 sc1\n\t" \
  "s_mov_b32 s84, 196608\n\t" \
  "buffer_store_dword v31, v194, s[44:47], s84 offen offset:448 sc1\n\t" \
  "v_mul_f32 v60, s81, v60\n\t" \
  "v_mul_f32 v61, s81, v61\n\t" \
  "v_mul_f32 v62, s81, v62\n\t" \
  "v_mul_f32 v63, s81, v63\n\t" \
  "s_mov_b32 s84, 1048576\n\t" \
  "buffer_store_dword v60, v194, s[44:47], s84 offen offset:448 sc1\n\t" \
  "s_mov_b32 s84, 1114112\n\t" \
  "buffer_store_dword v61, v194, s[44:47], s84 offen offset:448 sc1\n\t" \
  "s_mov_b32 s84, 1179648\n\t" \
  "buffer_store_dword v62, v194, s[44:47], s84 offen offset:448 sc1\n\t" \
  "s_mov_b32 s84, 1245184\n\t" \
  "buffer_store_dword v63, v194, s[44:47], s84 offen offset:448 sc1\n\t" \
  "v_mul_f32 v92, s81, v92\n\t" \
  "v_mul_f32 v93, s81, v93\n\t" \
  "v_mul_f32 v94, s81, v94\n\t" \
  "v_mul_f32 v95, s81, v95\n\t" \
  "s_mov_b32 s84, 2097152\n\t" \
  "buffer_store_dword v92, v194, s[44:47], s84 offen offset:448 sc1\n\t" \
  "s_mov_b32 s84, 2162688\n\t" \
  "buffer_store_dword v93, v194, s[44:47], s84 offen offset:448 sc1\n\t" \
  "s_mov_b32 s84, 2228224\n\t" \
  "buffer_store_dword v94, v194, s[44:47], s84 offen offset:448 sc1\n\t" \
  "s_mov_b32 s84, 2293760\n\t" \
  "buffer_store_dword v95, v194, s[44:47], s84 offen offset:448 sc1\n\t" \
  "v_mul_f32 v124, s81, v124\n\t" \
  "v_mul_f32 v125, s81, v125\n\t" \
  "v_mul_f32 v126, s81, v126\n\t" \
  "v_mul_f32 v127, s81, v127\n\t" \
  "s_mov_b32 s84, 3145728\n\t" \
  "buffer_store_dword v124, v194, s[44:47], s84 offen offset:448 sc1\n\t" \
  "s_mov_b32 s84, 3211264\n\t" \
  "buffer_store_dword v125, v194, s[44:47], s84 offen offset:448 sc1\n\t" \
  "s_mov_b32 s84, 3276800\n\t" \
  "buffer_store_dword v126, v194, s[44:47], s84 offen offset:448 sc1\n\t" \
  "s_mov_b32 s84, 3342336\n\t" \
  "buffer_store_dword v127, v194, s[44:47], s84 offen offset:448 sc1\n\t" \
  "s_endpgm\n\t" \
  "L_slow_0:\n\t" \
  "L_ap2_0:\n\t" \
  "global_load_dword v196, v195, s[26:27] offset:1024 sc1\n\t" \
  "s_branch L_apr2_0\n\t" \
  "L_ap3_0:\n\t" \
  "global_load_dword v196, v195, s[26:27] offset:1024 sc1\n\t" \
  "s_branch L_apr3_0\n\t" \
  "L_ap5_0:\n\t" \
  "global_load_dword v196, v195, s[26:27] offset:2048 sc1\n\t" \
  "s_branch L_apr5_0\n\t" \
  "L_ap6_0:\n\t" \
  "global_load_dword v196, v195, s[26:27] offset:2048 sc1\n\t" \
  "s_branch L_apr6_0\n\t" \
  "L_ap7_0:\n\t" \
  "global_load_dword v196, v195, s[26:27] offset:2048 sc1\n\t" \
  "s_branch L_apr7_0\n\t" \
  "L_ap8_0:\n\t" \
  "global_load_dword v196, v195, s[26:27] offset:2048 sc1\n\t" \
  "s_branch L_apr8_0\n\t" \
  "L_ap9_0:\n\t" \
  "global_load_dword v196, v195, s[26:27] offset:2048 sc1\n\t" \
  "s_branch L_apr9_0\n\t" \
  "L_ap10_0:\n\t" \
  "global_load_dword v196, v195, s[26:27] offset:2048 sc1\n\t" \
  "s_branch L_apr10_0\n\t" \
  "L_ap12_0:\n\t" \
  "global_load_dword v196, v195, s[26:27] offset:3072 sc1\n\t" \
  "s_branch L_apr12_0\n\t" \
  "L_ap13_0:\n\t" \
  "global_load_dword v196, v195, s[26:27] offset:3072 sc1\n\t" \
  "s_branch L_apr13_0\n\t" \
  "L_ap14_0:\n\t" \
  "global_load_dword v196, v195, s[26:27] offset:3072 sc1\n\t" \
  "s_branch L_apr14_0\n\t" \
  "L_ap15_0:\n\t" \
  "global_load_dword v196, v195, s[26:27] offset:3072 sc1\n\t" \
  "s_branch L_apr15_0\n\t" \
  "L_ap16_0:\n\t" \
  "global_load_dword v196, v195, s[26:27] offset:3072 sc1\n\t" \
  "s_branch L_apr16_0\n\t" \
  "L_ap17_0:\n\t" \
  "global_load_dword v196, v195, s[26:27] offset:3072 sc1\n\t" \
  "s_branch L_apr17_0\n\t" \
  "L_au2_0:\n\t" \
  "s_waitcnt vmcnt(1)\n\t" \
  "v_cmp_le_u32 vcc, s79, v196\n\t" \
  "s_nop 4\n\t" \
  "s_andn2_b32 s85, vcc_lo, s96\n\t" \
  "s_or_b32 s96, s96, vcc_lo\n\t" \
  "s_and_b32 exec_lo, s85, 0xfffffff\n\t" \
  "s_mov_b32 exec_hi, 0\n\t" \
  "s_cbranch_execz L_u5_0\n\t" \
  "s_mov_b32 s84, 458752\n\t" \
  "buffer_load_dword v210, v198, s[36:39], s84 offen \n\t" \
  "L_u5_0:\n\t" \
  "s_mov_b64 exec, -1\n\t" \
  "s_branch L_aur2_0\n\t" \
  "L_au3_0:\n\t" \
  "s_waitcnt vmcnt(12)\n\t" \
  "v_cmp_le_u32 vcc, s79, v196\n\t" \
  "s_nop 4\n\t" \
  "s_andn2_b32 s85, vcc_lo, s96\n\t" \
  "s_or_b32 s96, s96, vcc_lo\n\t" \
  "s_and_b32 exec_lo, s85, 0xfffffff\n\t" \
  "s_mov_b32 exec_hi, 0\n\t" \
  "s_cbranch_execz L_u6_0\n\t" \
  "s_mov_b32 s84, 458752\n\t" \
  "buffer_load_dword v210, v198, s[36:39], s84 offen \n\t" \
  "L_u6_0:\n\t" \
  "s_mov_b64 exec, -1\n\t" \
  "s_branch L_aur3_0\n\t" \
  "L_au5_0:\n\t" \
  "s_waitcnt vmcnt(4)\n\t" \
  "v_cmp_le_u32 vcc, s79, v196\n\t" \
  "s_nop 4\n\t" \
  "s_andn2_b32 s85, vcc_lo, s8\n\t" \
  "s_or_b32 s8, s8, vcc_lo\n\t" \
  "s_and_b32 exec_lo, s85, 0xfffffff\n\t" \
  "s_mov_b32 exec_hi, 0\n\t" \
  "s_cbranch_execz L_u7_0\n\t" \
  "s_mov_b32 s84, 917504\n\t" \
  "buffer_load_dword v210, v198, s[36:39], s84 offen \n\t" \
  "L_u7_0:\n\t" \
  "s_mov_b64 exec, -1\n\t" \
  "s_branch L_aur5_0\n\t" \
  "L_au6_0:\n\t" \
  "s_waitcnt vmcnt(9)\n\t" \
  "v_cmp_le_u32 vcc, s79, v196\n\t" \
  "s_nop 4\n\t" \
  "s_andn2_b32 s85, vcc_lo, s8\n\t" \
  "s_or_b32 s8, s8, vcc_lo\n\t" \
  "s_and_b32 exec_lo, s85, 0xfffffff\n\t" \
  "s_mov_b32 exec_hi, 0\n\t" \
  "s_cbranch_execz L_u8_0\n\t" \
  "s_mov_b32 s84, 917504\n\t" \
  "buffer_load_dword v210, v198, s[36:39], s84 offen \n\t" \
  "L_u8_0:\n\t" \
  "s_mov_b64 exec, -1\n\t" \
  "s_branch L_aur6_0\n\t" \
  "L_au7_0:\n\t" \
  "s_waitcnt vmcnt(10)\n\t" \
  "v_cmp_le_u32 vcc, s79, v196\n\t" \
  "s_nop 4\n\t" \
  "s_andn2_b32 s85, vcc_lo, s8\n\t" \
  "s_or_b32 s8, s8, vcc_lo\n\t" \
  "s_and_b32 exec_lo, s85, 0xfffffff\n\t" \
  "s_mov_b32 exec_hi, 0\n\t" \
  "s_cbranch_execz L_u9_0\n\t" \
  "s_mov_b32 s84, 917504\n\t" \
  "buffer_load_dword v210, v198, s[36:39], s84 offen \n\t" \
  "L_u9_0:\n\t" \
  "s_mov_b64 exec, -1\n\t" \
  "s_branch L_aur7_0\n\t" \
  "L_au8_0:\n\t" \
  "s_waitcnt vmcnt(13)\n\t" \
  "v_cmp_le_u32 vcc, s79, v196\n\t" \
  "s_nop 4\n\t" \
  "s_andn2_b32 s85, vcc_lo, s8\n\t" \
  "s_or_b32 s8, s8, vcc_lo\n\t" \
  "s_and_b32 exec_lo, s85, 0xfffffff\n\t" \
  "s_mov_b32 exec_hi, 0\n\t" \
  "s_cbranch_execz L_u10_0\n\t" \
  "s_mov_b32 s84, 917504\n\t" \
  "buffer_load_dword v210, v198, s[36:39], s84 offen \n\t" \
  "L_u10_0:\n\t" \
  "s_mov_b64 exec, -1\n\t" \
  "s_branch L_aur8_0\n\t" \
  "L_au9_0:\n\t" \
  "s_waitcnt vmcnt(2)\n\t" \
  "v_cmp_le_u32 vcc, s79, v196\n\t" \
  "s_nop 4\n\t" \
  "s_andn2_b32 s85, vcc_lo, s8\n\t" \
  "s_or_b32 s8, s8, vcc_lo\n\t" \
  "s_and_b32 exec_lo, s85, 0xfffffff\n\t" \
  "s_mov_b32 exec_hi, 0\n\t" \
  "s_cbranch_execz L_u11_0\n\t" \
  "s_mov_b32 s84, 917504\n\t" \
  "buffer_load_dword v210, v198, s[36:39], s84 offen \n\t" \
  "L_u11_0:\n\t" \
  "s_mov_b64 exec, -1\n\t" \
  "s_branch L_aur9_0\n\t" \
  "L_au10_0:\n\t" \
  "s_waitcnt vmcnt(13)\n\t" \
  "v_cmp_le_u32 vcc, s79, v196\n\t" \
  "s_nop 4\n\t" \
  "s_andn2_b32 s85, vcc_lo, s8\n\t" \
  "s_or_b32 s8, s8, vcc_lo\n\t" \
  "s_and_b32 exec_lo, s85, 0xfffffff\n\t" \
  "s_mov_b32 exec_hi, 0\n\t" \
  "s_cbranch_execz L_u12_0\n\t" \
  "s_mov_b32 s84, 917504\n\t" \
  "buffer_load_dword v210, v198, s[36:39], s84 offen \n\t" \
  "L_u12_0:\n\t" \
  "s_mov_b64 exec, -1\n\t" \
  "s_branch L_aur10_0\n\t" \
  "L_au12_0:\n\t" \
  "s_waitcnt vmcnt(8)\n\t" \
  "v_cmp_le_u32 vcc, s79, v196\n\t" \
  "s_nop 4\n\t" \
  "s_andn2_b32 s85, vcc_lo, s9\n\t" \
  "s_or_b32 s9, s9, vcc_lo\n\t" \
  "s_and_b32 exec_lo, s85, 0xfffffff\n\t" \
  "s_mov_b32 exec_hi, 0\n\t" \
  "s_cbranch_execz L_u13_0\n\t" \
  "s_mov_b32 s84, 1376256\n\t" \
  "buffer_load_dword v210, v198, s[36:39], s84 offen \n\t" \
  "L_u13_0:\n\t" \
  "s_mov_b64 exec, -1\n\t" \
  "s_branch L_aur12_0\n\t" \
  "L_au13_0:\n\t" \
  "s_waitcnt vmcnt(2)\n\t" \
  "v_cmp_le_u32 vcc, s79, v196\n\t" \
  "s_nop 4\n\t" \
  "s_andn2_b32 s85, vcc_lo, s9\n\t" \
  "s_or_b32 s9, s9, vcc_lo\n\t" \
  "s_and_b32 exec_lo, s85, 0xfffffff\n\t" \
  "s_mov_b32 exec_hi, 0\n\t" \
  "s_cbranch_execz L_u14_0\n\t" \
  "s_mov_b32 s84, 1376256\n\t" \
  "buffer_load_dword v210, v198, s[36:39], s84 offen \n\t" \
  "L_u14_0:\n\t" \
  "s_mov_b64 exec, -1\n\t" \
  "s_branch L_aur13_0\n\t" \
  "L_au14_0:\n\t" \
  "s_waitcnt vmcnt(13)\n\t" \
  "v_cmp_le_u32 vcc, s79, v196\n\t" \
  "s_nop 4\n\t" \
  "s_andn2_b32 s85, vcc_lo, s9\n\t" \
  "s_or_b32 s9, s9, vcc_lo\n\t" \
  "s_and_b32 exec_lo, s85, 0xfffffff\n\t" \
  "s_mov_b32 exec_hi, 0\n\t" \
  "s_cbranch_execz L_u15_0\n\t" \
  "s_mov_b32 s84, 1376256\n\t" \
  "buffer_load_dword v210, v198, s[36:39], s84 offen \n\t" \
  "L_u15_0:\n\t" \
  "s_mov_b64 exec, -1\n\t" \
  "s_branch L_aur14_0\n\t" \
  "L_au15_0:\n\t" \
  "s_waitcnt vmcnt(10)\n\t" \
  "v_cmp_le_u32 vcc, s79, v196\n\t" \
  "s_nop 4\n\t" \
  "s_andn2_b32 s85, vcc_lo, s9\n\t" \
  "s_or_b32 s9, s9, vcc_lo\n\t" \
  "s_and_b32 exec_lo, s85, 0xfffffff\n\t" \
  "s_mov_b32 exec_hi, 0\n\t" \
  "s_cbranch_execz L_u16_0\n\t" \
  "s_mov_b32 s84, 1376256\n\t" \
  "buffer_load_dword v210, v198, s[36:39], s84 offen \n\t" \
  "L_u16_0:\n\t" \
  "s_mov_b64 exec, -1\n\t" \
  "s_branch L_aur15_0\n\t" \
  "L_au16_0:\n\t" \
  "s_waitcnt vmcnt(9)\n\t" \
  "v_cmp_le_u32 vcc, s79, v196\n\t" \
  "s_nop 4\n\t" \
  "s_andn2_b32 s85, vcc_lo, s9\n\t" \
  "s_or_b32 s9, s9, vcc_lo\n\t" \
  "s_and_b32 exec_lo, s85, 0xfffffff\n\t" \
  "s_mov_b32 exec_hi, 0\n\t" \
  "s_cbranch_execz L_u17_0\n\t" \
  "s_mov_b32 s84, 1376256\n\t" \
  "buffer_load_dword v210, v198, s[36:39], s84 offen \n\t" \
  "L_u17_0:\n\t" \
  "s_mov_b64 exec, -1\n\t" \
  "s_branch L_aur16_0\n\t" \
  "L_au17_0:\n\t" \
  "s_waitcnt vmcnt(10)\n\t" \
  "v_cmp_le_u32 vcc, s79, v196\n\t" \
  "s_nop 4\n\t" \
  "s_andn2_b32 s85, vcc_lo, s9\n\t" \
  "s_or_b32 s9, s9, vcc_lo\n\t" \
  "s_and_b32 exec_lo, s85, 0xfffffff\n\t" \
  "s_mov_b32 exec_hi, 0\n\t" \
  "s_cbranch_execz L_u18_0\n\t" \
  "s_mov_b32 s84, 1376256\n\t" \
  "buffer_load_dword v210, v198, s[36:39], s84 offen \n\t" \
  "L_u18_0:\n\t" \
  "s_mov_b64 exec, -1\n\t" \
  "s_branch L_aur17_0\n\t" \
  "L_v4_0:\n\t" \
  "s_mov_b32 s101, 0\n\t" \
  "s_mov_b32 s98, s87\n\t" \
  "L_vl4_0:\n\t" \
  "s_memrealtime s[88:89]\n\t" \
  "s_cmp_ge_u32 s82, 18\n\t" \
  "s_cbranch_scc1 L_vn4_0\n\t" \
  "s_waitcnt lgkmcnt(0)\n\t" \
  "s_sub_u32 s84, s88, s14\n\t" \
  "s_cmp_lt_u32 s84, 520\n\t" \
  "s_cbranch_scc1 L_vn4_0\n\t" \
  "v_readlane_b32 s85, v211, s82\n\t" \
  "s_add_u32 s82, s82, 4\n\t" \
  "s_nop 3\n\t" \
  "buffer_load_dword v210, v197, s[60:63], s85 offen\n\t" \
  "L_vn4_0:\n\t" \
  "s_mov_b32 s87, 0\n\t" \
  "buffer_load_dwordx4 a[128:131], v188, s[36:39], s87 offen offset:0 sc0 sc1\n\t" \
  "buffer_load_dwordx4 a[132:135], v188, s[36:39], s87 offen offset:1024 sc0 sc1\n\t" \
  "buffer_load_dwordx4 a[136:139], v188, s[36:39], s87 offen offset:2048 sc0 sc1\n\t" \
  "buffer_load_dwordx4 a[140:143], v188, s[36:39], s87 offen offset:3072 sc0 sc1\n\t" \
  "buffer_load_dwordx4 a[144:147], v189, s[36:39], s87 offen offset:0 sc0 sc1\n\t" \
  "buffer_load_dwordx4 a[148:151], v189, s[36:39], s87 offen offset:1024 sc0 sc1\n\t" \
  "buffer_load_dwordx4 a[152:155], v189, s[36:39], s87 offen offset:2048 sc0 sc1\n\t" \
  "buffer_load_dwordx4 a[156:159], v189, s[36:39], s87 offen offset:3072 sc0 sc1\n\t" \
  "global_load_dword v226, v195, s[26:27] offset:0 sc0 sc1\n\t" \
  "s_memrealtime s[88:89]\n\t" \
  "s_waitcnt vmcnt(0) lgkmcnt(0)\n\t" \
  "v_accvgpr_read_b32 v237, a128\n\t" \
  "v_accvgpr_read_b32 v238, a129\n\t" \
  "v_accvgpr_read_b32 v239, a130\n\t" \
  "v_accvgpr_read_b32 v240, a131\n\t" \
  "v_min3_u32 v236, v237, v238, v237\n\t" \
  "v_min3_u32 v236, v239, v240, v236\n\t" \
  "v_accvgpr_read_b32 v237, a132\n\t" \
  "v_accvgpr_read_b32 v238, a133\n\t" \
  "v_accvgpr_read_b32 v239, a134\n\t" \
  "v_accvgpr_read_b32 v240, a135\n\t" \
  "v_min3_u32 v236, v237, v238, v236\n\t" \
  "v_min3_u32 v236, v239, v240, v236\n\t" \
  "v_accvgpr_read_b32 v237, a136\n\t" \
  "v_accvgpr_read_b32 v238, a137\n\t" \
  "v_accvgpr_read_b32 v239, a138\n\t" \
  "v_accvgpr_read_b32 v240, a139\n\t" \
  "v_min3_u32 v236, v237, v238, v236\n\t" \
  "v_min3_u32 v236, v239, v240, v236\n\t" \
  "v_accvgpr_read_b32 v237, a140\n\t" \
  "v_accvgpr_read_b32 v238, a141\n\t" \
  "v_accvgpr_read_b32 v239, a142\n\t" \
  "v_accvgpr_read_b32 v240, a143\n\t" \
  "v_min3_u32 v236, v237, v238, v236\n\t" \
  "v_min3_u32 v236, v239, v240, v236\n\t" \
  "v_accvgpr_read_b32 v237, a144\n\t" \
  "v_accvgpr_read_b32 v238, a145\n\t" \
  "v_accvgpr_read_b32 v239, a146\n\t" \
  "v_accvgpr_read_b32 v240, a147\n\t" \
  "v_min3_u32 v236, v237, v238, v236\n\t" \
  "v_min3_u32 v236, v239, v240, v236\n\t" \
  "v_accvgpr_read_b32 v237, a148\n\t" \
  "v_accvgpr_read_b32 v238, a149\n\t" \
  "v_accvgpr_read_b32 v239, a150\n\t" \
  "v_accvgpr_read_b32 v240, a151\n\t" \
  "v_min3_u32 v236, v237, v238, v236\n\t" \
  "v_min3_u32 v236, v239, v240, v236\n\t" \
  "v_accvgpr_read_b32 v237, a152\n\t" \
  "v_accvgpr_read_b32 v238, a153\n\t" \
  "v_accvgpr_read_b32 v239, a154\n\t" \
  "v_accvgpr_read_b32 v240, a155\n\t" \
  "v_min3_u32 v236, v237, v238, v236\n\t" \
  "v_min3_u32 v236, v239, v240, v236\n\t" \
  "v_accvgpr_read_b32 v237, a156\n\t" \
  "v_accvgpr_read_b32 v238, a157\n\t" \
  "v_accvgpr_read_b32 v239, a158\n\t" \
  "v_accvgpr_read_b32 v240, a159\n\t" \
  "v_min3_u32 v236, v237, v238, v236\n\t" \
  "v_min3_u32 v236, v239, v240, v236\n\t" \
  "v_cmp_eq_u32 vcc, 0, v236\n\t" \
  "s_nop 4\n\t" \
  "s_cmp_eq_u64 vcc, 0\n\t" \
  "s_cbranch_scc1 L_vd4_0\n\t" \
  "s_cmp_eq_u32 s101, 15\n\t" \
  "s_cbranch_scc1 L_vd4_0\n\t" \
  "v_cmp_le_u32 vcc, s79, v226\n\t" \
  "s_nop 4\n\t" \
  "s_bfe_u32 s101, vcc_lo, 262144\n\t" \
  "s_sub_u32 s84, s88, s14\n\t" \
  "s_or_b32 s84, s84, s97\n\t" \
  "s_cmp_lt_u32 s84, 50000000\n\t" \
  "s_cbranch_scc1 L_vl4_0\n\t" \
  "s_mov_b32 s97, -1\n\t" \
  "s_mov_b32 s81, 0x7fc00000\n\t" \
  "L_vd4_0:\n\t" \
  "s_mov_b64 exec, 0xfffffff\n\t" \
  "s_mov_b32 s84, 0\n\t" \
  "buffer_load_dword v210, v198, s[36:39], s84 offen sc1\n\t" \
  "s_mov_b64 exec, -1\n\t" \
  "s_mov_b32 s87, s98\n\t" \
  "s_branch L_vr4_0\n\t" \
  "L_v5_0:\n\t" \
  "s_mov_b32 s101, 0\n\t" \
  "s_mov_b32 s98, s87\n\t" \
  "L_vl5_0:\n\t" \
  "s_memrealtime s[88:89]\n\t" \
  "s_cmp_ge_u32 s82, 18\n\t" \
  "s_cbranch_scc1 L_vn5_0\n\t" \
  "s_waitcnt lgkmcnt(0)\n\t" \
  "s_sub_u32 s84, s88, s14\n\t" \
  "s_cmp_lt_u32 s84, 520\n\t" \
  "s_cbranch_scc1 L_vn5_0\n\t" \
  "v_readlane_b32 s85, v211, s82\n\t" \
  "s_add_u32 s82, s82, 4\n\t" \
  "s_nop 3\n\t" \
  "buffer_load_dword v210, v197, s[60:63], s85 offen\n\t" \
  "L_vn5_0:\n\t" \
  "s_mov_b32 s87, 65536\n\t" \
  "buffer_load_dwordx4 a[160:163], v188, s[36:39], s87 offen offset:0 sc0 sc1\n\t" \
  "buffer_load_dwordx4 a[164:167], v188, s[36:39], s87 offen offset:1024 sc0 sc1\n\t" \
  "buffer_load_dwordx4 a[168:171], v188, s[36:39], s87 offen offset:2048 sc0 sc1\n\t" \
  "buffer_load_dwordx4 a[172:175], v188, s[36:39], s87 offen offset:3072 sc0 sc1\n\t" \
  "buffer_load_dwordx4 a[176:179], v189, s[36:39], s87 offen offset:0 sc0 sc1\n\t" \
  "buffer_load_dwordx4 a[180:183], v189, s[36:39], s87 offen offset:1024 sc0 sc1\n\t" \
  "buffer_load_dwordx4 a[184:187], v189, s[36:39], s87 offen offset:2048 sc0 sc1\n\t" \
  "buffer_load_dwordx4 a[188:191], v189, s[36:39], s87 offen offset:3072 sc0 sc1\n\t" \
  "global_load_dword v226, v195, s[26:27] offset:0 sc0 sc1\n\t" \
  "s_memrealtime s[88:89]\n\t" \
  "s_waitcnt vmcnt(0) lgkmcnt(0)\n\t" \
  "v_accvgpr_read_b32 v237, a160\n\t" \
  "v_accvgpr_read_b32 v238, a161\n\t" \
  "v_accvgpr_read_b32 v239, a162\n\t" \
  "v_accvgpr_read_b32 v240, a163\n\t" \
  "v_min3_u32 v236, v237, v238, v237\n\t" \
  "v_min3_u32 v236, v239, v240, v236\n\t" \
  "v_accvgpr_read_b32 v237, a164\n\t" \
  "v_accvgpr_read_b32 v238, a165\n\t" \
  "v_accvgpr_read_b32 v239, a166\n\t" \
  "v_accvgpr_read_b32 v240, a167\n\t" \
  "v_min3_u32 v236, v237, v238, v236\n\t" \
  "v_min3_u32 v236, v239, v240, v236\n\t" \
  "v_accvgpr_read_b32 v237, a168\n\t" \
  "v_accvgpr_read_b32 v238, a169\n\t" \
  "v_accvgpr_read_b32 v239, a170\n\t" \
  "v_accvgpr_read_b32 v240, a171\n\t" \
  "v_min3_u32 v236, v237, v238, v236\n\t" \
  "v_min3_u32 v236, v239, v240, v236\n\t" \
  "v_accvgpr_read_b32 v237, a172\n\t" \
  "v_accvgpr_read_b32 v238, a173\n\t" \
  "v_accvgpr_read_b32 v239, a174\n\t" \
  "v_accvgpr_read_b32 v240, a175\n\t" \
  "v_min3_u32 v236, v237, v238, v236\n\t" \
  "v_min3_u32 v236, v239, v240, v236\n\t" \
  "v_accvgpr_read_b32 v237, a176\n\t" \
  "v_accvgpr_read_b32 v238, a177\n\t" \
  "v_accvgpr_read_b32 v239, a178\n\t" \
  "v_accvgpr_read_b32 v240, a179\n\t" \
  "v_min3_u32 v236, v237, v238, v236\n\t" \
  "v_min3_u32 v236, v239, v240, v236\n\t" \
  "v_accvgpr_read_b32 v237, a180\n\t" \
  "v_accvgpr_read_b32 v238, a181\n\t" \
  "v_accvgpr_read_b32 v239, a182\n\t" \
  "v_accvgpr_read_b32 v240, a183\n\t" \
  "v_min3_u32 v236, v237, v238, v236\n\t" \
  "v_min3_u32 v236, v239, v240, v236\n\t" \
  "v_accvgpr_read_b32 v237, a184\n\t" \
  "v_accvgpr_read_b32 v238, a185\n\t" \
  "v_accvgpr_read_b32 v239, a186\n\t" \
  "v_accvgpr_read_b32 v240, a187\n\t" \
  "v_min3_u32 v236, v237, v238, v236\n\t" \
  "v_min3_u32 v236, v239, v240, v236\n\t" \
  "v_accvgpr_read_b32 v237, a188\n\t" \
  "v_accvgpr_read_b32 v238, a189\n\t" \
  "v_accvgpr_read_b32 v239, a190\n\t" \
  "v_accvgpr_read_b32 v240, a191\n\t" \
  "v_min3_u32 v236, v237, v238, v236\n\t" \
  "v_min3_u32 v236, v239, v240, v236\n\t" \
  "v_cmp_eq_u32 vcc, 0, v236\n\t" \
  "s_nop 4\n\t" \
  "s_cmp_eq_u64 vcc, 0\n\t" \
  "s_cbranch_scc1 L_vd5_0\n\t" \
  "s_cmp_eq_u32 s101, 15\n\t" \
  "s_cbranch_scc1 L_vd5_0\n\t" \
  "v_cmp_le_u32 vcc, s79, v226\n\t" \
  "s_nop 4\n\t" \
  "s_bfe_u32 s101, vcc_lo, 262148\n\t" \
  "s_sub_u32 s84, s88, s14\n\t" \
  "s_or_b32 s84, s84, s97\n\t" \
  "s_cmp_lt_u32 s84, 50000000\n\t" \
  "s_cbranch_scc1 L_vl5_0\n\t" \
  "s_mov_b32 s97, -1\n\t" \
  "s_mov_b32 s81, 0x7fc00000\n\t" \
  "L_vd5_0:\n\t" \
  "s_mov_b64 exec, 0xfffffff\n\t" \
  "s_mov_b32 s84, 0\n\t" \
  "buffer_load_dword v210, v198, s[36:39], s84 offen sc1\n\t" \
  "s_mov_b64 exec, -1\n\t" \
  "s_mov_b32 s87, s98\n\t" \
  "s_branch L_vr5_0\n\t" \
  "L_v6_0:\n\t" \
  "s_mov_b32 s101, 0\n\t" \
  "s_mov_b32 s98, s87\n\t" \
  "L_vl6_0:\n\t" \
  "s_memrealtime s[88:89]\n\t" \
  "s_cmp_ge_u32 s82, 18\n\t" \
  "s_cbranch_scc1 L_vn6_0\n\t" \
  "s_waitcnt lgkmcnt(0)\n\t" \
  "s_sub_u32 s84, s88, s14\n\t" \
  "s_cmp_lt_u32 s84, 520\n\t" \
  "s_cbranch_scc1 L_vn6_0\n\t" \
  "v_readlane_b32 s85, v211, s82\n\t" \
  "s_add_u32 s82, s82, 4\n\t" \
  "s_nop 3\n\t" \
  "buffer_load_dword v210, v197, s[60:63], s85 offen\n\t" \
  "L_vn6_0:\n\t" \
  "s_mov_b32 s87, 131072\n\t" \
  "buffer_load_dwordx4 a[192:195], v188, s[36:39], s87 offen offset:0 sc0 sc1\n\t" \
  "buffer_load_dwordx4 a[196:199], v188, s[36:39], s87 offen offset:1024 sc0 sc1\n\t" \
  "buffer_load_dwordx4 a[200:203], v188, s[36:39], s87 offen offset:2048 sc0 sc1\n\t" \
  "buffer_load_dwordx4 a[204:207], v188, s[36:39], s87 offen offset:3072 sc0 sc1\n\t" \
  "buffer_load_dwordx4 a[208:211], v189, s[36:39], s87 offen offset:0 sc0 sc1\n\t" \
  "buffer_load_dwordx4 a[212:215], v189, s[36:39], s87 offen offset:1024 sc0 sc1\n\t" \
  "buffer_load_dwordx4 a[216:219], v189, s[36:39], s87 offen offset:2048 sc0 sc1\n\t" \
  "buffer_load_dwordx4 a[220:223], v189, s[36:39], s87 offen offset:3072 sc0 sc1\n\t" \
  "global_load_dword v226, v195, s[26:27] offset:0 sc0 sc1\n\t" \
  "s_memrealtime s[88:89]\n\t" \
  "s_waitcnt vmcnt(0) lgkmcnt(0)\n\t" \
  "v_accvgpr_read_b32 v237, a192\n\t" \
  "v_accvgpr_read_b32 v238, a193\n\t" \
  "v_accvgpr_read_b32 v239, a194\n\t" \
  "v_accvgpr_read_b32 v240, a195\n\t" \
  "v_min3_u32 v236, v237, v238, v237\n\t" \
  "v_min3_u32 v236, v239, v240, v236\n\t" \
  "v_accvgpr_read_b32 v237, a196\n\t" \
  "v_accvgpr_read_b32 v238, a197\n\t" \
  "v_accvgpr_read_b32 v239, a198\n\t" \
  "v_accvgpr_read_b32 v240, a199\n\t" \
  "v_min3_u32 v236, v237, v238, v236\n\t" \
  "v_min3_u32 v236, v239, v240, v236\n\t" \
  "v_accvgpr_read_b32 v237, a200\n\t" \
  "v_accvgpr_read_b32 v238, a201\n\t" \
  "v_accvgpr_read_b32 v239, a202\n\t" \
  "v_accvgpr_read_b32 v240, a203\n\t" \
  "v_min3_u32 v236, v237, v238, v236\n\t" \
  "v_min3_u32 v236, v239, v240, v236\n\t" \
  "v_accvgpr_read_b32 v237, a204\n\t" \
  "v_accvgpr_read_b32 v238, a205\n\t" \
  "v_accvgpr_read_b32 v239, a206\n\t" \
  "v_accvgpr_read_b32 v240, a207\n\t" \
  "v_min3_u32 v236, v237, v238, v236\n\t" \
  "v_min3_u32 v236, v239, v240, v236\n\t" \
  "v_accvgpr_read_b32 v237, a208\n\t" \
  "v_accvgpr_read_b32 v238, a209\n\t" \
  "v_accvgpr_read_b32 v239, a210\n\t" \
  "v_accvgpr_read_b32 v240, a211\n\t" \
  "v_min3_u32 v236, v237, v238, v236\n\t" \
  "v_min3_u32 v236, v239, v240, v236\n\t" \
  "v_accvgpr_read_b32 v237, a212\n\t" \
  "v_accvgpr_read_b32 v238, a213\n\t" \
  "v_accvgpr_read_b32 v239, a214\n\t" \
  "v_accvgpr_read_b32 v240, a215\n\t" \
  "v_min3_u32 v236, v237, v238, v236\n\t" \
  "v_min3_u32 v236, v239, v240, v236\n\t" \
  "v_accvgpr_read_b32 v237, a216\n\t" \
  "v_accvgpr_read_b32 v238, a217\n\t" \
  "v_accvgpr_read_b32 v239, a218\n\t" \
  "v_accvgpr_read_b32 v240, a219\n\t" \
  "v_min3_u32 v236, v237, v238, v236\n\t" \
  "v_min3_u32 v236, v239, v240, v236\n\t" \
  "v_accvgpr_read_b32 v237, a220\n\t" \
  "v_accvgpr_read_b32 v238, a221\n\t" \
  "v_accvgpr_read_b32 v239, a222\n\t" \
  "v_accvgpr_read_b32 v240, a223\n\t" \
  "v_min3_u32 v236, v237, v238, v236\n\t" \
  "v_min3_u32 v236, v239, v240, v236\n\t" \
  "v_cmp_eq_u32 vcc, 0, v236\n\t" \
  "s_nop 4\n\t" \
  "s_cmp_eq_u64 vcc, 0\n\t" \
  "s_cbranch_scc1 L_vd6_0\n\t" \
  "s_cmp_eq_u32 s101, 15\n\t" \
  "s_cbranch_scc1 L_vd6_0\n\t" \
  "v_cmp_le_u32 vcc, s79, v226\n\t" \
  "s_nop 4\n\t" \
  "s_bfe_u32 s101, vcc_lo, 262152\n\t" \
  "s_sub_u32 s84, s88, s14\n\t" \
  "s_or_b32 s84, s84, s97\n\t" \
  "s_cmp_lt_u32 s84, 50000000\n\t" \
  "s_cbranch_scc1 L_vl6_0\n\t" \
  "s_mov_b32 s97, -1\n\t" \
  "s_mov_b32 s81, 0x7fc00000\n\t" \
  "L_vd6_0:\n\t" \
  "s_mov_b64 exec, 0xfffffff\n\t" \
  "s_mov_b32 s84, 0\n\t" \
  "buffer_load_dword v210, v198, s[36:39], s84 offen sc1\n\t" \
  "s_mov_b64 exec, -1\n\t" \
  "s_mov_b32 s87, s98\n\t" \
  "s_branch L_vr6_0\n\t" \
  "L_v7_0:\n\t" \
  "s_mov_b32 s101, 0\n\t" \
  "s_mov_b32 s98, s87\n\t" \
  "L_vl7_0:\n\t" \
  "s_memrealtime s[88:89]\n\t" \
  "s_cmp_ge_u32 s82, 18\n\t" \
  "s_cbranch_scc1 L_vn7_0\n\t" \
  "s_waitcnt lgkmcnt(0)\n\t" \
  "s_sub_u32 s84, s88, s14\n\t" \
  "s_cmp_lt_u32 s84, 520\n\t" \
  "s_cbranch_scc1 L_vn7_0\n\t" \
  "v_readlane_b32 s85, v211, s82\n\t" \
  "s_add_u32 s82, s82, 4\n\t" \
  "s_nop 3\n\t" \
  "buffer_load_dword v210, v197, s[60:63], s85 offen\n\t" \
  "L_vn7_0:\n\t" \
  "s_mov_b32 s87, 196608\n\t" \
  "buffer_load_dwordx4 a[224:227], v188, s[36:39], s87 offen offset:0 sc0 sc1\n\t" \
  "buffer_load_dwordx4 a[228:231], v188, s[36:39], s87 offen offset:1024 sc0 sc1\n\t" \
  "buffer_load_dwordx4 a[232:235], v188, s[36:39], s87 offen offset:2048 sc0 sc1\n\t" \
  "buffer_load_dwordx4 a[236:239], v188, s[36:39], s87 offen offset:3072 sc0 sc1\n\t" \
  "buffer_load_dwordx4 a[240:243], v189, s[36:39], s87 offen offset:0 sc0 sc1\n\t" \
  "buffer_load_dwordx4 a[244:247], v189, s[36:39], s87 offen offset:1024 sc0 sc1\n\t" \
  "buffer_load_dwordx4 a[248:251], v189, s[36:39], s87 offen offset:2048 sc0 sc1\n\t" \
  "buffer_load_dwordx4 a[252:255], v189, s[36:39], s87 offen offset:3072 sc0 sc1\n\t" \
  "global_load_dword v226, v195, s[26:27] offset:0 sc0 sc1\n\t" \
  "s_memrealtime s[88:89]\n\t" \
  "s_waitcnt vmcnt(0) lgkmcnt(0)\n\t" \
  "v_accvgpr_read_b32 v237, a224\n\t" \
  "v_accvgpr_read_b32 v238, a225\n\t" \
  "v_accvgpr_read_b32 v239, a226\n\t" \
  "v_accvgpr_read_b32 v240, a227\n\t" \
  "v_min3_u32 v236, v237, v238, v237\n\t" \
  "v_min3_u32 v236, v239, v240, v236\n\t" \
  "v_accvgpr_read_b32 v237, a228\n\t" \
  "v_accvgpr_read_b32 v238, a229\n\t" \
  "v_accvgpr_read_b32 v239, a230\n\t" \
  "v_accvgpr_read_b32 v240, a231\n\t" \
  "v_min3_u32 v236, v237, v238, v236\n\t" \
  "v_min3_u32 v236, v239, v240, v236\n\t" \
  "v_accvgpr_read_b32 v237, a232\n\t" \
  "v_accvgpr_read_b32 v238, a233\n\t" \
  "v_accvgpr_read_b32 v239, a234\n\t" \
  "v_accvgpr_read_b32 v240, a235\n\t" \
  "v_min3_u32 v236, v237, v238, v236\n\t" \
  "v_min3_u32 v236, v239, v240, v236\n\t" \
  "v_accvgpr_read_b32 v237, a236\n\t" \
  "v_accvgpr_read_b32 v238, a237\n\t" \
  "v_accvgpr_read_b32 v239, a238\n\t" \
  "v_accvgpr_read_b32 v240, a239\n\t" \
  "v_min3_u32 v236, v237, v238, v236\n\t" \
  "v_min3_u32 v236, v239, v240, v236\n\t" \
  "v_accvgpr_read_b32 v237, a240\n\t" \
  "v_accvgpr_read_b32 v238, a241\n\t" \
  "v_accvgpr_read_b32 v239, a242\n\t" \
  "v_accvgpr_read_b32 v240, a243\n\t" \
  "v_min3_u32 v236, v237, v238, v236\n\t" \
  "v_min3_u32 v236, v239, v240, v236\n\t" \
  "v_accvgpr_read_b32 v237, a244\n\t" \
  "v_accvgpr_read_b32 v238, a245\n\t" \
  "v_accvgpr_read_b32 v239, a246\n\t" \
  "v_accvgpr_read_b32 v240, a247\n\t" \
  "v_min3_u32 v236, v237, v238, v236\n\t" \
  "v_min3_u32 v236, v239, v240, v236\n\t" \
  "v_accvgpr_read_b32 v237, a248\n\t" \
  "v_accvgpr_read_b32 v238, a249\n\t" \
  "v_accvgpr_read_b32 v239, a250\n\t" \
  "v_accvgpr_read_b32 v240, a251\n\t" \
  "v_min3_u32 v236, v237, v238, v236\n\t" \
  "v_min3_u32 v236, v239, v240, v236\n\t" \
  "v_accvgpr_read_b32 v237, a252\n\t" \
  "v_accvgpr_read_b32 v238, a253\n\t" \
  "v_accvgpr_read_b32 v239, a254\n\t" \
  "v_accvgpr_read_b32 v240, a255\n\t" \
  "v_min3_u32 v236, v237, v238, v236\n\t" \
  "v_min3_u32 v236, v239, v240, v236\n\t" \
  "v_cmp_eq_u32 vcc, 0, v236\n\t" \
  "s_nop 4\n\t" \
  "s_cmp_eq_u64 vcc, 0\n\t" \
  "s_cbranch_scc1 L_vd7_0\n\t" \
  "s_cmp_eq_u32 s101, 15\n\t" \
  "s_cbranch_scc1 L_vd7_0\n\t" \
  "v_cmp_le_u32 vcc, s79, v226\n\t" \
  "s_nop 4\n\t" \
  "s_bfe_u32 s101, vcc_lo, 262156\n\t" \
  "s_sub_u32 s84, s88, s14\n\t" \
  "s_or_b32 s84, s84, s97\n\t" \
  "s_cmp_lt_u32 s84, 50000000\n\t" \
  "s_cbranch_scc1 L_vl7_0\n\t" \
  "s_mov_b32 s97, -1\n\t" \
  "s_mov_b32 s81, 0x7fc00000\n\t" \
  "L_vd7_0:\n\t" \
  "s_mov_b64 exec, 0xfffffff\n\t" \
  "s_mov_b32 s84, 0\n\t" \
  "buffer_load_dword v210, v198, s[36:39], s84 offen sc1\n\t" \
  "s_mov_b64 exec, -1\n\t" \
  "s_mov_b32 s87, s98\n\t" \
  "s_branch L_vr7_0\n\t" \
  "L_v8_0:\n\t" \
  "s_mov_b32 s101, 0\n\t" \
  "s_mov_b32 s98, s87\n\t" \
  "L_vl8_0:\n\t" \
  "s_memrealtime s[88:89]\n\t" \
  "s_cmp_ge_u32 s82, 18\n\t" \
  "s_cbranch_scc1 L_vn8_0\n\t" \
  "s_waitcnt lgkmcnt(0)\n\t" \
  "s_sub_u32 s84, s88, s14\n\t" \
  "s_cmp_lt_u32 s84, 520\n\t" \
  "s_cbranch_scc1 L_vn8_0\n\t" \
  "v_readlane_b32 s85, v211, s82\n\t" \
  "s_add_u32 s82, s82, 4\n\t" \
  "s_nop 3\n\t" \
  "buffer_load_dword v210, v197, s[60:63], s85 offen\n\t" \
  "L_vn8_0:\n\t" \
  "s_mov_b32 s87, 262144\n\t" \
  "buffer_load_dwordx4 a[0:3], v188, s[36:39], s87 offen offset:0 sc0 sc1\n\t" \
  "buffer_load_dwordx4 a[4:7], v188, s[36:39], s87 offen offset:1024 sc0 sc1\n\t" \
  "buffer_load_dwordx4 a[8:11], v188, s[36:39], s87 offen offset:2048 sc0 sc1\n\t" \
  "buffer_load_dwordx4 a[12:15], v188, s[36:39], s87 offen offset:3072 sc0 sc1\n\t" \
  "buffer_load_dwordx4 a[16:19], v189, s[36:39], s87 offen offset:0 sc0 sc1\n\t" \
  "buffer_load_dwordx4 a[20:23], v189, s[36:39], s87 offen offset:1024 sc0 sc1\n\t" \
  "buffer_load_dwordx4 a[24:27], v189, s[36:39], s87 offen offset:2048 sc0 sc1\n\t" \
  "buffer_load_dwordx4 a[28:31], v189, s[36:39], s87 offen offset:3072 sc0 sc1\n\t" \
  "global_load_dword v226, v195, s[26:27] offset:0 sc0 sc1\n\t" \
  "s_memrealtime s[88:89]\n\t" \
  "s_waitcnt vmcnt(0) lgkmcnt(0)\n\t" \
  "v_accvgpr_read_b32 v237, a0\n\t" \
  "v_accvgpr_read_b32 v238, a1\n\t" \
  "v_accvgpr_read_b32 v239, a2\n\t" \
  "v_accvgpr_read_b32 v240, a3\n\t" \
  "v_min3_u32 v236, v237, v238, v237\n\t" \
  "v_min3_u32 v236, v239, v240, v236\n\t" \
  "v_accvgpr_read_b32 v237, a4\n\t" \
  "v_accvgpr_read_b32 v238, a5\n\t" \
  "v_accvgpr_read_b32 v239, a6\n\t" \
  "v_accvgpr_read_b32 v240, a7\n\t" \
  "v_min3_u32 v236, v237, v238, v236\n\t" \
  "v_min3_u32 v236, v239, v240, v236\n\t" \
  "v_accvgpr_read_b32 v237, a8\n\t" \
  "v_accvgpr_read_b32 v238, a9\n\t" \
  "v_accvgpr_read_b32 v239, a10\n\t" \
  "v_accvgpr_read_b32 v240, a11\n\t" \
  "v_min3_u32 v236, v237, v238, v236\n\t" \
  "v_min3_u32 v236, v239, v240, v236\n\t" \
  "v_accvgpr_read_b32 v237, a12\n\t" \
  "v_accvgpr_read_b32 v238, a13\n\t" \
  "v_accvgpr_read_b32 v239, a14\n\t" \
  "v_accvgpr_read_b32 v240, a15\n\t" \
  "v_min3_u32 v236, v237, v238, v236\n\t" \
  "v_min3_u32 v236, v239, v240, v236\n\t" \
  "v_accvgpr_read_b32 v237, a16\n\t" \
  "v_accvgpr_read_b32 v238, a17\n\t" \
  "v_accvgpr_read_b32 v239, a18\n\t" \
  "v_accvgpr_read_b32 v240, a19\n\t" \
  "v_min3_u32 v236, v237, v238, v236\n\t" \
  "v_min3_u32 v236, v239, v240, v236\n\t" \
  "v_accvgpr_read_b32 v237, a20\n\t" \
  "v_accvgpr_read_b32 v238, a21\n\t" \
  "v_accvgpr_read_b32 v239, a22\n\t" \
  "v_accvgpr_read_b32 v240, a23\n\t" \
  "v_min3_u32 v236, v237, v238, v236\n\t" \
  "v_min3_u32 v236, v239, v240, v236\n\t" \
  "v_accvgpr_read_b32 v237, a24\n\t" \
  "v_accvgpr_read_b32 v238, a25\n\t" \
  "v_accvgpr_read_b32 v239, a26\n\t" \
  "v_accvgpr_read_b32 v240, a27\n\t" \
  "v_min3_u32 v236, v237, v238, v236\n\t" \
  "v_min3_u32 v236, v239, v240, v236\n\t" \
  "v_accvgpr_read_b32 v237, a28\n\t" \
  "v_accvgpr_read_b32 v238, a29\n\t" \
  "v_accvgpr_read_b32 v239, a30\n\t" \
  "v_accvgpr_read_b32 v240, a31\n\t" \
  "v_min3_u32 v236, v237, v238, v236\n\t" \
  "v_min3_u32 v236, v239, v240, v236\n\t" \
  "v_cmp_eq_u32 vcc, 0, v236\n\t" \
  "s_nop 4\n\t" \
  "s_cmp_eq_u64 vcc, 0\n\t" \
  "s_cbranch_scc1 L_vd8_0\n\t" \
  "s_cmp_eq_u32 s101, 15\n\t" \
  "s_cbranch_scc1 L_vd8_0\n\t" \
  "v_cmp_le_u32 vcc, s79, v226\n\t" \
  "s_nop 4\n\t" \
  "s_bfe_u32 s101, vcc_lo, 262160\n\t" \
  "s_sub_u32 s84, s88, s14\n\t" \
  "s_or_b32 s84, s84, s97\n\t" \
  "s_cmp_lt_u32 s84, 50000000\n\t" \
  "s_cbranch_scc1 L_vl8_0\n\t" \
  "s_mov_b32 s97, -1\n\t" \
  "s_mov_b32 s81, 0x7fc00000\n\t" \
  "L_vd8_0:\n\t" \
  "s_mov_b64 exec, 0xfffffff\n\t" \
  "s_mov_b32 s84, 0\n\t" \
  "buffer_load_dword v210, v198, s[36:39], s84 offen sc1\n\t" \
  "s_mov_b64 exec, -1\n\t" \
  "s_mov_b32 s87, s98\n\t" \
  "s_branch L_vr8_0\n\t" \
  "L_v9_0:\n\t" \
  "s_mov_b32 s101, 0\n\t" \
  "s_mov_b32 s98, s87\n\t" \
  "L_vl9_0:\n\t" \
  "s_memrealtime s[88:89]\n\t" \
  "s_cmp_ge_u32 s82, 18\n\t" \
  "s_cbranch_scc1 L_vn9_0\n\t" \
  "s_waitcnt lgkmcnt(0)\n\t" \
  "s_sub_u32 s84, s88, s14\n\t" \
  "s_cmp_lt_u32 s84, 520\n\t" \
  "s_cbranch_scc1 L_vn9_0\n\t" \
  "v_readlane_b32 s85, v211, s82\n\t" \
  "s_add_u32 s82, s82, 4\n\t" \
  "s_nop 3\n\t" \
  "buffer_load_dword v210, v197, s[60:63], s85 offen\n\t" \
  "L_vn9_0:\n\t" \
  "s_mov_b32 s87, 327680\n\t" \
  "buffer_load_dwordx4 a[32:35], v188, s[36:39], s87 offen offset:0 sc0 sc1\n\t" \
  "buffer_load_dwordx4 a[36:39], v188, s[36:39], s87 offen offset:1024 sc0 sc1\n\t" \
  "buffer_load_dwordx4 a[40:43], v188, s[36:39], s87 offen offset:2048 sc0 sc1\n\t" \
  "buffer_load_dwordx4 a[44:47], v188, s[36:39], s87 offen offset:3072 sc0 sc1\n\t" \
  "buffer_load_dwordx4 a[48:51], v189, s[36:39], s87 offen offset:0 sc0 sc1\n\t" \
  "buffer_load_dwordx4 a[52:55], v189, s[36:39], s87 offen offset:1024 sc0 sc1\n\t" \
  "buffer_load_dwordx4 a[56:59], v189, s[36:39], s87 offen offset:2048 sc0 sc1\n\t" \
  "buffer_load_dwordx4 a[60:63], v189, s[36:39], s87 offen offset:3072 sc0 sc1\n\t" \
  "global_load_dword v226, v195, s[26:27] offset:0 sc0 sc1\n\t" \
  "s_memrealtime s[88:89]\n\t" \
  "s_waitcnt vmcnt(0) lgkmcnt(0)\n\t" \
  "v_accvgpr_read_b32 v237, a32\n\t" \
  "v_accvgpr_read_b32 v238, a33\n\t" \
  "v_accvgpr_read_b32 v239, a34\n\t" \
  "v_accvgpr_read_b32 v240, a35\n\t" \
  "v_min3_u32 v236, v237, v238, v237\n\t" \
  "v_min3_u32 v236, v239, v240, v236\n\t" \
  "v_accvgpr_read_b32 v237, a36\n\t" \
  "v_accvgpr_read_b32 v238, a37\n\t" \
  "v_accvgpr_read_b32 v239, a38\n\t" \
  "v_accvgpr_read_b32 v240, a39\n\t" \
  "v_min3_u32 v236, v237, v238, v236\n\t" \
  "v_min3_u32 v236, v239, v240, v236\n\t" \
  "v_accvgpr_read_b32 v237, a40\n\t" \
  "v_accvgpr_read_b32 v238, a41\n\t" \
  "v_accvgpr_read_b32 v239, a42\n\t" \
  "v_accvgpr_read_b32 v240, a43\n\t" \
  "v_min3_u32 v236, v237, v238, v236\n\t" \
  "v_min3_u32 v236, v239, v240, v236\n\t" \
  "v_accvgpr_read_b32 v237, a44\n\t" \
  "v_accvgpr_read_b32 v238, a45\n\t" \
  "v_accvgpr_read_b32 v239, a46\n\t" \
  "v_accvgpr_read_b32 v240, a47\n\t" \
  "v_min3_u32 v236, v237, v238, v236\n\t" \
  "v_min3_u32 v236, v239, v240, v236\n\t" \
  "v_accvgpr_read_b32 v237, a48\n\t" \
  "v_accvgpr_read_b32 v238, a49\n\t" \
  "v_accvgpr_read_b32 v239, a50\n\t" \
  "v_accvgpr_read_b32 v240, a51\n\t" \
  "v_min3_u32 v236, v237, v238, v236\n\t" \
  "v_min3_u32 v236, v239, v240, v236\n\t" \
  "v_accvgpr_read_b32 v237, a52\n\t" \
  "v_accvgpr_read_b32 v238, a53\n\t" \
  "v_accvgpr_read_b32 v239, a54\n\t" \
  "v_accvgpr_read_b32 v240, a55\n\t" \
  "v_min3_u32 v236, v237, v238, v236\n\t" \
  "v_min3_u32 v236, v239, v240, v236\n\t" \
  "v_accvgpr_read_b32 v237, a56\n\t" \
  "v_accvgpr_read_b32 v238, a57\n\t" \
  "v_accvgpr_read_b32 v239, a58\n\t" \
  "v_accvgpr_read_b32 v240, a59\n\t" \
  "v_min3_u32 v236, v237, v238, v236\n\t" \
  "v_min3_u32 v236, v239, v240, v236\n\t" \
  "v_accvgpr_read_b32 v237, a60\n\t" \
  "v_accvgpr_read_b32 v238, a61\n\t" \
  "v_accvgpr_read_b32 v239, a62\n\t" \
  "v_accvgpr_read_b32 v240, a63\n\t" \
  "v_min3_u32 v236, v237, v238, v236\n\t" \
  "v_min3_u32 v236, v239, v240, v236\n\t" \
  "v_cmp_eq_u32 vcc, 0, v236\n\t" \
  "s_nop 4\n\t" \
  "s_cmp_eq_u64 vcc, 0\n\t" \
  "s_cbranch_scc1 L_vd9_0\n\t" \
  "s_cmp_eq_u32 s101, 15\n\t" \
  "s_cbranch_scc1 L_vd9_0\n\t" \
  "v_cmp_le_u32 vcc, s79, v226\n\t" \
  "s_nop 4\n\t" \
  "s_bfe_u32 s101, vcc_lo, 262164\n\t" \
  "s_sub_u32 s84, s88, s14\n\t" \
  "s_or_b32 s84, s84, s97\n\t" \
  "s_cmp_lt_u32 s84, 50000000\n\t" \
  "s_cbranch_scc1 L_vl9_0\n\t" \
  "s_mov_b32 s97, -1\n\t" \
  "s_mov_b32 s81, 0x7fc00000\n\t" \
  "L_vd9_0:\n\t" \
  "s_mov_b64 exec, 0xfffffff\n\t" \
  "s_mov_b32 s84, 0\n\t" \
  "buffer_load_dword v210, v198, s[36:39], s84 offen sc1\n\t" \
  "s_mov_b64 exec, -1\n\t" \
  "s_mov_b32 s87, s98\n\t" \
  "s_branch L_vr9_0\n\t" \
  "L_v10_0:\n\t" \
  "s_mov_b32 s101, 0\n\t" \
  "s_mov_b32 s98, s87\n\t" \
  "L_vl10_0:\n\t" \
  "s_memrealtime s[88:89]\n\t" \
  "s_cmp_ge_u32 s82, 18\n\t" \
  "s_cbranch_scc1 L_vn10_0\n\t" \
  "s_waitcnt lgkmcnt(0)\n\t" \
  "s_sub_u32 s84, s88, s14\n\t" \
  "s_cmp_lt_u32 s84, 520\n\t" \
  "s_cbranch_scc1 L_vn10_0\n\t" \
  "v_readlane_b32 s85, v211, s82\n\t" \
  "s_add_u32 s82, s82, 4\n\t" \
  "s_nop 3\n\t" \
  "buffer_load_dword v210, v197, s[60:63], s85 offen\n\t" \
  "L_vn10_0:\n\t" \
  "s_mov_b32 s87, 393216\n\t" \
  "buffer_load_dwordx4 a[64:67], v188, s[36:39], s87 offen offset:0 sc0 sc1\n\t" \
  "buffer_load_dwordx4 a[68:71], v188, s[36:39], s87 offen offset:1024 sc0 sc1\n\t" \
  "buffer_load_dwordx4 a[72:75], v188, s[36:39], s87 offen offset:2048 sc0 sc1\n\t" \
  "buffer_load_dwordx4 a[76:79], v188, s[36:39], s87 offen offset:3072 sc0 sc1\n\t" \
  "buffer_load_dwordx4 a[80:83], v189, s[36:39], s87 offen offset:0 sc0 sc1\n\t" \
  "buffer_load_dwordx4 a[84:87], v189, s[36:39], s87 offen offset:1024 sc0 sc1\n\t" \
  "buffer_load_dwordx4 a[88:91], v189, s[36:39], s87 offen offset:2048 sc0 sc1\n\t" \
  "buffer_load_dwordx4 a[92:95], v189, s[36:39], s87 offen offset:3072 sc0 sc1\n\t" \
  "global_load_dword v226, v195, s[26:27] offset:0 sc0 sc1\n\t" \
  "s_memrealtime s[88:89]\n\t" \
  "s_waitcnt vmcnt(0) lgkmcnt(0)\n\t" \
  "v_accvgpr_read_b32 v237, a64\n\t" \
  "v_accvgpr_read_b32 v238, a65\n\t" \
  "v_accvgpr_read_b32 v239, a66\n\t" \
  "v_accvgpr_read_b32 v240, a67\n\t" \
  "v_min3_u32 v236, v237, v238, v237\n\t" \
  "v_min3_u32 v236, v239, v240, v236\n\t" \
  "v_accvgpr_read_b32 v237, a68\n\t" \
  "v_accvgpr_read_b32 v238, a69\n\t" \
  "v_accvgpr_read_b32 v239, a70\n\t" \
  "v_accvgpr_read_b32 v240, a71\n\t" \
  "v_min3_u32 v236, v237, v238, v236\n\t" \
  "v_min3_u32 v236, v239, v240, v236\n\t" \
  "v_accvgpr_read_b32 v237, a72\n\t" \
  "v_accvgpr_read_b32 v238, a73\n\t" \
  "v_accvgpr_read_b32 v239, a74\n\t" \
  "v_accvgpr_read_b32 v240, a75\n\t" \
  "v_min3_u32 v236, v237, v238, v236\n\t" \
  "v_min3_u32 v236, v239, v240, v236\n\t" \
  "v_accvgpr_read_b32 v237, a76\n\t" \
  "v_accvgpr_read_b32 v238, a77\n\t" \
  "v_accvgpr_read_b32 v239, a78\n\t" \
  "v_accvgpr_read_b32 v240, a79\n\t" \
  "v_min3_u32 v236, v237, v238, v236\n\t" \
  "v_min3_u32 v236, v239, v240, v236\n\t" \
  "v_accvgpr_read_b32 v237, a80\n\t" \
  "v_accvgpr_read_b32 v238, a81\n\t" \
  "v_accvgpr_read_b32 v239, a82\n\t" \
  "v_accvgpr_read_b32 v240, a83\n\t" \
  "v_min3_u32 v236, v237, v238, v236\n\t" \
  "v_min3_u32 v236, v239, v240, v236\n\t" \
  "v_accvgpr_read_b32 v237, a84\n\t" \
  "v_accvgpr_read_b32 v238, a85\n\t" \
  "v_accvgpr_read_b32 v239, a86\n\t" \
  "v_accvgpr_read_b32 v240, a87\n\t" \
  "v_min3_u32 v236, v237, v238, v236\n\t" \
  "v_min3_u32 v236, v239, v240, v236\n\t" \
  "v_accvgpr_read_b32 v237, a88\n\t" \
  "v_accvgpr_read_b32 v238, a89\n\t" \
  "v_accvgpr_read_b32 v239, a90\n\t" \
  "v_accvgpr_read_b32 v240, a91\n\t" \
  "v_min3_u32 v236, v237, v238, v236\n\t" \
  "v_min3_u32 v236, v239, v240, v236\n\t" \
  "v_accvgpr_read_b32 v237, a92\n\t" \
  "v_accvgpr_read_b32 v238, a93\n\t" \
  "v_accvgpr_read_b32 v239, a94\n\t" \
  "v_accvgpr_read_b32 v240, a95\n\t" \
  "v_min3_u32 v236, v237, v238, v236\n\t" \
  "v_min3_u32 v236, v239, v240, v236\n\t" \
  "v_cmp_eq_u32 vcc, 0, v236\n\t" \
  "s_nop 4\n\t" \
  "s_cmp_eq_u64 vcc, 0\n\t" \
  "s_cbranch_scc1 L_vd10_0\n\t" \
  "s_cmp_eq_u32 s101, 15\n\t" \
  "s_cbranch_scc1 L_vd10_0\n\t" \
  "v_cmp_le_u32 vcc, s79, v226\n\t" \
  "s_nop 4\n\t" \
  "s_bfe_u32 s101, vcc_lo, 262168\n\t" \
  "s_sub_u32 s84, s88, s14\n\t" \
  "s_or_b32 s84, s84, s97\n\t" \
  "s_cmp_lt_u32 s84, 50000000\n\t" \
  "s_cbranch_scc1 L_vl10_0\n\t" \
  "s_mov_b32 s97, -1\n\t" \
  "s_mov_b32 s81, 0x7fc00000\n\t" \
  "L_vd10_0:\n\t" \
  "s_mov_b64 exec, 0xfffffff\n\t" \
  "s_mov_b32 s84, 0\n\t" \
  "buffer_load_dword v210, v198, s[36:39], s84 offen sc1\n\t" \
  "s_mov_b64 exec, -1\n\t" \
  "s_mov_b32 s87, s98\n\t" \
  "s_branch L_vr10_0\n\t" \
  "L_q11_0:\n\t" \
  "global_load_dword v226, v195, s[26:27] offset:1024 sc0 sc1\n\t" \
  "s_memrealtime s[88:89]\n\t" \
  "s_waitcnt vmcnt(0) lgkmcnt(0)\n\t" \
  "v_cmp_le_u32 vcc, s79, v226\n\t" \
  "s_nop 4\n\t" \
  "s_andn2_b32 s85, vcc_lo, s96\n\t" \
  "s_or_b32 s96, s96, vcc_lo\n\t" \
  "s_and_b32 exec_lo, s85, 0xfffffff\n\t" \
  "s_mov_b32 exec_hi, 0\n\t" \
  "s_cbranch_execz L_u19_0\n\t" \
  "s_mov_b32 s84, 458752\n\t" \
  "buffer_load_dword v210, v198, s[36:39], s84 offen \n\t" \
  "L_u19_0:\n\t" \
  "s_mov_b64 exec, -1\n\t" \
  "s_bfe_u32 s84, s96, 262144\n\t" \
  "s_cmp_eq_u32 s84, 15\n\t" \
  "s_cbranch_scc1 L_qr11_0\n\t" \
  "s_sub_u32 s84, s88, s14\n\t" \
  "s_or_b32 s84, s84, s97\n\t" \
  "s_cmp_lt_u32 s84, 50000000\n\t" \
  "s_cbranch_scc1 L_q11_0\n\t" \
  "s_mov_b32 s97, -1\n\t" \
  "s_mov_b32 s81, 0x7fc00000\n\t" \
  "s_branch L_qr11_0\n\t" \
  "L_q12_0:\n\t" \
  "global_load_dword v226, v195, s[26:27] offset:1024 sc0 sc1\n\t" \
  "s_memrealtime s[88:89]\n\t" \
  "s_waitcnt vmcnt(0) lgkmcnt(0)\n\t" \
  "v_cmp_le_u32 vcc, s79, v226\n\t" \
  "s_nop 4\n\t" \
  "s_andn2_b32 s85, vcc_lo, s96\n\t" \
  "s_or_b32 s96, s96, vcc_lo\n\t" \
  "s_and_b32 exec_lo, s85, 0xfffffff\n\t" \
  "s_mov_b32 exec_hi, 0\n\t" \
  "s_cbranch_execz L_u20_0\n\t" \
  "s_mov_b32 s84, 458752\n\t" \
  "buffer_load_dword v210, v198, s[36:39], s84 offen \n\t" \
  "L_u20_0:\n\t" \
  "s_mov_b64 exec, -1\n\t" \
  "s_bfe_u32 s84, s96, 262148\n\t" \
  "s_cmp_eq_u32 s84, 15\n\t" \
  "s_cbranch_scc1 L_qr12_0\n\t" \
  "s_sub_u32 s84, s88, s14\n\t" \
  "s_or_b32 s84, s84, s97\n\t" \
  "s_cmp_lt_u32 s84, 50000000\n\t" \
  "s_cbranch_scc1 L_q12_0\n\t" \
  "s_mov_b32 s97, -1\n\t" \
  "s_mov_b32 s81, 0x7fc00000\n\t" \
  "s_branch L_qr12_0\n\t" \
  "L_q13_0:\n\t" \
  "global_load_dword v226, v195, s[26:27] offset:1024 sc0 sc1\n\t" \
  "s_memrealtime s[88:89]\n\t" \
  "s_waitcnt vmcnt(0) lgkmcnt(0)\n\t" \
  "v_cmp_le_u32 vcc, s79, v226\n\t" \
  "s_nop 4\n\t" \
  "s_andn2_b32 s85, vcc_lo, s96\n\t" \
  "s_or_b32 s96, s96, vcc_lo\n\t" \
  "s_and_b32 exec_lo, s85, 0xfffffff\n\t" \
  "s_mov_b32 exec_hi, 0\n\t" \
  "s_cbranch_execz L_u21_0\n\t" \
  "s_mov_b32 s84, 458752\n\t" \
  "buffer_load_dword v210, v198, s[36:39], s84 offen \n\t" \
  "L_u21_0:\n\t" \
  "s_mov_b64 exec, -1\n\t" \
  "s_bfe_u32 s84, s96, 262152\n\t" \
  "s_cmp_eq_u32 s84, 15\n\t" \
  "s_cbranch_scc1 L_qr13_0\n\t" \
  "s_sub_u32 s84, s88, s14\n\t" \
  "s_or_b32 s84, s84, s97\n\t" \
  "s_cmp_lt_u32 s84, 50000000\n\t" \
  "s_cbranch_scc1 L_q13_0\n\t" \
  "s_mov_b32 s97, -1\n\t" \
  "s_mov_b32 s81, 0x7fc00000\n\t" \
  "s_branch L_qr13_0\n\t" \
  "L_q14_0:\n\t" \
  "global_load_dword v226, v195, s[26:27] offset:1024 sc0 sc1\n\t" \
  "s_memrealtime s[88:89]\n\t" \
  "s_waitcnt vmcnt(0) lgkmcnt(0)\n\t" \
  "v_cmp_le_u32 vcc, s79, v226\n\t" \
  "s_nop 4\n\t" \
  "s_andn2_b32 s85, vcc_lo, s96\n\t" \
  "s_or_b32 s96, s96, vcc_lo\n\t" \
  "s_and_b32 exec_lo, s85, 0xfffffff\n\t" \
  "s_mov_b32 exec_hi, 0\n\t" \
  "s_cbranch_execz L_u22_0\n\t" \
  "s_mov_b32 s84, 458752\n\t" \
  "buffer_load_dword v210, v198, s[36:39], s84 offen \n\t" \
  "L_u22_0:\n\t" \
  "s_mov_b64 exec, -1\n\t" \
  "s_bfe_u32 s84, s96, 262156\n\t" \
  "s_cmp_eq_u32 s84, 15\n\t" \
  "s_cbranch_scc1 L_qr14_0\n\t" \
  "s_sub_u32 s84, s88, s14\n\t" \
  "s_or_b32 s84, s84, s97\n\t" \
  "s_cmp_lt_u32 s84, 50000000\n\t" \
  "s_cbranch_scc1 L_q14_0\n\t" \
  "s_mov_b32 s97, -1\n\t" \
  "s_mov_b32 s81, 0x7fc00000\n\t" \
  "s_branch L_qr14_0\n\t" \
  "L_q15_0:\n\t" \
  "global_load_dword v226, v195, s[26:27] offset:1024 sc0 sc1\n\t" \
  "s_memrealtime s[88:89]\n\t" \
  "s_waitcnt vmcnt(0) lgkmcnt(0)\n\t" \
  "v_cmp_le_u32 vcc, s79, v226\n\t" \
  "s_nop 4\n\t" \
  "s_andn2_b32 s85, vcc_lo, s96\n\t" \
  "s_or_b32 s96, s96, vcc_lo\n\t" \
  "s_and_b32 exec_lo, s85, 0xfffffff\n\t" \
  "s_mov_b32 exec_hi, 0\n\t" \
  "s_cbranch_execz L_u23_0\n\t" \
  "s_mov_b32 s84, 458752\n\t" \
  "buffer_load_dword v210, v198, s[36:39], s84 offen \n\t" \
  "L_u23_0:\n\t" \
  "s_mov_b64 exec, -1\n\t" \
  "s_bfe_u32 s84, s96, 262160\n\t" \
  "s_cmp_eq_u32 s84, 15\n\t" \
  "s_cbranch_scc1 L_qr15_0\n\t" \
  "s_sub_u32 s84, s88, s14\n\t" \
  "s_or_b32 s84, s84, s97\n\t" \
  "s_cmp_lt_u32 s84, 50000000\n\t" \
  "s_cbranch_scc1 L_q15_0\n\t" \
  "s_mov_b32 s97, -1\n\t" \
  "s_mov_b32 s81, 0x7fc00000\n\t" \
  "s_branch L_qr15_0\n\t" \
  "L_q16_0:\n\t" \
  "global_load_dword v226, v195, s[26:27] offset:1024 sc0 sc1\n\t" \
  "s_memrealtime s[88:89]\n\t" \
  "s_waitcnt vmcnt(0) lgkmcnt(0)\n\t" \
  "v_cmp_le_u32 vcc, s79, v226\n\t" \
  "s_nop 4\n\t" \
  "s_andn2_b32 s85, vcc_lo, s96\n\t" \
  "s_or_b32 s96, s96, vcc_lo\n\t" \
  "s_and_b32 exec_lo, s85, 0xfffffff\n\t" \
  "s_mov_b32 exec_hi, 0\n\t" \
  "s_cbranch_execz L_u24_0\n\t" \
  "s_mov_b32 s84, 458752\n\t" \
  "buffer_load_dword v210, v198, s[36:39], s84 offen \n\t" \
  "L_u24_0:\n\t" \
  "s_mov_b64 exec, -1\n\t" \
  "s_bfe_u32 s84, s96, 262164\n\t" \
  "s_cmp_eq_u32 s84, 15\n\t" \
  "s_cbranch_scc1 L_qr16_0\n\t" \
  "s_sub_u32 s84, s88, s14\n\t" \
  "s_or_b32 s84, s84, s97\n\t" \
  "s_cmp_lt_u32 s84, 50000000\n\t" \
  "s_cbranch_scc1 L_q16_0\n\t" \
  "s_mov_b32 s97, -1\n\t" \
  "s_mov_b32 s81, 0x7fc00000\n\t" \
  "s_branch L_qr16_0\n\t" \
  "L_q17_0:\n\t" \
  "global_load_dword v226, v195, s[26:27] offset:1024 sc0 sc1\n\t" \
  "s_memrealtime s[88:89]\n\t" \
  "s_waitcnt vmcnt(0) lgkmcnt(0)\n\t" \
  "v_cmp_le_u32 vcc, s79, v226\n\t" \
  "s_nop 4\n\t" \
  "s_andn2_b32 s85, vcc_lo, s96\n\t" \
  "s_or_b32 s96, s96, vcc_lo\n\t" \
  "s_and_b32 exec_lo, s85, 0xfffffff\n\t" \
  "s_mov_b32 exec_hi, 0\n\t" \
  "s_cbranch_execz L_u25_0\n\t" \
  "s_mov_b32 s84, 458752\n\t" \
  "buffer_load_dword v210, v198, s[36:39], s84 offen \n\t" \
  "L_u25_0:\n\t" \
  "s_mov_b64 exec, -1\n\t" \
  "s_bfe_u32 s84, s96, 262168\n\t" \
  "s_cmp_eq_u32 s84, 15\n\t" \
  "s_cbranch_scc1 L_qr17_0\n\t" \
  "s_sub_u32 s84, s88, s14\n\t" \
  "s_or_b32 s84, s84, s97\n\t" \
  "s_cmp_lt_u32 s84, 50000000\n\t" \
  "s_cbranch_scc1 L_q17_0\n\t" \
  "s_mov_b32 s97, -1\n\t" \
  "s_mov_b32 s81, 0x7fc00000\n\t" \
  "s_branch L_qr17_0\n\t" \
  "L_q18_0:\n\t" \
  "global_load_dword v226, v195, s[26:27] offset:2048 sc0 sc1\n\t" \
  "s_memrealtime s[88:89]\n\t" \
  "s_waitcnt vmcnt(0) lgkmcnt(0)\n\t" \
  "v_cmp_le_u32 vcc, s79, v226\n\t" \
  "s_nop 4\n\t" \
  "s_andn2_b32 s85, vcc_lo, s8\n\t" \
  "s_or_b32 s8, s8, vcc_lo\n\t" \
  "s_and_b32 exec_lo, s85, 0xfffffff\n\t" \
  "s_mov_b32 exec_hi, 0\n\t" \
  "s_cbranch_execz L_u26_0\n\t" \
  "s_mov_b32 s84, 917504\n\t" \
  "buffer_load_dword v210, v198, s[36:39], s84 offen \n\t" \
  "L_u26_0:\n\t" \
  "s_mov_b64 exec, -1\n\t" \
  "s_bfe_u32 s84, s8, 262144\n\t" \
  "s_cmp_eq_u32 s84, 15\n\t" \
  "s_cbranch_scc1 L_qr18_0\n\t" \
  "s_sub_u32 s84, s88, s14\n\t" \
  "s_or_b32 s84, s84, s97\n\t" \
  "s_cmp_lt_u32 s84, 50000000\n\t" \
  "s_cbranch_scc1 L_q18_0\n\t" \
  "s_mov_b32 s97, -1\n\t" \
  "s_mov_b32 s81, 0x7fc00000\n\t" \
  "s_branch L_qr18_0\n\t" \
  "L_q19_0:\n\t" \
  "global_load_dword v226, v195, s[26:27] offset:2048 sc0 sc1\n\t" \
  "s_memrealtime s[88:89]\n\t" \
  "s_waitcnt vmcnt(0) lgkmcnt(0)\n\t" \
  "v_cmp_le_u32 vcc, s79, v226\n\t" \
  "s_nop 4\n\t" \
  "s_andn2_b32 s85, vcc_lo, s8\n\t" \
  "s_or_b32 s8, s8, vcc_lo\n\t" \
  "s_and_b32 exec_lo, s85, 0xfffffff\n\t" \
  "s_mov_b32 exec_hi, 0\n\t" \
  "s_cbranch_execz L_u27_0\n\t" \
  "s_mov_b32 s84, 917504\n\t" \
  "buffer_load_dword v210, v198, s[36:39], s84 offen \n\t" \
  "L_u27_0:\n\t" \
  "s_mov_b64 exec, -1\n\t" \
  "s_bfe_u32 s84, s8, 262148\n\t" \
  "s_cmp_eq_u32 s84, 15\n\t" \
  "s_cbranch_scc1 L_qr19_0\n\t" \
  "s_sub_u32 s84, s88, s14\n\t" \
  "s_or_b32 s84, s84, s97\n\t" \
  "s_cmp_lt_u32 s84, 50000000\n\t" \
  "s_cbranch_scc1 L_q19_0\n\t" \
  "s_mov_b32 s97, -1\n\t" \
  "s_mov_b32 s81, 0x7fc00000\n\t" \
  "s_branch L_qr19_0\n\t" \
  "L_q20_0:\n\t" \
  "global_load_dword v226, v195, s[26:27] offset:2048 sc0 sc1\n\t" \
  "s_memrealtime s[88:89]\n\t" \
  "s_waitcnt vmcnt(0) lgkmcnt(0)\n\t" \
  "v_cmp_le_u32 vcc, s79, v226\n\t" \
  "s_nop 4\n\t" \
  "s_andn2_b32 s85, vcc_lo, s8\n\t" \
  "s_or_b32 s8, s8, vcc_lo\n\t" \
  "s_and_b32 exec_lo, s85, 0xfffffff\n\t" \
  "s_mov_b32 exec_hi, 0\n\t" \
  "s_cbranch_execz L_u28_0\n\t" \
  "s_mov_b32 s84, 917504\n\t" \
  "buffer_load_dword v210, v198, s[36:39], s84 offen \n\t" \
  "L_u28_0:\n\t" \
  "s_mov_b64 exec, -1\n\t" \
  "s_bfe_u32 s84, s8, 262152\n\t" \
  "s_cmp_eq_u32 s84, 15\n\t" \
  "s_cbranch_scc1 L_qr20_0\n\t" \
  "s_sub_u32 s84, s88, s14\n\t" \
  "s_or_b32 s84, s84, s97\n\t" \
  "s_cmp_lt_u32 s84, 50000000\n\t" \
  "s_cbranch_scc1 L_q20_0\n\t" \
  "s_mov_b32 s97, -1\n\t" \
  "s_mov_b32 s81, 0x7fc00000\n\t" \
  "s_branch L_qr20_0\n\t" \
  "L_q21_0:\n\t" \
  "global_load_dword v226, v195, s[26:27] offset:2048 sc0 sc1\n\t" \
  "s_memrealtime s[88:89]\n\t" \
  "s_waitcnt vmcnt(0) lgkmcnt(0)\n\t" \
  "v_cmp_le_u32 vcc, s79, v226\n\t" \
  "s_nop 4\n\t" \
  "s_andn2_b32 s85, vcc_lo, s8\n\t" \
  "s_or_b32 s8, s8, vcc_lo\n\t" \
  "s_and_b32 exec_lo, s85, 0xfffffff\n\t" \
  "s_mov_b32 exec_hi, 0\n\t" \
  "s_cbranch_execz L_u29_0\n\t" \
  "s_mov_b32 s84, 917504\n\t" \
  "buffer_load_dword v210, v198, s[36:39], s84 offen \n\t" \
  "L_u29_0:\n\t" \
  "s_mov_b64 exec, -1\n\t" \
  "s_bfe_u32 s84, s8, 262156\n\t" \
  "s_cmp_eq_u32 s84, 15\n\t" \
  "s_cbranch_scc1 L_qr21_0\n\t" \
  "s_sub_u32 s84, s88, s14\n\t" \
  "s_or_b32 s84, s84, s97\n\t" \
  "s_cmp_lt_u32 s84, 50000000\n\t" \
  "s_cbranch_scc1 L_q21_0\n\t" \
  "s_mov_b32 s97, -1\n\t" \
  "s_mov_b32 s81, 0x7fc00000\n\t" \
  "s_branch L_qr21_0\n\t" \
  "L_q22_0:\n\t" \
  "global_load_dword v226, v195, s[26:27] offset:2048 sc0 sc1\n\t" \
  "s_memrealtime s[88:89]\n\t" \
  "s_waitcnt vmcnt(0) lgkmcnt(0)\n\t" \
  "v_cmp_le_u32 vcc, s79, v226\n\t" \
  "s_nop 4\n\t" \
  "s_andn2_b32 s85, vcc_lo, s8\n\t" \
  "s_or_b32 s8, s8, vcc_lo\n\t" \
  "s_and_b32 exec_lo, s85, 0xfffffff\n\t" \
  "s_mov_b32 exec_hi, 0\n\t" \
  "s_cbranch_execz L_u30_0\n\t" \
  "s_mov_b32 s84, 917504\n\t" \
  "buffer_load_dword v210, v198, s[36:39], s84 offen \n\t" \
  "L_u30_0:\n\t" \
  "s_mov_b64 exec, -1\n\t" \
  "s_bfe_u32 s84, s8, 262160\n\t" \
  "s_cmp_eq_u32 s84, 15\n\t" \
  "s_cbranch_scc1 L_qr22_0\n\t" \
  "s_sub_u32 s84, s88, s14\n\t" \
  "s_or_b32 s84, s84, s97\n\t" \
  "s_cmp_lt_u32 s84, 50000000\n\t" \
  "s_cbranch_scc1 L_q22_0\n\t" \
  "s_mov_b32 s97, -1\n\t" \
  "s_mov_b32 s81, 0x7fc00000\n\t" \
  "s_branch L_qr22_0\n\t" \
  "L_q23_0:\n\t" \
  "global_load_dword v226, v195, s[26:27] offset:2048 sc0 sc1\n\t" \
  "s_memrealtime s[88:89]\n\t" \
  "s_waitcnt vmcnt(0) lgkmcnt(0)\n\t" \
  "v_cmp_le_u32 vcc, s79, v226\n\t" \
  "s_nop 4\n\t" \
  "s_andn2_b32 s85, vcc_lo, s8\n\t" \
  "s_or_b32 s8, s8, vcc_lo\n\t" \
  "s_and_b32 exec_lo, s85, 0xfffffff\n\t" \
  "s_mov_b32 exec_hi, 0\n\t" \
  "s_cbranch_execz L_u31_0\n\t" \
  "s_mov_b32 s84, 917504\n\t" \
  "buffer_load_dword v210, v198, s[36:39], s84 offen \n\t" \
  "L_u31_0:\n\t" \
  "s_mov_b64 exec, -1\n\t" \
  "s_bfe_u32 s84, s8, 262164\n\t" \
  "s_cmp_eq_u32 s84, 15\n\t" \
  "s_cbranch_scc1 L_qr23_0\n\t" \
  "s_sub_u32 s84, s88, s14\n\t" \
  "s_or_b32 s84, s84, s97\n\t" \
  "s_cmp_lt_u32 s84, 50000000\n\t" \
  "s_cbranch_scc1 L_q23_0\n\t" \
  "s_mov_b32 s97, -1\n\t" \
  "s_mov_b32 s81, 0x7fc00000\n\t" \
  "s_branch L_qr23_0\n\t" \
  "L_q24_0:\n\t" \
  "global_load_dword v226, v195, s[26:27] offset:2048 sc0 sc1\n\t" \
  "s_memrealtime s[88:89]\n\t" \
  "s_waitcnt vmcnt(0) lgkmcnt(0)\n\t" \
  "v_cmp_le_u32 vcc, s79, v226\n\t" \
  "s_nop 4\n\t" \
  "s_andn2_b32 s85, vcc_lo, s8\n\t" \
  "s_or_b32 s8, s8, vcc_lo\n\t" \
  "s_and_b32 exec_lo, s85, 0xfffffff\n\t" \
  "s_mov_b32 exec_hi, 0\n\t" \
  "s_cbranch_execz L_u32_0\n\t" \
  "s_mov_b32 s84, 917504\n\t" \
  "buffer_load_dword v210, v198, s[36:39], s84 offen \n\t" \
  "L_u32_0:\n\t" \
  "s_mov_b64 exec, -1\n\t" \
  "s_bfe_u32 s84, s8, 262168\n\t" \
  "s_cmp_eq_u32 s84, 15\n\t" \
  "s_cbranch_scc1 L_qr24_0\n\t" \
  "s_sub_u32 s84, s88, s14\n\t" \
  "s_or_b32 s84, s84, s97\n\t" \
  "s_cmp_lt_u32 s84, 50000000\n\t" \
  "s_cbranch_scc1 L_q24_0\n\t" \
  "s_mov_b32 s97, -1\n\t" \
  "s_mov_b32 s81, 0x7fc00000\n\t" \
  "s_branch L_qr24_0\n\t" \
  "L_q25_0:\n\t" \
  "global_load_dword v226, v195, s[26:27] offset:3072 sc0 sc1\n\t" \
  "s_memrealtime s[88:89]\n\t" \
  "s_waitcnt vmcnt(0) lgkmcnt(0)\n\t" \
  "v_cmp_le_u32 vcc, s79, v226\n\t" \
  "s_nop 4\n\t" \
  "s_andn2_b32 s85, vcc_lo, s9\n\t" \
  "s_or_b32 s9, s9, vcc_lo\n\t" \
  "s_and_b32 exec_lo, s85, 0xfffffff\n\t" \
  "s_mov_b32 exec_hi, 0\n\t" \
  "s_cbranch_execz L_u33_0\n\t" \
  "s_mov_b32 s84, 1376256\n\t" \
  "buffer_load_dword v210, v198, s[36:39], s84 offen \n\t" \
  "L_u33_0:\n\t" \
  "s_mov_b64 exec, -1\n\t" \
  "s_bfe_u32 s84, s9, 262144\n\t" \
  "s_cmp_eq_u32 s84, 15\n\t" \
  "s_cbranch_scc1 L_qr25_0\n\t" \
  "s_sub_u32 s84, s88, s14\n\t" \
  "s_or_b32 s84, s84, s97\n\t" \
  "s_cmp_lt_u32 s84, 50000000\n\t" \
  "s_cbranch_scc1 L_q25_0\n\t" \
  "s_mov_b32 s97, -1\n\t" \
  "s_mov_b32 s81, 0x7fc00000\n\t" \
  "s_branch L_qr25_0\n\t" \
  "L_q26_0:\n\t" \
  "global_load_dword v226, v195, s[26:27] offset:3072 sc0 sc1\n\t" \
  "s_memrealtime s[88:89]\n\t" \
  "s_waitcnt vmcnt(0) lgkmcnt(0)\n\t" \
  "v_cmp_le_u32 vcc, s79, v226\n\t" \
  "s_nop 4\n\t" \
  "s_andn2_b32 s85, vcc_lo, s9\n\t" \
  "s_or_b32 s9, s9, vcc_lo\n\t" \
  "s_and_b32 exec_lo, s85, 0xfffffff\n\t" \
  "s_mov_b32 exec_hi, 0\n\t" \
  "s_cbranch_execz L_u34_0\n\t" \
  "s_mov_b32 s84, 1376256\n\t" \
  "buffer_load_dword v210, v198, s[36:39], s84 offen \n\t" \
  "L_u34_0:\n\t" \
  "s_mov_b64 exec, -1\n\t" \
  "s_bfe_u32 s84, s9, 262148\n\t" \
  "s_cmp_eq_u32 s84, 15\n\t" \
  "s_cbranch_scc1 L_qr26_0\n\t" \
  "s_sub_u32 s84, s88, s14\n\t" \
  "s_or_b32 s84, s84, s97\n\t" \
  "s_cmp_lt_u32 s84, 50000000\n\t" \
  "s_cbranch_scc1 L_q26_0\n\t" \
  "s_mov_b32 s97, -1\n\t" \
  "s_mov_b32 s81, 0x7fc00000\n\t" \
  "s_branch L_qr26_0\n\t" \
  "L_q27_0:\n\t" \
  "global_load_dword v226, v195, s[26:27] offset:3072 sc0 sc1\n\t" \
  "s_memrealtime s[88:89]\n\t" \
  "s_waitcnt vmcnt(0) lgkmcnt(0)\n\t" \
  "v_cmp_le_u32 vcc, s79, v226\n\t" \
  "s_nop 4\n\t" \
  "s_andn2_b32 s85, vcc_lo, s9\n\t" \
  "s_or_b32 s9, s9, vcc_lo\n\t" \
  "s_and_b32 exec_lo, s85, 0xfffffff\n\t" \
  "s_mov_b32 exec_hi, 0\n\t" \
  "s_cbranch_execz L_u35_0\n\t" \
  "s_mov_b32 s84, 1376256\n\t" \
  "buffer_load_dword v210, v198, s[36:39], s84 offen \n\t" \
  "L_u35_0:\n\t" \
  "s_mov_b64 exec, -1\n\t" \
  "s_bfe_u32 s84, s9, 262152\n\t" \
  "s_cmp_eq_u32 s84, 15\n\t" \
  "s_cbranch_scc1 L_qr27_0\n\t" \
  "s_sub_u32 s84, s88, s14\n\t" \
  "s_or_b32 s84, s84, s97\n\t" \
  "s_cmp_lt_u32 s84, 50000000\n\t" \
  "s_cbranch_scc1 L_q27_0\n\t" \
  "s_mov_b32 s97, -1\n\t" \
  "s_mov_b32 s81, 0x7fc00000\n\t" \
  "s_branch L_qr27_0\n\t" \
  "L_q28_0:\n\t" \
  "global_load_dword v226, v195, s[26:27] offset:3072 sc0 sc1\n\t" \
  "s_memrealtime s[88:89]\n\t" \
  "s_waitcnt vmcnt(0) lgkmcnt(0)\n\t" \
  "v_cmp_le_u32 vcc, s79, v226\n\t" \
  "s_nop 4\n\t" \
  "s_andn2_b32 s85, vcc_lo, s9\n\t" \
  "s_or_b32 s9, s9, vcc_lo\n\t" \
  "s_and_b32 exec_lo, s85, 0xfffffff\n\t" \
  "s_mov_b32 exec_hi, 0\n\t" \
  "s_cbranch_execz L_u36_0\n\t" \
  "s_mov_b32 s84, 1376256\n\t" \
  "buffer_load_dword v210, v198, s[36:39], s84 offen \n\t" \
  "L_u36_0:\n\t" \
  "s_mov_b64 exec, -1\n\t" \
  "s_bfe_u32 s84, s9, 262156\n\t" \
  "s_cmp_eq_u32 s84, 15\n\t" \
  "s_cbranch_scc1 L_qr28_0\n\t" \
  "s_sub_u32 s84, s88, s14\n\t" \
  "s_or_b32 s84, s84, s97\n\t" \
  "s_cmp_lt_u32 s84, 50000000\n\t" \
  "s_cbranch_scc1 L_q28_0\n\t" \
  "s_mov_b32 s97, -1\n\t" \
  "s_mov_b32 s81, 0x7fc00000\n\t" \
  "s_branch L_qr28_0\n\t" \
  "L_q29_0:\n\t" \
  "global_load_dword v226, v195, s[26:27] offset:3072 sc0 sc1\n\t" \
  "s_memrealtime s[88:89]\n\t" \
  "s_waitcnt vmcnt(0) lgkmcnt(0)\n\t" \
  "v_cmp_le_u32 vcc, s79, v226\n\t" \
  "s_nop 4\n\t" \
  "s_andn2_b32 s85, vcc_lo, s9\n\t" \
  "s_or_b32 s9, s9, vcc_lo\n\t" \
  "s_and_b32 exec_lo, s85, 0xfffffff\n\t" \
  "s_mov_b32 exec_hi, 0\n\t" \
  "s_cbranch_execz L_u37_0\n\t" \
  "s_mov_b32 s84, 1376256\n\t" \
  "buffer_load_dword v210, v198, s[36:39], s84 offen \n\t" \
  "L_u37_0:\n\t" \
  "s_mov_b64 exec, -1\n\t" \
  "s_bfe_u32 s84, s9, 262160\n\t" \
  "s_cmp_eq_u32 s84, 15\n\t" \
  "s_cbranch_scc1 L_qr29_0\n\t" \
  "s_sub_u32 s84, s88, s14\n\t" \
  "s_or_b32 s84, s84, s97\n\t" \
  "s_cmp_lt_u32 s84, 50000000\n\t" \
  "s_cbranch_scc1 L_q29_0\n\t" \
  "s_mov_b32 s97, -1\n\t" \
  "s_mov_b32 s81, 0x7fc00000\n\t" \
  "s_branch L_qr29_0\n\t" \
  "L_q30_0:\n\t" \
  "global_load_dword v226, v195, s[26:27] offset:3072 sc0 sc1\n\t" \
  "s_memrealtime s[88:89]\n\t" \
  "s_waitcnt vmcnt(0) lgkmcnt(0)\n\t" \
  "v_cmp_le_u32 vcc, s79, v226\n\t" \
  "s_nop 4\n\t" \
  "s_andn2_b32 s85, vcc_lo, s9\n\t" \
  "s_or_b32 s9, s9, vcc_lo\n\t" \
  "s_and_b32 exec_lo, s85, 0xfffffff\n\t" \
  "s_mov_b32 exec_hi, 0\n\t" \
  "s_cbranch_execz L_u38_0\n\t" \
  "s_mov_b32 s84, 1376256\n\t" \
  "buffer_load_dword v210, v198, s[36:39], s84 offen \n\t" \
  "L_u38_0:\n\t" \
  "s_mov_b64 exec, -1\n\t" \
  "s_bfe_u32 s84, s9, 262164\n\t" \
  "s_cmp_eq_u32 s84, 15\n\t" \
  "s_cbranch_scc1 L_qr30_0\n\t" \
  "s_sub_u32 s84, s88, s14\n\t" \
  "s_or_b32 s84, s84, s97\n\t" \
  "s_cmp_lt_u32 s84, 50000000\n\t" \
  "s_cbranch_scc1 L_q30_0\n\t" \
  "s_mov_b32 s97, -1\n\t" \
  "s_mov_b32 s81, 0x7fc00000\n\t" \
  "s_branch L_qr30_0\n\t" \
  "L_q31_0:\n\t" \
  "global_load_dword v226, v195, s[26:27] offset:3072 sc0 sc1\n\t" \
  "s_memrealtime s[88:89]\n\t" \
  "s_waitcnt vmcnt(0) lgkmcnt(0)\n\t" \
  "v_cmp_le_u32 vcc, s79, v226\n\t" \
  "s_nop 4\n\t" \
  "s_andn2_b32 s85, vcc_lo, s9\n\t" \
  "s_or_b32 s9, s9, vcc_lo\n\t" \
  "s_and_b32 exec_lo, s85, 0xfffffff\n\t" \
  "s_mov_b32 exec_hi, 0\n\t" \
  "s_cbranch_execz L_u39_0\n\t" \
  "s_mov_b32 s84, 1376256\n\t" \
  "buffer_load_dword v210, v198, s[36:39], s84 offen \n\t" \
  "L_u39_0:\n\t" \
  "s_mov_b64 exec, -1\n\t" \
  "s_bfe_u32 s84, s9, 262168\n\t" \
  "s_cmp_eq_u32 s84, 15\n\t" \
  "s_cbranch_scc1 L_qr31_0\n\t" \
  "s_sub_u32 s84, s88, s14\n\t" \
  "s_or_b32 s84, s84, s97\n\t" \
  "s_cmp_lt_u32 s84, 50000000\n\t" \
  "s_cbranch_scc1 L_q31_0\n\t" \
  "s_mov_b32 s97, -1\n\t" \
  "s_mov_b32 s81, 0x7fc00000\n\t" \
  "s_branch L_qr31_0\n\t" \
  "H_PH1:\n\t" \
  "s_add_u32 m0, s77, 0\n\t" \
  "s_add_u32 s90, s75, 0\n\t" \
  "buffer_load_dwordx4 v186, s[40:43], s90 offen offset:0  lds\n\t" \
  "s_add_u32 m0, s77, 0\n\t" \
  "s_add_u32 s90, s75, 0\n\t" \
  "buffer_load_dwordx4 v187, s[40:43], s90 offen offset:1024  lds\n\t" \
  "s_add_u32 m0, s78, 65536\n\t" \
  "s_movk_i32 s84, 0\n\t" \
  "buffer_load_dwordx4 v190, s[32:35], s84 offen offset:0 lds\n\t" \
  "s_add_u32 m0, s78, 65536\n\t" \
  "s_movk_i32 s84, 3072\n\t" \
  "buffer_load_dwordx4 v191, s[32:35], s84 offen offset:1024 lds\n\t" \
  "s_add_u32 m0, s78, 65536\n\t" \
  "s_movk_i32 s84, 6144\n\t" \
  "buffer_load_dwordx4 v190, s[32:35], s84 offen offset:2048 lds\n\t" \
  "s_add_u32 m0, s78, 65536\n\t" \
  "s_movk_i32 s84, 9216\n\t" \
  "buffer_load_dwordx4 v191, s[32:35], s84 offen offset:3072 lds\n\t" \
  "s_add_u32 m0, s78, 69632\n\t" \
  "s_movk_i32 s84, 16384\n\t" \
  "buffer_load_dwordx4 v190, s[32:35], s84 offen offset:0 lds\n\t" \
  "s_add_u32 m0, s78, 69632\n\t" \
  "s_movk_i32 s84, 19456\n\t" \
  "buffer_load_dwordx4 v191, s[32:35], s84 offen offset:1024 lds\n\t" \
  "s_add_u32 m0, s78, 69632\n\t" \
  "s_movk_i32 s84, 22528\n\t" \
  "buffer_load_dwordx4 v190, s[32:35], s84 offen offset:2048 lds\n\t" \
  "s_add_u32 m0, s78, 69632\n\t" \
  "s_movk_i32 s84, 25600\n\t" \
  "buffer_load_dwordx4 v191, s[32:35], s84 offen offset:3072 lds\n\t" \
  "s_add_u32 m0, s77, 16384\n\t" \
  "s_add_u32 s90, s75, 2097152\n\t" \
  "buffer_load_dwordx4 v186, s[40:43], s90 offen offset:0  lds\n\t" \
  "s_add_u32 m0, s77, 16384\n\t" \
  "s_add_u32 s90, s75, 2097152\n\t" \
  "buffer_load_dwordx4 v187, s[40:43], s90 offen offset:1024  lds\n\t" \
  "s_add_u32 m0, s78, 98304\n\t" \
  "s_movk_i32 s84, 128\n\t" \
  "buffer_load_dwordx4 v190, s[32:35], s84 offen offset:0 lds\n\t" \
  "s_add_u32 m0, s78, 98304\n\t" \
  "s_movk_i32 s84, 3200\n\t" \
  "buffer_load_dwordx4 v191, s[32:35], s84 offen offset:1024 lds\n\t" \
  "s_add_u32 m0, s78, 98304\n\t" \
  "s_movk_i32 s84, 6272\n\t" \
  "buffer_load_dwordx4 v190, s[32:35], s84 offen offset:2048 lds\n\t" \
  "s_add_u32 m0, s78, 98304\n\t" \
  "s_movk_i32 s84, 9344\n\t" \
  "buffer_load_dwordx4 v191, s[32:35], s84 offen offset:3072 lds\n\t" \
  "s_add_u32 m0, s78, 102400\n\t" \
  "s_movk_i32 s84, 16512\n\t" \
  "buffer_load_dwordx4 v190, s[32:35], s84 offen offset:0 lds\n\t" \
  "s_add_u32 m0, s78, 102400\n\t" \
  "s_movk_i32 s84, 19584\n\t" \
  "buffer_load_dwordx4 v191, s[32:35], s84 offen offset:1024 lds\n\t" \
  "s_add_u32 m0, s78, 102400\n\t" \
  "s_movk_i32 s84, 22656\n\t" \
  "buffer_load_dwordx4 v190, s[32:35], s84 offen offset:2048 lds\n\t" \
  "s_add_u32 m0, s78, 102400\n\t" \
  "s_movk_i32 s84, 25728\n\t" \
  "buffer_load_dwordx4 v191, s[32:35], s84 offen offset:3072 lds\n\t" \
  "v_lshlrev_b32 v194, 2, v213\n\t" \
  "v_lshl_add_u32 v194, v214, 18, v194\n\t" \
  "s_lshl_b32 s86, s80, 22\n\t" \
  "s_lshl_b32 s85, s30, 9\n\t" \
  "s_add_u32 s86, s86, s85\n\t" \
  "s_add_u32 s44, s24, s86\n\t" \
  "s_addc_u32 s45, s25, 0\n\t" \
  "s_and_b32 s45, s45, 0xffff\n\t" \
  "s_mov_b32 s46, 4129280\n\t" \
  "s_mov_b32 s47, 0x20000\n\t" \
  "s_lshl_b32 s86, s98, 2\n\t" \
  "v_add_u32 v197, s86, v214\n\t" \
  "v_lshlrev_b32 v197, 14, v197\n\t" \
  "v_lshl_add_u32 v197, v213, 7, v197\n\t" \
  "s_lshl2_add_u32 s86, s31, s29\n\t" \
  "s_lshl_b32 s86, s86, 4\n\t" \
  "v_add_u32 v241, s86, v213\n\t" \
  "v_lshlrev_b32 v241, 14, v241\n\t" \
  "v_min_u32 v217, 27, v212\n\t" \
  "s_lshl_b32 s86, s80, 7\n\t" \
  "s_add_u32 s86, s86, 6291456\n\t" \
  "v_lshl_add_u32 v195, v217, 2, s86\n\t" \
  "v_lshrrev_b32 v218, 2, v217\n\t" \
  "v_lshlrev_b32 v218, 16, v218\n\t" \
  "v_and_b32 v217, 3, v217\n\t" \
  "v_lshl_add_u32 v218, v217, 11, v218\n\t" \
  "s_lshl_b32 s86, s99, 7\n\t" \
  "v_add_u32 v198, s86, v218\n\t" \
  "v_subrev_u32 v217, 4, v212\n\t" \
  "v_and_b32 v217, 63, v217\n\t" \
  "v_mul_u32_u24 v218, 37, v217\n\t" \
  "v_lshrrev_b32 v218, 8, v218\n\t" \
  "v_mul_u32_u24 v219, 7, v218\n\t" \
  "v_sub_u32 v217, v217, v219\n\t" \
  "v_add3_u32 v217, v217, s28, 1\n\t" \
  "v_and_b32 v217, 7, v217\n\t" \
  "v_lshl_add_u32 v217, v217, 2, v218\n\t" \
  "s_lshl_b32 s84, s28, 2\n\t" \
  "v_add_u32 v218, s84, v212\n\t" \
  "v_cmp_gt_u32 vcc, 4, v212\n\t" \
  "s_nop 1\n\t" \
  "v_cndmask_b32 v217, v217, v218, vcc\n\t" \
  "v_lshlrev_b32 v211, 21, v217\n\t" \
  "s_waitcnt vmcnt(10)\n\t" \
  "s_cmp_eq_u32 s29, 0\n\t" \
  "s_cbranch_scc0 L_fl0_2\n\t" \
  "s_mov_b64 exec, 1\n\t" \
  "buffer_store_dword v208, v209, s[56:59], 0 offen sc0 sc1\n\t" \
  "s_mov_b64 exec, -1\n\t" \
  "L_fl0_2:\n\t" \
  "ds_read_b128 a[0:3], v192 offset:0\n\t" \
  "ds_read_b128 a[4:7], v193 offset:0\n\t" \
  "ds_read_b128 a[8:11], v192 offset:2048\n\t" \
  "ds_read_b128 a[12:15], v193 offset:2048\n\t" \
  "ds_read_b128 a[16:19], v192 offset:4096\n\t" \
  "ds_read_b128 a[20:23], v193 offset:4096\n\t" \
  "ds_read_b128 a[24:27], v192 offset:6144\n\t" \
  "ds_read_b128 a[28:31], v193 offset:6144\n\t" \
  "s_waitcnt lgkmcnt(0)\n\t" \
  "s_add_u32 m0, s76, 32768\n\t" \
  "s_add_u32 s90, s75, 4194304\n\t" \
  "buffer_load_dwordx4 v184, s[40:43], s90 offen offset:0  lds\n\t" \
  "s_add_u32 m0, s76, 32768\n\t" \
  "s_add_u32 s90, s75, 4194304\n\t" \
  "buffer_load_dwordx4 v185, s[40:43], s90 offen offset:1024  lds\n\t" \
  "s_add_u32 m0, s76, 49152\n\t" \
  "s_add_u32 s90, s75, 6291456\n\t" \
  "buffer_load_dwordx4 v184, s[40:43], s90 offen offset:0  lds\n\t" \
  "s_add_u32 m0, s76, 49152\n\t" \
  "s_add_u32 s90, s75, 6291456\n\t" \
  "buffer_load_dwordx4 v185, s[40:43], s90 offen offset:1024  lds\n\t" \
  "s_cmp_eq_u32 s29, 1\n\t" \
  "s_cbranch_scc0 L_ps1_2\n\t" \
  "buffer_store_dwordx4 v[200:203], v188, s[52:55], 0 offen offset:0 sc0 sc1\n\t" \
  "buffer_store_dwordx4 v[204:207], v188, s[52:55], 0 offen offset:1024 sc0 sc1\n\t" \
  "L_ps1_2:\n\t" \
  "s_barrier\n\t" \
  "ds_read_b64_tr_b8 v[128:129], v160 offset:0\n\t" \
  "ds_read_b64_tr_b8 v[130:131], v160 offset:1024\n\t" \
  "ds_read_b64_tr_b8 v[132:133], v160 offset:8192\n\t" \
  "ds_read_b64_tr_b8 v[134:135], v160 offset:9216\n\t" \
  "ds_read_b64_tr_b8 v[136:137], v161 offset:0\n\t" \
  "ds_read_b64_tr_b8 v[138:139], v161 offset:1024\n\t" \
  "ds_read_b64_tr_b8 v[140:141], v161 offset:8192\n\t" \
  "ds_read_b64_tr_b8 v[142:143], v161 offset:9216\n\t" \
  "s_waitcnt lgkmcnt(4)\n\t" \
  "v_mfma_f32_16x16x128_f8f6f4 v[0:3], a[0:7], v[128:135], 0 cbsz:0 blgp:1\n\t" \
  "ds_read_b64_tr_b8 v[144:145], v162 offset:0\n\t" \
  "s_add_u32 m0, s78, 131072\n\t" \
  "s_movk_i32 s84, 256\n\t" \
  "buffer_load_dwordx4 v190, s[32:35], s84 offen offset:0 lds\n\t" \
  "v_mfma_f32_16x16x128_f8f6f4 v[32:35], a[8:15], v[128:135], 0 cbsz:0 blgp:1\n\t" \
  "ds_read_b64_tr_b8 v[146:147], v162 offset:1024\n\t" \
  "s_add_u32 s99, s29, 4\n\t" \
  "s_min_u32 s99, s99, 31\n\t" \
  "v_readlane_b32 s99, v211, s99\n\t" \
  "s_mov_b32 exec_lo, s15\n\t" \
  "s_mov_b32 exec_hi, 0\n\t" \
  "s_cbranch_execz L_c4_2\n\t" \
  "s_nop 1\n\t" \
  "buffer_load_dword v210, v222, s[60:63], s99 offen\n\t" \
  "L_c4_2:\n\t" \
  "s_mov_b64 exec, -1\n\t" \
  "v_mfma_f32_16x16x128_f8f6f4 v[64:67], a[16:23], v[128:135], 0 cbsz:0 blgp:1\n\t" \
  "ds_read_b64_tr_b8 v[148:149], v162 offset:8192\n\t" \
  "s_add_u32 m0, s78, 131072\n\t" \
  "s_movk_i32 s84, 3328\n\t" \
  "buffer_load_dwordx4 v191, s[32:35], s84 offen offset:1024 lds\n\t" \
  "v_mfma_f32_16x16x128_f8f6f4 v[96:99], a[24:31], v[128:135], 0 cbsz:0 blgp:1\n\t" \
  "ds_read_b64_tr_b8 v[150:151], v162 offset:9216\n\t" \
  "s_add_u32 s99, s29, 8\n\t" \
  "s_min_u32 s99, s99, 31\n\t" \
  "v_readlane_b32 s99, v211, s99\n\t" \
  "s_mov_b32 exec_lo, s15\n\t" \
  "s_mov_b32 exec_hi, 0\n\t" \
  "s_cbranch_execz L_c8_2\n\t" \
  "s_nop 1\n\t" \
  "buffer_load_dword v210, v222, s[60:63], s99 offen\n\t" \
  "L_c8_2:\n\t" \
  "s_mov_b64 exec, -1\n\t" \
  "s_waitcnt lgkmcnt(4)\n\t" \
  "v_mfma_f32_16x16x128_f8f6f4 v[4:7], a[0:7], v[136:143], 0 cbsz:0 blgp:1\n\t" \
  "ds_read_b64_tr_b8 v[152:153], v163 offset:0\n\t" \
  "s_add_u32 m0, s78, 131072\n\t" \
  "s_movk_i32 s84, 6400\n\t" \
  "buffer_load_dwordx4 v190, s[32:35], s84 offen offset:2048 lds\n\t" \
  "v_mfma_f32_16x16x128_f8f6f4 v[36:39], a[8:15], v[136:143], 0 cbsz:0 blgp:1\n\t" \
  "ds_read_b64_tr_b8 v[154:155], v163 offset:1024\n\t" \
  "v_mfma_f32_16x16x128_f8f6f4 v[68:71], a[16:23], v[136:143], 0 cbsz:0 blgp:1\n\t" \
  "ds_read_b64_tr_b8 v[156:157], v163 offset:8192\n\t" \
  "s_add_u32 m0, s78, 131072\n\t" \
  "s_movk_i32 s84, 9472\n\t" \
  "buffer_load_dwordx4 v191, s[32:35], s84 offen offset:3072 lds\n\t" \
  "v_mfma_f32_16x16x128_f8f6f4 v[100:103], a[24:31], v[136:143], 0 cbsz:0 blgp:1\n\t" \
  "ds_read_b64_tr_b8 v[158:159], v163 offset:9216\n\t" \
  "s_waitcnt lgkmcnt(4)\n\t" \
  "v_mfma_f32_16x16x128_f8f6f4 v[8:11], a[0:7], v[144:151], 0 cbsz:0 blgp:1\n\t" \
  "ds_read_b64_tr_b8 v[128:129], v164 offset:0\n\t" \
  "s_waitcnt vmcnt(14)\n\t" \
  "ds_read_b128 a[32:35], v192 offset:32768\n\t" \
  "s_add_u32 m0, s78, 135168\n\t" \
  "s_movk_i32 s84, 16640\n\t" \
  "buffer_load_dwordx4 v190, s[32:35], s84 offen offset:0 lds\n\t" \
  "v_mfma_f32_16x16x128_f8f6f4 v[40:43], a[8:15], v[144:151], 0 cbsz:0 blgp:1\n\t" \
  "ds_read_b64_tr_b8 v[130:131], v164 offset:1024\n\t" \
  "ds_read_b128 a[36:39], v193 offset:32768\n\t" \
  "v_mfma_f32_16x16x128_f8f6f4 v[72:75], a[16:23], v[144:151], 0 cbsz:0 blgp:1\n\t" \
  "ds_read_b64_tr_b8 v[132:133], v164 offset:8192\n\t" \
  "s_add_u32 m0, s78, 135168\n\t" \
  "s_movk_i32 s84, 19712\n\t" \
  "buffer_load_dwordx4 v191, s[32:35], s84 offen offset:1024 lds\n\t" \
  "v_mfma_f32_16x16x128_f8f6f4 v[104:107], a[24:31], v[144:151], 0 cbsz:0 blgp:1\n\t" \
  "ds_read_b64_tr_b8 v[134:135], v164 offset:9216\n\t" \
  "s_waitcnt lgkmcnt(6)\n\t" \
  "v_mfma_f32_16x16x128_f8f6f4 v[12:15], a[0:7], v[152:159], 0 cbsz:0 blgp:1\n\t" \
  "ds_read_b64_tr_b8 v[136:137], v165 offset:0\n\t" \
  "s_waitcnt vmcnt(14)\n\t" \
  "ds_read_b128 a[40:43], v192 offset:34816\n\t" \
  "s_add_u32 m0, s78, 135168\n\t" \
  "s_movk_i32 s84, 22784\n\t" \
  "buffer_load_dwordx4 v190, s[32:35], s84 offen offset:2048 lds\n\t" \
  "v_mfma_f32_16x16x128_f8f6f4 v[44:47], a[8:15], v[152:159], 0 cbsz:0 blgp:1\n\t" \
  "ds_read_b64_tr_b8 v[138:139], v165 offset:1024\n\t" \
  "ds_read_b128 a[44:47], v193 offset:34816\n\t" \
  "v_mfma_f32_16x16x128_f8f6f4 v[76:79], a[16:23], v[152:159], 0 cbsz:0 blgp:1\n\t" \
  "ds_read_b64_tr_b8 v[140:141], v165 offset:8192\n\t" \
  "s_add_u32 m0, s78, 135168\n\t" \
  "s_movk_i32 s84, 25856\n\t" \
  "buffer_load_dwordx4 v191, s[32:35], s84 offen offset:3072 lds\n\t" \
  "v_mfma_f32_16x16x128_f8f6f4 v[108:111], a[24:31], v[152:159], 0 cbsz:0 blgp:1\n\t" \
  "ds_read_b64_tr_b8 v[142:143], v165 offset:9216\n\t" \
  "s_waitcnt lgkmcnt(6)\n\t" \
  "v_mfma_f32_16x16x128_f8f6f4 v[16:19], a[0:7], v[128:135], 0 cbsz:0 blgp:1\n\t" \
  "ds_read_b64_tr_b8 v[144:145], v166 offset:0\n\t" \
  "s_waitcnt vmcnt(14)\n\t" \
  "ds_read_b128 a[48:51], v192 offset:36864\n\t" \
  "s_add_u32 m0, s78, 65536\n\t" \
  "s_movk_i32 s84, 384\n\t" \
  "buffer_load_dwordx4 v190, s[32:35], s84 offen offset:0 lds\n\t" \
  "v_mfma_f32_16x16x128_f8f6f4 v[48:51], a[8:15], v[128:135], 0 cbsz:0 blgp:1\n\t" \
  "ds_read_b64_tr_b8 v[146:147], v166 offset:1024\n\t" \
  "ds_read_b128 a[52:55], v193 offset:36864\n\t" \
  "v_mfma_f32_16x16x128_f8f6f4 v[80:83], a[16:23], v[128:135], 0 cbsz:0 blgp:1\n\t" \
  "ds_read_b64_tr_b8 v[148:149], v166 offset:8192\n\t" \
  "s_add_u32 m0, s78, 65536\n\t" \
  "s_movk_i32 s84, 3456\n\t" \
  "buffer_load_dwordx4 v191, s[32:35], s84 offen offset:1024 lds\n\t" \
  "v_mfma_f32_16x16x128_f8f6f4 v[112:115], a[24:31], v[128:135], 0 cbsz:0 blgp:1\n\t" \
  "ds_read_b64_tr_b8 v[150:151], v166 offset:9216\n\t" \
  "s_waitcnt lgkmcnt(6)\n\t" \
  "v_mfma_f32_16x16x128_f8f6f4 v[20:23], a[0:7], v[136:143], 0 cbsz:0 blgp:1\n\t" \
  "ds_read_b64_tr_b8 v[152:153], v167 offset:0\n\t" \
  "s_waitcnt vmcnt(14)\n\t" \
  "ds_read_b128 a[56:59], v192 offset:38912\n\t" \
  "s_add_u32 m0, s78, 65536\n\t" \
  "s_movk_i32 s84, 6528\n\t" \
  "buffer_load_dwordx4 v190, s[32:35], s84 offen offset:2048 lds\n\t" \
  "v_mfma_f32_16x16x128_f8f6f4 v[52:55], a[8:15], v[136:143], 0 cbsz:0 blgp:1\n\t" \
  "ds_read_b64_tr_b8 v[154:155], v167 offset:1024\n\t" \
  "ds_read_b128 a[60:63], v193 offset:38912\n\t" \
  "v_mfma_f32_16x16x128_f8f6f4 v[84:87], a[16:23], v[136:143], 0 cbsz:0 blgp:1\n\t" \
  "ds_read_b64_tr_b8 v[156:157], v167 offset:8192\n\t" \
  "s_add_u32 m0, s78, 65536\n\t" \
  "s_movk_i32 s84, 9600\n\t" \
  "buffer_load_dwordx4 v191, s[32:35], s84 offen offset:3072 lds\n\t" \
  "v_mfma_f32_16x16x128_f8f6f4 v[116:119], a[24:31], v[136:143], 0 cbsz:0 blgp:1\n\t" \
  "ds_read_b64_tr_b8 v[158:159], v167 offset:9216\n\t" \
  "s_waitcnt lgkmcnt(6)\n\t" \
  "v_mfma_f32_16x16x128_f8f6f4 v[24:27], a[0:7], v[144:151], 0 cbsz:0 blgp:1\n\t" \
  "s_waitcnt lgkmcnt(2)\n\t" \
  "s_barrier\n\t" \
  "ds_read_b64_tr_b8 v[128:129], v160 offset:16384\n\t" \
  "v_mfma_f32_16x16x128_f8f6f4 v[56:59], a[8:15], v[144:151], 0 cbsz:0 blgp:1\n\t" \
  "ds_read_b64_tr_b8 v[130:131], v160 offset:17408\n\t" \
  "s_add_u32 m0, s78, 69632\n\t" \
  "s_movk_i32 s84, 16768\n\t" \
  "buffer_load_dwordx4 v190, s[32:35], s84 offen offset:0 lds\n\t" \
  "v_mfma_f32_16x16x128_f8f6f4 v[88:91], a[16:23], v[144:151], 0 cbsz:0 blgp:1\n\t" \
  "ds_read_b64_tr_b8 v[132:133], v160 offset:24576\n\t" \
  "v_mfma_f32_16x16x128_f8f6f4 v[120:123], a[24:31], v[144:151], 0 cbsz:0 blgp:1\n\t" \
  "ds_read_b64_tr_b8 v[134:135], v160 offset:25600\n\t" \
  "s_add_u32 m0, s78, 69632\n\t" \
  "s_movk_i32 s84, 19840\n\t" \
  "buffer_load_dwordx4 v191, s[32:35], s84 offen offset:1024 lds\n\t" \
  "s_waitcnt lgkmcnt(4)\n\t" \
  "v_mfma_f32_16x16x128_f8f6f4 v[28:31], a[0:7], v[152:159], 0 cbsz:0 blgp:1\n\t" \
  "ds_read_b64_tr_b8 v[136:137], v161 offset:16384\n\t" \
  "v_mfma_f32_16x16x128_f8f6f4 v[60:63], a[8:15], v[152:159], 0 cbsz:0 blgp:1\n\t" \
  "ds_read_b64_tr_b8 v[138:139], v161 offset:17408\n\t" \
  "s_add_u32 m0, s78, 69632\n\t" \
  "s_movk_i32 s84, 22912\n\t" \
  "buffer_load_dwordx4 v190, s[32:35], s84 offen offset:2048 lds\n\t" \
  "v_mfma_f32_16x16x128_f8f6f4 v[92:95], a[16:23], v[152:159], 0 cbsz:0 blgp:1\n\t" \
  "ds_read_b64_tr_b8 v[140:141], v161 offset:24576\n\t" \
  "s_cmp_eq_u32 s29, 3\n\t" \
  "s_cbranch_scc0 L_ct_2\n\t" \
  "v_mov_b32 v212, s83\n\t" \
  "s_mov_b64 exec, 1\n\t" \
  "global_store_dword v212, v208, s[26:27] sc1\n\t" \
  "s_mov_b64 exec, -1\n\t" \
  "L_ct_2:\n\t" \
  "v_mfma_f32_16x16x128_f8f6f4 v[124:127], a[24:31], v[152:159], 0 cbsz:0 blgp:1\n\t" \
  "ds_read_b64_tr_b8 v[142:143], v161 offset:25600\n\t" \
  "s_add_u32 m0, s78, 69632\n\t" \
  "s_movk_i32 s84, 25984\n\t" \
  "buffer_load_dwordx4 v191, s[32:35], s84 offen offset:3072 lds\n\t" \
  "s_waitcnt lgkmcnt(4)\n\t" \
  "v_mfma_f32_16x16x128_f8f6f4 v[0:3], a[32:39], v[128:135], v[0:3] cbsz:0 blgp:1\n\t" \
  "ds_read_b64_tr_b8 v[144:145], v162 offset:16384\n\t" \
  "v_mfma_f32_16x16x128_f8f6f4 v[32:35], a[40:47], v[128:135], v[32:35] cbsz:0 blgp:1\n\t" \
  "ds_read_b64_tr_b8 v[146:147], v162 offset:17408\n\t" \
  "v_mfma_f32_16x16x128_f8f6f4 v[64:67], a[48:55], v[128:135], v[64:67] cbsz:0 blgp:1\n\t" \
  "ds_read_b64_tr_b8 v[148:149], v162 offset:24576\n\t" \
  "s_add_u32 m0, s77, 32768\n\t" \
  "s_add_u32 s90, s75, 4194304\n\t" \
  "buffer_load_dwordx4 v186, s[40:43], s90 offen offset:0  lds\n\t" \
  "v_mfma_f32_16x16x128_f8f6f4 v[96:99], a[56:63], v[128:135], v[96:99] cbsz:0 blgp:1\n\t" \
  "ds_read_b64_tr_b8 v[150:151], v162 offset:25600\n\t" \
  "s_add_u32 m0, s77, 32768\n\t" \
  "s_add_u32 s90, s75, 4194304\n\t" \
  "buffer_load_dwordx4 v187, s[40:43], s90 offen offset:1024  lds\n\t" \
  "s_waitcnt lgkmcnt(4)\n\t" \
  "v_mfma_f32_16x16x128_f8f6f4 v[4:7], a[32:39], v[136:143], v[4:7] cbsz:0 blgp:1\n\t" \
  "ds_read_b64_tr_b8 v[152:153], v163 offset:16384\n\t" \
  "s_add_u32 m0, s77, 49152\n\t" \
  "s_add_u32 s90, s75, 6291456\n\t" \
  "buffer_load_dwordx4 v186, s[40:43], s90 offen offset:0  lds\n\t" \
  "v_mfma_f32_16x16x128_f8f6f4 v[36:39], a[40:47], v[136:143], v[36:39] cbsz:0 blgp:1\n\t" \
  "ds_read_b64_tr_b8 v[154:155], v163 offset:17408\n\t" \
  "s_add_u32 m0, s77, 49152\n\t" \
  "s_add_u32 s90, s75, 6291456\n\t" \
  "buffer_load_dwordx4 v187, s[40:43], s90 offen offset:1024  lds\n\t" \
  "v_mfma_f32_16x16x128_f8f6f4 v[68:71], a[48:55], v[136:143], v[68:71] cbsz:0 blgp:1\n\t" \
  "ds_read_b64_tr_b8 v[156:157], v163 offset:24576\n\t" \
  "v_mfma_f32_16x16x128_f8f6f4 v[100:103], a[56:63], v[136:143], v[100:103] cbsz:0 blgp:1\n\t" \
  "ds_read_b64_tr_b8 v[158:159], v163 offset:25600\n\t" \
  "s_waitcnt lgkmcnt(4)\n\t" \
  "v_mfma_f32_16x16x128_f8f6f4 v[8:11], a[32:39], v[144:151], v[8:11] cbsz:0 blgp:1\n\t" \
  "ds_read_b64_tr_b8 v[128:129], v164 offset:16384\n\t" \
  "s_waitcnt vmcnt(18)\n\t" \
  "ds_read_b128 a[64:67], v224 offset:0\n\t" \
  "v_mfma_f32_16x16x128_f8f6f4 v[40:43], a[40:47], v[144:151], v[40:43] cbsz:0 blgp:1\n\t" \
  "ds_read_b64_tr_b8 v[130:131], v164 offset:17408\n\t" \
  "ds_read_b128 a[68:71], v225 offset:0\n\t" \
  "v_mfma_f32_16x16x128_f8f6f4 v[72:75], a[48:55], v[144:151], v[72:75] cbsz:0 blgp:1\n\t" \
  "ds_read_b64_tr_b8 v[132:133], v164 offset:24576\n\t" \
  "s_waitcnt vmcnt(16)\n\t" \
  "ds_read_b128 a[72:75], v224 offset:2048\n\t" \
  "v_mfma_f32_16x16x128_f8f6f4 v[104:107], a[56:63], v[144:151], v[104:107] cbsz:0 blgp:1\n\t" \
  "ds_read_b64_tr_b8 v[134:135], v164 offset:25600\n\t" \
  "ds_read_b128 a[76:79], v225 offset:2048\n\t" \
  "s_waitcnt lgkmcnt(8)\n\t" \
  "v_mfma_f32_16x16x128_f8f6f4 v[12:15], a[32:39], v[152:159], v[12:15] cbsz:0 blgp:1\n\t" \
  "ds_read_b64_tr_b8 v[136:137], v165 offset:16384\n\t" \
  "s_waitcnt vmcnt(14)\n\t" \
  "ds_read_b128 a[80:83], v224 offset:4096\n\t" \
  "v_mfma_f32_16x16x128_f8f6f4 v[44:47], a[40:47], v[152:159], v[44:47] cbsz:0 blgp:1\n\t" \
  "ds_read_b64_tr_b8 v[138:139], v165 offset:17408\n\t" \
  "ds_read_b128 a[84:87], v225 offset:4096\n\t" \
  "v_mfma_f32_16x16x128_f8f6f4 v[76:79], a[48:55], v[152:159], v[76:79] cbsz:0 blgp:1\n\t" \
  "ds_read_b64_tr_b8 v[140:141], v165 offset:24576\n\t" \
  "s_waitcnt vmcnt(12)\n\t" \
  "ds_read_b128 a[88:91], v224 offset:6144\n\t" \
  "v_mfma_f32_16x16x128_f8f6f4 v[108:111], a[56:63], v[152:159], v[108:111] cbsz:0 blgp:1\n\t" \
  "ds_read_b64_tr_b8 v[142:143], v165 offset:25600\n\t" \
  "ds_read_b128 a[92:95], v225 offset:6144\n\t" \
  "s_waitcnt vmcnt(10)\n\t" \
  "ds_read_b128 a[96:99], v192 offset:0\n\t" \
  "s_waitcnt lgkmcnt(10)\n\t" \
  "v_mfma_f32_16x16x128_f8f6f4 v[16:19], a[32:39], v[128:135], v[16:19] cbsz:0 blgp:1\n\t" \
  "ds_read_b64_tr_b8 v[144:145], v166 offset:16384\n\t" \
  "ds_read_b128 a[100:103], v193 offset:0\n\t" \
  "v_mfma_f32_16x16x128_f8f6f4 v[48:51], a[40:47], v[128:135], v[48:51] cbsz:0 blgp:1\n\t" \
  "ds_read_b64_tr_b8 v[146:147], v166 offset:17408\n\t" \
  "s_waitcnt vmcnt(8)\n\t" \
  "ds_read_b128 a[104:107], v192 offset:2048\n\t" \
  "v_mfma_f32_16x16x128_f8f6f4 v[80:83], a[48:55], v[128:135], v[80:83] cbsz:0 blgp:1\n\t" \
  "ds_read_b64_tr_b8 v[148:149], v166 offset:24576\n\t" \
  "ds_read_b128 a[108:111], v193 offset:2048\n\t" \
  "s_mov_b64 exec, 0xfffffff\n\t" \
  "s_mov_b32 s84, 0\n\t" \
  "buffer_load_dword v210, v198, s[36:39], s84 offen sc1\n\t" \
  "s_mov_b64 exec, -1\n\t" \
  "v_mfma_f32_16x16x128_f8f6f4 v[112:115], a[56:63], v[128:135], v[112:115] cbsz:0 blgp:1\n\t" \
  "ds_read_b64_tr_b8 v[150:151], v166 offset:25600\n\t" \
  "s_waitcnt vmcnt(7)\n\t" \
  "ds_read_b128 a[112:115], v192 offset:4096\n\t" \
  "s_mov_b32 s87, 0\n\t" \
  "buffer_load_dwordx4 a[128:131], v188, s[36:39], s87 offen offset:0 sc1\n\t" \
  "s_waitcnt lgkmcnt(10)\n\t" \
  "v_mfma_f32_16x16x128_f8f6f4 v[20:23], a[32:39], v[136:143], v[20:23] cbsz:0 blgp:1\n\t" \
  "ds_read_b64_tr_b8 v[152:153], v167 offset:16384\n\t" \
  "ds_read_b128 a[116:119], v193 offset:4096\n\t" \
  "buffer_load_dwordx4 a[132:135], v188, s[36:39], s87 offen offset:1024 sc1\n\t" \
  "v_mfma_f32_16x16x128_f8f6f4 v[52:55], a[40:47], v[136:143], v[52:55] cbsz:0 blgp:1\n\t" \
  "ds_read_b64_tr_b8 v[154:155], v167 offset:17408\n\t" \
  "s_waitcnt vmcnt(7)\n\t" \
  "ds_read_b128 a[120:123], v192 offset:6144\n\t" \
  "buffer_load_dwordx4 a[136:139], v188, s[36:39], s87 offen offset:2048 sc1\n\t" \
  "v_mfma_f32_16x16x128_f8f6f4 v[84:87], a[48:55], v[136:143], v[84:87] cbsz:0 blgp:1\n\t" \
  "ds_read_b64_tr_b8 v[156:157], v167 offset:24576\n\t" \
  "ds_read_b128 a[124:127], v193 offset:6144\n\t" \
  "buffer_load_dwordx4 a[140:143], v188, s[36:39], s87 offen offset:3072 sc1\n\t" \
  "v_mfma_f32_16x16x128_f8f6f4 v[116:119], a[56:63], v[136:143], v[116:119] cbsz:0 blgp:1\n\t" \
  "ds_read_b64_tr_b8 v[158:159], v167 offset:25600\n\t" \
  "buffer_load_dwordx4 a[144:147], v189, s[36:39], s87 offen offset:0 sc1\n\t" \
  "s_waitcnt lgkmcnt(8)\n\t" \
  "v_mfma_f32_16x16x128_f8f6f4 v[24:27], a[32:39], v[144:151], v[24:27] cbsz:0 blgp:1\n\t" \
  "s_waitcnt vmcnt(8) lgkmcnt(1)\n\t" \
  "s_barrier\n\t" \
  "ds_read_b64_tr_b8 v[128:129], v160 offset:32768\n\t" \
  "v_mfma_f32_16x16x128_f8f6f4 v[56:59], a[40:47], v[144:151], v[56:59] cbsz:0 blgp:1\n\t" \
  "ds_read_b64_tr_b8 v[130:131], v160 offset:33792\n\t" \
  "s_add_u32 m0, s76, 65536\n\t" \
  "s_add_u32 s90, s68, 0\n\t" \
  "buffer_load_dwordx4 v184, s[40:43], s90 offen offset:0  lds\n\t" \
  "s_add_u32 m0, s76, 65536\n\t" \
  "s_add_u32 s90, s68, 0\n\t" \
  "buffer_load_dwordx4 v185, s[40:43], s90 offen offset:1024  lds\n\t" \
  "s_add_u32 m0, s76, 81920\n\t" \
  "s_add_u32 s90, s69, 0\n\t" \
  "buffer_load_dwordx4 v184, s[40:43], s90 offen offset:0  lds\n\t" \
  "s_add_u32 m0, s76, 81920\n\t" \
  "s_add_u32 s90, s69, 0\n\t" \
  "buffer_load_dwordx4 v185, s[40:43], s90 offen offset:1024  lds\n\t" \
  "s_add_u32 s99, s29, 12\n\t" \
  "s_min_u32 s99, s99, 31\n\t" \
  "v_readlane_b32 s99, v211, s99\n\t" \
  "s_mov_b32 exec_lo, s15\n\t" \
  "s_mov_b32 exec_hi, 0\n\t" \
  "s_cbranch_execz L_c12_2\n\t" \
  "s_nop 1\n\t" \
  "buffer_load_dword v210, v222, s[60:63], s99 offen\n\t" \
  "L_c12_2:\n\t" \
  "s_mov_b64 exec, -1\n\t" \
  "v_mfma_f32_16x16x128_f8f6f4 v[88:91], a[48:55], v[144:151], v[88:91] cbsz:0 blgp:1\n\t" \
  "ds_read_b64_tr_b8 v[132:133], v160 offset:40960\n\t" \
  "s_cmp_eq_u32 s29, 2\n\t" \
  "s_cbranch_scc0 L_ps2_2\n\t" \
  "buffer_store_dwordx4 v[200:203], v188, s[52:55], 0 offen offset:0 sc0 sc1\n\t" \
  "buffer_store_dwordx4 v[204:207], v188, s[52:55], 0 offen offset:1024 sc0 sc1\n\t" \
  "L_ps2_2:\n\t" \
  "s_cmp_lt_u32 s29, 2\n\t" \
  "s_cbranch_scc1 L_sk_2\n\t" \
  "s_getpc_b64 s[88:89]\n\t" \
  "L_skp_2:\n\t" \
  "s_lshl_b32 s84, s29, 13\n\t" \
  "s_add_u32 s84, s84, L_slow_2-L_skp_2-16384\n\t" \
  "s_add_u32 s88, s88, s84\n\t" \
  "s_addc_u32 s89, s89, 0\n\t" \
  "s_mov_b64 exec, 0xff\n\t" \
  "global_load_dword v210, v234, s[88:89]\n\t" \
  "s_mov_b64 exec, -1\n\t" \
  "L_sk_2:\n\t" \
  "buffer_load_dwordx4 a[148:151], v189, s[36:39], s87 offen offset:1024 sc1\n\t" \
  "v_mfma_f32_16x16x128_f8f6f4 v[120:123], a[56:63], v[144:151], v[120:123] cbsz:0 blgp:1\n\t" \
  "ds_read_b64_tr_b8 v[134:135], v160 offset:41984\n\t" \
  "buffer_load_dwordx4 a[152:155], v189, s[36:39], s87 offen offset:2048 sc1\n\t" \
  "s_waitcnt lgkmcnt(4)\n\t" \
  "v_mfma_f32_16x16x128_f8f6f4 v[28:31], a[32:39], v[152:159], v[28:31] cbsz:0 blgp:1\n\t" \
  "ds_read_b64_tr_b8 v[136:137], v161 offset:32768\n\t" \
  "s_cmp_eq_u32 s29, 1\n\t" \
  "s_cbranch_scc0 L_fl1_2\n\t" \
  "s_waitcnt vmcnt(32)\n\t" \
  "s_mov_b64 exec, 1\n\t" \
  "buffer_store_dword v208, v209, s[56:59], 0 offen sc0 sc1\n\t" \
  "s_mov_b64 exec, -1\n\t" \
  "L_fl1_2:\n\t" \
  "buffer_load_dwordx4 a[156:159], v189, s[36:39], s87 offen offset:3072 sc1\n\t" \
  "v_mfma_f32_16x16x128_f8f6f4 v[60:63], a[40:47], v[152:159], v[60:63] cbsz:0 blgp:1\n\t" \
  "ds_read_b64_tr_b8 v[138:139], v161 offset:33792\n\t" \
  "v_mfma_f32_16x16x128_f8f6f4 v[92:95], a[48:55], v[152:159], v[92:95] cbsz:0 blgp:1\n\t" \
  "ds_read_b64_tr_b8 v[140:141], v161 offset:40960\n\t" \
  "v_mfma_f32_16x16x128_f8f6f4 v[124:127], a[56:63], v[152:159], v[124:127] cbsz:0 blgp:1\n\t" \
  "ds_read_b64_tr_b8 v[142:143], v161 offset:41984\n\t" \
  "s_waitcnt lgkmcnt(4)\n\t" \
  "v_mfma_f32_16x16x128_f8f6f4 v[0:3], a[64:71], v[128:135], v[0:3] cbsz:0 blgp:1\n\t" \
  "ds_read_b64_tr_b8 v[144:145], v162 offset:32768\n\t" \
  "v_mfma_f32_16x16x128_f8f6f4 v[32:35], a[72:79], v[128:135], v[32:35] cbsz:0 blgp:1\n\t" \
  "ds_read_b64_tr_b8 v[146:147], v162 offset:33792\n\t" \
  "v_mfma_f32_16x16x128_f8f6f4 v[64:67], a[80:87], v[128:135], v[64:67] cbsz:0 blgp:1\n\t" \
  "ds_read_b64_tr_b8 v[148:149], v162 offset:40960\n\t" \
  "v_mfma_f32_16x16x128_f8f6f4 v[96:99], a[88:95], v[128:135], v[96:99] cbsz:0 blgp:1\n\t" \
  "ds_read_b64_tr_b8 v[150:151], v162 offset:41984\n\t" \
  "s_waitcnt lgkmcnt(4)\n\t" \
  "v_mfma_f32_16x16x128_f8f6f4 v[4:7], a[64:71], v[136:143], v[4:7] cbsz:0 blgp:1\n\t" \
  "ds_read_b64_tr_b8 v[152:153], v163 offset:32768\n\t" \
  "v_mfma_f32_16x16x128_f8f6f4 v[36:39], a[72:79], v[136:143], v[36:39] cbsz:0 blgp:1\n\t" \
  "ds_read_b64_tr_b8 v[154:155], v163 offset:33792\n\t" \
  "v_mfma_f32_16x16x128_f8f6f4 v[68:71], a[80:87], v[136:143], v[68:71] cbsz:0 blgp:1\n\t" \
  "ds_read_b64_tr_b8 v[156:157], v163 offset:40960\n\t" \
  "v_mfma_f32_16x16x128_f8f6f4 v[100:103], a[88:95], v[136:143], v[100:103] cbsz:0 blgp:1\n\t" \
  "ds_read_b64_tr_b8 v[158:159], v163 offset:41984\n\t" \
  "s_waitcnt lgkmcnt(4)\n\t" \
  "v_mfma_f32_16x16x128_f8f6f4 v[8:11], a[64:71], v[144:151], v[8:11] cbsz:0 blgp:1\n\t" \
  "ds_read_b64_tr_b8 v[128:129], v164 offset:32768\n\t" \
  "s_waitcnt vmcnt(11)\n\t" \
  "v_accvgpr_read_b32 v229, a128\n\t" \
  "v_accvgpr_read_b32 v230, a129\n\t" \
  "v_mfma_f32_16x16x128_f8f6f4 v[40:43], a[72:79], v[144:151], v[40:43] cbsz:0 blgp:1\n\t" \
  "ds_read_b64_tr_b8 v[130:131], v164 offset:33792\n\t" \
  "v_accvgpr_read_b32 v231, a130\n\t" \
  "v_accvgpr_read_b32 v232, a131\n\t" \
  "v_mfma_f32_16x16x128_f8f6f4 v[72:75], a[80:87], v[144:151], v[72:75] cbsz:0 blgp:1\n\t" \
  "ds_read_b64_tr_b8 v[132:133], v164 offset:40960\n\t" \
  "v_min3_u32 v228, v229, v230, v229\n\t" \
  "v_min3_u32 v228, v231, v232, v228\n\t" \
  "v_mfma_f32_16x16x128_f8f6f4 v[104:107], a[88:95], v[144:151], v[104:107] cbsz:0 blgp:1\n\t" \
  "ds_read_b64_tr_b8 v[134:135], v164 offset:41984\n\t" \
  "s_waitcnt vmcnt(10)\n\t" \
  "v_accvgpr_read_b32 v229, a132\n\t" \
  "v_accvgpr_read_b32 v230, a133\n\t" \
  "s_waitcnt lgkmcnt(4)\n\t" \
  "v_mfma_f32_16x16x128_f8f6f4 v[12:15], a[64:71], v[152:159], v[12:15] cbsz:0 blgp:1\n\t" \
  "ds_read_b64_tr_b8 v[136:137], v165 offset:32768\n\t" \
  "v_accvgpr_read_b32 v231, a134\n\t" \
  "v_accvgpr_read_b32 v232, a135\n\t" \
  "v_mfma_f32_16x16x128_f8f6f4 v[44:47], a[72:79], v[152:159], v[44:47] cbsz:0 blgp:1\n\t" \
  "ds_read_b64_tr_b8 v[138:139], v165 offset:33792\n\t" \
  "v_min3_u32 v228, v229, v230, v228\n\t" \
  "v_min3_u32 v228, v231, v232, v228\n\t" \
  "v_mfma_f32_16x16x128_f8f6f4 v[76:79], a[80:87], v[152:159], v[76:79] cbsz:0 blgp:1\n\t" \
  "ds_read_b64_tr_b8 v[140:141], v165 offset:40960\n\t" \
  "s_waitcnt vmcnt(9)\n\t" \
  "v_accvgpr_read_b32 v229, a136\n\t" \
  "v_accvgpr_read_b32 v230, a137\n\t" \
  "v_mfma_f32_16x16x128_f8f6f4 v[108:111], a[88:95], v[152:159], v[108:111] cbsz:0 blgp:1\n\t" \
  "ds_read_b64_tr_b8 v[142:143], v165 offset:41984\n\t" \
  "v_accvgpr_read_b32 v231, a138\n\t" \
  "v_accvgpr_read_b32 v232, a139\n\t" \
  "s_waitcnt lgkmcnt(4)\n\t" \
  "v_mfma_f32_16x16x128_f8f6f4 v[16:19], a[64:71], v[128:135], v[16:19] cbsz:0 blgp:1\n\t" \
  "ds_read_b64_tr_b8 v[144:145], v166 offset:32768\n\t" \
  "v_min3_u32 v228, v229, v230, v228\n\t" \
  "v_min3_u32 v228, v231, v232, v228\n\t" \
  "v_mfma_f32_16x16x128_f8f6f4 v[48:51], a[72:79], v[128:135], v[48:51] cbsz:0 blgp:1\n\t" \
  "ds_read_b64_tr_b8 v[146:147], v166 offset:33792\n\t" \
  "s_waitcnt vmcnt(8)\n\t" \
  "v_accvgpr_read_b32 v229, a140\n\t" \
  "v_accvgpr_read_b32 v230, a141\n\t" \
  "s_add_u32 m0, s77, 65536\n\t" \
  "s_add_u32 s90, s68, 0\n\t" \
  "buffer_load_dwordx4 v186, s[40:43], s90 offen offset:0  lds\n\t" \
  "v_mfma_f32_16x16x128_f8f6f4 v[80:83], a[80:87], v[128:135], v[80:83] cbsz:0 blgp:1\n\t" \
  "ds_read_b64_tr_b8 v[148:149], v166 offset:40960\n\t" \
  "v_accvgpr_read_b32 v231, a142\n\t" \
  "v_accvgpr_read_b32 v232, a143\n\t" \
  "v_mfma_f32_16x16x128_f8f6f4 v[112:115], a[88:95], v[128:135], v[112:115] cbsz:0 blgp:1\n\t" \
  "ds_read_b64_tr_b8 v[150:151], v166 offset:41984\n\t" \
  "v_min3_u32 v228, v229, v230, v228\n\t" \
  "v_min3_u32 v228, v231, v232, v228\n\t" \
  "s_add_u32 m0, s77, 65536\n\t" \
  "s_add_u32 s90, s68, 0\n\t" \
  "buffer_load_dwordx4 v187, s[40:43], s90 offen offset:1024  lds\n\t" \
  "s_waitcnt lgkmcnt(4)\n\t" \
  "v_mfma_f32_16x16x128_f8f6f4 v[20:23], a[64:71], v[136:143], v[20:23] cbsz:0 blgp:1\n\t" \
  "ds_read_b64_tr_b8 v[152:153], v167 offset:32768\n\t" \
  "s_waitcnt vmcnt(9)\n\t" \
  "v_accvgpr_read_b32 v229, a144\n\t" \
  "v_accvgpr_read_b32 v230, a145\n\t" \
  "v_mfma_f32_16x16x128_f8f6f4 v[52:55], a[72:79], v[136:143], v[52:55] cbsz:0 blgp:1\n\t" \
  "ds_read_b64_tr_b8 v[154:155], v167 offset:33792\n\t" \
  "v_accvgpr_read_b32 v231, a146\n\t" \
  "v_accvgpr_read_b32 v232, a147\n\t" \
  "s_add_u32 m0, s77, 81920\n\t" \
  "s_add_u32 s90, s69, 0\n\t" \
  "buffer_load_dwordx4 v186, s[40:43], s90 offen offset:0  lds\n\t" \
  "v_mfma_f32_16x16x128_f8f6f4 v[84:87], a[80:87], v[136:143], v[84:87] cbsz:0 blgp:1\n\t" \
  "ds_read_b64_tr_b8 v[156:157], v167 offset:40960\n\t" \
  "v_min3_u32 v228, v229, v230, v228\n\t" \
  "v_min3_u32 v228, v231, v232, v228\n\t" \
  "v_mfma_f32_16x16x128_f8f6f4 v[116:119], a[88:95], v[136:143], v[116:119] cbsz:0 blgp:1\n\t" \
  "ds_read_b64_tr_b8 v[158:159], v167 offset:41984\n\t" \
  "s_waitcnt vmcnt(5)\n\t" \
  "v_accvgpr_read_b32 v229, a148\n\t" \
  "v_accvgpr_read_b32 v230, a149\n\t" \
  "s_add_u32 m0, s77, 81920\n\t" \
  "s_add_u32 s90, s69, 0\n\t" \
  "buffer_load_dwordx4 v187, s[40:43], s90 offen offset:1024  lds\n\t" \
  "s_waitcnt lgkmcnt(4)\n\t" \
  "v_mfma_f32_16x16x128_f8f6f4 v[24:27], a[64:71], v[144:151], v[24:27] cbsz:0 blgp:1\n\t" \
  "s_barrier\n\t" \
  "ds_read_b64_tr_b8 v[128:129], v160 offset:49152\n\t" \
  "v_accvgpr_read_b32 v231, a150\n\t" \
  "v_accvgpr_read_b32 v232, a151\n\t" \
  "v_mfma_f32_16x16x128_f8f6f4 v[56:59], a[72:79], v[144:151], v[56:59] cbsz:0 blgp:1\n\t" \
  "ds_read_b64_tr_b8 v[130:131], v160 offset:50176\n\t" \
  "v_min3_u32 v228, v229, v230, v228\n\t" \
  "v_min3_u32 v228, v231, v232, v228\n\t" \
  "v_mfma_f32_16x16x128_f8f6f4 v[88:91], a[80:87], v[144:151], v[88:91] cbsz:0 blgp:1\n\t" \
  "ds_read_b64_tr_b8 v[132:133], v160 offset:57344\n\t" \
  "s_waitcnt vmcnt(5)\n\t" \
  "v_accvgpr_read_b32 v229, a152\n\t" \
  "v_accvgpr_read_b32 v230, a153\n\t" \
  "v_mfma_f32_16x16x128_f8f6f4 v[120:123], a[88:95], v[144:151], v[120:123] cbsz:0 blgp:1\n\t" \
  "ds_read_b64_tr_b8 v[134:135], v160 offset:58368\n\t" \
  "v_accvgpr_read_b32 v231, a154\n\t" \
  "v_accvgpr_read_b32 v232, a155\n\t" \
  "s_waitcnt lgkmcnt(4)\n\t" \
  "v_mfma_f32_16x16x128_f8f6f4 v[28:31], a[64:71], v[152:159], v[28:31] cbsz:0 blgp:1\n\t" \
  "ds_read_b64_tr_b8 v[136:137], v161 offset:49152\n\t" \
  "v_min3_u32 v228, v229, v230, v228\n\t" \
  "v_min3_u32 v228, v231, v232, v228\n\t" \
  "s_add_u32 m0, s76, 98304\n\t" \
  "s_add_u32 s90, s70, 0\n\t" \
  "buffer_load_dwordx4 v184, s[40:43], s90 offen offset:0  lds\n\t" \
  "v_mfma_f32_16x16x128_f8f6f4 v[60:63], a[72:79], v[152:159], v[60:63] cbsz:0 blgp:1\n\t" \
  "ds_read_b64_tr_b8 v[138:139], v161 offset:50176\n\t" \
  "s_waitcnt vmcnt(5)\n\t" \
  "v_accvgpr_read_b32 v229, a156\n\t" \
  "v_accvgpr_read_b32 v230, a157\n\t" \
  "s_add_u32 m0, s76, 98304\n\t" \
  "s_add_u32 s90, s70, 0\n\t" \
  "buffer_load_dwordx4 v185, s[40:43], s90 offen offset:1024  lds\n\t" \
  "v_mfma_f32_16x16x128_f8f6f4 v[92:95], a[80:87], v[152:159], v[92:95] cbsz:0 blgp:1\n\t" \
  "ds_read_b64_tr_b8 v[140:141], v161 offset:57344\n\t" \
  "v_accvgpr_read_b32 v231, a158\n\t" \
  "v_accvgpr_read_b32 v232, a159\n\t" \
  "s_add_u32 m0, s76, 114688\n\t" \
  "s_add_u32 s90, s71, 0\n\t" \
  "buffer_load_dwordx4 v184, s[40:43], s90 offen offset:0  lds\n\t" \
  "v_mfma_f32_16x16x128_f8f6f4 v[124:127], a[88:95], v[152:159], v[124:127] cbsz:0 blgp:1\n\t" \
  "ds_read_b64_tr_b8 v[142:143], v161 offset:58368\n\t" \
  "v_min3_u32 v228, v229, v230, v228\n\t" \
  "v_min3_u32 v228, v231, v232, v228\n\t" \
  "s_add_u32 m0, s76, 114688\n\t" \
  "s_add_u32 s90, s71, 0\n\t" \
  "buffer_load_dwordx4 v185, s[40:43], s90 offen offset:1024  lds\n\t" \
  "s_waitcnt lgkmcnt(4)\n\t" \
  "v_mfma_f32_16x16x128_f8f6f4 v[0:3], a[96:103], v[128:135], v[0:3] cbsz:0 blgp:1\n\t" \
  "ds_read_b64_tr_b8 v[144:145], v162 offset:49152\n\t" \
  "s_add_u32 m0, s76, 131072\n\t" \
  "s_add_u32 s90, s72, 0\n\t" \
  "buffer_load_dwordx4 v184, s[40:43], s90 offen offset:0  lds\n\t" \
  "v_mfma_f32_16x16x128_f8f6f4 v[32:35], a[104:111], v[128:135], v[32:35] cbsz:0 blgp:1\n\t" \
  "ds_read_b64_tr_b8 v[146:147], v162 offset:50176\n\t" \
  "v_cmp_eq_u32 vcc, 0, v228\n\t" \
  "s_nop 4\n\t" \
  "s_cmp_lg_u64 vcc, 0\n\t" \
  "s_cbranch_scc1 L_v4_2\n\t" \
  "L_vr4_2:\n\t" \
  "s_add_u32 m0, s76, 131072\n\t" \
  "s_add_u32 s90, s72, 0\n\t" \
  "buffer_load_dwordx4 v185, s[40:43], s90 offen offset:1024  lds\n\t" \
  "v_mfma_f32_16x16x128_f8f6f4 v[64:67], a[112:119], v[128:135], v[64:67] cbsz:0 blgp:1\n\t" \
  "ds_read_b64_tr_b8 v[148:149], v162 offset:57344\n\t" \
  "s_add_u32 m0, s76, 147456\n\t" \
  "s_add_u32 s90, s73, 0\n\t" \
  "buffer_load_dwordx4 v184, s[40:43], s90 offen offset:0  lds\n\t" \
  "v_mfma_f32_16x16x128_f8f6f4 v[96:99], a[120:127], v[128:135], v[96:99] cbsz:0 blgp:1\n\t" \
  "ds_read_b64_tr_b8 v[150:151], v162 offset:58368\n\t" \
  "s_add_u32 m0, s76, 147456\n\t" \
  "s_add_u32 s90, s73, 0\n\t" \
  "buffer_load_dwordx4 v185, s[40:43], s90 offen offset:1024  lds\n\t" \
  "s_waitcnt lgkmcnt(4)\n\t" \
  "v_mfma_f32_16x16x128_f8f6f4 v[4:7], a[96:103], v[136:143], v[4:7] cbsz:0 blgp:1\n\t" \
  "ds_read_b64_tr_b8 v[152:153], v163 offset:49152\n\t" \
  "s_mov_b32 s87, 65536\n\t" \
  "buffer_load_dwordx4 a[160:163], v188, s[36:39], s87 offen offset:0 sc1\n\t" \
  "v_mfma_f32_16x16x128_f8f6f4 v[36:39], a[104:111], v[136:143], v[36:39] cbsz:0 blgp:1\n\t" \
  "ds_read_b64_tr_b8 v[154:155], v163 offset:50176\n\t" \
  "v_mfma_f32_16x16x128_f8f6f4 v[68:71], a[112:119], v[136:143], v[68:71] cbsz:0 blgp:1\n\t" \
  "ds_read_b64_tr_b8 v[156:157], v163 offset:57344\n\t" \
  "buffer_load_dwordx4 a[164:167], v188, s[36:39], s87 offen offset:1024 sc1\n\t" \
  "v_mfma_f32_16x16x128_f8f6f4 v[100:103], a[120:127], v[136:143], v[100:103] cbsz:0 blgp:1\n\t" \
  "ds_read_b64_tr_b8 v[158:159], v163 offset:58368\n\t" \
  "s_waitcnt lgkmcnt(4)\n\t" \
  "v_mfma_f32_16x16x128_f8f6f4 v[8:11], a[96:103], v[144:151], v[8:11] cbsz:0 blgp:1\n\t" \
  "ds_read_b64_tr_b8 v[128:129], v164 offset:49152\n\t" \
  "buffer_load_dwordx4 a[168:171], v188, s[36:39], s87 offen offset:2048 sc1\n\t" \
  "v_mfma_f32_16x16x128_f8f6f4 v[40:43], a[104:111], v[144:151], v[40:43] cbsz:0 blgp:1\n\t" \
  "ds_read_b64_tr_b8 v[130:131], v164 offset:50176\n\t" \
  "v_mfma_f32_16x16x128_f8f6f4 v[72:75], a[112:119], v[144:151], v[72:75] cbsz:0 blgp:1\n\t" \
  "ds_read_b64_tr_b8 v[132:133], v164 offset:57344\n\t" \
  "buffer_load_dwordx4 a[172:175], v188, s[36:39], s87 offen offset:3072 sc1\n\t" \
  "v_mfma_f32_16x16x128_f8f6f4 v[104:107], a[120:127], v[144:151], v[104:107] cbsz:0 blgp:1\n\t" \
  "ds_read_b64_tr_b8 v[134:135], v164 offset:58368\n\t" \
  "s_waitcnt lgkmcnt(4)\n\t" \
  "v_mfma_f32_16x16x128_f8f6f4 v[12:15], a[96:103], v[152:159], v[12:15] cbsz:0 blgp:1\n\t" \
  "ds_read_b64_tr_b8 v[136:137], v165 offset:49152\n\t" \
  "s_cmp_eq_u32 s29, 3\n\t" \
  "s_cbranch_scc0 L_ps3_2\n\t" \
  "buffer_store_dwordx4 v[200:203], v188, s[52:55], 0 offen offset:0 sc0 sc1\n\t" \
  "buffer_store_dwordx4 v[204:207], v188, s[52:55], 0 offen offset:1024 sc0 sc1\n\t" \
  "L_ps3_2:\n\t" \
  "buffer_load_dwordx4 a[176:179], v189, s[36:39], s87 offen offset:0 sc1\n\t" \
  "v_mfma_f32_16x16x128_f8f6f4 v[44:47], a[104:111], v[152:159], v[44:47] cbsz:0 blgp:1\n\t" \
  "ds_read_b64_tr_b8 v[138:139], v165 offset:50176\n\t" \
  "v_mfma_f32_16x16x128_f8f6f4 v[76:79], a[112:119], v[152:159], v[76:79] cbsz:0 blgp:1\n\t" \
  "ds_read_b64_tr_b8 v[140:141], v165 offset:57344\n\t" \
  "buffer_load_dwordx4 a[180:183], v189, s[36:39], s87 offen offset:1024 sc1\n\t" \
  "v_mfma_f32_16x16x128_f8f6f4 v[108:111], a[120:127], v[152:159], v[108:111] cbsz:0 blgp:1\n\t" \
  "ds_read_b64_tr_b8 v[142:143], v165 offset:58368\n\t" \
  "s_waitcnt lgkmcnt(4)\n\t" \
  "v_mfma_f32_16x16x128_f8f6f4 v[16:19], a[96:103], v[128:135], v[16:19] cbsz:0 blgp:1\n\t" \
  "ds_read_b64_tr_b8 v[144:145], v166 offset:49152\n\t" \
  "buffer_load_dwordx4 a[184:187], v189, s[36:39], s87 offen offset:2048 sc1\n\t" \
  "v_mfma_f32_16x16x128_f8f6f4 v[48:51], a[104:111], v[128:135], v[48:51] cbsz:0 blgp:1\n\t" \
  "ds_read_b64_tr_b8 v[146:147], v166 offset:50176\n\t" \
  "v_mfma_f32_16x16x128_f8f6f4 v[80:83], a[112:119], v[128:135], v[80:83] cbsz:0 blgp:1\n\t" \
  "ds_read_b64_tr_b8 v[148:149], v166 offset:57344\n\t" \
  "buffer_load_dwordx4 a[188:191], v189, s[36:39], s87 offen offset:3072 sc1\n\t" \
  "v_mfma_f32_16x16x128_f8f6f4 v[112:115], a[120:127], v[128:135], v[112:115] cbsz:0 blgp:1\n\t" \
  "ds_read_b64_tr_b8 v[150:151], v166 offset:58368\n\t" \
  "s_waitcnt lgkmcnt(4)\n\t" \
  "v_mfma_f32_16x16x128_f8f6f4 v[20:23], a[96:103], v[136:143], v[20:23] cbsz:0 blgp:1\n\t" \
  "ds_read_b64_tr_b8 v[152:153], v167 offset:49152\n\t" \
  "s_mov_b32 s87, 131072\n\t" \
  "buffer_load_dwordx4 a[192:195], v188, s[36:39], s87 offen offset:0 sc1\n\t" \
  "v_mfma_f32_16x16x128_f8f6f4 v[52:55], a[104:111], v[136:143], v[52:55] cbsz:0 blgp:1\n\t" \
  "ds_read_b64_tr_b8 v[154:155], v167 offset:50176\n\t" \
  "v_mfma_f32_16x16x128_f8f6f4 v[84:87], a[112:119], v[136:143], v[84:87] cbsz:0 blgp:1\n\t" \
  "ds_read_b64_tr_b8 v[156:157], v167 offset:57344\n\t" \
  "buffer_load_dwordx4 a[196:199], v188, s[36:39], s87 offen offset:1024 sc1\n\t" \
  "v_mfma_f32_16x16x128_f8f6f4 v[116:119], a[120:127], v[136:143], v[116:119] cbsz:0 blgp:1\n\t" \
  "ds_read_b64_tr_b8 v[158:159], v167 offset:58368\n\t" \
  "s_waitcnt lgkmcnt(4)\n\t" \
  "v_mfma_f32_16x16x128_f8f6f4 v[24:27], a[96:103], v[144:151], v[24:27] cbsz:0 blgp:1\n\t" \
  "s_waitcnt vmcnt(20)\n\t" \
  "s_barrier\n\t" \
  "ds_read_b64_tr_b8 v[128:129], v168 offset:0\n\t" \
  "v_mfma_f32_16x16x128_f8f6f4 v[56:59], a[104:111], v[144:151], v[56:59] cbsz:0 blgp:1\n\t" \
  "ds_read_b64_tr_b8 v[130:131], v168 offset:1024\n\t" \
  "buffer_load_dwordx4 a[200:203], v188, s[36:39], s87 offen offset:2048 sc1\n\t" \
  "v_mfma_f32_16x16x128_f8f6f4 v[88:91], a[112:119], v[144:151], v[88:91] cbsz:0 blgp:1\n\t" \
  "ds_read_b64_tr_b8 v[132:133], v168 offset:8192\n\t" \
  "s_cmp_eq_u32 s29, 2\n\t" \
  "s_cbranch_scc0 L_fl2_2\n\t" \
  "s_waitcnt vmcnt(26)\n\t" \
  "s_mov_b64 exec, 1\n\t" \
  "buffer_store_dword v208, v209, s[56:59], 0 offen sc0 sc1\n\t" \
  "s_mov_b64 exec, -1\n\t" \
  "L_fl2_2:\n\t" \
  "v_mfma_f32_16x16x128_f8f6f4 v[120:123], a[120:127], v[144:151], v[120:123] cbsz:0 blgp:1\n\t" \
  "ds_read_b64_tr_b8 v[134:135], v168 offset:9216\n\t" \
  "buffer_load_dwordx4 a[204:207], v188, s[36:39], s87 offen offset:3072 sc1\n\t" \
  "s_waitcnt lgkmcnt(4)\n\t" \
  "v_mfma_f32_16x16x128_f8f6f4 v[28:31], a[96:103], v[152:159], v[28:31] cbsz:0 blgp:1\n\t" \
  "ds_read_b64_tr_b8 v[136:137], v169 offset:0\n\t" \
  "v_mfma_f32_16x16x128_f8f6f4 v[60:63], a[104:111], v[152:159], v[60:63] cbsz:0 blgp:1\n\t" \
  "ds_read_b64_tr_b8 v[138:139], v169 offset:1024\n\t" \
  "s_waitcnt vmcnt(11)\n\t" \
  "v_accvgpr_read_b32 v229, a160\n\t" \
  "v_accvgpr_read_b32 v230, a161\n\t" \
  "buffer_load_dwordx4 a[208:211], v189, s[36:39], s87 offen offset:0 sc1\n\t" \
  "v_mfma_f32_16x16x128_f8f6f4 v[92:95], a[112:119], v[152:159], v[92:95] cbsz:0 blgp:1\n\t" \
  "ds_read_b64_tr_b8 v[140:141], v169 offset:8192\n\t" \
  "v_accvgpr_read_b32 v231, a162\n\t" \
  "v_accvgpr_read_b32 v232, a163\n\t" \
  "v_mfma_f32_16x16x128_f8f6f4 v[124:127], a[120:127], v[152:159], v[124:127] cbsz:0 blgp:1\n\t" \
  "ds_read_b64_tr_b8 v[142:143], v169 offset:9216\n\t" \
  "v_min3_u32 v228, v229, v230, v229\n\t" \
  "v_min3_u32 v228, v231, v232, v228\n\t" \
  "buffer_load_dwordx4 a[212:215], v189, s[36:39], s87 offen offset:1024 sc1\n\t" \
  "s_waitcnt lgkmcnt(4)\n\t" \
  "v_mfma_f32_16x16x128_f8f6f4 v[0:3], a[128:135], v[128:135], v[0:3] cbsz:0 blgp:1\n\t" \
  "ds_read_b64_tr_b8 v[144:145], v170 offset:0\n\t" \
  "s_waitcnt vmcnt(12)\n\t" \
  "v_accvgpr_read_b32 v229, a164\n\t" \
  "v_accvgpr_read_b32 v230, a165\n\t" \
  "v_mfma_f32_16x16x128_f8f6f4 v[32:35], a[136:143], v[128:135], v[32:35] cbsz:0 blgp:1\n\t" \
  "ds_read_b64_tr_b8 v[146:147], v170 offset:1024\n\t" \
  "v_accvgpr_read_b32 v231, a166\n\t" \
  "v_accvgpr_read_b32 v232, a167\n\t" \
  "buffer_load_dwordx4 a[216:219], v189, s[36:39], s87 offen offset:2048 sc1\n\t" \
  "v_mfma_f32_16x16x128_f8f6f4 v[64:67], a[144:151], v[128:135], v[64:67] cbsz:0 blgp:1\n\t" \
  "ds_read_b64_tr_b8 v[148:149], v170 offset:8192\n\t" \
  "v_min3_u32 v228, v229, v230, v228\n\t" \
  "v_min3_u32 v228, v231, v232, v228\n\t" \
  "v_mfma_f32_16x16x128_f8f6f4 v[96:99], a[152:159], v[128:135], v[96:99] cbsz:0 blgp:1\n\t" \
  "ds_read_b64_tr_b8 v[150:151], v170 offset:9216\n\t" \
  "s_waitcnt vmcnt(12)\n\t" \
  "v_accvgpr_read_b32 v229, a168\n\t" \
  "v_accvgpr_read_b32 v230, a169\n\t" \
  "buffer_load_dwordx4 a[220:223], v189, s[36:39], s87 offen offset:3072 sc1\n\t" \
  "s_waitcnt lgkmcnt(4)\n\t" \
  "v_mfma_f32_16x16x128_f8f6f4 v[4:7], a[128:135], v[136:143], v[4:7] cbsz:0 blgp:1\n\t" \
  "ds_read_b64_tr_b8 v[152:153], v171 offset:0\n\t" \
  "v_accvgpr_read_b32 v231, a170\n\t" \
  "v_accvgpr_read_b32 v232, a171\n\t" \
  "v_mfma_f32_16x16x128_f8f6f4 v[36:39], a[136:143], v[136:143], v[36:39] cbsz:0 blgp:1\n\t" \
  "ds_read_b64_tr_b8 v[154:155], v171 offset:1024\n\t" \
  "v_min3_u32 v228, v229, v230, v228\n\t" \
  "v_min3_u32 v228, v231, v232, v228\n\t" \
  "s_mov_b32 s87, 196608\n\t" \
  "buffer_load_dwordx4 a[224:227], v188, s[36:39], s87 offen offset:0 sc1\n\t" \
  "v_mfma_f32_16x16x128_f8f6f4 v[68:71], a[144:151], v[136:143], v[68:71] cbsz:0 blgp:1\n\t" \
  "ds_read_b64_tr_b8 v[156:157], v171 offset:8192\n\t" \
  "s_waitcnt vmcnt(13)\n\t" \
  "v_accvgpr_read_b32 v229, a172\n\t" \
  "v_accvgpr_read_b32 v230, a173\n\t" \
  "v_mfma_f32_16x16x128_f8f6f4 v[100:103], a[152:159], v[136:143], v[100:103] cbsz:0 blgp:1\n\t" \
  "ds_read_b64_tr_b8 v[158:159], v171 offset:9216\n\t" \
  "v_accvgpr_read_b32 v231, a174\n\t" \
  "v_accvgpr_read_b32 v232, a175\n\t" \
  "s_waitcnt lgkmcnt(4)\n\t" \
  "v_mfma_f32_16x16x128_f8f6f4 v[8:11], a[128:135], v[144:151], v[8:11] cbsz:0 blgp:1\n\t" \
  "ds_read_b64_tr_b8 v[128:129], v172 offset:0\n\t" \
  "v_min3_u32 v228, v229, v230, v228\n\t" \
  "v_min3_u32 v228, v231, v232, v228\n\t" \
  "buffer_load_dwordx4 a[228:231], v188, s[36:39], s87 offen offset:1024 sc1\n\t" \
  "v_mfma_f32_16x16x128_f8f6f4 v[40:43], a[136:143], v[144:151], v[40:43] cbsz:0 blgp:1\n\t" \
  "ds_read_b64_tr_b8 v[130:131], v172 offset:1024\n\t" \
  "s_waitcnt vmcnt(13)\n\t" \
  "v_accvgpr_read_b32 v229, a176\n\t" \
  "v_accvgpr_read_b32 v230, a177\n\t" \
  "v_mfma_f32_16x16x128_f8f6f4 v[72:75], a[144:151], v[144:151], v[72:75] cbsz:0 blgp:1\n\t" \
  "ds_read_b64_tr_b8 v[132:133], v172 offset:8192\n\t" \
  "v_accvgpr_read_b32 v231, a178\n\t" \
  "v_accvgpr_read_b32 v232, a179\n\t" \
  "v_mfma_f32_16x16x128_f8f6f4 v[104:107], a[152:159], v[144:151], v[104:107] cbsz:0 blgp:1\n\t" \
  "ds_read_b64_tr_b8 v[134:135], v172 offset:9216\n\t" \
  "v_min3_u32 v228, v229, v230, v228\n\t" \
  "v_min3_u32 v228, v231, v232, v228\n\t" \
  "buffer_load_dwordx4 a[232:235], v188, s[36:39], s87 offen offset:2048 sc1\n\t" \
  "s_waitcnt lgkmcnt(4)\n\t" \
  "v_mfma_f32_16x16x128_f8f6f4 v[12:15], a[128:135], v[152:159], v[12:15] cbsz:0 blgp:1\n\t" \
  "ds_read_b64_tr_b8 v[136:137], v173 offset:0\n\t" \
  "s_waitcnt vmcnt(13)\n\t" \
  "v_accvgpr_read_b32 v229, a180\n\t" \
  "v_accvgpr_read_b32 v230, a181\n\t" \
  "v_mfma_f32_16x16x128_f8f6f4 v[44:47], a[136:143], v[152:159], v[44:47] cbsz:0 blgp:1\n\t" \
  "ds_read_b64_tr_b8 v[138:139], v173 offset:1024\n\t" \
  "v_accvgpr_read_b32 v231, a182\n\t" \
  "v_accvgpr_read_b32 v232, a183\n\t" \
  "v_mfma_f32_16x16x128_f8f6f4 v[76:79], a[144:151], v[152:159], v[76:79] cbsz:0 blgp:1\n\t" \
  "ds_read_b64_tr_b8 v[140:141], v173 offset:8192\n\t" \
  "v_min3_u32 v228, v229, v230, v228\n\t" \
  "v_min3_u32 v228, v231, v232, v228\n\t" \
  "buffer_load_dwordx4 a[236:239], v188, s[36:39], s87 offen offset:3072 sc1\n\t" \
  "v_mfma_f32_16x16x128_f8f6f4 v[108:111], a[152:159], v[152:159], v[108:111] cbsz:0 blgp:1\n\t" \
  "ds_read_b64_tr_b8 v[142:143], v173 offset:9216\n\t" \
  "s_waitcnt vmcnt(13)\n\t" \
  "v_accvgpr_read_b32 v229, a184\n\t" \
  "v_accvgpr_read_b32 v230, a185\n\t" \
  "s_waitcnt lgkmcnt(4)\n\t" \
  "v_mfma_f32_16x16x128_f8f6f4 v[16:19], a[128:135], v[128:135], v[16:19] cbsz:0 blgp:1\n\t" \
  "ds_read_b64_tr_b8 v[144:145], v174 offset:0\n\t" \
  "v_accvgpr_read_b32 v231, a186\n\t" \
  "v_accvgpr_read_b32 v232, a187\n\t" \
  "s_add_u32 m0, s77, 98304\n\t" \
  "s_add_u32 s90, s70, 0\n\t" \
  "buffer_load_dwordx4 v186, s[40:43], s90 offen offset:0  lds\n\t" \
  "v_mfma_f32_16x16x128_f8f6f4 v[48:51], a[136:143], v[128:135], v[48:51] cbsz:0 blgp:1\n\t" \
  "ds_read_b64_tr_b8 v[146:147], v174 offset:1024\n\t" \
  "v_min3_u32 v228, v229, v230, v228\n\t" \
  "v_min3_u32 v228, v231, v232, v228\n\t" \
  "buffer_load_dwordx4 a[240:243], v189, s[36:39], s87 offen offset:0 sc1\n\t" \
  "v_mfma_f32_16x16x128_f8f6f4 v[80:83], a[144:151], v[128:135], v[80:83] cbsz:0 blgp:1\n\t" \
  "ds_read_b64_tr_b8 v[148:149], v174 offset:8192\n\t" \
  "s_waitcnt vmcnt(14)\n\t" \
  "v_accvgpr_read_b32 v229, a188\n\t" \
  "v_accvgpr_read_b32 v230, a189\n\t" \
  "s_add_u32 m0, s77, 98304\n\t" \
  "s_add_u32 s90, s70, 0\n\t" \
  "buffer_load_dwordx4 v187, s[40:43], s90 offen offset:1024  lds\n\t" \
  "v_mfma_f32_16x16x128_f8f6f4 v[112:115], a[152:159], v[128:135], v[112:115] cbsz:0 blgp:1\n\t" \
  "ds_read_b64_tr_b8 v[150:151], v174 offset:9216\n\t" \
  "v_accvgpr_read_b32 v231, a190\n\t" \
  "v_accvgpr_read_b32 v232, a191\n\t" \
  "s_add_u32 m0, s77, 114688\n\t" \
  "s_add_u32 s90, s71, 0\n\t" \
  "buffer_load_dwordx4 v186, s[40:43], s90 offen offset:0  lds\n\t" \
  "s_waitcnt lgkmcnt(4)\n\t" \
  "v_mfma_f32_16x16x128_f8f6f4 v[20:23], a[128:135], v[136:143], v[20:23] cbsz:0 blgp:1\n\t" \
  "ds_read_b64_tr_b8 v[152:153], v175 offset:0\n\t" \
  "v_min3_u32 v228, v229, v230, v228\n\t" \
  "v_min3_u32 v228, v231, v232, v228\n\t" \
  "s_add_u32 m0, s77, 131072\n\t" \
  "s_add_u32 s90, s72, 0\n\t" \
  "buffer_load_dwordx4 v186, s[40:43], s90 offen offset:0  lds\n\t" \
  "v_mfma_f32_16x16x128_f8f6f4 v[52:55], a[136:143], v[136:143], v[52:55] cbsz:0 blgp:1\n\t" \
  "ds_read_b64_tr_b8 v[154:155], v175 offset:1024\n\t" \
  "s_add_u32 m0, s77, 114688\n\t" \
  "s_add_u32 s90, s71, 0\n\t" \
  "buffer_load_dwordx4 v187, s[40:43], s90 offen offset:1024  lds\n\t" \
  "v_mfma_f32_16x16x128_f8f6f4 v[84:87], a[144:151], v[136:143], v[84:87] cbsz:0 blgp:1\n\t" \
  "ds_read_b64_tr_b8 v[156:157], v175 offset:8192\n\t" \
  "v_cmp_eq_u32 vcc, 0, v228\n\t" \
  "s_nop 4\n\t" \
  "s_cmp_lg_u64 vcc, 0\n\t" \
  "s_cbranch_scc1 L_v5_2\n\t" \
  "L_vr5_2:\n\t" \
  "s_add_u32 m0, s77, 131072\n\t" \
  "s_add_u32 s90, s72, 0\n\t" \
  "buffer_load_dwordx4 v187, s[40:43], s90 offen offset:1024  lds\n\t" \
  "v_mfma_f32_16x16x128_f8f6f4 v[116:119], a[152:159], v[136:143], v[116:119] cbsz:0 blgp:1\n\t" \
  "ds_read_b64_tr_b8 v[158:159], v175 offset:9216\n\t" \
  "buffer_load_dwordx4 a[244:247], v189, s[36:39], s87 offen offset:1024 sc1\n\t" \
  "s_waitcnt lgkmcnt(4)\n\t" \
  "v_mfma_f32_16x16x128_f8f6f4 v[24:27], a[128:135], v[144:151], v[24:27] cbsz:0 blgp:1\n\t" \
  "s_barrier\n\t" \
  "ds_read_b64_tr_b8 v[128:129], v168 offset:16384\n\t" \
  "v_mfma_f32_16x16x128_f8f6f4 v[56:59], a[136:143], v[144:151], v[56:59] cbsz:0 blgp:1\n\t" \
  "ds_read_b64_tr_b8 v[130:131], v168 offset:17408\n\t" \
  "v_mfma_f32_16x16x128_f8f6f4 v[88:91], a[144:151], v[144:151], v[88:91] cbsz:0 blgp:1\n\t" \
  "ds_read_b64_tr_b8 v[132:133], v168 offset:24576\n\t" \
  "buffer_load_dwordx4 a[248:251], v189, s[36:39], s87 offen offset:2048 sc1\n\t" \
  "v_mfma_f32_16x16x128_f8f6f4 v[120:123], a[152:159], v[144:151], v[120:123] cbsz:0 blgp:1\n\t" \
  "ds_read_b64_tr_b8 v[134:135], v168 offset:25600\n\t" \
  "s_waitcnt lgkmcnt(4)\n\t" \
  "v_mfma_f32_16x16x128_f8f6f4 v[28:31], a[128:135], v[152:159], v[28:31] cbsz:0 blgp:1\n\t" \
  "ds_read_b64_tr_b8 v[136:137], v169 offset:16384\n\t" \
  "buffer_load_dwordx4 a[252:255], v189, s[36:39], s87 offen offset:3072 sc1\n\t" \
  "v_mfma_f32_16x16x128_f8f6f4 v[60:63], a[136:143], v[152:159], v[60:63] cbsz:0 blgp:1\n\t" \
  "ds_read_b64_tr_b8 v[138:139], v169 offset:17408\n\t" \
  "s_waitcnt vmcnt(21)\n\t" \
  "v_accvgpr_read_b32 v229, a192\n\t" \
  "v_accvgpr_read_b32 v230, a193\n\t" \
  "s_mov_b32 s87, 262144\n\t" \
  "buffer_load_dwordx4 a[0:3], v188, s[36:39], s87 offen offset:0 sc1\n\t" \
  "v_mfma_f32_16x16x128_f8f6f4 v[92:95], a[144:151], v[152:159], v[92:95] cbsz:0 blgp:1\n\t" \
  "ds_read_b64_tr_b8 v[140:141], v169 offset:24576\n\t" \
  "v_accvgpr_read_b32 v231, a194\n\t" \
  "v_accvgpr_read_b32 v232, a195\n\t" \
  "v_mfma_f32_16x16x128_f8f6f4 v[124:127], a[152:159], v[152:159], v[124:127] cbsz:0 blgp:1\n\t" \
  "ds_read_b64_tr_b8 v[142:143], v169 offset:25600\n\t" \
  "v_min3_u32 v228, v229, v230, v229\n\t" \
  "v_min3_u32 v228, v231, v232, v228\n\t" \
  "s_waitcnt lgkmcnt(4)\n\t" \
  "v_mfma_f32_16x16x128_f8f6f4 v[0:3], a[160:167], v[128:135], v[0:3] cbsz:0 blgp:1\n\t" \
  "ds_read_b64_tr_b8 v[144:145], v170 offset:16384\n\t" \
  "s_waitcnt vmcnt(21)\n\t" \
  "v_accvgpr_read_b32 v229, a196\n\t" \
  "v_accvgpr_read_b32 v230, a197\n\t" \
  "buffer_load_dwordx4 a[4:7], v188, s[36:39], s87 offen offset:1024 sc1\n\t" \
  "v_mfma_f32_16x16x128_f8f6f4 v[32:35], a[168:175], v[128:135], v[32:35] cbsz:0 blgp:1\n\t" \
  "ds_read_b64_tr_b8 v[146:147], v170 offset:17408\n\t" \
  "v_accvgpr_read_b32 v231, a198\n\t" \
  "v_accvgpr_read_b32 v232, a199\n\t" \
  "v_mfma_f32_16x16x128_f8f6f4 v[64:67], a[176:183], v[128:135], v[64:67] cbsz:0 blgp:1\n\t" \
  "ds_read_b64_tr_b8 v[148:149], v170 offset:24576\n\t" \
  "v_min3_u32 v228, v229, v230, v228\n\t" \
  "v_min3_u32 v228, v231, v232, v228\n\t" \
  "v_mfma_f32_16x16x128_f8f6f4 v[96:99], a[184:191], v[128:135], v[96:99] cbsz:0 blgp:1\n\t" \
  "ds_read_b64_tr_b8 v[150:151], v170 offset:25600\n\t" \
  "s_waitcnt vmcnt(21)\n\t" \
  "v_accvgpr_read_b32 v229, a200\n\t" \
  "v_accvgpr_read_b32 v230, a201\n\t" \
  "buffer_load_dwordx4 a[8:11], v188, s[36:39], s87 offen offset:2048 sc1\n\t" \
  "s_waitcnt lgkmcnt(4)\n\t" \
  "v_mfma_f32_16x16x128_f8f6f4 v[4:7], a[160:167], v[136:143], v[4:7] cbsz:0 blgp:1\n\t" \
  "ds_read_b64_tr_b8 v[152:153], v171 offset:16384\n\t" \
  "v_accvgpr_read_b32 v231, a202\n\t" \
  "v_accvgpr_read_b32 v232, a203\n\t" \
  "s_add_u32 m0, s77, 147456\n\t" \
  "s_add_u32 s90, s73, 0\n\t" \
  "buffer_load_dwordx4 v186, s[40:43], s90 offen offset:0  lds\n\t" \
  "v_mfma_f32_16x16x128_f8f6f4 v[36:39], a[168:175], v[136:143], v[36:39] cbsz:0 blgp:1\n\t" \
  "ds_read_b64_tr_b8 v[154:155], v171 offset:17408\n\t" \
  "v_min3_u32 v228, v229, v230, v228\n\t" \
  "v_min3_u32 v228, v231, v232, v228\n\t" \
  "v_mfma_f32_16x16x128_f8f6f4 v[68:71], a[176:183], v[136:143], v[68:71] cbsz:0 blgp:1\n\t" \
  "ds_read_b64_tr_b8 v[156:157], v171 offset:24576\n\t" \
  "s_waitcnt vmcnt(22)\n\t" \
  "v_accvgpr_read_b32 v229, a204\n\t" \
  "v_accvgpr_read_b32 v230, a205\n\t" \
  "s_add_u32 m0, s77, 147456\n\t" \
  "s_add_u32 s90, s73, 0\n\t" \
  "buffer_load_dwordx4 v187, s[40:43], s90 offen offset:1024  lds\n\t" \
  "v_mfma_f32_16x16x128_f8f6f4 v[100:103], a[184:191], v[136:143], v[100:103] cbsz:0 blgp:1\n\t" \
  "ds_read_b64_tr_b8 v[158:159], v171 offset:25600\n\t" \
  "v_accvgpr_read_b32 v231, a206\n\t" \
  "v_accvgpr_read_b32 v232, a207\n\t" \
  "buffer_load_dwordx4 a[12:15], v188, s[36:39], s87 offen offset:3072 sc1\n\t" \
  "s_waitcnt lgkmcnt(4)\n\t" \
  "v_mfma_f32_16x16x128_f8f6f4 v[8:11], a[160:167], v[144:151], v[8:11] cbsz:0 blgp:1\n\t" \
  "ds_read_b64_tr_b8 v[128:129], v172 offset:16384\n\t" \
  "v_min3_u32 v228, v229, v230, v228\n\t" \
  "v_min3_u32 v228, v231, v232, v228\n\t" \
  "v_mfma_f32_16x16x128_f8f6f4 v[40:43], a[168:175], v[144:151], v[40:43] cbsz:0 blgp:1\n\t" \
  "ds_read_b64_tr_b8 v[130:131], v172 offset:17408\n\t" \
  "s_waitcnt vmcnt(23)\n\t" \
  "v_accvgpr_read_b32 v229, a208\n\t" \
  "v_accvgpr_read_b32 v230, a209\n\t" \
  "v_mfma_f32_16x16x128_f8f6f4 v[72:75], a[176:183], v[144:151], v[72:75] cbsz:0 blgp:1\n\t" \
  "ds_read_b64_tr_b8 v[132:133], v172 offset:24576\n\t" \
  "v_accvgpr_read_b32 v231, a210\n\t" \
  "v_accvgpr_read_b32 v232, a211\n\t" \
  "buffer_load_dwordx4 a[16:19], v189, s[36:39], s87 offen offset:0 sc1\n\t" \
  "v_mfma_f32_16x16x128_f8f6f4 v[104:107], a[184:191], v[144:151], v[104:107] cbsz:0 blgp:1\n\t" \
  "ds_read_b64_tr_b8 v[134:135], v172 offset:25600\n\t" \
  "v_min3_u32 v228, v229, v230, v228\n\t" \
  "v_min3_u32 v228, v231, v232, v228\n\t" \
  "s_waitcnt lgkmcnt(4)\n\t" \
  "v_mfma_f32_16x16x128_f8f6f4 v[12:15], a[160:167], v[152:159], v[12:15] cbsz:0 blgp:1\n\t" \
  "ds_read_b64_tr_b8 v[136:137], v173 offset:16384\n\t" \
  "s_waitcnt vmcnt(23)\n\t" \
  "v_accvgpr_read_b32 v229, a212\n\t" \
  "v_accvgpr_read_b32 v230, a213\n\t" \
  "s_cmp_eq_u32 s29, 3\n\t" \
  "s_cbranch_scc0 L_fl3_2\n\t" \
  "s_waitcnt vmcnt(33)\n\t" \
  "s_mov_b64 exec, 1\n\t" \
  "buffer_store_dword v208, v209, s[56:59], 0 offen sc0 sc1\n\t" \
  "s_mov_b64 exec, -1\n\t" \
  "L_fl3_2:\n\t" \
  "v_mfma_f32_16x16x128_f8f6f4 v[44:47], a[168:175], v[152:159], v[44:47] cbsz:0 blgp:1\n\t" \
  "ds_read_b64_tr_b8 v[138:139], v173 offset:17408\n\t" \
  "v_accvgpr_read_b32 v231, a214\n\t" \
  "v_accvgpr_read_b32 v232, a215\n\t" \
  "buffer_load_dwordx4 a[20:23], v189, s[36:39], s87 offen offset:1024 sc1\n\t" \
  "v_mfma_f32_16x16x128_f8f6f4 v[76:79], a[176:183], v[152:159], v[76:79] cbsz:0 blgp:1\n\t" \
  "ds_read_b64_tr_b8 v[140:141], v173 offset:24576\n\t" \
  "v_min3_u32 v228, v229, v230, v228\n\t" \
  "v_min3_u32 v228, v231, v232, v228\n\t" \
  "v_mfma_f32_16x16x128_f8f6f4 v[108:111], a[184:191], v[152:159], v[108:111] cbsz:0 blgp:1\n\t" \
  "ds_read_b64_tr_b8 v[142:143], v173 offset:25600\n\t" \
  "s_waitcnt vmcnt(23)\n\t" \
  "v_accvgpr_read_b32 v229, a216\n\t" \
  "v_accvgpr_read_b32 v230, a217\n\t" \
  "s_waitcnt lgkmcnt(4)\n\t" \
  "v_mfma_f32_16x16x128_f8f6f4 v[16:19], a[160:167], v[128:135], v[16:19] cbsz:0 blgp:1\n\t" \
  "ds_read_b64_tr_b8 v[144:145], v174 offset:16384\n\t" \
  "v_accvgpr_read_b32 v231, a218\n\t" \
  "v_accvgpr_read_b32 v232, a219\n\t" \
  "buffer_load_dwordx4 a[24:27], v189, s[36:39], s87 offen offset:2048 sc1\n\t" \
  "v_mfma_f32_16x16x128_f8f6f4 v[48:51], a[168:175], v[128:135], v[48:51] cbsz:0 blgp:1\n\t" \
  "ds_read_b64_tr_b8 v[146:147], v174 offset:17408\n\t" \
  "v_min3_u32 v228, v229, v230, v228\n\t" \
  "v_min3_u32 v228, v231, v232, v228\n\t" \
  "v_mfma_f32_16x16x128_f8f6f4 v[80:83], a[176:183], v[128:135], v[80:83] cbsz:0 blgp:1\n\t" \
  "ds_read_b64_tr_b8 v[148:149], v174 offset:24576\n\t" \
  "s_waitcnt vmcnt(23)\n\t" \
  "v_accvgpr_read_b32 v229, a220\n\t" \
  "v_accvgpr_read_b32 v230, a221\n\t" \
  "v_mfma_f32_16x16x128_f8f6f4 v[112:115], a[184:191], v[128:135], v[112:115] cbsz:0 blgp:1\n\t" \
  "ds_read_b64_tr_b8 v[150:151], v174 offset:25600\n\t" \
  "v_accvgpr_read_b32 v231, a222\n\t" \
  "v_accvgpr_read_b32 v232, a223\n\t" \
  "buffer_load_dwordx4 a[28:31], v189, s[36:39], s87 offen offset:3072 sc1\n\t" \
  "s_waitcnt lgkmcnt(4)\n\t" \
  "v_mfma_f32_16x16x128_f8f6f4 v[20:23], a[160:167], v[136:143], v[20:23] cbsz:0 blgp:1\n\t" \
  "ds_read_b64_tr_b8 v[152:153], v175 offset:16384\n\t" \
  "v_min3_u32 v228, v229, v230, v228\n\t" \
  "v_min3_u32 v228, v231, v232, v228\n\t" \
  "s_mov_b32 s87, 327680\n\t" \
  "buffer_load_dwordx4 a[32:35], v188, s[36:39], s87 offen offset:0 sc1\n\t" \
  "v_mfma_f32_16x16x128_f8f6f4 v[52:55], a[168:175], v[136:143], v[52:55] cbsz:0 blgp:1\n\t" \
  "ds_read_b64_tr_b8 v[154:155], v175 offset:17408\n\t" \
  "v_mfma_f32_16x16x128_f8f6f4 v[84:87], a[176:183], v[136:143], v[84:87] cbsz:0 blgp:1\n\t" \
  "ds_read_b64_tr_b8 v[156:157], v175 offset:24576\n\t" \
  "v_cmp_eq_u32 vcc, 0, v228\n\t" \
  "s_nop 4\n\t" \
  "s_cmp_lg_u64 vcc, 0\n\t" \
  "s_cbranch_scc1 L_v6_2\n\t" \
  "L_vr6_2:\n\t" \
  "v_mfma_f32_16x16x128_f8f6f4 v[116:119], a[184:191], v[136:143], v[116:119] cbsz:0 blgp:1\n\t" \
  "ds_read_b64_tr_b8 v[158:159], v175 offset:25600\n\t" \
  "buffer_load_dwordx4 a[36:39], v188, s[36:39], s87 offen offset:1024 sc1\n\t" \
  "s_waitcnt lgkmcnt(4)\n\t" \
  "v_mfma_f32_16x16x128_f8f6f4 v[24:27], a[160:167], v[144:151], v[24:27] cbsz:0 blgp:1\n\t" \
  "s_waitcnt vmcnt(19)\n\t" \
  "s_barrier\n\t" \
  "ds_read_b64_tr_b8 v[128:129], v168 offset:32768\n\t" \
  "v_mfma_f32_16x16x128_f8f6f4 v[56:59], a[168:175], v[144:151], v[56:59] cbsz:0 blgp:1\n\t" \
  "ds_read_b64_tr_b8 v[130:131], v168 offset:33792\n\t" \
  "v_mfma_f32_16x16x128_f8f6f4 v[88:91], a[176:183], v[144:151], v[88:91] cbsz:0 blgp:1\n\t" \
  "ds_read_b64_tr_b8 v[132:133], v168 offset:40960\n\t" \
  "buffer_load_dwordx4 a[40:43], v188, s[36:39], s87 offen offset:2048 sc1\n\t" \
  "v_mfma_f32_16x16x128_f8f6f4 v[120:123], a[184:191], v[144:151], v[120:123] cbsz:0 blgp:1\n\t" \
  "ds_read_b64_tr_b8 v[134:135], v168 offset:41984\n\t" \
  "s_waitcnt lgkmcnt(4)\n\t" \
  "v_mfma_f32_16x16x128_f8f6f4 v[28:31], a[160:167], v[152:159], v[28:31] cbsz:0 blgp:1\n\t" \
  "ds_read_b64_tr_b8 v[136:137], v169 offset:32768\n\t" \
  "v_mfma_f32_16x16x128_f8f6f4 v[60:63], a[168:175], v[152:159], v[60:63] cbsz:0 blgp:1\n\t" \
  "ds_read_b64_tr_b8 v[138:139], v169 offset:33792\n\t" \
  "v_accvgpr_read_b32 v229, a224\n\t" \
  "v_accvgpr_read_b32 v230, a225\n\t" \
  "buffer_load_dwordx4 a[44:47], v188, s[36:39], s87 offen offset:3072 sc1\n\t" \
  "v_mfma_f32_16x16x128_f8f6f4 v[92:95], a[176:183], v[152:159], v[92:95] cbsz:0 blgp:1\n\t" \
  "ds_read_b64_tr_b8 v[140:141], v169 offset:40960\n\t" \
  "v_accvgpr_read_b32 v231, a226\n\t" \
  "v_accvgpr_read_b32 v232, a227\n\t" \
  "v_mfma_f32_16x16x128_f8f6f4 v[124:127], a[184:191], v[152:159], v[124:127] cbsz:0 blgp:1\n\t" \
  "ds_read_b64_tr_b8 v[142:143], v169 offset:41984\n\t" \
  "v_min3_u32 v228, v229, v230, v229\n\t" \
  "v_min3_u32 v228, v231, v232, v228\n\t" \
  "s_waitcnt lgkmcnt(4)\n\t" \
  "v_mfma_f32_16x16x128_f8f6f4 v[0:3], a[192:199], v[128:135], v[0:3] cbsz:0 blgp:1\n\t" \
  "ds_read_b64_tr_b8 v[144:145], v170 offset:32768\n\t" \
  "v_accvgpr_read_b32 v229, a228\n\t" \
  "v_accvgpr_read_b32 v230, a229\n\t" \
  "buffer_load_dwordx4 a[48:51], v189, s[36:39], s87 offen offset:0 sc1\n\t" \
  "v_mfma_f32_16x16x128_f8f6f4 v[32:35], a[200:207], v[128:135], v[32:35] cbsz:0 blgp:1\n\t" \
  "ds_read_b64_tr_b8 v[146:147], v170 offset:33792\n\t" \
  "v_accvgpr_read_b32 v231, a230\n\t" \
  "v_accvgpr_read_b32 v232, a231\n\t" \
  "v_mfma_f32_16x16x128_f8f6f4 v[64:67], a[208:215], v[128:135], v[64:67] cbsz:0 blgp:1\n\t" \
  "ds_read_b64_tr_b8 v[148:149], v170 offset:40960\n\t" \
  "v_min3_u32 v228, v229, v230, v228\n\t" \
  "v_min3_u32 v228, v231, v232, v228\n\t" \
  "v_mfma_f32_16x16x128_f8f6f4 v[96:99], a[216:223], v[128:135], v[96:99] cbsz:0 blgp:1\n\t" \
  "ds_read_b64_tr_b8 v[150:151], v170 offset:41984\n\t" \
  "v_accvgpr_read_b32 v229, a232\n\t" \
  "v_accvgpr_read_b32 v230, a233\n\t" \
  "buffer_load_dwordx4 a[52:55], v189, s[36:39], s87 offen offset:1024 sc1\n\t" \
  "s_waitcnt lgkmcnt(4)\n\t" \
  "v_mfma_f32_16x16x128_f8f6f4 v[4:7], a[192:199], v[136:143], v[4:7] cbsz:0 blgp:1\n\t" \
  "ds_read_b64_tr_b8 v[152:153], v171 offset:32768\n\t" \
  "v_accvgpr_read_b32 v231, a234\n\t" \
  "v_accvgpr_read_b32 v232, a235\n\t" \
  "v_mfma_f32_16x16x128_f8f6f4 v[36:39], a[200:207], v[136:143], v[36:39] cbsz:0 blgp:1\n\t" \
  "ds_read_b64_tr_b8 v[154:155], v171 offset:33792\n\t" \
  "v_min3_u32 v228, v229, v230, v228\n\t" \
  "v_min3_u32 v228, v231, v232, v228\n\t" \
  "v_mfma_f32_16x16x128_f8f6f4 v[68:71], a[208:215], v[136:143], v[68:71] cbsz:0 blgp:1\n\t" \
  "ds_read_b64_tr_b8 v[156:157], v171 offset:40960\n\t" \
  "v_accvgpr_read_b32 v229, a236\n\t" \
  "v_accvgpr_read_b32 v230, a237\n\t" \
  "buffer_load_dwordx4 a[56:59], v189, s[36:39], s87 offen offset:2048 sc1\n\t" \
  "v_mfma_f32_16x16x128_f8f6f4 v[100:103], a[216:223], v[136:143], v[100:103] cbsz:0 blgp:1\n\t" \
  "ds_read_b64_tr_b8 v[158:159], v171 offset:41984\n\t" \
  "v_accvgpr_read_b32 v231, a238\n\t" \
  "v_accvgpr_read_b32 v232, a239\n\t" \
  "s_waitcnt lgkmcnt(4)\n\t" \
  "v_mfma_f32_16x16x128_f8f6f4 v[8:11], a[192:199], v[144:151], v[8:11] cbsz:0 blgp:1\n\t" \
  "ds_read_b64_tr_b8 v[128:129], v172 offset:32768\n\t" \
  "v_min3_u32 v228, v229, v230, v228\n\t" \
  "v_min3_u32 v228, v231, v232, v228\n\t" \
  "s_add_u32 m0, s76, 0\n\t" \
  "s_add_u32 s90, s74, 0\n\t" \
  "buffer_load_dwordx4 v184, s[40:43], s90 offen offset:0  lds\n\t" \
  "v_mfma_f32_16x16x128_f8f6f4 v[40:43], a[200:207], v[144:151], v[40:43] cbsz:0 blgp:1\n\t" \
  "ds_read_b64_tr_b8 v[130:131], v172 offset:33792\n\t" \
  "v_accvgpr_read_b32 v229, a240\n\t" \
  "v_accvgpr_read_b32 v230, a241\n\t" \
  "s_add_u32 m0, s76, 0\n\t" \
  "s_add_u32 s90, s74, 0\n\t" \
  "buffer_load_dwordx4 v185, s[40:43], s90 offen offset:1024  lds\n\t" \
  "v_mfma_f32_16x16x128_f8f6f4 v[72:75], a[208:215], v[144:151], v[72:75] cbsz:0 blgp:1\n\t" \
  "ds_read_b64_tr_b8 v[132:133], v172 offset:40960\n\t" \
  "v_accvgpr_read_b32 v231, a242\n\t" \
  "v_accvgpr_read_b32 v232, a243\n\t" \
  "s_cmp_gt_u32 s29, 1\n\t" \
  "s_cbranch_scc1 L_k202_2\n\t" \
  "s_getpc_b64 s[88:89]\n\t" \
  "s_lshl_b32 s84, s29, 13\n\t" \
  "s_add_u32 s84, s84, 2048\n\t" \
  "s_add_u32 s88, s88, s84\n\t" \
  "s_addc_u32 s89, s89, 0\n\t" \
  "s_mov_b64 exec, 0xff\n\t" \
  "global_load_dword v210, v234, s[88:89]\n\t" \
  "s_mov_b64 exec, -1\n\t" \
  "L_k202_2:\n\t" \
  "s_add_u32 m0, s76, 16384\n\t" \
  "s_add_u32 s90, s68, 2097152\n\t" \
  "buffer_load_dwordx4 v184, s[40:43], s90 offen offset:0  lds\n\t" \
  "v_mfma_f32_16x16x128_f8f6f4 v[104:107], a[216:223], v[144:151], v[104:107] cbsz:0 blgp:1\n\t" \
  "ds_read_b64_tr_b8 v[134:135], v172 offset:41984\n\t" \
  "v_min3_u32 v228, v229, v230, v228\n\t" \
  "v_min3_u32 v228, v231, v232, v228\n\t" \
  "s_add_u32 m0, s76, 16384\n\t" \
  "s_add_u32 s90, s68, 2097152\n\t" \
  "buffer_load_dwordx4 v185, s[40:43], s90 offen offset:1024  lds\n\t" \
  "s_waitcnt lgkmcnt(4)\n\t" \
  "v_mfma_f32_16x16x128_f8f6f4 v[12:15], a[192:199], v[152:159], v[12:15] cbsz:0 blgp:1\n\t" \
  "ds_read_b64_tr_b8 v[136:137], v173 offset:32768\n\t" \
  "s_waitcnt vmcnt(23)\n\t" \
  "v_accvgpr_read_b32 v229, a244\n\t" \
  "v_accvgpr_read_b32 v230, a245\n\t" \
  "s_add_u32 m0, s76, 32768\n\t" \
  "s_add_u32 s90, s69, 2097152\n\t" \
  "buffer_load_dwordx4 v184, s[40:43], s90 offen offset:0  lds\n\t" \
  "v_mfma_f32_16x16x128_f8f6f4 v[44:47], a[200:207], v[152:159], v[44:47] cbsz:0 blgp:1\n\t" \
  "ds_read_b64_tr_b8 v[138:139], v173 offset:33792\n\t" \
  "v_accvgpr_read_b32 v231, a246\n\t" \
  "v_accvgpr_read_b32 v232, a247\n\t" \
  "s_add_u32 m0, s76, 32768\n\t" \
  "s_add_u32 s90, s69, 2097152\n\t" \
  "buffer_load_dwordx4 v185, s[40:43], s90 offen offset:1024  lds\n\t" \
  "v_mfma_f32_16x16x128_f8f6f4 v[76:79], a[208:215], v[152:159], v[76:79] cbsz:0 blgp:1\n\t" \
  "ds_read_b64_tr_b8 v[140:141], v173 offset:40960\n\t" \
  "v_min3_u32 v228, v229, v230, v228\n\t" \
  "v_min3_u32 v228, v231, v232, v228\n\t" \
  "s_add_u32 m0, s76, 49152\n\t" \
  "s_add_u32 s90, s70, 2097152\n\t" \
  "buffer_load_dwordx4 v184, s[40:43], s90 offen offset:0  lds\n\t" \
  "v_mfma_f32_16x16x128_f8f6f4 v[108:111], a[216:223], v[152:159], v[108:111] cbsz:0 blgp:1\n\t" \
  "ds_read_b64_tr_b8 v[142:143], v173 offset:41984\n\t" \
  "s_waitcnt vmcnt(25)\n\t" \
  "v_accvgpr_read_b32 v229, a248\n\t" \
  "v_accvgpr_read_b32 v230, a249\n\t" \
  "s_add_u32 m0, s76, 49152\n\t" \
  "s_add_u32 s90, s70, 2097152\n\t" \
  "buffer_load_dwordx4 v185, s[40:43], s90 offen offset:1024  lds\n\t" \
  "s_waitcnt lgkmcnt(4)\n\t" \
  "v_mfma_f32_16x16x128_f8f6f4 v[16:19], a[192:199], v[128:135], v[16:19] cbsz:0 blgp:1\n\t" \
  "ds_read_b64_tr_b8 v[144:145], v174 offset:32768\n\t" \
  "v_accvgpr_read_b32 v231, a250\n\t" \
  "v_accvgpr_read_b32 v232, a251\n\t" \
  "s_add_u32 s99, s29, 16\n\t" \
  "s_min_u32 s99, s99, 31\n\t" \
  "v_readlane_b32 s99, v211, s99\n\t" \
  "s_mov_b32 exec_lo, s15\n\t" \
  "s_mov_b32 exec_hi, 0\n\t" \
  "s_cbranch_execz L_c16_2\n\t" \
  "s_nop 1\n\t" \
  "buffer_load_dword v210, v222, s[60:63], s99 offen\n\t" \
  "L_c16_2:\n\t" \
  "s_mov_b64 exec, -1\n\t" \
  "v_mfma_f32_16x16x128_f8f6f4 v[48:51], a[200:207], v[128:135], v[48:51] cbsz:0 blgp:1\n\t" \
  "ds_read_b64_tr_b8 v[146:147], v174 offset:33792\n\t" \
  "v_min3_u32 v228, v229, v230, v228\n\t" \
  "v_min3_u32 v228, v231, v232, v228\n\t" \
  "buffer_load_dwordx4 a[60:63], v189, s[36:39], s87 offen offset:3072 sc1\n\t" \
  "v_mfma_f32_16x16x128_f8f6f4 v[80:83], a[208:215], v[128:135], v[80:83] cbsz:0 blgp:1\n\t" \
  "ds_read_b64_tr_b8 v[148:149], v174 offset:40960\n\t" \
  "s_waitcnt vmcnt(26)\n\t" \
  "v_accvgpr_read_b32 v229, a252\n\t" \
  "v_accvgpr_read_b32 v230, a253\n\t" \
  "v_mfma_f32_16x16x128_f8f6f4 v[112:115], a[216:223], v[128:135], v[112:115] cbsz:0 blgp:1\n\t" \
  "ds_read_b64_tr_b8 v[150:151], v174 offset:41984\n\t" \
  "v_accvgpr_read_b32 v231, a254\n\t" \
  "v_accvgpr_read_b32 v232, a255\n\t" \
  "s_waitcnt lgkmcnt(4)\n\t" \
  "v_mfma_f32_16x16x128_f8f6f4 v[20:23], a[192:199], v[136:143], v[20:23] cbsz:0 blgp:1\n\t" \
  "ds_read_b64_tr_b8 v[152:153], v175 offset:32768\n\t" \
  "v_min3_u32 v228, v229, v230, v228\n\t" \
  "v_min3_u32 v228, v231, v232, v228\n\t" \
  "v_mfma_f32_16x16x128_f8f6f4 v[52:55], a[200:207], v[136:143], v[52:55] cbsz:0 blgp:1\n\t" \
  "ds_read_b64_tr_b8 v[154:155], v175 offset:33792\n\t" \
  "s_mov_b32 s64, 0\n\t" \
  "v_mfma_f32_16x16x128_f8f6f4 v[84:87], a[208:215], v[136:143], v[84:87] cbsz:0 blgp:1\n\t" \
  "ds_read_b64_tr_b8 v[156:157], v175 offset:40960\n\t" \
  "v_cmp_eq_u32 vcc, 0, v228\n\t" \
  "s_nop 4\n\t" \
  "s_cmp_lg_u64 vcc, 0\n\t" \
  "s_cbranch_scc1 L_v7_2\n\t" \
  "L_vr7_2:\n\t" \
  "global_load_dword v196, v195, s[26:27] offset:0 sc1\n\t" \
  "v_mfma_f32_16x16x128_f8f6f4 v[116:119], a[216:223], v[136:143], v[116:119] cbsz:0 blgp:1\n\t" \
  "ds_read_b64_tr_b8 v[158:159], v175 offset:41984\n\t" \
  "s_waitcnt lgkmcnt(4)\n\t" \
  "v_mfma_f32_16x16x128_f8f6f4 v[24:27], a[192:199], v[144:151], v[24:27] cbsz:0 blgp:1\n\t" \
  "s_barrier\n\t" \
  "ds_read_b64_tr_b8 v[128:129], v168 offset:49152\n\t" \
  "v_mfma_f32_16x16x128_f8f6f4 v[56:59], a[200:207], v[144:151], v[56:59] cbsz:0 blgp:1\n\t" \
  "ds_read_b64_tr_b8 v[130:131], v168 offset:50176\n\t" \
  "v_mfma_f32_16x16x128_f8f6f4 v[88:91], a[208:215], v[144:151], v[88:91] cbsz:0 blgp:1\n\t" \
  "ds_read_b64_tr_b8 v[132:133], v168 offset:57344\n\t" \
  "v_mfma_f32_16x16x128_f8f6f4 v[120:123], a[216:223], v[144:151], v[120:123] cbsz:0 blgp:1\n\t" \
  "ds_read_b64_tr_b8 v[134:135], v168 offset:58368\n\t" \
  "s_waitcnt lgkmcnt(4)\n\t" \
  "v_mfma_f32_16x16x128_f8f6f4 v[28:31], a[192:199], v[152:159], v[28:31] cbsz:0 blgp:1\n\t" \
  "ds_read_b64_tr_b8 v[136:137], v169 offset:49152\n\t" \
  "v_mfma_f32_16x16x128_f8f6f4 v[60:63], a[200:207], v[152:159], v[60:63] cbsz:0 blgp:1\n\t" \
  "ds_read_b64_tr_b8 v[138:139], v169 offset:50176\n\t" \
  "s_waitcnt vmcnt(26)\n\t" \
  "v_accvgpr_read_b32 v229, a0\n\t" \
  "v_accvgpr_read_b32 v230, a1\n\t" \
  "v_mfma_f32_16x16x128_f8f6f4 v[92:95], a[208:215], v[152:159], v[92:95] cbsz:0 blgp:1\n\t" \
  "ds_read_b64_tr_b8 v[140:141], v169 offset:57344\n\t" \
  "v_accvgpr_read_b32 v231, a2\n\t" \
  "v_accvgpr_read_b32 v232, a3\n\t" \
  "v_mfma_f32_16x16x128_f8f6f4 v[124:127], a[216:223], v[152:159], v[124:127] cbsz:0 blgp:1\n\t" \
  "ds_read_b64_tr_b8 v[142:143], v169 offset:58368\n\t" \
  "v_min3_u32 v228, v229, v230, v229\n\t" \
  "v_min3_u32 v228, v231, v232, v228\n\t" \
  "s_waitcnt lgkmcnt(4)\n\t" \
  "v_mfma_f32_16x16x128_f8f6f4 v[0:3], a[224:231], v[128:135], v[0:3] cbsz:0 blgp:1\n\t" \
  "ds_read_b64_tr_b8 v[144:145], v170 offset:49152\n\t" \
  "s_waitcnt vmcnt(25)\n\t" \
  "v_accvgpr_read_b32 v229, a4\n\t" \
  "v_accvgpr_read_b32 v230, a5\n\t" \
  "v_mfma_f32_16x16x128_f8f6f4 v[32:35], a[232:239], v[128:135], v[32:35] cbsz:0 blgp:1\n\t" \
  "ds_read_b64_tr_b8 v[146:147], v170 offset:50176\n\t" \
  "v_accvgpr_read_b32 v231, a6\n\t" \
  "v_accvgpr_read_b32 v232, a7\n\t" \
  "v_mfma_f32_16x16x128_f8f6f4 v[64:67], a[240:247], v[128:135], v[64:67] cbsz:0 blgp:1\n\t" \
  "ds_read_b64_tr_b8 v[148:149], v170 offset:57344\n\t" \
  "v_min3_u32 v228, v229, v230, v228\n\t" \
  "v_min3_u32 v228, v231, v232, v228\n\t" \
  "v_mfma_f32_16x16x128_f8f6f4 v[96:99], a[248:255], v[128:135], v[96:99] cbsz:0 blgp:1\n\t" \
  "ds_read_b64_tr_b8 v[150:151], v170 offset:58368\n\t" \
  "s_waitcnt vmcnt(24)\n\t" \
  "v_accvgpr_read_b32 v229, a8\n\t" \
  "v_accvgpr_read_b32 v230, a9\n\t" \
  "s_waitcnt lgkmcnt(4)\n\t" \
  "v_mfma_f32_16x16x128_f8f6f4 v[4:7], a[224:231], v[136:143], v[4:7] cbsz:0 blgp:1\n\t" \
  "ds_read_b64_tr_b8 v[152:153], v171 offset:49152\n\t" \
  "v_accvgpr_read_b32 v231, a10\n\t" \
  "v_accvgpr_read_b32 v232, a11\n\t" \
  "v_mfma_f32_16x16x128_f8f6f4 v[36:39], a[232:239], v[136:143], v[36:39] cbsz:0 blgp:1\n\t" \
  "ds_read_b64_tr_b8 v[154:155], v171 offset:50176\n\t" \
  "v_min3_u32 v228, v229, v230, v228\n\t" \
  "v_min3_u32 v228, v231, v232, v228\n\t" \
  "v_mfma_f32_16x16x128_f8f6f4 v[68:71], a[240:247], v[136:143], v[68:71] cbsz:0 blgp:1\n\t" \
  "ds_read_b64_tr_b8 v[156:157], v171 offset:57344\n\t" \
  "s_waitcnt vmcnt(21)\n\t" \
  "v_accvgpr_read_b32 v229, a12\n\t" \
  "v_accvgpr_read_b32 v230, a13\n\t" \
  "v_mfma_f32_16x16x128_f8f6f4 v[100:103], a[248:255], v[136:143], v[100:103] cbsz:0 blgp:1\n\t" \
  "ds_read_b64_tr_b8 v[158:159], v171 offset:58368\n\t" \
  "v_accvgpr_read_b32 v231, a14\n\t" \
  "v_accvgpr_read_b32 v232, a15\n\t" \
  "s_waitcnt lgkmcnt(4)\n\t" \
  "v_mfma_f32_16x16x128_f8f6f4 v[8:11], a[224:231], v[144:151], v[8:11] cbsz:0 blgp:1\n\t" \
  "ds_read_b64_tr_b8 v[128:129], v172 offset:49152\n\t" \
  "v_min3_u32 v228, v229, v230, v228\n\t" \
  "v_min3_u32 v228, v231, v232, v228\n\t" \
  "v_mfma_f32_16x16x128_f8f6f4 v[40:43], a[232:239], v[144:151], v[40:43] cbsz:0 blgp:1\n\t" \
  "ds_read_b64_tr_b8 v[130:131], v172 offset:50176\n\t" \
  "s_waitcnt vmcnt(20)\n\t" \
  "v_accvgpr_read_b32 v229, a16\n\t" \
  "v_accvgpr_read_b32 v230, a17\n\t" \
  "v_mfma_f32_16x16x128_f8f6f4 v[72:75], a[240:247], v[144:151], v[72:75] cbsz:0 blgp:1\n\t" \
  "ds_read_b64_tr_b8 v[132:133], v172 offset:57344\n\t" \
  "v_accvgpr_read_b32 v231, a18\n\t" \
  "v_accvgpr_read_b32 v232, a19\n\t" \
  "s_waitcnt vmcnt(0)\n\t" \
  "v_cmp_le_u32 vcc, s79, v196\n\t" \
  "s_nop 4\n\t" \
  "s_andn2_b32 s85, vcc_lo, s64\n\t" \
  "s_or_b32 s64, s64, vcc_lo\n\t" \
  "v_mfma_f32_16x16x128_f8f6f4 v[104:107], a[248:255], v[144:151], v[104:107] cbsz:0 blgp:1\n\t" \
  "ds_read_b64_tr_b8 v[134:135], v172 offset:58368\n\t" \
  "v_min3_u32 v228, v229, v230, v228\n\t" \
  "v_min3_u32 v228, v231, v232, v228\n\t" \
  "s_mov_b32 s96, 0\n\t" \
  "s_waitcnt lgkmcnt(4)\n\t" \
  "v_mfma_f32_16x16x128_f8f6f4 v[12:15], a[224:231], v[152:159], v[12:15] cbsz:0 blgp:1\n\t" \
  "ds_read_b64_tr_b8 v[136:137], v173 offset:49152\n\t" \
  "v_accvgpr_read_b32 v229, a20\n\t" \
  "v_accvgpr_read_b32 v230, a21\n\t" \
  "global_load_dword v196, v195, s[26:27] offset:1024 sc1\n\t" \
  "v_mfma_f32_16x16x128_f8f6f4 v[44:47], a[232:239], v[152:159], v[44:47] cbsz:0 blgp:1\n\t" \
  "ds_read_b64_tr_b8 v[138:139], v173 offset:50176\n\t" \
  "v_accvgpr_read_b32 v231, a22\n\t" \
  "v_accvgpr_read_b32 v232, a23\n\t" \
  "v_mfma_f32_16x16x128_f8f6f4 v[76:79], a[240:247], v[152:159], v[76:79] cbsz:0 blgp:1\n\t" \
  "ds_read_b64_tr_b8 v[140:141], v173 offset:57344\n\t" \
  "v_min3_u32 v228, v229, v230, v228\n\t" \
  "v_min3_u32 v228, v231, v232, v228\n\t" \
  "v_mfma_f32_16x16x128_f8f6f4 v[108:111], a[248:255], v[152:159], v[108:111] cbsz:0 blgp:1\n\t" \
  "ds_read_b64_tr_b8 v[142:143], v173 offset:58368\n\t" \
  "v_accvgpr_read_b32 v229, a24\n\t" \
  "v_accvgpr_read_b32 v230, a25\n\t" \
  "s_waitcnt lgkmcnt(4)\n\t" \
  "v_mfma_f32_16x16x128_f8f6f4 v[16:19], a[224:231], v[128:135], v[16:19] cbsz:0 blgp:1\n\t" \
  "ds_read_b64_tr_b8 v[144:145], v174 offset:49152\n\t" \
  "v_accvgpr_read_b32 v231, a26\n\t" \
  "v_accvgpr_read_b32 v232, a27\n\t" \
  "s_bfe_u32 s84, s64, 262168\n\t" \
  "s_cmp_eq_u32 s84, 15\n\t" \
  "s_cbranch_scc0 L_v10_2\n\t" \
  "L_vr10_2:\n\t" \
  "s_mov_b32 s87, 393216\n\t" \
  "buffer_load_dwordx4 a[64:67], v188, s[36:39], s87 offen offset:0 sc1\n\t" \
  "v_mfma_f32_16x16x128_f8f6f4 v[48:51], a[232:239], v[128:135], v[48:51] cbsz:0 blgp:1\n\t" \
  "ds_read_b64_tr_b8 v[146:147], v174 offset:50176\n\t" \
  "v_min3_u32 v228, v229, v230, v228\n\t" \
  "v_min3_u32 v228, v231, v232, v228\n\t" \
  "v_mfma_f32_16x16x128_f8f6f4 v[80:83], a[240:247], v[128:135], v[80:83] cbsz:0 blgp:1\n\t" \
  "ds_read_b64_tr_b8 v[148:149], v174 offset:57344\n\t" \
  "v_accvgpr_read_b32 v229, a28\n\t" \
  "v_accvgpr_read_b32 v230, a29\n\t" \
  "buffer_load_dwordx4 a[68:71], v188, s[36:39], s87 offen offset:1024 sc1\n\t" \
  "v_mfma_f32_16x16x128_f8f6f4 v[112:115], a[248:255], v[128:135], v[112:115] cbsz:0 blgp:1\n\t" \
  "ds_read_b64_tr_b8 v[150:151], v174 offset:58368\n\t" \
  "v_accvgpr_read_b32 v231, a30\n\t" \
  "v_accvgpr_read_b32 v232, a31\n\t" \
  "s_waitcnt lgkmcnt(4)\n\t" \
  "v_mfma_f32_16x16x128_f8f6f4 v[20:23], a[224:231], v[136:143], v[20:23] cbsz:0 blgp:1\n\t" \
  "ds_read_b64_tr_b8 v[152:153], v175 offset:49152\n\t" \
  "v_min3_u32 v228, v229, v230, v228\n\t" \
  "v_min3_u32 v228, v231, v232, v228\n\t" \
  "buffer_load_dwordx4 a[72:75], v188, s[36:39], s87 offen offset:2048 sc1\n\t" \
  "v_mfma_f32_16x16x128_f8f6f4 v[52:55], a[232:239], v[136:143], v[52:55] cbsz:0 blgp:1\n\t" \
  "ds_read_b64_tr_b8 v[154:155], v175 offset:50176\n\t" \
  "v_mfma_f32_16x16x128_f8f6f4 v[84:87], a[240:247], v[136:143], v[84:87] cbsz:0 blgp:1\n\t" \
  "ds_read_b64_tr_b8 v[156:157], v175 offset:57344\n\t" \
  "v_cmp_eq_u32 vcc, 0, v228\n\t" \
  "s_nop 4\n\t" \
  "s_cmp_lg_u64 vcc, 0\n\t" \
  "s_cbranch_scc1 L_v8_2\n\t" \
  "L_vr8_2:\n\t" \
  "buffer_load_dwordx4 a[76:79], v188, s[36:39], s87 offen offset:3072 sc1\n\t" \
  "v_mfma_f32_16x16x128_f8f6f4 v[116:119], a[248:255], v[136:143], v[116:119] cbsz:0 blgp:1\n\t" \
  "ds_read_b64_tr_b8 v[158:159], v175 offset:58368\n\t" \
  "s_waitcnt lgkmcnt(4)\n\t" \
  "v_mfma_f32_16x16x128_f8f6f4 v[24:27], a[224:231], v[144:151], v[24:27] cbsz:0 blgp:1\n\t" \
  "s_barrier\n\t" \
  "ds_read_b64_tr_b8 v[128:129], v176 offset:0\n\t" \
  "v_mfma_f32_16x16x128_f8f6f4 v[56:59], a[232:239], v[144:151], v[56:59] cbsz:0 blgp:1\n\t" \
  "ds_read_b64_tr_b8 v[130:131], v176 offset:1024\n\t" \
  "buffer_load_dwordx4 a[80:83], v189, s[36:39], s87 offen offset:0 sc1\n\t" \
  "v_mfma_f32_16x16x128_f8f6f4 v[88:91], a[240:247], v[144:151], v[88:91] cbsz:0 blgp:1\n\t" \
  "ds_read_b64_tr_b8 v[132:133], v176 offset:8192\n\t" \
  "v_mfma_f32_16x16x128_f8f6f4 v[120:123], a[248:255], v[144:151], v[120:123] cbsz:0 blgp:1\n\t" \
  "ds_read_b64_tr_b8 v[134:135], v176 offset:9216\n\t" \
  "buffer_load_dwordx4 a[84:87], v189, s[36:39], s87 offen offset:1024 sc1\n\t" \
  "s_waitcnt lgkmcnt(4)\n\t" \
  "v_mfma_f32_16x16x128_f8f6f4 v[28:31], a[224:231], v[152:159], v[28:31] cbsz:0 blgp:1\n\t" \
  "ds_read_b64_tr_b8 v[136:137], v177 offset:0\n\t" \
  "v_mfma_f32_16x16x128_f8f6f4 v[60:63], a[232:239], v[152:159], v[60:63] cbsz:0 blgp:1\n\t" \
  "ds_read_b64_tr_b8 v[138:139], v177 offset:1024\n\t" \
  "v_accvgpr_read_b32 v229, a32\n\t" \
  "v_accvgpr_read_b32 v230, a33\n\t" \
  "buffer_load_dwordx4 a[88:91], v189, s[36:39], s87 offen offset:2048 sc1\n\t" \
  "v_mfma_f32_16x16x128_f8f6f4 v[92:95], a[240:247], v[152:159], v[92:95] cbsz:0 blgp:1\n\t" \
  "ds_read_b64_tr_b8 v[140:141], v177 offset:8192\n\t" \
  "v_accvgpr_read_b32 v231, a34\n\t" \
  "v_accvgpr_read_b32 v232, a35\n\t" \
  "v_mfma_f32_16x16x128_f8f6f4 v[124:127], a[248:255], v[152:159], v[124:127] cbsz:0 blgp:1\n\t" \
  "ds_read_b64_tr_b8 v[142:143], v177 offset:9216\n\t" \
  "v_min3_u32 v228, v229, v230, v229\n\t" \
  "v_min3_u32 v228, v231, v232, v228\n\t" \
  "s_add_u32 m0, s77, 0\n\t" \
  "s_add_u32 s90, s74, 0\n\t" \
  "buffer_load_dwordx4 v186, s[40:43], s90 offen offset:0  lds\n\t" \
  "s_waitcnt lgkmcnt(4)\n\t" \
  "v_mfma_f32_16x16x128_f8f6f4 v[0:3], a[0:7], v[128:135], v[0:3] cbsz:0 blgp:1\n\t" \
  "ds_read_b64_tr_b8 v[144:145], v178 offset:0\n\t" \
  "v_accvgpr_read_b32 v229, a36\n\t" \
  "v_accvgpr_read_b32 v230, a37\n\t" \
  "s_waitcnt vmcnt(8)\n\t" \
  "v_cmp_le_u32 vcc, s79, v196\n\t" \
  "s_nop 4\n\t" \
  "s_andn2_b32 s85, vcc_lo, s96\n\t" \
  "s_or_b32 s96, s96, vcc_lo\n\t" \
  "s_and_b32 exec_lo, s85, 0xfffffff\n\t" \
  "s_mov_b32 exec_hi, 0\n\t" \
  "s_cbranch_execz L_u2_2\n\t" \
  "s_mov_b32 s84, 458752\n\t" \
  "buffer_load_dword v210, v198, s[36:39], s84 offen \n\t" \
  "L_u2_2:\n\t" \
  "s_mov_b64 exec, -1\n\t" \
  "s_add_u32 m0, s77, 16384\n\t" \
  "s_add_u32 s90, s68, 2097152\n\t" \
  "buffer_load_dwordx4 v186, s[40:43], s90 offen offset:0  lds\n\t" \
  "v_mfma_f32_16x16x128_f8f6f4 v[32:35], a[8:15], v[128:135], v[32:35] cbsz:0 blgp:1\n\t" \
  "ds_read_b64_tr_b8 v[146:147], v178 offset:1024\n\t" \
  "v_accvgpr_read_b32 v231, a38\n\t" \
  "v_accvgpr_read_b32 v232, a39\n\t" \
  "s_add_u32 m0, s77, 0\n\t" \
  "s_add_u32 s90, s74, 0\n\t" \
  "buffer_load_dwordx4 v187, s[40:43], s90 offen offset:1024  lds\n\t" \
  "v_mfma_f32_16x16x128_f8f6f4 v[64:67], a[16:23], v[128:135], v[64:67] cbsz:0 blgp:1\n\t" \
  "ds_read_b64_tr_b8 v[148:149], v178 offset:8192\n\t" \
  "v_min3_u32 v228, v229, v230, v228\n\t" \
  "v_min3_u32 v228, v231, v232, v228\n\t" \
  "s_add_u32 m0, s77, 16384\n\t" \
  "s_add_u32 s90, s68, 2097152\n\t" \
  "buffer_load_dwordx4 v187, s[40:43], s90 offen offset:1024  lds\n\t" \
  "v_mfma_f32_16x16x128_f8f6f4 v[96:99], a[24:31], v[128:135], v[96:99] cbsz:0 blgp:1\n\t" \
  "ds_read_b64_tr_b8 v[150:151], v178 offset:9216\n\t" \
  "v_accvgpr_read_b32 v229, a40\n\t" \
  "v_accvgpr_read_b32 v230, a41\n\t" \
  "s_add_u32 m0, s77, 32768\n\t" \
  "s_add_u32 s90, s69, 2097152\n\t" \
  "buffer_load_dwordx4 v186, s[40:43], s90 offen offset:0  lds\n\t" \
  "s_waitcnt lgkmcnt(4)\n\t" \
  "v_mfma_f32_16x16x128_f8f6f4 v[4:7], a[0:7], v[136:143], v[4:7] cbsz:0 blgp:1\n\t" \
  "ds_read_b64_tr_b8 v[152:153], v179 offset:0\n\t" \
  "v_accvgpr_read_b32 v231, a42\n\t" \
  "v_accvgpr_read_b32 v232, a43\n\t" \
  "s_add_u32 m0, s77, 49152\n\t" \
  "s_add_u32 s90, s70, 2097152\n\t" \
  "buffer_load_dwordx4 v186, s[40:43], s90 offen offset:0  lds\n\t" \
  "v_mfma_f32_16x16x128_f8f6f4 v[36:39], a[8:15], v[136:143], v[36:39] cbsz:0 blgp:1\n\t" \
  "ds_read_b64_tr_b8 v[154:155], v179 offset:1024\n\t" \
  "v_min3_u32 v228, v229, v230, v228\n\t" \
  "v_min3_u32 v228, v231, v232, v228\n\t" \
  "s_add_u32 m0, s77, 32768\n\t" \
  "s_add_u32 s90, s69, 2097152\n\t" \
  "buffer_load_dwordx4 v187, s[40:43], s90 offen offset:1024  lds\n\t" \
  "v_mfma_f32_16x16x128_f8f6f4 v[68:71], a[16:23], v[136:143], v[68:71] cbsz:0 blgp:1\n\t" \
  "ds_read_b64_tr_b8 v[156:157], v179 offset:8192\n\t" \
  "v_accvgpr_read_b32 v229, a44\n\t" \
  "v_accvgpr_read_b32 v230, a45\n\t" \
  "s_add_u32 m0, s77, 49152\n\t" \
  "s_add_u32 s90, s70, 2097152\n\t" \
  "buffer_load_dwordx4 v187, s[40:43], s90 offen offset:1024  lds\n\t" \
  "v_mfma_f32_16x16x128_f8f6f4 v[100:103], a[24:31], v[136:143], v[100:103] cbsz:0 blgp:1\n\t" \
  "ds_read_b64_tr_b8 v[158:159], v179 offset:9216\n\t" \
  "v_accvgpr_read_b32 v231, a46\n\t" \
  "v_accvgpr_read_b32 v232, a47\n\t" \
  "buffer_load_dwordx4 a[92:95], v189, s[36:39], s87 offen offset:3072 sc1\n\t" \
  "s_waitcnt lgkmcnt(4)\n\t" \
  "v_mfma_f32_16x16x128_f8f6f4 v[8:11], a[0:7], v[144:151], v[8:11] cbsz:0 blgp:1\n\t" \
  "ds_read_b64_tr_b8 v[128:129], v180 offset:0\n\t" \
  "v_min3_u32 v228, v229, v230, v228\n\t" \
  "v_min3_u32 v228, v231, v232, v228\n\t" \
  "v_mfma_f32_16x16x128_f8f6f4 v[40:43], a[8:15], v[144:151], v[40:43] cbsz:0 blgp:1\n\t" \
  "ds_read_b64_tr_b8 v[130:131], v180 offset:1024\n\t" \
  "v_accvgpr_read_b32 v229, a48\n\t" \
  "v_accvgpr_read_b32 v230, a49\n\t" \
  "s_bfe_u32 s84, s96, 262144\n\t" \
  "s_cmp_eq_u32 s84, 15\n\t" \
  "s_cbranch_scc0 L_q11_2\n\t" \
  "L_qr11_2:\n\t" \
  "s_mov_b32 s87, 458752\n\t" \
  "buffer_load_dwordx4 a[96:99], v188, s[36:39], s87 offen offset:0 \n\t" \
  "v_mfma_f32_16x16x128_f8f6f4 v[72:75], a[16:23], v[144:151], v[72:75] cbsz:0 blgp:1\n\t" \
  "ds_read_b64_tr_b8 v[132:133], v180 offset:8192\n\t" \
  "v_accvgpr_read_b32 v231, a50\n\t" \
  "v_accvgpr_read_b32 v232, a51\n\t" \
  "v_mfma_f32_16x16x128_f8f6f4 v[104:107], a[24:31], v[144:151], v[104:107] cbsz:0 blgp:1\n\t" \
  "ds_read_b64_tr_b8 v[134:135], v180 offset:9216\n\t" \
  "v_min3_u32 v228, v229, v230, v228\n\t" \
  "v_min3_u32 v228, v231, v232, v228\n\t" \
  "buffer_load_dwordx4 a[100:103], v188, s[36:39], s87 offen offset:1024 \n\t" \
  "s_waitcnt lgkmcnt(4)\n\t" \
  "v_mfma_f32_16x16x128_f8f6f4 v[12:15], a[0:7], v[152:159], v[12:15] cbsz:0 blgp:1\n\t" \
  "ds_read_b64_tr_b8 v[136:137], v181 offset:0\n\t" \
  "v_accvgpr_read_b32 v229, a52\n\t" \
  "v_accvgpr_read_b32 v230, a53\n\t" \
  "s_cmp_eq_u32 s96, -1\n\t" \
  "s_cbranch_scc0 L_ap2_2\n\t" \
  "L_apr2_2:\n\t" \
  "v_mfma_f32_16x16x128_f8f6f4 v[44:47], a[8:15], v[152:159], v[44:47] cbsz:0 blgp:1\n\t" \
  "ds_read_b64_tr_b8 v[138:139], v181 offset:1024\n\t" \
  "v_accvgpr_read_b32 v231, a54\n\t" \
  "v_accvgpr_read_b32 v232, a55\n\t" \
  "buffer_load_dwordx4 a[104:107], v188, s[36:39], s87 offen offset:2048 \n\t" \
  "v_mfma_f32_16x16x128_f8f6f4 v[76:79], a[16:23], v[152:159], v[76:79] cbsz:0 blgp:1\n\t" \
  "ds_read_b64_tr_b8 v[140:141], v181 offset:8192\n\t" \
  "v_min3_u32 v228, v229, v230, v228\n\t" \
  "v_min3_u32 v228, v231, v232, v228\n\t" \
  "v_mfma_f32_16x16x128_f8f6f4 v[108:111], a[24:31], v[152:159], v[108:111] cbsz:0 blgp:1\n\t" \
  "ds_read_b64_tr_b8 v[142:143], v181 offset:9216\n\t" \
  "v_accvgpr_read_b32 v229, a56\n\t" \
  "v_accvgpr_read_b32 v230, a57\n\t" \
  "buffer_load_dwordx4 a[108:111], v188, s[36:39], s87 offen offset:3072 \n\t" \
  "s_waitcnt lgkmcnt(4)\n\t" \
  "v_mfma_f32_16x16x128_f8f6f4 v[16:19], a[0:7], v[128:135], v[16:19] cbsz:0 blgp:1\n\t" \
  "ds_read_b64_tr_b8 v[144:145], v182 offset:0\n\t" \
  "v_accvgpr_read_b32 v231, a58\n\t" \
  "v_accvgpr_read_b32 v232, a59\n\t" \
  "v_mfma_f32_16x16x128_f8f6f4 v[48:51], a[8:15], v[128:135], v[48:51] cbsz:0 blgp:1\n\t" \
  "ds_read_b64_tr_b8 v[146:147], v182 offset:1024\n\t" \
  "v_min3_u32 v228, v229, v230, v228\n\t" \
  "v_min3_u32 v228, v231, v232, v228\n\t" \
  "buffer_load_dwordx4 a[112:115], v189, s[36:39], s87 offen offset:0 \n\t" \
  "v_mfma_f32_16x16x128_f8f6f4 v[80:83], a[16:23], v[128:135], v[80:83] cbsz:0 blgp:1\n\t" \
  "ds_read_b64_tr_b8 v[148:149], v182 offset:8192\n\t" \
  "v_accvgpr_read_b32 v229, a60\n\t" \
  "v_accvgpr_read_b32 v230, a61\n\t" \
  "v_mfma_f32_16x16x128_f8f6f4 v[112:115], a[24:31], v[128:135], v[112:115] cbsz:0 blgp:1\n\t" \
  "ds_read_b64_tr_b8 v[150:151], v182 offset:9216\n\t" \
  "v_accvgpr_read_b32 v231, a62\n\t" \
  "v_accvgpr_read_b32 v232, a63\n\t" \
  "buffer_load_dwordx4 a[116:119], v189, s[36:39], s87 offen offset:1024 \n\t" \
  "s_waitcnt lgkmcnt(4)\n\t" \
  "v_mfma_f32_16x16x128_f8f6f4 v[20:23], a[0:7], v[136:143], v[20:23] cbsz:0 blgp:1\n\t" \
  "ds_read_b64_tr_b8 v[152:153], v183 offset:0\n\t" \
  "v_min3_u32 v228, v229, v230, v228\n\t" \
  "v_min3_u32 v228, v231, v232, v228\n\t" \
  "v_mfma_f32_16x16x128_f8f6f4 v[52:55], a[8:15], v[136:143], v[52:55] cbsz:0 blgp:1\n\t" \
  "ds_read_b64_tr_b8 v[154:155], v183 offset:1024\n\t" \
  "buffer_load_dwordx4 a[120:123], v189, s[36:39], s87 offen offset:2048 \n\t" \
  "v_mfma_f32_16x16x128_f8f6f4 v[84:87], a[16:23], v[136:143], v[84:87] cbsz:0 blgp:1\n\t" \
  "ds_read_b64_tr_b8 v[156:157], v183 offset:8192\n\t" \
  "v_cmp_eq_u32 vcc, 0, v228\n\t" \
  "s_nop 4\n\t" \
  "s_cmp_lg_u64 vcc, 0\n\t" \
  "s_cbranch_scc1 L_v9_2\n\t" \
  "L_vr9_2:\n\t" \
  "v_mfma_f32_16x16x128_f8f6f4 v[116:119], a[24:31], v[136:143], v[116:119] cbsz:0 blgp:1\n\t" \
  "ds_read_b64_tr_b8 v[158:159], v183 offset:9216\n\t" \
  "buffer_load_dwordx4 a[124:127], v189, s[36:39], s87 offen offset:3072 \n\t" \
  "s_waitcnt lgkmcnt(4)\n\t" \
  "v_mfma_f32_16x16x128_f8f6f4 v[24:27], a[0:7], v[144:151], v[24:27] cbsz:0 blgp:1\n\t" \
  "s_barrier\n\t" \
  "ds_read_b64_tr_b8 v[128:129], v176 offset:16384\n\t" \
  "v_mfma_f32_16x16x128_f8f6f4 v[56:59], a[8:15], v[144:151], v[56:59] cbsz:0 blgp:1\n\t" \
  "ds_read_b64_tr_b8 v[130:131], v176 offset:17408\n\t" \
  "s_bfe_u32 s84, s96, 262148\n\t" \
  "s_cmp_eq_u32 s84, 15\n\t" \
  "s_cbranch_scc0 L_q12_2\n\t" \
  "L_qr12_2:\n\t" \
  "s_mov_b32 s87, 524288\n\t" \
  "buffer_load_dwordx4 a[128:131], v188, s[36:39], s87 offen offset:0 \n\t" \
  "v_mfma_f32_16x16x128_f8f6f4 v[88:91], a[16:23], v[144:151], v[88:91] cbsz:0 blgp:1\n\t" \
  "ds_read_b64_tr_b8 v[132:133], v176 offset:24576\n\t" \
  "v_mfma_f32_16x16x128_f8f6f4 v[120:123], a[24:31], v[144:151], v[120:123] cbsz:0 blgp:1\n\t" \
  "ds_read_b64_tr_b8 v[134:135], v176 offset:25600\n\t" \
  "buffer_load_dwordx4 a[132:135], v188, s[36:39], s87 offen offset:1024 \n\t" \
  "s_waitcnt lgkmcnt(4)\n\t" \
  "v_mfma_f32_16x16x128_f8f6f4 v[28:31], a[0:7], v[152:159], v[28:31] cbsz:0 blgp:1\n\t" \
  "ds_read_b64_tr_b8 v[136:137], v177 offset:16384\n\t" \
  "v_mfma_f32_16x16x128_f8f6f4 v[60:63], a[8:15], v[152:159], v[60:63] cbsz:0 blgp:1\n\t" \
  "ds_read_b64_tr_b8 v[138:139], v177 offset:17408\n\t" \
  "buffer_load_dwordx4 a[136:139], v188, s[36:39], s87 offen offset:2048 \n\t" \
  "v_mfma_f32_16x16x128_f8f6f4 v[92:95], a[16:23], v[152:159], v[92:95] cbsz:0 blgp:1\n\t" \
  "ds_read_b64_tr_b8 v[140:141], v177 offset:24576\n\t" \
  "v_mfma_f32_16x16x128_f8f6f4 v[124:127], a[24:31], v[152:159], v[124:127] cbsz:0 blgp:1\n\t" \
  "ds_read_b64_tr_b8 v[142:143], v177 offset:25600\n\t" \
  "buffer_load_dwordx4 a[140:143], v188, s[36:39], s87 offen offset:3072 \n\t" \
  "s_waitcnt lgkmcnt(4)\n\t" \
  "v_mfma_f32_16x16x128_f8f6f4 v[0:3], a[32:39], v[128:135], v[0:3] cbsz:0 blgp:1\n\t" \
  "ds_read_b64_tr_b8 v[144:145], v178 offset:16384\n\t" \
  "s_cmp_eq_u32 s96, -1\n\t" \
  "s_cbranch_scc0 L_au2_2\n\t" \
  "L_aur2_2:\n\t" \
  "v_mfma_f32_16x16x128_f8f6f4 v[32:35], a[40:47], v[128:135], v[32:35] cbsz:0 blgp:1\n\t" \
  "ds_read_b64_tr_b8 v[146:147], v178 offset:17408\n\t" \
  "buffer_load_dwordx4 a[144:147], v189, s[36:39], s87 offen offset:0 \n\t" \
  "v_mfma_f32_16x16x128_f8f6f4 v[64:67], a[48:55], v[128:135], v[64:67] cbsz:0 blgp:1\n\t" \
  "ds_read_b64_tr_b8 v[148:149], v178 offset:24576\n\t" \
  "v_mfma_f32_16x16x128_f8f6f4 v[96:99], a[56:63], v[128:135], v[96:99] cbsz:0 blgp:1\n\t" \
  "ds_read_b64_tr_b8 v[150:151], v178 offset:25600\n\t" \
  "buffer_load_dwordx4 a[148:151], v189, s[36:39], s87 offen offset:1024 \n\t" \
  "s_waitcnt lgkmcnt(4)\n\t" \
  "v_mfma_f32_16x16x128_f8f6f4 v[4:7], a[32:39], v[136:143], v[4:7] cbsz:0 blgp:1\n\t" \
  "ds_read_b64_tr_b8 v[152:153], v179 offset:16384\n\t" \
  "v_mfma_f32_16x16x128_f8f6f4 v[36:39], a[40:47], v[136:143], v[36:39] cbsz:0 blgp:1\n\t" \
  "ds_read_b64_tr_b8 v[154:155], v179 offset:17408\n\t" \
  "buffer_load_dwordx4 a[152:155], v189, s[36:39], s87 offen offset:2048 \n\t" \
  "v_mfma_f32_16x16x128_f8f6f4 v[68:71], a[48:55], v[136:143], v[68:71] cbsz:0 blgp:1\n\t" \
  "ds_read_b64_tr_b8 v[156:157], v179 offset:24576\n\t" \
  "v_mfma_f32_16x16x128_f8f6f4 v[100:103], a[56:63], v[136:143], v[100:103] cbsz:0 blgp:1\n\t" \
  "ds_read_b64_tr_b8 v[158:159], v179 offset:25600\n\t" \
  "buffer_load_dwordx4 a[156:159], v189, s[36:39], s87 offen offset:3072 \n\t" \
  "s_waitcnt lgkmcnt(4)\n\t" \
  "v_mfma_f32_16x16x128_f8f6f4 v[8:11], a[32:39], v[144:151], v[8:11] cbsz:0 blgp:1\n\t" \
  "ds_read_b64_tr_b8 v[128:129], v180 offset:16384\n\t" \
  "v_mfma_f32_16x16x128_f8f6f4 v[40:43], a[40:47], v[144:151], v[40:43] cbsz:0 blgp:1\n\t" \
  "ds_read_b64_tr_b8 v[130:131], v180 offset:17408\n\t" \
  "s_bfe_u32 s84, s96, 262152\n\t" \
  "s_cmp_eq_u32 s84, 15\n\t" \
  "s_cbranch_scc0 L_q13_2\n\t" \
  "L_qr13_2:\n\t" \
  "s_mov_b32 s87, 589824\n\t" \
  "buffer_load_dwordx4 a[160:163], v188, s[36:39], s87 offen offset:0 \n\t" \
  "v_mfma_f32_16x16x128_f8f6f4 v[72:75], a[48:55], v[144:151], v[72:75] cbsz:0 blgp:1\n\t" \
  "ds_read_b64_tr_b8 v[132:133], v180 offset:24576\n\t" \
  "v_mfma_f32_16x16x128_f8f6f4 v[104:107], a[56:63], v[144:151], v[104:107] cbsz:0 blgp:1\n\t" \
  "ds_read_b64_tr_b8 v[134:135], v180 offset:25600\n\t" \
  "buffer_load_dwordx4 a[164:167], v188, s[36:39], s87 offen offset:1024 \n\t" \
  "s_waitcnt lgkmcnt(4)\n\t" \
  "v_mfma_f32_16x16x128_f8f6f4 v[12:15], a[32:39], v[152:159], v[12:15] cbsz:0 blgp:1\n\t" \
  "ds_read_b64_tr_b8 v[136:137], v181 offset:16384\n\t" \
  "s_cmp_eq_u32 s96, -1\n\t" \
  "s_cbranch_scc0 L_ap3_2\n\t" \
  "L_apr3_2:\n\t" \
  "s_cmp_eq_u32 s29, 3\n\t" \
  "s_cbranch_scc0 L_z_2\n\t" \
  "s_cmp_lt_u32 s10, 224\n\t" \
  "s_cbranch_scc0 L_z_2\n\t" \
  "v_mov_b32 v200, 0\n\t" \
  "v_mov_b32 v201, 0\n\t" \
  "v_mov_b32 v202, 0\n\t" \
  "v_mov_b32 v203, 0\n\t" \
  "v_add_u32 v212, s100, v188\n\t" \
  "global_store_dwordx4 v212, v[200:203], s[26:27] sc1\n\t" \
  "global_store_dwordx4 v212, v[200:203], s[26:27] offset:1024 sc1\n\t" \
  "L_z_2:\n\t" \
  "v_mfma_f32_16x16x128_f8f6f4 v[44:47], a[40:47], v[152:159], v[44:47] cbsz:0 blgp:1\n\t" \
  "ds_read_b64_tr_b8 v[138:139], v181 offset:17408\n\t" \
  "buffer_load_dwordx4 a[168:171], v188, s[36:39], s87 offen offset:2048 \n\t" \
  "v_mfma_f32_16x16x128_f8f6f4 v[76:79], a[48:55], v[152:159], v[76:79] cbsz:0 blgp:1\n\t" \
  "ds_read_b64_tr_b8 v[140:141], v181 offset:24576\n\t" \
  "v_mfma_f32_16x16x128_f8f6f4 v[108:111], a[56:63], v[152:159], v[108:111] cbsz:0 blgp:1\n\t" \
  "ds_read_b64_tr_b8 v[142:143], v181 offset:25600\n\t" \
  "buffer_load_dwordx4 a[172:175], v188, s[36:39], s87 offen offset:3072 \n\t" \
  "s_waitcnt lgkmcnt(4)\n\t" \
  "v_mfma_f32_16x16x128_f8f6f4 v[16:19], a[32:39], v[128:135], v[16:19] cbsz:0 blgp:1\n\t" \
  "ds_read_b64_tr_b8 v[144:145], v182 offset:16384\n\t" \
  "v_mfma_f32_16x16x128_f8f6f4 v[48:51], a[40:47], v[128:135], v[48:51] cbsz:0 blgp:1\n\t" \
  "ds_read_b64_tr_b8 v[146:147], v182 offset:17408\n\t" \
  "buffer_load_dwordx4 a[176:179], v189, s[36:39], s87 offen offset:0 \n\t" \
  "v_mfma_f32_16x16x128_f8f6f4 v[80:83], a[48:55], v[128:135], v[80:83] cbsz:0 blgp:1\n\t" \
  "ds_read_b64_tr_b8 v[148:149], v182 offset:24576\n\t" \
  "v_mfma_f32_16x16x128_f8f6f4 v[112:115], a[56:63], v[128:135], v[112:115] cbsz:0 blgp:1\n\t" \
  "ds_read_b64_tr_b8 v[150:151], v182 offset:25600\n\t" \
  "buffer_load_dwordx4 a[180:183], v189, s[36:39], s87 offen offset:1024 \n\t" \
  "s_waitcnt lgkmcnt(4)\n\t" \
  "v_mfma_f32_16x16x128_f8f6f4 v[20:23], a[32:39], v[136:143], v[20:23] cbsz:0 blgp:1\n\t" \
  "ds_read_b64_tr_b8 v[152:153], v183 offset:16384\n\t" \
  "s_add_u32 m0, s76, 65536\n\t" \
  "s_add_u32 s90, s71, 2097152\n\t" \
  "buffer_load_dwordx4 v184, s[40:43], s90 offen offset:0  lds\n\t" \
  "v_mfma_f32_16x16x128_f8f6f4 v[52:55], a[40:47], v[136:143], v[52:55] cbsz:0 blgp:1\n\t" \
  "ds_read_b64_tr_b8 v[154:155], v183 offset:17408\n\t" \
  "s_add_u32 m0, s76, 65536\n\t" \
  "s_add_u32 s90, s71, 2097152\n\t" \
  "buffer_load_dwordx4 v185, s[40:43], s90 offen offset:1024  lds\n\t" \
  "v_mfma_f32_16x16x128_f8f6f4 v[84:87], a[48:55], v[136:143], v[84:87] cbsz:0 blgp:1\n\t" \
  "ds_read_b64_tr_b8 v[156:157], v183 offset:24576\n\t" \
  "s_add_u32 m0, s76, 81920\n\t" \
  "s_add_u32 s90, s72, 2097152\n\t" \
  "buffer_load_dwordx4 v184, s[40:43], s90 offen offset:0  lds\n\t" \
  "v_mfma_f32_16x16x128_f8f6f4 v[116:119], a[56:63], v[136:143], v[116:119] cbsz:0 blgp:1\n\t" \
  "ds_read_b64_tr_b8 v[158:159], v183 offset:25600\n\t" \
  "s_add_u32 m0, s76, 81920\n\t" \
  "s_add_u32 s90, s72, 2097152\n\t" \
  "buffer_load_dwordx4 v185, s[40:43], s90 offen offset:1024  lds\n\t" \
  "s_waitcnt lgkmcnt(4)\n\t" \
  "v_mfma_f32_16x16x128_f8f6f4 v[24:27], a[32:39], v[144:151], v[24:27] cbsz:0 blgp:1\n\t" \
  "s_waitcnt vmcnt(32)\n\t" \
  "s_barrier\n\t" \
  "ds_read_b64_tr_b8 v[128:129], v160 offset:0\n\t" \
  "v_mfma_f32_16x16x128_f8f6f4 v[56:59], a[40:47], v[144:151], v[56:59] cbsz:0 blgp:1\n\t" \
  "ds_read_b64_tr_b8 v[130:131], v160 offset:1024\n\t" \
  "s_add_u32 m0, s76, 98304\n\t" \
  "s_add_u32 s90, s73, 2097152\n\t" \
  "buffer_load_dwordx4 v184, s[40:43], s90 offen offset:0  lds\n\t" \
  "v_mfma_f32_16x16x128_f8f6f4 v[88:91], a[48:55], v[144:151], v[88:91] cbsz:0 blgp:1\n\t" \
  "ds_read_b64_tr_b8 v[132:133], v160 offset:8192\n\t" \
  "s_add_u32 m0, s76, 98304\n\t" \
  "s_add_u32 s90, s73, 2097152\n\t" \
  "buffer_load_dwordx4 v185, s[40:43], s90 offen offset:1024  lds\n\t" \
  "v_mfma_f32_16x16x128_f8f6f4 v[120:123], a[56:63], v[144:151], v[120:123] cbsz:0 blgp:1\n\t" \
  "ds_read_b64_tr_b8 v[134:135], v160 offset:9216\n\t" \
  "s_add_u32 m0, s76, 114688\n\t" \
  "s_add_u32 s90, s74, 2097152\n\t" \
  "buffer_load_dwordx4 v184, s[40:43], s90 offen offset:0  lds\n\t" \
  "s_waitcnt lgkmcnt(4)\n\t" \
  "v_mfma_f32_16x16x128_f8f6f4 v[28:31], a[32:39], v[152:159], v[28:31] cbsz:0 blgp:1\n\t" \
  "ds_read_b64_tr_b8 v[136:137], v161 offset:0\n\t" \
  "s_add_u32 m0, s76, 114688\n\t" \
  "s_add_u32 s90, s74, 2097152\n\t" \
  "buffer_load_dwordx4 v185, s[40:43], s90 offen offset:1024  lds\n\t" \
  "v_mfma_f32_16x16x128_f8f6f4 v[60:63], a[40:47], v[152:159], v[60:63] cbsz:0 blgp:1\n\t" \
  "ds_read_b64_tr_b8 v[138:139], v161 offset:1024\n\t" \
  "s_add_u32 s99, s29, 20\n\t" \
  "s_min_u32 s99, s99, 31\n\t" \
  "v_readlane_b32 s99, v211, s99\n\t" \
  "s_mov_b32 exec_lo, s15\n\t" \
  "s_mov_b32 exec_hi, 0\n\t" \
  "s_cbranch_execz L_c20_2\n\t" \
  "s_nop 1\n\t" \
  "buffer_load_dword v210, v222, s[60:63], s99 offen\n\t" \
  "L_c20_2:\n\t" \
  "s_mov_b64 exec, -1\n\t" \
  "v_mfma_f32_16x16x128_f8f6f4 v[92:95], a[48:55], v[152:159], v[92:95] cbsz:0 blgp:1\n\t" \
  "ds_read_b64_tr_b8 v[140:141], v161 offset:8192\n\t" \
  "buffer_load_dwordx4 a[184:187], v189, s[36:39], s87 offen offset:2048 \n\t" \
  "v_mfma_f32_16x16x128_f8f6f4 v[124:127], a[56:63], v[152:159], v[124:127] cbsz:0 blgp:1\n\t" \
  "ds_read_b64_tr_b8 v[142:143], v161 offset:9216\n\t" \
  "s_waitcnt lgkmcnt(4)\n\t" \
  "v_mfma_f32_16x16x128_f8f6f4 v[0:3], a[64:71], v[128:135], v[0:3] cbsz:0 blgp:1\n\t" \
  "ds_read_b64_tr_b8 v[144:145], v162 offset:0\n\t" \
  "s_cmp_eq_u32 s96, -1\n\t" \
  "s_cbranch_scc0 L_au3_2\n\t" \
  "L_aur3_2:\n\t" \
  "v_mfma_f32_16x16x128_f8f6f4 v[32:35], a[72:79], v[128:135], v[32:35] cbsz:0 blgp:1\n\t" \
  "ds_read_b64_tr_b8 v[146:147], v162 offset:1024\n\t" \
  "v_mfma_f32_16x16x128_f8f6f4 v[64:67], a[80:87], v[128:135], v[64:67] cbsz:0 blgp:1\n\t" \
  "ds_read_b64_tr_b8 v[148:149], v162 offset:8192\n\t" \
  "s_waitcnt vmcnt(31)\n\t" \
  "v_mfma_f32_16x16x128_f8f6f4 v[96:99], a[88:95], v[128:135], v[96:99] cbsz:0 blgp:1\n\t" \
  "ds_read_b64_tr_b8 v[150:151], v162 offset:9216\n\t" \
  "s_waitcnt lgkmcnt(4)\n\t" \
  "v_mfma_f32_16x16x128_f8f6f4 v[4:7], a[64:71], v[136:143], v[4:7] cbsz:0 blgp:1\n\t" \
  "ds_read_b64_tr_b8 v[152:153], v163 offset:0\n\t" \
  "v_mfma_f32_16x16x128_f8f6f4 v[36:39], a[72:79], v[136:143], v[36:39] cbsz:0 blgp:1\n\t" \
  "ds_read_b64_tr_b8 v[154:155], v163 offset:1024\n\t" \
  "v_mfma_f32_16x16x128_f8f6f4 v[68:71], a[80:87], v[136:143], v[68:71] cbsz:0 blgp:1\n\t" \
  "ds_read_b64_tr_b8 v[156:157], v163 offset:8192\n\t" \
  "v_mfma_f32_16x16x128_f8f6f4 v[100:103], a[88:95], v[136:143], v[100:103] cbsz:0 blgp:1\n\t" \
  "ds_read_b64_tr_b8 v[158:159], v163 offset:9216\n\t" \
  "s_waitcnt lgkmcnt(4)\n\t" \
  "v_mfma_f32_16x16x128_f8f6f4 v[8:11], a[64:71], v[144:151], v[8:11] cbsz:0 blgp:1\n\t" \
  "ds_read_b64_tr_b8 v[128:129], v164 offset:0\n\t" \
  "v_mfma_f32_16x16x128_f8f6f4 v[40:43], a[72:79], v[144:151], v[40:43] cbsz:0 blgp:1\n\t" \
  "ds_read_b64_tr_b8 v[130:131], v164 offset:1024\n\t" \
  "v_mfma_f32_16x16x128_f8f6f4 v[72:75], a[80:87], v[144:151], v[72:75] cbsz:0 blgp:1\n\t" \
  "ds_read_b64_tr_b8 v[132:133], v164 offset:8192\n\t" \
  "v_mfma_f32_16x16x128_f8f6f4 v[104:107], a[88:95], v[144:151], v[104:107] cbsz:0 blgp:1\n\t" \
  "ds_read_b64_tr_b8 v[134:135], v164 offset:9216\n\t" \
  "s_waitcnt lgkmcnt(4)\n\t" \
  "v_mfma_f32_16x16x128_f8f6f4 v[12:15], a[64:71], v[152:159], v[12:15] cbsz:0 blgp:1\n\t" \
  "ds_read_b64_tr_b8 v[136:137], v165 offset:0\n\t" \
  "v_mfma_f32_16x16x128_f8f6f4 v[44:47], a[72:79], v[152:159], v[44:47] cbsz:0 blgp:1\n\t" \
  "ds_read_b64_tr_b8 v[138:139], v165 offset:1024\n\t" \
  "v_mfma_f32_16x16x128_f8f6f4 v[76:79], a[80:87], v[152:159], v[76:79] cbsz:0 blgp:1\n\t" \
  "ds_read_b64_tr_b8 v[140:141], v165 offset:8192\n\t" \
  "v_mfma_f32_16x16x128_f8f6f4 v[108:111], a[88:95], v[152:159], v[108:111] cbsz:0 blgp:1\n\t" \
  "ds_read_b64_tr_b8 v[142:143], v165 offset:9216\n\t" \
  "s_waitcnt lgkmcnt(4)\n\t" \
  "v_mfma_f32_16x16x128_f8f6f4 v[16:19], a[64:71], v[128:135], v[16:19] cbsz:0 blgp:1\n\t" \
  "ds_read_b64_tr_b8 v[144:145], v166 offset:0\n\t" \
  "v_mfma_f32_16x16x128_f8f6f4 v[48:51], a[72:79], v[128:135], v[48:51] cbsz:0 blgp:1\n\t" \
  "ds_read_b64_tr_b8 v[146:147], v166 offset:1024\n\t" \
  "v_mfma_f32_16x16x128_f8f6f4 v[80:83], a[80:87], v[128:135], v[80:83] cbsz:0 blgp:1\n\t" \
  "ds_read_b64_tr_b8 v[148:149], v166 offset:8192\n\t" \
  "v_mfma_f32_16x16x128_f8f6f4 v[112:115], a[88:95], v[128:135], v[112:115] cbsz:0 blgp:1\n\t" \
  "ds_read_b64_tr_b8 v[150:151], v166 offset:9216\n\t" \
  "s_mov_b32 s8, 0\n\t" \
  "s_waitcnt lgkmcnt(4)\n\t" \
  "v_mfma_f32_16x16x128_f8f6f4 v[20:23], a[64:71], v[136:143], v[20:23] cbsz:0 blgp:1\n\t" \
  "ds_read_b64_tr_b8 v[152:153], v167 offset:0\n\t" \
  "global_load_dword v196, v195, s[26:27] offset:2048 sc1\n\t" \
  "v_mfma_f32_16x16x128_f8f6f4 v[52:55], a[72:79], v[136:143], v[52:55] cbsz:0 blgp:1\n\t" \
  "ds_read_b64_tr_b8 v[154:155], v167 offset:1024\n\t" \
  "v_mfma_f32_16x16x128_f8f6f4 v[84:87], a[80:87], v[136:143], v[84:87] cbsz:0 blgp:1\n\t" \
  "ds_read_b64_tr_b8 v[156:157], v167 offset:8192\n\t" \
  "v_mfma_f32_16x16x128_f8f6f4 v[116:119], a[88:95], v[136:143], v[116:119] cbsz:0 blgp:1\n\t" \
  "ds_read_b64_tr_b8 v[158:159], v167 offset:9216\n\t" \
  "s_waitcnt lgkmcnt(4)\n\t" \
  "v_mfma_f32_16x16x128_f8f6f4 v[24:27], a[64:71], v[144:151], v[24:27] cbsz:0 blgp:1\n\t" \
  "s_barrier\n\t" \
  "ds_read_b64_tr_b8 v[128:129], v160 offset:16384\n\t" \
  "v_mfma_f32_16x16x128_f8f6f4 v[56:59], a[72:79], v[144:151], v[56:59] cbsz:0 blgp:1\n\t" \
  "ds_read_b64_tr_b8 v[130:131], v160 offset:17408\n\t" \
  "v_mfma_f32_16x16x128_f8f6f4 v[88:91], a[80:87], v[144:151], v[88:91] cbsz:0 blgp:1\n\t" \
  "ds_read_b64_tr_b8 v[132:133], v160 offset:24576\n\t" \
  "v_mfma_f32_16x16x128_f8f6f4 v[120:123], a[88:95], v[144:151], v[120:123] cbsz:0 blgp:1\n\t" \
  "ds_read_b64_tr_b8 v[134:135], v160 offset:25600\n\t" \
  "s_waitcnt lgkmcnt(4)\n\t" \
  "v_mfma_f32_16x16x128_f8f6f4 v[28:31], a[64:71], v[152:159], v[28:31] cbsz:0 blgp:1\n\t" \
  "ds_read_b64_tr_b8 v[136:137], v161 offset:16384\n\t" \
  "v_mfma_f32_16x16x128_f8f6f4 v[60:63], a[72:79], v[152:159], v[60:63] cbsz:0 blgp:1\n\t" \
  "ds_read_b64_tr_b8 v[138:139], v161 offset:17408\n\t" \
  "buffer_load_dwordx4 a[188:191], v189, s[36:39], s87 offen offset:3072 \n\t" \
  "v_mfma_f32_16x16x128_f8f6f4 v[92:95], a[80:87], v[152:159], v[92:95] cbsz:0 blgp:1\n\t" \
  "ds_read_b64_tr_b8 v[140:141], v161 offset:24576\n\t" \
  "v_mfma_f32_16x16x128_f8f6f4 v[124:127], a[88:95], v[152:159], v[124:127] cbsz:0 blgp:1\n\t" \
  "ds_read_b64_tr_b8 v[142:143], v161 offset:25600\n\t" \
  "s_bfe_u32 s84, s96, 262156\n\t" \
  "s_cmp_eq_u32 s84, 15\n\t" \
  "s_cbranch_scc0 L_q14_2\n\t" \
  "L_qr14_2:\n\t" \
  "s_mov_b32 s87, 655360\n\t" \
  "buffer_load_dwordx4 a[192:195], v188, s[36:39], s87 offen offset:0 \n\t" \
  "s_waitcnt vmcnt(32) lgkmcnt(4)\n\t" \
  "v_mfma_f32_16x16x128_f8f6f4 v[0:3], a[96:103], v[128:135], v[0:3] cbsz:0 blgp:1\n\t" \
  "ds_read_b64_tr_b8 v[144:145], v162 offset:16384\n\t" \
  "s_waitcnt vmcnt(30)\n\t" \
  "v_mfma_f32_16x16x128_f8f6f4 v[32:35], a[104:111], v[128:135], v[32:35] cbsz:0 blgp:1\n\t" \
  "ds_read_b64_tr_b8 v[146:147], v162 offset:17408\n\t" \
  "buffer_load_dwordx4 a[196:199], v188, s[36:39], s87 offen offset:1024 \n\t" \
  "s_waitcnt vmcnt(29)\n\t" \
  "v_mfma_f32_16x16x128_f8f6f4 v[64:67], a[112:119], v[128:135], v[64:67] cbsz:0 blgp:1\n\t" \
  "ds_read_b64_tr_b8 v[148:149], v162 offset:24576\n\t" \
  "s_waitcnt vmcnt(27)\n\t" \
  "v_mfma_f32_16x16x128_f8f6f4 v[96:99], a[120:127], v[128:135], v[96:99] cbsz:0 blgp:1\n\t" \
  "ds_read_b64_tr_b8 v[150:151], v162 offset:25600\n\t" \
  "buffer_load_dwordx4 a[200:203], v188, s[36:39], s87 offen offset:2048 \n\t" \
  "s_waitcnt lgkmcnt(4)\n\t" \
  "v_mfma_f32_16x16x128_f8f6f4 v[4:7], a[96:103], v[136:143], v[4:7] cbsz:0 blgp:1\n\t" \
  "ds_read_b64_tr_b8 v[152:153], v163 offset:16384\n\t" \
  "v_mfma_f32_16x16x128_f8f6f4 v[36:39], a[104:111], v[136:143], v[36:39] cbsz:0 blgp:1\n\t" \
  "ds_read_b64_tr_b8 v[154:155], v163 offset:17408\n\t" \
  "buffer_load_dwordx4 a[204:207], v188, s[36:39], s87 offen offset:3072 \n\t" \
  "v_mfma_f32_16x16x128_f8f6f4 v[68:71], a[112:119], v[136:143], v[68:71] cbsz:0 blgp:1\n\t" \
  "ds_read_b64_tr_b8 v[156:157], v163 offset:24576\n\t" \
  "v_mfma_f32_16x16x128_f8f6f4 v[100:103], a[120:127], v[136:143], v[100:103] cbsz:0 blgp:1\n\t" \
  "ds_read_b64_tr_b8 v[158:159], v163 offset:25600\n\t" \
  "buffer_load_dwordx4 a[208:211], v189, s[36:39], s87 offen offset:0 \n\t" \
  "s_waitcnt lgkmcnt(4)\n\t" \
  "v_mfma_f32_16x16x128_f8f6f4 v[8:11], a[96:103], v[144:151], v[8:11] cbsz:0 blgp:1\n\t" \
  "ds_read_b64_tr_b8 v[128:129], v164 offset:16384\n\t" \
  "s_waitcnt vmcnt(6)\n\t" \
  "v_cmp_le_u32 vcc, s79, v196\n\t" \
  "s_nop 4\n\t" \
  "s_andn2_b32 s85, vcc_lo, s8\n\t" \
  "s_or_b32 s8, s8, vcc_lo\n\t" \
  "s_and_b32 exec_lo, s85, 0xfffffff\n\t" \
  "s_mov_b32 exec_hi, 0\n\t" \
  "s_cbranch_execz L_u3_2\n\t" \
  "s_mov_b32 s84, 917504\n\t" \
  "buffer_load_dword v210, v198, s[36:39], s84 offen \n\t" \
  "L_u3_2:\n\t" \
  "s_mov_b64 exec, -1\n\t" \
  "v_mfma_f32_16x16x128_f8f6f4 v[40:43], a[104:111], v[144:151], v[40:43] cbsz:0 blgp:1\n\t" \
  "ds_read_b64_tr_b8 v[130:131], v164 offset:17408\n\t" \
  "buffer_load_dwordx4 a[212:215], v189, s[36:39], s87 offen offset:1024 \n\t" \
  "v_mfma_f32_16x16x128_f8f6f4 v[72:75], a[112:119], v[144:151], v[72:75] cbsz:0 blgp:1\n\t" \
  "ds_read_b64_tr_b8 v[132:133], v164 offset:24576\n\t" \
  "v_mfma_f32_16x16x128_f8f6f4 v[104:107], a[120:127], v[144:151], v[104:107] cbsz:0 blgp:1\n\t" \
  "ds_read_b64_tr_b8 v[134:135], v164 offset:25600\n\t" \
  "buffer_load_dwordx4 a[216:219], v189, s[36:39], s87 offen offset:2048 \n\t" \
  "s_waitcnt lgkmcnt(4)\n\t" \
  "v_mfma_f32_16x16x128_f8f6f4 v[12:15], a[96:103], v[152:159], v[12:15] cbsz:0 blgp:1\n\t" \
  "ds_read_b64_tr_b8 v[136:137], v165 offset:16384\n\t" \
  "s_add_u32 m0, s77, 65536\n\t" \
  "s_add_u32 s90, s71, 2097152\n\t" \
  "buffer_load_dwordx4 v186, s[40:43], s90 offen offset:0  lds\n\t" \
  "v_mfma_f32_16x16x128_f8f6f4 v[44:47], a[104:111], v[152:159], v[44:47] cbsz:0 blgp:1\n\t" \
  "ds_read_b64_tr_b8 v[138:139], v165 offset:17408\n\t" \
  "s_add_u32 m0, s77, 81920\n\t" \
  "s_add_u32 s90, s72, 2097152\n\t" \
  "buffer_load_dwordx4 v186, s[40:43], s90 offen offset:0  lds\n\t" \
  "v_mfma_f32_16x16x128_f8f6f4 v[76:79], a[112:119], v[152:159], v[76:79] cbsz:0 blgp:1\n\t" \
  "ds_read_b64_tr_b8 v[140:141], v165 offset:24576\n\t" \
  "s_add_u32 m0, s77, 65536\n\t" \
  "s_add_u32 s90, s71, 2097152\n\t" \
  "buffer_load_dwordx4 v187, s[40:43], s90 offen offset:1024  lds\n\t" \
  "v_mfma_f32_16x16x128_f8f6f4 v[108:111], a[120:127], v[152:159], v[108:111] cbsz:0 blgp:1\n\t" \
  "ds_read_b64_tr_b8 v[142:143], v165 offset:25600\n\t" \
  "s_add_u32 m0, s77, 81920\n\t" \
  "s_add_u32 s90, s72, 2097152\n\t" \
  "buffer_load_dwordx4 v187, s[40:43], s90 offen offset:1024  lds\n\t" \
  "s_waitcnt lgkmcnt(4)\n\t" \
  "v_mfma_f32_16x16x128_f8f6f4 v[16:19], a[96:103], v[128:135], v[16:19] cbsz:0 blgp:1\n\t" \
  "ds_read_b64_tr_b8 v[144:145], v166 offset:16384\n\t" \
  "s_add_u32 m0, s77, 98304\n\t" \
  "s_add_u32 s90, s73, 2097152\n\t" \
  "buffer_load_dwordx4 v186, s[40:43], s90 offen offset:0  lds\n\t" \
  "v_mfma_f32_16x16x128_f8f6f4 v[48:51], a[104:111], v[128:135], v[48:51] cbsz:0 blgp:1\n\t" \
  "ds_read_b64_tr_b8 v[146:147], v166 offset:17408\n\t" \
  "s_add_u32 m0, s77, 114688\n\t" \
  "s_add_u32 s90, s74, 2097152\n\t" \
  "buffer_load_dwordx4 v186, s[40:43], s90 offen offset:0  lds\n\t" \
  "v_mfma_f32_16x16x128_f8f6f4 v[80:83], a[112:119], v[128:135], v[80:83] cbsz:0 blgp:1\n\t" \
  "ds_read_b64_tr_b8 v[148:149], v166 offset:24576\n\t" \
  "s_add_u32 m0, s77, 98304\n\t" \
  "s_add_u32 s90, s73, 2097152\n\t" \
  "buffer_load_dwordx4 v187, s[40:43], s90 offen offset:1024  lds\n\t" \
  "v_mfma_f32_16x16x128_f8f6f4 v[112:115], a[120:127], v[128:135], v[112:115] cbsz:0 blgp:1\n\t" \
  "ds_read_b64_tr_b8 v[150:151], v166 offset:25600\n\t" \
  "s_add_u32 m0, s77, 114688\n\t" \
  "s_add_u32 s90, s74, 2097152\n\t" \
  "buffer_load_dwordx4 v187, s[40:43], s90 offen offset:1024  lds\n\t" \
  "s_waitcnt lgkmcnt(4)\n\t" \
  "v_mfma_f32_16x16x128_f8f6f4 v[20:23], a[96:103], v[136:143], v[20:23] cbsz:0 blgp:1\n\t" \
  "ds_read_b64_tr_b8 v[152:153], v167 offset:16384\n\t" \
  "s_cmp_eq_u32 s8, -1\n\t" \
  "s_cbranch_scc0 L_ap5_2\n\t" \
  "L_apr5_2:\n\t" \
  "buffer_load_dwordx4 a[220:223], v189, s[36:39], s87 offen offset:3072 \n\t" \
  "v_mfma_f32_16x16x128_f8f6f4 v[52:55], a[104:111], v[136:143], v[52:55] cbsz:0 blgp:1\n\t" \
  "ds_read_b64_tr_b8 v[154:155], v167 offset:17408\n\t" \
  "v_mfma_f32_16x16x128_f8f6f4 v[84:87], a[112:119], v[136:143], v[84:87] cbsz:0 blgp:1\n\t" \
  "ds_read_b64_tr_b8 v[156:157], v167 offset:24576\n\t" \
  "s_bfe_u32 s84, s96, 262160\n\t" \
  "s_cmp_eq_u32 s84, 15\n\t" \
  "s_cbranch_scc0 L_q15_2\n\t" \
  "L_qr15_2:\n\t" \
  "s_mov_b32 s87, 720896\n\t" \
  "buffer_load_dwordx4 a[224:227], v188, s[36:39], s87 offen offset:0 \n\t" \
  "v_mfma_f32_16x16x128_f8f6f4 v[116:119], a[120:127], v[136:143], v[116:119] cbsz:0 blgp:1\n\t" \
  "ds_read_b64_tr_b8 v[158:159], v167 offset:25600\n\t" \
  "s_waitcnt lgkmcnt(4)\n\t" \
  "v_mfma_f32_16x16x128_f8f6f4 v[24:27], a[96:103], v[144:151], v[24:27] cbsz:0 blgp:1\n\t" \
  "s_barrier\n\t" \
  "ds_read_b64_tr_b8 v[128:129], v160 offset:32768\n\t" \
  "v_mfma_f32_16x16x128_f8f6f4 v[56:59], a[104:111], v[144:151], v[56:59] cbsz:0 blgp:1\n\t" \
  "ds_read_b64_tr_b8 v[130:131], v160 offset:33792\n\t" \
  "buffer_load_dwordx4 a[228:231], v188, s[36:39], s87 offen offset:1024 \n\t" \
  "v_mfma_f32_16x16x128_f8f6f4 v[88:91], a[112:119], v[144:151], v[88:91] cbsz:0 blgp:1\n\t" \
  "ds_read_b64_tr_b8 v[132:133], v160 offset:40960\n\t" \
  "v_mfma_f32_16x16x128_f8f6f4 v[120:123], a[120:127], v[144:151], v[120:123] cbsz:0 blgp:1\n\t" \
  "ds_read_b64_tr_b8 v[134:135], v160 offset:41984\n\t" \
  "buffer_load_dwordx4 a[232:235], v188, s[36:39], s87 offen offset:2048 \n\t" \
  "s_waitcnt lgkmcnt(4)\n\t" \
  "v_mfma_f32_16x16x128_f8f6f4 v[28:31], a[96:103], v[152:159], v[28:31] cbsz:0 blgp:1\n\t" \
  "ds_read_b64_tr_b8 v[136:137], v161 offset:32768\n\t" \
  "v_mfma_f32_16x16x128_f8f6f4 v[60:63], a[104:111], v[152:159], v[60:63] cbsz:0 blgp:1\n\t" \
  "ds_read_b64_tr_b8 v[138:139], v161 offset:33792\n\t" \
  "buffer_load_dwordx4 a[236:239], v188, s[36:39], s87 offen offset:3072 \n\t" \
  "v_mfma_f32_16x16x128_f8f6f4 v[92:95], a[112:119], v[152:159], v[92:95] cbsz:0 blgp:1\n\t" \
  "ds_read_b64_tr_b8 v[140:141], v161 offset:40960\n\t" \
  "v_mfma_f32_16x16x128_f8f6f4 v[124:127], a[120:127], v[152:159], v[124:127] cbsz:0 blgp:1\n\t" \
  "ds_read_b64_tr_b8 v[142:143], v161 offset:41984\n\t" \
  "buffer_load_dwordx4 a[240:243], v189, s[36:39], s87 offen offset:0 \n\t" \
  "s_waitcnt lgkmcnt(4)\n\t" \
  "v_mfma_f32_16x16x128_f8f6f4 v[0:3], a[128:135], v[128:135], v[0:3] cbsz:0 blgp:1\n\t" \
  "ds_read_b64_tr_b8 v[144:145], v162 offset:32768\n\t" \
  "v_mfma_f32_16x16x128_f8f6f4 v[32:35], a[136:143], v[128:135], v[32:35] cbsz:0 blgp:1\n\t" \
  "ds_read_b64_tr_b8 v[146:147], v162 offset:33792\n\t" \
  "buffer_load_dwordx4 a[244:247], v189, s[36:39], s87 offen offset:1024 \n\t" \
  "v_mfma_f32_16x16x128_f8f6f4 v[64:67], a[144:151], v[128:135], v[64:67] cbsz:0 blgp:1\n\t" \
  "ds_read_b64_tr_b8 v[148:149], v162 offset:40960\n\t" \
  "v_mfma_f32_16x16x128_f8f6f4 v[96:99], a[152:159], v[128:135], v[96:99] cbsz:0 blgp:1\n\t" \
  "ds_read_b64_tr_b8 v[150:151], v162 offset:41984\n\t" \
  "buffer_load_dwordx4 a[248:251], v189, s[36:39], s87 offen offset:2048 \n\t" \
  "s_waitcnt lgkmcnt(4)\n\t" \
  "v_mfma_f32_16x16x128_f8f6f4 v[4:7], a[128:135], v[136:143], v[4:7] cbsz:0 blgp:1\n\t" \
  "ds_read_b64_tr_b8 v[152:153], v163 offset:32768\n\t" \
  "v_mfma_f32_16x16x128_f8f6f4 v[36:39], a[136:143], v[136:143], v[36:39] cbsz:0 blgp:1\n\t" \
  "ds_read_b64_tr_b8 v[154:155], v163 offset:33792\n\t" \
  "buffer_load_dwordx4 a[252:255], v189, s[36:39], s87 offen offset:3072 \n\t" \
  "v_mfma_f32_16x16x128_f8f6f4 v[68:71], a[144:151], v[136:143], v[68:71] cbsz:0 blgp:1\n\t" \
  "ds_read_b64_tr_b8 v[156:157], v163 offset:40960\n\t" \
  "v_mfma_f32_16x16x128_f8f6f4 v[100:103], a[152:159], v[136:143], v[100:103] cbsz:0 blgp:1\n\t" \
  "ds_read_b64_tr_b8 v[158:159], v163 offset:41984\n\t" \
  "s_bfe_u32 s84, s96, 262164\n\t" \
  "s_cmp_eq_u32 s84, 15\n\t" \
  "s_cbranch_scc0 L_q16_2\n\t" \
  "L_qr16_2:\n\t" \
  "s_mov_b32 s87, 786432\n\t" \
  "buffer_load_dwordx4 a[0:3], v188, s[36:39], s87 offen offset:0 \n\t" \
  "s_waitcnt lgkmcnt(4)\n\t" \
  "v_mfma_f32_16x16x128_f8f6f4 v[8:11], a[128:135], v[144:151], v[8:11] cbsz:0 blgp:1\n\t" \
  "ds_read_b64_tr_b8 v[128:129], v164 offset:32768\n\t" \
  "s_cmp_eq_u32 s8, -1\n\t" \
  "s_cbranch_scc0 L_au5_2\n\t" \
  "L_aur5_2:\n\t" \
  "v_mfma_f32_16x16x128_f8f6f4 v[40:43], a[136:143], v[144:151], v[40:43] cbsz:0 blgp:1\n\t" \
  "ds_read_b64_tr_b8 v[130:131], v164 offset:33792\n\t" \
  "buffer_load_dwordx4 a[4:7], v188, s[36:39], s87 offen offset:1024 \n\t" \
  "v_mfma_f32_16x16x128_f8f6f4 v[72:75], a[144:151], v[144:151], v[72:75] cbsz:0 blgp:1\n\t" \
  "ds_read_b64_tr_b8 v[132:133], v164 offset:40960\n\t" \
  "v_mfma_f32_16x16x128_f8f6f4 v[104:107], a[152:159], v[144:151], v[104:107] cbsz:0 blgp:1\n\t" \
  "ds_read_b64_tr_b8 v[134:135], v164 offset:41984\n\t" \
  "buffer_load_dwordx4 a[8:11], v188, s[36:39], s87 offen offset:2048 \n\t" \
  "s_waitcnt lgkmcnt(4)\n\t" \
  "v_mfma_f32_16x16x128_f8f6f4 v[12:15], a[128:135], v[152:159], v[12:15] cbsz:0 blgp:1\n\t" \
  "ds_read_b64_tr_b8 v[136:137], v165 offset:32768\n\t" \
  "v_mfma_f32_16x16x128_f8f6f4 v[44:47], a[136:143], v[152:159], v[44:47] cbsz:0 blgp:1\n\t" \
  "ds_read_b64_tr_b8 v[138:139], v165 offset:33792\n\t" \
  "buffer_load_dwordx4 a[12:15], v188, s[36:39], s87 offen offset:3072 \n\t" \
  "v_mfma_f32_16x16x128_f8f6f4 v[76:79], a[144:151], v[152:159], v[76:79] cbsz:0 blgp:1\n\t" \
  "ds_read_b64_tr_b8 v[140:141], v165 offset:40960\n\t" \
  "v_mfma_f32_16x16x128_f8f6f4 v[108:111], a[152:159], v[152:159], v[108:111] cbsz:0 blgp:1\n\t" \
  "ds_read_b64_tr_b8 v[142:143], v165 offset:41984\n\t" \
  "buffer_load_dwordx4 a[16:19], v189, s[36:39], s87 offen offset:0 \n\t" \
  "s_waitcnt lgkmcnt(4)\n\t" \
  "v_mfma_f32_16x16x128_f8f6f4 v[16:19], a[128:135], v[128:135], v[16:19] cbsz:0 blgp:1\n\t" \
  "ds_read_b64_tr_b8 v[144:145], v166 offset:32768\n\t" \
  "v_mfma_f32_16x16x128_f8f6f4 v[48:51], a[136:143], v[128:135], v[48:51] cbsz:0 blgp:1\n\t" \
  "ds_read_b64_tr_b8 v[146:147], v166 offset:33792\n\t" \
  "buffer_load_dwordx4 a[20:23], v189, s[36:39], s87 offen offset:1024 \n\t" \
  "v_mfma_f32_16x16x128_f8f6f4 v[80:83], a[144:151], v[128:135], v[80:83] cbsz:0 blgp:1\n\t" \
  "ds_read_b64_tr_b8 v[148:149], v166 offset:40960\n\t" \
  "v_mfma_f32_16x16x128_f8f6f4 v[112:115], a[152:159], v[128:135], v[112:115] cbsz:0 blgp:1\n\t" \
  "ds_read_b64_tr_b8 v[150:151], v166 offset:41984\n\t" \
  "buffer_load_dwordx4 a[24:27], v189, s[36:39], s87 offen offset:2048 \n\t" \
  "s_waitcnt lgkmcnt(4)\n\t" \
  "v_mfma_f32_16x16x128_f8f6f4 v[20:23], a[128:135], v[136:143], v[20:23] cbsz:0 blgp:1\n\t" \
  "ds_read_b64_tr_b8 v[152:153], v167 offset:32768\n\t" \
  "s_cmp_eq_u32 s8, -1\n\t" \
  "s_cbranch_scc0 L_ap6_2\n\t" \
  "L_apr6_2:\n\t" \
  "v_mfma_f32_16x16x128_f8f6f4 v[52:55], a[136:143], v[136:143], v[52:55] cbsz:0 blgp:1\n\t" \
  "ds_read_b64_tr_b8 v[154:155], v167 offset:33792\n\t" \
  "buffer_load_dwordx4 a[28:31], v189, s[36:39], s87 offen offset:3072 \n\t" \
  "v_mfma_f32_16x16x128_f8f6f4 v[84:87], a[144:151], v[136:143], v[84:87] cbsz:0 blgp:1\n\t" \
  "ds_read_b64_tr_b8 v[156:157], v167 offset:40960\n\t" \
  "v_mfma_f32_16x16x128_f8f6f4 v[116:119], a[152:159], v[136:143], v[116:119] cbsz:0 blgp:1\n\t" \
  "ds_read_b64_tr_b8 v[158:159], v167 offset:41984\n\t" \
  "s_bfe_u32 s84, s96, 262168\n\t" \
  "s_cmp_eq_u32 s84, 15\n\t" \
  "s_cbranch_scc0 L_q17_2\n\t" \
  "L_qr17_2:\n\t" \
  "s_mov_b32 s87, 851968\n\t" \
  "buffer_load_dwordx4 a[32:35], v188, s[36:39], s87 offen offset:0 \n\t" \
  "s_waitcnt lgkmcnt(4)\n\t" \
  "v_mfma_f32_16x16x128_f8f6f4 v[24:27], a[128:135], v[144:151], v[24:27] cbsz:0 blgp:1\n\t" \
  "s_barrier\n\t" \
  "ds_read_b64_tr_b8 v[128:129], v160 offset:49152\n\t" \
  "v_mfma_f32_16x16x128_f8f6f4 v[56:59], a[136:143], v[144:151], v[56:59] cbsz:0 blgp:1\n\t" \
  "ds_read_b64_tr_b8 v[130:131], v160 offset:50176\n\t" \
  "buffer_load_dwordx4 a[36:39], v188, s[36:39], s87 offen offset:1024 \n\t" \
  "v_mfma_f32_16x16x128_f8f6f4 v[88:91], a[144:151], v[144:151], v[88:91] cbsz:0 blgp:1\n\t" \
  "ds_read_b64_tr_b8 v[132:133], v160 offset:57344\n\t" \
  "v_mfma_f32_16x16x128_f8f6f4 v[120:123], a[152:159], v[144:151], v[120:123] cbsz:0 blgp:1\n\t" \
  "ds_read_b64_tr_b8 v[134:135], v160 offset:58368\n\t" \
  "buffer_load_dwordx4 a[40:43], v188, s[36:39], s87 offen offset:2048 \n\t" \
  "s_waitcnt lgkmcnt(4)\n\t" \
  "v_mfma_f32_16x16x128_f8f6f4 v[28:31], a[128:135], v[152:159], v[28:31] cbsz:0 blgp:1\n\t" \
  "ds_read_b64_tr_b8 v[136:137], v161 offset:49152\n\t" \
  "v_mfma_f32_16x16x128_f8f6f4 v[60:63], a[136:143], v[152:159], v[60:63] cbsz:0 blgp:1\n\t" \
  "ds_read_b64_tr_b8 v[138:139], v161 offset:50176\n\t" \
  "buffer_load_dwordx4 a[44:47], v188, s[36:39], s87 offen offset:3072 \n\t" \
  "v_mfma_f32_16x16x128_f8f6f4 v[92:95], a[144:151], v[152:159], v[92:95] cbsz:0 blgp:1\n\t" \
  "ds_read_b64_tr_b8 v[140:141], v161 offset:57344\n\t" \
  "v_mfma_f32_16x16x128_f8f6f4 v[124:127], a[152:159], v[152:159], v[124:127] cbsz:0 blgp:1\n\t" \
  "ds_read_b64_tr_b8 v[142:143], v161 offset:58368\n\t" \
  "buffer_load_dwordx4 a[48:51], v189, s[36:39], s87 offen offset:0 \n\t" \
  "s_waitcnt lgkmcnt(4)\n\t" \
  "v_mfma_f32_16x16x128_f8f6f4 v[0:3], a[160:167], v[128:135], v[0:3] cbsz:0 blgp:1\n\t" \
  "ds_read_b64_tr_b8 v[144:145], v162 offset:49152\n\t" \
  "s_add_u32 m0, s76, 131072\n\t" \
  "s_add_u32 s90, s68, 4194304\n\t" \
  "buffer_load_dwordx4 v184, s[40:43], s90 offen offset:0  lds\n\t" \
  "v_mfma_f32_16x16x128_f8f6f4 v[32:35], a[168:175], v[128:135], v[32:35] cbsz:0 blgp:1\n\t" \
  "ds_read_b64_tr_b8 v[146:147], v162 offset:50176\n\t" \
  "s_add_u32 m0, s76, 131072\n\t" \
  "s_add_u32 s90, s68, 4194304\n\t" \
  "buffer_load_dwordx4 v185, s[40:43], s90 offen offset:1024  lds\n\t" \
  "v_mfma_f32_16x16x128_f8f6f4 v[64:67], a[176:183], v[128:135], v[64:67] cbsz:0 blgp:1\n\t" \
  "ds_read_b64_tr_b8 v[148:149], v162 offset:57344\n\t" \
  "s_cmp_gt_u32 s29, 1\n\t" \
  "s_cbranch_scc1 L_k418_2\n\t" \
  "s_getpc_b64 s[88:89]\n\t" \
  "s_lshl_b32 s84, s29, 13\n\t" \
  "s_add_u32 s84, s84, 2048\n\t" \
  "s_add_u32 s88, s88, s84\n\t" \
  "s_addc_u32 s89, s89, 0\n\t" \
  "s_mov_b64 exec, 0xff\n\t" \
  "global_load_dword v210, v234, s[88:89]\n\t" \
  "s_mov_b64 exec, -1\n\t" \
  "L_k418_2:\n\t" \
  "s_add_u32 m0, s76, 147456\n\t" \
  "s_add_u32 s90, s69, 4194304\n\t" \
  "buffer_load_dwordx4 v184, s[40:43], s90 offen offset:0  lds\n\t" \
  "s_waitcnt vmcnt(40)\n\t" \
  "v_mfma_f32_16x16x128_f8f6f4 v[96:99], a[184:191], v[128:135], v[96:99] cbsz:0 blgp:1\n\t" \
  "ds_read_b64_tr_b8 v[150:151], v162 offset:58368\n\t" \
  "s_add_u32 m0, s76, 147456\n\t" \
  "s_add_u32 s90, s69, 4194304\n\t" \
  "buffer_load_dwordx4 v185, s[40:43], s90 offen offset:1024  lds\n\t" \
  "s_waitcnt lgkmcnt(4)\n\t" \
  "v_mfma_f32_16x16x128_f8f6f4 v[4:7], a[160:167], v[136:143], v[4:7] cbsz:0 blgp:1\n\t" \
  "ds_read_b64_tr_b8 v[152:153], v163 offset:49152\n\t" \
  "s_add_u32 m0, s76, 0\n\t" \
  "s_add_u32 s90, s70, 4194304\n\t" \
  "buffer_load_dwordx4 v184, s[40:43], s90 offen offset:0  lds\n\t" \
  "v_mfma_f32_16x16x128_f8f6f4 v[36:39], a[168:175], v[136:143], v[36:39] cbsz:0 blgp:1\n\t" \
  "ds_read_b64_tr_b8 v[154:155], v163 offset:50176\n\t" \
  "s_add_u32 m0, s76, 0\n\t" \
  "s_add_u32 s90, s70, 4194304\n\t" \
  "buffer_load_dwordx4 v185, s[40:43], s90 offen offset:1024  lds\n\t" \
  "v_mfma_f32_16x16x128_f8f6f4 v[68:71], a[176:183], v[136:143], v[68:71] cbsz:0 blgp:1\n\t" \
  "ds_read_b64_tr_b8 v[156:157], v163 offset:57344\n\t" \
  "s_add_u32 m0, s76, 16384\n\t" \
  "s_add_u32 s90, s71, 4194304\n\t" \
  "buffer_load_dwordx4 v184, s[40:43], s90 offen offset:0  lds\n\t" \
  "v_mfma_f32_16x16x128_f8f6f4 v[100:103], a[184:191], v[136:143], v[100:103] cbsz:0 blgp:1\n\t" \
  "ds_read_b64_tr_b8 v[158:159], v163 offset:58368\n\t" \
  "s_add_u32 m0, s76, 16384\n\t" \
  "s_add_u32 s90, s71, 4194304\n\t" \
  "buffer_load_dwordx4 v185, s[40:43], s90 offen offset:1024  lds\n\t" \
  "s_waitcnt lgkmcnt(4)\n\t" \
  "v_mfma_f32_16x16x128_f8f6f4 v[8:11], a[160:167], v[144:151], v[8:11] cbsz:0 blgp:1\n\t" \
  "ds_read_b64_tr_b8 v[128:129], v164 offset:49152\n\t" \
  "s_cmp_eq_u32 s8, -1\n\t" \
  "s_cbranch_scc0 L_au6_2\n\t" \
  "L_aur6_2:\n\t" \
  "s_add_u32 s99, s29, 24\n\t" \
  "s_min_u32 s99, s99, 31\n\t" \
  "v_readlane_b32 s99, v211, s99\n\t" \
  "s_mov_b32 exec_lo, s15\n\t" \
  "s_mov_b32 exec_hi, 0\n\t" \
  "s_cbranch_execz L_c24_2\n\t" \
  "s_nop 1\n\t" \
  "buffer_load_dword v210, v222, s[60:63], s99 offen\n\t" \
  "L_c24_2:\n\t" \
  "s_mov_b64 exec, -1\n\t" \
  "v_mfma_f32_16x16x128_f8f6f4 v[40:43], a[168:175], v[144:151], v[40:43] cbsz:0 blgp:1\n\t" \
  "ds_read_b64_tr_b8 v[130:131], v164 offset:50176\n\t" \
  "buffer_load_dwordx4 a[52:55], v189, s[36:39], s87 offen offset:1024 \n\t" \
  "v_mfma_f32_16x16x128_f8f6f4 v[72:75], a[176:183], v[144:151], v[72:75] cbsz:0 blgp:1\n\t" \
  "ds_read_b64_tr_b8 v[132:133], v164 offset:57344\n\t" \
  "v_mfma_f32_16x16x128_f8f6f4 v[104:107], a[184:191], v[144:151], v[104:107] cbsz:0 blgp:1\n\t" \
  "ds_read_b64_tr_b8 v[134:135], v164 offset:58368\n\t" \
  "s_waitcnt lgkmcnt(4)\n\t" \
  "v_mfma_f32_16x16x128_f8f6f4 v[12:15], a[160:167], v[152:159], v[12:15] cbsz:0 blgp:1\n\t" \
  "ds_read_b64_tr_b8 v[136:137], v165 offset:49152\n\t" \
  "v_mfma_f32_16x16x128_f8f6f4 v[44:47], a[168:175], v[152:159], v[44:47] cbsz:0 blgp:1\n\t" \
  "ds_read_b64_tr_b8 v[138:139], v165 offset:50176\n\t" \
  "v_mfma_f32_16x16x128_f8f6f4 v[76:79], a[176:183], v[152:159], v[76:79] cbsz:0 blgp:1\n\t" \
  "ds_read_b64_tr_b8 v[140:141], v165 offset:57344\n\t" \
  "v_mfma_f32_16x16x128_f8f6f4 v[108:111], a[184:191], v[152:159], v[108:111] cbsz:0 blgp:1\n\t" \
  "ds_read_b64_tr_b8 v[142:143], v165 offset:58368\n\t" \
  "s_waitcnt lgkmcnt(4)\n\t" \
  "v_mfma_f32_16x16x128_f8f6f4 v[16:19], a[160:167], v[128:135], v[16:19] cbsz:0 blgp:1\n\t" \
  "ds_read_b64_tr_b8 v[144:145], v166 offset:49152\n\t" \
  "v_mfma_f32_16x16x128_f8f6f4 v[48:51], a[168:175], v[128:135], v[48:51] cbsz:0 blgp:1\n\t" \
  "ds_read_b64_tr_b8 v[146:147], v166 offset:50176\n\t" \
  "v_mfma_f32_16x16x128_f8f6f4 v[80:83], a[176:183], v[128:135], v[80:83] cbsz:0 blgp:1\n\t" \
  "ds_read_b64_tr_b8 v[148:149], v166 offset:57344\n\t" \
  "v_mfma_f32_16x16x128_f8f6f4 v[112:115], a[184:191], v[128:135], v[112:115] cbsz:0 blgp:1\n\t" \
  "ds_read_b64_tr_b8 v[150:151], v166 offset:58368\n\t" \
  "s_waitcnt lgkmcnt(4)\n\t" \
  "v_mfma_f32_16x16x128_f8f6f4 v[20:23], a[160:167], v[136:143], v[20:23] cbsz:0 blgp:1\n\t" \
  "ds_read_b64_tr_b8 v[152:153], v167 offset:49152\n\t" \
  "s_cmp_eq_u32 s8, -1\n\t" \
  "s_cbranch_scc0 L_ap7_2\n\t" \
  "L_apr7_2:\n\t" \
  "v_mfma_f32_16x16x128_f8f6f4 v[52:55], a[168:175], v[136:143], v[52:55] cbsz:0 blgp:1\n\t" \
  "ds_read_b64_tr_b8 v[154:155], v167 offset:50176\n\t" \
  "v_mfma_f32_16x16x128_f8f6f4 v[84:87], a[176:183], v[136:143], v[84:87] cbsz:0 blgp:1\n\t" \
  "ds_read_b64_tr_b8 v[156:157], v167 offset:57344\n\t" \
  "v_mfma_f32_16x16x128_f8f6f4 v[116:119], a[184:191], v[136:143], v[116:119] cbsz:0 blgp:1\n\t" \
  "ds_read_b64_tr_b8 v[158:159], v167 offset:58368\n\t" \
  "s_waitcnt lgkmcnt(4)\n\t" \
  "v_mfma_f32_16x16x128_f8f6f4 v[24:27], a[160:167], v[144:151], v[24:27] cbsz:0 blgp:1\n\t" \
  "s_waitcnt vmcnt(36)\n\t" \
  "s_barrier\n\t" \
  "ds_read_b64_tr_b8 v[128:129], v168 offset:0\n\t" \
  "v_mfma_f32_16x16x128_f8f6f4 v[56:59], a[168:175], v[144:151], v[56:59] cbsz:0 blgp:1\n\t" \
  "ds_read_b64_tr_b8 v[130:131], v168 offset:1024\n\t" \
  "v_mfma_f32_16x16x128_f8f6f4 v[88:91], a[176:183], v[144:151], v[88:91] cbsz:0 blgp:1\n\t" \
  "ds_read_b64_tr_b8 v[132:133], v168 offset:8192\n\t" \
  "v_mfma_f32_16x16x128_f8f6f4 v[120:123], a[184:191], v[144:151], v[120:123] cbsz:0 blgp:1\n\t" \
  "ds_read_b64_tr_b8 v[134:135], v168 offset:9216\n\t" \
  "s_waitcnt lgkmcnt(4)\n\t" \
  "v_mfma_f32_16x16x128_f8f6f4 v[28:31], a[160:167], v[152:159], v[28:31] cbsz:0 blgp:1\n\t" \
  "ds_read_b64_tr_b8 v[136:137], v169 offset:0\n\t" \
  "v_mfma_f32_16x16x128_f8f6f4 v[60:63], a[168:175], v[152:159], v[60:63] cbsz:0 blgp:1\n\t" \
  "ds_read_b64_tr_b8 v[138:139], v169 offset:1024\n\t" \
  "v_mfma_f32_16x16x128_f8f6f4 v[92:95], a[176:183], v[152:159], v[92:95] cbsz:0 blgp:1\n\t" \
  "ds_read_b64_tr_b8 v[140:141], v169 offset:8192\n\t" \
  "v_mfma_f32_16x16x128_f8f6f4 v[124:127], a[184:191], v[152:159], v[124:127] cbsz:0 blgp:1\n\t" \
  "ds_read_b64_tr_b8 v[142:143], v169 offset:9216\n\t" \
  "s_waitcnt lgkmcnt(4)\n\t" \
  "v_mfma_f32_16x16x128_f8f6f4 v[0:3], a[192:199], v[128:135], v[0:3] cbsz:0 blgp:1\n\t" \
  "ds_read_b64_tr_b8 v[144:145], v170 offset:0\n\t" \
  "v_mfma_f32_16x16x128_f8f6f4 v[32:35], a[200:207], v[128:135], v[32:35] cbsz:0 blgp:1\n\t" \
  "ds_read_b64_tr_b8 v[146:147], v170 offset:1024\n\t" \
  "v_mfma_f32_16x16x128_f8f6f4 v[64:67], a[208:215], v[128:135], v[64:67] cbsz:0 blgp:1\n\t" \
  "ds_read_b64_tr_b8 v[148:149], v170 offset:8192\n\t" \
  "s_waitcnt vmcnt(30)\n\t" \
  "v_mfma_f32_16x16x128_f8f6f4 v[96:99], a[216:223], v[128:135], v[96:99] cbsz:0 blgp:1\n\t" \
  "ds_read_b64_tr_b8 v[150:151], v170 offset:9216\n\t" \
  "s_waitcnt lgkmcnt(4)\n\t" \
  "v_mfma_f32_16x16x128_f8f6f4 v[4:7], a[192:199], v[136:143], v[4:7] cbsz:0 blgp:1\n\t" \
  "ds_read_b64_tr_b8 v[152:153], v171 offset:0\n\t" \
  "v_mfma_f32_16x16x128_f8f6f4 v[36:39], a[200:207], v[136:143], v[36:39] cbsz:0 blgp:1\n\t" \
  "ds_read_b64_tr_b8 v[154:155], v171 offset:1024\n\t" \
  "v_mfma_f32_16x16x128_f8f6f4 v[68:71], a[208:215], v[136:143], v[68:71] cbsz:0 blgp:1\n\t" \
  "ds_read_b64_tr_b8 v[156:157], v171 offset:8192\n\t" \
  "v_mfma_f32_16x16x128_f8f6f4 v[100:103], a[216:223], v[136:143], v[100:103] cbsz:0 blgp:1\n\t" \
  "ds_read_b64_tr_b8 v[158:159], v171 offset:9216\n\t" \
  "s_waitcnt lgkmcnt(4)\n\t" \
  "v_mfma_f32_16x16x128_f8f6f4 v[8:11], a[192:199], v[144:151], v[8:11] cbsz:0 blgp:1\n\t" \
  "ds_read_b64_tr_b8 v[128:129], v172 offset:0\n\t" \
  "s_cmp_eq_u32 s8, -1\n\t" \
  "s_cbranch_scc0 L_au7_2\n\t" \
  "L_aur7_2:\n\t" \
  "buffer_load_dwordx4 a[56:59], v189, s[36:39], s87 offen offset:2048 \n\t" \
  "v_mfma_f32_16x16x128_f8f6f4 v[40:43], a[200:207], v[144:151], v[40:43] cbsz:0 blgp:1\n\t" \
  "ds_read_b64_tr_b8 v[130:131], v172 offset:1024\n\t" \
  "v_mfma_f32_16x16x128_f8f6f4 v[72:75], a[208:215], v[144:151], v[72:75] cbsz:0 blgp:1\n\t" \
  "ds_read_b64_tr_b8 v[132:133], v172 offset:8192\n\t" \
  "buffer_load_dwordx4 a[60:63], v189, s[36:39], s87 offen offset:3072 \n\t" \
  "v_mfma_f32_16x16x128_f8f6f4 v[104:107], a[216:223], v[144:151], v[104:107] cbsz:0 blgp:1\n\t" \
  "ds_read_b64_tr_b8 v[134:135], v172 offset:9216\n\t" \
  "s_waitcnt lgkmcnt(4)\n\t" \
  "v_mfma_f32_16x16x128_f8f6f4 v[12:15], a[192:199], v[152:159], v[12:15] cbsz:0 blgp:1\n\t" \
  "ds_read_b64_tr_b8 v[136:137], v173 offset:0\n\t" \
  "s_bfe_u32 s84, s8, 262144\n\t" \
  "s_cmp_eq_u32 s84, 15\n\t" \
  "s_cbranch_scc0 L_q18_2\n\t" \
  "L_qr18_2:\n\t" \
  "s_mov_b32 s87, 917504\n\t" \
  "buffer_load_dwordx4 a[64:67], v188, s[36:39], s87 offen offset:0 \n\t" \
  "v_mfma_f32_16x16x128_f8f6f4 v[44:47], a[200:207], v[152:159], v[44:47] cbsz:0 blgp:1\n\t" \
  "ds_read_b64_tr_b8 v[138:139], v173 offset:1024\n\t" \
  "v_mfma_f32_16x16x128_f8f6f4 v[76:79], a[208:215], v[152:159], v[76:79] cbsz:0 blgp:1\n\t" \
  "ds_read_b64_tr_b8 v[140:141], v173 offset:8192\n\t" \
  "buffer_load_dwordx4 a[68:71], v188, s[36:39], s87 offen offset:1024 \n\t" \
  "v_mfma_f32_16x16x128_f8f6f4 v[108:111], a[216:223], v[152:159], v[108:111] cbsz:0 blgp:1\n\t" \
  "ds_read_b64_tr_b8 v[142:143], v173 offset:9216\n\t" \
  "s_waitcnt lgkmcnt(4)\n\t" \
  "v_mfma_f32_16x16x128_f8f6f4 v[16:19], a[192:199], v[128:135], v[16:19] cbsz:0 blgp:1\n\t" \
  "ds_read_b64_tr_b8 v[144:145], v174 offset:0\n\t" \
  "buffer_load_dwordx4 a[72:75], v188, s[36:39], s87 offen offset:2048 \n\t" \
  "v_mfma_f32_16x16x128_f8f6f4 v[48:51], a[200:207], v[128:135], v[48:51] cbsz:0 blgp:1\n\t" \
  "ds_read_b64_tr_b8 v[146:147], v174 offset:1024\n\t" \
  "v_mfma_f32_16x16x128_f8f6f4 v[80:83], a[208:215], v[128:135], v[80:83] cbsz:0 blgp:1\n\t" \
  "ds_read_b64_tr_b8 v[148:149], v174 offset:8192\n\t" \
  "buffer_load_dwordx4 a[76:79], v188, s[36:39], s87 offen offset:3072 \n\t" \
  "v_mfma_f32_16x16x128_f8f6f4 v[112:115], a[216:223], v[128:135], v[112:115] cbsz:0 blgp:1\n\t" \
  "ds_read_b64_tr_b8 v[150:151], v174 offset:9216\n\t" \
  "s_waitcnt lgkmcnt(4)\n\t" \
  "v_mfma_f32_16x16x128_f8f6f4 v[20:23], a[192:199], v[136:143], v[20:23] cbsz:0 blgp:1\n\t" \
  "ds_read_b64_tr_b8 v[152:153], v175 offset:0\n\t" \
  "s_cmp_eq_u32 s8, -1\n\t" \
  "s_cbranch_scc0 L_ap8_2\n\t" \
  "L_apr8_2:\n\t" \
  "buffer_load_dwordx4 a[80:83], v189, s[36:39], s87 offen offset:0 \n\t" \
  "v_mfma_f32_16x16x128_f8f6f4 v[52:55], a[200:207], v[136:143], v[52:55] cbsz:0 blgp:1\n\t" \
  "ds_read_b64_tr_b8 v[154:155], v175 offset:1024\n\t" \
  "v_mfma_f32_16x16x128_f8f6f4 v[84:87], a[208:215], v[136:143], v[84:87] cbsz:0 blgp:1\n\t" \
  "ds_read_b64_tr_b8 v[156:157], v175 offset:8192\n\t" \
  "buffer_load_dwordx4 a[84:87], v189, s[36:39], s87 offen offset:1024 \n\t" \
  "v_mfma_f32_16x16x128_f8f6f4 v[116:119], a[216:223], v[136:143], v[116:119] cbsz:0 blgp:1\n\t" \
  "ds_read_b64_tr_b8 v[158:159], v175 offset:9216\n\t" \
  "s_add_u32 m0, s77, 131072\n\t" \
  "s_add_u32 s90, s68, 4194304\n\t" \
  "buffer_load_dwordx4 v186, s[40:43], s90 offen offset:0  lds\n\t" \
  "s_waitcnt lgkmcnt(4)\n\t" \
  "v_mfma_f32_16x16x128_f8f6f4 v[24:27], a[192:199], v[144:151], v[24:27] cbsz:0 blgp:1\n\t" \
  "s_barrier\n\t" \
  "ds_read_b64_tr_b8 v[128:129], v168 offset:16384\n\t" \
  "v_mfma_f32_16x16x128_f8f6f4 v[56:59], a[200:207], v[144:151], v[56:59] cbsz:0 blgp:1\n\t" \
  "ds_read_b64_tr_b8 v[130:131], v168 offset:17408\n\t" \
  "s_add_u32 m0, s77, 131072\n\t" \
  "s_add_u32 s90, s68, 4194304\n\t" \
  "buffer_load_dwordx4 v187, s[40:43], s90 offen offset:1024  lds\n\t" \
  "v_mfma_f32_16x16x128_f8f6f4 v[88:91], a[208:215], v[144:151], v[88:91] cbsz:0 blgp:1\n\t" \
  "ds_read_b64_tr_b8 v[132:133], v168 offset:24576\n\t" \
  "s_add_u32 m0, s77, 147456\n\t" \
  "s_add_u32 s90, s69, 4194304\n\t" \
  "buffer_load_dwordx4 v186, s[40:43], s90 offen offset:0  lds\n\t" \
  "v_mfma_f32_16x16x128_f8f6f4 v[120:123], a[216:223], v[144:151], v[120:123] cbsz:0 blgp:1\n\t" \
  "ds_read_b64_tr_b8 v[134:135], v168 offset:25600\n\t" \
  "s_add_u32 m0, s77, 0\n\t" \
  "s_add_u32 s90, s70, 4194304\n\t" \
  "buffer_load_dwordx4 v186, s[40:43], s90 offen offset:0  lds\n\t" \
  "s_waitcnt lgkmcnt(4)\n\t" \
  "v_mfma_f32_16x16x128_f8f6f4 v[28:31], a[192:199], v[152:159], v[28:31] cbsz:0 blgp:1\n\t" \
  "ds_read_b64_tr_b8 v[136:137], v169 offset:16384\n\t" \
  "s_add_u32 m0, s77, 147456\n\t" \
  "s_add_u32 s90, s69, 4194304\n\t" \
  "buffer_load_dwordx4 v187, s[40:43], s90 offen offset:1024  lds\n\t" \
  "v_mfma_f32_16x16x128_f8f6f4 v[60:63], a[200:207], v[152:159], v[60:63] cbsz:0 blgp:1\n\t" \
  "ds_read_b64_tr_b8 v[138:139], v169 offset:17408\n\t" \
  "s_add_u32 m0, s77, 0\n\t" \
  "s_add_u32 s90, s70, 4194304\n\t" \
  "buffer_load_dwordx4 v187, s[40:43], s90 offen offset:1024  lds\n\t" \
  "v_mfma_f32_16x16x128_f8f6f4 v[92:95], a[208:215], v[152:159], v[92:95] cbsz:0 blgp:1\n\t" \
  "ds_read_b64_tr_b8 v[140:141], v169 offset:24576\n\t" \
  "s_add_u32 m0, s77, 16384\n\t" \
  "s_add_u32 s90, s71, 4194304\n\t" \
  "buffer_load_dwordx4 v186, s[40:43], s90 offen offset:0  lds\n\t" \
  "v_mfma_f32_16x16x128_f8f6f4 v[124:127], a[216:223], v[152:159], v[124:127] cbsz:0 blgp:1\n\t" \
  "ds_read_b64_tr_b8 v[142:143], v169 offset:25600\n\t" \
  "buffer_load_dwordx4 a[88:91], v189, s[36:39], s87 offen offset:2048 \n\t" \
  "s_waitcnt vmcnt(44) lgkmcnt(4)\n\t" \
  "v_mfma_f32_16x16x128_f8f6f4 v[0:3], a[224:231], v[128:135], v[0:3] cbsz:0 blgp:1\n\t" \
  "ds_read_b64_tr_b8 v[144:145], v170 offset:16384\n\t" \
  "s_add_u32 m0, s77, 16384\n\t" \
  "s_add_u32 s90, s71, 4194304\n\t" \
  "buffer_load_dwordx4 v187, s[40:43], s90 offen offset:1024  lds\n\t" \
  "s_waitcnt vmcnt(43)\n\t" \
  "v_mfma_f32_16x16x128_f8f6f4 v[32:35], a[232:239], v[128:135], v[32:35] cbsz:0 blgp:1\n\t" \
  "ds_read_b64_tr_b8 v[146:147], v170 offset:17408\n\t" \
  "buffer_load_dwordx4 a[92:95], v189, s[36:39], s87 offen offset:3072 \n\t" \
  "s_waitcnt vmcnt(42)\n\t" \
  "v_mfma_f32_16x16x128_f8f6f4 v[64:67], a[240:247], v[128:135], v[64:67] cbsz:0 blgp:1\n\t" \
  "ds_read_b64_tr_b8 v[148:149], v170 offset:24576\n\t" \
  "s_waitcnt vmcnt(40)\n\t" \
  "v_mfma_f32_16x16x128_f8f6f4 v[96:99], a[248:255], v[128:135], v[96:99] cbsz:0 blgp:1\n\t" \
  "ds_read_b64_tr_b8 v[150:151], v170 offset:25600\n\t" \
  "s_bfe_u32 s84, s8, 262148\n\t" \
  "s_cmp_eq_u32 s84, 15\n\t" \
  "s_cbranch_scc0 L_q19_2\n\t" \
  "L_qr19_2:\n\t" \
  "s_mov_b32 s87, 983040\n\t" \
  "buffer_load_dwordx4 a[96:99], v188, s[36:39], s87 offen offset:0 \n\t" \
  "s_waitcnt lgkmcnt(4)\n\t" \
  "v_mfma_f32_16x16x128_f8f6f4 v[4:7], a[224:231], v[136:143], v[4:7] cbsz:0 blgp:1\n\t" \
  "ds_read_b64_tr_b8 v[152:153], v171 offset:16384\n\t" \
  "v_mfma_f32_16x16x128_f8f6f4 v[36:39], a[232:239], v[136:143], v[36:39] cbsz:0 blgp:1\n\t" \
  "ds_read_b64_tr_b8 v[154:155], v171 offset:17408\n\t" \
  "buffer_load_dwordx4 a[100:103], v188, s[36:39], s87 offen offset:1024 \n\t" \
  "v_mfma_f32_16x16x128_f8f6f4 v[68:71], a[240:247], v[136:143], v[68:71] cbsz:0 blgp:1\n\t" \
  "ds_read_b64_tr_b8 v[156:157], v171 offset:24576\n\t" \
  "v_mfma_f32_16x16x128_f8f6f4 v[100:103], a[248:255], v[136:143], v[100:103] cbsz:0 blgp:1\n\t" \
  "ds_read_b64_tr_b8 v[158:159], v171 offset:25600\n\t" \
  "buffer_load_dwordx4 a[104:107], v188, s[36:39], s87 offen offset:2048 \n\t" \
  "s_waitcnt lgkmcnt(4)\n\t" \
  "v_mfma_f32_16x16x128_f8f6f4 v[8:11], a[224:231], v[144:151], v[8:11] cbsz:0 blgp:1\n\t" \
  "ds_read_b64_tr_b8 v[128:129], v172 offset:16384\n\t" \
  "s_cmp_eq_u32 s8, -1\n\t" \
  "s_cbranch_scc0 L_au8_2\n\t" \
  "L_aur8_2:\n\t" \
  "v_mfma_f32_16x16x128_f8f6f4 v[40:43], a[232:239], v[144:151], v[40:43] cbsz:0 blgp:1\n\t" \
  "ds_read_b64_tr_b8 v[130:131], v172 offset:17408\n\t" \
  "buffer_load_dwordx4 a[108:111], v188, s[36:39], s87 offen offset:3072 \n\t" \
  "v_mfma_f32_16x16x128_f8f6f4 v[72:75], a[240:247], v[144:151], v[72:75] cbsz:0 blgp:1\n\t" \
  "ds_read_b64_tr_b8 v[132:133], v172 offset:24576\n\t" \
  "v_mfma_f32_16x16x128_f8f6f4 v[104:107], a[248:255], v[144:151], v[104:107] cbsz:0 blgp:1\n\t" \
  "ds_read_b64_tr_b8 v[134:135], v172 offset:25600\n\t" \
  "buffer_load_dwordx4 a[112:115], v189, s[36:39], s87 offen offset:0 \n\t" \
  "s_waitcnt lgkmcnt(4)\n\t" \
  "v_mfma_f32_16x16x128_f8f6f4 v[12:15], a[224:231], v[152:159], v[12:15] cbsz:0 blgp:1\n\t" \
  "ds_read_b64_tr_b8 v[136:137], v173 offset:16384\n\t" \
  "v_mfma_f32_16x16x128_f8f6f4 v[44:47], a[232:239], v[152:159], v[44:47] cbsz:0 blgp:1\n\t" \
  "ds_read_b64_tr_b8 v[138:139], v173 offset:17408\n\t" \
  "buffer_load_dwordx4 a[116:119], v189, s[36:39], s87 offen offset:1024 \n\t" \
  "v_mfma_f32_16x16x128_f8f6f4 v[76:79], a[240:247], v[152:159], v[76:79] cbsz:0 blgp:1\n\t" \
  "ds_read_b64_tr_b8 v[140:141], v173 offset:24576\n\t" \
  "v_mfma_f32_16x16x128_f8f6f4 v[108:111], a[248:255], v[152:159], v[108:111] cbsz:0 blgp:1\n\t" \
  "ds_read_b64_tr_b8 v[142:143], v173 offset:25600\n\t" \
  "buffer_load_dwordx4 a[120:123], v189, s[36:39], s87 offen offset:2048 \n\t" \
  "s_waitcnt lgkmcnt(4)\n\t" \
  "v_mfma_f32_16x16x128_f8f6f4 v[16:19], a[224:231], v[128:135], v[16:19] cbsz:0 blgp:1\n\t" \
  "ds_read_b64_tr_b8 v[144:145], v174 offset:16384\n\t" \
  "v_mfma_f32_16x16x128_f8f6f4 v[48:51], a[232:239], v[128:135], v[48:51] cbsz:0 blgp:1\n\t" \
  "ds_read_b64_tr_b8 v[146:147], v174 offset:17408\n\t" \
  "buffer_load_dwordx4 a[124:127], v189, s[36:39], s87 offen offset:3072 \n\t" \
  "v_mfma_f32_16x16x128_f8f6f4 v[80:83], a[240:247], v[128:135], v[80:83] cbsz:0 blgp:1\n\t" \
  "ds_read_b64_tr_b8 v[148:149], v174 offset:24576\n\t" \
  "v_mfma_f32_16x16x128_f8f6f4 v[112:115], a[248:255], v[128:135], v[112:115] cbsz:0 blgp:1\n\t" \
  "ds_read_b64_tr_b8 v[150:151], v174 offset:25600\n\t" \
  "s_bfe_u32 s84, s8, 262152\n\t" \
  "s_cmp_eq_u32 s84, 15\n\t" \
  "s_cbranch_scc0 L_q20_2\n\t" \
  "L_qr20_2:\n\t" \
  "s_mov_b32 s87, 1048576\n\t" \
  "buffer_load_dwordx4 a[128:131], v188, s[36:39], s87 offen offset:0 \n\t" \
  "s_waitcnt lgkmcnt(4)\n\t" \
  "v_mfma_f32_16x16x128_f8f6f4 v[20:23], a[224:231], v[136:143], v[20:23] cbsz:0 blgp:1\n\t" \
  "ds_read_b64_tr_b8 v[152:153], v175 offset:16384\n\t" \
  "v_mfma_f32_16x16x128_f8f6f4 v[52:55], a[232:239], v[136:143], v[52:55] cbsz:0 blgp:1\n\t" \
  "ds_read_b64_tr_b8 v[154:155], v175 offset:17408\n\t" \
  "buffer_load_dwordx4 a[132:135], v188, s[36:39], s87 offen offset:1024 \n\t" \
  "v_mfma_f32_16x16x128_f8f6f4 v[84:87], a[240:247], v[136:143], v[84:87] cbsz:0 blgp:1\n\t" \
  "ds_read_b64_tr_b8 v[156:157], v175 offset:24576\n\t" \
  "s_mov_b32 s9, 0\n\t" \
  "v_mfma_f32_16x16x128_f8f6f4 v[116:119], a[248:255], v[136:143], v[116:119] cbsz:0 blgp:1\n\t" \
  "ds_read_b64_tr_b8 v[158:159], v175 offset:25600\n\t" \
  "global_load_dword v196, v195, s[26:27] offset:3072 sc1\n\t" \
  "buffer_load_dwordx4 a[136:139], v188, s[36:39], s87 offen offset:2048 \n\t" \
  "s_waitcnt lgkmcnt(4)\n\t" \
  "v_mfma_f32_16x16x128_f8f6f4 v[24:27], a[224:231], v[144:151], v[24:27] cbsz:0 blgp:1\n\t" \
  "s_barrier\n\t" \
  "ds_read_b64_tr_b8 v[128:129], v168 offset:32768\n\t" \
  "v_mfma_f32_16x16x128_f8f6f4 v[56:59], a[232:239], v[144:151], v[56:59] cbsz:0 blgp:1\n\t" \
  "ds_read_b64_tr_b8 v[130:131], v168 offset:33792\n\t" \
  "buffer_load_dwordx4 a[140:143], v188, s[36:39], s87 offen offset:3072 \n\t" \
  "v_mfma_f32_16x16x128_f8f6f4 v[88:91], a[240:247], v[144:151], v[88:91] cbsz:0 blgp:1\n\t" \
  "ds_read_b64_tr_b8 v[132:133], v168 offset:40960\n\t" \
  "v_mfma_f32_16x16x128_f8f6f4 v[120:123], a[248:255], v[144:151], v[120:123] cbsz:0 blgp:1\n\t" \
  "ds_read_b64_tr_b8 v[134:135], v168 offset:41984\n\t" \
  "buffer_load_dwordx4 a[144:147], v189, s[36:39], s87 offen offset:0 \n\t" \
  "s_waitcnt lgkmcnt(4)\n\t" \
  "v_mfma_f32_16x16x128_f8f6f4 v[28:31], a[224:231], v[152:159], v[28:31] cbsz:0 blgp:1\n\t" \
  "ds_read_b64_tr_b8 v[136:137], v169 offset:32768\n\t" \
  "v_mfma_f32_16x16x128_f8f6f4 v[60:63], a[232:239], v[152:159], v[60:63] cbsz:0 blgp:1\n\t" \
  "ds_read_b64_tr_b8 v[138:139], v169 offset:33792\n\t" \
  "buffer_load_dwordx4 a[148:151], v189, s[36:39], s87 offen offset:1024 \n\t" \
  "v_mfma_f32_16x16x128_f8f6f4 v[92:95], a[240:247], v[152:159], v[92:95] cbsz:0 blgp:1\n\t" \
  "ds_read_b64_tr_b8 v[140:141], v169 offset:40960\n\t" \
  "v_mfma_f32_16x16x128_f8f6f4 v[124:127], a[248:255], v[152:159], v[124:127] cbsz:0 blgp:1\n\t" \
  "ds_read_b64_tr_b8 v[142:143], v169 offset:41984\n\t" \
  "buffer_load_dwordx4 a[152:155], v189, s[36:39], s87 offen offset:2048 \n\t" \
  "s_waitcnt vmcnt(54) lgkmcnt(4)\n\t" \
  "v_mfma_f32_16x16x128_f8f6f4 v[0:3], a[0:7], v[128:135], v[0:3] cbsz:0 blgp:1\n\t" \
  "ds_read_b64_tr_b8 v[144:145], v170 offset:32768\n\t" \
  "s_waitcnt vmcnt(52)\n\t" \
  "v_mfma_f32_16x16x128_f8f6f4 v[32:35], a[8:15], v[128:135], v[32:35] cbsz:0 blgp:1\n\t" \
  "ds_read_b64_tr_b8 v[146:147], v170 offset:33792\n\t" \
  "buffer_load_dwordx4 a[156:159], v189, s[36:39], s87 offen offset:3072 \n\t" \
  "s_waitcnt vmcnt(51)\n\t" \
  "v_mfma_f32_16x16x128_f8f6f4 v[64:67], a[16:23], v[128:135], v[64:67] cbsz:0 blgp:1\n\t" \
  "ds_read_b64_tr_b8 v[148:149], v170 offset:40960\n\t" \
  "s_waitcnt vmcnt(49)\n\t" \
  "v_mfma_f32_16x16x128_f8f6f4 v[96:99], a[24:31], v[128:135], v[96:99] cbsz:0 blgp:1\n\t" \
  "ds_read_b64_tr_b8 v[150:151], v170 offset:41984\n\t" \
  "s_bfe_u32 s84, s8, 262156\n\t" \
  "s_cmp_eq_u32 s84, 15\n\t" \
  "s_cbranch_scc0 L_q21_2\n\t" \
  "L_qr21_2:\n\t" \
  "s_mov_b32 s87, 1114112\n\t" \
  "buffer_load_dwordx4 a[160:163], v188, s[36:39], s87 offen offset:0 \n\t" \
  "s_waitcnt lgkmcnt(4)\n\t" \
  "v_mfma_f32_16x16x128_f8f6f4 v[4:7], a[0:7], v[136:143], v[4:7] cbsz:0 blgp:1\n\t" \
  "ds_read_b64_tr_b8 v[152:153], v171 offset:32768\n\t" \
  "v_mfma_f32_16x16x128_f8f6f4 v[36:39], a[8:15], v[136:143], v[36:39] cbsz:0 blgp:1\n\t" \
  "ds_read_b64_tr_b8 v[154:155], v171 offset:33792\n\t" \
  "buffer_load_dwordx4 a[164:167], v188, s[36:39], s87 offen offset:1024 \n\t" \
  "v_mfma_f32_16x16x128_f8f6f4 v[68:71], a[16:23], v[136:143], v[68:71] cbsz:0 blgp:1\n\t" \
  "ds_read_b64_tr_b8 v[156:157], v171 offset:40960\n\t" \
  "v_mfma_f32_16x16x128_f8f6f4 v[100:103], a[24:31], v[136:143], v[100:103] cbsz:0 blgp:1\n\t" \
  "ds_read_b64_tr_b8 v[158:159], v171 offset:41984\n\t" \
  "buffer_load_dwordx4 a[168:171], v188, s[36:39], s87 offen offset:2048 \n\t" \
  "s_waitcnt lgkmcnt(4)\n\t" \
  "v_mfma_f32_16x16x128_f8f6f4 v[8:11], a[0:7], v[144:151], v[8:11] cbsz:0 blgp:1\n\t" \
  "ds_read_b64_tr_b8 v[128:129], v172 offset:32768\n\t" \
  "v_mfma_f32_16x16x128_f8f6f4 v[40:43], a[8:15], v[144:151], v[40:43] cbsz:0 blgp:1\n\t" \
  "ds_read_b64_tr_b8 v[130:131], v172 offset:33792\n\t" \
  "buffer_load_dwordx4 a[172:175], v188, s[36:39], s87 offen offset:3072 \n\t" \
  "v_mfma_f32_16x16x128_f8f6f4 v[72:75], a[16:23], v[144:151], v[72:75] cbsz:0 blgp:1\n\t" \
  "ds_read_b64_tr_b8 v[132:133], v172 offset:40960\n\t" \
  "v_mfma_f32_16x16x128_f8f6f4 v[104:107], a[24:31], v[144:151], v[104:107] cbsz:0 blgp:1\n\t" \
  "ds_read_b64_tr_b8 v[134:135], v172 offset:41984\n\t" \
  "s_waitcnt vmcnt(10)\n\t" \
  "v_cmp_le_u32 vcc, s79, v196\n\t" \
  "s_nop 4\n\t" \
  "s_andn2_b32 s85, vcc_lo, s9\n\t" \
  "s_or_b32 s9, s9, vcc_lo\n\t" \
  "s_and_b32 exec_lo, s85, 0xfffffff\n\t" \
  "s_mov_b32 exec_hi, 0\n\t" \
  "s_cbranch_execz L_u4_2\n\t" \
  "s_mov_b32 s84, 1376256\n\t" \
  "buffer_load_dword v210, v198, s[36:39], s84 offen \n\t" \
  "L_u4_2:\n\t" \
  "s_mov_b64 exec, -1\n\t" \
  "buffer_load_dwordx4 a[176:179], v189, s[36:39], s87 offen offset:0 \n\t" \
  "s_waitcnt lgkmcnt(4)\n\t" \
  "v_mfma_f32_16x16x128_f8f6f4 v[12:15], a[0:7], v[152:159], v[12:15] cbsz:0 blgp:1\n\t" \
  "ds_read_b64_tr_b8 v[136:137], v173 offset:32768\n\t" \
  "s_add_u32 m0, s76, 32768\n\t" \
  "s_add_u32 s90, s72, 4194304\n\t" \
  "buffer_load_dwordx4 v184, s[40:43], s90 offen offset:0  lds\n\t" \
  "v_mfma_f32_16x16x128_f8f6f4 v[44:47], a[8:15], v[152:159], v[44:47] cbsz:0 blgp:1\n\t" \
  "ds_read_b64_tr_b8 v[138:139], v173 offset:33792\n\t" \
  "s_add_u32 m0, s76, 32768\n\t" \
  "s_add_u32 s90, s72, 4194304\n\t" \
  "buffer_load_dwordx4 v185, s[40:43], s90 offen offset:1024  lds\n\t" \
  "v_mfma_f32_16x16x128_f8f6f4 v[76:79], a[16:23], v[152:159], v[76:79] cbsz:0 blgp:1\n\t" \
  "ds_read_b64_tr_b8 v[140:141], v173 offset:40960\n\t" \
  "s_add_u32 m0, s76, 49152\n\t" \
  "s_add_u32 s90, s73, 4194304\n\t" \
  "buffer_load_dwordx4 v184, s[40:43], s90 offen offset:0  lds\n\t" \
  "v_mfma_f32_16x16x128_f8f6f4 v[108:111], a[24:31], v[152:159], v[108:111] cbsz:0 blgp:1\n\t" \
  "ds_read_b64_tr_b8 v[142:143], v173 offset:41984\n\t" \
  "s_add_u32 m0, s76, 49152\n\t" \
  "s_add_u32 s90, s73, 4194304\n\t" \
  "buffer_load_dwordx4 v185, s[40:43], s90 offen offset:1024  lds\n\t" \
  "s_waitcnt lgkmcnt(4)\n\t" \
  "v_mfma_f32_16x16x128_f8f6f4 v[16:19], a[0:7], v[128:135], v[16:19] cbsz:0 blgp:1\n\t" \
  "ds_read_b64_tr_b8 v[144:145], v174 offset:32768\n\t" \
  "s_add_u32 m0, s76, 65536\n\t" \
  "s_add_u32 s90, s74, 4194304\n\t" \
  "buffer_load_dwordx4 v184, s[40:43], s90 offen offset:0  lds\n\t" \
  "v_mfma_f32_16x16x128_f8f6f4 v[48:51], a[8:15], v[128:135], v[48:51] cbsz:0 blgp:1\n\t" \
  "ds_read_b64_tr_b8 v[146:147], v174 offset:33792\n\t" \
  "s_add_u32 m0, s76, 65536\n\t" \
  "s_add_u32 s90, s74, 4194304\n\t" \
  "buffer_load_dwordx4 v185, s[40:43], s90 offen offset:1024  lds\n\t" \
  "v_mfma_f32_16x16x128_f8f6f4 v[80:83], a[16:23], v[128:135], v[80:83] cbsz:0 blgp:1\n\t" \
  "ds_read_b64_tr_b8 v[148:149], v174 offset:40960\n\t" \
  "s_add_u32 s90, s68, 6291456\n\t" \
  "s_mov_b64 exec, 0xffff\n\t" \
  "buffer_load_dword v210, v241, s[40:43], s90 offen\n\t" \
  "s_mov_b64 exec, -1\n\t" \
  "v_mfma_f32_16x16x128_f8f6f4 v[112:115], a[24:31], v[128:135], v[112:115] cbsz:0 blgp:1\n\t" \
  "ds_read_b64_tr_b8 v[150:151], v174 offset:41984\n\t" \
  "s_add_u32 s99, s29, 28\n\t" \
  "s_min_u32 s99, s99, 31\n\t" \
  "v_readlane_b32 s99, v211, s99\n\t" \
  "s_mov_b32 exec_lo, s15\n\t" \
  "s_mov_b32 exec_hi, 0\n\t" \
  "s_cbranch_execz L_c28_2\n\t" \
  "s_nop 1\n\t" \
  "buffer_load_dword v210, v222, s[60:63], s99 offen\n\t" \
  "L_c28_2:\n\t" \
  "s_mov_b64 exec, -1\n\t" \
  "s_waitcnt lgkmcnt(4)\n\t" \
  "v_mfma_f32_16x16x128_f8f6f4 v[20:23], a[0:7], v[136:143], v[20:23] cbsz:0 blgp:1\n\t" \
  "ds_read_b64_tr_b8 v[152:153], v175 offset:32768\n\t" \
  "buffer_load_dwordx4 a[180:183], v189, s[36:39], s87 offen offset:1024 \n\t" \
  "v_mfma_f32_16x16x128_f8f6f4 v[52:55], a[8:15], v[136:143], v[52:55] cbsz:0 blgp:1\n\t" \
  "ds_read_b64_tr_b8 v[154:155], v175 offset:33792\n\t" \
  "v_mfma_f32_16x16x128_f8f6f4 v[84:87], a[16:23], v[136:143], v[84:87] cbsz:0 blgp:1\n\t" \
  "ds_read_b64_tr_b8 v[156:157], v175 offset:40960\n\t" \
  "v_mfma_f32_16x16x128_f8f6f4 v[116:119], a[24:31], v[136:143], v[116:119] cbsz:0 blgp:1\n\t" \
  "ds_read_b64_tr_b8 v[158:159], v175 offset:41984\n\t" \
  "s_cmp_eq_u32 s9, -1\n\t" \
  "s_cbranch_scc0 L_ap10_2\n\t" \
  "L_apr10_2:\n\t" \
  "s_waitcnt lgkmcnt(4)\n\t" \
  "v_mfma_f32_16x16x128_f8f6f4 v[24:27], a[0:7], v[144:151], v[24:27] cbsz:0 blgp:1\n\t" \
  "s_barrier\n\t" \
  "ds_read_b64_tr_b8 v[128:129], v168 offset:49152\n\t" \
  "v_mfma_f32_16x16x128_f8f6f4 v[56:59], a[8:15], v[144:151], v[56:59] cbsz:0 blgp:1\n\t" \
  "ds_read_b64_tr_b8 v[130:131], v168 offset:50176\n\t" \
  "v_mfma_f32_16x16x128_f8f6f4 v[88:91], a[16:23], v[144:151], v[88:91] cbsz:0 blgp:1\n\t" \
  "ds_read_b64_tr_b8 v[132:133], v168 offset:57344\n\t" \
  "v_mfma_f32_16x16x128_f8f6f4 v[120:123], a[24:31], v[144:151], v[120:123] cbsz:0 blgp:1\n\t" \
  "ds_read_b64_tr_b8 v[134:135], v168 offset:58368\n\t" \
  "s_waitcnt lgkmcnt(4)\n\t" \
  "v_mfma_f32_16x16x128_f8f6f4 v[28:31], a[0:7], v[152:159], v[28:31] cbsz:0 blgp:1\n\t" \
  "ds_read_b64_tr_b8 v[136:137], v169 offset:49152\n\t" \
  "v_mfma_f32_16x16x128_f8f6f4 v[60:63], a[8:15], v[152:159], v[60:63] cbsz:0 blgp:1\n\t" \
  "ds_read_b64_tr_b8 v[138:139], v169 offset:50176\n\t" \
  "v_mfma_f32_16x16x128_f8f6f4 v[92:95], a[16:23], v[152:159], v[92:95] cbsz:0 blgp:1\n\t" \
  "ds_read_b64_tr_b8 v[140:141], v169 offset:57344\n\t" \
  "v_mfma_f32_16x16x128_f8f6f4 v[124:127], a[24:31], v[152:159], v[124:127] cbsz:0 blgp:1\n\t" \
  "ds_read_b64_tr_b8 v[142:143], v169 offset:58368\n\t" \
  "s_waitcnt lgkmcnt(4)\n\t" \
  "v_mfma_f32_16x16x128_f8f6f4 v[0:3], a[32:39], v[128:135], v[0:3] cbsz:0 blgp:1\n\t" \
  "ds_read_b64_tr_b8 v[144:145], v170 offset:49152\n\t" \
  "v_mfma_f32_16x16x128_f8f6f4 v[32:35], a[40:47], v[128:135], v[32:35] cbsz:0 blgp:1\n\t" \
  "ds_read_b64_tr_b8 v[146:147], v170 offset:50176\n\t" \
  "v_mfma_f32_16x16x128_f8f6f4 v[64:67], a[48:55], v[128:135], v[64:67] cbsz:0 blgp:1\n\t" \
  "ds_read_b64_tr_b8 v[148:149], v170 offset:57344\n\t" \
  "v_mfma_f32_16x16x128_f8f6f4 v[96:99], a[56:63], v[128:135], v[96:99] cbsz:0 blgp:1\n\t" \
  "ds_read_b64_tr_b8 v[150:151], v170 offset:58368\n\t" \
  "s_waitcnt lgkmcnt(4)\n\t" \
  "v_mfma_f32_16x16x128_f8f6f4 v[4:7], a[32:39], v[136:143], v[4:7] cbsz:0 blgp:1\n\t" \
  "ds_read_b64_tr_b8 v[152:153], v171 offset:49152\n\t" \
  "v_mfma_f32_16x16x128_f8f6f4 v[36:39], a[40:47], v[136:143], v[36:39] cbsz:0 blgp:1\n\t" \
  "ds_read_b64_tr_b8 v[154:155], v171 offset:50176\n\t" \
  "v_mfma_f32_16x16x128_f8f6f4 v[68:71], a[48:55], v[136:143], v[68:71] cbsz:0 blgp:1\n\t" \
  "ds_read_b64_tr_b8 v[156:157], v171 offset:57344\n\t" \
  "v_mfma_f32_16x16x128_f8f6f4 v[100:103], a[56:63], v[136:143], v[100:103] cbsz:0 blgp:1\n\t" \
  "ds_read_b64_tr_b8 v[158:159], v171 offset:58368\n\t" \
  "s_waitcnt lgkmcnt(4)\n\t" \
  "v_mfma_f32_16x16x128_f8f6f4 v[8:11], a[32:39], v[144:151], v[8:11] cbsz:0 blgp:1\n\t" \
  "ds_read_b64_tr_b8 v[128:129], v172 offset:49152\n\t" \
  "v_mfma_f32_16x16x128_f8f6f4 v[40:43], a[40:47], v[144:151], v[40:43] cbsz:0 blgp:1\n\t" \
  "ds_read_b64_tr_b8 v[130:131], v172 offset:50176\n\t" \
  "v_mfma_f32_16x16x128_f8f6f4 v[72:75], a[48:55], v[144:151], v[72:75] cbsz:0 blgp:1\n\t" \
  "ds_read_b64_tr_b8 v[132:133], v172 offset:57344\n\t" \
  "v_mfma_f32_16x16x128_f8f6f4 v[104:107], a[56:63], v[144:151], v[104:107] cbsz:0 blgp:1\n\t" \
  "ds_read_b64_tr_b8 v[134:135], v172 offset:58368\n\t" \
  "s_cmp_eq_u32 s9, -1\n\t" \
  "s_cbranch_scc0 L_au10_2\n\t" \
  "L_aur10_2:\n\t" \
  "s_waitcnt lgkmcnt(4)\n\t" \
  "v_mfma_f32_16x16x128_f8f6f4 v[12:15], a[32:39], v[152:159], v[12:15] cbsz:0 blgp:1\n\t" \
  "ds_read_b64_tr_b8 v[136:137], v173 offset:49152\n\t" \
  "v_mfma_f32_16x16x128_f8f6f4 v[44:47], a[40:47], v[152:159], v[44:47] cbsz:0 blgp:1\n\t" \
  "ds_read_b64_tr_b8 v[138:139], v173 offset:50176\n\t" \
  "v_mfma_f32_16x16x128_f8f6f4 v[76:79], a[48:55], v[152:159], v[76:79] cbsz:0 blgp:1\n\t" \
  "ds_read_b64_tr_b8 v[140:141], v173 offset:57344\n\t" \
  "v_mfma_f32_16x16x128_f8f6f4 v[108:111], a[56:63], v[152:159], v[108:111] cbsz:0 blgp:1\n\t" \
  "ds_read_b64_tr_b8 v[142:143], v173 offset:58368\n\t" \
  "s_waitcnt lgkmcnt(4)\n\t" \
  "v_mfma_f32_16x16x128_f8f6f4 v[16:19], a[32:39], v[128:135], v[16:19] cbsz:0 blgp:1\n\t" \
  "ds_read_b64_tr_b8 v[144:145], v174 offset:49152\n\t" \
  "v_mfma_f32_16x16x128_f8f6f4 v[48:51], a[40:47], v[128:135], v[48:51] cbsz:0 blgp:1\n\t" \
  "ds_read_b64_tr_b8 v[146:147], v174 offset:50176\n\t" \
  "v_mfma_f32_16x16x128_f8f6f4 v[80:83], a[48:55], v[128:135], v[80:83] cbsz:0 blgp:1\n\t" \
  "ds_read_b64_tr_b8 v[148:149], v174 offset:57344\n\t" \
  "v_mfma_f32_16x16x128_f8f6f4 v[112:115], a[56:63], v[128:135], v[112:115] cbsz:0 blgp:1\n\t" \
  "ds_read_b64_tr_b8 v[150:151], v174 offset:58368\n\t" \
  "buffer_load_dwordx4 a[184:187], v189, s[36:39], s87 offen offset:2048 \n\t" \
  "s_waitcnt lgkmcnt(4)\n\t" \
  "v_mfma_f32_16x16x128_f8f6f4 v[20:23], a[32:39], v[136:143], v[20:23] cbsz:0 blgp:1\n\t" \
  "ds_read_b64_tr_b8 v[152:153], v175 offset:49152\n\t" \
  "v_mfma_f32_16x16x128_f8f6f4 v[52:55], a[40:47], v[136:143], v[52:55] cbsz:0 blgp:1\n\t" \
  "ds_read_b64_tr_b8 v[154:155], v175 offset:50176\n\t" \
  "buffer_load_dwordx4 a[188:191], v189, s[36:39], s87 offen offset:3072 \n\t" \
  "v_mfma_f32_16x16x128_f8f6f4 v[84:87], a[48:55], v[136:143], v[84:87] cbsz:0 blgp:1\n\t" \
  "ds_read_b64_tr_b8 v[156:157], v175 offset:57344\n\t" \
  "v_mfma_f32_16x16x128_f8f6f4 v[116:119], a[56:63], v[136:143], v[116:119] cbsz:0 blgp:1\n\t" \
  "ds_read_b64_tr_b8 v[158:159], v175 offset:58368\n\t" \
  "s_cmp_eq_u32 s9, -1\n\t" \
  "s_cbranch_scc0 L_ap11_2\n\t" \
  "L_apr11_2:\n\t" \
  "s_bfe_u32 s84, s8, 262160\n\t" \
  "s_cmp_eq_u32 s84, 15\n\t" \
  "s_cbranch_scc0 L_q22_2\n\t" \
  "L_qr22_2:\n\t" \
  "s_mov_b32 s87, 1179648\n\t" \
  "buffer_load_dwordx4 a[192:195], v188, s[36:39], s87 offen offset:0 \n\t" \
  "s_waitcnt lgkmcnt(4)\n\t" \
  "v_mfma_f32_16x16x128_f8f6f4 v[24:27], a[32:39], v[144:151], v[24:27] cbsz:0 blgp:1\n\t" \
  "s_barrier\n\t" \
  "ds_read_b64_tr_b8 v[128:129], v176 offset:0\n\t" \
  "v_mfma_f32_16x16x128_f8f6f4 v[56:59], a[40:47], v[144:151], v[56:59] cbsz:0 blgp:1\n\t" \
  "ds_read_b64_tr_b8 v[130:131], v176 offset:1024\n\t" \
  "buffer_load_dwordx4 a[196:199], v188, s[36:39], s87 offen offset:1024 \n\t" \
  "v_mfma_f32_16x16x128_f8f6f4 v[88:91], a[48:55], v[144:151], v[88:91] cbsz:0 blgp:1\n\t" \
  "ds_read_b64_tr_b8 v[132:133], v176 offset:8192\n\t" \
  "v_mfma_f32_16x16x128_f8f6f4 v[120:123], a[56:63], v[144:151], v[120:123] cbsz:0 blgp:1\n\t" \
  "ds_read_b64_tr_b8 v[134:135], v176 offset:9216\n\t" \
  "buffer_load_dwordx4 a[200:203], v188, s[36:39], s87 offen offset:2048 \n\t" \
  "s_waitcnt lgkmcnt(4)\n\t" \
  "v_mfma_f32_16x16x128_f8f6f4 v[28:31], a[32:39], v[152:159], v[28:31] cbsz:0 blgp:1\n\t" \
  "ds_read_b64_tr_b8 v[136:137], v177 offset:0\n\t" \
  "v_mfma_f32_16x16x128_f8f6f4 v[60:63], a[40:47], v[152:159], v[60:63] cbsz:0 blgp:1\n\t" \
  "ds_read_b64_tr_b8 v[138:139], v177 offset:1024\n\t" \
  "buffer_load_dwordx4 a[204:207], v188, s[36:39], s87 offen offset:3072 \n\t" \
  "v_mfma_f32_16x16x128_f8f6f4 v[92:95], a[48:55], v[152:159], v[92:95] cbsz:0 blgp:1\n\t" \
  "ds_read_b64_tr_b8 v[140:141], v177 offset:8192\n\t" \
  "v_mfma_f32_16x16x128_f8f6f4 v[124:127], a[56:63], v[152:159], v[124:127] cbsz:0 blgp:1\n\t" \
  "ds_read_b64_tr_b8 v[142:143], v177 offset:9216\n\t" \
  "buffer_load_dwordx4 a[208:211], v189, s[36:39], s87 offen offset:0 \n\t" \
  "s_waitcnt lgkmcnt(4)\n\t" \
  "v_mfma_f32_16x16x128_f8f6f4 v[0:3], a[64:71], v[128:135], v[0:3] cbsz:0 blgp:1\n\t" \
  "ds_read_b64_tr_b8 v[144:145], v178 offset:0\n\t" \
  "v_mfma_f32_16x16x128_f8f6f4 v[32:35], a[72:79], v[128:135], v[32:35] cbsz:0 blgp:1\n\t" \
  "ds_read_b64_tr_b8 v[146:147], v178 offset:1024\n\t" \
  "buffer_load_dwordx4 a[212:215], v189, s[36:39], s87 offen offset:1024 \n\t" \
  "v_mfma_f32_16x16x128_f8f6f4 v[64:67], a[80:87], v[128:135], v[64:67] cbsz:0 blgp:1\n\t" \
  "ds_read_b64_tr_b8 v[148:149], v178 offset:8192\n\t" \
  "s_add_u32 m0, s77, 32768\n\t" \
  "s_add_u32 s90, s72, 4194304\n\t" \
  "buffer_load_dwordx4 v186, s[40:43], s90 offen offset:0  lds\n\t" \
  "v_mfma_f32_16x16x128_f8f6f4 v[96:99], a[88:95], v[128:135], v[96:99] cbsz:0 blgp:1\n\t" \
  "ds_read_b64_tr_b8 v[150:151], v178 offset:9216\n\t" \
  "s_add_u32 m0, s77, 49152\n\t" \
  "s_add_u32 s90, s73, 4194304\n\t" \
  "buffer_load_dwordx4 v186, s[40:43], s90 offen offset:0  lds\n\t" \
  "s_waitcnt lgkmcnt(4)\n\t" \
  "v_mfma_f32_16x16x128_f8f6f4 v[4:7], a[64:71], v[136:143], v[4:7] cbsz:0 blgp:1\n\t" \
  "ds_read_b64_tr_b8 v[152:153], v179 offset:0\n\t" \
  "s_add_u32 m0, s77, 32768\n\t" \
  "s_add_u32 s90, s72, 4194304\n\t" \
  "buffer_load_dwordx4 v187, s[40:43], s90 offen offset:1024  lds\n\t" \
  "v_mfma_f32_16x16x128_f8f6f4 v[36:39], a[72:79], v[136:143], v[36:39] cbsz:0 blgp:1\n\t" \
  "ds_read_b64_tr_b8 v[154:155], v179 offset:1024\n\t" \
  "s_add_u32 m0, s77, 49152\n\t" \
  "s_add_u32 s90, s73, 4194304\n\t" \
  "buffer_load_dwordx4 v187, s[40:43], s90 offen offset:1024  lds\n\t" \
  "v_mfma_f32_16x16x128_f8f6f4 v[68:71], a[80:87], v[136:143], v[68:71] cbsz:0 blgp:1\n\t" \
  "ds_read_b64_tr_b8 v[156:157], v179 offset:8192\n\t" \
  "s_add_u32 m0, s77, 65536\n\t" \
  "s_add_u32 s90, s74, 4194304\n\t" \
  "buffer_load_dwordx4 v186, s[40:43], s90 offen offset:0  lds\n\t" \
  "v_mfma_f32_16x16x128_f8f6f4 v[100:103], a[88:95], v[136:143], v[100:103] cbsz:0 blgp:1\n\t" \
  "ds_read_b64_tr_b8 v[158:159], v179 offset:9216\n\t" \
  "s_add_u32 m0, s76, 81920\n\t" \
  "s_add_u32 s90, s68, 6291456\n\t" \
  "buffer_load_dwordx4 v184, s[40:43], s90 offen offset:0  lds\n\t" \
  "s_waitcnt lgkmcnt(4)\n\t" \
  "v_mfma_f32_16x16x128_f8f6f4 v[8:11], a[64:71], v[144:151], v[8:11] cbsz:0 blgp:1\n\t" \
  "ds_read_b64_tr_b8 v[128:129], v180 offset:0\n\t" \
  "s_add_u32 m0, s77, 65536\n\t" \
  "s_add_u32 s90, s74, 4194304\n\t" \
  "buffer_load_dwordx4 v187, s[40:43], s90 offen offset:1024  lds\n\t" \
  "v_mfma_f32_16x16x128_f8f6f4 v[40:43], a[72:79], v[144:151], v[40:43] cbsz:0 blgp:1\n\t" \
  "ds_read_b64_tr_b8 v[130:131], v180 offset:1024\n\t" \
  "s_add_u32 m0, s76, 81920\n\t" \
  "s_add_u32 s90, s68, 6291456\n\t" \
  "buffer_load_dwordx4 v185, s[40:43], s90 offen offset:1024  lds\n\t" \
  "v_mfma_f32_16x16x128_f8f6f4 v[72:75], a[80:87], v[144:151], v[72:75] cbsz:0 blgp:1\n\t" \
  "ds_read_b64_tr_b8 v[132:133], v180 offset:8192\n\t" \
  "buffer_load_dwordx4 a[216:219], v189, s[36:39], s87 offen offset:2048 \n\t" \
  "v_mfma_f32_16x16x128_f8f6f4 v[104:107], a[88:95], v[144:151], v[104:107] cbsz:0 blgp:1\n\t" \
  "ds_read_b64_tr_b8 v[134:135], v180 offset:9216\n\t" \
  "s_cmp_eq_u32 s9, -1\n\t" \
  "s_cbranch_scc0 L_au11_2\n\t" \
  "L_aur11_2:\n\t" \
  "s_add_u32 m0, s77, 81920\n\t" \
  "s_add_u32 s90, s68, 6291456\n\t" \
  "buffer_load_dwordx4 v186, s[40:43], s90 offen offset:0  lds\n\t" \
  "s_waitcnt lgkmcnt(4)\n\t" \
  "v_mfma_f32_16x16x128_f8f6f4 v[12:15], a[64:71], v[152:159], v[12:15] cbsz:0 blgp:1\n\t" \
  "ds_read_b64_tr_b8 v[136:137], v181 offset:0\n\t" \
  "buffer_load_dwordx4 a[220:223], v189, s[36:39], s87 offen offset:3072 \n\t" \
  "v_mfma_f32_16x16x128_f8f6f4 v[44:47], a[72:79], v[152:159], v[44:47] cbsz:0 blgp:1\n\t" \
  "ds_read_b64_tr_b8 v[138:139], v181 offset:1024\n\t" \
  "s_add_u32 m0, s77, 81920\n\t" \
  "s_add_u32 s90, s68, 6291456\n\t" \
  "buffer_load_dwordx4 v187, s[40:43], s90 offen offset:1024  lds\n\t" \
  "v_mfma_f32_16x16x128_f8f6f4 v[76:79], a[80:87], v[152:159], v[76:79] cbsz:0 blgp:1\n\t" \
  "ds_read_b64_tr_b8 v[140:141], v181 offset:8192\n\t" \
  "s_bfe_u32 s84, s8, 262164\n\t" \
  "s_cmp_eq_u32 s84, 15\n\t" \
  "s_cbranch_scc0 L_q23_2\n\t" \
  "L_qr23_2:\n\t" \
  "s_mov_b32 s87, 1245184\n\t" \
  "buffer_load_dwordx4 a[224:227], v188, s[36:39], s87 offen offset:0 \n\t" \
  "v_mfma_f32_16x16x128_f8f6f4 v[108:111], a[88:95], v[152:159], v[108:111] cbsz:0 blgp:1\n\t" \
  "ds_read_b64_tr_b8 v[142:143], v181 offset:9216\n\t" \
  "s_waitcnt lgkmcnt(4)\n\t" \
  "v_mfma_f32_16x16x128_f8f6f4 v[16:19], a[64:71], v[128:135], v[16:19] cbsz:0 blgp:1\n\t" \
  "ds_read_b64_tr_b8 v[144:145], v182 offset:0\n\t" \
  "buffer_load_dwordx4 a[228:231], v188, s[36:39], s87 offen offset:1024 \n\t" \
  "v_mfma_f32_16x16x128_f8f6f4 v[48:51], a[72:79], v[128:135], v[48:51] cbsz:0 blgp:1\n\t" \
  "ds_read_b64_tr_b8 v[146:147], v182 offset:1024\n\t" \
  "v_mfma_f32_16x16x128_f8f6f4 v[80:83], a[80:87], v[128:135], v[80:83] cbsz:0 blgp:1\n\t" \
  "ds_read_b64_tr_b8 v[148:149], v182 offset:8192\n\t" \
  "buffer_load_dwordx4 a[232:235], v188, s[36:39], s87 offen offset:2048 \n\t" \
  "v_mfma_f32_16x16x128_f8f6f4 v[112:115], a[88:95], v[128:135], v[112:115] cbsz:0 blgp:1\n\t" \
  "ds_read_b64_tr_b8 v[150:151], v182 offset:9216\n\t" \
  "s_waitcnt lgkmcnt(4)\n\t" \
  "v_mfma_f32_16x16x128_f8f6f4 v[20:23], a[64:71], v[136:143], v[20:23] cbsz:0 blgp:1\n\t" \
  "ds_read_b64_tr_b8 v[152:153], v183 offset:0\n\t" \
  "buffer_load_dwordx4 a[236:239], v188, s[36:39], s87 offen offset:3072 \n\t" \
  "v_mfma_f32_16x16x128_f8f6f4 v[52:55], a[72:79], v[136:143], v[52:55] cbsz:0 blgp:1\n\t" \
  "ds_read_b64_tr_b8 v[154:155], v183 offset:1024\n\t" \
  "v_mfma_f32_16x16x128_f8f6f4 v[84:87], a[80:87], v[136:143], v[84:87] cbsz:0 blgp:1\n\t" \
  "ds_read_b64_tr_b8 v[156:157], v183 offset:8192\n\t" \
  "buffer_load_dwordx4 a[240:243], v189, s[36:39], s87 offen offset:0 \n\t" \
  "v_mfma_f32_16x16x128_f8f6f4 v[116:119], a[88:95], v[136:143], v[116:119] cbsz:0 blgp:1\n\t" \
  "ds_read_b64_tr_b8 v[158:159], v183 offset:9216\n\t" \
  "s_cmp_eq_u32 s9, -1\n\t" \
  "s_cbranch_scc0 L_ap12_2\n\t" \
  "L_apr12_2:\n\t" \
  "s_waitcnt lgkmcnt(4)\n\t" \
  "v_mfma_f32_16x16x128_f8f6f4 v[24:27], a[64:71], v[144:151], v[24:27] cbsz:0 blgp:1\n\t" \
  "s_barrier\n\t" \
  "ds_read_b64_tr_b8 v[128:129], v176 offset:16384\n\t" \
  "v_mfma_f32_16x16x128_f8f6f4 v[56:59], a[72:79], v[144:151], v[56:59] cbsz:0 blgp:1\n\t" \
  "ds_read_b64_tr_b8 v[130:131], v176 offset:17408\n\t" \
  "buffer_load_dwordx4 a[244:247], v189, s[36:39], s87 offen offset:1024 \n\t" \
  "v_mfma_f32_16x16x128_f8f6f4 v[88:91], a[80:87], v[144:151], v[88:91] cbsz:0 blgp:1\n\t" \
  "ds_read_b64_tr_b8 v[132:133], v176 offset:24576\n\t" \
  "v_mfma_f32_16x16x128_f8f6f4 v[120:123], a[88:95], v[144:151], v[120:123] cbsz:0 blgp:1\n\t" \
  "ds_read_b64_tr_b8 v[134:135], v176 offset:25600\n\t" \
  "buffer_load_dwordx4 a[248:251], v189, s[36:39], s87 offen offset:2048 \n\t" \
  "s_waitcnt lgkmcnt(4)\n\t" \
  "v_mfma_f32_16x16x128_f8f6f4 v[28:31], a[64:71], v[152:159], v[28:31] cbsz:0 blgp:1\n\t" \
  "ds_read_b64_tr_b8 v[136:137], v177 offset:16384\n\t" \
  "v_mfma_f32_16x16x128_f8f6f4 v[60:63], a[72:79], v[152:159], v[60:63] cbsz:0 blgp:1\n\t" \
  "ds_read_b64_tr_b8 v[138:139], v177 offset:17408\n\t" \
  "buffer_load_dwordx4 a[252:255], v189, s[36:39], s87 offen offset:3072 \n\t" \
  "v_mfma_f32_16x16x128_f8f6f4 v[92:95], a[80:87], v[152:159], v[92:95] cbsz:0 blgp:1\n\t" \
  "ds_read_b64_tr_b8 v[140:141], v177 offset:24576\n\t" \
  "v_mfma_f32_16x16x128_f8f6f4 v[124:127], a[88:95], v[152:159], v[124:127] cbsz:0 blgp:1\n\t" \
  "ds_read_b64_tr_b8 v[142:143], v177 offset:25600\n\t" \
  "s_bfe_u32 s84, s8, 262168\n\t" \
  "s_cmp_eq_u32 s84, 15\n\t" \
  "s_cbranch_scc0 L_q24_2\n\t" \
  "L_qr24_2:\n\t" \
  "s_mov_b32 s87, 1310720\n\t" \
  "buffer_load_dwordx4 a[0:3], v188, s[36:39], s87 offen offset:0 \n\t" \
  "s_waitcnt lgkmcnt(4)\n\t" \
  "v_mfma_f32_16x16x128_f8f6f4 v[0:3], a[96:103], v[128:135], v[0:3] cbsz:0 blgp:1\n\t" \
  "ds_read_b64_tr_b8 v[144:145], v178 offset:16384\n\t" \
  "v_mfma_f32_16x16x128_f8f6f4 v[32:35], a[104:111], v[128:135], v[32:35] cbsz:0 blgp:1\n\t" \
  "ds_read_b64_tr_b8 v[146:147], v178 offset:17408\n\t" \
  "buffer_load_dwordx4 a[4:7], v188, s[36:39], s87 offen offset:1024 \n\t" \
  "v_mfma_f32_16x16x128_f8f6f4 v[64:67], a[112:119], v[128:135], v[64:67] cbsz:0 blgp:1\n\t" \
  "ds_read_b64_tr_b8 v[148:149], v178 offset:24576\n\t" \
  "v_mfma_f32_16x16x128_f8f6f4 v[96:99], a[120:127], v[128:135], v[96:99] cbsz:0 blgp:1\n\t" \
  "ds_read_b64_tr_b8 v[150:151], v178 offset:25600\n\t" \
  "buffer_load_dwordx4 a[8:11], v188, s[36:39], s87 offen offset:2048 \n\t" \
  "s_waitcnt lgkmcnt(4)\n\t" \
  "v_mfma_f32_16x16x128_f8f6f4 v[4:7], a[96:103], v[136:143], v[4:7] cbsz:0 blgp:1\n\t" \
  "ds_read_b64_tr_b8 v[152:153], v179 offset:16384\n\t" \
  "v_mfma_f32_16x16x128_f8f6f4 v[36:39], a[104:111], v[136:143], v[36:39] cbsz:0 blgp:1\n\t" \
  "ds_read_b64_tr_b8 v[154:155], v179 offset:17408\n\t" \
  "buffer_load_dwordx4 a[12:15], v188, s[36:39], s87 offen offset:3072 \n\t" \
  "v_mfma_f32_16x16x128_f8f6f4 v[68:71], a[112:119], v[136:143], v[68:71] cbsz:0 blgp:1\n\t" \
  "ds_read_b64_tr_b8 v[156:157], v179 offset:24576\n\t" \
  "v_mfma_f32_16x16x128_f8f6f4 v[100:103], a[120:127], v[136:143], v[100:103] cbsz:0 blgp:1\n\t" \
  "ds_read_b64_tr_b8 v[158:159], v179 offset:25600\n\t" \
  "buffer_load_dwordx4 a[16:19], v189, s[36:39], s87 offen offset:0 \n\t" \
  "s_waitcnt lgkmcnt(4)\n\t" \
  "v_mfma_f32_16x16x128_f8f6f4 v[8:11], a[96:103], v[144:151], v[8:11] cbsz:0 blgp:1\n\t" \
  "ds_read_b64_tr_b8 v[128:129], v180 offset:16384\n\t" \
  "v_mfma_f32_16x16x128_f8f6f4 v[40:43], a[104:111], v[144:151], v[40:43] cbsz:0 blgp:1\n\t" \
  "ds_read_b64_tr_b8 v[130:131], v180 offset:17408\n\t" \
  "buffer_load_dwordx4 a[20:23], v189, s[36:39], s87 offen offset:1024 \n\t" \
  "v_mfma_f32_16x16x128_f8f6f4 v[72:75], a[112:119], v[144:151], v[72:75] cbsz:0 blgp:1\n\t" \
  "ds_read_b64_tr_b8 v[132:133], v180 offset:24576\n\t" \
  "v_mfma_f32_16x16x128_f8f6f4 v[104:107], a[120:127], v[144:151], v[104:107] cbsz:0 blgp:1\n\t" \
  "ds_read_b64_tr_b8 v[134:135], v180 offset:25600\n\t" \
  "s_cmp_eq_u32 s9, -1\n\t" \
  "s_cbranch_scc0 L_au12_2\n\t" \
  "L_aur12_2:\n\t" \
  "buffer_load_dwordx4 a[24:27], v189, s[36:39], s87 offen offset:2048 \n\t" \
  "s_waitcnt lgkmcnt(4)\n\t" \
  "v_mfma_f32_16x16x128_f8f6f4 v[12:15], a[96:103], v[152:159], v[12:15] cbsz:0 blgp:1\n\t" \
  "ds_read_b64_tr_b8 v[136:137], v181 offset:16384\n\t" \
  "v_mfma_f32_16x16x128_f8f6f4 v[44:47], a[104:111], v[152:159], v[44:47] cbsz:0 blgp:1\n\t" \
  "ds_read_b64_tr_b8 v[138:139], v181 offset:17408\n\t" \
  "buffer_load_dwordx4 a[28:31], v189, s[36:39], s87 offen offset:3072 \n\t" \
  "v_mfma_f32_16x16x128_f8f6f4 v[76:79], a[112:119], v[152:159], v[76:79] cbsz:0 blgp:1\n\t" \
  "ds_read_b64_tr_b8 v[140:141], v181 offset:24576\n\t" \
  "v_mfma_f32_16x16x128_f8f6f4 v[108:111], a[120:127], v[152:159], v[108:111] cbsz:0 blgp:1\n\t" \
  "ds_read_b64_tr_b8 v[142:143], v181 offset:25600\n\t" \
  "s_bfe_u32 s84, s9, 262144\n\t" \
  "s_cmp_eq_u32 s84, 15\n\t" \
  "s_cbranch_scc0 L_q25_2\n\t" \
  "L_qr25_2:\n\t" \
  "s_mov_b32 s87, 1376256\n\t" \
  "buffer_load_dwordx4 a[32:35], v188, s[36:39], s87 offen offset:0 \n\t" \
  "s_waitcnt lgkmcnt(4)\n\t" \
  "v_mfma_f32_16x16x128_f8f6f4 v[16:19], a[96:103], v[128:135], v[16:19] cbsz:0 blgp:1\n\t" \
  "ds_read_b64_tr_b8 v[144:145], v182 offset:16384\n\t" \
  "v_mfma_f32_16x16x128_f8f6f4 v[48:51], a[104:111], v[128:135], v[48:51] cbsz:0 blgp:1\n\t" \
  "ds_read_b64_tr_b8 v[146:147], v182 offset:17408\n\t" \
  "buffer_load_dwordx4 a[36:39], v188, s[36:39], s87 offen offset:1024 \n\t" \
  "v_mfma_f32_16x16x128_f8f6f4 v[80:83], a[112:119], v[128:135], v[80:83] cbsz:0 blgp:1\n\t" \
  "ds_read_b64_tr_b8 v[148:149], v182 offset:24576\n\t" \
  "v_mfma_f32_16x16x128_f8f6f4 v[112:115], a[120:127], v[128:135], v[112:115] cbsz:0 blgp:1\n\t" \
  "ds_read_b64_tr_b8 v[150:151], v182 offset:25600\n\t" \
  "buffer_load_dwordx4 a[40:43], v188, s[36:39], s87 offen offset:2048 \n\t" \
  "s_waitcnt lgkmcnt(4)\n\t" \
  "v_mfma_f32_16x16x128_f8f6f4 v[20:23], a[96:103], v[136:143], v[20:23] cbsz:0 blgp:1\n\t" \
  "ds_read_b64_tr_b8 v[152:153], v183 offset:16384\n\t" \
  "v_mfma_f32_16x16x128_f8f6f4 v[52:55], a[104:111], v[136:143], v[52:55] cbsz:0 blgp:1\n\t" \
  "ds_read_b64_tr_b8 v[154:155], v183 offset:17408\n\t" \
  "buffer_load_dwordx4 a[44:47], v188, s[36:39], s87 offen offset:3072 \n\t" \
  "v_mfma_f32_16x16x128_f8f6f4 v[84:87], a[112:119], v[136:143], v[84:87] cbsz:0 blgp:1\n\t" \
  "ds_read_b64_tr_b8 v[156:157], v183 offset:24576\n\t" \
  "v_mfma_f32_16x16x128_f8f6f4 v[116:119], a[120:127], v[136:143], v[116:119] cbsz:0 blgp:1\n\t" \
  "ds_read_b64_tr_b8 v[158:159], v183 offset:25600\n\t" \
  "s_cmp_eq_u32 s9, -1\n\t" \
  "s_cbranch_scc0 L_ap13_2\n\t" \
  "L_apr13_2:\n\t" \
  "buffer_load_dwordx4 a[48:51], v189, s[36:39], s87 offen offset:0 \n\t" \
  "s_waitcnt lgkmcnt(4)\n\t" \
  "v_mfma_f32_16x16x128_f8f6f4 v[24:27], a[96:103], v[144:151], v[24:27] cbsz:0 blgp:1\n\t" \
  "s_barrier\n\t" \
  "ds_read_b64_tr_b8 v[128:129], v160 offset:0\n\t" \
  "v_mfma_f32_16x16x128_f8f6f4 v[56:59], a[104:111], v[144:151], v[56:59] cbsz:0 blgp:1\n\t" \
  "ds_read_b64_tr_b8 v[130:131], v160 offset:1024\n\t" \
  "s_add_u32 m0, s76, 98304\n\t" \
  "s_add_u32 s90, s69, 6291456\n\t" \
  "buffer_load_dwordx4 v184, s[40:43], s90 offen offset:0  lds\n\t" \
  "v_mfma_f32_16x16x128_f8f6f4 v[88:91], a[112:119], v[144:151], v[88:91] cbsz:0 blgp:1\n\t" \
  "ds_read_b64_tr_b8 v[132:133], v160 offset:8192\n\t" \
  "s_add_u32 m0, s76, 98304\n\t" \
  "s_add_u32 s90, s69, 6291456\n\t" \
  "buffer_load_dwordx4 v185, s[40:43], s90 offen offset:1024  lds\n\t" \
  "v_mfma_f32_16x16x128_f8f6f4 v[120:123], a[120:127], v[144:151], v[120:123] cbsz:0 blgp:1\n\t" \
  "ds_read_b64_tr_b8 v[134:135], v160 offset:9216\n\t" \
  "s_add_u32 m0, s76, 114688\n\t" \
  "s_add_u32 s90, s70, 6291456\n\t" \
  "buffer_load_dwordx4 v184, s[40:43], s90 offen offset:0  lds\n\t" \
  "s_waitcnt lgkmcnt(4)\n\t" \
  "v_mfma_f32_16x16x128_f8f6f4 v[28:31], a[96:103], v[152:159], v[28:31] cbsz:0 blgp:1\n\t" \
  "ds_read_b64_tr_b8 v[136:137], v161 offset:0\n\t" \
  "s_add_u32 m0, s76, 114688\n\t" \
  "s_add_u32 s90, s70, 6291456\n\t" \
  "buffer_load_dwordx4 v185, s[40:43], s90 offen offset:1024  lds\n\t" \
  "v_mfma_f32_16x16x128_f8f6f4 v[60:63], a[104:111], v[152:159], v[60:63] cbsz:0 blgp:1\n\t" \
  "ds_read_b64_tr_b8 v[138:139], v161 offset:1024\n\t" \
  "s_add_u32 m0, s76, 131072\n\t" \
  "s_add_u32 s90, s71, 6291456\n\t" \
  "buffer_load_dwordx4 v184, s[40:43], s90 offen offset:0  lds\n\t" \
  "v_mfma_f32_16x16x128_f8f6f4 v[92:95], a[112:119], v[152:159], v[92:95] cbsz:0 blgp:1\n\t" \
  "ds_read_b64_tr_b8 v[140:141], v161 offset:8192\n\t" \
  "s_add_u32 m0, s76, 131072\n\t" \
  "s_add_u32 s90, s71, 6291456\n\t" \
  "buffer_load_dwordx4 v185, s[40:43], s90 offen offset:1024  lds\n\t" \
  "v_mfma_f32_16x16x128_f8f6f4 v[124:127], a[120:127], v[152:159], v[124:127] cbsz:0 blgp:1\n\t" \
  "ds_read_b64_tr_b8 v[142:143], v161 offset:9216\n\t" \
  "s_add_u32 s90, s72, 6291456\n\t" \
  "s_mov_b64 exec, 0xffff\n\t" \
  "buffer_load_dword v210, v241, s[40:43], s90 offen\n\t" \
  "s_mov_b64 exec, -1\n\t" \
  "s_waitcnt lgkmcnt(4)\n\t" \
  "v_mfma_f32_16x16x128_f8f6f4 v[0:3], a[128:135], v[128:135], v[0:3] cbsz:0 blgp:1\n\t" \
  "ds_read_b64_tr_b8 v[144:145], v162 offset:0\n\t" \
  "buffer_load_dwordx4 a[52:55], v189, s[36:39], s87 offen offset:1024 \n\t" \
  "s_waitcnt vmcnt(63)\n\t" \
  "v_mfma_f32_16x16x128_f8f6f4 v[32:35], a[136:143], v[128:135], v[32:35] cbsz:0 blgp:1\n\t" \
  "ds_read_b64_tr_b8 v[146:147], v162 offset:1024\n\t" \
  "v_mfma_f32_16x16x128_f8f6f4 v[64:67], a[144:151], v[128:135], v[64:67] cbsz:0 blgp:1\n\t" \
  "ds_read_b64_tr_b8 v[148:149], v162 offset:8192\n\t" \
  "s_waitcnt vmcnt(62)\n\t" \
  "v_mfma_f32_16x16x128_f8f6f4 v[96:99], a[152:159], v[128:135], v[96:99] cbsz:0 blgp:1\n\t" \
  "ds_read_b64_tr_b8 v[150:151], v162 offset:9216\n\t" \
  "s_waitcnt lgkmcnt(4)\n\t" \
  "v_mfma_f32_16x16x128_f8f6f4 v[4:7], a[128:135], v[136:143], v[4:7] cbsz:0 blgp:1\n\t" \
  "ds_read_b64_tr_b8 v[152:153], v163 offset:0\n\t" \
  "v_mfma_f32_16x16x128_f8f6f4 v[36:39], a[136:143], v[136:143], v[36:39] cbsz:0 blgp:1\n\t" \
  "ds_read_b64_tr_b8 v[154:155], v163 offset:1024\n\t" \
  "v_mfma_f32_16x16x128_f8f6f4 v[68:71], a[144:151], v[136:143], v[68:71] cbsz:0 blgp:1\n\t" \
  "ds_read_b64_tr_b8 v[156:157], v163 offset:8192\n\t" \
  "v_mfma_f32_16x16x128_f8f6f4 v[100:103], a[152:159], v[136:143], v[100:103] cbsz:0 blgp:1\n\t" \
  "ds_read_b64_tr_b8 v[158:159], v163 offset:9216\n\t" \
  "s_waitcnt lgkmcnt(4)\n\t" \
  "v_mfma_f32_16x16x128_f8f6f4 v[8:11], a[128:135], v[144:151], v[8:11] cbsz:0 blgp:1\n\t" \
  "ds_read_b64_tr_b8 v[128:129], v164 offset:0\n\t" \
  "v_mfma_f32_16x16x128_f8f6f4 v[40:43], a[136:143], v[144:151], v[40:43] cbsz:0 blgp:1\n\t" \
  "ds_read_b64_tr_b8 v[130:131], v164 offset:1024\n\t" \
  "v_mfma_f32_16x16x128_f8f6f4 v[72:75], a[144:151], v[144:151], v[72:75] cbsz:0 blgp:1\n\t" \
  "ds_read_b64_tr_b8 v[132:133], v164 offset:8192\n\t" \
  "v_mfma_f32_16x16x128_f8f6f4 v[104:107], a[152:159], v[144:151], v[104:107] cbsz:0 blgp:1\n\t" \
  "ds_read_b64_tr_b8 v[134:135], v164 offset:9216\n\t" \
  "s_cmp_eq_u32 s9, -1\n\t" \
  "s_cbranch_scc0 L_au13_2\n\t" \
  "L_aur13_2:\n\t" \
  "s_waitcnt lgkmcnt(4)\n\t" \
  "v_mfma_f32_16x16x128_f8f6f4 v[12:15], a[128:135], v[152:159], v[12:15] cbsz:0 blgp:1\n\t" \
  "ds_read_b64_tr_b8 v[136:137], v165 offset:0\n\t" \
  "v_mfma_f32_16x16x128_f8f6f4 v[44:47], a[136:143], v[152:159], v[44:47] cbsz:0 blgp:1\n\t" \
  "ds_read_b64_tr_b8 v[138:139], v165 offset:1024\n\t" \
  "v_mfma_f32_16x16x128_f8f6f4 v[76:79], a[144:151], v[152:159], v[76:79] cbsz:0 blgp:1\n\t" \
  "ds_read_b64_tr_b8 v[140:141], v165 offset:8192\n\t" \
  "v_mfma_f32_16x16x128_f8f6f4 v[108:111], a[152:159], v[152:159], v[108:111] cbsz:0 blgp:1\n\t" \
  "ds_read_b64_tr_b8 v[142:143], v165 offset:9216\n\t" \
  "s_waitcnt lgkmcnt(4)\n\t" \
  "v_mfma_f32_16x16x128_f8f6f4 v[16:19], a[128:135], v[128:135], v[16:19] cbsz:0 blgp:1\n\t" \
  "ds_read_b64_tr_b8 v[144:145], v166 offset:0\n\t" \
  "v_mfma_f32_16x16x128_f8f6f4 v[48:51], a[136:143], v[128:135], v[48:51] cbsz:0 blgp:1\n\t" \
  "ds_read_b64_tr_b8 v[146:147], v166 offset:1024\n\t" \
  "v_mfma_f32_16x16x128_f8f6f4 v[80:83], a[144:151], v[128:135], v[80:83] cbsz:0 blgp:1\n\t" \
  "ds_read_b64_tr_b8 v[148:149], v166 offset:8192\n\t" \
  "v_mfma_f32_16x16x128_f8f6f4 v[112:115], a[152:159], v[128:135], v[112:115] cbsz:0 blgp:1\n\t" \
  "ds_read_b64_tr_b8 v[150:151], v166 offset:9216\n\t" \
  "s_waitcnt lgkmcnt(4)\n\t" \
  "v_mfma_f32_16x16x128_f8f6f4 v[20:23], a[128:135], v[136:143], v[20:23] cbsz:0 blgp:1\n\t" \
  "ds_read_b64_tr_b8 v[152:153], v167 offset:0\n\t" \
  "v_mfma_f32_16x16x128_f8f6f4 v[52:55], a[136:143], v[136:143], v[52:55] cbsz:0 blgp:1\n\t" \
  "ds_read_b64_tr_b8 v[154:155], v167 offset:1024\n\t" \
  "v_mfma_f32_16x16x128_f8f6f4 v[84:87], a[144:151], v[136:143], v[84:87] cbsz:0 blgp:1\n\t" \
  "ds_read_b64_tr_b8 v[156:157], v167 offset:8192\n\t" \
  "v_mfma_f32_16x16x128_f8f6f4 v[116:119], a[152:159], v[136:143], v[116:119] cbsz:0 blgp:1\n\t" \
  "ds_read_b64_tr_b8 v[158:159], v167 offset:9216\n\t" \
  "s_cmp_eq_u32 s9, -1\n\t" \
  "s_cbranch_scc0 L_ap14_2\n\t" \
  "L_apr14_2:\n\t" \
  "s_waitcnt lgkmcnt(4)\n\t" \
  "v_mfma_f32_16x16x128_f8f6f4 v[24:27], a[128:135], v[144:151], v[24:27] cbsz:0 blgp:1\n\t" \
  "s_barrier\n\t" \
  "ds_read_b64_tr_b8 v[128:129], v160 offset:16384\n\t" \
  "v_mfma_f32_16x16x128_f8f6f4 v[56:59], a[136:143], v[144:151], v[56:59] cbsz:0 blgp:1\n\t" \
  "ds_read_b64_tr_b8 v[130:131], v160 offset:17408\n\t" \
  "v_mfma_f32_16x16x128_f8f6f4 v[88:91], a[144:151], v[144:151], v[88:91] cbsz:0 blgp:1\n\t" \
  "ds_read_b64_tr_b8 v[132:133], v160 offset:24576\n\t" \
  "v_mfma_f32_16x16x128_f8f6f4 v[120:123], a[152:159], v[144:151], v[120:123] cbsz:0 blgp:1\n\t" \
  "ds_read_b64_tr_b8 v[134:135], v160 offset:25600\n\t" \
  "s_waitcnt lgkmcnt(4)\n\t" \
  "v_mfma_f32_16x16x128_f8f6f4 v[28:31], a[128:135], v[152:159], v[28:31] cbsz:0 blgp:1\n\t" \
  "ds_read_b64_tr_b8 v[136:137], v161 offset:16384\n\t" \
  "v_mfma_f32_16x16x128_f8f6f4 v[60:63], a[136:143], v[152:159], v[60:63] cbsz:0 blgp:1\n\t" \
  "ds_read_b64_tr_b8 v[138:139], v161 offset:17408\n\t" \
  "v_mfma_f32_16x16x128_f8f6f4 v[92:95], a[144:151], v[152:159], v[92:95] cbsz:0 blgp:1\n\t" \
  "ds_read_b64_tr_b8 v[140:141], v161 offset:24576\n\t" \
  "v_mfma_f32_16x16x128_f8f6f4 v[124:127], a[152:159], v[152:159], v[124:127] cbsz:0 blgp:1\n\t" \
  "ds_read_b64_tr_b8 v[142:143], v161 offset:25600\n\t" \
  "s_waitcnt vmcnt(60) lgkmcnt(4)\n\t" \
  "v_mfma_f32_16x16x128_f8f6f4 v[0:3], a[160:167], v[128:135], v[0:3] cbsz:0 blgp:1\n\t" \
  "ds_read_b64_tr_b8 v[144:145], v162 offset:16384\n\t" \
  "s_add_u32 s90, s73, 6291456\n\t" \
  "s_mov_b64 exec, 0xffff\n\t" \
  "buffer_load_dword v210, v241, s[40:43], s90 offen\n\t" \
  "s_mov_b64 exec, -1\n\t" \
  "s_waitcnt vmcnt(59)\n\t" \
  "v_mfma_f32_16x16x128_f8f6f4 v[32:35], a[168:175], v[128:135], v[32:35] cbsz:0 blgp:1\n\t" \
  "ds_read_b64_tr_b8 v[146:147], v162 offset:17408\n\t" \
  "s_add_u32 s90, s74, 6291456\n\t" \
  "s_mov_b64 exec, 0xffff\n\t" \
  "buffer_load_dword v210, v241, s[40:43], s90 offen\n\t" \
  "s_mov_b64 exec, -1\n\t" \
  "s_waitcnt vmcnt(51)\n\t" \
  "v_mfma_f32_16x16x128_f8f6f4 v[64:67], a[176:183], v[128:135], v[64:67] cbsz:0 blgp:1\n\t" \
  "ds_read_b64_tr_b8 v[148:149], v162 offset:24576\n\t" \
  "s_cmp_gt_u32 s29, 1\n\t" \
  "s_cbranch_scc1 L_k674_2\n\t" \
  "s_getpc_b64 s[88:89]\n\t" \
  "s_lshl_b32 s84, s29, 13\n\t" \
  "s_add_u32 s84, s84, 2048\n\t" \
  "s_add_u32 s88, s88, s84\n\t" \
  "s_addc_u32 s89, s89, 0\n\t" \
  "s_mov_b64 exec, 0xff\n\t" \
  "global_load_dword v210, v234, s[88:89]\n\t" \
  "s_mov_b64 exec, -1\n\t" \
  "L_k674_2:\n\t" \
  "buffer_load_dwordx4 a[56:59], v189, s[36:39], s87 offen offset:2048 \n\t" \
  "s_waitcnt vmcnt(50)\n\t" \
  "v_mfma_f32_16x16x128_f8f6f4 v[96:99], a[184:191], v[128:135], v[96:99] cbsz:0 blgp:1\n\t" \
  "ds_read_b64_tr_b8 v[150:151], v162 offset:25600\n\t" \
  "s_waitcnt lgkmcnt(4)\n\t" \
  "v_mfma_f32_16x16x128_f8f6f4 v[4:7], a[160:167], v[136:143], v[4:7] cbsz:0 blgp:1\n\t" \
  "ds_read_b64_tr_b8 v[152:153], v163 offset:16384\n\t" \
  "v_mfma_f32_16x16x128_f8f6f4 v[36:39], a[168:175], v[136:143], v[36:39] cbsz:0 blgp:1\n\t" \
  "ds_read_b64_tr_b8 v[154:155], v163 offset:17408\n\t" \
  "v_mfma_f32_16x16x128_f8f6f4 v[68:71], a[176:183], v[136:143], v[68:71] cbsz:0 blgp:1\n\t" \
  "ds_read_b64_tr_b8 v[156:157], v163 offset:24576\n\t" \
  "v_mfma_f32_16x16x128_f8f6f4 v[100:103], a[184:191], v[136:143], v[100:103] cbsz:0 blgp:1\n\t" \
  "ds_read_b64_tr_b8 v[158:159], v163 offset:25600\n\t" \
  "s_waitcnt lgkmcnt(4)\n\t" \
  "v_mfma_f32_16x16x128_f8f6f4 v[8:11], a[160:167], v[144:151], v[8:11] cbsz:0 blgp:1\n\t" \
  "ds_read_b64_tr_b8 v[128:129], v164 offset:16384\n\t" \
  "v_mfma_f32_16x16x128_f8f6f4 v[40:43], a[168:175], v[144:151], v[40:43] cbsz:0 blgp:1\n\t" \
  "ds_read_b64_tr_b8 v[130:131], v164 offset:17408\n\t" \
  "v_mfma_f32_16x16x128_f8f6f4 v[72:75], a[176:183], v[144:151], v[72:75] cbsz:0 blgp:1\n\t" \
  "ds_read_b64_tr_b8 v[132:133], v164 offset:24576\n\t" \
  "v_mfma_f32_16x16x128_f8f6f4 v[104:107], a[184:191], v[144:151], v[104:107] cbsz:0 blgp:1\n\t" \
  "ds_read_b64_tr_b8 v[134:135], v164 offset:25600\n\t" \
  "s_cmp_eq_u32 s9, -1\n\t" \
  "s_cbranch_scc0 L_au14_2\n\t" \
  "L_aur14_2:\n\t" \
  "s_waitcnt lgkmcnt(4)\n\t" \
  "v_mfma_f32_16x16x128_f8f6f4 v[12:15], a[160:167], v[152:159], v[12:15] cbsz:0 blgp:1\n\t" \
  "ds_read_b64_tr_b8 v[136:137], v165 offset:16384\n\t" \
  "v_mfma_f32_16x16x128_f8f6f4 v[44:47], a[168:175], v[152:159], v[44:47] cbsz:0 blgp:1\n\t" \
  "ds_read_b64_tr_b8 v[138:139], v165 offset:17408\n\t" \
  "v_mfma_f32_16x16x128_f8f6f4 v[76:79], a[176:183], v[152:159], v[76:79] cbsz:0 blgp:1\n\t" \
  "ds_read_b64_tr_b8 v[140:141], v165 offset:24576\n\t" \
  "v_mfma_f32_16x16x128_f8f6f4 v[108:111], a[184:191], v[152:159], v[108:111] cbsz:0 blgp:1\n\t" \
  "ds_read_b64_tr_b8 v[142:143], v165 offset:25600\n\t" \
  "s_waitcnt lgkmcnt(4)\n\t" \
  "v_mfma_f32_16x16x128_f8f6f4 v[16:19], a[160:167], v[128:135], v[16:19] cbsz:0 blgp:1\n\t" \
  "ds_read_b64_tr_b8 v[144:145], v166 offset:16384\n\t" \
  "v_mfma_f32_16x16x128_f8f6f4 v[48:51], a[168:175], v[128:135], v[48:51] cbsz:0 blgp:1\n\t" \
  "ds_read_b64_tr_b8 v[146:147], v166 offset:17408\n\t" \
  "v_mfma_f32_16x16x128_f8f6f4 v[80:83], a[176:183], v[128:135], v[80:83] cbsz:0 blgp:1\n\t" \
  "ds_read_b64_tr_b8 v[148:149], v166 offset:24576\n\t" \
  "v_mfma_f32_16x16x128_f8f6f4 v[112:115], a[184:191], v[128:135], v[112:115] cbsz:0 blgp:1\n\t" \
  "ds_read_b64_tr_b8 v[150:151], v166 offset:25600\n\t" \
  "s_add_u32 m0, s77, 98304\n\t" \
  "s_add_u32 s90, s69, 6291456\n\t" \
  "buffer_load_dwordx4 v186, s[40:43], s90 offen offset:0  lds\n\t" \
  "s_waitcnt lgkmcnt(4)\n\t" \
  "v_mfma_f32_16x16x128_f8f6f4 v[20:23], a[160:167], v[136:143], v[20:23] cbsz:0 blgp:1\n\t" \
  "ds_read_b64_tr_b8 v[152:153], v167 offset:16384\n\t" \
  "s_add_u32 m0, s77, 114688\n\t" \
  "s_add_u32 s90, s70, 6291456\n\t" \
  "buffer_load_dwordx4 v186, s[40:43], s90 offen offset:0  lds\n\t" \
  "v_mfma_f32_16x16x128_f8f6f4 v[52:55], a[168:175], v[136:143], v[52:55] cbsz:0 blgp:1\n\t" \
  "ds_read_b64_tr_b8 v[154:155], v167 offset:17408\n\t" \
  "s_add_u32 m0, s77, 98304\n\t" \
  "s_add_u32 s90, s69, 6291456\n\t" \
  "buffer_load_dwordx4 v187, s[40:43], s90 offen offset:1024  lds\n\t" \
  "v_mfma_f32_16x16x128_f8f6f4 v[84:87], a[176:183], v[136:143], v[84:87] cbsz:0 blgp:1\n\t" \
  "ds_read_b64_tr_b8 v[156:157], v167 offset:24576\n\t" \
  "s_add_u32 m0, s77, 114688\n\t" \
  "s_add_u32 s90, s70, 6291456\n\t" \
  "buffer_load_dwordx4 v187, s[40:43], s90 offen offset:1024  lds\n\t" \
  "v_mfma_f32_16x16x128_f8f6f4 v[116:119], a[184:191], v[136:143], v[116:119] cbsz:0 blgp:1\n\t" \
  "ds_read_b64_tr_b8 v[158:159], v167 offset:25600\n\t" \
  "s_cmp_eq_u32 s9, -1\n\t" \
  "s_cbranch_scc0 L_ap15_2\n\t" \
  "L_apr15_2:\n\t" \
  "s_add_u32 m0, s77, 131072\n\t" \
  "s_add_u32 s90, s71, 6291456\n\t" \
  "buffer_load_dwordx4 v186, s[40:43], s90 offen offset:0  lds\n\t" \
  "s_waitcnt lgkmcnt(4)\n\t" \
  "v_mfma_f32_16x16x128_f8f6f4 v[24:27], a[160:167], v[144:151], v[24:27] cbsz:0 blgp:1\n\t" \
  "s_waitcnt vmcnt(46)\n\t" \
  "s_barrier\n\t" \
  "ds_read_b64_tr_b8 v[128:129], v160 offset:32768\n\t" \
  "v_mfma_f32_16x16x128_f8f6f4 v[56:59], a[168:175], v[144:151], v[56:59] cbsz:0 blgp:1\n\t" \
  "ds_read_b64_tr_b8 v[130:131], v160 offset:33792\n\t" \
  "s_add_u32 m0, s77, 131072\n\t" \
  "s_add_u32 s90, s71, 6291456\n\t" \
  "buffer_load_dwordx4 v187, s[40:43], s90 offen offset:1024  lds\n\t" \
  "v_mfma_f32_16x16x128_f8f6f4 v[88:91], a[176:183], v[144:151], v[88:91] cbsz:0 blgp:1\n\t" \
  "ds_read_b64_tr_b8 v[132:133], v160 offset:40960\n\t" \
  "s_add_u32 m0, s76, 147456\n\t" \
  "s_add_u32 s90, s72, 6291456\n\t" \
  "buffer_load_dwordx4 v184, s[40:43], s90 offen offset:0  lds\n\t" \
  "v_mfma_f32_16x16x128_f8f6f4 v[120:123], a[184:191], v[144:151], v[120:123] cbsz:0 blgp:1\n\t" \
  "ds_read_b64_tr_b8 v[134:135], v160 offset:41984\n\t" \
  "buffer_load_dwordx4 a[60:63], v189, s[36:39], s87 offen offset:3072 \n\t" \
  "s_waitcnt lgkmcnt(4)\n\t" \
  "v_mfma_f32_16x16x128_f8f6f4 v[28:31], a[160:167], v[152:159], v[28:31] cbsz:0 blgp:1\n\t" \
  "ds_read_b64_tr_b8 v[136:137], v161 offset:32768\n\t" \
  "s_add_u32 m0, s76, 147456\n\t" \
  "s_add_u32 s90, s72, 6291456\n\t" \
  "buffer_load_dwordx4 v185, s[40:43], s90 offen offset:1024  lds\n\t" \
  "v_mfma_f32_16x16x128_f8f6f4 v[60:63], a[168:175], v[152:159], v[60:63] cbsz:0 blgp:1\n\t" \
  "ds_read_b64_tr_b8 v[138:139], v161 offset:33792\n\t" \
  "s_bfe_u32 s84, s9, 262148\n\t" \
  "s_cmp_eq_u32 s84, 15\n\t" \
  "s_cbranch_scc0 L_q26_2\n\t" \
  "L_qr26_2:\n\t" \
  "s_mov_b32 s87, 1441792\n\t" \
  "buffer_load_dwordx4 a[64:67], v188, s[36:39], s87 offen offset:0 \n\t" \
  "v_mfma_f32_16x16x128_f8f6f4 v[92:95], a[176:183], v[152:159], v[92:95] cbsz:0 blgp:1\n\t" \
  "ds_read_b64_tr_b8 v[140:141], v161 offset:40960\n\t" \
  "s_add_u32 m0, s77, 147456\n\t" \
  "s_add_u32 s90, s72, 6291456\n\t" \
  "buffer_load_dwordx4 v186, s[40:43], s90 offen offset:0  lds\n\t" \
  "v_mfma_f32_16x16x128_f8f6f4 v[124:127], a[184:191], v[152:159], v[124:127] cbsz:0 blgp:1\n\t" \
  "ds_read_b64_tr_b8 v[142:143], v161 offset:41984\n\t" \
  "buffer_load_dwordx4 a[68:71], v188, s[36:39], s87 offen offset:1024 \n\t" \
  "s_waitcnt lgkmcnt(4)\n\t" \
  "v_mfma_f32_16x16x128_f8f6f4 v[0:3], a[192:199], v[128:135], v[0:3] cbsz:0 blgp:1\n\t" \
  "ds_read_b64_tr_b8 v[144:145], v162 offset:32768\n\t" \
  "s_add_u32 m0, s77, 147456\n\t" \
  "s_add_u32 s90, s72, 6291456\n\t" \
  "buffer_load_dwordx4 v187, s[40:43], s90 offen offset:1024  lds\n\t" \
  "v_mfma_f32_16x16x128_f8f6f4 v[32:35], a[200:207], v[128:135], v[32:35] cbsz:0 blgp:1\n\t" \
  "ds_read_b64_tr_b8 v[146:147], v162 offset:33792\n\t" \
  "buffer_load_dwordx4 a[72:75], v188, s[36:39], s87 offen offset:2048 \n\t" \
  "v_mfma_f32_16x16x128_f8f6f4 v[64:67], a[208:215], v[128:135], v[64:67] cbsz:0 blgp:1\n\t" \
  "ds_read_b64_tr_b8 v[148:149], v162 offset:40960\n\t" \
  "s_waitcnt vmcnt(47)\n\t" \
  "v_mfma_f32_16x16x128_f8f6f4 v[96:99], a[216:223], v[128:135], v[96:99] cbsz:0 blgp:1\n\t" \
  "ds_read_b64_tr_b8 v[150:151], v162 offset:41984\n\t" \
  "buffer_load_dwordx4 a[76:79], v188, s[36:39], s87 offen offset:3072 \n\t" \
  "s_waitcnt lgkmcnt(4)\n\t" \
  "v_mfma_f32_16x16x128_f8f6f4 v[4:7], a[192:199], v[136:143], v[4:7] cbsz:0 blgp:1\n\t" \
  "ds_read_b64_tr_b8 v[152:153], v163 offset:32768\n\t" \
  "v_mfma_f32_16x16x128_f8f6f4 v[36:39], a[200:207], v[136:143], v[36:39] cbsz:0 blgp:1\n\t" \
  "ds_read_b64_tr_b8 v[154:155], v163 offset:33792\n\t" \
  "buffer_load_dwordx4 a[80:83], v189, s[36:39], s87 offen offset:0 \n\t" \
  "v_mfma_f32_16x16x128_f8f6f4 v[68:71], a[208:215], v[136:143], v[68:71] cbsz:0 blgp:1\n\t" \
  "ds_read_b64_tr_b8 v[156:157], v163 offset:40960\n\t" \
  "v_mfma_f32_16x16x128_f8f6f4 v[100:103], a[216:223], v[136:143], v[100:103] cbsz:0 blgp:1\n\t" \
  "ds_read_b64_tr_b8 v[158:159], v163 offset:41984\n\t" \
  "buffer_load_dwordx4 a[84:87], v189, s[36:39], s87 offen offset:1024 \n\t" \
  "s_waitcnt lgkmcnt(4)\n\t" \
  "v_mfma_f32_16x16x128_f8f6f4 v[8:11], a[192:199], v[144:151], v[8:11] cbsz:0 blgp:1\n\t" \
  "ds_read_b64_tr_b8 v[128:129], v164 offset:32768\n\t" \
  "v_mfma_f32_16x16x128_f8f6f4 v[40:43], a[200:207], v[144:151], v[40:43] cbsz:0 blgp:1\n\t" \
  "ds_read_b64_tr_b8 v[130:131], v164 offset:33792\n\t" \
  "buffer_load_dwordx4 a[88:91], v189, s[36:39], s87 offen offset:2048 \n\t" \
  "v_mfma_f32_16x16x128_f8f6f4 v[72:75], a[208:215], v[144:151], v[72:75] cbsz:0 blgp:1\n\t" \
  "ds_read_b64_tr_b8 v[132:133], v164 offset:40960\n\t" \
  "v_mfma_f32_16x16x128_f8f6f4 v[104:107], a[216:223], v[144:151], v[104:107] cbsz:0 blgp:1\n\t" \
  "ds_read_b64_tr_b8 v[134:135], v164 offset:41984\n\t" \
  "s_cmp_eq_u32 s9, -1\n\t" \
  "s_cbranch_scc0 L_au15_2\n\t" \
  "L_aur15_2:\n\t" \
  "buffer_load_dwordx4 a[92:95], v189, s[36:39], s87 offen offset:3072 \n\t" \
  "s_waitcnt lgkmcnt(4)\n\t" \
  "v_mfma_f32_16x16x128_f8f6f4 v[12:15], a[192:199], v[152:159], v[12:15] cbsz:0 blgp:1\n\t" \
  "ds_read_b64_tr_b8 v[136:137], v165 offset:32768\n\t" \
  "v_mfma_f32_16x16x128_f8f6f4 v[44:47], a[200:207], v[152:159], v[44:47] cbsz:0 blgp:1\n\t" \
  "ds_read_b64_tr_b8 v[138:139], v165 offset:33792\n\t" \
  "s_bfe_u32 s84, s9, 262152\n\t" \
  "s_cmp_eq_u32 s84, 15\n\t" \
  "s_cbranch_scc0 L_q27_2\n\t" \
  "L_qr27_2:\n\t" \
  "s_mov_b32 s87, 1507328\n\t" \
  "buffer_load_dwordx4 a[96:99], v188, s[36:39], s87 offen offset:0 \n\t" \
  "v_mfma_f32_16x16x128_f8f6f4 v[76:79], a[208:215], v[152:159], v[76:79] cbsz:0 blgp:1\n\t" \
  "ds_read_b64_tr_b8 v[140:141], v165 offset:40960\n\t" \
  "v_mfma_f32_16x16x128_f8f6f4 v[108:111], a[216:223], v[152:159], v[108:111] cbsz:0 blgp:1\n\t" \
  "ds_read_b64_tr_b8 v[142:143], v165 offset:41984\n\t" \
  "buffer_load_dwordx4 a[100:103], v188, s[36:39], s87 offen offset:1024 \n\t" \
  "s_waitcnt lgkmcnt(4)\n\t" \
  "v_mfma_f32_16x16x128_f8f6f4 v[16:19], a[192:199], v[128:135], v[16:19] cbsz:0 blgp:1\n\t" \
  "ds_read_b64_tr_b8 v[144:145], v166 offset:32768\n\t" \
  "v_mfma_f32_16x16x128_f8f6f4 v[48:51], a[200:207], v[128:135], v[48:51] cbsz:0 blgp:1\n\t" \
  "ds_read_b64_tr_b8 v[146:147], v166 offset:33792\n\t" \
  "s_add_u32 m0, s76, 0\n\t" \
  "s_add_u32 s90, s73, 6291456\n\t" \
  "buffer_load_dwordx4 v184, s[40:43], s90 offen offset:0  lds\n\t" \
  "v_mfma_f32_16x16x128_f8f6f4 v[80:83], a[208:215], v[128:135], v[80:83] cbsz:0 blgp:1\n\t" \
  "ds_read_b64_tr_b8 v[148:149], v166 offset:40960\n\t" \
  "buffer_load_dwordx4 a[104:107], v188, s[36:39], s87 offen offset:2048 \n\t" \
  "v_mfma_f32_16x16x128_f8f6f4 v[112:115], a[216:223], v[128:135], v[112:115] cbsz:0 blgp:1\n\t" \
  "ds_read_b64_tr_b8 v[150:151], v166 offset:41984\n\t" \
  "s_add_u32 m0, s76, 0\n\t" \
  "s_add_u32 s90, s73, 6291456\n\t" \
  "buffer_load_dwordx4 v185, s[40:43], s90 offen offset:1024  lds\n\t" \
  "s_waitcnt lgkmcnt(4)\n\t" \
  "v_mfma_f32_16x16x128_f8f6f4 v[20:23], a[192:199], v[136:143], v[20:23] cbsz:0 blgp:1\n\t" \
  "ds_read_b64_tr_b8 v[152:153], v167 offset:32768\n\t" \
  "buffer_load_dwordx4 a[108:111], v188, s[36:39], s87 offen offset:3072 \n\t" \
  "v_mfma_f32_16x16x128_f8f6f4 v[52:55], a[200:207], v[136:143], v[52:55] cbsz:0 blgp:1\n\t" \
  "ds_read_b64_tr_b8 v[154:155], v167 offset:33792\n\t" \
  "s_add_u32 m0, s77, 0\n\t" \
  "s_add_u32 s90, s73, 6291456\n\t" \
  "buffer_load_dwordx4 v186, s[40:43], s90 offen offset:0  lds\n\t" \
  "v_mfma_f32_16x16x128_f8f6f4 v[84:87], a[208:215], v[136:143], v[84:87] cbsz:0 blgp:1\n\t" \
  "ds_read_b64_tr_b8 v[156:157], v167 offset:40960\n\t" \
  "buffer_load_dwordx4 a[112:115], v189, s[36:39], s87 offen offset:0 \n\t" \
  "v_mfma_f32_16x16x128_f8f6f4 v[116:119], a[216:223], v[136:143], v[116:119] cbsz:0 blgp:1\n\t" \
  "ds_read_b64_tr_b8 v[158:159], v167 offset:41984\n\t" \
  "s_cmp_eq_u32 s9, -1\n\t" \
  "s_cbranch_scc0 L_ap16_2\n\t" \
  "L_apr16_2:\n\t" \
  "s_add_u32 m0, s77, 0\n\t" \
  "s_add_u32 s90, s73, 6291456\n\t" \
  "buffer_load_dwordx4 v187, s[40:43], s90 offen offset:1024  lds\n\t" \
  "s_waitcnt lgkmcnt(4)\n\t" \
  "v_mfma_f32_16x16x128_f8f6f4 v[24:27], a[192:199], v[144:151], v[24:27] cbsz:0 blgp:1\n\t" \
  "s_barrier\n\t" \
  "ds_read_b64_tr_b8 v[128:129], v160 offset:49152\n\t" \
  "v_mfma_f32_16x16x128_f8f6f4 v[56:59], a[200:207], v[144:151], v[56:59] cbsz:0 blgp:1\n\t" \
  "ds_read_b64_tr_b8 v[130:131], v160 offset:50176\n\t" \
  "s_add_u32 m0, s76, 16384\n\t" \
  "s_add_u32 s90, s74, 6291456\n\t" \
  "buffer_load_dwordx4 v184, s[40:43], s90 offen offset:0  lds\n\t" \
  "v_mfma_f32_16x16x128_f8f6f4 v[88:91], a[208:215], v[144:151], v[88:91] cbsz:0 blgp:1\n\t" \
  "ds_read_b64_tr_b8 v[132:133], v160 offset:57344\n\t" \
  "buffer_load_dwordx4 a[116:119], v189, s[36:39], s87 offen offset:1024 \n\t" \
  "v_mfma_f32_16x16x128_f8f6f4 v[120:123], a[216:223], v[144:151], v[120:123] cbsz:0 blgp:1\n\t" \
  "ds_read_b64_tr_b8 v[134:135], v160 offset:58368\n\t" \
  "s_add_u32 m0, s76, 16384\n\t" \
  "s_add_u32 s90, s74, 6291456\n\t" \
  "buffer_load_dwordx4 v185, s[40:43], s90 offen offset:1024  lds\n\t" \
  "s_waitcnt lgkmcnt(4)\n\t" \
  "v_mfma_f32_16x16x128_f8f6f4 v[28:31], a[192:199], v[152:159], v[28:31] cbsz:0 blgp:1\n\t" \
  "ds_read_b64_tr_b8 v[136:137], v161 offset:49152\n\t" \
  "buffer_load_dwordx4 a[120:123], v189, s[36:39], s87 offen offset:2048 \n\t" \
  "v_mfma_f32_16x16x128_f8f6f4 v[60:63], a[200:207], v[152:159], v[60:63] cbsz:0 blgp:1\n\t" \
  "ds_read_b64_tr_b8 v[138:139], v161 offset:50176\n\t" \
  "s_add_u32 m0, s77, 16384\n\t" \
  "s_add_u32 s90, s74, 6291456\n\t" \
  "buffer_load_dwordx4 v186, s[40:43], s90 offen offset:0  lds\n\t" \
  "v_mfma_f32_16x16x128_f8f6f4 v[92:95], a[208:215], v[152:159], v[92:95] cbsz:0 blgp:1\n\t" \
  "ds_read_b64_tr_b8 v[140:141], v161 offset:57344\n\t" \
  "buffer_load_dwordx4 a[124:127], v189, s[36:39], s87 offen offset:3072 \n\t" \
  "v_mfma_f32_16x16x128_f8f6f4 v[124:127], a[216:223], v[152:159], v[124:127] cbsz:0 blgp:1\n\t" \
  "ds_read_b64_tr_b8 v[142:143], v161 offset:58368\n\t" \
  "s_add_u32 m0, s77, 16384\n\t" \
  "s_add_u32 s90, s74, 6291456\n\t" \
  "buffer_load_dwordx4 v187, s[40:43], s90 offen offset:1024  lds\n\t" \
  "s_waitcnt vmcnt(63) lgkmcnt(4)\n\t" \
  "v_mfma_f32_16x16x128_f8f6f4 v[0:3], a[224:231], v[128:135], v[0:3] cbsz:0 blgp:1\n\t" \
  "ds_read_b64_tr_b8 v[144:145], v162 offset:49152\n\t" \
  "s_bfe_u32 s84, s9, 262156\n\t" \
  "s_cmp_eq_u32 s84, 15\n\t" \
  "s_cbranch_scc0 L_q28_2\n\t" \
  "L_qr28_2:\n\t" \
  "s_mov_b32 s87, 1572864\n\t" \
  "buffer_load_dwordx4 a[128:131], v188, s[36:39], s87 offen offset:0 \n\t" \
  "v_mfma_f32_16x16x128_f8f6f4 v[32:35], a[232:239], v[128:135], v[32:35] cbsz:0 blgp:1\n\t" \
  "ds_read_b64_tr_b8 v[146:147], v162 offset:50176\n\t" \
  "s_waitcnt vmcnt(62)\n\t" \
  "v_mfma_f32_16x16x128_f8f6f4 v[64:67], a[240:247], v[128:135], v[64:67] cbsz:0 blgp:1\n\t" \
  "ds_read_b64_tr_b8 v[148:149], v162 offset:57344\n\t" \
  "buffer_load_dwordx4 a[132:135], v188, s[36:39], s87 offen offset:1024 \n\t" \
  "s_waitcnt vmcnt(61)\n\t" \
  "v_mfma_f32_16x16x128_f8f6f4 v[96:99], a[248:255], v[128:135], v[96:99] cbsz:0 blgp:1\n\t" \
  "ds_read_b64_tr_b8 v[150:151], v162 offset:58368\n\t" \
  "s_waitcnt lgkmcnt(4)\n\t" \
  "v_mfma_f32_16x16x128_f8f6f4 v[4:7], a[224:231], v[136:143], v[4:7] cbsz:0 blgp:1\n\t" \
  "ds_read_b64_tr_b8 v[152:153], v163 offset:49152\n\t" \
  "buffer_load_dwordx4 a[136:139], v188, s[36:39], s87 offen offset:2048 \n\t" \
  "v_mfma_f32_16x16x128_f8f6f4 v[36:39], a[232:239], v[136:143], v[36:39] cbsz:0 blgp:1\n\t" \
  "ds_read_b64_tr_b8 v[154:155], v163 offset:50176\n\t" \
  "v_mfma_f32_16x16x128_f8f6f4 v[68:71], a[240:247], v[136:143], v[68:71] cbsz:0 blgp:1\n\t" \
  "ds_read_b64_tr_b8 v[156:157], v163 offset:57344\n\t" \
  "buffer_load_dwordx4 a[140:143], v188, s[36:39], s87 offen offset:3072 \n\t" \
  "v_mfma_f32_16x16x128_f8f6f4 v[100:103], a[248:255], v[136:143], v[100:103] cbsz:0 blgp:1\n\t" \
  "ds_read_b64_tr_b8 v[158:159], v163 offset:58368\n\t" \
  "s_waitcnt lgkmcnt(4)\n\t" \
  "v_mfma_f32_16x16x128_f8f6f4 v[8:11], a[224:231], v[144:151], v[8:11] cbsz:0 blgp:1\n\t" \
  "ds_read_b64_tr_b8 v[128:129], v164 offset:49152\n\t" \
  "buffer_load_dwordx4 a[144:147], v189, s[36:39], s87 offen offset:0 \n\t" \
  "v_mfma_f32_16x16x128_f8f6f4 v[40:43], a[232:239], v[144:151], v[40:43] cbsz:0 blgp:1\n\t" \
  "ds_read_b64_tr_b8 v[130:131], v164 offset:50176\n\t" \
  "v_mfma_f32_16x16x128_f8f6f4 v[72:75], a[240:247], v[144:151], v[72:75] cbsz:0 blgp:1\n\t" \
  "ds_read_b64_tr_b8 v[132:133], v164 offset:57344\n\t" \
  "buffer_load_dwordx4 a[148:151], v189, s[36:39], s87 offen offset:1024 \n\t" \
  "v_mfma_f32_16x16x128_f8f6f4 v[104:107], a[248:255], v[144:151], v[104:107] cbsz:0 blgp:1\n\t" \
  "ds_read_b64_tr_b8 v[134:135], v164 offset:58368\n\t" \
  "s_cmp_eq_u32 s9, -1\n\t" \
  "s_cbranch_scc0 L_au16_2\n\t" \
  "L_aur16_2:\n\t" \
  "s_waitcnt lgkmcnt(4)\n\t" \
  "v_mfma_f32_16x16x128_f8f6f4 v[12:15], a[224:231], v[152:159], v[12:15] cbsz:0 blgp:1\n\t" \
  "ds_read_b64_tr_b8 v[136:137], v165 offset:49152\n\t" \
  "buffer_load_dwordx4 a[152:155], v189, s[36:39], s87 offen offset:2048 \n\t" \
  "v_mfma_f32_16x16x128_f8f6f4 v[44:47], a[232:239], v[152:159], v[44:47] cbsz:0 blgp:1\n\t" \
  "ds_read_b64_tr_b8 v[138:139], v165 offset:50176\n\t" \
  "v_mfma_f32_16x16x128_f8f6f4 v[76:79], a[240:247], v[152:159], v[76:79] cbsz:0 blgp:1\n\t" \
  "ds_read_b64_tr_b8 v[140:141], v165 offset:57344\n\t" \
  "buffer_load_dwordx4 a[156:159], v189, s[36:39], s87 offen offset:3072 \n\t" \
  "v_mfma_f32_16x16x128_f8f6f4 v[108:111], a[248:255], v[152:159], v[108:111] cbsz:0 blgp:1\n\t" \
  "ds_read_b64_tr_b8 v[142:143], v165 offset:58368\n\t" \
  "s_waitcnt lgkmcnt(4)\n\t" \
  "v_mfma_f32_16x16x128_f8f6f4 v[16:19], a[224:231], v[128:135], v[16:19] cbsz:0 blgp:1\n\t" \
  "ds_read_b64_tr_b8 v[144:145], v166 offset:49152\n\t" \
  "s_bfe_u32 s84, s9, 262160\n\t" \
  "s_cmp_eq_u32 s84, 15\n\t" \
  "s_cbranch_scc0 L_q29_2\n\t" \
  "L_qr29_2:\n\t" \
  "s_mov_b32 s87, 1638400\n\t" \
  "buffer_load_dwordx4 a[160:163], v188, s[36:39], s87 offen offset:0 \n\t" \
  "v_mfma_f32_16x16x128_f8f6f4 v[48:51], a[232:239], v[128:135], v[48:51] cbsz:0 blgp:1\n\t" \
  "ds_read_b64_tr_b8 v[146:147], v166 offset:50176\n\t" \
  "v_mfma_f32_16x16x128_f8f6f4 v[80:83], a[240:247], v[128:135], v[80:83] cbsz:0 blgp:1\n\t" \
  "ds_read_b64_tr_b8 v[148:149], v166 offset:57344\n\t" \
  "buffer_load_dwordx4 a[164:167], v188, s[36:39], s87 offen offset:1024 \n\t" \
  "v_mfma_f32_16x16x128_f8f6f4 v[112:115], a[248:255], v[128:135], v[112:115] cbsz:0 blgp:1\n\t" \
  "ds_read_b64_tr_b8 v[150:151], v166 offset:58368\n\t" \
  "s_waitcnt lgkmcnt(4)\n\t" \
  "v_mfma_f32_16x16x128_f8f6f4 v[20:23], a[224:231], v[136:143], v[20:23] cbsz:0 blgp:1\n\t" \
  "ds_read_b64_tr_b8 v[152:153], v167 offset:49152\n\t" \
  "buffer_load_dwordx4 a[168:171], v188, s[36:39], s87 offen offset:2048 \n\t" \
  "v_mfma_f32_16x16x128_f8f6f4 v[52:55], a[232:239], v[136:143], v[52:55] cbsz:0 blgp:1\n\t" \
  "ds_read_b64_tr_b8 v[154:155], v167 offset:50176\n\t" \
  "v_mfma_f32_16x16x128_f8f6f4 v[84:87], a[240:247], v[136:143], v[84:87] cbsz:0 blgp:1\n\t" \
  "ds_read_b64_tr_b8 v[156:157], v167 offset:57344\n\t" \
  "buffer_load_dwordx4 a[172:175], v188, s[36:39], s87 offen offset:3072 \n\t" \
  "v_mfma_f32_16x16x128_f8f6f4 v[116:119], a[248:255], v[136:143], v[116:119] cbsz:0 blgp:1\n\t" \
  "ds_read_b64_tr_b8 v[158:159], v167 offset:58368\n\t" \
  "s_cmp_eq_u32 s9, -1\n\t" \
  "s_cbranch_scc0 L_ap17_2\n\t" \
  "L_apr17_2:\n\t" \
  "s_waitcnt lgkmcnt(4)\n\t" \
  "v_mfma_f32_16x16x128_f8f6f4 v[24:27], a[224:231], v[144:151], v[24:27] cbsz:0 blgp:1\n\t" \
  "s_barrier\n\t" \
  "ds_read_b64_tr_b8 v[128:129], v168 offset:0\n\t" \
  "v_mfma_f32_16x16x128_f8f6f4 v[56:59], a[232:239], v[144:151], v[56:59] cbsz:0 blgp:1\n\t" \
  "ds_read_b64_tr_b8 v[130:131], v168 offset:1024\n\t" \
  "buffer_load_dwordx4 a[176:179], v189, s[36:39], s87 offen offset:0 \n\t" \
  "v_mfma_f32_16x16x128_f8f6f4 v[88:91], a[240:247], v[144:151], v[88:91] cbsz:0 blgp:1\n\t" \
  "ds_read_b64_tr_b8 v[132:133], v168 offset:8192\n\t" \
  "v_mfma_f32_16x16x128_f8f6f4 v[120:123], a[248:255], v[144:151], v[120:123] cbsz:0 blgp:1\n\t" \
  "ds_read_b64_tr_b8 v[134:135], v168 offset:9216\n\t" \
  "buffer_load_dwordx4 a[180:183], v189, s[36:39], s87 offen offset:1024 \n\t" \
  "s_waitcnt lgkmcnt(4)\n\t" \
  "v_mfma_f32_16x16x128_f8f6f4 v[28:31], a[224:231], v[152:159], v[28:31] cbsz:0 blgp:1\n\t" \
  "s_barrier\n\t" \
  "ds_read_b64_tr_b8 v[136:137], v168 offset:16384\n\t" \
  "v_mfma_f32_16x16x128_f8f6f4 v[60:63], a[232:239], v[152:159], v[60:63] cbsz:0 blgp:1\n\t" \
  "ds_read_b64_tr_b8 v[138:139], v168 offset:17408\n\t" \
  "buffer_load_dwordx4 a[184:187], v189, s[36:39], s87 offen offset:2048 \n\t" \
  "v_mfma_f32_16x16x128_f8f6f4 v[92:95], a[240:247], v[152:159], v[92:95] cbsz:0 blgp:1\n\t" \
  "ds_read_b64_tr_b8 v[140:141], v168 offset:24576\n\t" \
  "v_mfma_f32_16x16x128_f8f6f4 v[124:127], a[248:255], v[152:159], v[124:127] cbsz:0 blgp:1\n\t" \
  "ds_read_b64_tr_b8 v[142:143], v168 offset:25600\n\t" \
  "buffer_load_dwordx4 a[188:191], v189, s[36:39], s87 offen offset:3072 \n\t" \
  "s_waitcnt vmcnt(63) lgkmcnt(4)\n\t" \
  "v_mfma_f32_16x16x128_f8f6f4 v[0:3], a[0:7], v[128:135], v[0:3] cbsz:0 blgp:1\n\t" \
  "s_waitcnt vmcnt(48)\n\t" \
  "s_barrier\n\t" \
  "ds_read_b64_tr_b8 v[144:145], v168 offset:32768\n\t" \
  "v_mfma_f32_16x16x128_f8f6f4 v[32:35], a[8:15], v[128:135], v[32:35] cbsz:0 blgp:1\n\t" \
  "ds_read_b64_tr_b8 v[146:147], v168 offset:33792\n\t" \
  "s_bfe_u32 s84, s9, 262164\n\t" \
  "s_cmp_eq_u32 s84, 15\n\t" \
  "s_cbranch_scc0 L_q30_2\n\t" \
  "L_qr30_2:\n\t" \
  "s_mov_b32 s87, 1703936\n\t" \
  "buffer_load_dwordx4 a[192:195], v188, s[36:39], s87 offen offset:0 \n\t" \
  "v_mfma_f32_16x16x128_f8f6f4 v[64:67], a[16:23], v[128:135], v[64:67] cbsz:0 blgp:1\n\t" \
  "ds_read_b64_tr_b8 v[148:149], v168 offset:40960\n\t" \
  "v_mfma_f32_16x16x128_f8f6f4 v[96:99], a[24:31], v[128:135], v[96:99] cbsz:0 blgp:1\n\t" \
  "ds_read_b64_tr_b8 v[150:151], v168 offset:41984\n\t" \
  "buffer_load_dwordx4 a[196:199], v188, s[36:39], s87 offen offset:1024 \n\t" \
  "s_waitcnt lgkmcnt(4)\n\t" \
  "v_mfma_f32_16x16x128_f8f6f4 v[0:3], a[32:39], v[136:143], v[0:3] cbsz:0 blgp:1\n\t" \
  "ds_read_b64_tr_b8 v[152:153], v169 offset:0\n\t" \
  "v_mfma_f32_16x16x128_f8f6f4 v[32:35], a[40:47], v[136:143], v[32:35] cbsz:0 blgp:1\n\t" \
  "ds_read_b64_tr_b8 v[154:155], v169 offset:1024\n\t" \
  "buffer_load_dwordx4 a[200:203], v188, s[36:39], s87 offen offset:2048 \n\t" \
  "v_mfma_f32_16x16x128_f8f6f4 v[64:67], a[48:55], v[136:143], v[64:67] cbsz:0 blgp:1\n\t" \
  "ds_read_b64_tr_b8 v[156:157], v169 offset:8192\n\t" \
  "s_waitcnt vmcnt(46)\n\t" \
  "v_mfma_f32_16x16x128_f8f6f4 v[96:99], a[56:63], v[136:143], v[96:99] cbsz:0 blgp:1\n\t" \
  "ds_read_b64_tr_b8 v[158:159], v169 offset:9216\n\t" \
  "buffer_load_dwordx4 a[204:207], v188, s[36:39], s87 offen offset:3072 \n\t" \
  "s_waitcnt vmcnt(43) lgkmcnt(4)\n\t" \
  "v_mfma_f32_16x16x128_f8f6f4 v[0:3], a[64:71], v[144:151], v[0:3] cbsz:0 blgp:1\n\t" \
  "s_barrier\n\t" \
  "ds_read_b64_tr_b8 v[128:129], v168 offset:49152\n\t" \
  "s_waitcnt vmcnt(40)\n\t" \
  "v_mfma_f32_16x16x128_f8f6f4 v[32:35], a[72:79], v[144:151], v[32:35] cbsz:0 blgp:1\n\t" \
  "ds_read_b64_tr_b8 v[130:131], v168 offset:50176\n\t" \
  "buffer_load_dwordx4 a[208:211], v189, s[36:39], s87 offen offset:0 \n\t" \
  "s_waitcnt vmcnt(39)\n\t" \
  "v_mfma_f32_16x16x128_f8f6f4 v[64:67], a[80:87], v[144:151], v[64:67] cbsz:0 blgp:1\n\t" \
  "ds_read_b64_tr_b8 v[132:133], v168 offset:57344\n\t" \
  "s_waitcnt vmcnt(37)\n\t" \
  "v_mfma_f32_16x16x128_f8f6f4 v[96:99], a[88:95], v[144:151], v[96:99] cbsz:0 blgp:1\n\t" \
  "ds_read_b64_tr_b8 v[134:135], v168 offset:58368\n\t" \
  "s_cmp_eq_u32 s9, -1\n\t" \
  "s_cbranch_scc0 L_au17_2\n\t" \
  "L_aur17_2:\n\t" \
  "buffer_load_dwordx4 a[212:215], v189, s[36:39], s87 offen offset:1024 \n\t" \
  "s_waitcnt lgkmcnt(4)\n\t" \
  "v_mfma_f32_16x16x128_f8f6f4 v[4:7], a[0:7], v[152:159], v[4:7] cbsz:0 blgp:1\n\t" \
  "ds_read_b64_tr_b8 v[136:137], v169 offset:16384\n\t" \
  "v_mfma_f32_16x16x128_f8f6f4 v[36:39], a[8:15], v[152:159], v[36:39] cbsz:0 blgp:1\n\t" \
  "ds_read_b64_tr_b8 v[138:139], v169 offset:17408\n\t" \
  "buffer_load_dwordx4 a[216:219], v189, s[36:39], s87 offen offset:2048 \n\t" \
  "v_mfma_f32_16x16x128_f8f6f4 v[68:71], a[16:23], v[152:159], v[68:71] cbsz:0 blgp:1\n\t" \
  "ds_read_b64_tr_b8 v[140:141], v169 offset:24576\n\t" \
  "v_mfma_f32_16x16x128_f8f6f4 v[100:103], a[24:31], v[152:159], v[100:103] cbsz:0 blgp:1\n\t" \
  "ds_read_b64_tr_b8 v[142:143], v169 offset:25600\n\t" \
  "buffer_load_dwordx4 a[220:223], v189, s[36:39], s87 offen offset:3072 \n\t" \
  "s_waitcnt vmcnt(38) lgkmcnt(4)\n\t" \
  "v_mfma_f32_16x16x128_f8f6f4 v[0:3], a[96:103], v[128:135], v[0:3] cbsz:0 blgp:1\n\t" \
  "s_barrier\n\t" \
  "ds_read_b64_tr_b8 v[144:145], v176 offset:0\n\t" \
  "s_waitcnt vmcnt(34)\n\t" \
  "v_mfma_f32_16x16x128_f8f6f4 v[32:35], a[104:111], v[128:135], v[32:35] cbsz:0 blgp:1\n\t" \
  "ds_read_b64_tr_b8 v[146:147], v176 offset:1024\n\t" \
  "s_bfe_u32 s84, s9, 262168\n\t" \
  "s_cmp_eq_u32 s84, 15\n\t" \
  "s_cbranch_scc0 L_q31_2\n\t" \
  "L_qr31_2:\n\t" \
  "s_mov_b32 s87, 1769472\n\t" \
  "buffer_load_dwordx4 a[224:227], v188, s[36:39], s87 offen offset:0 \n\t" \
  "s_waitcnt vmcnt(30)\n\t" \
  "v_mfma_f32_16x16x128_f8f6f4 v[64:67], a[112:119], v[128:135], v[64:67] cbsz:0 blgp:1\n\t" \
  "ds_read_b64_tr_b8 v[148:149], v176 offset:8192\n\t" \
  "s_waitcnt vmcnt(26)\n\t" \
  "v_mfma_f32_16x16x128_f8f6f4 v[96:99], a[120:127], v[128:135], v[96:99] cbsz:0 blgp:1\n\t" \
  "ds_read_b64_tr_b8 v[150:151], v176 offset:9216\n\t" \
  "buffer_load_dwordx4 a[228:231], v188, s[36:39], s87 offen offset:1024 \n\t" \
  "s_waitcnt lgkmcnt(4)\n\t" \
  "v_mfma_f32_16x16x128_f8f6f4 v[4:7], a[32:39], v[136:143], v[4:7] cbsz:0 blgp:1\n\t" \
  "ds_read_b64_tr_b8 v[152:153], v169 offset:32768\n\t" \
  "v_mfma_f32_16x16x128_f8f6f4 v[36:39], a[40:47], v[136:143], v[36:39] cbsz:0 blgp:1\n\t" \
  "ds_read_b64_tr_b8 v[154:155], v169 offset:33792\n\t" \
  "buffer_load_dwordx4 a[232:235], v188, s[36:39], s87 offen offset:2048 \n\t" \
  "v_mfma_f32_16x16x128_f8f6f4 v[68:71], a[48:55], v[136:143], v[68:71] cbsz:0 blgp:1\n\t" \
  "ds_read_b64_tr_b8 v[156:157], v169 offset:40960\n\t" \
  "v_mfma_f32_16x16x128_f8f6f4 v[100:103], a[56:63], v[136:143], v[100:103] cbsz:0 blgp:1\n\t" \
  "ds_read_b64_tr_b8 v[158:159], v169 offset:41984\n\t" \
  "buffer_load_dwordx4 a[236:239], v188, s[36:39], s87 offen offset:3072 \n\t" \
  "s_waitcnt vmcnt(26) lgkmcnt(4)\n\t" \
  "v_mfma_f32_16x16x128_f8f6f4 v[0:3], a[128:135], v[144:151], v[0:3] cbsz:0 blgp:1\n\t" \
  "ds_read_b64_tr_b8 v[128:129], v170 offset:0\n\t" \
  "s_waitcnt vmcnt(24)\n\t" \
  "v_mfma_f32_16x16x128_f8f6f4 v[32:35], a[136:143], v[144:151], v[32:35] cbsz:0 blgp:1\n\t" \
  "ds_read_b64_tr_b8 v[130:131], v170 offset:1024\n\t" \
  "buffer_load_dwordx4 a[240:243], v189, s[36:39], s87 offen offset:0 \n\t" \
  "s_waitcnt vmcnt(23)\n\t" \
  "v_mfma_f32_16x16x128_f8f6f4 v[64:67], a[144:151], v[144:151], v[64:67] cbsz:0 blgp:1\n\t" \
  "ds_read_b64_tr_b8 v[132:133], v170 offset:8192\n\t" \
  "s_waitcnt vmcnt(21)\n\t" \
  "v_mfma_f32_16x16x128_f8f6f4 v[96:99], a[152:159], v[144:151], v[96:99] cbsz:0 blgp:1\n\t" \
  "ds_read_b64_tr_b8 v[134:135], v170 offset:9216\n\t" \
  "buffer_load_dwordx4 a[244:247], v189, s[36:39], s87 offen offset:1024 \n\t" \
  "s_waitcnt lgkmcnt(4)\n\t" \
  "v_mfma_f32_16x16x128_f8f6f4 v[4:7], a[64:71], v[152:159], v[4:7] cbsz:0 blgp:1\n\t" \
  "s_barrier\n\t" \
  "ds_read_b64_tr_b8 v[136:137], v176 offset:16384\n\t" \
  "v_mfma_f32_16x16x128_f8f6f4 v[36:39], a[72:79], v[152:159], v[36:39] cbsz:0 blgp:1\n\t" \
  "ds_read_b64_tr_b8 v[138:139], v176 offset:17408\n\t" \
  "buffer_load_dwordx4 a[248:251], v189, s[36:39], s87 offen offset:2048 \n\t" \
  "v_mfma_f32_16x16x128_f8f6f4 v[68:71], a[80:87], v[152:159], v[68:71] cbsz:0 blgp:1\n\t" \
  "ds_read_b64_tr_b8 v[140:141], v176 offset:24576\n\t" \
  "v_mfma_f32_16x16x128_f8f6f4 v[100:103], a[88:95], v[152:159], v[100:103] cbsz:0 blgp:1\n\t" \
  "ds_read_b64_tr_b8 v[142:143], v176 offset:25600\n\t" \
  "buffer_load_dwordx4 a[252:255], v189, s[36:39], s87 offen offset:3072 \n\t" \
  "s_waitcnt lgkmcnt(4)\n\t" \
  "v_mfma_f32_16x16x128_f8f6f4 v[8:11], a[0:7], v[128:135], v[8:11] cbsz:0 blgp:1\n\t" \
  "ds_read_b64_tr_b8 v[144:145], v169 offset:49152\n\t" \
  "v_mfma_f32_16x16x128_f8f6f4 v[40:43], a[8:15], v[128:135], v[40:43] cbsz:0 blgp:1\n\t" \
  "ds_read_b64_tr_b8 v[146:147], v169 offset:50176\n\t" \
  "v_mfma_f32_16x16x128_f8f6f4 v[72:75], a[16:23], v[128:135], v[72:75] cbsz:0 blgp:1\n\t" \
  "ds_read_b64_tr_b8 v[148:149], v169 offset:57344\n\t" \
  "v_mfma_f32_16x16x128_f8f6f4 v[104:107], a[24:31], v[128:135], v[104:107] cbsz:0 blgp:1\n\t" \
  "ds_read_b64_tr_b8 v[150:151], v169 offset:58368\n\t" \
  "s_waitcnt vmcnt(22) lgkmcnt(4)\n\t" \
  "v_mfma_f32_16x16x128_f8f6f4 v[0:3], a[160:167], v[136:143], v[0:3] cbsz:0 blgp:1\n\t" \
  "ds_read_b64_tr_b8 v[152:153], v170 offset:16384\n\t" \
  "s_waitcnt vmcnt(20)\n\t" \
  "v_mfma_f32_16x16x128_f8f6f4 v[32:35], a[168:175], v[136:143], v[32:35] cbsz:0 blgp:1\n\t" \
  "ds_read_b64_tr_b8 v[154:155], v170 offset:17408\n\t" \
  "s_waitcnt vmcnt(18)\n\t" \
  "v_mfma_f32_16x16x128_f8f6f4 v[64:67], a[176:183], v[136:143], v[64:67] cbsz:0 blgp:1\n\t" \
  "ds_read_b64_tr_b8 v[156:157], v170 offset:24576\n\t" \
  "s_waitcnt vmcnt(16)\n\t" \
  "v_mfma_f32_16x16x128_f8f6f4 v[96:99], a[184:191], v[136:143], v[96:99] cbsz:0 blgp:1\n\t" \
  "ds_read_b64_tr_b8 v[158:159], v170 offset:25600\n\t" \
  "s_waitcnt lgkmcnt(4)\n\t" \
  "v_mfma_f32_16x16x128_f8f6f4 v[4:7], a[96:103], v[144:151], v[4:7] cbsz:0 blgp:1\n\t" \
  "s_barrier\n\t" \
  "ds_read_b64_tr_b8 v[128:129], v160 offset:0\n\t" \
  "v_mfma_f32_16x16x128_f8f6f4 v[36:39], a[104:111], v[144:151], v[36:39] cbsz:0 blgp:1\n\t" \
  "ds_read_b64_tr_b8 v[130:131], v160 offset:1024\n\t" \
  "v_mfma_f32_16x16x128_f8f6f4 v[68:71], a[112:119], v[144:151], v[68:71] cbsz:0 blgp:1\n\t" \
  "ds_read_b64_tr_b8 v[132:133], v160 offset:8192\n\t" \
  "v_mfma_f32_16x16x128_f8f6f4 v[100:103], a[120:127], v[144:151], v[100:103] cbsz:0 blgp:1\n\t" \
  "ds_read_b64_tr_b8 v[134:135], v160 offset:9216\n\t" \
  "s_waitcnt lgkmcnt(4)\n\t" \
  "v_mfma_f32_16x16x128_f8f6f4 v[8:11], a[32:39], v[152:159], v[8:11] cbsz:0 blgp:1\n\t" \
  "ds_read_b64_tr_b8 v[136:137], v177 offset:0\n\t" \
  "v_mfma_f32_16x16x128_f8f6f4 v[40:43], a[40:47], v[152:159], v[40:43] cbsz:0 blgp:1\n\t" \
  "ds_read_b64_tr_b8 v[138:139], v177 offset:1024\n\t" \
  "v_mfma_f32_16x16x128_f8f6f4 v[72:75], a[48:55], v[152:159], v[72:75] cbsz:0 blgp:1\n\t" \
  "ds_read_b64_tr_b8 v[140:141], v177 offset:8192\n\t" \
  "v_mfma_f32_16x16x128_f8f6f4 v[104:107], a[56:63], v[152:159], v[104:107] cbsz:0 blgp:1\n\t" \
  "ds_read_b64_tr_b8 v[142:143], v177 offset:9216\n\t" \
  "s_waitcnt vmcnt(14) lgkmcnt(4)\n\t" \
  "v_mfma_f32_16x16x128_f8f6f4 v[0:3], a[192:199], v[128:135], v[0:3] cbsz:0 blgp:1\n\t" \
  "ds_read_b64_tr_b8 v[144:145], v170 offset:32768\n\t" \
  "s_waitcnt vmcnt(12)\n\t" \
  "v_mfma_f32_16x16x128_f8f6f4 v[32:35], a[200:207], v[128:135], v[32:35] cbsz:0 blgp:1\n\t" \
  "ds_read_b64_tr_b8 v[146:147], v170 offset:33792\n\t" \
  "s_waitcnt vmcnt(10)\n\t" \
  "v_mfma_f32_16x16x128_f8f6f4 v[64:67], a[208:215], v[128:135], v[64:67] cbsz:0 blgp:1\n\t" \
  "ds_read_b64_tr_b8 v[148:149], v170 offset:40960\n\t" \
  "s_waitcnt vmcnt(8)\n\t" \
  "v_mfma_f32_16x16x128_f8f6f4 v[96:99], a[216:223], v[128:135], v[96:99] cbsz:0 blgp:1\n\t" \
  "ds_read_b64_tr_b8 v[150:151], v170 offset:41984\n\t" \
  "s_waitcnt lgkmcnt(4)\n\t" \
  "v_mfma_f32_16x16x128_f8f6f4 v[4:7], a[128:135], v[136:143], v[4:7] cbsz:0 blgp:1\n\t" \
  "ds_read_b64_tr_b8 v[152:153], v171 offset:0\n\t" \
  "v_mfma_f32_16x16x128_f8f6f4 v[36:39], a[136:143], v[136:143], v[36:39] cbsz:0 blgp:1\n\t" \
  "ds_read_b64_tr_b8 v[154:155], v171 offset:1024\n\t" \
  "v_mfma_f32_16x16x128_f8f6f4 v[68:71], a[144:151], v[136:143], v[68:71] cbsz:0 blgp:1\n\t" \
  "ds_read_b64_tr_b8 v[156:157], v171 offset:8192\n\t" \
  "v_mfma_f32_16x16x128_f8f6f4 v[100:103], a[152:159], v[136:143], v[100:103] cbsz:0 blgp:1\n\t" \
  "ds_read_b64_tr_b8 v[158:159], v171 offset:9216\n\t" \
  "s_waitcnt lgkmcnt(4)\n\t" \
  "v_mfma_f32_16x16x128_f8f6f4 v[8:11], a[64:71], v[144:151], v[8:11] cbsz:0 blgp:1\n\t" \
  "s_barrier\n\t" \
  "ds_read_b64_tr_b8 v[128:129], v160 offset:16384\n\t" \
  "v_mfma_f32_16x16x128_f8f6f4 v[40:43], a[72:79], v[144:151], v[40:43] cbsz:0 blgp:1\n\t" \
  "ds_read_b64_tr_b8 v[130:131], v160 offset:17408\n\t" \
  "v_mfma_f32_16x16x128_f8f6f4 v[72:75], a[80:87], v[144:151], v[72:75] cbsz:0 blgp:1\n\t" \
  "ds_read_b64_tr_b8 v[132:133], v160 offset:24576\n\t" \
  "v_mfma_f32_16x16x128_f8f6f4 v[104:107], a[88:95], v[144:151], v[104:107] cbsz:0 blgp:1\n\t" \
  "ds_read_b64_tr_b8 v[134:135], v160 offset:25600\n\t" \
  "s_waitcnt lgkmcnt(4)\n\t" \
  "v_mfma_f32_16x16x128_f8f6f4 v[12:15], a[0:7], v[152:159], v[12:15] cbsz:0 blgp:1\n\t" \
  "ds_read_b64_tr_b8 v[136:137], v177 offset:16384\n\t" \
  "v_mfma_f32_16x16x128_f8f6f4 v[44:47], a[8:15], v[152:159], v[44:47] cbsz:0 blgp:1\n\t" \
  "ds_read_b64_tr_b8 v[138:139], v177 offset:17408\n\t" \
  "v_mfma_f32_16x16x128_f8f6f4 v[76:79], a[16:23], v[152:159], v[76:79] cbsz:0 blgp:1\n\t" \
  "ds_read_b64_tr_b8 v[140:141], v177 offset:24576\n\t" \
  "v_mfma_f32_16x16x128_f8f6f4 v[108:111], a[24:31], v[152:159], v[108:111] cbsz:0 blgp:1\n\t" \
  "ds_read_b64_tr_b8 v[142:143], v177 offset:25600\n\t" \
  "s_waitcnt vmcnt(6) lgkmcnt(4)\n\t" \
  "v_mfma_f32_16x16x128_f8f6f4 v[0:3], a[224:231], v[128:135], v[0:3] cbsz:0 blgp:1\n\t" \
  "ds_read_b64_tr_b8 v[144:145], v170 offset:49152\n\t" \
  "s_waitcnt vmcnt(4)\n\t" \
  "v_mfma_f32_16x16x128_f8f6f4 v[32:35], a[232:239], v[128:135], v[32:35] cbsz:0 blgp:1\n\t" \
  "ds_read_b64_tr_b8 v[146:147], v170 offset:50176\n\t" \
  "s_waitcnt vmcnt(2)\n\t" \
  "v_mfma_f32_16x16x128_f8f6f4 v[64:67], a[240:247], v[128:135], v[64:67] cbsz:0 blgp:1\n\t" \
  "ds_read_b64_tr_b8 v[148:149], v170 offset:57344\n\t" \
  "s_waitcnt vmcnt(0)\n\t" \
  "v_mfma_f32_16x16x128_f8f6f4 v[96:99], a[248:255], v[128:135], v[96:99] cbsz:0 blgp:1\n\t" \
  "ds_read_b64_tr_b8 v[150:151], v170 offset:58368\n\t" \
  "s_waitcnt lgkmcnt(4)\n\t" \
  "v_mfma_f32_16x16x128_f8f6f4 v[4:7], a[160:167], v[136:143], v[4:7] cbsz:0 blgp:1\n\t" \
  "ds_read_b64_tr_b8 v[152:153], v171 offset:16384\n\t" \
  "v_mfma_f32_16x16x128_f8f6f4 v[36:39], a[168:175], v[136:143], v[36:39] cbsz:0 blgp:1\n\t" \
  "ds_read_b64_tr_b8 v[154:155], v171 offset:17408\n\t" \
  "v_mfma_f32_16x16x128_f8f6f4 v[68:71], a[176:183], v[136:143], v[68:71] cbsz:0 blgp:1\n\t" \
  "ds_read_b64_tr_b8 v[156:157], v171 offset:24576\n\t" \
  "v_mul_f32 v0, s81, v0\n\t" \
  "v_mfma_f32_16x16x128_f8f6f4 v[100:103], a[184:191], v[136:143], v[100:103] cbsz:0 blgp:1\n\t" \
  "ds_read_b64_tr_b8 v[158:159], v171 offset:25600\n\t" \
  "s_mov_b32 s84, 0\n\t" \
  "buffer_store_dword v0, v194, s[44:47], s84 offen offset:0 sc1\n\t" \
  "v_mul_f32 v1, s81, v1\n\t" \
  "s_waitcnt lgkmcnt(4)\n\t" \
  "v_mfma_f32_16x16x128_f8f6f4 v[8:11], a[96:103], v[144:151], v[8:11] cbsz:0 blgp:1\n\t" \
  "ds_read_b64_tr_b8 v[128:129], v161 offset:0\n\t" \
  "s_mov_b32 s84, 65536\n\t" \
  "buffer_store_dword v1, v194, s[44:47], s84 offen offset:0 sc1\n\t" \
  "v_mul_f32 v2, s81, v2\n\t" \
  "v_mfma_f32_16x16x128_f8f6f4 v[40:43], a[104:111], v[144:151], v[40:43] cbsz:0 blgp:1\n\t" \
  "ds_read_b64_tr_b8 v[130:131], v161 offset:1024\n\t" \
  "s_mov_b32 s84, 131072\n\t" \
  "buffer_store_dword v2, v194, s[44:47], s84 offen offset:0 sc1\n\t" \
  "v_mul_f32 v3, s81, v3\n\t" \
  "v_mfma_f32_16x16x128_f8f6f4 v[72:75], a[112:119], v[144:151], v[72:75] cbsz:0 blgp:1\n\t" \
  "ds_read_b64_tr_b8 v[132:133], v161 offset:8192\n\t" \
  "s_mov_b32 s84, 196608\n\t" \
  "buffer_store_dword v3, v194, s[44:47], s84 offen offset:0 sc1\n\t" \
  "v_mfma_f32_16x16x128_f8f6f4 v[104:107], a[120:127], v[144:151], v[104:107] cbsz:0 blgp:1\n\t" \
  "ds_read_b64_tr_b8 v[134:135], v161 offset:9216\n\t" \
  "s_waitcnt lgkmcnt(4)\n\t" \
  "v_mfma_f32_16x16x128_f8f6f4 v[12:15], a[32:39], v[152:159], v[12:15] cbsz:0 blgp:1\n\t" \
  "ds_read_b64_tr_b8 v[136:137], v178 offset:0\n\t" \
  "v_mul_f32 v32, s81, v32\n\t" \
  "v_mfma_f32_16x16x128_f8f6f4 v[44:47], a[40:47], v[152:159], v[44:47] cbsz:0 blgp:1\n\t" \
  "ds_read_b64_tr_b8 v[138:139], v178 offset:1024\n\t" \
  "s_mov_b32 s84, 1048576\n\t" \
  "buffer_store_dword v32, v194, s[44:47], s84 offen offset:0 sc1\n\t" \
  "v_mul_f32 v33, s81, v33\n\t" \
  "v_mfma_f32_16x16x128_f8f6f4 v[76:79], a[48:55], v[152:159], v[76:79] cbsz:0 blgp:1\n\t" \
  "ds_read_b64_tr_b8 v[140:141], v178 offset:8192\n\t" \
  "s_mov_b32 s84, 1114112\n\t" \
  "buffer_store_dword v33, v194, s[44:47], s84 offen offset:0 sc1\n\t" \
  "v_mul_f32 v34, s81, v34\n\t" \
  "v_mfma_f32_16x16x128_f8f6f4 v[108:111], a[56:63], v[152:159], v[108:111] cbsz:0 blgp:1\n\t" \
  "ds_read_b64_tr_b8 v[142:143], v178 offset:9216\n\t" \
  "s_mov_b32 s84, 1179648\n\t" \
  "buffer_store_dword v34, v194, s[44:47], s84 offen offset:0 sc1\n\t" \
  "v_mul_f32 v35, s81, v35\n\t" \
  "s_waitcnt lgkmcnt(4)\n\t" \
  "v_mfma_f32_16x16x128_f8f6f4 v[4:7], a[192:199], v[128:135], v[4:7] cbsz:0 blgp:1\n\t" \
  "ds_read_b64_tr_b8 v[144:145], v171 offset:32768\n\t" \
  "s_mov_b32 s84, 1245184\n\t" \
  "buffer_store_dword v35, v194, s[44:47], s84 offen offset:0 sc1\n\t" \
  "v_mfma_f32_16x16x128_f8f6f4 v[36:39], a[200:207], v[128:135], v[36:39] cbsz:0 blgp:1\n\t" \
  "ds_read_b64_tr_b8 v[146:147], v171 offset:33792\n\t" \
  "v_mfma_f32_16x16x128_f8f6f4 v[68:71], a[208:215], v[128:135], v[68:71] cbsz:0 blgp:1\n\t" \
  "ds_read_b64_tr_b8 v[148:149], v171 offset:40960\n\t" \
  "v_mul_f32 v64, s81, v64\n\t" \
  "v_mfma_f32_16x16x128_f8f6f4 v[100:103], a[216:223], v[128:135], v[100:103] cbsz:0 blgp:1\n\t" \
  "ds_read_b64_tr_b8 v[150:151], v171 offset:41984\n\t" \
  "s_mov_b32 s84, 2097152\n\t" \
  "buffer_store_dword v64, v194, s[44:47], s84 offen offset:0 sc1\n\t" \
  "v_mul_f32 v65, s81, v65\n\t" \
  "s_waitcnt lgkmcnt(4)\n\t" \
  "v_mfma_f32_16x16x128_f8f6f4 v[8:11], a[128:135], v[136:143], v[8:11] cbsz:0 blgp:1\n\t" \
  "ds_read_b64_tr_b8 v[152:153], v172 offset:0\n\t" \
  "s_mov_b32 s84, 2162688\n\t" \
  "buffer_store_dword v65, v194, s[44:47], s84 offen offset:0 sc1\n\t" \
  "v_mul_f32 v66, s81, v66\n\t" \
  "v_mfma_f32_16x16x128_f8f6f4 v[40:43], a[136:143], v[136:143], v[40:43] cbsz:0 blgp:1\n\t" \
  "ds_read_b64_tr_b8 v[154:155], v172 offset:1024\n\t" \
  "s_mov_b32 s84, 2228224\n\t" \
  "buffer_store_dword v66, v194, s[44:47], s84 offen offset:0 sc1\n\t" \
  "v_mul_f32 v67, s81, v67\n\t" \
  "v_mfma_f32_16x16x128_f8f6f4 v[72:75], a[144:151], v[136:143], v[72:75] cbsz:0 blgp:1\n\t" \
  "ds_read_b64_tr_b8 v[156:157], v172 offset:8192\n\t" \
  "s_mov_b32 s84, 2293760\n\t" \
  "buffer_store_dword v67, v194, s[44:47], s84 offen offset:0 sc1\n\t" \
  "v_mfma_f32_16x16x128_f8f6f4 v[104:107], a[152:159], v[136:143], v[104:107] cbsz:0 blgp:1\n\t" \
  "ds_read_b64_tr_b8 v[158:159], v172 offset:9216\n\t" \
  "s_waitcnt lgkmcnt(4)\n\t" \
  "v_mfma_f32_16x16x128_f8f6f4 v[12:15], a[64:71], v[144:151], v[12:15] cbsz:0 blgp:1\n\t" \
  "ds_read_b64_tr_b8 v[128:129], v161 offset:16384\n\t" \
  "v_mul_f32 v96, s81, v96\n\t" \
  "v_mfma_f32_16x16x128_f8f6f4 v[44:47], a[72:79], v[144:151], v[44:47] cbsz:0 blgp:1\n\t" \
  "ds_read_b64_tr_b8 v[130:131], v161 offset:17408\n\t" \
  "s_mov_b32 s84, 3145728\n\t" \
  "buffer_store_dword v96, v194, s[44:47], s84 offen offset:0 sc1\n\t" \
  "v_mul_f32 v97, s81, v97\n\t" \
  "v_mfma_f32_16x16x128_f8f6f4 v[76:79], a[80:87], v[144:151], v[76:79] cbsz:0 blgp:1\n\t" \
  "ds_read_b64_tr_b8 v[132:133], v161 offset:24576\n\t" \
  "s_mov_b32 s84, 3211264\n\t" \
  "buffer_store_dword v97, v194, s[44:47], s84 offen offset:0 sc1\n\t" \
  "v_mul_f32 v98, s81, v98\n\t" \
  "v_mfma_f32_16x16x128_f8f6f4 v[108:111], a[88:95], v[144:151], v[108:111] cbsz:0 blgp:1\n\t" \
  "ds_read_b64_tr_b8 v[134:135], v161 offset:25600\n\t" \
  "s_mov_b32 s84, 3276800\n\t" \
  "buffer_store_dword v98, v194, s[44:47], s84 offen offset:0 sc1\n\t" \
  "v_mul_f32 v99, s81, v99\n\t" \
  "s_waitcnt lgkmcnt(4)\n\t" \
  "v_mfma_f32_16x16x128_f8f6f4 v[16:19], a[0:7], v[152:159], v[16:19] cbsz:0 blgp:1\n\t" \
  "ds_read_b64_tr_b8 v[136:137], v178 offset:16384\n\t" \
  "s_mov_b32 s84, 3342336\n\t" \
  "buffer_store_dword v99, v194, s[44:47], s84 offen offset:0 sc1\n\t" \
  "v_mfma_f32_16x16x128_f8f6f4 v[48:51], a[8:15], v[152:159], v[48:51] cbsz:0 blgp:1\n\t" \
  "ds_read_b64_tr_b8 v[138:139], v178 offset:17408\n\t" \
  "v_mfma_f32_16x16x128_f8f6f4 v[80:83], a[16:23], v[152:159], v[80:83] cbsz:0 blgp:1\n\t" \
  "ds_read_b64_tr_b8 v[140:141], v178 offset:24576\n\t" \
  "v_mfma_f32_16x16x128_f8f6f4 v[112:115], a[24:31], v[152:159], v[112:115] cbsz:0 blgp:1\n\t" \
  "ds_read_b64_tr_b8 v[142:143], v178 offset:25600\n\t" \
  "s_waitcnt lgkmcnt(4)\n\t" \
  "v_mfma_f32_16x16x128_f8f6f4 v[4:7], a[224:231], v[128:135], v[4:7] cbsz:0 blgp:1\n\t" \
  "ds_read_b64_tr_b8 v[144:145], v171 offset:49152\n\t" \
  "v_mfma_f32_16x16x128_f8f6f4 v[36:39], a[232:239], v[128:135], v[36:39] cbsz:0 blgp:1\n\t" \
  "ds_read_b64_tr_b8 v[146:147], v171 offset:50176\n\t" \
  "v_mfma_f32_16x16x128_f8f6f4 v[68:71], a[240:247], v[128:135], v[68:71] cbsz:0 blgp:1\n\t" \
  "ds_read_b64_tr_b8 v[148:149], v171 offset:57344\n\t" \
  "v_mfma_f32_16x16x128_f8f6f4 v[100:103], a[248:255], v[128:135], v[100:103] cbsz:0 blgp:1\n\t" \
  "ds_read_b64_tr_b8 v[150:151], v171 offset:58368\n\t" \
  "s_waitcnt lgkmcnt(4)\n\t" \
  "v_mfma_f32_16x16x128_f8f6f4 v[8:11], a[160:167], v[136:143], v[8:11] cbsz:0 blgp:1\n\t" \
  "ds_read_b64_tr_b8 v[152:153], v172 offset:16384\n\t" \
  "v_mfma_f32_16x16x128_f8f6f4 v[40:43], a[168:175], v[136:143], v[40:43] cbsz:0 blgp:1\n\t" \
  "ds_read_b64_tr_b8 v[154:155], v172 offset:17408\n\t" \
  "v_mfma_f32_16x16x128_f8f6f4 v[72:75], a[176:183], v[136:143], v[72:75] cbsz:0 blgp:1\n\t" \
  "ds_read_b64_tr_b8 v[156:157], v172 offset:24576\n\t" \
  "v_mul_f32 v4, s81, v4\n\t" \
  "v_mfma_f32_16x16x128_f8f6f4 v[104:107], a[184:191], v[136:143], v[104:107] cbsz:0 blgp:1\n\t" \
  "ds_read_b64_tr_b8 v[158:159], v172 offset:25600\n\t" \
  "s_mov_b32 s84, 0\n\t" \
  "buffer_store_dword v4, v194, s[44:47], s84 offen offset:64 sc1\n\t" \
  "v_mul_f32 v5, s81, v5\n\t" \
  "s_waitcnt lgkmcnt(4)\n\t" \
  "v_mfma_f32_16x16x128_f8f6f4 v[12:15], a[96:103], v[144:151], v[12:15] cbsz:0 blgp:1\n\t" \
  "ds_read_b64_tr_b8 v[128:129], v162 offset:0\n\t" \
  "s_mov_b32 s84, 65536\n\t" \
  "buffer_store_dword v5, v194, s[44:47], s84 offen offset:64 sc1\n\t" \
  "v_mul_f32 v6, s81, v6\n\t" \
  "v_mfma_f32_16x16x128_f8f6f4 v[44:47], a[104:111], v[144:151], v[44:47] cbsz:0 blgp:1\n\t" \
  "ds_read_b64_tr_b8 v[130:131], v162 offset:1024\n\t" \
  "s_mov_b32 s84, 131072\n\t" \
  "buffer_store_dword v6, v194, s[44:47], s84 offen offset:64 sc1\n\t" \
  "v_mul_f32 v7, s81, v7\n\t" \
  "v_mfma_f32_16x16x128_f8f6f4 v[76:79], a[112:119], v[144:151], v[76:79] cbsz:0 blgp:1\n\t" \
  "ds_read_b64_tr_b8 v[132:133], v162 offset:8192\n\t" \
  "s_mov_b32 s84, 196608\n\t" \
  "buffer_store_dword v7, v194, s[44:47], s84 offen offset:64 sc1\n\t" \
  "v_mfma_f32_16x16x128_f8f6f4 v[108:111], a[120:127], v[144:151], v[108:111] cbsz:0 blgp:1\n\t" \
  "ds_read_b64_tr_b8 v[134:135], v162 offset:9216\n\t" \
  "s_waitcnt lgkmcnt(4)\n\t" \
  "v_mfma_f32_16x16x128_f8f6f4 v[16:19], a[32:39], v[152:159], v[16:19] cbsz:0 blgp:1\n\t" \
  "ds_read_b64_tr_b8 v[136:137], v179 offset:0\n\t" \
  "v_mul_f32 v36, s81, v36\n\t" \
  "v_mfma_f32_16x16x128_f8f6f4 v[48:51], a[40:47], v[152:159], v[48:51] cbsz:0 blgp:1\n\t" \
  "ds_read_b64_tr_b8 v[138:139], v179 offset:1024\n\t" \
  "s_mov_b32 s84, 1048576\n\t" \
  "buffer_store_dword v36, v194, s[44:47], s84 offen offset:64 sc1\n\t" \
  "v_mul_f32 v37, s81, v37\n\t" \
  "v_mfma_f32_16x16x128_f8f6f4 v[80:83], a[48:55], v[152:159], v[80:83] cbsz:0 blgp:1\n\t" \
  "ds_read_b64_tr_b8 v[140:141], v179 offset:8192\n\t" \
  "s_mov_b32 s84, 1114112\n\t" \
  "buffer_store_dword v37, v194, s[44:47], s84 offen offset:64 sc1\n\t" \
  "v_mul_f32 v38, s81, v38\n\t" \
  "v_mfma_f32_16x16x128_f8f6f4 v[112:115], a[56:63], v[152:159], v[112:115] cbsz:0 blgp:1\n\t" \
  "ds_read_b64_tr_b8 v[142:143], v179 offset:9216\n\t" \
  "s_mov_b32 s84, 1179648\n\t" \
  "buffer_store_dword v38, v194, s[44:47], s84 offen offset:64 sc1\n\t" \
  "v_mul_f32 v39, s81, v39\n\t" \
  "s_waitcnt lgkmcnt(4)\n\t" \
  "v_mfma_f32_16x16x128_f8f6f4 v[8:11], a[192:199], v[128:135], v[8:11] cbsz:0 blgp:1\n\t" \
  "ds_read_b64_tr_b8 v[144:145], v172 offset:32768\n\t" \
  "s_mov_b32 s84, 1245184\n\t" \
  "buffer_store_dword v39, v194, s[44:47], s84 offen offset:64 sc1\n\t" \
  "v_mfma_f32_16x16x128_f8f6f4 v[40:43], a[200:207], v[128:135], v[40:43] cbsz:0 blgp:1\n\t" \
  "ds_read_b64_tr_b8 v[146:147], v172 offset:33792\n\t" \
  "v_mfma_f32_16x16x128_f8f6f4 v[72:75], a[208:215], v[128:135], v[72:75] cbsz:0 blgp:1\n\t" \
  "ds_read_b64_tr_b8 v[148:149], v172 offset:40960\n\t" \
  "v_mul_f32 v68, s81, v68\n\t" \
  "v_mfma_f32_16x16x128_f8f6f4 v[104:107], a[216:223], v[128:135], v[104:107] cbsz:0 blgp:1\n\t" \
  "ds_read_b64_tr_b8 v[150:151], v172 offset:41984\n\t" \
  "s_mov_b32 s84, 2097152\n\t" \
  "buffer_store_dword v68, v194, s[44:47], s84 offen offset:64 sc1\n\t" \
  "v_mul_f32 v69, s81, v69\n\t" \
  "s_waitcnt lgkmcnt(4)\n\t" \
  "v_mfma_f32_16x16x128_f8f6f4 v[12:15], a[128:135], v[136:143], v[12:15] cbsz:0 blgp:1\n\t" \
  "ds_read_b64_tr_b8 v[152:153], v173 offset:0\n\t" \
  "s_mov_b32 s84, 2162688\n\t" \
  "buffer_store_dword v69, v194, s[44:47], s84 offen offset:64 sc1\n\t" \
  "v_mul_f32 v70, s81, v70\n\t" \
  "v_mfma_f32_16x16x128_f8f6f4 v[44:47], a[136:143], v[136:143], v[44:47] cbsz:0 blgp:1\n\t" \
  "ds_read_b64_tr_b8 v[154:155], v173 offset:1024\n\t" \
  "s_mov_b32 s84, 2228224\n\t" \
  "buffer_store_dword v70, v194, s[44:47], s84 offen offset:64 sc1\n\t" \
  "v_mul_f32 v71, s81, v71\n\t" \
  "v_mfma_f32_16x16x128_f8f6f4 v[76:79], a[144:151], v[136:143], v[76:79] cbsz:0 blgp:1\n\t" \
  "ds_read_b64_tr_b8 v[156:157], v173 offset:8192\n\t" \
  "s_mov_b32 s84, 2293760\n\t" \
  "buffer_store_dword v71, v194, s[44:47], s84 offen offset:64 sc1\n\t" \
  "v_mfma_f32_16x16x128_f8f6f4 v[108:111], a[152:159], v[136:143], v[108:111] cbsz:0 blgp:1\n\t" \
  "ds_read_b64_tr_b8 v[158:159], v173 offset:9216\n\t" \
  "s_waitcnt lgkmcnt(4)\n\t" \
  "v_mfma_f32_16x16x128_f8f6f4 v[16:19], a[64:71], v[144:151], v[16:19] cbsz:0 blgp:1\n\t" \
  "ds_read_b64_tr_b8 v[128:129], v162 offset:16384\n\t" \
  "v_mul_f32 v100, s81, v100\n\t" \
  "v_mfma_f32_16x16x128_f8f6f4 v[48:51], a[72:79], v[144:151], v[48:51] cbsz:0 blgp:1\n\t" \
  "ds_read_b64_tr_b8 v[130:131], v162 offset:17408\n\t" \
  "s_mov_b32 s84, 3145728\n\t" \
  "buffer_store_dword v100, v194, s[44:47], s84 offen offset:64 sc1\n\t" \
  "v_mul_f32 v101, s81, v101\n\t" \
  "v_mfma_f32_16x16x128_f8f6f4 v[80:83], a[80:87], v[144:151], v[80:83] cbsz:0 blgp:1\n\t" \
  "ds_read_b64_tr_b8 v[132:133], v162 offset:24576\n\t" \
  "s_mov_b32 s84, 3211264\n\t" \
  "buffer_store_dword v101, v194, s[44:47], s84 offen offset:64 sc1\n\t" \
  "v_mul_f32 v102, s81, v102\n\t" \
  "v_mfma_f32_16x16x128_f8f6f4 v[112:115], a[88:95], v[144:151], v[112:115] cbsz:0 blgp:1\n\t" \
  "ds_read_b64_tr_b8 v[134:135], v162 offset:25600\n\t" \
  "s_mov_b32 s84, 3276800\n\t" \
  "buffer_store_dword v102, v194, s[44:47], s84 offen offset:64 sc1\n\t" \
  "v_mul_f32 v103, s81, v103\n\t" \
  "s_waitcnt lgkmcnt(4)\n\t" \
  "v_mfma_f32_16x16x128_f8f6f4 v[20:23], a[0:7], v[152:159], v[20:23] cbsz:0 blgp:1\n\t" \
  "ds_read_b64_tr_b8 v[136:137], v179 offset:16384\n\t" \
  "s_mov_b32 s84, 3342336\n\t" \
  "buffer_store_dword v103, v194, s[44:47], s84 offen offset:64 sc1\n\t" \
  "v_mfma_f32_16x16x128_f8f6f4 v[52:55], a[8:15], v[152:159], v[52:55] cbsz:0 blgp:1\n\t" \
  "ds_read_b64_tr_b8 v[138:139], v179 offset:17408\n\t" \
  "v_mfma_f32_16x16x128_f8f6f4 v[84:87], a[16:23], v[152:159], v[84:87] cbsz:0 blgp:1\n\t" \
  "ds_read_b64_tr_b8 v[140:141], v179 offset:24576\n\t" \
  "v_mfma_f32_16x16x128_f8f6f4 v[116:119], a[24:31], v[152:159], v[116:119] cbsz:0 blgp:1\n\t" \
  "ds_read_b64_tr_b8 v[142:143], v179 offset:25600\n\t" \
  "s_waitcnt lgkmcnt(4)\n\t" \
  "v_mfma_f32_16x16x128_f8f6f4 v[8:11], a[224:231], v[128:135], v[8:11] cbsz:0 blgp:1\n\t" \
  "ds_read_b64_tr_b8 v[144:145], v172 offset:49152\n\t" \
  "v_mfma_f32_16x16x128_f8f6f4 v[40:43], a[232:239], v[128:135], v[40:43] cbsz:0 blgp:1\n\t" \
  "ds_read_b64_tr_b8 v[146:147], v172 offset:50176\n\t" \
  "v_mfma_f32_16x16x128_f8f6f4 v[72:75], a[240:247], v[128:135], v[72:75] cbsz:0 blgp:1\n\t" \
  "ds_read_b64_tr_b8 v[148:149], v172 offset:57344\n\t" \
  "v_mfma_f32_16x16x128_f8f6f4 v[104:107], a[248:255], v[128:135], v[104:107] cbsz:0 blgp:1\n\t" \
  "ds_read_b64_tr_b8 v[150:151], v172 offset:58368\n\t" \
  "s_waitcnt lgkmcnt(4)\n\t" \
  "v_mfma_f32_16x16x128_f8f6f4 v[12:15], a[160:167], v[136:143], v[12:15] cbsz:0 blgp:1\n\t" \
  "ds_read_b64_tr_b8 v[152:153], v173 offset:16384\n\t" \
  "v_mfma_f32_16x16x128_f8f6f4 v[44:47], a[168:175], v[136:143], v[44:47] cbsz:0 blgp:1\n\t" \
  "ds_read_b64_tr_b8 v[154:155], v173 offset:17408\n\t" \
  "v_mfma_f32_16x16x128_f8f6f4 v[76:79], a[176:183], v[136:143], v[76:79] cbsz:0 blgp:1\n\t" \
  "ds_read_b64_tr_b8 v[156:157], v173 offset:24576\n\t" \
  "v_mul_f32 v8, s81, v8\n\t" \
  "v_mfma_f32_16x16x128_f8f6f4 v[108:111], a[184:191], v[136:143], v[108:111] cbsz:0 blgp:1\n\t" \
  "ds_read_b64_tr_b8 v[158:159], v173 offset:25600\n\t" \
  "s_mov_b32 s84, 0\n\t" \
  "buffer_store_dword v8, v194, s[44:47], s84 offen offset:128 sc1\n\t" \
  "v_mul_f32 v9, s81, v9\n\t" \
  "s_waitcnt lgkmcnt(4)\n\t" \
  "v_mfma_f32_16x16x128_f8f6f4 v[16:19], a[96:103], v[144:151], v[16:19] cbsz:0 blgp:1\n\t" \
  "ds_read_b64_tr_b8 v[128:129], v163 offset:0\n\t" \
  "s_mov_b32 s84, 65536\n\t" \
  "buffer_store_dword v9, v194, s[44:47], s84 offen offset:128 sc1\n\t" \
  "v_mul_f32 v10, s81, v10\n\t" \
  "v_mfma_f32_16x16x128_f8f6f4 v[48:51], a[104:111], v[144:151], v[48:51] cbsz:0 blgp:1\n\t" \
  "ds_read_b64_tr_b8 v[130:131], v163 offset:1024\n\t" \
  "s_mov_b32 s84, 131072\n\t" \
  "buffer_store_dword v10, v194, s[44:47], s84 offen offset:128 sc1\n\t" \
  "v_mul_f32 v11, s81, v11\n\t" \
  "v_mfma_f32_16x16x128_f8f6f4 v[80:83], a[112:119], v[144:151], v[80:83] cbsz:0 blgp:1\n\t" \
  "ds_read_b64_tr_b8 v[132:133], v163 offset:8192\n\t" \
  "s_mov_b32 s84, 196608\n\t" \
  "buffer_store_dword v11, v194, s[44:47], s84 offen offset:128 sc1\n\t" \
  "v_mfma_f32_16x16x128_f8f6f4 v[112:115], a[120:127], v[144:151], v[112:115] cbsz:0 blgp:1\n\t" \
  "ds_read_b64_tr_b8 v[134:135], v163 offset:9216\n\t" \
  "s_waitcnt lgkmcnt(4)\n\t" \
  "v_mfma_f32_16x16x128_f8f6f4 v[20:23], a[32:39], v[152:159], v[20:23] cbsz:0 blgp:1\n\t" \
  "ds_read_b64_tr_b8 v[136:137], v180 offset:0\n\t" \
  "v_mul_f32 v40, s81, v40\n\t" \
  "v_mfma_f32_16x16x128_f8f6f4 v[52:55], a[40:47], v[152:159], v[52:55] cbsz:0 blgp:1\n\t" \
  "ds_read_b64_tr_b8 v[138:139], v180 offset:1024\n\t" \
  "s_mov_b32 s84, 1048576\n\t" \
  "buffer_store_dword v40, v194, s[44:47], s84 offen offset:128 sc1\n\t" \
  "v_mul_f32 v41, s81, v41\n\t" \
  "v_mfma_f32_16x16x128_f8f6f4 v[84:87], a[48:55], v[152:159], v[84:87] cbsz:0 blgp:1\n\t" \
  "ds_read_b64_tr_b8 v[140:141], v180 offset:8192\n\t" \
  "s_mov_b32 s84, 1114112\n\t" \
  "buffer_store_dword v41, v194, s[44:47], s84 offen offset:128 sc1\n\t" \
  "v_mul_f32 v42, s81, v42\n\t" \
  "v_mfma_f32_16x16x128_f8f6f4 v[116:119], a[56:63], v[152:159], v[116:119] cbsz:0 blgp:1\n\t" \
  "ds_read_b64_tr_b8 v[142:143], v180 offset:9216\n\t" \
  "s_mov_b32 s84, 1179648\n\t" \
  "buffer_store_dword v42, v194, s[44:47], s84 offen offset:128 sc1\n\t" \
  "v_mul_f32 v43, s81, v43\n\t" \
  "s_waitcnt lgkmcnt(4)\n\t" \
  "v_mfma_f32_16x16x128_f8f6f4 v[12:15], a[192:199], v[128:135], v[12:15] cbsz:0 blgp:1\n\t" \
  "ds_read_b64_tr_b8 v[144:145], v173 offset:32768\n\t" \
  "s_mov_b32 s84, 1245184\n\t" \
  "buffer_store_dword v43, v194, s[44:47], s84 offen offset:128 sc1\n\t" \
  "v_mfma_f32_16x16x128_f8f6f4 v[44:47], a[200:207], v[128:135], v[44:47] cbsz:0 blgp:1\n\t" \
  "ds_read_b64_tr_b8 v[146:147], v173 offset:33792\n\t" \
  "v_mfma_f32_16x16x128_f8f6f4 v[76:79], a[208:215], v[128:135], v[76:79] cbsz:0 blgp:1\n\t" \
  "ds_read_b64_tr_b8 v[148:149], v173 offset:40960\n\t" \
  "v_mul_f32 v72, s81, v72\n\t" \
  "v_mfma_f32_16x16x128_f8f6f4 v[108:111], a[216:223], v[128:135], v[108:111] cbsz:0 blgp:1\n\t" \
  "ds_read_b64_tr_b8 v[150:151], v173 offset:41984\n\t" \
  "s_mov_b32 s84, 2097152\n\t" \
  "buffer_store_dword v72, v194, s[44:47], s84 offen offset:128 sc1\n\t" \
  "v_mul_f32 v73, s81, v73\n\t" \
  "s_waitcnt lgkmcnt(4)\n\t" \
  "v_mfma_f32_16x16x128_f8f6f4 v[16:19], a[128:135], v[136:143], v[16:19] cbsz:0 blgp:1\n\t" \
  "ds_read_b64_tr_b8 v[152:153], v174 offset:0\n\t" \
  "s_mov_b32 s84, 2162688\n\t" \
  "buffer_store_dword v73, v194, s[44:47], s84 offen offset:128 sc1\n\t" \
  "v_mul_f32 v74, s81, v74\n\t" \
  "v_mfma_f32_16x16x128_f8f6f4 v[48:51], a[136:143], v[136:143], v[48:51] cbsz:0 blgp:1\n\t" \
  "ds_read_b64_tr_b8 v[154:155], v174 offset:1024\n\t" \
  "s_mov_b32 s84, 2228224\n\t" \
  "buffer_store_dword v74, v194, s[44:47], s84 offen offset:128 sc1\n\t" \
  "v_mul_f32 v75, s81, v75\n\t" \
  "v_mfma_f32_16x16x128_f8f6f4 v[80:83], a[144:151], v[136:143], v[80:83] cbsz:0 blgp:1\n\t" \
  "ds_read_b64_tr_b8 v[156:157], v174 offset:8192\n\t" \
  "s_mov_b32 s84, 2293760\n\t" \
  "buffer_store_dword v75, v194, s[44:47], s84 offen offset:128 sc1\n\t" \
  "v_mfma_f32_16x16x128_f8f6f4 v[112:115], a[152:159], v[136:143], v[112:115] cbsz:0 blgp:1\n\t" \
  "ds_read_b64_tr_b8 v[158:159], v174 offset:9216\n\t" \
  "s_waitcnt lgkmcnt(4)\n\t" \
  "v_mfma_f32_16x16x128_f8f6f4 v[20:23], a[64:71], v[144:151], v[20:23] cbsz:0 blgp:1\n\t" \
  "ds_read_b64_tr_b8 v[128:129], v163 offset:16384\n\t" \
  "v_mul_f32 v104, s81, v104\n\t" \
  "v_mfma_f32_16x16x128_f8f6f4 v[52:55], a[72:79], v[144:151], v[52:55] cbsz:0 blgp:1\n\t" \
  "ds_read_b64_tr_b8 v[130:131], v163 offset:17408\n\t" \
  "s_mov_b32 s84, 3145728\n\t" \
  "buffer_store_dword v104, v194, s[44:47], s84 offen offset:128 sc1\n\t" \
  "v_mul_f32 v105, s81, v105\n\t" \
  "v_mfma_f32_16x16x128_f8f6f4 v[84:87], a[80:87], v[144:151], v[84:87] cbsz:0 blgp:1\n\t" \
  "ds_read_b64_tr_b8 v[132:133], v163 offset:24576\n\t" \
  "s_mov_b32 s84, 3211264\n\t" \
  "buffer_store_dword v105, v194, s[44:47], s84 offen offset:128 sc1\n\t" \
  "v_mul_f32 v106, s81, v106\n\t" \
  "v_mfma_f32_16x16x128_f8f6f4 v[116:119], a[88:95], v[144:151], v[116:119] cbsz:0 blgp:1\n\t" \
  "ds_read_b64_tr_b8 v[134:135], v163 offset:25600\n\t" \
  "s_mov_b32 s84, 3276800\n\t" \
  "buffer_store_dword v106, v194, s[44:47], s84 offen offset:128 sc1\n\t" \
  "v_mul_f32 v107, s81, v107\n\t" \
  "s_waitcnt lgkmcnt(4)\n\t" \
  "v_mfma_f32_16x16x128_f8f6f4 v[24:27], a[0:7], v[152:159], v[24:27] cbsz:0 blgp:1\n\t" \
  "ds_read_b64_tr_b8 v[136:137], v180 offset:16384\n\t" \
  "s_mov_b32 s84, 3342336\n\t" \
  "buffer_store_dword v107, v194, s[44:47], s84 offen offset:128 sc1\n\t" \
  "v_mfma_f32_16x16x128_f8f6f4 v[56:59], a[8:15], v[152:159], v[56:59] cbsz:0 blgp:1\n\t" \
  "ds_read_b64_tr_b8 v[138:139], v180 offset:17408\n\t" \
  "v_mfma_f32_16x16x128_f8f6f4 v[88:91], a[16:23], v[152:159], v[88:91] cbsz:0 blgp:1\n\t" \
  "ds_read_b64_tr_b8 v[140:141], v180 offset:24576\n\t" \
  "v_mfma_f32_16x16x128_f8f6f4 v[120:123], a[24:31], v[152:159], v[120:123] cbsz:0 blgp:1\n\t" \
  "ds_read_b64_tr_b8 v[142:143], v180 offset:25600\n\t" \
  "s_waitcnt lgkmcnt(4)\n\t" \
  "v_mfma_f32_16x16x128_f8f6f4 v[12:15], a[224:231], v[128:135], v[12:15] cbsz:0 blgp:1\n\t" \
  "ds_read_b64_tr_b8 v[144:145], v173 offset:49152\n\t" \
  "v_mfma_f32_16x16x128_f8f6f4 v[44:47], a[232:239], v[128:135], v[44:47] cbsz:0 blgp:1\n\t" \
  "ds_read_b64_tr_b8 v[146:147], v173 offset:50176\n\t" \
  "v_mfma_f32_16x16x128_f8f6f4 v[76:79], a[240:247], v[128:135], v[76:79] cbsz:0 blgp:1\n\t" \
  "ds_read_b64_tr_b8 v[148:149], v173 offset:57344\n\t" \
  "v_mfma_f32_16x16x128_f8f6f4 v[108:111], a[248:255], v[128:135], v[108:111] cbsz:0 blgp:1\n\t" \
  "ds_read_b64_tr_b8 v[150:151], v173 offset:58368\n\t" \
  "s_waitcnt lgkmcnt(4)\n\t" \
  "v_mfma_f32_16x16x128_f8f6f4 v[16:19], a[160:167], v[136:143], v[16:19] cbsz:0 blgp:1\n\t" \
  "ds_read_b64_tr_b8 v[152:153], v174 offset:16384\n\t" \
  "v_mfma_f32_16x16x128_f8f6f4 v[48:51], a[168:175], v[136:143], v[48:51] cbsz:0 blgp:1\n\t" \
  "ds_read_b64_tr_b8 v[154:155], v174 offset:17408\n\t" \
  "v_mfma_f32_16x16x128_f8f6f4 v[80:83], a[176:183], v[136:143], v[80:83] cbsz:0 blgp:1\n\t" \
  "ds_read_b64_tr_b8 v[156:157], v174 offset:24576\n\t" \
  "v_mul_f32 v12, s81, v12\n\t" \
  "v_mfma_f32_16x16x128_f8f6f4 v[112:115], a[184:191], v[136:143], v[112:115] cbsz:0 blgp:1\n\t" \
  "ds_read_b64_tr_b8 v[158:159], v174 offset:25600\n\t" \
  "s_mov_b32 s84, 0\n\t" \
  "buffer_store_dword v12, v194, s[44:47], s84 offen offset:192 sc1\n\t" \
  "v_mul_f32 v13, s81, v13\n\t" \
  "s_waitcnt lgkmcnt(4)\n\t" \
  "v_mfma_f32_16x16x128_f8f6f4 v[20:23], a[96:103], v[144:151], v[20:23] cbsz:0 blgp:1\n\t" \
  "ds_read_b64_tr_b8 v[128:129], v164 offset:0\n\t" \
  "s_mov_b32 s84, 65536\n\t" \
  "buffer_store_dword v13, v194, s[44:47], s84 offen offset:192 sc1\n\t" \
  "v_mul_f32 v14, s81, v14\n\t" \
  "v_mfma_f32_16x16x128_f8f6f4 v[52:55], a[104:111], v[144:151], v[52:55] cbsz:0 blgp:1\n\t" \
  "ds_read_b64_tr_b8 v[130:131], v164 offset:1024\n\t" \
  "s_mov_b32 s84, 131072\n\t" \
  "buffer_store_dword v14, v194, s[44:47], s84 offen offset:192 sc1\n\t" \
  "v_mul_f32 v15, s81, v15\n\t" \
  "v_mfma_f32_16x16x128_f8f6f4 v[84:87], a[112:119], v[144:151], v[84:87] cbsz:0 blgp:1\n\t" \
  "ds_read_b64_tr_b8 v[132:133], v164 offset:8192\n\t" \
  "s_mov_b32 s84, 196608\n\t" \
  "buffer_store_dword v15, v194, s[44:47], s84 offen offset:192 sc1\n\t" \
  "v_mfma_f32_16x16x128_f8f6f4 v[116:119], a[120:127], v[144:151], v[116:119] cbsz:0 blgp:1\n\t" \
  "ds_read_b64_tr_b8 v[134:135], v164 offset:9216\n\t" \
  "s_waitcnt lgkmcnt(4)\n\t" \
  "v_mfma_f32_16x16x128_f8f6f4 v[24:27], a[32:39], v[152:159], v[24:27] cbsz:0 blgp:1\n\t" \
  "ds_read_b64_tr_b8 v[136:137], v181 offset:0\n\t" \
  "v_mul_f32 v44, s81, v44\n\t" \
  "v_mfma_f32_16x16x128_f8f6f4 v[56:59], a[40:47], v[152:159], v[56:59] cbsz:0 blgp:1\n\t" \
  "ds_read_b64_tr_b8 v[138:139], v181 offset:1024\n\t" \
  "s_mov_b32 s84, 1048576\n\t" \
  "buffer_store_dword v44, v194, s[44:47], s84 offen offset:192 sc1\n\t" \
  "v_mul_f32 v45, s81, v45\n\t" \
  "v_mfma_f32_16x16x128_f8f6f4 v[88:91], a[48:55], v[152:159], v[88:91] cbsz:0 blgp:1\n\t" \
  "ds_read_b64_tr_b8 v[140:141], v181 offset:8192\n\t" \
  "s_mov_b32 s84, 1114112\n\t" \
  "buffer_store_dword v45, v194, s[44:47], s84 offen offset:192 sc1\n\t" \
  "v_mul_f32 v46, s81, v46\n\t" \
  "v_mfma_f32_16x16x128_f8f6f4 v[120:123], a[56:63], v[152:159], v[120:123] cbsz:0 blgp:1\n\t" \
  "ds_read_b64_tr_b8 v[142:143], v181 offset:9216\n\t" \
  "s_mov_b32 s84, 1179648\n\t" \
  "buffer_store_dword v46, v194, s[44:47], s84 offen offset:192 sc1\n\t" \
  "v_mul_f32 v47, s81, v47\n\t" \
  "s_waitcnt lgkmcnt(4)\n\t" \
  "v_mfma_f32_16x16x128_f8f6f4 v[16:19], a[192:199], v[128:135], v[16:19] cbsz:0 blgp:1\n\t" \
  "ds_read_b64_tr_b8 v[144:145], v174 offset:32768\n\t" \
  "s_mov_b32 s84, 1245184\n\t" \
  "buffer_store_dword v47, v194, s[44:47], s84 offen offset:192 sc1\n\t" \
  "v_mfma_f32_16x16x128_f8f6f4 v[48:51], a[200:207], v[128:135], v[48:51] cbsz:0 blgp:1\n\t" \
  "ds_read_b64_tr_b8 v[146:147], v174 offset:33792\n\t" \
  "v_mfma_f32_16x16x128_f8f6f4 v[80:83], a[208:215], v[128:135], v[80:83] cbsz:0 blgp:1\n\t" \
  "ds_read_b64_tr_b8 v[148:149], v174 offset:40960\n\t" \
  "v_mul_f32 v76, s81, v76\n\t" \
  "v_mfma_f32_16x16x128_f8f6f4 v[112:115], a[216:223], v[128:135], v[112:115] cbsz:0 blgp:1\n\t" \
  "ds_read_b64_tr_b8 v[150:151], v174 offset:41984\n\t" \
  "s_mov_b32 s84, 2097152\n\t" \
  "buffer_store_dword v76, v194, s[44:47], s84 offen offset:192 sc1\n\t" \
  "v_mul_f32 v77, s81, v77\n\t" \
  "s_waitcnt lgkmcnt(4)\n\t" \
  "v_mfma_f32_16x16x128_f8f6f4 v[20:23], a[128:135], v[136:143], v[20:23] cbsz:0 blgp:1\n\t" \
  "ds_read_b64_tr_b8 v[152:153], v175 offset:0\n\t" \
  "s_mov_b32 s84, 2162688\n\t" \
  "buffer_store_dword v77, v194, s[44:47], s84 offen offset:192 sc1\n\t" \
  "v_mul_f32 v78, s81, v78\n\t" \
  "v_mfma_f32_16x16x128_f8f6f4 v[52:55], a[136:143], v[136:143], v[52:55] cbsz:0 blgp:1\n\t" \
  "ds_read_b64_tr_b8 v[154:155], v175 offset:1024\n\t" \
  "s_mov_b32 s84, 2228224\n\t" \
  "buffer_store_dword v78, v194, s[44:47], s84 offen offset:192 sc1\n\t" \
  "v_mul_f32 v79, s81, v79\n\t" \
  "v_mfma_f32_16x16x128_f8f6f4 v[84:87], a[144:151], v[136:143], v[84:87] cbsz:0 blgp:1\n\t" \
  "ds_read_b64_tr_b8 v[156:157], v175 offset:8192\n\t" \
  "s_mov_b32 s84, 2293760\n\t" \
  "buffer_store_dword v79, v194, s[44:47], s84 offen offset:192 sc1\n\t" \
  "v_mfma_f32_16x16x128_f8f6f4 v[116:119], a[152:159], v[136:143], v[116:119] cbsz:0 blgp:1\n\t" \
  "ds_read_b64_tr_b8 v[158:159], v175 offset:9216\n\t" \
  "s_waitcnt lgkmcnt(4)\n\t" \
  "v_mfma_f32_16x16x128_f8f6f4 v[24:27], a[64:71], v[144:151], v[24:27] cbsz:0 blgp:1\n\t" \
  "ds_read_b64_tr_b8 v[128:129], v164 offset:16384\n\t" \
  "v_mul_f32 v108, s81, v108\n\t" \
  "v_mfma_f32_16x16x128_f8f6f4 v[56:59], a[72:79], v[144:151], v[56:59] cbsz:0 blgp:1\n\t" \
  "ds_read_b64_tr_b8 v[130:131], v164 offset:17408\n\t" \
  "s_mov_b32 s84, 3145728\n\t" \
  "buffer_store_dword v108, v194, s[44:47], s84 offen offset:192 sc1\n\t" \
  "v_mul_f32 v109, s81, v109\n\t" \
  "v_mfma_f32_16x16x128_f8f6f4 v[88:91], a[80:87], v[144:151], v[88:91] cbsz:0 blgp:1\n\t" \
  "ds_read_b64_tr_b8 v[132:133], v164 offset:24576\n\t" \
  "s_mov_b32 s84, 3211264\n\t" \
  "buffer_store_dword v109, v194, s[44:47], s84 offen offset:192 sc1\n\t" \
  "v_mul_f32 v110, s81, v110\n\t" \
  "v_mfma_f32_16x16x128_f8f6f4 v[120:123], a[88:95], v[144:151], v[120:123] cbsz:0 blgp:1\n\t" \
  "ds_read_b64_tr_b8 v[134:135], v164 offset:25600\n\t" \
  "s_mov_b32 s84, 3276800\n\t" \
  "buffer_store_dword v110, v194, s[44:47], s84 offen offset:192 sc1\n\t" \
  "v_mul_f32 v111, s81, v111\n\t" \
  "s_waitcnt lgkmcnt(4)\n\t" \
  "v_mfma_f32_16x16x128_f8f6f4 v[28:31], a[0:7], v[152:159], v[28:31] cbsz:0 blgp:1\n\t" \
  "ds_read_b64_tr_b8 v[136:137], v181 offset:16384\n\t" \
  "s_mov_b32 s84, 3342336\n\t" \
  "buffer_store_dword v111, v194, s[44:47], s84 offen offset:192 sc1\n\t" \
  "v_mfma_f32_16x16x128_f8f6f4 v[60:63], a[8:15], v[152:159], v[60:63] cbsz:0 blgp:1\n\t" \
  "ds_read_b64_tr_b8 v[138:139], v181 offset:17408\n\t" \
  "v_mfma_f32_16x16x128_f8f6f4 v[92:95], a[16:23], v[152:159], v[92:95] cbsz:0 blgp:1\n\t" \
  "ds_read_b64_tr_b8 v[140:141], v181 offset:24576\n\t" \
  "v_mfma_f32_16x16x128_f8f6f4 v[124:127], a[24:31], v[152:159], v[124:127] cbsz:0 blgp:1\n\t" \
  "ds_read_b64_tr_b8 v[142:143], v181 offset:25600\n\t" \
  "s_waitcnt lgkmcnt(4)\n\t" \
  "v_mfma_f32_16x16x128_f8f6f4 v[16:19], a[224:231], v[128:135], v[16:19] cbsz:0 blgp:1\n\t" \
  "ds_read_b64_tr_b8 v[144:145], v174 offset:49152\n\t" \
  "v_mfma_f32_16x16x128_f8f6f4 v[48:51], a[232:239], v[128:135], v[48:51] cbsz:0 blgp:1\n\t" \
  "ds_read_b64_tr_b8 v[146:147], v174 offset:50176\n\t" \
  "v_mfma_f32_16x16x128_f8f6f4 v[80:83], a[240:247], v[128:135], v[80:83] cbsz:0 blgp:1\n\t" \
  "ds_read_b64_tr_b8 v[148:149], v174 offset:57344\n\t" \
  "v_mfma_f32_16x16x128_f8f6f4 v[112:115], a[248:255], v[128:135], v[112:115] cbsz:0 blgp:1\n\t" \
  "ds_read_b64_tr_b8 v[150:151], v174 offset:58368\n\t" \
  "s_waitcnt lgkmcnt(4)\n\t" \
  "v_mfma_f32_16x16x128_f8f6f4 v[20:23], a[160:167], v[136:143], v[20:23] cbsz:0 blgp:1\n\t" \
  "ds_read_b64_tr_b8 v[152:153], v175 offset:16384\n\t" \
  "v_mfma_f32_16x16x128_f8f6f4 v[52:55], a[168:175], v[136:143], v[52:55] cbsz:0 blgp:1\n\t" \
  "ds_read_b64_tr_b8 v[154:155], v175 offset:17408\n\t" \
  "v_mfma_f32_16x16x128_f8f6f4 v[84:87], a[176:183], v[136:143], v[84:87] cbsz:0 blgp:1\n\t" \
  "ds_read_b64_tr_b8 v[156:157], v175 offset:24576\n\t" \
  "v_mul_f32 v16, s81, v16\n\t" \
  "v_mfma_f32_16x16x128_f8f6f4 v[116:119], a[184:191], v[136:143], v[116:119] cbsz:0 blgp:1\n\t" \
  "ds_read_b64_tr_b8 v[158:159], v175 offset:25600\n\t" \
  "s_mov_b32 s84, 0\n\t" \
  "buffer_store_dword v16, v194, s[44:47], s84 offen offset:256 sc1\n\t" \
  "v_mul_f32 v17, s81, v17\n\t" \
  "s_waitcnt lgkmcnt(4)\n\t" \
  "v_mfma_f32_16x16x128_f8f6f4 v[24:27], a[96:103], v[144:151], v[24:27] cbsz:0 blgp:1\n\t" \
  "ds_read_b64_tr_b8 v[128:129], v165 offset:0\n\t" \
  "s_mov_b32 s84, 65536\n\t" \
  "buffer_store_dword v17, v194, s[44:47], s84 offen offset:256 sc1\n\t" \
  "v_mul_f32 v18, s81, v18\n\t" \
  "v_mfma_f32_16x16x128_f8f6f4 v[56:59], a[104:111], v[144:151], v[56:59] cbsz:0 blgp:1\n\t" \
  "ds_read_b64_tr_b8 v[130:131], v165 offset:1024\n\t" \
  "s_mov_b32 s84, 131072\n\t" \
  "buffer_store_dword v18, v194, s[44:47], s84 offen offset:256 sc1\n\t" \
  "v_mul_f32 v19, s81, v19\n\t" \
  "v_mfma_f32_16x16x128_f8f6f4 v[88:91], a[112:119], v[144:151], v[88:91] cbsz:0 blgp:1\n\t" \
  "ds_read_b64_tr_b8 v[132:133], v165 offset:8192\n\t" \
  "s_mov_b32 s84, 196608\n\t" \
  "buffer_store_dword v19, v194, s[44:47], s84 offen offset:256 sc1\n\t" \
  "v_mfma_f32_16x16x128_f8f6f4 v[120:123], a[120:127], v[144:151], v[120:123] cbsz:0 blgp:1\n\t" \
  "ds_read_b64_tr_b8 v[134:135], v165 offset:9216\n\t" \
  "s_waitcnt lgkmcnt(4)\n\t" \
  "v_mfma_f32_16x16x128_f8f6f4 v[28:31], a[32:39], v[152:159], v[28:31] cbsz:0 blgp:1\n\t" \
  "ds_read_b64_tr_b8 v[136:137], v182 offset:0\n\t" \
  "v_mul_f32 v48, s81, v48\n\t" \
  "v_mfma_f32_16x16x128_f8f6f4 v[60:63], a[40:47], v[152:159], v[60:63] cbsz:0 blgp:1\n\t" \
  "ds_read_b64_tr_b8 v[138:139], v182 offset:1024\n\t" \
  "s_mov_b32 s84, 1048576\n\t" \
  "buffer_store_dword v48, v194, s[44:47], s84 offen offset:256 sc1\n\t" \
  "v_mul_f32 v49, s81, v49\n\t" \
  "v_mfma_f32_16x16x128_f8f6f4 v[92:95], a[48:55], v[152:159], v[92:95] cbsz:0 blgp:1\n\t" \
  "ds_read_b64_tr_b8 v[140:141], v182 offset:8192\n\t" \
  "s_mov_b32 s84, 1114112\n\t" \
  "buffer_store_dword v49, v194, s[44:47], s84 offen offset:256 sc1\n\t" \
  "v_mul_f32 v50, s81, v50\n\t" \
  "v_mfma_f32_16x16x128_f8f6f4 v[124:127], a[56:63], v[152:159], v[124:127] cbsz:0 blgp:1\n\t" \
  "ds_read_b64_tr_b8 v[142:143], v182 offset:9216\n\t" \
  "s_mov_b32 s84, 1179648\n\t" \
  "buffer_store_dword v50, v194, s[44:47], s84 offen offset:256 sc1\n\t" \
  "v_mul_f32 v51, s81, v51\n\t" \
  "s_waitcnt lgkmcnt(4)\n\t" \
  "v_mfma_f32_16x16x128_f8f6f4 v[20:23], a[192:199], v[128:135], v[20:23] cbsz:0 blgp:1\n\t" \
  "ds_read_b64_tr_b8 v[144:145], v175 offset:32768\n\t" \
  "s_mov_b32 s84, 1245184\n\t" \
  "buffer_store_dword v51, v194, s[44:47], s84 offen offset:256 sc1\n\t" \
  "v_mfma_f32_16x16x128_f8f6f4 v[52:55], a[200:207], v[128:135], v[52:55] cbsz:0 blgp:1\n\t" \
  "ds_read_b64_tr_b8 v[146:147], v175 offset:33792\n\t" \
  "v_mfma_f32_16x16x128_f8f6f4 v[84:87], a[208:215], v[128:135], v[84:87] cbsz:0 blgp:1\n\t" \
  "ds_read_b64_tr_b8 v[148:149], v175 offset:40960\n\t" \
  "v_mul_f32 v80, s81, v80\n\t" \
  "v_mfma_f32_16x16x128_f8f6f4 v[116:119], a[216:223], v[128:135], v[116:119] cbsz:0 blgp:1\n\t" \
  "ds_read_b64_tr_b8 v[150:151], v175 offset:41984\n\t" \
  "s_mov_b32 s84, 2097152\n\t" \
  "buffer_store_dword v80, v194, s[44:47], s84 offen offset:256 sc1\n\t" \
  "v_mul_f32 v81, s81, v81\n\t" \
  "s_waitcnt lgkmcnt(4)\n\t" \
  "v_mfma_f32_16x16x128_f8f6f4 v[24:27], a[128:135], v[136:143], v[24:27] cbsz:0 blgp:1\n\t" \
  "ds_read_b64_tr_b8 v[152:153], v165 offset:16384\n\t" \
  "s_mov_b32 s84, 2162688\n\t" \
  "buffer_store_dword v81, v194, s[44:47], s84 offen offset:256 sc1\n\t" \
  "v_mul_f32 v82, s81, v82\n\t" \
  "v_mfma_f32_16x16x128_f8f6f4 v[56:59], a[136:143], v[136:143], v[56:59] cbsz:0 blgp:1\n\t" \
  "ds_read_b64_tr_b8 v[154:155], v165 offset:17408\n\t" \
  "s_mov_b32 s84, 2228224\n\t" \
  "buffer_store_dword v82, v194, s[44:47], s84 offen offset:256 sc1\n\t" \
  "v_mul_f32 v83, s81, v83\n\t" \
  "v_mfma_f32_16x16x128_f8f6f4 v[88:91], a[144:151], v[136:143], v[88:91] cbsz:0 blgp:1\n\t" \
  "ds_read_b64_tr_b8 v[156:157], v165 offset:24576\n\t" \
  "s_mov_b32 s84, 2293760\n\t" \
  "buffer_store_dword v83, v194, s[44:47], s84 offen offset:256 sc1\n\t" \
  "v_mfma_f32_16x16x128_f8f6f4 v[120:123], a[152:159], v[136:143], v[120:123] cbsz:0 blgp:1\n\t" \
  "ds_read_b64_tr_b8 v[158:159], v165 offset:25600\n\t" \
  "s_waitcnt lgkmcnt(4)\n\t" \
  "v_mfma_f32_16x16x128_f8f6f4 v[28:31], a[64:71], v[144:151], v[28:31] cbsz:0 blgp:1\n\t" \
  "ds_read_b64_tr_b8 v[128:129], v182 offset:16384\n\t" \
  "v_mul_f32 v112, s81, v112\n\t" \
  "v_mfma_f32_16x16x128_f8f6f4 v[60:63], a[72:79], v[144:151], v[60:63] cbsz:0 blgp:1\n\t" \
  "ds_read_b64_tr_b8 v[130:131], v182 offset:17408\n\t" \
  "s_mov_b32 s84, 3145728\n\t" \
  "buffer_store_dword v112, v194, s[44:47], s84 offen offset:256 sc1\n\t" \
  "v_mul_f32 v113, s81, v113\n\t" \
  "v_mfma_f32_16x16x128_f8f6f4 v[92:95], a[80:87], v[144:151], v[92:95] cbsz:0 blgp:1\n\t" \
  "ds_read_b64_tr_b8 v[132:133], v182 offset:24576\n\t" \
  "s_mov_b32 s84, 3211264\n\t" \
  "buffer_store_dword v113, v194, s[44:47], s84 offen offset:256 sc1\n\t" \
  "v_mul_f32 v114, s81, v114\n\t" \
  "v_mfma_f32_16x16x128_f8f6f4 v[124:127], a[88:95], v[144:151], v[124:127] cbsz:0 blgp:1\n\t" \
  "ds_read_b64_tr_b8 v[134:135], v182 offset:25600\n\t" \
  "s_mov_b32 s84, 3276800\n\t" \
  "buffer_store_dword v114, v194, s[44:47], s84 offen offset:256 sc1\n\t" \
  "v_mul_f32 v115, s81, v115\n\t" \
  "s_waitcnt lgkmcnt(4)\n\t" \
  "v_mfma_f32_16x16x128_f8f6f4 v[20:23], a[224:231], v[152:159], v[20:23] cbsz:0 blgp:1\n\t" \
  "ds_read_b64_tr_b8 v[136:137], v175 offset:49152\n\t" \
  "s_mov_b32 s84, 3342336\n\t" \
  "buffer_store_dword v115, v194, s[44:47], s84 offen offset:256 sc1\n\t" \
  "v_mfma_f32_16x16x128_f8f6f4 v[52:55], a[232:239], v[152:159], v[52:55] cbsz:0 blgp:1\n\t" \
  "ds_read_b64_tr_b8 v[138:139], v175 offset:50176\n\t" \
  "v_mfma_f32_16x16x128_f8f6f4 v[84:87], a[240:247], v[152:159], v[84:87] cbsz:0 blgp:1\n\t" \
  "ds_read_b64_tr_b8 v[140:141], v175 offset:57344\n\t" \
  "v_mfma_f32_16x16x128_f8f6f4 v[116:119], a[248:255], v[152:159], v[116:119] cbsz:0 blgp:1\n\t" \
  "ds_read_b64_tr_b8 v[142:143], v175 offset:58368\n\t" \
  "s_waitcnt lgkmcnt(4)\n\t" \
  "v_mfma_f32_16x16x128_f8f6f4 v[24:27], a[160:167], v[128:135], v[24:27] cbsz:0 blgp:1\n\t" \
  "ds_read_b64_tr_b8 v[144:145], v166 offset:0\n\t" \
  "v_mfma_f32_16x16x128_f8f6f4 v[56:59], a[168:175], v[128:135], v[56:59] cbsz:0 blgp:1\n\t" \
  "ds_read_b64_tr_b8 v[146:147], v166 offset:1024\n\t" \
  "v_mfma_f32_16x16x128_f8f6f4 v[88:91], a[176:183], v[128:135], v[88:91] cbsz:0 blgp:1\n\t" \
  "ds_read_b64_tr_b8 v[148:149], v166 offset:8192\n\t" \
  "v_mul_f32 v20, s81, v20\n\t" \
  "v_mfma_f32_16x16x128_f8f6f4 v[120:123], a[184:191], v[128:135], v[120:123] cbsz:0 blgp:1\n\t" \
  "ds_read_b64_tr_b8 v[150:151], v166 offset:9216\n\t" \
  "s_mov_b32 s84, 0\n\t" \
  "buffer_store_dword v20, v194, s[44:47], s84 offen offset:320 sc1\n\t" \
  "v_mul_f32 v21, s81, v21\n\t" \
  "s_waitcnt lgkmcnt(4)\n\t" \
  "v_mfma_f32_16x16x128_f8f6f4 v[28:31], a[96:103], v[136:143], v[28:31] cbsz:0 blgp:1\n\t" \
  "ds_read_b64_tr_b8 v[152:153], v183 offset:0\n\t" \
  "s_mov_b32 s84, 65536\n\t" \
  "buffer_store_dword v21, v194, s[44:47], s84 offen offset:320 sc1\n\t" \
  "v_mul_f32 v22, s81, v22\n\t" \
  "v_mfma_f32_16x16x128_f8f6f4 v[60:63], a[104:111], v[136:143], v[60:63] cbsz:0 blgp:1\n\t" \
  "ds_read_b64_tr_b8 v[154:155], v183 offset:1024\n\t" \
  "s_mov_b32 s84, 131072\n\t" \
  "buffer_store_dword v22, v194, s[44:47], s84 offen offset:320 sc1\n\t" \
  "v_mul_f32 v23, s81, v23\n\t" \
  "v_mfma_f32_16x16x128_f8f6f4 v[92:95], a[112:119], v[136:143], v[92:95] cbsz:0 blgp:1\n\t" \
  "ds_read_b64_tr_b8 v[156:157], v183 offset:8192\n\t" \
  "s_mov_b32 s84, 196608\n\t" \
  "buffer_store_dword v23, v194, s[44:47], s84 offen offset:320 sc1\n\t" \
  "v_mfma_f32_16x16x128_f8f6f4 v[124:127], a[120:127], v[136:143], v[124:127] cbsz:0 blgp:1\n\t" \
  "ds_read_b64_tr_b8 v[158:159], v183 offset:9216\n\t" \
  "s_waitcnt lgkmcnt(4)\n\t" \
  "v_mfma_f32_16x16x128_f8f6f4 v[24:27], a[192:199], v[144:151], v[24:27] cbsz:0 blgp:1\n\t" \
  "ds_read_b64_tr_b8 v[128:129], v166 offset:16384\n\t" \
  "v_mul_f32 v52, s81, v52\n\t" \
  "v_mfma_f32_16x16x128_f8f6f4 v[56:59], a[200:207], v[144:151], v[56:59] cbsz:0 blgp:1\n\t" \
  "ds_read_b64_tr_b8 v[130:131], v166 offset:17408\n\t" \
  "s_mov_b32 s84, 1048576\n\t" \
  "buffer_store_dword v52, v194, s[44:47], s84 offen offset:320 sc1\n\t" \
  "v_mul_f32 v53, s81, v53\n\t" \
  "v_mfma_f32_16x16x128_f8f6f4 v[88:91], a[208:215], v[144:151], v[88:91] cbsz:0 blgp:1\n\t" \
  "ds_read_b64_tr_b8 v[132:133], v166 offset:24576\n\t" \
  "s_mov_b32 s84, 1114112\n\t" \
  "buffer_store_dword v53, v194, s[44:47], s84 offen offset:320 sc1\n\t" \
  "v_mul_f32 v54, s81, v54\n\t" \
  "v_mfma_f32_16x16x128_f8f6f4 v[120:123], a[216:223], v[144:151], v[120:123] cbsz:0 blgp:1\n\t" \
  "ds_read_b64_tr_b8 v[134:135], v166 offset:25600\n\t" \
  "s_mov_b32 s84, 1179648\n\t" \
  "buffer_store_dword v54, v194, s[44:47], s84 offen offset:320 sc1\n\t" \
  "v_mul_f32 v55, s81, v55\n\t" \
  "s_waitcnt lgkmcnt(4)\n\t" \
  "v_mfma_f32_16x16x128_f8f6f4 v[28:31], a[128:135], v[152:159], v[28:31] cbsz:0 blgp:1\n\t" \
  "ds_read_b64_tr_b8 v[136:137], v183 offset:16384\n\t" \
  "s_mov_b32 s84, 1245184\n\t" \
  "buffer_store_dword v55, v194, s[44:47], s84 offen offset:320 sc1\n\t" \
  "v_mfma_f32_16x16x128_f8f6f4 v[60:63], a[136:143], v[152:159], v[60:63] cbsz:0 blgp:1\n\t" \
  "ds_read_b64_tr_b8 v[138:139], v183 offset:17408\n\t" \
  "v_mfma_f32_16x16x128_f8f6f4 v[92:95], a[144:151], v[152:159], v[92:95] cbsz:0 blgp:1\n\t" \
  "ds_read_b64_tr_b8 v[140:141], v183 offset:24576\n\t" \
  "v_mul_f32 v84, s81, v84\n\t" \
  "v_mfma_f32_16x16x128_f8f6f4 v[124:127], a[152:159], v[152:159], v[124:127] cbsz:0 blgp:1\n\t" \
  "ds_read_b64_tr_b8 v[142:143], v183 offset:25600\n\t" \
  "s_mov_b32 s84, 2097152\n\t" \
  "buffer_store_dword v84, v194, s[44:47], s84 offen offset:320 sc1\n\t" \
  "v_mul_f32 v85, s81, v85\n\t" \
  "s_waitcnt lgkmcnt(4)\n\t" \
  "v_mfma_f32_16x16x128_f8f6f4 v[24:27], a[224:231], v[128:135], v[24:27] cbsz:0 blgp:1\n\t" \
  "ds_read_b64_tr_b8 v[144:145], v167 offset:0\n\t" \
  "s_mov_b32 s84, 2162688\n\t" \
  "buffer_store_dword v85, v194, s[44:47], s84 offen offset:320 sc1\n\t" \
  "v_mul_f32 v86, s81, v86\n\t" \
  "v_mfma_f32_16x16x128_f8f6f4 v[56:59], a[232:239], v[128:135], v[56:59] cbsz:0 blgp:1\n\t" \
  "ds_read_b64_tr_b8 v[146:147], v167 offset:1024\n\t" \
  "s_mov_b32 s84, 2228224\n\t" \
  "buffer_store_dword v86, v194, s[44:47], s84 offen offset:320 sc1\n\t" \
  "v_mul_f32 v87, s81, v87\n\t" \
  "v_mfma_f32_16x16x128_f8f6f4 v[88:91], a[240:247], v[128:135], v[88:91] cbsz:0 blgp:1\n\t" \
  "ds_read_b64_tr_b8 v[148:149], v167 offset:8192\n\t" \
  "s_mov_b32 s84, 2293760\n\t" \
  "buffer_store_dword v87, v194, s[44:47], s84 offen offset:320 sc1\n\t" \
  "v_mfma_f32_16x16x128_f8f6f4 v[120:123], a[248:255], v[128:135], v[120:123] cbsz:0 blgp:1\n\t" \
  "ds_read_b64_tr_b8 v[150:151], v167 offset:9216\n\t" \
  "s_waitcnt lgkmcnt(4)\n\t" \
  "v_mfma_f32_16x16x128_f8f6f4 v[28:31], a[160:167], v[136:143], v[28:31] cbsz:0 blgp:1\n\t" \
  "ds_read_b64_tr_b8 v[152:153], v167 offset:16384\n\t" \
  "v_mul_f32 v116, s81, v116\n\t" \
  "v_mfma_f32_16x16x128_f8f6f4 v[60:63], a[168:175], v[136:143], v[60:63] cbsz:0 blgp:1\n\t" \
  "ds_read_b64_tr_b8 v[154:155], v167 offset:17408\n\t" \
  "s_mov_b32 s84, 3145728\n\t" \
  "buffer_store_dword v116, v194, s[44:47], s84 offen offset:320 sc1\n\t" \
  "v_mul_f32 v117, s81, v117\n\t" \
  "v_mfma_f32_16x16x128_f8f6f4 v[92:95], a[176:183], v[136:143], v[92:95] cbsz:0 blgp:1\n\t" \
  "ds_read_b64_tr_b8 v[156:157], v167 offset:24576\n\t" \
  "s_mov_b32 s84, 3211264\n\t" \
  "buffer_store_dword v117, v194, s[44:47], s84 offen offset:320 sc1\n\t" \
  "v_mul_f32 v118, s81, v118\n\t" \
  "v_mfma_f32_16x16x128_f8f6f4 v[124:127], a[184:191], v[136:143], v[124:127] cbsz:0 blgp:1\n\t" \
  "ds_read_b64_tr_b8 v[158:159], v167 offset:25600\n\t" \
  "s_mov_b32 s84, 3276800\n\t" \
  "buffer_store_dword v118, v194, s[44:47], s84 offen offset:320 sc1\n\t" \
  "v_mul_f32 v119, s81, v119\n\t" \
  "s_waitcnt lgkmcnt(4)\n\t" \
  "v_mfma_f32_16x16x128_f8f6f4 v[28:31], a[192:199], v[144:151], v[28:31] cbsz:0 blgp:1\n\t" \
  "s_mov_b32 s84, 3342336\n\t" \
  "buffer_store_dword v119, v194, s[44:47], s84 offen offset:320 sc1\n\t" \
  "v_mfma_f32_16x16x128_f8f6f4 v[60:63], a[200:207], v[144:151], v[60:63] cbsz:0 blgp:1\n\t" \
  "v_mfma_f32_16x16x128_f8f6f4 v[92:95], a[208:215], v[144:151], v[92:95] cbsz:0 blgp:1\n\t" \
  "v_mul_f32 v24, s81, v24\n\t" \
  "v_mfma_f32_16x16x128_f8f6f4 v[124:127], a[216:223], v[144:151], v[124:127] cbsz:0 blgp:1\n\t" \
  "s_mov_b32 s84, 0\n\t" \
  "buffer_store_dword v24, v194, s[44:47], s84 offen offset:384 sc1\n\t" \
  "v_mul_f32 v25, s81, v25\n\t" \
  "s_waitcnt lgkmcnt(0)\n\t" \
  "v_mfma_f32_16x16x128_f8f6f4 v[28:31], a[224:231], v[152:159], v[28:31] cbsz:0 blgp:1\n\t" \
  "s_mov_b32 s84, 65536\n\t" \
  "buffer_store_dword v25, v194, s[44:47], s84 offen offset:384 sc1\n\t" \
  "v_mul_f32 v26, s81, v26\n\t" \
  "v_mfma_f32_16x16x128_f8f6f4 v[60:63], a[232:239], v[152:159], v[60:63] cbsz:0 blgp:1\n\t" \
  "s_mov_b32 s84, 131072\n\t" \
  "buffer_store_dword v26, v194, s[44:47], s84 offen offset:384 sc1\n\t" \
  "v_mul_f32 v27, s81, v27\n\t" \
  "v_mfma_f32_16x16x128_f8f6f4 v[92:95], a[240:247], v[152:159], v[92:95] cbsz:0 blgp:1\n\t" \
  "s_mov_b32 s84, 196608\n\t" \
  "buffer_store_dword v27, v194, s[44:47], s84 offen offset:384 sc1\n\t" \
  "v_mfma_f32_16x16x128_f8f6f4 v[124:127], a[248:255], v[152:159], v[124:127] cbsz:0 blgp:1\n\t" \
  "s_nop 7\n\t" \
  "s_nop 3\n\t" \
  "v_mul_f32 v56, s81, v56\n\t" \
  "v_mul_f32 v57, s81, v57\n\t" \
  "v_mul_f32 v58, s81, v58\n\t" \
  "v_mul_f32 v59, s81, v59\n\t" \
  "s_mov_b32 s84, 1048576\n\t" \
  "buffer_store_dword v56, v194, s[44:47], s84 offen offset:384 sc1\n\t" \
  "s_mov_b32 s84, 1114112\n\t" \
  "buffer_store_dword v57, v194, s[44:47], s84 offen offset:384 sc1\n\t" \
  "s_mov_b32 s84, 1179648\n\t" \
  "buffer_store_dword v58, v194, s[44:47], s84 offen offset:384 sc1\n\t" \
  "s_mov_b32 s84, 1245184\n\t" \
  "buffer_store_dword v59, v194, s[44:47], s84 offen offset:384 sc1\n\t" \
  "v_mul_f32 v88, s81, v88\n\t" \
  "v_mul_f32 v89, s81, v89\n\t" \
  "v_mul_f32 v90, s81, v90\n\t" \
  "v_mul_f32 v91, s81, v91\n\t" \
  "s_mov_b32 s84, 2097152\n\t" \
  "buffer_store_dword v88, v194, s[44:47], s84 offen offset:384 sc1\n\t" \
  "s_mov_b32 s84, 2162688\n\t" \
  "buffer_store_dword v89, v194, s[44:47], s84 offen offset:384 sc1\n\t" \
  "s_mov_b32 s84, 2228224\n\t" \
  "buffer_store_dword v90, v194, s[44:47], s84 offen offset:384 sc1\n\t" \
  "s_mov_b32 s84, 2293760\n\t" \
  "buffer_store_dword v91, v194, s[44:47], s84 offen offset:384 sc1\n\t" \
  "v_mul_f32 v120, s81, v120\n\t" \
  "v_mul_f32 v121, s81, v121\n\t" \
  "v_mul_f32 v122, s81, v122\n\t" \
  "v_mul_f32 v123, s81, v123\n\t" \
  "s_mov_b32 s84, 3145728\n\t" \
  "buffer_store_dword v120, v194, s[44:47], s84 offen offset:384 sc1\n\t" \
  "s_mov_b32 s84, 3211264\n\t" \
  "buffer_store_dword v121, v194, s[44:47], s84 offen offset:384 sc1\n\t" \
  "s_mov_b32 s84, 3276800\n\t" \
  "buffer_store_dword v122, v194, s[44:47], s84 offen offset:384 sc1\n\t" \
  "s_mov_b32 s84, 3342336\n\t" \
  "buffer_store_dword v123, v194, s[44:47], s84 offen offset:384 sc1\n\t" \
  "v_mul_f32 v28, s81, v28\n\t" \
  "v_mul_f32 v29, s81, v29\n\t" \
  "v_mul_f32 v30, s81, v30\n\t" \
  "v_mul_f32 v31, s81, v31\n\t" \
  "s_mov_b32 s84, 0\n\t" \
  "buffer_store_dword v28, v194, s[44:47], s84 offen offset:448 sc1\n\t" \
  "s_mov_b32 s84, 65536\n\t" \
  "buffer_store_dword v29, v194, s[44:47], s84 offen offset:448 sc1\n\t" \
  "s_mov_b32 s84, 131072\n\t" \
  "buffer_store_dword v30, v194, s[44:47], s84 offen offset:448 sc1\n\t" \
  "s_mov_b32 s84, 196608\n\t" \
  "buffer_store_dword v31, v194, s[44:47], s84 offen offset:448 sc1\n\t" \
  "v_mul_f32 v60, s81, v60\n\t" \
  "v_mul_f32 v61, s81, v61\n\t" \
  "v_mul_f32 v62, s81, v62\n\t" \
  "v_mul_f32 v63, s81, v63\n\t" \
  "s_mov_b32 s84, 1048576\n\t" \
  "buffer_store_dword v60, v194, s[44:47], s84 offen offset:448 sc1\n\t" \
  "s_mov_b32 s84, 1114112\n\t" \
  "buffer_store_dword v61, v194, s[44:47], s84 offen offset:448 sc1\n\t" \
  "s_mov_b32 s84, 1179648\n\t" \
  "buffer_store_dword v62, v194, s[44:47], s84 offen offset:448 sc1\n\t" \
  "s_mov_b32 s84, 1245184\n\t" \
  "buffer_store_dword v63, v194, s[44:47], s84 offen offset:448 sc1\n\t" \
  "v_mul_f32 v92, s81, v92\n\t" \
  "v_mul_f32 v93, s81, v93\n\t" \
  "v_mul_f32 v94, s81, v94\n\t" \
  "v_mul_f32 v95, s81, v95\n\t" \
  "s_mov_b32 s84, 2097152\n\t" \
  "buffer_store_dword v92, v194, s[44:47], s84 offen offset:448 sc1\n\t" \
  "s_mov_b32 s84, 2162688\n\t" \
  "buffer_store_dword v93, v194, s[44:47], s84 offen offset:448 sc1\n\t" \
  "s_mov_b32 s84, 2228224\n\t" \
  "buffer_store_dword v94, v194, s[44:47], s84 offen offset:448 sc1\n\t" \
  "s_mov_b32 s84, 2293760\n\t" \
  "buffer_store_dword v95, v194, s[44:47], s84 offen offset:448 sc1\n\t" \
  "v_mul_f32 v124, s81, v124\n\t" \
  "v_mul_f32 v125, s81, v125\n\t" \
  "v_mul_f32 v126, s81, v126\n\t" \
  "v_mul_f32 v127, s81, v127\n\t" \
  "s_mov_b32 s84, 3145728\n\t" \
  "buffer_store_dword v124, v194, s[44:47], s84 offen offset:448 sc1\n\t" \
  "s_mov_b32 s84, 3211264\n\t" \
  "buffer_store_dword v125, v194, s[44:47], s84 offen offset:448 sc1\n\t" \
  "s_mov_b32 s84, 3276800\n\t" \
  "buffer_store_dword v126, v194, s[44:47], s84 offen offset:448 sc1\n\t" \
  "s_mov_b32 s84, 3342336\n\t" \
  "buffer_store_dword v127, v194, s[44:47], s84 offen offset:448 sc1\n\t" \
  "s_endpgm\n\t" \
  "L_slow_2:\n\t" \
  "L_ap2_2:\n\t" \
  "global_load_dword v196, v195, s[26:27] offset:1024 sc1\n\t" \
  "s_branch L_apr2_2\n\t" \
  "L_ap3_2:\n\t" \
  "global_load_dword v196, v195, s[26:27] offset:1024 sc1\n\t" \
  "s_branch L_apr3_2\n\t" \
  "L_ap5_2:\n\t" \
  "global_load_dword v196, v195, s[26:27] offset:2048 sc1\n\t" \
  "s_branch L_apr5_2\n\t" \
  "L_ap6_2:\n\t" \
  "global_load_dword v196, v195, s[26:27] offset:2048 sc1\n\t" \
  "s_branch L_apr6_2\n\t" \
  "L_ap7_2:\n\t" \
  "global_load_dword v196, v195, s[26:27] offset:2048 sc1\n\t" \
  "s_branch L_apr7_2\n\t" \
  "L_ap8_2:\n\t" \
  "global_load_dword v196, v195, s[26:27] offset:2048 sc1\n\t" \
  "s_branch L_apr8_2\n\t" \
  "L_ap10_2:\n\t" \
  "global_load_dword v196, v195, s[26:27] offset:3072 sc1\n\t" \
  "s_branch L_apr10_2\n\t" \
  "L_ap11_2:\n\t" \
  "global_load_dword v196, v195, s[26:27] offset:3072 sc1\n\t" \
  "s_branch L_apr11_2\n\t" \
  "L_ap12_2:\n\t" \
  "global_load_dword v196, v195, s[26:27] offset:3072 sc1\n\t" \
  "s_branch L_apr12_2\n\t" \
  "L_ap13_2:\n\t" \
  "global_load_dword v196, v195, s[26:27] offset:3072 sc1\n\t" \
  "s_branch L_apr13_2\n\t" \
  "L_ap14_2:\n\t" \
  "global_load_dword v196, v195, s[26:27] offset:3072 sc1\n\t" \
  "s_branch L_apr14_2\n\t" \
  "L_ap15_2:\n\t" \
  "global_load_dword v196, v195, s[26:27] offset:3072 sc1\n\t" \
  "s_branch L_apr15_2\n\t" \
  "L_ap16_2:\n\t" \
  "global_load_dword v196, v195, s[26:27] offset:3072 sc1\n\t" \
  "s_branch L_apr16_2\n\t" \
  "L_ap17_2:\n\t" \
  "global_load_dword v196, v195, s[26:27] offset:3072 sc1\n\t" \
  "s_branch L_apr17_2\n\t" \
  "L_au2_2:\n\t" \
  "s_waitcnt vmcnt(10)\n\t" \
  "v_cmp_le_u32 vcc, s79, v196\n\t" \
  "s_nop 4\n\t" \
  "s_andn2_b32 s85, vcc_lo, s96\n\t" \
  "s_or_b32 s96, s96, vcc_lo\n\t" \
  "s_and_b32 exec_lo, s85, 0xfffffff\n\t" \
  "s_mov_b32 exec_hi, 0\n\t" \
  "s_cbranch_execz L_u5_2\n\t" \
  "s_mov_b32 s84, 458752\n\t" \
  "buffer_load_dword v210, v198, s[36:39], s84 offen \n\t" \
  "L_u5_2:\n\t" \
  "s_mov_b64 exec, -1\n\t" \
  "s_branch L_aur2_2\n\t" \
  "L_au3_2:\n\t" \
  "s_waitcnt vmcnt(13)\n\t" \
  "v_cmp_le_u32 vcc, s79, v196\n\t" \
  "s_nop 4\n\t" \
  "s_andn2_b32 s85, vcc_lo, s96\n\t" \
  "s_or_b32 s96, s96, vcc_lo\n\t" \
  "s_and_b32 exec_lo, s85, 0xfffffff\n\t" \
  "s_mov_b32 exec_hi, 0\n\t" \
  "s_cbranch_execz L_u6_2\n\t" \
  "s_mov_b32 s84, 458752\n\t" \
  "buffer_load_dword v210, v198, s[36:39], s84 offen \n\t" \
  "L_u6_2:\n\t" \
  "s_mov_b64 exec, -1\n\t" \
  "s_branch L_aur3_2\n\t" \
  "L_au5_2:\n\t" \
  "s_waitcnt vmcnt(10)\n\t" \
  "v_cmp_le_u32 vcc, s79, v196\n\t" \
  "s_nop 4\n\t" \
  "s_andn2_b32 s85, vcc_lo, s8\n\t" \
  "s_or_b32 s8, s8, vcc_lo\n\t" \
  "s_and_b32 exec_lo, s85, 0xfffffff\n\t" \
  "s_mov_b32 exec_hi, 0\n\t" \
  "s_cbranch_execz L_u7_2\n\t" \
  "s_mov_b32 s84, 917504\n\t" \
  "buffer_load_dword v210, v198, s[36:39], s84 offen \n\t" \
  "L_u7_2:\n\t" \
  "s_mov_b64 exec, -1\n\t" \
  "s_branch L_aur5_2\n\t" \
  "L_au6_2:\n\t" \
  "s_waitcnt vmcnt(14)\n\t" \
  "v_cmp_le_u32 vcc, s79, v196\n\t" \
  "s_nop 4\n\t" \
  "s_andn2_b32 s85, vcc_lo, s8\n\t" \
  "s_or_b32 s8, s8, vcc_lo\n\t" \
  "s_and_b32 exec_lo, s85, 0xfffffff\n\t" \
  "s_mov_b32 exec_hi, 0\n\t" \
  "s_cbranch_execz L_u8_2\n\t" \
  "s_mov_b32 s84, 917504\n\t" \
  "buffer_load_dword v210, v198, s[36:39], s84 offen \n\t" \
  "L_u8_2:\n\t" \
  "s_mov_b64 exec, -1\n\t" \
  "s_branch L_aur6_2\n\t" \
  "L_au7_2:\n\t" \
  "s_waitcnt vmcnt(0)\n\t" \
  "v_cmp_le_u32 vcc, s79, v196\n\t" \
  "s_nop 4\n\t" \
  "s_andn2_b32 s85, vcc_lo, s8\n\t" \
  "s_or_b32 s8, s8, vcc_lo\n\t" \
  "s_and_b32 exec_lo, s85, 0xfffffff\n\t" \
  "s_mov_b32 exec_hi, 0\n\t" \
  "s_cbranch_execz L_u9_2\n\t" \
  "s_mov_b32 s84, 917504\n\t" \
  "buffer_load_dword v210, v198, s[36:39], s84 offen \n\t" \
  "L_u9_2:\n\t" \
  "s_mov_b64 exec, -1\n\t" \
  "s_branch L_aur7_2\n\t" \
  "L_au8_2:\n\t" \
  "s_waitcnt vmcnt(15)\n\t" \
  "v_cmp_le_u32 vcc, s79, v196\n\t" \
  "s_nop 4\n\t" \
  "s_andn2_b32 s85, vcc_lo, s8\n\t" \
  "s_or_b32 s8, s8, vcc_lo\n\t" \
  "s_and_b32 exec_lo, s85, 0xfffffff\n\t" \
  "s_mov_b32 exec_hi, 0\n\t" \
  "s_cbranch_execz L_u10_2\n\t" \
  "s_mov_b32 s84, 917504\n\t" \
  "buffer_load_dword v210, v198, s[36:39], s84 offen \n\t" \
  "L_u10_2:\n\t" \
  "s_mov_b64 exec, -1\n\t" \
  "s_branch L_aur8_2\n\t" \
  "L_au10_2:\n\t" \
  "s_waitcnt vmcnt(0)\n\t" \
  "v_cmp_le_u32 vcc, s79, v196\n\t" \
  "s_nop 4\n\t" \
  "s_andn2_b32 s85, vcc_lo, s9\n\t" \
  "s_or_b32 s9, s9, vcc_lo\n\t" \
  "s_and_b32 exec_lo, s85, 0xfffffff\n\t" \
  "s_mov_b32 exec_hi, 0\n\t" \
  "s_cbranch_execz L_u11_2\n\t" \
  "s_mov_b32 s84, 1376256\n\t" \
  "buffer_load_dword v210, v198, s[36:39], s84 offen \n\t" \
  "L_u11_2:\n\t" \
  "s_mov_b64 exec, -1\n\t" \
  "s_branch L_aur10_2\n\t" \
  "L_au11_2:\n\t" \
  "s_waitcnt vmcnt(15)\n\t" \
  "v_cmp_le_u32 vcc, s79, v196\n\t" \
  "s_nop 4\n\t" \
  "s_andn2_b32 s85, vcc_lo, s9\n\t" \
  "s_or_b32 s9, s9, vcc_lo\n\t" \
  "s_and_b32 exec_lo, s85, 0xfffffff\n\t" \
  "s_mov_b32 exec_hi, 0\n\t" \
  "s_cbranch_execz L_u12_2\n\t" \
  "s_mov_b32 s84, 1376256\n\t" \
  "buffer_load_dword v210, v198, s[36:39], s84 offen \n\t" \
  "L_u12_2:\n\t" \
  "s_mov_b64 exec, -1\n\t" \
  "s_branch L_aur11_2\n\t" \
  "L_au12_2:\n\t" \
  "s_waitcnt vmcnt(9)\n\t" \
  "v_cmp_le_u32 vcc, s79, v196\n\t" \
  "s_nop 4\n\t" \
  "s_andn2_b32 s85, vcc_lo, s9\n\t" \
  "s_or_b32 s9, s9, vcc_lo\n\t" \
  "s_and_b32 exec_lo, s85, 0xfffffff\n\t" \
  "s_mov_b32 exec_hi, 0\n\t" \
  "s_cbranch_execz L_u13_2\n\t" \
  "s_mov_b32 s84, 1376256\n\t" \
  "buffer_load_dword v210, v198, s[36:39], s84 offen \n\t" \
  "L_u13_2:\n\t" \
  "s_mov_b64 exec, -1\n\t" \
  "s_branch L_aur12_2\n\t" \
  "L_au13_2:\n\t" \
  "s_waitcnt vmcnt(9)\n\t" \
  "v_cmp_le_u32 vcc, s79, v196\n\t" \
  "s_nop 4\n\t" \
  "s_andn2_b32 s85, vcc_lo, s9\n\t" \
  "s_or_b32 s9, s9, vcc_lo\n\t" \
  "s_and_b32 exec_lo, s85, 0xfffffff\n\t" \
  "s_mov_b32 exec_hi, 0\n\t" \
  "s_cbranch_execz L_u14_2\n\t" \
  "s_mov_b32 s84, 1376256\n\t" \
  "buffer_load_dword v210, v198, s[36:39], s84 offen \n\t" \
  "L_u14_2:\n\t" \
  "s_mov_b64 exec, -1\n\t" \
  "s_branch L_aur13_2\n\t" \
  "L_au14_2:\n\t" \
  "s_waitcnt vmcnt(3)\n\t" \
  "v_cmp_le_u32 vcc, s79, v196\n\t" \
  "s_nop 4\n\t" \
  "s_andn2_b32 s85, vcc_lo, s9\n\t" \
  "s_or_b32 s9, s9, vcc_lo\n\t" \
  "s_and_b32 exec_lo, s85, 0xfffffff\n\t" \
  "s_mov_b32 exec_hi, 0\n\t" \
  "s_cbranch_execz L_u15_2\n\t" \
  "s_mov_b32 s84, 1376256\n\t" \
  "buffer_load_dword v210, v198, s[36:39], s84 offen \n\t" \
  "L_u15_2:\n\t" \
  "s_mov_b64 exec, -1\n\t" \
  "s_branch L_aur14_2\n\t" \
  "L_au15_2:\n\t" \
  "s_waitcnt vmcnt(14)\n\t" \
  "v_cmp_le_u32 vcc, s79, v196\n\t" \
  "s_nop 4\n\t" \
  "s_andn2_b32 s85, vcc_lo, s9\n\t" \
  "s_or_b32 s9, s9, vcc_lo\n\t" \
  "s_and_b32 exec_lo, s85, 0xfffffff\n\t" \
  "s_mov_b32 exec_hi, 0\n\t" \
  "s_cbranch_execz L_u16_2\n\t" \
  "s_mov_b32 s84, 1376256\n\t" \
  "buffer_load_dword v210, v198, s[36:39], s84 offen \n\t" \
  "L_u16_2:\n\t" \
  "s_mov_b64 exec, -1\n\t" \
  "s_branch L_aur15_2\n\t" \
  "L_au16_2:\n\t" \
  "s_waitcnt vmcnt(14)\n\t" \
  "v_cmp_le_u32 vcc, s79, v196\n\t" \
  "s_nop 4\n\t" \
  "s_andn2_b32 s85, vcc_lo, s9\n\t" \
  "s_or_b32 s9, s9, vcc_lo\n\t" \
  "s_and_b32 exec_lo, s85, 0xfffffff\n\t" \
  "s_mov_b32 exec_hi, 0\n\t" \
  "s_cbranch_execz L_u17_2\n\t" \
  "s_mov_b32 s84, 1376256\n\t" \
  "buffer_load_dword v210, v198, s[36:39], s84 offen \n\t" \
  "L_u17_2:\n\t" \
  "s_mov_b64 exec, -1\n\t" \
  "s_branch L_aur16_2\n\t" \
  "L_au17_2:\n\t" \
  "s_waitcnt vmcnt(9)\n\t" \
  "v_cmp_le_u32 vcc, s79, v196\n\t" \
  "s_nop 4\n\t" \
  "s_andn2_b32 s85, vcc_lo, s9\n\t" \
  "s_or_b32 s9, s9, vcc_lo\n\t" \
  "s_and_b32 exec_lo, s85, 0xfffffff\n\t" \
  "s_mov_b32 exec_hi, 0\n\t" \
  "s_cbranch_execz L_u18_2\n\t" \
  "s_mov_b32 s84, 1376256\n\t" \
  "buffer_load_dword v210, v198, s[36:39], s84 offen \n\t" \
  "L_u18_2:\n\t" \
  "s_mov_b64 exec, -1\n\t" \
  "s_branch L_aur17_2\n\t" \
  "L_v4_2:\n\t" \
  "s_mov_b32 s101, 0\n\t" \
  "s_mov_b32 s98, s87\n\t" \
  "L_vl4_2:\n\t" \
  "s_memrealtime s[88:89]\n\t" \
  "s_cmp_ge_u32 s82, 18\n\t" \
  "s_cbranch_scc1 L_vn4_2\n\t" \
  "s_waitcnt lgkmcnt(0)\n\t" \
  "s_sub_u32 s84, s88, s14\n\t" \
  "s_cmp_lt_u32 s84, 520\n\t" \
  "s_cbranch_scc1 L_vn4_2\n\t" \
  "v_readlane_b32 s85, v211, s82\n\t" \
  "s_add_u32 s82, s82, 4\n\t" \
  "s_nop 3\n\t" \
  "buffer_load_dword v210, v197, s[60:63], s85 offen\n\t" \
  "L_vn4_2:\n\t" \
  "s_mov_b32 s87, 0\n\t" \
  "buffer_load_dwordx4 a[128:131], v188, s[36:39], s87 offen offset:0 sc0 sc1\n\t" \
  "buffer_load_dwordx4 a[132:135], v188, s[36:39], s87 offen offset:1024 sc0 sc1\n\t" \
  "buffer_load_dwordx4 a[136:139], v188, s[36:39], s87 offen offset:2048 sc0 sc1\n\t" \
  "buffer_load_dwordx4 a[140:143], v188, s[36:39], s87 offen offset:3072 sc0 sc1\n\t" \
  "buffer_load_dwordx4 a[144:147], v189, s[36:39], s87 offen offset:0 sc0 sc1\n\t" \
  "buffer_load_dwordx4 a[148:151], v189, s[36:39], s87 offen offset:1024 sc0 sc1\n\t" \
  "buffer_load_dwordx4 a[152:155], v189, s[36:39], s87 offen offset:2048 sc0 sc1\n\t" \
  "buffer_load_dwordx4 a[156:159], v189, s[36:39], s87 offen offset:3072 sc0 sc1\n\t" \
  "global_load_dword v226, v195, s[26:27] offset:0 sc0 sc1\n\t" \
  "s_memrealtime s[88:89]\n\t" \
  "s_waitcnt vmcnt(0) lgkmcnt(0)\n\t" \
  "v_accvgpr_read_b32 v237, a128\n\t" \
  "v_accvgpr_read_b32 v238, a129\n\t" \
  "v_accvgpr_read_b32 v239, a130\n\t" \
  "v_accvgpr_read_b32 v240, a131\n\t" \
  "v_min3_u32 v236, v237, v238, v237\n\t" \
  "v_min3_u32 v236, v239, v240, v236\n\t" \
  "v_accvgpr_read_b32 v237, a132\n\t" \
  "v_accvgpr_read_b32 v238, a133\n\t" \
  "v_accvgpr_read_b32 v239, a134\n\t" \
  "v_accvgpr_read_b32 v240, a135\n\t" \
  "v_min3_u32 v236, v237, v238, v236\n\t" \
  "v_min3_u32 v236, v239, v240, v236\n\t" \
  "v_accvgpr_read_b32 v237, a136\n\t" \
  "v_accvgpr_read_b32 v238, a137\n\t" \
  "v_accvgpr_read_b32 v239, a138\n\t" \
  "v_accvgpr_read_b32 v240, a139\n\t" \
  "v_min3_u32 v236, v237, v238, v236\n\t" \
  "v_min3_u32 v236, v239, v240, v236\n\t" \
  "v_accvgpr_read_b32 v237, a140\n\t" \
  "v_accvgpr_read_b32 v238, a141\n\t" \
  "v_accvgpr_read_b32 v239, a142\n\t" \
  "v_accvgpr_read_b32 v240, a143\n\t" \
  "v_min3_u32 v236, v237, v238, v236\n\t" \
  "v_min3_u32 v236, v239, v240, v236\n\t" \
  "v_accvgpr_read_b32 v237, a144\n\t" \
  "v_accvgpr_read_b32 v238, a145\n\t" \
  "v_accvgpr_read_b32 v239, a146\n\t" \
  "v_accvgpr_read_b32 v240, a147\n\t" \
  "v_min3_u32 v236, v237, v238, v236\n\t" \
  "v_min3_u32 v236, v239, v240, v236\n\t" \
  "v_accvgpr_read_b32 v237, a148\n\t" \
  "v_accvgpr_read_b32 v238, a149\n\t" \
  "v_accvgpr_read_b32 v239, a150\n\t" \
  "v_accvgpr_read_b32 v240, a151\n\t" \
  "v_min3_u32 v236, v237, v238, v236\n\t" \
  "v_min3_u32 v236, v239, v240, v236\n\t" \
  "v_accvgpr_read_b32 v237, a152\n\t" \
  "v_accvgpr_read_b32 v238, a153\n\t" \
  "v_accvgpr_read_b32 v239, a154\n\t" \
  "v_accvgpr_read_b32 v240, a155\n\t" \
  "v_min3_u32 v236, v237, v238, v236\n\t" \
  "v_min3_u32 v236, v239, v240, v236\n\t" \
  "v_accvgpr_read_b32 v237, a156\n\t" \
  "v_accvgpr_read_b32 v238, a157\n\t" \
  "v_accvgpr_read_b32 v239, a158\n\t" \
  "v_accvgpr_read_b32 v240, a159\n\t" \
  "v_min3_u32 v236, v237, v238, v236\n\t" \
  "v_min3_u32 v236, v239, v240, v236\n\t" \
  "v_cmp_eq_u32 vcc, 0, v236\n\t" \
  "s_nop 4\n\t" \
  "s_cmp_eq_u64 vcc, 0\n\t" \
  "s_cbranch_scc1 L_vd4_2\n\t" \
  "s_cmp_eq_u32 s101, 15\n\t" \
  "s_cbranch_scc1 L_vd4_2\n\t" \
  "v_cmp_le_u32 vcc, s79, v226\n\t" \
  "s_nop 4\n\t" \
  "s_bfe_u32 s101, vcc_lo, 262144\n\t" \
  "s_sub_u32 s84, s88, s14\n\t" \
  "s_or_b32 s84, s84, s97\n\t" \
  "s_cmp_lt_u32 s84, 50000000\n\t" \
  "s_cbranch_scc1 L_vl4_2\n\t" \
  "s_mov_b32 s97, -1\n\t" \
  "s_mov_b32 s81, 0x7fc00000\n\t" \
  "L_vd4_2:\n\t" \
  "s_mov_b64 exec, 0xfffffff\n\t" \
  "s_mov_b32 s84, 0\n\t" \
  "buffer_load_dword v210, v198, s[36:39], s84 offen sc1\n\t" \
  "s_mov_b64 exec, -1\n\t" \
  "s_mov_b32 s87, s98\n\t" \
  "s_branch L_vr4_2\n\t" \
  "L_v5_2:\n\t" \
  "s_mov_b32 s101, 0\n\t" \
  "s_mov_b32 s98, s87\n\t" \
  "L_vl5_2:\n\t" \
  "s_memrealtime s[88:89]\n\t" \
  "s_cmp_ge_u32 s82, 18\n\t" \
  "s_cbranch_scc1 L_vn5_2\n\t" \
  "s_waitcnt lgkmcnt(0)\n\t" \
  "s_sub_u32 s84, s88, s14\n\t" \
  "s_cmp_lt_u32 s84, 520\n\t" \
  "s_cbranch_scc1 L_vn5_2\n\t" \
  "v_readlane_b32 s85, v211, s82\n\t" \
  "s_add_u32 s82, s82, 4\n\t" \
  "s_nop 3\n\t" \
  "buffer_load_dword v210, v197, s[60:63], s85 offen\n\t" \
  "L_vn5_2:\n\t" \
  "s_mov_b32 s87, 65536\n\t" \
  "buffer_load_dwordx4 a[160:163], v188, s[36:39], s87 offen offset:0 sc0 sc1\n\t" \
  "buffer_load_dwordx4 a[164:167], v188, s[36:39], s87 offen offset:1024 sc0 sc1\n\t" \
  "buffer_load_dwordx4 a[168:171], v188, s[36:39], s87 offen offset:2048 sc0 sc1\n\t" \
  "buffer_load_dwordx4 a[172:175], v188, s[36:39], s87 offen offset:3072 sc0 sc1\n\t" \
  "buffer_load_dwordx4 a[176:179], v189, s[36:39], s87 offen offset:0 sc0 sc1\n\t" \
  "buffer_load_dwordx4 a[180:183], v189, s[36:39], s87 offen offset:1024 sc0 sc1\n\t" \
  "buffer_load_dwordx4 a[184:187], v189, s[36:39], s87 offen offset:2048 sc0 sc1\n\t" \
  "buffer_load_dwordx4 a[188:191], v189, s[36:39], s87 offen offset:3072 sc0 sc1\n\t" \
  "global_load_dword v226, v195, s[26:27] offset:0 sc0 sc1\n\t" \
  "s_memrealtime s[88:89]\n\t" \
  "s_waitcnt vmcnt(0) lgkmcnt(0)\n\t" \
  "v_accvgpr_read_b32 v237, a160\n\t" \
  "v_accvgpr_read_b32 v238, a161\n\t" \
  "v_accvgpr_read_b32 v239, a162\n\t" \
  "v_accvgpr_read_b32 v240, a163\n\t" \
  "v_min3_u32 v236, v237, v238, v237\n\t" \
  "v_min3_u32 v236, v239, v240, v236\n\t" \
  "v_accvgpr_read_b32 v237, a164\n\t" \
  "v_accvgpr_read_b32 v238, a165\n\t" \
  "v_accvgpr_read_b32 v239, a166\n\t" \
  "v_accvgpr_read_b32 v240, a167\n\t" \
  "v_min3_u32 v236, v237, v238, v236\n\t" \
  "v_min3_u32 v236, v239, v240, v236\n\t" \
  "v_accvgpr_read_b32 v237, a168\n\t" \
  "v_accvgpr_read_b32 v238, a169\n\t" \
  "v_accvgpr_read_b32 v239, a170\n\t" \
  "v_accvgpr_read_b32 v240, a171\n\t" \
  "v_min3_u32 v236, v237, v238, v236\n\t" \
  "v_min3_u32 v236, v239, v240, v236\n\t" \
  "v_accvgpr_read_b32 v237, a172\n\t" \
  "v_accvgpr_read_b32 v238, a173\n\t" \
  "v_accvgpr_read_b32 v239, a174\n\t" \
  "v_accvgpr_read_b32 v240, a175\n\t" \
  "v_min3_u32 v236, v237, v238, v236\n\t" \
  "v_min3_u32 v236, v239, v240, v236\n\t" \
  "v_accvgpr_read_b32 v237, a176\n\t" \
  "v_accvgpr_read_b32 v238, a177\n\t" \
  "v_accvgpr_read_b32 v239, a178\n\t" \
  "v_accvgpr_read_b32 v240, a179\n\t" \
  "v_min3_u32 v236, v237, v238, v236\n\t" \
  "v_min3_u32 v236, v239, v240, v236\n\t" \
  "v_accvgpr_read_b32 v237, a180\n\t" \
  "v_accvgpr_read_b32 v238, a181\n\t" \
  "v_accvgpr_read_b32 v239, a182\n\t" \
  "v_accvgpr_read_b32 v240, a183\n\t" \
  "v_min3_u32 v236, v237, v238, v236\n\t" \
  "v_min3_u32 v236, v239, v240, v236\n\t" \
  "v_accvgpr_read_b32 v237, a184\n\t" \
  "v_accvgpr_read_b32 v238, a185\n\t" \
  "v_accvgpr_read_b32 v239, a186\n\t" \
  "v_accvgpr_read_b32 v240, a187\n\t" \
  "v_min3_u32 v236, v237, v238, v236\n\t" \
  "v_min3_u32 v236, v239, v240, v236\n\t" \
  "v_accvgpr_read_b32 v237, a188\n\t" \
  "v_accvgpr_read_b32 v238, a189\n\t" \
  "v_accvgpr_read_b32 v239, a190\n\t" \
  "v_accvgpr_read_b32 v240, a191\n\t" \
  "v_min3_u32 v236, v237, v238, v236\n\t" \
  "v_min3_u32 v236, v239, v240, v236\n\t" \
  "v_cmp_eq_u32 vcc, 0, v236\n\t" \
  "s_nop 4\n\t" \
  "s_cmp_eq_u64 vcc, 0\n\t" \
  "s_cbranch_scc1 L_vd5_2\n\t" \
  "s_cmp_eq_u32 s101, 15\n\t" \
  "s_cbranch_scc1 L_vd5_2\n\t" \
  "v_cmp_le_u32 vcc, s79, v226\n\t" \
  "s_nop 4\n\t" \
  "s_bfe_u32 s101, vcc_lo, 262148\n\t" \
  "s_sub_u32 s84, s88, s14\n\t" \
  "s_or_b32 s84, s84, s97\n\t" \
  "s_cmp_lt_u32 s84, 50000000\n\t" \
  "s_cbranch_scc1 L_vl5_2\n\t" \
  "s_mov_b32 s97, -1\n\t" \
  "s_mov_b32 s81, 0x7fc00000\n\t" \
  "L_vd5_2:\n\t" \
  "s_mov_b64 exec, 0xfffffff\n\t" \
  "s_mov_b32 s84, 0\n\t" \
  "buffer_load_dword v210, v198, s[36:39], s84 offen sc1\n\t" \
  "s_mov_b64 exec, -1\n\t" \
  "s_mov_b32 s87, s98\n\t" \
  "s_branch L_vr5_2\n\t" \
  "L_v6_2:\n\t" \
  "s_mov_b32 s101, 0\n\t" \
  "s_mov_b32 s98, s87\n\t" \
  "L_vl6_2:\n\t" \
  "s_memrealtime s[88:89]\n\t" \
  "s_cmp_ge_u32 s82, 18\n\t" \
  "s_cbranch_scc1 L_vn6_2\n\t" \
  "s_waitcnt lgkmcnt(0)\n\t" \
  "s_sub_u32 s84, s88, s14\n\t" \
  "s_cmp_lt_u32 s84, 520\n\t" \
  "s_cbranch_scc1 L_vn6_2\n\t" \
  "v_readlane_b32 s85, v211, s82\n\t" \
  "s_add_u32 s82, s82, 4\n\t" \
  "s_nop 3\n\t" \
  "buffer_load_dword v210, v197, s[60:63], s85 offen\n\t" \
  "L_vn6_2:\n\t" \
  "s_mov_b32 s87, 131072\n\t" \
  "buffer_load_dwordx4 a[192:195], v188, s[36:39], s87 offen offset:0 sc0 sc1\n\t" \
  "buffer_load_dwordx4 a[196:199], v188, s[36:39], s87 offen offset:1024 sc0 sc1\n\t" \
  "buffer_load_dwordx4 a[200:203], v188, s[36:39], s87 offen offset:2048 sc0 sc1\n\t" \
  "buffer_load_dwordx4 a[204:207], v188, s[36:39], s87 offen offset:3072 sc0 sc1\n\t" \
  "buffer_load_dwordx4 a[208:211], v189, s[36:39], s87 offen offset:0 sc0 sc1\n\t" \
  "buffer_load_dwordx4 a[212:215], v189, s[36:39], s87 offen offset:1024 sc0 sc1\n\t" \
  "buffer_load_dwordx4 a[216:219], v189, s[36:39], s87 offen offset:2048 sc0 sc1\n\t" \
  "buffer_load_dwordx4 a[220:223], v189, s[36:39], s87 offen offset:3072 sc0 sc1\n\t" \
  "global_load_dword v226, v195, s[26:27] offset:0 sc0 sc1\n\t" \
  "s_memrealtime s[88:89]\n\t" \
  "s_waitcnt vmcnt(0) lgkmcnt(0)\n\t" \
  "v_accvgpr_read_b32 v237, a192\n\t" \
  "v_accvgpr_read_b32 v238, a193\n\t" \
  "v_accvgpr_read_b32 v239, a194\n\t" \
  "v_accvgpr_read_b32 v240, a195\n\t" \
  "v_min3_u32 v236, v237, v238, v237\n\t" \
  "v_min3_u32 v236, v239, v240, v236\n\t" \
  "v_accvgpr_read_b32 v237, a196\n\t" \
  "v_accvgpr_read_b32 v238, a197\n\t" \
  "v_accvgpr_read_b32 v239, a198\n\t" \
  "v_accvgpr_read_b32 v240, a199\n\t" \
  "v_min3_u32 v236, v237, v238, v236\n\t" \
  "v_min3_u32 v236, v239, v240, v236\n\t" \
  "v_accvgpr_read_b32 v237, a200\n\t" \
  "v_accvgpr_read_b32 v238, a201\n\t" \
  "v_accvgpr_read_b32 v239, a202\n\t" \
  "v_accvgpr_read_b32 v240, a203\n\t" \
  "v_min3_u32 v236, v237, v238, v236\n\t" \
  "v_min3_u32 v236, v239, v240, v236\n\t" \
  "v_accvgpr_read_b32 v237, a204\n\t" \
  "v_accvgpr_read_b32 v238, a205\n\t" \
  "v_accvgpr_read_b32 v239, a206\n\t" \
  "v_accvgpr_read_b32 v240, a207\n\t" \
  "v_min3_u32 v236, v237, v238, v236\n\t" \
  "v_min3_u32 v236, v239, v240, v236\n\t" \
  "v_accvgpr_read_b32 v237, a208\n\t" \
  "v_accvgpr_read_b32 v238, a209\n\t" \
  "v_accvgpr_read_b32 v239, a210\n\t" \
  "v_accvgpr_read_b32 v240, a211\n\t" \
  "v_min3_u32 v236, v237, v238, v236\n\t" \
  "v_min3_u32 v236, v239, v240, v236\n\t" \
  "v_accvgpr_read_b32 v237, a212\n\t" \
  "v_accvgpr_read_b32 v238, a213\n\t" \
  "v_accvgpr_read_b32 v239, a214\n\t" \
  "v_accvgpr_read_b32 v240, a215\n\t" \
  "v_min3_u32 v236, v237, v238, v236\n\t" \
  "v_min3_u32 v236, v239, v240, v236\n\t" \
  "v_accvgpr_read_b32 v237, a216\n\t" \
  "v_accvgpr_read_b32 v238, a217\n\t" \
  "v_accvgpr_read_b32 v239, a218\n\t" \
  "v_accvgpr_read_b32 v240, a219\n\t" \
  "v_min3_u32 v236, v237, v238, v236\n\t" \
  "v_min3_u32 v236, v239, v240, v236\n\t" \
  "v_accvgpr_read_b32 v237, a220\n\t" \
  "v_accvgpr_read_b32 v238, a221\n\t" \
  "v_accvgpr_read_b32 v239, a222\n\t" \
  "v_accvgpr_read_b32 v240, a223\n\t" \
  "v_min3_u32 v236, v237, v238, v236\n\t" \
  "v_min3_u32 v236, v239, v240, v236\n\t" \
  "v_cmp_eq_u32 vcc, 0, v236\n\t" \
  "s_nop 4\n\t" \
  "s_cmp_eq_u64 vcc, 0\n\t" \
  "s_cbranch_scc1 L_vd6_2\n\t" \
  "s_cmp_eq_u32 s101, 15\n\t" \
  "s_cbranch_scc1 L_vd6_2\n\t" \
  "v_cmp_le_u32 vcc, s79, v226\n\t" \
  "s_nop 4\n\t" \
  "s_bfe_u32 s101, vcc_lo, 262152\n\t" \
  "s_sub_u32 s84, s88, s14\n\t" \
  "s_or_b32 s84, s84, s97\n\t" \
  "s_cmp_lt_u32 s84, 50000000\n\t" \
  "s_cbranch_scc1 L_vl6_2\n\t" \
  "s_mov_b32 s97, -1\n\t" \
  "s_mov_b32 s81, 0x7fc00000\n\t" \
  "L_vd6_2:\n\t" \
  "s_mov_b64 exec, 0xfffffff\n\t" \
  "s_mov_b32 s84, 0\n\t" \
  "buffer_load_dword v210, v198, s[36:39], s84 offen sc1\n\t" \
  "s_mov_b64 exec, -1\n\t" \
  "s_mov_b32 s87, s98\n\t" \
  "s_branch L_vr6_2\n\t" \
  "L_v7_2:\n\t" \
  "s_mov_b32 s101, 0\n\t" \
  "s_mov_b32 s98, s87\n\t" \
  "L_vl7_2:\n\t" \
  "s_memrealtime s[88:89]\n\t" \
  "s_cmp_ge_u32 s82, 18\n\t" \
  "s_cbranch_scc1 L_vn7_2\n\t" \
  "s_waitcnt lgkmcnt(0)\n\t" \
  "s_sub_u32 s84, s88, s14\n\t" \
  "s_cmp_lt_u32 s84, 520\n\t" \
  "s_cbranch_scc1 L_vn7_2\n\t" \
  "v_readlane_b32 s85, v211, s82\n\t" \
  "s_add_u32 s82, s82, 4\n\t" \
  "s_nop 3\n\t" \
  "buffer_load_dword v210, v197, s[60:63], s85 offen\n\t" \
  "L_vn7_2:\n\t" \
  "s_mov_b32 s87, 196608\n\t" \
  "buffer_load_dwordx4 a[224:227], v188, s[36:39], s87 offen offset:0 sc0 sc1\n\t" \
  "buffer_load_dwordx4 a[228:231], v188, s[36:39], s87 offen offset:1024 sc0 sc1\n\t" \
  "buffer_load_dwordx4 a[232:235], v188, s[36:39], s87 offen offset:2048 sc0 sc1\n\t" \
  "buffer_load_dwordx4 a[236:239], v188, s[36:39], s87 offen offset:3072 sc0 sc1\n\t" \
  "buffer_load_dwordx4 a[240:243], v189, s[36:39], s87 offen offset:0 sc0 sc1\n\t" \
  "buffer_load_dwordx4 a[244:247], v189, s[36:39], s87 offen offset:1024 sc0 sc1\n\t" \
  "buffer_load_dwordx4 a[248:251], v189, s[36:39], s87 offen offset:2048 sc0 sc1\n\t" \
  "buffer_load_dwordx4 a[252:255], v189, s[36:39], s87 offen offset:3072 sc0 sc1\n\t" \
  "global_load_dword v226, v195, s[26:27] offset:0 sc0 sc1\n\t" \
  "s_memrealtime s[88:89]\n\t" \
  "s_waitcnt vmcnt(0) lgkmcnt(0)\n\t" \
  "v_accvgpr_read_b32 v237, a224\n\t" \
  "v_accvgpr_read_b32 v238, a225\n\t" \
  "v_accvgpr_read_b32 v239, a226\n\t" \
  "v_accvgpr_read_b32 v240, a227\n\t" \
  "v_min3_u32 v236, v237, v238, v237\n\t" \
  "v_min3_u32 v236, v239, v240, v236\n\t" \
  "v_accvgpr_read_b32 v237, a228\n\t" \
  "v_accvgpr_read_b32 v238, a229\n\t" \
  "v_accvgpr_read_b32 v239, a230\n\t" \
  "v_accvgpr_read_b32 v240, a231\n\t" \
  "v_min3_u32 v236, v237, v238, v236\n\t" \
  "v_min3_u32 v236, v239, v240, v236\n\t" \
  "v_accvgpr_read_b32 v237, a232\n\t" \
  "v_accvgpr_read_b32 v238, a233\n\t" \
  "v_accvgpr_read_b32 v239, a234\n\t" \
  "v_accvgpr_read_b32 v240, a235\n\t" \
  "v_min3_u32 v236, v237, v238, v236\n\t" \
  "v_min3_u32 v236, v239, v240, v236\n\t" \
  "v_accvgpr_read_b32 v237, a236\n\t" \
  "v_accvgpr_read_b32 v238, a237\n\t" \
  "v_accvgpr_read_b32 v239, a238\n\t" \
  "v_accvgpr_read_b32 v240, a239\n\t" \
  "v_min3_u32 v236, v237, v238, v236\n\t" \
  "v_min3_u32 v236, v239, v240, v236\n\t" \
  "v_accvgpr_read_b32 v237, a240\n\t" \
  "v_accvgpr_read_b32 v238, a241\n\t" \
  "v_accvgpr_read_b32 v239, a242\n\t" \
  "v_accvgpr_read_b32 v240, a243\n\t" \
  "v_min3_u32 v236, v237, v238, v236\n\t" \
  "v_min3_u32 v236, v239, v240, v236\n\t" \
  "v_accvgpr_read_b32 v237, a244\n\t" \
  "v_accvgpr_read_b32 v238, a245\n\t" \
  "v_accvgpr_read_b32 v239, a246\n\t" \
  "v_accvgpr_read_b32 v240, a247\n\t" \
  "v_min3_u32 v236, v237, v238, v236\n\t" \
  "v_min3_u32 v236, v239, v240, v236\n\t" \
  "v_accvgpr_read_b32 v237, a248\n\t" \
  "v_accvgpr_read_b32 v238, a249\n\t" \
  "v_accvgpr_read_b32 v239, a250\n\t" \
  "v_accvgpr_read_b32 v240, a251\n\t" \
  "v_min3_u32 v236, v237, v238, v236\n\t" \
  "v_min3_u32 v236, v239, v240, v236\n\t" \
  "v_accvgpr_read_b32 v237, a252\n\t" \
  "v_accvgpr_read_b32 v238, a253\n\t" \
  "v_accvgpr_read_b32 v239, a254\n\t" \
  "v_accvgpr_read_b32 v240, a255\n\t" \
  "v_min3_u32 v236, v237, v238, v236\n\t" \
  "v_min3_u32 v236, v239, v240, v236\n\t" \
  "v_cmp_eq_u32 vcc, 0, v236\n\t" \
  "s_nop 4\n\t" \
  "s_cmp_eq_u64 vcc, 0\n\t" \
  "s_cbranch_scc1 L_vd7_2\n\t" \
  "s_cmp_eq_u32 s101, 15\n\t" \
  "s_cbranch_scc1 L_vd7_2\n\t" \
  "v_cmp_le_u32 vcc, s79, v226\n\t" \
  "s_nop 4\n\t" \
  "s_bfe_u32 s101, vcc_lo, 262156\n\t" \
  "s_sub_u32 s84, s88, s14\n\t" \
  "s_or_b32 s84, s84, s97\n\t" \
  "s_cmp_lt_u32 s84, 50000000\n\t" \
  "s_cbranch_scc1 L_vl7_2\n\t" \
  "s_mov_b32 s97, -1\n\t" \
  "s_mov_b32 s81, 0x7fc00000\n\t" \
  "L_vd7_2:\n\t" \
  "s_mov_b64 exec, 0xfffffff\n\t" \
  "s_mov_b32 s84, 0\n\t" \
  "buffer_load_dword v210, v198, s[36:39], s84 offen sc1\n\t" \
  "s_mov_b64 exec, -1\n\t" \
  "s_mov_b32 s87, s98\n\t" \
  "s_branch L_vr7_2\n\t" \
  "L_v8_2:\n\t" \
  "s_mov_b32 s101, 0\n\t" \
  "s_mov_b32 s98, s87\n\t" \
  "L_vl8_2:\n\t" \
  "s_memrealtime s[88:89]\n\t" \
  "s_cmp_ge_u32 s82, 18\n\t" \
  "s_cbranch_scc1 L_vn8_2\n\t" \
  "s_waitcnt lgkmcnt(0)\n\t" \
  "s_sub_u32 s84, s88, s14\n\t" \
  "s_cmp_lt_u32 s84, 520\n\t" \
  "s_cbranch_scc1 L_vn8_2\n\t" \
  "v_readlane_b32 s85, v211, s82\n\t" \
  "s_add_u32 s82, s82, 4\n\t" \
  "s_nop 3\n\t" \
  "buffer_load_dword v210, v197, s[60:63], s85 offen\n\t" \
  "L_vn8_2:\n\t" \
  "s_mov_b32 s87, 262144\n\t" \
  "buffer_load_dwordx4 a[0:3], v188, s[36:39], s87 offen offset:0 sc0 sc1\n\t" \
  "buffer_load_dwordx4 a[4:7], v188, s[36:39], s87 offen offset:1024 sc0 sc1\n\t" \
  "buffer_load_dwordx4 a[8:11], v188, s[36:39], s87 offen offset:2048 sc0 sc1\n\t" \
  "buffer_load_dwordx4 a[12:15], v188, s[36:39], s87 offen offset:3072 sc0 sc1\n\t" \
  "buffer_load_dwordx4 a[16:19], v189, s[36:39], s87 offen offset:0 sc0 sc1\n\t" \
  "buffer_load_dwordx4 a[20:23], v189, s[36:39], s87 offen offset:1024 sc0 sc1\n\t" \
  "buffer_load_dwordx4 a[24:27], v189, s[36:39], s87 offen offset:2048 sc0 sc1\n\t" \
  "buffer_load_dwordx4 a[28:31], v189, s[36:39], s87 offen offset:3072 sc0 sc1\n\t" \
  "global_load_dword v226, v195, s[26:27] offset:0 sc0 sc1\n\t" \
  "s_memrealtime s[88:89]\n\t" \
  "s_waitcnt vmcnt(0) lgkmcnt(0)\n\t" \
  "v_accvgpr_read_b32 v237, a0\n\t" \
  "v_accvgpr_read_b32 v238, a1\n\t" \
  "v_accvgpr_read_b32 v239, a2\n\t" \
  "v_accvgpr_read_b32 v240, a3\n\t" \
  "v_min3_u32 v236, v237, v238, v237\n\t" \
  "v_min3_u32 v236, v239, v240, v236\n\t" \
  "v_accvgpr_read_b32 v237, a4\n\t" \
  "v_accvgpr_read_b32 v238, a5\n\t" \
  "v_accvgpr_read_b32 v239, a6\n\t" \
  "v_accvgpr_read_b32 v240, a7\n\t" \
  "v_min3_u32 v236, v237, v238, v236\n\t" \
  "v_min3_u32 v236, v239, v240, v236\n\t" \
  "v_accvgpr_read_b32 v237, a8\n\t" \
  "v_accvgpr_read_b32 v238, a9\n\t" \
  "v_accvgpr_read_b32 v239, a10\n\t" \
  "v_accvgpr_read_b32 v240, a11\n\t" \
  "v_min3_u32 v236, v237, v238, v236\n\t" \
  "v_min3_u32 v236, v239, v240, v236\n\t" \
  "v_accvgpr_read_b32 v237, a12\n\t" \
  "v_accvgpr_read_b32 v238, a13\n\t" \
  "v_accvgpr_read_b32 v239, a14\n\t" \
  "v_accvgpr_read_b32 v240, a15\n\t" \
  "v_min3_u32 v236, v237, v238, v236\n\t" \
  "v_min3_u32 v236, v239, v240, v236\n\t" \
  "v_accvgpr_read_b32 v237, a16\n\t" \
  "v_accvgpr_read_b32 v238, a17\n\t" \
  "v_accvgpr_read_b32 v239, a18\n\t" \
  "v_accvgpr_read_b32 v240, a19\n\t" \
  "v_min3_u32 v236, v237, v238, v236\n\t" \
  "v_min3_u32 v236, v239, v240, v236\n\t" \
  "v_accvgpr_read_b32 v237, a20\n\t" \
  "v_accvgpr_read_b32 v238, a21\n\t" \
  "v_accvgpr_read_b32 v239, a22\n\t" \
  "v_accvgpr_read_b32 v240, a23\n\t" \
  "v_min3_u32 v236, v237, v238, v236\n\t" \
  "v_min3_u32 v236, v239, v240, v236\n\t" \
  "v_accvgpr_read_b32 v237, a24\n\t" \
  "v_accvgpr_read_b32 v238, a25\n\t" \
  "v_accvgpr_read_b32 v239, a26\n\t" \
  "v_accvgpr_read_b32 v240, a27\n\t" \
  "v_min3_u32 v236, v237, v238, v236\n\t" \
  "v_min3_u32 v236, v239, v240, v236\n\t" \
  "v_accvgpr_read_b32 v237, a28\n\t" \
  "v_accvgpr_read_b32 v238, a29\n\t" \
  "v_accvgpr_read_b32 v239, a30\n\t" \
  "v_accvgpr_read_b32 v240, a31\n\t" \
  "v_min3_u32 v236, v237, v238, v236\n\t" \
  "v_min3_u32 v236, v239, v240, v236\n\t" \
  "v_cmp_eq_u32 vcc, 0, v236\n\t" \
  "s_nop 4\n\t" \
  "s_cmp_eq_u64 vcc, 0\n\t" \
  "s_cbranch_scc1 L_vd8_2\n\t" \
  "s_cmp_eq_u32 s101, 15\n\t" \
  "s_cbranch_scc1 L_vd8_2\n\t" \
  "v_cmp_le_u32 vcc, s79, v226\n\t" \
  "s_nop 4\n\t" \
  "s_bfe_u32 s101, vcc_lo, 262160\n\t" \
  "s_sub_u32 s84, s88, s14\n\t" \
  "s_or_b32 s84, s84, s97\n\t" \
  "s_cmp_lt_u32 s84, 50000000\n\t" \
  "s_cbranch_scc1 L_vl8_2\n\t" \
  "s_mov_b32 s97, -1\n\t" \
  "s_mov_b32 s81, 0x7fc00000\n\t" \
  "L_vd8_2:\n\t" \
  "s_mov_b64 exec, 0xfffffff\n\t" \
  "s_mov_b32 s84, 0\n\t" \
  "buffer_load_dword v210, v198, s[36:39], s84 offen sc1\n\t" \
  "s_mov_b64 exec, -1\n\t" \
  "s_mov_b32 s87, s98\n\t" \
  "s_branch L_vr8_2\n\t" \
  "L_v9_2:\n\t" \
  "s_mov_b32 s101, 0\n\t" \
  "s_mov_b32 s98, s87\n\t" \
  "L_vl9_2:\n\t" \
  "s_memrealtime s[88:89]\n\t" \
  "s_cmp_ge_u32 s82, 18\n\t" \
  "s_cbranch_scc1 L_vn9_2\n\t" \
  "s_waitcnt lgkmcnt(0)\n\t" \
  "s_sub_u32 s84, s88, s14\n\t" \
  "s_cmp_lt_u32 s84, 520\n\t" \
  "s_cbranch_scc1 L_vn9_2\n\t" \
  "v_readlane_b32 s85, v211, s82\n\t" \
  "s_add_u32 s82, s82, 4\n\t" \
  "s_nop 3\n\t" \
  "buffer_load_dword v210, v197, s[60:63], s85 offen\n\t" \
  "L_vn9_2:\n\t" \
  "s_mov_b32 s87, 327680\n\t" \
  "buffer_load_dwordx4 a[32:35], v188, s[36:39], s87 offen offset:0 sc0 sc1\n\t" \
  "buffer_load_dwordx4 a[36:39], v188, s[36:39], s87 offen offset:1024 sc0 sc1\n\t" \
  "buffer_load_dwordx4 a[40:43], v188, s[36:39], s87 offen offset:2048 sc0 sc1\n\t" \
  "buffer_load_dwordx4 a[44:47], v188, s[36:39], s87 offen offset:3072 sc0 sc1\n\t" \
  "buffer_load_dwordx4 a[48:51], v189, s[36:39], s87 offen offset:0 sc0 sc1\n\t" \
  "buffer_load_dwordx4 a[52:55], v189, s[36:39], s87 offen offset:1024 sc0 sc1\n\t" \
  "buffer_load_dwordx4 a[56:59], v189, s[36:39], s87 offen offset:2048 sc0 sc1\n\t" \
  "buffer_load_dwordx4 a[60:63], v189, s[36:39], s87 offen offset:3072 sc0 sc1\n\t" \
  "global_load_dword v226, v195, s[26:27] offset:0 sc0 sc1\n\t" \
  "s_memrealtime s[88:89]\n\t" \
  "s_waitcnt vmcnt(0) lgkmcnt(0)\n\t" \
  "v_accvgpr_read_b32 v237, a32\n\t" \
  "v_accvgpr_read_b32 v238, a33\n\t" \
  "v_accvgpr_read_b32 v239, a34\n\t" \
  "v_accvgpr_read_b32 v240, a35\n\t" \
  "v_min3_u32 v236, v237, v238, v237\n\t" \
  "v_min3_u32 v236, v239, v240, v236\n\t" \
  "v_accvgpr_read_b32 v237, a36\n\t" \
  "v_accvgpr_read_b32 v238, a37\n\t" \
  "v_accvgpr_read_b32 v239, a38\n\t" \
  "v_accvgpr_read_b32 v240, a39\n\t" \
  "v_min3_u32 v236, v237, v238, v236\n\t" \
  "v_min3_u32 v236, v239, v240, v236\n\t" \
  "v_accvgpr_read_b32 v237, a40\n\t" \
  "v_accvgpr_read_b32 v238, a41\n\t" \
  "v_accvgpr_read_b32 v239, a42\n\t" \
  "v_accvgpr_read_b32 v240, a43\n\t" \
  "v_min3_u32 v236, v237, v238, v236\n\t" \
  "v_min3_u32 v236, v239, v240, v236\n\t" \
  "v_accvgpr_read_b32 v237, a44\n\t" \
  "v_accvgpr_read_b32 v238, a45\n\t" \
  "v_accvgpr_read_b32 v239, a46\n\t" \
  "v_accvgpr_read_b32 v240, a47\n\t" \
  "v_min3_u32 v236, v237, v238, v236\n\t" \
  "v_min3_u32 v236, v239, v240, v236\n\t" \
  "v_accvgpr_read_b32 v237, a48\n\t" \
  "v_accvgpr_read_b32 v238, a49\n\t" \
  "v_accvgpr_read_b32 v239, a50\n\t" \
  "v_accvgpr_read_b32 v240, a51\n\t" \
  "v_min3_u32 v236, v237, v238, v236\n\t" \
  "v_min3_u32 v236, v239, v240, v236\n\t" \
  "v_accvgpr_read_b32 v237, a52\n\t" \
  "v_accvgpr_read_b32 v238, a53\n\t" \
  "v_accvgpr_read_b32 v239, a54\n\t" \
  "v_accvgpr_read_b32 v240, a55\n\t" \
  "v_min3_u32 v236, v237, v238, v236\n\t" \
  "v_min3_u32 v236, v239, v240, v236\n\t" \
  "v_accvgpr_read_b32 v237, a56\n\t" \
  "v_accvgpr_read_b32 v238, a57\n\t" \
  "v_accvgpr_read_b32 v239, a58\n\t" \
  "v_accvgpr_read_b32 v240, a59\n\t" \
  "v_min3_u32 v236, v237, v238, v236\n\t" \
  "v_min3_u32 v236, v239, v240, v236\n\t" \
  "v_accvgpr_read_b32 v237, a60\n\t" \
  "v_accvgpr_read_b32 v238, a61\n\t" \
  "v_accvgpr_read_b32 v239, a62\n\t" \
  "v_accvgpr_read_b32 v240, a63\n\t" \
  "v_min3_u32 v236, v237, v238, v236\n\t" \
  "v_min3_u32 v236, v239, v240, v236\n\t" \
  "v_cmp_eq_u32 vcc, 0, v236\n\t" \
  "s_nop 4\n\t" \
  "s_cmp_eq_u64 vcc, 0\n\t" \
  "s_cbranch_scc1 L_vd9_2\n\t" \
  "s_cmp_eq_u32 s101, 15\n\t" \
  "s_cbranch_scc1 L_vd9_2\n\t" \
  "v_cmp_le_u32 vcc, s79, v226\n\t" \
  "s_nop 4\n\t" \
  "s_bfe_u32 s101, vcc_lo, 262164\n\t" \
  "s_sub_u32 s84, s88, s14\n\t" \
  "s_or_b32 s84, s84, s97\n\t" \
  "s_cmp_lt_u32 s84, 50000000\n\t" \
  "s_cbranch_scc1 L_vl9_2\n\t" \
  "s_mov_b32 s97, -1\n\t" \
  "s_mov_b32 s81, 0x7fc00000\n\t" \
  "L_vd9_2:\n\t" \
  "s_mov_b64 exec, 0xfffffff\n\t" \
  "s_mov_b32 s84, 0\n\t" \
  "buffer_load_dword v210, v198, s[36:39], s84 offen sc1\n\t" \
  "s_mov_b64 exec, -1\n\t" \
  "s_mov_b32 s87, s98\n\t" \
  "s_branch L_vr9_2\n\t" \
  "L_v10_2:\n\t" \
  "s_mov_b32 s101, 0\n\t" \
  "s_mov_b32 s98, s87\n\t" \
  "L_vl10_2:\n\t" \
  "s_memrealtime s[88:89]\n\t" \
  "s_cmp_ge_u32 s82, 18\n\t" \
  "s_cbranch_scc1 L_vn10_2\n\t" \
  "s_waitcnt lgkmcnt(0)\n\t" \
  "s_sub_u32 s84, s88, s14\n\t" \
  "s_cmp_lt_u32 s84, 520\n\t" \
  "s_cbranch_scc1 L_vn10_2\n\t" \
  "v_readlane_b32 s85, v211, s82\n\t" \
  "s_add_u32 s82, s82, 4\n\t" \
  "s_nop 3\n\t" \
  "buffer_load_dword v210, v197, s[60:63], s85 offen\n\t" \
  "L_vn10_2:\n\t" \
  "s_mov_b32 s87, 393216\n\t" \
  "buffer_load_dwordx4 a[64:67], v188, s[36:39], s87 offen offset:0 sc0 sc1\n\t" \
  "buffer_load_dwordx4 a[68:71], v188, s[36:39], s87 offen offset:1024 sc0 sc1\n\t" \
  "buffer_load_dwordx4 a[72:75], v188, s[36:39], s87 offen offset:2048 sc0 sc1\n\t" \
  "buffer_load_dwordx4 a[76:79], v188, s[36:39], s87 offen offset:3072 sc0 sc1\n\t" \
  "buffer_load_dwordx4 a[80:83], v189, s[36:39], s87 offen offset:0 sc0 sc1\n\t" \
  "buffer_load_dwordx4 a[84:87], v189, s[36:39], s87 offen offset:1024 sc0 sc1\n\t" \
  "buffer_load_dwordx4 a[88:91], v189, s[36:39], s87 offen offset:2048 sc0 sc1\n\t" \
  "buffer_load_dwordx4 a[92:95], v189, s[36:39], s87 offen offset:3072 sc0 sc1\n\t" \
  "global_load_dword v226, v195, s[26:27] offset:0 sc0 sc1\n\t" \
  "s_memrealtime s[88:89]\n\t" \
  "s_waitcnt vmcnt(0) lgkmcnt(0)\n\t" \
  "v_accvgpr_read_b32 v237, a64\n\t" \
  "v_accvgpr_read_b32 v238, a65\n\t" \
  "v_accvgpr_read_b32 v239, a66\n\t" \
  "v_accvgpr_read_b32 v240, a67\n\t" \
  "v_min3_u32 v236, v237, v238, v237\n\t" \
  "v_min3_u32 v236, v239, v240, v236\n\t" \
  "v_accvgpr_read_b32 v237, a68\n\t" \
  "v_accvgpr_read_b32 v238, a69\n\t" \
  "v_accvgpr_read_b32 v239, a70\n\t" \
  "v_accvgpr_read_b32 v240, a71\n\t" \
  "v_min3_u32 v236, v237, v238, v236\n\t" \
  "v_min3_u32 v236, v239, v240, v236\n\t" \
  "v_accvgpr_read_b32 v237, a72\n\t" \
  "v_accvgpr_read_b32 v238, a73\n\t" \
  "v_accvgpr_read_b32 v239, a74\n\t" \
  "v_accvgpr_read_b32 v240, a75\n\t" \
  "v_min3_u32 v236, v237, v238, v236\n\t" \
  "v_min3_u32 v236, v239, v240, v236\n\t" \
  "v_accvgpr_read_b32 v237, a76\n\t" \
  "v_accvgpr_read_b32 v238, a77\n\t" \
  "v_accvgpr_read_b32 v239, a78\n\t" \
  "v_accvgpr_read_b32 v240, a79\n\t" \
  "v_min3_u32 v236, v237, v238, v236\n\t" \
  "v_min3_u32 v236, v239, v240, v236\n\t" \
  "v_accvgpr_read_b32 v237, a80\n\t" \
  "v_accvgpr_read_b32 v238, a81\n\t" \
  "v_accvgpr_read_b32 v239, a82\n\t" \
  "v_accvgpr_read_b32 v240, a83\n\t" \
  "v_min3_u32 v236, v237, v238, v236\n\t" \
  "v_min3_u32 v236, v239, v240, v236\n\t" \
  "v_accvgpr_read_b32 v237, a84\n\t" \
  "v_accvgpr_read_b32 v238, a85\n\t" \
  "v_accvgpr_read_b32 v239, a86\n\t" \
  "v_accvgpr_read_b32 v240, a87\n\t" \
  "v_min3_u32 v236, v237, v238, v236\n\t" \
  "v_min3_u32 v236, v239, v240, v236\n\t" \
  "v_accvgpr_read_b32 v237, a88\n\t" \
  "v_accvgpr_read_b32 v238, a89\n\t" \
  "v_accvgpr_read_b32 v239, a90\n\t" \
  "v_accvgpr_read_b32 v240, a91\n\t" \
  "v_min3_u32 v236, v237, v238, v236\n\t" \
  "v_min3_u32 v236, v239, v240, v236\n\t" \
  "v_accvgpr_read_b32 v237, a92\n\t" \
  "v_accvgpr_read_b32 v238, a93\n\t" \
  "v_accvgpr_read_b32 v239, a94\n\t" \
  "v_accvgpr_read_b32 v240, a95\n\t" \
  "v_min3_u32 v236, v237, v238, v236\n\t" \
  "v_min3_u32 v236, v239, v240, v236\n\t" \
  "v_cmp_eq_u32 vcc, 0, v236\n\t" \
  "s_nop 4\n\t" \
  "s_cmp_eq_u64 vcc, 0\n\t" \
  "s_cbranch_scc1 L_vd10_2\n\t" \
  "s_cmp_eq_u32 s101, 15\n\t" \
  "s_cbranch_scc1 L_vd10_2\n\t" \
  "v_cmp_le_u32 vcc, s79, v226\n\t" \
  "s_nop 4\n\t" \
  "s_bfe_u32 s101, vcc_lo, 262168\n\t" \
  "s_sub_u32 s84, s88, s14\n\t" \
  "s_or_b32 s84, s84, s97\n\t" \
  "s_cmp_lt_u32 s84, 50000000\n\t" \
  "s_cbranch_scc1 L_vl10_2\n\t" \
  "s_mov_b32 s97, -1\n\t" \
  "s_mov_b32 s81, 0x7fc00000\n\t" \
  "L_vd10_2:\n\t" \
  "s_mov_b64 exec, 0xfffffff\n\t" \
  "s_mov_b32 s84, 0\n\t" \
  "buffer_load_dword v210, v198, s[36:39], s84 offen sc1\n\t" \
  "s_mov_b64 exec, -1\n\t" \
  "s_mov_b32 s87, s98\n\t" \
  "s_branch L_vr10_2\n\t" \
  "L_q11_2:\n\t" \
  "global_load_dword v226, v195, s[26:27] offset:1024 sc0 sc1\n\t" \
  "s_memrealtime s[88:89]\n\t" \
  "s_waitcnt vmcnt(0) lgkmcnt(0)\n\t" \
  "v_cmp_le_u32 vcc, s79, v226\n\t" \
  "s_nop 4\n\t" \
  "s_andn2_b32 s85, vcc_lo, s96\n\t" \
  "s_or_b32 s96, s96, vcc_lo\n\t" \
  "s_and_b32 exec_lo, s85, 0xfffffff\n\t" \
  "s_mov_b32 exec_hi, 0\n\t" \
  "s_cbranch_execz L_u19_2\n\t" \
  "s_mov_b32 s84, 458752\n\t" \
  "buffer_load_dword v210, v198, s[36:39], s84 offen \n\t" \
  "L_u19_2:\n\t" \
  "s_mov_b64 exec, -1\n\t" \
  "s_bfe_u32 s84, s96, 262144\n\t" \
  "s_cmp_eq_u32 s84, 15\n\t" \
  "s_cbranch_scc1 L_qr11_2\n\t" \
  "s_sub_u32 s84, s88, s14\n\t" \
  "s_or_b32 s84, s84, s97\n\t" \
  "s_cmp_lt_u32 s84, 50000000\n\t" \
  "s_cbranch_scc1 L_q11_2\n\t" \
  "s_mov_b32 s97, -1\n\t" \
  "s_mov_b32 s81, 0x7fc00000\n\t" \
  "s_branch L_qr11_2\n\t" \
  "L_q12_2:\n\t" \
  "global_load_dword v226, v195, s[26:27] offset:1024 sc0 sc1\n\t" \
  "s_memrealtime s[88:89]\n\t" \
  "s_waitcnt vmcnt(0) lgkmcnt(0)\n\t" \
  "v_cmp_le_u32 vcc, s79, v226\n\t" \
  "s_nop 4\n\t" \
  "s_andn2_b32 s85, vcc_lo, s96\n\t" \
  "s_or_b32 s96, s96, vcc_lo\n\t" \
  "s_and_b32 exec_lo, s85, 0xfffffff\n\t" \
  "s_mov_b32 exec_hi, 0\n\t" \
  "s_cbranch_execz L_u20_2\n\t" \
  "s_mov_b32 s84, 458752\n\t" \
  "buffer_load_dword v210, v198, s[36:39], s84 offen \n\t" \
  "L_u20_2:\n\t" \
  "s_mov_b64 exec, -1\n\t" \
  "s_bfe_u32 s84, s96, 262148\n\t" \
  "s_cmp_eq_u32 s84, 15\n\t" \
  "s_cbranch_scc1 L_qr12_2\n\t" \
  "s_sub_u32 s84, s88, s14\n\t" \
  "s_or_b32 s84, s84, s97\n\t" \
  "s_cmp_lt_u32 s84, 50000000\n\t" \
  "s_cbranch_scc1 L_q12_2\n\t" \
  "s_mov_b32 s97, -1\n\t" \
  "s_mov_b32 s81, 0x7fc00000\n\t" \
  "s_branch L_qr12_2\n\t" \
  "L_q13_2:\n\t" \
  "global_load_dword v226, v195, s[26:27] offset:1024 sc0 sc1\n\t" \
  "s_memrealtime s[88:89]\n\t" \
  "s_waitcnt vmcnt(0) lgkmcnt(0)\n\t" \
  "v_cmp_le_u32 vcc, s79, v226\n\t" \
  "s_nop 4\n\t" \
  "s_andn2_b32 s85, vcc_lo, s96\n\t" \
  "s_or_b32 s96, s96, vcc_lo\n\t" \
  "s_and_b32 exec_lo, s85, 0xfffffff\n\t" \
  "s_mov_b32 exec_hi, 0\n\t" \
  "s_cbranch_execz L_u21_2\n\t" \
  "s_mov_b32 s84, 458752\n\t" \
  "buffer_load_dword v210, v198, s[36:39], s84 offen \n\t" \
  "L_u21_2:\n\t" \
  "s_mov_b64 exec, -1\n\t" \
  "s_bfe_u32 s84, s96, 262152\n\t" \
  "s_cmp_eq_u32 s84, 15\n\t" \
  "s_cbranch_scc1 L_qr13_2\n\t" \
  "s_sub_u32 s84, s88, s14\n\t" \
  "s_or_b32 s84, s84, s97\n\t" \
  "s_cmp_lt_u32 s84, 50000000\n\t" \
  "s_cbranch_scc1 L_q13_2\n\t" \
  "s_mov_b32 s97, -1\n\t" \
  "s_mov_b32 s81, 0x7fc00000\n\t" \
  "s_branch L_qr13_2\n\t" \
  "L_q14_2:\n\t" \
  "global_load_dword v226, v195, s[26:27] offset:1024 sc0 sc1\n\t" \
  "s_memrealtime s[88:89]\n\t" \
  "s_waitcnt vmcnt(0) lgkmcnt(0)\n\t" \
  "v_cmp_le_u32 vcc, s79, v226\n\t" \
  "s_nop 4\n\t" \
  "s_andn2_b32 s85, vcc_lo, s96\n\t" \
  "s_or_b32 s96, s96, vcc_lo\n\t" \
  "s_and_b32 exec_lo, s85, 0xfffffff\n\t" \
  "s_mov_b32 exec_hi, 0\n\t" \
  "s_cbranch_execz L_u22_2\n\t" \
  "s_mov_b32 s84, 458752\n\t" \
  "buffer_load_dword v210, v198, s[36:39], s84 offen \n\t" \
  "L_u22_2:\n\t" \
  "s_mov_b64 exec, -1\n\t" \
  "s_bfe_u32 s84, s96, 262156\n\t" \
  "s_cmp_eq_u32 s84, 15\n\t" \
  "s_cbranch_scc1 L_qr14_2\n\t" \
  "s_sub_u32 s84, s88, s14\n\t" \
  "s_or_b32 s84, s84, s97\n\t" \
  "s_cmp_lt_u32 s84, 50000000\n\t" \
  "s_cbranch_scc1 L_q14_2\n\t" \
  "s_mov_b32 s97, -1\n\t" \
  "s_mov_b32 s81, 0x7fc00000\n\t" \
  "s_branch L_qr14_2\n\t" \
  "L_q15_2:\n\t" \
  "global_load_dword v226, v195, s[26:27] offset:1024 sc0 sc1\n\t" \
  "s_memrealtime s[88:89]\n\t" \
  "s_waitcnt vmcnt(0) lgkmcnt(0)\n\t" \
  "v_cmp_le_u32 vcc, s79, v226\n\t" \
  "s_nop 4\n\t" \
  "s_andn2_b32 s85, vcc_lo, s96\n\t" \
  "s_or_b32 s96, s96, vcc_lo\n\t" \
  "s_and_b32 exec_lo, s85, 0xfffffff\n\t" \
  "s_mov_b32 exec_hi, 0\n\t" \
  "s_cbranch_execz L_u23_2\n\t" \
  "s_mov_b32 s84, 458752\n\t" \
  "buffer_load_dword v210, v198, s[36:39], s84 offen \n\t" \
  "L_u23_2:\n\t" \
  "s_mov_b64 exec, -1\n\t" \
  "s_bfe_u32 s84, s96, 262160\n\t" \
  "s_cmp_eq_u32 s84, 15\n\t" \
  "s_cbranch_scc1 L_qr15_2\n\t" \
  "s_sub_u32 s84, s88, s14\n\t" \
  "s_or_b32 s84, s84, s97\n\t" \
  "s_cmp_lt_u32 s84, 50000000\n\t" \
  "s_cbranch_scc1 L_q15_2\n\t" \
  "s_mov_b32 s97, -1\n\t" \
  "s_mov_b32 s81, 0x7fc00000\n\t" \
  "s_branch L_qr15_2\n\t" \
  "L_q16_2:\n\t" \
  "global_load_dword v226, v195, s[26:27] offset:1024 sc0 sc1\n\t" \
  "s_memrealtime s[88:89]\n\t" \
  "s_waitcnt vmcnt(0) lgkmcnt(0)\n\t" \
  "v_cmp_le_u32 vcc, s79, v226\n\t" \
  "s_nop 4\n\t" \
  "s_andn2_b32 s85, vcc_lo, s96\n\t" \
  "s_or_b32 s96, s96, vcc_lo\n\t" \
  "s_and_b32 exec_lo, s85, 0xfffffff\n\t" \
  "s_mov_b32 exec_hi, 0\n\t" \
  "s_cbranch_execz L_u24_2\n\t" \
  "s_mov_b32 s84, 458752\n\t" \
  "buffer_load_dword v210, v198, s[36:39], s84 offen \n\t" \
  "L_u24_2:\n\t" \
  "s_mov_b64 exec, -1\n\t" \
  "s_bfe_u32 s84, s96, 262164\n\t" \
  "s_cmp_eq_u32 s84, 15\n\t" \
  "s_cbranch_scc1 L_qr16_2\n\t" \
  "s_sub_u32 s84, s88, s14\n\t" \
  "s_or_b32 s84, s84, s97\n\t" \
  "s_cmp_lt_u32 s84, 50000000\n\t" \
  "s_cbranch_scc1 L_q16_2\n\t" \
  "s_mov_b32 s97, -1\n\t" \
  "s_mov_b32 s81, 0x7fc00000\n\t" \
  "s_branch L_qr16_2\n\t" \
  "L_q17_2:\n\t" \
  "global_load_dword v226, v195, s[26:27] offset:1024 sc0 sc1\n\t" \
  "s_memrealtime s[88:89]\n\t" \
  "s_waitcnt vmcnt(0) lgkmcnt(0)\n\t" \
  "v_cmp_le_u32 vcc, s79, v226\n\t" \
  "s_nop 4\n\t" \
  "s_andn2_b32 s85, vcc_lo, s96\n\t" \
  "s_or_b32 s96, s96, vcc_lo\n\t" \
  "s_and_b32 exec_lo, s85, 0xfffffff\n\t" \
  "s_mov_b32 exec_hi, 0\n\t" \
  "s_cbranch_execz L_u25_2\n\t" \
  "s_mov_b32 s84, 458752\n\t" \
  "buffer_load_dword v210, v198, s[36:39], s84 offen \n\t" \
  "L_u25_2:\n\t" \
  "s_mov_b64 exec, -1\n\t" \
  "s_bfe_u32 s84, s96, 262168\n\t" \
  "s_cmp_eq_u32 s84, 15\n\t" \
  "s_cbranch_scc1 L_qr17_2\n\t" \
  "s_sub_u32 s84, s88, s14\n\t" \
  "s_or_b32 s84, s84, s97\n\t" \
  "s_cmp_lt_u32 s84, 50000000\n\t" \
  "s_cbranch_scc1 L_q17_2\n\t" \
  "s_mov_b32 s97, -1\n\t" \
  "s_mov_b32 s81, 0x7fc00000\n\t" \
  "s_branch L_qr17_2\n\t" \
  "L_q18_2:\n\t" \
  "global_load_dword v226, v195, s[26:27] offset:2048 sc0 sc1\n\t" \
  "s_memrealtime s[88:89]\n\t" \
  "s_waitcnt vmcnt(0) lgkmcnt(0)\n\t" \
  "v_cmp_le_u32 vcc, s79, v226\n\t" \
  "s_nop 4\n\t" \
  "s_andn2_b32 s85, vcc_lo, s8\n\t" \
  "s_or_b32 s8, s8, vcc_lo\n\t" \
  "s_and_b32 exec_lo, s85, 0xfffffff\n\t" \
  "s_mov_b32 exec_hi, 0\n\t" \
  "s_cbranch_execz L_u26_2\n\t" \
  "s_mov_b32 s84, 917504\n\t" \
  "buffer_load_dword v210, v198, s[36:39], s84 offen \n\t" \
  "L_u26_2:\n\t" \
  "s_mov_b64 exec, -1\n\t" \
  "s_bfe_u32 s84, s8, 262144\n\t" \
  "s_cmp_eq_u32 s84, 15\n\t" \
  "s_cbranch_scc1 L_qr18_2\n\t" \
  "s_sub_u32 s84, s88, s14\n\t" \
  "s_or_b32 s84, s84, s97\n\t" \
  "s_cmp_lt_u32 s84, 50000000\n\t" \
  "s_cbranch_scc1 L_q18_2\n\t" \
  "s_mov_b32 s97, -1\n\t" \
  "s_mov_b32 s81, 0x7fc00000\n\t" \
  "s_branch L_qr18_2\n\t" \
  "L_q19_2:\n\t" \
  "global_load_dword v226, v195, s[26:27] offset:2048 sc0 sc1\n\t" \
  "s_memrealtime s[88:89]\n\t" \
  "s_waitcnt vmcnt(0) lgkmcnt(0)\n\t" \
  "v_cmp_le_u32 vcc, s79, v226\n\t" \
  "s_nop 4\n\t" \
  "s_andn2_b32 s85, vcc_lo, s8\n\t" \
  "s_or_b32 s8, s8, vcc_lo\n\t" \
  "s_and_b32 exec_lo, s85, 0xfffffff\n\t" \
  "s_mov_b32 exec_hi, 0\n\t" \
  "s_cbranch_execz L_u27_2\n\t" \
  "s_mov_b32 s84, 917504\n\t" \
  "buffer_load_dword v210, v198, s[36:39], s84 offen \n\t" \
  "L_u27_2:\n\t" \
  "s_mov_b64 exec, -1\n\t" \
  "s_bfe_u32 s84, s8, 262148\n\t" \
  "s_cmp_eq_u32 s84, 15\n\t" \
  "s_cbranch_scc1 L_qr19_2\n\t" \
  "s_sub_u32 s84, s88, s14\n\t" \
  "s_or_b32 s84, s84, s97\n\t" \
  "s_cmp_lt_u32 s84, 50000000\n\t" \
  "s_cbranch_scc1 L_q19_2\n\t" \
  "s_mov_b32 s97, -1\n\t" \
  "s_mov_b32 s81, 0x7fc00000\n\t" \
  "s_branch L_qr19_2\n\t" \
  "L_q20_2:\n\t" \
  "global_load_dword v226, v195, s[26:27] offset:2048 sc0 sc1\n\t" \
  "s_memrealtime s[88:89]\n\t" \
  "s_waitcnt vmcnt(0) lgkmcnt(0)\n\t" \
  "v_cmp_le_u32 vcc, s79, v226\n\t" \
  "s_nop 4\n\t" \
  "s_andn2_b32 s85, vcc_lo, s8\n\t" \
  "s_or_b32 s8, s8, vcc_lo\n\t" \
  "s_and_b32 exec_lo, s85, 0xfffffff\n\t" \
  "s_mov_b32 exec_hi, 0\n\t" \
  "s_cbranch_execz L_u28_2\n\t" \
  "s_mov_b32 s84, 917504\n\t" \
  "buffer_load_dword v210, v198, s[36:39], s84 offen \n\t" \
  "L_u28_2:\n\t" \
  "s_mov_b64 exec, -1\n\t" \
  "s_bfe_u32 s84, s8, 262152\n\t" \
  "s_cmp_eq_u32 s84, 15\n\t" \
  "s_cbranch_scc1 L_qr20_2\n\t" \
  "s_sub_u32 s84, s88, s14\n\t" \
  "s_or_b32 s84, s84, s97\n\t" \
  "s_cmp_lt_u32 s84, 50000000\n\t" \
  "s_cbranch_scc1 L_q20_2\n\t" \
  "s_mov_b32 s97, -1\n\t" \
  "s_mov_b32 s81, 0x7fc00000\n\t" \
  "s_branch L_qr20_2\n\t" \
  "L_q21_2:\n\t" \
  "global_load_dword v226, v195, s[26:27] offset:2048 sc0 sc1\n\t" \
  "s_memrealtime s[88:89]\n\t" \
  "s_waitcnt vmcnt(0) lgkmcnt(0)\n\t" \
  "v_cmp_le_u32 vcc, s79, v226\n\t" \
  "s_nop 4\n\t" \
  "s_andn2_b32 s85, vcc_lo, s8\n\t" \
  "s_or_b32 s8, s8, vcc_lo\n\t" \
  "s_and_b32 exec_lo, s85, 0xfffffff\n\t" \
  "s_mov_b32 exec_hi, 0\n\t" \
  "s_cbranch_execz L_u29_2\n\t" \
  "s_mov_b32 s84, 917504\n\t" \
  "buffer_load_dword v210, v198, s[36:39], s84 offen \n\t" \
  "L_u29_2:\n\t" \
  "s_mov_b64 exec, -1\n\t" \
  "s_bfe_u32 s84, s8, 262156\n\t" \
  "s_cmp_eq_u32 s84, 15\n\t" \
  "s_cbranch_scc1 L_qr21_2\n\t" \
  "s_sub_u32 s84, s88, s14\n\t" \
  "s_or_b32 s84, s84, s97\n\t" \
  "s_cmp_lt_u32 s84, 50000000\n\t" \
  "s_cbranch_scc1 L_q21_2\n\t" \
  "s_mov_b32 s97, -1\n\t" \
  "s_mov_b32 s81, 0x7fc00000\n\t" \
  "s_branch L_qr21_2\n\t" \
  "L_q22_2:\n\t" \
  "global_load_dword v226, v195, s[26:27] offset:2048 sc0 sc1\n\t" \
  "s_memrealtime s[88:89]\n\t" \
  "s_waitcnt vmcnt(0) lgkmcnt(0)\n\t" \
  "v_cmp_le_u32 vcc, s79, v226\n\t" \
  "s_nop 4\n\t" \
  "s_andn2_b32 s85, vcc_lo, s8\n\t" \
  "s_or_b32 s8, s8, vcc_lo\n\t" \
  "s_and_b32 exec_lo, s85, 0xfffffff\n\t" \
  "s_mov_b32 exec_hi, 0\n\t" \
  "s_cbranch_execz L_u30_2\n\t" \
  "s_mov_b32 s84, 917504\n\t" \
  "buffer_load_dword v210, v198, s[36:39], s84 offen \n\t" \
  "L_u30_2:\n\t" \
  "s_mov_b64 exec, -1\n\t" \
  "s_bfe_u32 s84, s8, 262160\n\t" \
  "s_cmp_eq_u32 s84, 15\n\t" \
  "s_cbranch_scc1 L_qr22_2\n\t" \
  "s_sub_u32 s84, s88, s14\n\t" \
  "s_or_b32 s84, s84, s97\n\t" \
  "s_cmp_lt_u32 s84, 50000000\n\t" \
  "s_cbranch_scc1 L_q22_2\n\t" \
  "s_mov_b32 s97, -1\n\t" \
  "s_mov_b32 s81, 0x7fc00000\n\t" \
  "s_branch L_qr22_2\n\t" \
  "L_q23_2:\n\t" \
  "global_load_dword v226, v195, s[26:27] offset:2048 sc0 sc1\n\t" \
  "s_memrealtime s[88:89]\n\t" \
  "s_waitcnt vmcnt(0) lgkmcnt(0)\n\t" \
  "v_cmp_le_u32 vcc, s79, v226\n\t" \
  "s_nop 4\n\t" \
  "s_andn2_b32 s85, vcc_lo, s8\n\t" \
  "s_or_b32 s8, s8, vcc_lo\n\t" \
  "s_and_b32 exec_lo, s85, 0xfffffff\n\t" \
  "s_mov_b32 exec_hi, 0\n\t" \
  "s_cbranch_execz L_u31_2\n\t" \
  "s_mov_b32 s84, 917504\n\t" \
  "buffer_load_dword v210, v198, s[36:39], s84 offen \n\t" \
  "L_u31_2:\n\t" \
  "s_mov_b64 exec, -1\n\t" \
  "s_bfe_u32 s84, s8, 262164\n\t" \
  "s_cmp_eq_u32 s84, 15\n\t" \
  "s_cbranch_scc1 L_qr23_2\n\t" \
  "s_sub_u32 s84, s88, s14\n\t" \
  "s_or_b32 s84, s84, s97\n\t" \
  "s_cmp_lt_u32 s84, 50000000\n\t" \
  "s_cbranch_scc1 L_q23_2\n\t" \
  "s_mov_b32 s97, -1\n\t" \
  "s_mov_b32 s81, 0x7fc00000\n\t" \
  "s_branch L_qr23_2\n\t" \
  "L_q24_2:\n\t" \
  "global_load_dword v226, v195, s[26:27] offset:2048 sc0 sc1\n\t" \
  "s_memrealtime s[88:89]\n\t" \
  "s_waitcnt vmcnt(0) lgkmcnt(0)\n\t" \
  "v_cmp_le_u32 vcc, s79, v226\n\t" \
  "s_nop 4\n\t" \
  "s_andn2_b32 s85, vcc_lo, s8\n\t" \
  "s_or_b32 s8, s8, vcc_lo\n\t" \
  "s_and_b32 exec_lo, s85, 0xfffffff\n\t" \
  "s_mov_b32 exec_hi, 0\n\t" \
  "s_cbranch_execz L_u32_2\n\t" \
  "s_mov_b32 s84, 917504\n\t" \
  "buffer_load_dword v210, v198, s[36:39], s84 offen \n\t" \
  "L_u32_2:\n\t" \
  "s_mov_b64 exec, -1\n\t" \
  "s_bfe_u32 s84, s8, 262168\n\t" \
  "s_cmp_eq_u32 s84, 15\n\t" \
  "s_cbranch_scc1 L_qr24_2\n\t" \
  "s_sub_u32 s84, s88, s14\n\t" \
  "s_or_b32 s84, s84, s97\n\t" \
  "s_cmp_lt_u32 s84, 50000000\n\t" \
  "s_cbranch_scc1 L_q24_2\n\t" \
  "s_mov_b32 s97, -1\n\t" \
  "s_mov_b32 s81, 0x7fc00000\n\t" \
  "s_branch L_qr24_2\n\t" \
  "L_q25_2:\n\t" \
  "global_load_dword v226, v195, s[26:27] offset:3072 sc0 sc1\n\t" \
  "s_memrealtime s[88:89]\n\t" \
  "s_waitcnt vmcnt(0) lgkmcnt(0)\n\t" \
  "v_cmp_le_u32 vcc, s79, v226\n\t" \
  "s_nop 4\n\t" \
  "s_andn2_b32 s85, vcc_lo, s9\n\t" \
  "s_or_b32 s9, s9, vcc_lo\n\t" \
  "s_and_b32 exec_lo, s85, 0xfffffff\n\t" \
  "s_mov_b32 exec_hi, 0\n\t" \
  "s_cbranch_execz L_u33_2\n\t" \
  "s_mov_b32 s84, 1376256\n\t" \
  "buffer_load_dword v210, v198, s[36:39], s84 offen \n\t" \
  "L_u33_2:\n\t" \
  "s_mov_b64 exec, -1\n\t" \
  "s_bfe_u32 s84, s9, 262144\n\t" \
  "s_cmp_eq_u32 s84, 15\n\t" \
  "s_cbranch_scc1 L_qr25_2\n\t" \
  "s_sub_u32 s84, s88, s14\n\t" \
  "s_or_b32 s84, s84, s97\n\t" \
  "s_cmp_lt_u32 s84, 50000000\n\t" \
  "s_cbranch_scc1 L_q25_2\n\t" \
  "s_mov_b32 s97, -1\n\t" \
  "s_mov_b32 s81, 0x7fc00000\n\t" \
  "s_branch L_qr25_2\n\t" \
  "L_q26_2:\n\t" \
  "global_load_dword v226, v195, s[26:27] offset:3072 sc0 sc1\n\t" \
  "s_memrealtime s[88:89]\n\t" \
  "s_waitcnt vmcnt(0) lgkmcnt(0)\n\t" \
  "v_cmp_le_u32 vcc, s79, v226\n\t" \
  "s_nop 4\n\t" \
  "s_andn2_b32 s85, vcc_lo, s9\n\t" \
  "s_or_b32 s9, s9, vcc_lo\n\t" \
  "s_and_b32 exec_lo, s85, 0xfffffff\n\t" \
  "s_mov_b32 exec_hi, 0\n\t" \
  "s_cbranch_execz L_u34_2\n\t" \
  "s_mov_b32 s84, 1376256\n\t" \
  "buffer_load_dword v210, v198, s[36:39], s84 offen \n\t" \
  "L_u34_2:\n\t" \
  "s_mov_b64 exec, -1\n\t" \
  "s_bfe_u32 s84, s9, 262148\n\t" \
  "s_cmp_eq_u32 s84, 15\n\t" \
  "s_cbranch_scc1 L_qr26_2\n\t" \
  "s_sub_u32 s84, s88, s14\n\t" \
  "s_or_b32 s84, s84, s97\n\t" \
  "s_cmp_lt_u32 s84, 50000000\n\t" \
  "s_cbranch_scc1 L_q26_2\n\t" \
  "s_mov_b32 s97, -1\n\t" \
  "s_mov_b32 s81, 0x7fc00000\n\t" \
  "s_branch L_qr26_2\n\t" \
  "L_q27_2:\n\t" \
  "global_load_dword v226, v195, s[26:27] offset:3072 sc0 sc1\n\t" \
  "s_memrealtime s[88:89]\n\t" \
  "s_waitcnt vmcnt(0) lgkmcnt(0)\n\t" \
  "v_cmp_le_u32 vcc, s79, v226\n\t" \
  "s_nop 4\n\t" \
  "s_andn2_b32 s85, vcc_lo, s9\n\t" \
  "s_or_b32 s9, s9, vcc_lo\n\t" \
  "s_and_b32 exec_lo, s85, 0xfffffff\n\t" \
  "s_mov_b32 exec_hi, 0\n\t" \
  "s_cbranch_execz L_u35_2\n\t" \
  "s_mov_b32 s84, 1376256\n\t" \
  "buffer_load_dword v210, v198, s[36:39], s84 offen \n\t" \
  "L_u35_2:\n\t" \
  "s_mov_b64 exec, -1\n\t" \
  "s_bfe_u32 s84, s9, 262152\n\t" \
  "s_cmp_eq_u32 s84, 15\n\t" \
  "s_cbranch_scc1 L_qr27_2\n\t" \
  "s_sub_u32 s84, s88, s14\n\t" \
  "s_or_b32 s84, s84, s97\n\t" \
  "s_cmp_lt_u32 s84, 50000000\n\t" \
  "s_cbranch_scc1 L_q27_2\n\t" \
  "s_mov_b32 s97, -1\n\t" \
  "s_mov_b32 s81, 0x7fc00000\n\t" \
  "s_branch L_qr27_2\n\t" \
  "L_q28_2:\n\t" \
  "global_load_dword v226, v195, s[26:27] offset:3072 sc0 sc1\n\t" \
  "s_memrealtime s[88:89]\n\t" \
  "s_waitcnt vmcnt(0) lgkmcnt(0)\n\t" \
  "v_cmp_le_u32 vcc, s79, v226\n\t" \
  "s_nop 4\n\t" \
  "s_andn2_b32 s85, vcc_lo, s9\n\t" \
  "s_or_b32 s9, s9, vcc_lo\n\t" \
  "s_and_b32 exec_lo, s85, 0xfffffff\n\t" \
  "s_mov_b32 exec_hi, 0\n\t" \
  "s_cbranch_execz L_u36_2\n\t" \
  "s_mov_b32 s84, 1376256\n\t" \
  "buffer_load_dword v210, v198, s[36:39], s84 offen \n\t" \
  "L_u36_2:\n\t" \
  "s_mov_b64 exec, -1\n\t" \
  "s_bfe_u32 s84, s9, 262156\n\t" \
  "s_cmp_eq_u32 s84, 15\n\t" \
  "s_cbranch_scc1 L_qr28_2\n\t" \
  "s_sub_u32 s84, s88, s14\n\t" \
  "s_or_b32 s84, s84, s97\n\t" \
  "s_cmp_lt_u32 s84, 50000000\n\t" \
  "s_cbranch_scc1 L_q28_2\n\t" \
  "s_mov_b32 s97, -1\n\t" \
  "s_mov_b32 s81, 0x7fc00000\n\t" \
  "s_branch L_qr28_2\n\t" \
  "L_q29_2:\n\t" \
  "global_load_dword v226, v195, s[26:27] offset:3072 sc0 sc1\n\t" \
  "s_memrealtime s[88:89]\n\t" \
  "s_waitcnt vmcnt(0) lgkmcnt(0)\n\t" \
  "v_cmp_le_u32 vcc, s79, v226\n\t" \
  "s_nop 4\n\t" \
  "s_andn2_b32 s85, vcc_lo, s9\n\t" \
  "s_or_b32 s9, s9, vcc_lo\n\t" \
  "s_and_b32 exec_lo, s85, 0xfffffff\n\t" \
  "s_mov_b32 exec_hi, 0\n\t" \
  "s_cbranch_execz L_u37_2\n\t" \
  "s_mov_b32 s84, 1376256\n\t" \
  "buffer_load_dword v210, v198, s[36:39], s84 offen \n\t" \
  "L_u37_2:\n\t" \
  "s_mov_b64 exec, -1\n\t" \
  "s_bfe_u32 s84, s9, 262160\n\t" \
  "s_cmp_eq_u32 s84, 15\n\t" \
  "s_cbranch_scc1 L_qr29_2\n\t" \
  "s_sub_u32 s84, s88, s14\n\t" \
  "s_or_b32 s84, s84, s97\n\t" \
  "s_cmp_lt_u32 s84, 50000000\n\t" \
  "s_cbranch_scc1 L_q29_2\n\t" \
  "s_mov_b32 s97, -1\n\t" \
  "s_mov_b32 s81, 0x7fc00000\n\t" \
  "s_branch L_qr29_2\n\t" \
  "L_q30_2:\n\t" \
  "global_load_dword v226, v195, s[26:27] offset:3072 sc0 sc1\n\t" \
  "s_memrealtime s[88:89]\n\t" \
  "s_waitcnt vmcnt(0) lgkmcnt(0)\n\t" \
  "v_cmp_le_u32 vcc, s79, v226\n\t" \
  "s_nop 4\n\t" \
  "s_andn2_b32 s85, vcc_lo, s9\n\t" \
  "s_or_b32 s9, s9, vcc_lo\n\t" \
  "s_and_b32 exec_lo, s85, 0xfffffff\n\t" \
  "s_mov_b32 exec_hi, 0\n\t" \
  "s_cbranch_execz L_u38_2\n\t" \
  "s_mov_b32 s84, 1376256\n\t" \
  "buffer_load_dword v210, v198, s[36:39], s84 offen \n\t" \
  "L_u38_2:\n\t" \
  "s_mov_b64 exec, -1\n\t" \
  "s_bfe_u32 s84, s9, 262164\n\t" \
  "s_cmp_eq_u32 s84, 15\n\t" \
  "s_cbranch_scc1 L_qr30_2\n\t" \
  "s_sub_u32 s84, s88, s14\n\t" \
  "s_or_b32 s84, s84, s97\n\t" \
  "s_cmp_lt_u32 s84, 50000000\n\t" \
  "s_cbranch_scc1 L_q30_2\n\t" \
  "s_mov_b32 s97, -1\n\t" \
  "s_mov_b32 s81, 0x7fc00000\n\t" \
  "s_branch L_qr30_2\n\t" \
  "L_q31_2:\n\t" \
  "global_load_dword v226, v195, s[26:27] offset:3072 sc0 sc1\n\t" \
  "s_memrealtime s[88:89]\n\t" \
  "s_waitcnt vmcnt(0) lgkmcnt(0)\n\t" \
  "v_cmp_le_u32 vcc, s79, v226\n\t" \
  "s_nop 4\n\t" \
  "s_andn2_b32 s85, vcc_lo, s9\n\t" \
  "s_or_b32 s9, s9, vcc_lo\n\t" \
  "s_and_b32 exec_lo, s85, 0xfffffff\n\t" \
  "s_mov_b32 exec_hi, 0\n\t" \
  "s_cbranch_execz L_u39_2\n\t" \
  "s_mov_b32 s84, 1376256\n\t" \
  "buffer_load_dword v210, v198, s[36:39], s84 offen \n\t" \
  "L_u39_2:\n\t" \
  "s_mov_b64 exec, -1\n\t" \
  "s_bfe_u32 s84, s9, 262168\n\t" \
  "s_cmp_eq_u32 s84, 15\n\t" \
  "s_cbranch_scc1 L_qr31_2\n\t" \
  "s_sub_u32 s84, s88, s14\n\t" \
  "s_or_b32 s84, s84, s97\n\t" \
  "s_cmp_lt_u32 s84, 50000000\n\t" \
  "s_cbranch_scc1 L_q31_2\n\t" \
  "s_mov_b32 s97, -1\n\t" \
  "s_mov_b32 s81, 0x7fc00000\n\t" \
  "s_branch L_qr31_2\n\t" \
  ".fill 6144, 4, 0xbf800000\n\t" \
  ""
#define KERN_CLOBBERS "v1", "v2", "v3", "v4", "v5", "v6", "v7", "v8", "v9", "v10", "v11", "v12", "v13", "v14", "v15", "v16", "v17", "v18", "v19", "v20", "v21", "v22", "v23", "v24", "v25", "v26", "v27", "v28", "v29", "v30", "v31", "v32", "v33", "v34", "v35", "v36", "v37", "v38", "v39", "v40", "v41", "v42", "v43", "v44", "v45", "v46", "v47", "v48", "v49", "v50", "v51", "v52", "v53", "v54", "v55", "v56", "v57", "v58", "v59", "v60", "v61", "v62", "v63", "v64", "v65", "v66", "v67", "v68", "v69", "v70", "v71", "v72", "v73", "v74", "v75", "v76", "v77", "v78", "v79", "v80", "v81", "v82", "v83", "v84", "v85", "v86", "v87", "v88", "v89", "v90", "v91", "v92", "v93", "v94", "v95", "v96", "v97", "v98", "v99", "v100", "v101", "v102", "v103", "v104", "v105", "v106", "v107", "v108", "v109", "v110", "v111", "v112", "v113", "v114", "v115", "v116", "v117", "v118", "v119", "v120", "v121", "v122", "v123", "v124", "v125", "v126", "v127", "v128", "v129", "v130", "v131", "v132", "v133", "v134", "v135", "v136", "v137", "v138", "v139", "v140", "v141", "v142", "v143", "v144", "v145", "v146", "v147", "v148", "v149", "v150", "v151", "v152", "v153", "v154", "v155", "v156", "v157", "v158", "v159", "v160", "v161", "v162", "v163", "v164", "v165", "v166", "v167", "v168", "v169", "v170", "v171", "v172", "v173", "v174", "v175", "v176", "v177", "v178", "v179", "v180", "v181", "v182", "v183", "v184", "v185", "v186", "v187", "v188", "v189", "v190", "v191", "v192", "v193", "v194", "v195", "v196", "v197", "v198", "v199", "v200", "v201", "v202", "v203", "v204", "v205", "v206", "v207", "v208", "v209", "v210", "v211", "v212", "v213", "v214", "v215", "v216", "v217", "v218", "v219", "v220", "v221", "v222", "v223", "v224", "v225", "v226", "v227", "v228", "v229", "v230", "v231", "v232", "v233", "v234", "v235", "v236", "v237", "v238", "v239", "v240", "v241", "v242", "v243", "a0", "a1", "a2", "a3", "a4", "a5", "a6", "a7", "a8", "a9", "a10", "a11", "a12", "a13", "a14", "a15", "a16", "a17", "a18", "a19", "a20", "a21", "a22", "a23", "a24", "a25", "a26", "a27", "a28", "a29", "a30", "a31", "a32", "a33", "a34", "a35", "a36", "a37", "a38", "a39", "a40", "a41", "a42", "a43", "a44", "a45", "a46", "a47", "a48", "a49", "a50", "a51", "a52", "a53", "a54", "a55", "a56", "a57", "a58", "a59", "a60", "a61", "a62", "a63", "a64", "a65", "a66", "a67", "a68", "a69", "a70", "a71", "a72", "a73", "a74", "a75", "a76", "a77", "a78", "a79", "a80", "a81", "a82", "a83", "a84", "a85", "a86", "a87", "a88", "a89", "a90", "a91", "a92", "a93", "a94", "a95", "a96", "a97", "a98", "a99", "a100", "a101", "a102", "a103", "a104", "a105", "a106", "a107", "a108", "a109", "a110", "a111", "a112", "a113", "a114", "a115", "a116", "a117", "a118", "a119", "a120", "a121", "a122", "a123", "a124", "a125", "a126", "a127", "a128", "a129", "a130", "a131", "a132", "a133", "a134", "a135", "a136", "a137", "a138", "a139", "a140", "a141", "a142", "a143", "a144", "a145", "a146", "a147", "a148", "a149", "a150", "a151", "a152", "a153", "a154", "a155", "a156", "a157", "a158", "a159", "a160", "a161", "a162", "a163", "a164", "a165", "a166", "a167", "a168", "a169", "a170", "a171", "a172", "a173", "a174", "a175", "a176", "a177", "a178", "a179", "a180", "a181", "a182", "a183", "a184", "a185", "a186", "a187", "a188", "a189", "a190", "a191", "a192", "a193", "a194", "a195", "a196", "a197", "a198", "a199", "a200", "a201", "a202", "a203", "a204", "a205", "a206", "a207", "a208", "a209", "a210", "a211", "a212", "a213", "a214", "a215", "a216", "a217", "a218", "a219", "a220", "a221", "a222", "a223", "a224", "a225", "a226", "a227", "a228", "a229", "a230", "a231", "a232", "a233", "a234", "a235", "a236", "a237", "a238", "a239", "a240", "a241", "a242", "a243", "a244", "a245", "a246", "a247", "a248", "a249", "a250", "a251", "a252", "a253", "a254", "a255", "s8", "s9", "s10", "s11", "s12", "s13", "s14", "s15", "s16", "s17", "s18", "s19", "s20", "s21", "s22", "s23", "s24", "s25", "s26", "s27", "s28", "s29", "s30", "s31", "s32", "s33", "s34", "s35", "s36", "s37", "s38", "s39", "s40", "s41", "s42", "s43", "s44", "s45", "s46", "s47", "s48", "s49", "s50", "s51", "s52", "s53", "s54", "s55", "s56", "s57", "s58", "s59", "s60", "s61", "s62", "s63", "s64", "s65", "s66", "s67", "s68", "s69", "s70", "s71", "s72", "s73", "s74", "s75", "s76", "s77", "s78", "s79", "s80", "s81", "s82", "s83", "s84", "s85", "s86", "s87", "s88", "s89", "s90", "s91", "s92", "s93", "s94", "s95", "s96", "s97", "s98", "s99", "s100", "s101", "vcc", "scc", "m0", "memory"


struct KParams {
  const void* x; const void* w; const void* sx; const void* sw;
  void* out; void* ws_self; int rank; int pad; void* wsr[7];
};

__global__ __launch_bounds__(KERN_THREADS, KERN_WPE) void a2a_gemm_kernel(KParams p) {
  __shared__ __attribute__((aligned(1024))) char lds[KERN_LDS];
  int tid = threadIdx.x;
  asm volatile(KERN_ASM
               : "+v"(tid)
               : "s"(__builtin_amdgcn_kernarg_segment_ptr()), "s"(blockIdx.x), "s"((unsigned)(size_t)lds)
               : KERN_CLOBBERS);
}

extern "C" void kernel_launch(int rank, int nranks, ncclComm_t comm, hipStream_t stream,
                              void* const* d_in, const int* in_sizes, int n_in,
                              void* d_out, int out_size, const int* in_dtypes, int out_dtype,
                              void* const* d_out_all, void* const* d_ws_all, size_t ws_bytes) {
  if (nranks != 8 || n_in != 4 || (unsigned)rank >= 8u || ws_bytes < (16u << 20) ||
      in_sizes[0] != 4096 * 512 || in_sizes[1] != 4096 * 16384 || out_size != 512 * 16384) {
    fprintf(stderr, "unexpected problem shape\n");
    return;
  }
  KParams p;
  p.x = d_in[0]; p.w = d_in[1]; p.sx = d_in[2]; p.sw = d_in[3];
  p.out = d_out; p.ws_self = d_ws_all[rank]; p.rank = rank; p.pad = 0;
  for (int j = 0; j < 7; ++j) p.wsr[j] = d_ws_all[(rank - 1 - j) & 7];
  hipLaunchKernelGGL(a2a_gemm_kernel, dim3(256), dim3(KERN_THREADS), 0, stream, p);
  hipError_t e = hipGetLastError();
  if (e != hipSuccess) fprintf(stderr, "launch error %s\n", hipGetErrorString(e));
}
```
